```python
import numpy as np

kernel_name = "gpu_gemm_rs_m4096_k4096_n4096_bf16_relu"


ABSMAX_RTOL = 0.008

COMPUTE_DTYPE = "bfloat16"
EPILOGUE = "relu"
IN_QUANT = None
OUT_QUANT = None
IN_DTYPES = {"x": "bfloat16", "w_mat": "bfloat16"}
WORLD_SIZE = 8

def _to_bf16(x):
    u = np.ascontiguousarray(x, dtype=np.float32).view(np.uint32)
    return ((u + 0x7FFF + ((u >> 16) & 1)) & 0xFFFF0000).view(np.float32)

def setup_inputs():

    x = _to_bf16(np.random.randn(4096, 4096).astype(np.float32))
    w_mat = _to_bf16((np.random.randn(4096, 4096) / np.sqrt(4096)).astype(np.float32))
    return {"x": x, "w_mat": w_mat}

def _epilogue(y):
    return np.maximum(y, 0.0).astype(np.float32)

def reference(x, w_mat):
    y = (x @ w_mat).astype(np.float32)
    return _epilogue(y)
```

```cpp
#include <hip/hip_runtime.h>
#include <rccl/rccl.h>
#include <cstdio>
#include <cstdint>

#define HIP_CHECK(expr) do { hipError_t _e = (expr); if (_e != hipSuccess) { \
    fprintf(stderr, "HIP error %s at %s:%d\n", hipGetErrorString(_e), __FILE__, __LINE__); \
    return; } } while (0)

namespace {

typedef unsigned u32;
typedef unsigned long long u64;
typedef u32 u32x4 __attribute__((ext_vector_type(4)));
typedef float f32x2 __attribute__((ext_vector_type(2)));
typedef float f32x16 __attribute__((ext_vector_type(16)));
typedef __bf16 bf16x2 __attribute__((ext_vector_type(2)));
typedef __bf16 bf16x8 __attribute__((ext_vector_type(8)));
typedef short s16x4 __attribute__((ext_vector_type(4)));
typedef short s16x8 __attribute__((ext_vector_type(8)));
#define LDSP __attribute__((address_space(3)))

constexpr int NR = 8;
constexpr u32 PIECE = 65536u;
constexpr u32 MIB = 1u << 20;
constexpr u32 XS_OFF = 0;
constexpr u32 XS_PAR = 6u * MIB;
constexpr u32 WS_OFF = 12u * MIB;
constexpr u32 PIN_OFF = 24u * MIB;
constexpr u32 PIN_PAR = 7u * MIB;
constexpr u32 POUT_OFF = 38u * MIB;
constexpr u32 FLG_OFF = 45u * MIB;
constexpr u32 FIN_OFF = FLG_OFF;
constexpr u32 FOUT_OFF = FLG_OFF + 16384u;
constexpr u32 FLOC_OFF = FLG_OFF + 32768u;
constexpr u32 CNT_OFF = FLG_OFF + 49152u;
constexpr u32 TMO_OFF = FLG_OFF + 57344u;
constexpr u32 DUMMY_OFF = FLG_OFF + 65536u;
constexpr u32 DBG_OFF = FLG_OFF + 131072u;
constexpr u32 WS_NEED = FLG_OFF + 262144u;

constexpr int N_PUSH_WG = 56;
constexpr int N_MAIN_WG = 256;
constexpr int N_FIN_WG = 96;
constexpr int N_WG = 2 * N_PUSH_WG + N_MAIN_WG + N_FIN_WG;
#define TAU 112
#define LAG 1
constexpr u32 STAGE_B = 16384u;
constexpr u32 LDS_BYTES = 4u * STAGE_B + 64u;
constexpr u64 DEADLINE = 100000000ull;

struct Params {
  const char* x;
  const char* w;
  char* out;
  char* ws[NR];
  int rank;
};

#define SB() __builtin_amdgcn_sched_barrier(0)
template <int N> __device__ __forceinline__ void waitv() { asm volatile("s_waitcnt vmcnt(%0)" :: "i"(N) : "memory"); }

__device__ __forceinline__ u64 now() { return __builtin_amdgcn_s_memrealtime(); }
template <typename T> __device__ __forceinline__ T* uni(T* p) {
  const u64 v = (u64)(uintptr_t)p;
  const u32 lo = __builtin_amdgcn_readfirstlane((u32)v), hi = __builtin_amdgcn_readfirstlane((u32)(v >> 32));
  return (T*)(uintptr_t)(((u64)hi << 32) | lo);
}
__device__ __forceinline__ bool expired(u64 t0) {
  return __builtin_amdgcn_readfirstlane((int)(now() - t0 > DEADLINE)) != 0;
}
__device__ __forceinline__ u32 ld_sys(const void* p) {
  return __hip_atomic_load((const u32*)p, __ATOMIC_RELAXED, __HIP_MEMORY_SCOPE_SYSTEM);
}
__device__ __forceinline__ void st_sys(void* p, u32 v) {
  __hip_atomic_store((u32*)p, v, __ATOMIC_RELAXED, __HIP_MEMORY_SCOPE_SYSTEM);
}
__device__ __forceinline__ void st_agt(void* p, u32 v) {
  __hip_atomic_store((u32*)p, v, __ATOMIC_RELAXED, __HIP_MEMORY_SCOPE_AGENT);
}
template <int IMM> __device__ __forceinline__ void ld16i(u32x4& v, u32 off, const char* base) {
  asm volatile("global_load_dwordx4 %0, %1, %2 offset:%3 sc0 sc1" : "=v"(v) : "v"(off), "s"(base), "i"(IMM) : "memory");
}
template <int IMM> __device__ __forceinline__ void st16(u32 off, u32x4 v, char* base) {
  asm volatile("global_store_dwordx4 %0, %1, %2 offset:%3 sc0 sc1\n\ts_nop 1" :: "v"(off), "v"(v), "s"(base), "i"(IMM) : "memory");
}
__device__ __forceinline__ void dma16x2(u32 lds, u32 o0, u32 o1, const char* base) {
  asm volatile("s_mov_b32 m0, %0\n\ts_nop 3\n\t"
               "global_load_lds_dwordx4 %1, %3 sc0 sc1\n\t"
               "global_load_lds_dwordx4 %2, %3 offset:1024 sc0 sc1"
               :: "s"(lds), "v"(o0), "v"(o1), "s"(base) : "memory");
}
__device__ __forceinline__ void dma4(u32 lds, u32 off, const char* base) {
  asm volatile("s_mov_b32 m0, %0\n\ts_nop 3\n\t"
               "global_load_lds_dword %1, %2 sc0 sc1"
               :: "s"(lds), "v"(off), "s"(base) : "memory");
}
__device__ __forceinline__ void st4_lane0(u32 off, u32 val, char* base) {
  asm volatile("s_mov_b64 exec, 1\n\ts_nop 3\n\tglobal_store_dword %0, %1, %2 sc0 sc1\n\ts_mov_b64 exec, -1"
               :: "v"(off), "v"(val), "s"(base) : "memory");
}

#define STAMP(ws, idx) do {} while (0)

constexpr int NPIECE = 46;

__device__ __forceinline__ void pusher(const Params& P, int wave, u32 lane, char* smem) {
  __builtin_amdgcn_s_setprio(3);
  const u64 tstart = now();
  const int s = P.rank;
  char* const myws = P.ws[s];
  const int idx = (blockIdx.x >> 3) * 4 + wave;
  const int d = 1 + idx % 7;
  const int p = (blockIdx.x & 7) * 4 + idx / 7;
  const int slot = d - 2;
  const u32 r = (u32)(s ^ d);
  if (r >= (u32)NR) return;
  char* const rws = uni(P.ws[r]);
  const char* const xin = P.x;
  const char* const win = P.w;
  const int rg = r >> 2, rb_ = (r >> 1) & 1, ra = r & 1;
  const int first = d == 1 ? 30 : 0;

  const u32 lb = (u32)(uintptr_t)smem + (u32)wave * 16384u;

  const u32 dl = (u32)p * 2048u + lane * 16u;
  const u32 xc = (lane & 3u) ^ ((lane >> 4) & 3u);
  const u32 x0 = ((u32)(2 * (p & 3)) * 16u + (lane >> 2)) * 1024u + (u32)(p >> 2) * 64u + xc * 16u;
  const u32 x1 = x0 + 16384u - 1024u;
  const u32 q4 = lane >> 4;
  const u32 v0 = (u32)(2 * p) & 7u, ks = (u32)p >> 2;
  const u32 w0 = (ks * 32u + 4u * v0 + q4) * 8192u + (((lane & 15u) ^ ((q4 << 2) | (v0 & 3u))) * 16u);
  const u32 w1 = (ks * 32u + 4u * (v0 + 1u) + q4) * 8192u + (((lane & 15u) ^ ((q4 << 2) | ((v0 + 1u) & 3u))) * 16u) - 1024u;
  const u32 poll_off = lane < 32u ? FLOC_OFF + (u32)((d - 1) * 8 + (p >> 2)) * 128u + (lane & 15u) * 4u : TMO_OFF;
  const u32 cnt_off = CNT_OFF + (u32)(blockIdx.x * 4 + wave) * 4u;

#define ISSUE_LOADS(QQ) do { \
    const int q_ = (QQ) > NPIECE - 1 ? NPIECE - 1 : (QQ); \
    const bool isx_ = q_ < 12 || (q_ >= 28 && q_ < 30), isw_ = !isx_ && q_ < 28; \
    const int jj_ = (q_ >> 1) + 1; \
    const int j_ = q_ < 12 ? jj_ + (jj_ >= d ? 1 : 0) : 0; \
    const u32 offx_ = ((r ^ (u32)j_) * 512u + (u32)rb_ * 256u + (u32)(q_ & 1) * 128u) * 1024u + (u32)ra * 512u; \
    const u32 offw_ = (u32)ra * 256u * 8192u + (u32)rg * 4096u + (u32)(q_ - 12) * 256u; \
    const u32 offo_ = POUT_OFF + (u32)((d - 1) * 16 + (q_ - 30)) * PIECE; \
    const u32 off_ = isx_ ? offx_ : (isw_ ? offw_ : offo_); \
    const char* base_ = isx_ ? xin : (isw_ ? win : (const char*)myws); \
    const u32 o0_ = (isx_ ? x0 : (isw_ ? w0 : dl)) + off_; \
    const u32 o1_ = (isx_ ? x1 : (isw_ ? w1 : dl)) + off_; \
    dma16x2(lb + (u32)((QQ) & 3) * 2048u, o0_, o1_, base_); \
  } while (0)

  dma4(lb + 12288u, cnt_off, myws);
  if (first == 0) { ISSUE_LOADS(0); ISSUE_LOADS(1); waitv<4>(); } else waitv<0>();
  const u32 call = __builtin_amdgcn_readfirstlane(*(volatile LDSP u32*)(uintptr_t)(lb + 12288u)) + 1u;
  const u32 par = call & 1u;
  const u32 callv = call;
  st4_lane0(cnt_off, callv, myws);
  STAMP(myws, 0);

  const u32 p4 = (u32)p * 4u;
#define FLAG(QQ) do { \
    const int q_ = (QQ); \
    const bool real_ = q_ >= 12 && q_ >= first; \
    const u32 o_ = !real_ ? DUMMY_OFF + (u32)(d - 1) * 4096u + (u32)p * 124u \
                 : (q_ < 30 ? FIN_OFF + (u32)(slot * 18 + (q_ - 12)) * 128u : FOUT_OFF + (u32)((d - 1) * 16 + (q_ - 30)) * 128u); \
    char* base_ = real_ ? rws : myws; \
    st4_lane0(p4 + o_, callv, base_); \
  } while (0)

  u32 ready = 0;
  bool dead = false;
  for (int i = first == 0 ? 0 : first - 2; i < NPIECE; ++i) {
    const int qn = i + 2;
    if (qn >= 30 && qn < NPIECE) {
      if (i >= first + 1) {
        const u32 pv0 = *(volatile LDSP u32*)(uintptr_t)(lb + 8192u + (u32)((i + 1) & 3) * 256u + lane * 4u);
        ready |= (u32)__builtin_amdgcn_ballot_w64(pv0 >= call);
      }
      if (!((ready >> (qn - 30)) & 1u)) {
        waitv<0>();
#pragma unroll
        for (int z = LAG; z >= 1; --z) FLAG(i - z);
        const u64 t0 = now();
        for (;;) {
          const u32 pv = ld_sys(myws + poll_off);
          ready |= (u32)__builtin_amdgcn_ballot_w64(pv >= call);
          if ((ready >> (qn - 30)) & 1u) break;
          if ((__builtin_amdgcn_ballot_w64(pv == call) >> 63) || expired(t0)) {
            st4_lane0(TMO_OFF, callv, myws);
            dead = true;
            break;
          }
          __builtin_amdgcn_s_sleep(4);
        }
        if (dead) break;
      }
    }
    if (TAU > 0 && first == 0) {
      const u32 target = (u32)i * (u32)TAU;
      while ((u32)__builtin_amdgcn_readfirstlane((u32)(now() - tstart)) < target) __builtin_amdgcn_s_sleep(1);
    }
    dma4(lb + 8192u + (u32)(i & 3) * 256u, poll_off, myws);
    ISSUE_LOADS(qn);
    if (i < first) continue;
    if (i >= first + 2) waitv<12>(); else waitv<5>();
    const u32 la = lb + (u32)(i & 3) * 2048u + lane * 16u;
    const u32x4 d0 = *(volatile LDSP u32x4*)(uintptr_t)la;
    const u32x4 d1 = *(volatile LDSP u32x4*)(uintptr_t)(la + 1024u);
    {
      const int jj = (i >> 1) + 1;
      const int j = i < 12 ? jj + (jj >= d ? 1 : 0) : 0;
      const u32 ox = XS_OFF + par * XS_PAR + (u32)((2 * j + (i & 1)) * 6 + slot) * PIECE;
      const u32 ow = WS_OFF + par * XS_PAR + (u32)((i - 12) * 6 + slot) * PIECE;
      const u32 oo = PIN_OFF + par * PIN_PAR + (u32)((d - 1) * 16 + (i - 30)) * PIECE;
      const u32 o = dl + ((i < 12 || (i >= 28 && i < 30)) ? ox : (i < 28 ? ow : oo));
      st16<0>(o, d0, rws);
      st16<1024>(o, d1, rws);
    }
    waitv<6 * LAG>();
    FLAG(i - LAG);
    if (i == 11) STAMP(myws, 1);
    if (i == 29) STAMP(myws, 2);
  }
  if (!dead) {
    if (LAG >= 3) { waitv<13>(); FLAG(NPIECE - 3); }
    if (LAG >= 2) { waitv<7>(); FLAG(NPIECE - 2); }
    waitv<0>();   FLAG(NPIECE - 1);
  }
  waitv<0>();
  STAMP(myws, 3);
}

struct Frag {
  bf16x8 x[2][2];
  bf16x8 w[2][2];
};

struct Loader {
  const char* base;
  u32 stride;
  u32 v[4];
  u32 m0base;
};

template <int STAGE> __device__ __forceinline__ void glds_pair(const Loader& L, int half) {
  const u32 m0v = L.m0base + STAGE * STAGE_B;
  if (half == 0)
    asm volatile("s_mov_b32 m0, %0\n\ts_nop 0\n\t"
                 "global_load_lds_dwordx4 %1, %3 sc0 sc1\n\t"
                 "global_load_lds_dwordx4 %2, %3 offset:1024 sc0 sc1"
                 :: "s"(m0v), "v"(L.v[0]), "v"(L.v[1]), "s"(L.base) : "memory");
  else
    asm volatile("s_mov_b32 m0, %0\n\ts_nop 0\n\t"
                 "global_load_lds_dwordx4 %1, %3 offset:2048 sc0 sc1\n\t"
                 "global_load_lds_dwordx4 %2, %3 offset:3072 sc0 sc1"
                 :: "s"(m0v), "v"(L.v[2]), "v"(L.v[3]), "s"(L.base) : "memory");
}

struct RdAddr { u32 xa[2]; u32 wa[2][2]; };

template <int STAGE> __device__ __forceinline__ bf16x8 rd_x(const RdAddr& A, int mt, int ks) {
  return *(const LDSP bf16x8*)(uintptr_t)(A.xa[ks] + (u32)(STAGE * STAGE_B + mt * 2048));
}
template <int STAGE> __device__ __forceinline__ bf16x8 rd_w(const RdAddr& A, int nt, int ks) {
  s16x4 lo = __builtin_amdgcn_ds_read_tr16_b64_v4i16((LDSP s16x4*)(uintptr_t)(A.wa[nt][0] + (u32)(STAGE * STAGE_B + 8192 + ks * 4096)));
  s16x4 hi = __builtin_amdgcn_ds_read_tr16_b64_v4i16((LDSP s16x4*)(uintptr_t)(A.wa[nt][1] + (u32)(STAGE * STAGE_B + 8192 + ks * 4096)));
  s16x8 v = __builtin_shufflevector(lo, hi, 0, 1, 2, 3, 4, 5, 6, 7);
  return __builtin_bit_cast(bf16x8, v);
}
#define MFMA(nt, mt, ks) acc[nt][mt] = __builtin_amdgcn_mfma_f32_32x32x16_bf16(cur.w[nt][ks], cur.x[mt][ks], acc[nt][mt], 0, 0, 0)

template <int S, int VM, bool LOAD, bool READ>
__device__ __forceinline__ void kstep(f32x16 (&acc)[2][2], const Frag& cur, Frag& nxt, const RdAddr& A, Loader& L) {
  constexpr int SN = (S + 1) & 3;
  if (READ) {
    __builtin_amdgcn_s_waitcnt((VM & 15) | (7 << 4) | (0 << 8) | ((VM >> 4) << 14));
    asm volatile("s_barrier" ::: "memory");
  } else {
    __builtin_amdgcn_s_waitcnt(15 | (7 << 4) | (0 << 8) | (3 << 14));
  }
  SB();
  MFMA(0, 0, 0); SB();
  if (READ) { nxt.w[0][0] = rd_w<SN>(A, 0, 0); nxt.x[0][0] = rd_x<SN>(A, 0, 0); } SB();
  MFMA(0, 1, 0); SB();
  if (READ) { nxt.x[1][0] = rd_x<SN>(A, 1, 0); nxt.w[1][0] = rd_w<SN>(A, 1, 0); } SB();
  MFMA(1, 0, 0); SB();
  if (LOAD) glds_pair<S>(L, 0);
  SB();
  MFMA(1, 1, 0); SB();
  if (READ) { nxt.w[0][1] = rd_w<SN>(A, 0, 1); nxt.x[0][1] = rd_x<SN>(A, 0, 1); } SB();
  MFMA(0, 0, 1); SB();
  if (READ) { nxt.x[1][1] = rd_x<SN>(A, 1, 1); nxt.w[1][1] = rd_w<SN>(A, 1, 1); } SB();
  MFMA(0, 1, 1); SB();
  if (LOAD) { glds_pair<S>(L, 1); L.base += L.stride; }
  SB();
  MFMA(1, 0, 1); SB();
  MFMA(1, 1, 1); SB();
}

template <int S> __device__ __forceinline__ void stage_load(Loader& L) {
  glds_pair<S>(L, 0); glds_pair<S>(L, 1); L.base += L.stride;
}

__device__ __forceinline__ void gemm_phase(f32x16 (&acc)[2][2], const RdAddr& A, Loader& L, int n, int tskip) {
  Frag f0, f1;
  L.base = uni(L.base + (tskip == 0 ? PIECE : 0u));
  L.stride = __builtin_amdgcn_readfirstlane(L.stride);
  asm volatile("s_nop 4" ::: "memory");
  stage_load<0>(L); stage_load<1>(L); stage_load<2>(L); stage_load<3>(L);
  waitv<12>();
  asm volatile("s_barrier" ::: "memory");
#pragma unroll
  for (int ks = 0; ks < 2; ++ks)
#pragma unroll
    for (int i = 0; i < 2; ++i) { f0.w[i][ks] = rd_w<0>(A, i, ks); f0.x[i][ks] = rd_x<0>(A, i, ks); }
  SB();
  for (int t = 0; t < n - 4; t += 4) {
    if (t + 4 == tskip) L.base += PIECE;
    kstep<0, 8, true, true>(acc, f0, f1, A, L);
    kstep<1, 8, true, true>(acc, f1, f0, A, L);
    kstep<2, 8, true, true>(acc, f0, f1, A, L);
    kstep<3, 8, true, true>(acc, f1, f0, A, L);
  }
  kstep<0, 8, false, true>(acc, f0, f1, A, L);
  kstep<1, 4, false, true>(acc, f1, f0, A, L);
  kstep<2, 0, false, true>(acc, f0, f1, A, L);
  kstep<3, 0, false, false>(acc, f1, f0, A, L);
}

__device__ __forceinline__ u32 pack2(float a, float b) {
  f32x2 t = {a, b};
  return __builtin_bit_cast(u32, __builtin_convertvector(t, bf16x2));
}
__device__ __forceinline__ float bf_lo(u32 v) { return __builtin_bit_cast(float, v << 16); }
__device__ __forceinline__ float bf_hi(u32 v) { return __builtin_bit_cast(float, v & 0xffff0000u); }

__device__ __forceinline__ void tile(const Params& P, int tid, int wave, u32 lane, char* smem) {
  const int r = P.rank;
  char* const myws = P.ws[r];
  const int g = r >> 2, b = (r >> 1) & 1, a = r & 1;
  const int bid = blockIdx.x;

  const u32 cnt_off = CNT_OFF + (u32)(bid * 4 + wave) * 4u;
  const u32 call = __builtin_amdgcn_readfirstlane(ld_sys(myws + cnt_off)) + 1u;
  if (lane == 0) st_agt(myws + cnt_off, call);
  const u32 par = call & 1u;
  STAMP(myws, 0);

  int row0, col0, nrem, tskip, j, e, cb, rb, dsrc;
  int nparts;
  if (tid < N_MAIN_WG) {
    const int xcd = tid & 7, o = tid >> 3, u = xcd >> 1;
    int rbi, cbi;
    if (u < 2) { rbi = o < 24 ? 1 + o % 3 : 0; cbi = o < 24 ? o / 3 : o - 24; }
    else { rbi = o & 3; cbi = o >> 2; }
    rb = u + 4 * rbi; cb = cbi * 2 + (xcd & 1);
    j = rb >> 1; e = rb & 1;
    row0 = (r ^ j) * 512 + b * 256 + e * 128; col0 = g * 2048 + cb * 128;
    nrem = j >= 2 ? 40 : 48; tskip = j >= 2 ? (j - 2) * 8 : 1000;
    nparts = j == 0 ? 1 : 0; dsrc = 1;
  } else {
    const int f = tid - N_MAIN_WG;
    const int qd = 1 + f / 32, rem = f & 31;
    e = (rem >> 3) & 1; cb = (rem & 7) * 2 + (rem >> 4);
    const int h = b ^ (qd & 1), g2 = g ^ (qd >> 1);
    row0 = r * 512 + h * 256 + e * 128; col0 = g2 * 2048 + cb * 128;
    nrem = 0; tskip = 1000; j = 0; rb = 0;
    nparts = 2; dsrc = qd * 2;
  }

  const u32 sbase = (u32)(uintptr_t)smem;
  const int wm = wave >> 1, wn = wave & 1;
  RdAddr A;
  {
    const u32 h = lane >> 5, m = lane & 31u, f = (lane >> 2) & 3u;
    A.xa[0] = sbase + ((u32)wm * 64u + m) * 64u + 16u * (h ^ f);
    A.xa[1] = sbase + ((u32)wm * 64u + m) * 64u + 16u * ((2u + h) ^ f);
    const u32 blk = (lane >> 4) & 1u, q = (lane & 15u) >> 2, pp = lane & 3u;
#pragma unroll
    for (int nt = 0; nt < 2; ++nt)
#pragma unroll
      for (int t = 0; t < 2; ++t) {
        const u32 row = 8u * h + 4u * t + q;
        const u32 ch = (u32)wn * 8u + (u32)nt * 4u + 2u * blk + (pp & 1u);
        A.wa[nt][t] = sbase + 256u * row + 16u * (ch ^ (((row & 3u) << 2) | ((row >> 2) & 3u))) + 8u * (pp >> 1);
      }
  }

  f32x16 acc[2][2];
#pragma unroll
  for (int i = 0; i < 2; ++i)
#pragma unroll
    for (int k = 0; k < 2; ++k)
#pragma unroll
      for (int z = 0; z < 16; ++z) acc[i][k][z] = 0.f;

  Loader L;
  L.m0base = sbase + (u32)wave * 4096u;
  bool dead = false;
  const int nph = nrem ? 2 : 1;
  for (int ph = 0; ph < nph; ++ph) {
    int n, ts;
    if (ph == 0) {
      n = 16; ts = 1000;
      if (wave < 2) {
        L.base = P.x + (size_t)((u32)(row0 + wave * 64) * 1024u);
        L.stride = 64u;
        const u32 c = (lane & 3u) ^ ((lane >> 4) & 3u);
#pragma unroll
        for (int k = 0; k < 4; ++k) L.v[k] = (lane >> 2) * 1024u + c * 16u + (u32)k * (16384u - 1024u);
      } else {
        L.base = P.w + (size_t)((u32)(wave - 2) * 16u * 8192u + (u32)col0 * 2u);
        L.stride = 32u * 8192u;
        const u32 q = lane >> 4;
#pragma unroll
        for (int k = 0; k < 4; ++k) L.v[k] = q * 8192u + (((lane & 15u) ^ ((q << 2) | (u32)k)) * 16u) + (u32)k * (4u * 8192u - 1024u);
      }
    } else {
      n = nrem; ts = tskip;
      STAMP(myws, 1);
      if (wave == 0) {
        const int fidx = j == 0 ? 16 + e : cb;
        const char* fp = myws + FIN_OFF + (u32)(((lane >> 5) * 18u + (u32)fidx) * 128u + (lane & 31u) * 4u);
        const u64 t0 = now();
        u32 st = 0;
        for (;;) {
          const u32 a0 = ld_sys(fp), a1 = ld_sys(fp + 2 * 18 * 128), a2 = ld_sys(fp + 4 * 18 * 128);
          const u32 tv = ld_sys(myws + TMO_OFF);
          const bool ok = a0 >= call && a1 >= call && a2 >= call;
          if (__builtin_amdgcn_ballot_w64(ok) == ~0ull) break;
          if (__builtin_amdgcn_readfirstlane(tv) == call || expired(t0)) {
            if (lane == 0) st_agt(myws + TMO_OFF, call);
            st = 1;
            break;
          }
          __builtin_amdgcn_s_sleep(4);
        }
        if (lane == 0) *(LDSP u32*)(uintptr_t)(sbase + 4u * STAGE_B) = st;
        __builtin_amdgcn_s_waitcnt(0);
      }
      asm volatile("s_barrier" ::: "memory");
      dead = __builtin_amdgcn_readfirstlane(*(volatile LDSP u32*)(uintptr_t)(sbase + 4u * STAGE_B)) != 0;
      STAMP(myws, 2);
      if (dead) break;
      if (wave < 2) L.base = myws + (size_t)(XS_OFF + par * XS_PAR + (u32)rb * 6u * PIECE + (u32)wave * 4096u);
      else          L.base = myws + (size_t)(WS_OFF + par * XS_PAR + (u32)cb * 6u * PIECE + (u32)(wave - 2) * 4096u);
      L.stride = 8192u;
#pragma unroll
      for (int k = 0; k < 4; ++k) L.v[k] = lane * 16u;
    }
    gemm_phase(acc, A, L, n, ts);
  }
  STAMP(myws, 3);

  const u32 tile_off = (u32)cb * PIECE + (u32)e * 32768u + (u32)wave * 8192u;
  const u32 l16 = lane * 16u;
  if (nparts == 0) {
    char* dst = myws + (size_t)(POUT_OFF + (u32)(j - 1) * 16u * PIECE + tile_off);
    char* fl = myws + (size_t)(FLOC_OFF + (u32)((j - 1) * 8 + e * 4 + wave) * 128u + (u32)cb * 4u);
    dst = uni(dst);
    asm volatile("s_nop 4" ::: "memory");
#pragma unroll
    for (int nt = 0; nt < 2; ++nt)
#pragma unroll
      for (int mt = 0; mt < 2; ++mt) {
        u32x4 v0, v1;
#pragma unroll
        for (int z = 0; z < 4; ++z) {
          v0[z] = dead ? 0x7fc07fc0u : pack2(acc[nt][mt][2 * z], acc[nt][mt][2 * z + 1]);
          v1[z] = dead ? 0x7fc07fc0u : pack2(acc[nt][mt][8 + 2 * z], acc[nt][mt][8 + 2 * z + 1]);
        }
        const u32 o = l16 + (u32)nt * 4096u;
        if (mt == 0) { st16<0>(o, v0, dst); st16<1024>(o, v1, dst); }
        else         { st16<2048>(o, v0, dst); st16<3072>(o, v1, dst); }
      }
    waitv<0>();
    if (lane == 0) st_sys(fl, call);
  } else {
    const u32 src = lane >> 2;
    const char* fp = (lane < (u32)(4 * nparts))
        ? myws + FOUT_OFF + ((u32)(dsrc - 1 + (int)src) * 16u + (u32)cb) * 128u + (u32)(e * 16 + wave * 4) * 4u + (lane & 3u) * 4u
        : myws + TMO_OFF;
    const u64 want = nparts == 2 ? 0xffull : 0xfull;
    if (!dead) {
      const u64 t0 = now();
      for (;;) {
        const u32 v = ld_sys(fp);
        const u64 ge = __builtin_amdgcn_ballot_w64(v >= call);
        const u64 eq = __builtin_amdgcn_ballot_w64(v == call);
        if (eq >> 63) { dead = true; break; }
        if ((ge & want) == want) break;
        if (expired(t0)) {
          if (lane == 0) st_agt(myws + TMO_OFF, call);
          dead = true;
          break;
        }
        __builtin_amdgcn_s_sleep(2);
      }
    }
    STAMP(myws, 4);
    if (!dead) {
      for (int s = 0; s < nparts; ++s) {
        const char* pb = uni(myws + (size_t)(PIN_OFF + par * PIN_PAR + (u32)(dsrc - 1 + s) * 16u * PIECE + tile_off));
        u32x4 pv[8];
        asm volatile("s_nop 4" ::: "memory");
        ld16i<0>(pv[0], l16, pb); ld16i<1024>(pv[1], l16, pb); ld16i<2048>(pv[2], l16, pb); ld16i<3072>(pv[3], l16, pb);
        const u32 l16b = l16 + 4096u;
        ld16i<0>(pv[4], l16b, pb); ld16i<1024>(pv[5], l16b, pb); ld16i<2048>(pv[6], l16b, pb); ld16i<3072>(pv[7], l16b, pb);
        asm volatile("s_waitcnt vmcnt(0)" : "+v"(pv[0]), "+v"(pv[1]), "+v"(pv[2]), "+v"(pv[3]),
                     "+v"(pv[4]), "+v"(pv[5]), "+v"(pv[6]), "+v"(pv[7]) :: "memory");
#pragma unroll
        for (int nt = 0; nt < 2; ++nt)
#pragma unroll
          for (int mt = 0; mt < 2; ++mt)
#pragma unroll
            for (int sx = 0; sx < 2; ++sx)
#pragma unroll
              for (int z = 0; z < 4; ++z) {
                const u32 u = pv[(nt * 2 + mt) * 2 + sx][z];
                acc[nt][mt][8 * sx + 2 * z] += bf_lo(u);
                acc[nt][mt][8 * sx + 2 * z + 1] += bf_hi(u);
              }
      }
    }
    const u32 h = lane >> 5, m = lane & 31u;
    char* ob = P.out + (size_t)((u32)(row0 - r * 512 + wm * 64) * 8192u + (u32)(col0 + wn * 64) * 2u);
#pragma unroll
    for (int nt = 0; nt < 2; ++nt)
#pragma unroll
      for (int mt = 0; mt < 2; ++mt)
#pragma unroll
        for (int sx = 0; sx < 2; ++sx) {
          u32x4 v;
#pragma unroll
          for (int z = 0; z < 4; ++z) {
            const float lo = acc[nt][mt][8 * sx + 2 * z], hi = acc[nt][mt][8 * sx + 2 * z + 1];
            v[z] = dead ? 0x7fc07fc0u : pack2(lo < 0.f ? 0.f : lo, hi < 0.f ? 0.f : hi);
          }
          *(u32x4*)(ob + (size_t)(((u32)mt * 32u + m) * 8192u + (u32)(nt * 32 + sx * 16) * 2u + h * 16u)) = v;
        }
  }
  STAMP(myws, 5);
}

__global__ __launch_bounds__(256, 2) void fused_gemm_rs(Params P) {
  extern __shared__ __attribute__((aligned(16))) char smem[];
  const u32 lane = threadIdx.x & 63u;
  const int wave = __builtin_amdgcn_readfirstlane(threadIdx.x >> 6);
  const int bid = blockIdx.x;
  if (bid < N_PUSH_WG) pusher(P, wave, lane, smem);
  else if (bid >= 256 && bid < 256 + N_PUSH_WG) {
    for (int i = 0; i < 6; ++i) __builtin_amdgcn_s_sleep(16);
  } else tile(P, bid < 256 ? bid - N_PUSH_WG : bid - 2 * N_PUSH_WG, wave, lane, smem);
}

}

extern "C" void kernel_launch(int rank, int nranks, ncclComm_t comm,
                              hipStream_t stream,
                              void* const* d_in, const int* in_sizes, int n_in,
                              void* d_out, int out_size,
                              const int* in_dtypes, int out_dtype,
                              void* const* d_out_all,
                              void* const* d_ws_all, size_t ws_bytes) {
  if (nranks != NR || n_in < 2 || ws_bytes < (size_t)WS_NEED || (unsigned)rank >= (unsigned)NR) {
    fprintf(stderr, "unsupported configuration\n");
    return;
  }
  Params P;
  P.x = (const char*)d_in[0];
  P.w = (const char*)d_in[1];
  P.out = (char*)d_out;
  for (int i = 0; i < NR; ++i) P.ws[i] = (char*)d_ws_all[i];
  P.rank = rank;
  HIP_CHECK(hipFuncSetAttribute((const void*)fused_gemm_rs, hipFuncAttributeMaxDynamicSharedMemorySize, (int)LDS_BYTES));
  hipLaunchKernelGGL(fused_gemm_rs, dim3(N_WG), dim3(256), LDS_BYTES, stream, P);
  HIP_CHECK(hipGetLastError());
}
```

```cpp
#include <hip/hip_runtime.h>
#include <rccl/rccl.h>
#include <cstdio>
#include <cstdint>

#define HIP_CHECK(expr) do { hipError_t _e = (expr); if (_e != hipSuccess) { \
    fprintf(stderr, "HIP error %s at %s:%d\n", hipGetErrorString(_e), __FILE__, __LINE__); \
    return; } } while (0)

namespace {

typedef unsigned u32;
typedef unsigned long long u64;
typedef u32 u32x4 __attribute__((ext_vector_type(4)));
typedef float f32x2 __attribute__((ext_vector_type(2)));
typedef float f32x16 __attribute__((ext_vector_type(16)));
typedef __bf16 bf16x2 __attribute__((ext_vector_type(2)));
typedef __bf16 bf16x8 __attribute__((ext_vector_type(8)));
typedef short s16x4 __attribute__((ext_vector_type(4)));
typedef short s16x8 __attribute__((ext_vector_type(8)));
#define LDSP __attribute__((address_space(3)))

constexpr int NR = 8;
constexpr u32 PIECE = 65536u;
constexpr u32 MIB = 1u << 20;
#define NS 2
#define OX 14
#define TAU 119
#define T0 100
#define PFG 0u
#define EARLY 100
#define TAU1 100
#define LAG 1
#define WIN 3
constexpr int NIN = 30;
constexpr int NOUT = 16 - NS;
constexpr int NOUT1 = 16 + 6 * NS;
constexpr u32 XS_OFF = 0;
constexpr u32 XS_PAR = 6u * MIB;
constexpr u32 WS_OFF = 12u * MIB;
constexpr u32 PIN_OFF = 24u * MIB;
constexpr u32 PIN_PAR = 7u * MIB;
constexpr u32 POUT_OFF = 38u * MIB;
constexpr u32 FLG_OFF = 45u * MIB;
constexpr u32 FIN_OFF = FLG_OFF;
constexpr u32 FLOC_OFF = FLG_OFF + 16384u;
constexpr u32 CNT_OFF = FLG_OFF + 32768u;
constexpr u32 TMO_OFF = FLG_OFF + 49152u;
constexpr u32 DUMMY_OFF = FLG_OFF + 65536u;
constexpr u32 DBG_OFF = FLG_OFF + 131072u;
constexpr u32 WS_NEED = FLG_OFF + 262144u;

constexpr int N_PUSH_WG = 56;
constexpr int N_MAIN_WG = 256;
constexpr int N_FIN_WG = 96;
constexpr int N_WG = 2 * N_PUSH_WG + N_MAIN_WG + N_FIN_WG;
constexpr u32 STAGE_B = 16384u;
constexpr u32 LDS_BYTES = 4u * STAGE_B + 1024u;
constexpr u64 DEADLINE = 100000000ull;
constexpr u32 DEAD = 0x7fffffffu;

struct Params {
  const char* x;
  const char* w;
  char* out;
  char* ws[NR];
  int rank;
};

#define SB() __builtin_amdgcn_sched_barrier(0)
template <int N> __device__ __forceinline__ void waitv() { asm volatile("s_waitcnt vmcnt(%0)" :: "i"(N) : "memory"); }

__device__ __forceinline__ u64 now() { return __builtin_amdgcn_s_memrealtime(); }
template <typename T> __device__ __forceinline__ T* uni(T* p) {
  const u64 v = (u64)(uintptr_t)p;
  const u32 lo = __builtin_amdgcn_readfirstlane((u32)v), hi = __builtin_amdgcn_readfirstlane((u32)(v >> 32));
  return (T*)(uintptr_t)(((u64)hi << 32) | lo);
}
__device__ __forceinline__ u32 ld_sys(const void* p) {
  return __hip_atomic_load((const u32*)p, __ATOMIC_RELAXED, __HIP_MEMORY_SCOPE_SYSTEM);
}
__device__ __forceinline__ void st_sys(void* p, u32 v) {
  __hip_atomic_store((u32*)p, v, __ATOMIC_RELAXED, __HIP_MEMORY_SCOPE_SYSTEM);
}
__device__ __forceinline__ void st_agt(void* p, u32 v) {
  __hip_atomic_store((u32*)p, v, __ATOMIC_RELAXED, __HIP_MEMORY_SCOPE_AGENT);
}
__device__ __forceinline__ bool give_up(u64 t0, const char* ws, u32 call) {
  const u64 dt = now() - t0;
  if (!__builtin_amdgcn_readfirstlane((int)(dt > 50000ull))) return false;
  __builtin_amdgcn_s_sleep(64);
  if (__builtin_amdgcn_readfirstlane((int)(dt > DEADLINE))) return true;
  return __builtin_amdgcn_readfirstlane(ld_sys(ws + TMO_OFF)) == call;
}
__device__ __forceinline__ void sleep_until(u64 tstart, u32 t) {
  for (;;) {
    const u32 el = (u32)__builtin_amdgcn_readfirstlane((u32)(now() - tstart));
    if (el >= t) break;
    if (t - el > 40u) __builtin_amdgcn_s_sleep(12); else __builtin_amdgcn_s_sleep(1);
  }
}
template <int IMM> __device__ __forceinline__ void st16(u32 off, u32x4 v, char* base) {
  asm volatile("global_store_dwordx4 %0, %1, %2 offset:%3 sc0 sc1\n\ts_nop 1" :: "v"(off), "v"(v), "s"(base), "i"(IMM) : "memory");
}
template <int IMM> __device__ __forceinline__ void st16p(u32 off, u32x4 v, char* base) {
  asm volatile("global_store_dwordx4 %0, %1, %2 offset:%3\n\ts_nop 1" :: "v"(off), "v"(v), "s"(base), "i"(IMM) : "memory");
}
#define OST st16
__device__ __forceinline__ void dma16x4(u32 lds, const u32 (&o)[4], const char* base) {
  asm volatile("s_mov_b32 m0, %0\n\ts_nop 3\n\t"
               "global_load_lds_dwordx4 %1, %5 sc0 sc1\n\t"
               "global_load_lds_dwordx4 %2, %5 offset:1024 sc0 sc1\n\t"
               "global_load_lds_dwordx4 %3, %5 offset:2048 sc0 sc1\n\t"
               "global_load_lds_dwordx4 %4, %5 offset:3072 sc0 sc1"
               :: "s"(lds), "v"(o[0]), "v"(o[1]), "v"(o[2]), "v"(o[3]), "s"(base) : "memory");
}
__device__ __forceinline__ void dma4(u32 lds, u32 off, const char* base) {
  asm volatile("s_mov_b32 m0, %0\n\ts_nop 3\n\t"
               "global_load_lds_dword %1, %2 sc0 sc1"
               :: "s"(lds), "v"(off), "s"(base) : "memory");
}
__device__ __forceinline__ void st4_lane0(u32 off, u32 val, char* base) {
  asm volatile("s_mov_b64 exec, 1\n\ts_nop 3\n\tglobal_store_dword %0, %1, %2 sc0 sc1\n\ts_mov_b64 exec, -1"
               :: "v"(off), "v"(val), "s"(base) : "memory");
}

#define STAMP(ws, idx) do {} while (0)

__device__ __forceinline__ void piece_of(int q, int d, bool& isx, int& idx) {
  const int t = q - NS, u = q - NS - 12;
  int j = (t >> 1) + 2;
  j += j >= d ? 1 : 0;
  j = j == 8 ? 1 : j;
  isx = (q >= NS && u < 0) || (u >= OX && u < OX + 2);
  idx = q < NS ? q : (u < 0 ? 2 * j + (t & 1) : (u < OX ? NS + u : (u < OX + 2 ? u - OX : NS + u - 2)));
}
__device__ __forceinline__ int slot_w(int k) { return k < NS ? k : (k - NS < OX ? 12 + k : 14 + k); }
__device__ __forceinline__ int slot_x(int j, int e, int d) {
  return j == 0 ? NS + 12 + OX + e : NS + 2 * (j == 1 ? 5 : j - 2 - (j > d ? 1 : 0)) + e;
}
__device__ __forceinline__ u32 lds_rd(u32 a) { return __builtin_amdgcn_readfirstlane(*(volatile LDSP u32*)(uintptr_t)a); }
__device__ __forceinline__ void lds_wr(u32 a, u32 v) { *(volatile LDSP u32*)(uintptr_t)a = v; }

__device__ __forceinline__ void mover(const Params& P, int wave, u32 lane, char* smem) {
  __builtin_amdgcn_s_setprio(3);
  const u64 tstart = now();
  const int s = P.rank;
  char* const myws = P.ws[s];
  const int pair = wave >> 1;
  const int d = 1 + ((int)(blockIdx.x >> 3) + 3 * pair) % 7;
  const int p = (blockIdx.x & 7) * 2 + pair;
  const u32 r = (u32)(s ^ d) & (u32)(NR - 1);
  const int nin = d == 1 ? 0 : NIN;
  const int np = d == 1 ? NOUT1 : NIN + NOUT;
  const u32 ring = (u32)(uintptr_t)smem + (u32)pair * 32768u;
  const u32 ctl = (u32)(uintptr_t)smem + 65536u + (u32)pair * 512u;
  const u32 cnt_off = CNT_OFF + (u32)(blockIdx.x * 4 + wave) * 4u;

  if ((wave & 1) == 0) {
    const char* const xin = P.x;
    const char* const win = P.w;
    const int rg = r >> 2, rb_ = (r >> 1) & 1, ra = r & 1;
    u32 xo[4], wo[4], oo[4];
    {
      const u32 xc = (lane & 3u) ^ ((lane >> 4) & 3u), q4 = lane >> 4;
#pragma unroll
      for (int k = 0; k < 4; ++k) {
        xo[k] = ((u32)(p & 1) * 64u + (u32)k * 16u + (lane >> 2)) * 1024u + (u32)(p >> 1) * 64u + xc * 16u - (u32)k * 1024u;
        wo[k] = ((u32)(p >> 1) * 32u + (u32)(p & 1) * 16u + (u32)k * 4u + q4) * 8192u
              + (((lane & 15u) ^ ((q4 << 2) | (u32)k)) * 16u) - (u32)k * 1024u;
        oo[k] = (u32)p * 4096u + lane * 16u;
      }
    }
    u32 poll_off;
    {
      const int i = (int)lane < np - nin ? (int)lane : 0;
      const int jj = d != 1 ? d : (i < 6 * NS ? 2 + i / NS : 1);
      const int kk = d != 1 ? NS + i : (i < 6 * NS ? i % NS : i - 6 * NS);
      poll_off = FLOC_OFF + (u32)((jj - 1) * 8 + (p >> 1)) * 128u + (u32)kk * 4u;
    }
#define ISSUE_LOADS(QQ) do { \
      const int q_ = (QQ); \
      const u32 dst_ = ring + (u32)(q_ & 7) * 4096u; \
      if (q_ < nin) { \
        bool isx_; int ix_; \
        piece_of(q_, d, isx_, ix_); \
        if (isx_) dma16x4(dst_, xo, xin + (size_t)(((r ^ (u32)(ix_ >> 1)) * 512u + (u32)rb_ * 256u + (u32)(ix_ & 1) * 128u) * 1024u + (u32)ra * 512u)); \
        else      dma16x4(dst_, wo, win + (size_t)((u32)ra * 256u * 8192u + (u32)rg * 4096u + (u32)(ra ? 15 - ix_ : ix_) * 256u)); \
      } else { \
        const int i_ = q_ - nin; \
        const int jj_ = d != 1 ? d : (i_ < 6 * NS ? 2 + i_ / NS : 1); \
        const int kk_ = d != 1 ? NS + i_ : (i_ < 6 * NS ? i_ % NS : i_ - 6 * NS); \
        dma16x4(dst_, oo, myws + (size_t)(POUT_OFF + (u32)((jj_ - 1) * 16 + kk_) * PIECE)); \
      } \
    } while (0)

    lds_wr(ctl, 0u);
    lds_wr(ctl + 256u + lane * 4u, 0u);
    asm volatile("s_waitcnt lgkmcnt(0)\n\ts_barrier" ::: "memory");
    dma4(ring + 7u * 4096u, cnt_off, myws);
    int q0 = 0;
    u32 pub = 0;
    if (nin) {
      ISSUE_LOADS(0); ISSUE_LOADS(1); ISSUE_LOADS(2);
      waitv<8>();
      lds_wr(ctl, 1u);
      q0 = 3; pub = 1;
    } else {
      waitv<0>();
    }
    const u32 call = lds_rd(ring + 7u * 4096u) + 1u;
    st4_lane0(cnt_off, call, myws);
    STAMP(myws, 0);

    if (!nin) sleep_until(tstart, (u32)(T0 + (NS + 2) * TAU + 500));
    u64 ready = 0;
    u32 cons = 0;
    bool dead = false;
    for (int q = q0; q < np; ++q) {
      bool rdy = true;
      if (q >= nin) {
        if (!((ready >> (q - nin)) & 1ull)) {
          const u32 pv = *(volatile LDSP u32*)(uintptr_t)(ctl + 256u + lane * 4u);
          ready |= __builtin_amdgcn_ballot_w64(pv >= call);
        }
        rdy = (ready >> (q - nin)) & 1ull;
      }
      if (cons + 8u <= (u32)q) cons = lds_rd(ctl + 4u);
      if (!rdy || cons + 8u <= (u32)q) {
        waitv<0>();
        pub = (u32)q;
        lds_wr(ctl, pub);
        const u64 t0 = now();
        for (;;) {
          if (!rdy) {
            const u32 pv = ld_sys(myws + poll_off);
            ready |= __builtin_amdgcn_ballot_w64(pv >= call);
            rdy = (ready >> (q - nin)) & 1ull;
          } else {
            __builtin_amdgcn_s_sleep(2);
          }
          cons = lds_rd(ctl + 4u);
          if (rdy && cons + 8u > (u32)q) break;
          if (give_up(t0, myws, call)) { dead = true; break; }
        }
        if (dead) break;
      }
      if (q + 4 >= nin) dma4(ctl + 256u, poll_off, myws);
      ISSUE_LOADS(q);
      if (q + 3 >= nin) waitv<10>(); else waitv<8>();
      if (q - 1 > (int)pub) { pub = (u32)(q - 1); lds_wr(ctl, pub); }
      if (q == nin - 1) STAMP(myws, 1);
    }
    waitv<0>();
    if (dead) st4_lane0(TMO_OFF, call, myws);
    lds_wr(ctl, dead ? DEAD : (u32)np);
    STAMP(myws, 3);
    return;
  }

  char* const rws = uni(P.ws[r]);
  lds_wr(ctl + 4u, 0u);
  asm volatile("s_waitcnt lgkmcnt(0)\n\ts_barrier" ::: "memory");
  const u32 call = __builtin_amdgcn_readfirstlane(ld_sys(myws + cnt_off)) + 1u;
  st4_lane0(cnt_off, call, myws);
  STAMP(myws, 0);
  const u32 par = call & 1u;
  const int slot = d - 2;
  const u32 dl = (u32)p * 4096u + lane * 16u;
  const u32 pin0 = PIN_OFF + par * PIN_PAR + (u32)(d == 1 ? 0 : NOUT1 + slot * NOUT) * PIECE;
#define FLAG(QQ) do { \
    const int q_ = (QQ); \
    const bool real_ = q_ >= 0 && q_ < nin; \
    const u32 o_ = real_ ? FIN_OFF + (u32)(slot * 32 + q_) * 64u + (u32)p * 4u : DUMMY_OFF + (u32)(blockIdx.x * 4 + wave) * 128u; \
    st4_lane0(o_, call, uni(real_ ? rws : myws)); \
  } while (0)

  u32 full = 0, tnext = 0;
  for (int q = 0; q < np; ++q) {
    {
      const u32 target = d == 1 ? tnext : (u32)q * (u32)TAU + (u32)T0;
      sleep_until(tstart, target);
    }
    if (full <= (u32)q) {
      full = lds_rd(ctl);
      if (full <= (u32)q) {
        const u64 t0 = now();
        bool drained = !(q > 0 && q < nin + LAG);
        for (;;) {
          __builtin_amdgcn_s_sleep(1);
          full = lds_rd(ctl);
          if (full > (u32)q) break;
          const u64 dt = now() - t0;
          if (!drained && __builtin_amdgcn_readfirstlane((int)(dt > 30ull))) {
            drained = true;
            waitv<0>();
#pragma unroll
            for (int z = LAG; z >= 1; --z) FLAG(q - z);
          }
          if (__builtin_amdgcn_readfirstlane((int)(dt > 2ull * DEADLINE - 1000ull))) { full = DEAD; break; }
        }
      }
      if (full == DEAD) break;
    }
    tnext = (u32)__builtin_amdgcn_readfirstlane((u32)(now() - tstart)) + (u32)TAU1;
    const u32 la = ring + (u32)(q & 7) * 4096u + lane * 16u;
    const u32x4 d0 = *(volatile LDSP u32x4*)(uintptr_t)la;
    const u32x4 d1 = *(volatile LDSP u32x4*)(uintptr_t)(la + 1024u);
    const u32x4 d2 = *(volatile LDSP u32x4*)(uintptr_t)(la + 2048u);
    const u32x4 d3 = *(volatile LDSP u32x4*)(uintptr_t)(la + 3072u);
    lds_wr(ctl + 4u, (u32)q + 1u);
    u32 o;
    if (q < nin) {
      bool isx; int ix;
      piece_of(q, d, isx, ix);
      o = (isx ? XS_OFF : WS_OFF) + par * XS_PAR + (u32)(ix * 6 + slot) * PIECE;
    } else {
      o = pin0 + (u32)(q - nin) * PIECE;
    }
    char* const dst = rws + (size_t)o;
    st16<0>(dl, d0, dst);
    st16<1024>(dl, d1, dst);
    st16<2048>(dl, d2, dst);
    st16<3072>(dl, d3, dst);
    if (nin && q < nin + LAG) {
      waitv<5 * LAG>();
      FLAG(q - LAG);
    } else {
      waitv<4 * WIN>();
    }
    if (q == NS + 11) STAMP(myws, 1);
    if (q == nin - 1) STAMP(myws, 2);
    if (q == np - 1) STAMP(myws, 4);
  }
  waitv<0>();
  STAMP(myws, 3);
}

__device__ __forceinline__ void agent(const Params& P, int wave, u32 lane, char* smem) {
  const u64 t0 = now();
  char* const myws = P.ws[P.rank];
  const int xcd = blockIdx.x & 7, w = (int)((blockIdx.x - 256) >> 3) * 4 + wave;
  if (w >= 24) return;
  const int sl = w % 6, qt = w / 6;
  const u32 cnt_off = CNT_OFF + (u32)(blockIdx.x * 4 + wave) * 4u;
  const u32 call = __builtin_amdgcn_readfirstlane(ld_sys(myws + cnt_off)) + 1u;
  if (lane == 0) st_agt(myws + cnt_off, call);
  const u32 par = call & 1u;
  const u32 lb = (u32)(uintptr_t)smem + (u32)wave * 4096u;
  const u32 lo = (u32)qt * 16384u + lane * PFG;
  for (int q = 0; q < NIN; ++q) {
    bool isx; int ix;
    piece_of(q, sl + 2, isx, ix);
    if (isx ? (ix & 3) != (xcd >> 1) : (ix & 1) != (xcd & 1)) continue;
    const char* fp = myws + FIN_OFF + (u32)(sl * 32 + q) * 64u + (u32)qt * 16u + (lane & 3u) * 4u;
    sleep_until(t0, (u32)(T0 + q * TAU + EARLY));
    for (;;) {
      const u32 v = ld_sys(fp);
      if ((__builtin_amdgcn_ballot_w64(v >= call) & 15ull) == 15ull) break;
      if (give_up(t0, myws, call)) return;
      __builtin_amdgcn_s_sleep(2);
    }
    const char* src = myws + (size_t)((isx ? XS_OFF : WS_OFF) + par * XS_PAR + (u32)(ix * 6 + sl) * PIECE);
#pragma unroll
    for (u32 i = 0; i < 16384u / (64u * (PFG ? PFG : 64u)); ++i) dma4(lb + i * 256u, lo + i * 64u * PFG, src);
  }
  waitv<0>();
}

struct Frag {
  bf16x8 x[2][2];
  bf16x8 w[2][2];
};

struct Loader {
  const char* base;
  u32 stride;
  u32 v[4];
  u32 m0base;
};

template <int STAGE> __device__ __forceinline__ void glds_pair(const Loader& L, int half) {
  const u32 m0v = L.m0base + STAGE * STAGE_B;
  if (half == 0)
    asm volatile("s_mov_b32 m0, %0\n\ts_nop 0\n\t"
                 "global_load_lds_dwordx4 %1, %3 sc0 sc1\n\t"
                 "global_load_lds_dwordx4 %2, %3 offset:1024 sc0 sc1"
                 :: "s"(m0v), "v"(L.v[0]), "v"(L.v[1]), "s"(L.base) : "memory");
  else
    asm volatile("s_mov_b32 m0, %0\n\ts_nop 0\n\t"
                 "global_load_lds_dwordx4 %1, %3 offset:2048 sc0 sc1\n\t"
                 "global_load_lds_dwordx4 %2, %3 offset:3072 sc0 sc1"
                 :: "s"(m0v), "v"(L.v[2]), "v"(L.v[3]), "s"(L.base) : "memory");
}

struct RdAddr { u32 xa[2]; u32 wa[2][2]; };

template <int STAGE> __device__ __forceinline__ bf16x8 rd_x(const RdAddr& A, int mt, int ks) {
  return *(const LDSP bf16x8*)(uintptr_t)(A.xa[ks] + (u32)(STAGE * STAGE_B + mt * 2048));
}
template <int STAGE> __device__ __forceinline__ bf16x8 rd_w(const RdAddr& A, int nt, int ks) {
  s16x4 lo = __builtin_amdgcn_ds_read_tr16_b64_v4i16((LDSP s16x4*)(uintptr_t)(A.wa[nt][0] + (u32)(STAGE * STAGE_B + 8192 + ks * 4096)));
  s16x4 hi = __builtin_amdgcn_ds_read_tr16_b64_v4i16((LDSP s16x4*)(uintptr_t)(A.wa[nt][1] + (u32)(STAGE * STAGE_B + 8192 + ks * 4096)));
  s16x8 v = __builtin_shufflevector(lo, hi, 0, 1, 2, 3, 4, 5, 6, 7);
  return __builtin_bit_cast(bf16x8, v);
}
#define MFMA(nt, mt, ks) acc[nt][mt] = __builtin_amdgcn_mfma_f32_32x32x16_bf16(cur.w[nt][ks], cur.x[mt][ks], acc[nt][mt], 0, 0, 0)

template <int S, int VM, bool LOAD, bool READ>
__device__ __forceinline__ void kstep(f32x16 (&acc)[2][2], const Frag& cur, Frag& nxt, const RdAddr& A, Loader& L) {
  constexpr int SN = (S + 1) & 3;
  if (READ) {
    __builtin_amdgcn_s_waitcnt((VM & 15) | (7 << 4) | (0 << 8) | ((VM >> 4) << 14));
    asm volatile("s_barrier" ::: "memory");
  } else {
    __builtin_amdgcn_s_waitcnt(15 | (7 << 4) | (0 << 8) | (3 << 14));
  }
  SB();
  MFMA(0, 0, 0); SB();
  if (READ) { nxt.w[0][0] = rd_w<SN>(A, 0, 0); nxt.x[0][0] = rd_x<SN>(A, 0, 0); } SB();
  MFMA(0, 1, 0); SB();
  if (READ) { nxt.x[1][0] = rd_x<SN>(A, 1, 0); nxt.w[1][0] = rd_w<SN>(A, 1, 0); } SB();
  MFMA(1, 0, 0); SB();
  if (LOAD) glds_pair<S>(L, 0);
  SB();
  MFMA(1, 1, 0); SB();
  if (READ) { nxt.w[0][1] = rd_w<SN>(A, 0, 1); nxt.x[0][1] = rd_x<SN>(A, 0, 1); } SB();
  MFMA(0, 0, 1); SB();
  if (READ) { nxt.x[1][1] = rd_x<SN>(A, 1, 1); nxt.w[1][1] = rd_w<SN>(A, 1, 1); } SB();
  MFMA(0, 1, 1); SB();
  if (LOAD) { glds_pair<S>(L, 1); L.base += L.stride; }
  SB();
  MFMA(1, 0, 1); SB();
  MFMA(1, 1, 1); SB();
}

template <int S> __device__ __forceinline__ void stage_load(Loader& L) {
  glds_pair<S>(L, 0); glds_pair<S>(L, 1); L.base += L.stride;
}

__device__ __forceinline__ void gemm_phase(f32x16 (&acc)[2][2], const RdAddr& A, Loader& L, int n, int tskip) {
  Frag f0, f1;
  L.base = uni(L.base + (tskip == 0 ? PIECE : 0u));
  L.stride = __builtin_amdgcn_readfirstlane(L.stride);
  asm volatile("s_nop 4" ::: "memory");
  stage_load<0>(L); stage_load<1>(L); stage_load<2>(L); stage_load<3>(L);
  waitv<12>();
  asm volatile("s_barrier" ::: "memory");
#pragma unroll
  for (int ks = 0; ks < 2; ++ks)
#pragma unroll
    for (int i = 0; i < 2; ++i) { f0.w[i][ks] = rd_w<0>(A, i, ks); f0.x[i][ks] = rd_x<0>(A, i, ks); }
  SB();
  for (int t = 0; t < n - 4; t += 4) {
    if (t + 4 == tskip) L.base += PIECE;
    kstep<0, 8, true, true>(acc, f0, f1, A, L);
    kstep<1, 8, true, true>(acc, f1, f0, A, L);
    kstep<2, 8, true, true>(acc, f0, f1, A, L);
    kstep<3, 8, true, true>(acc, f1, f0, A, L);
  }
  kstep<0, 8, false, true>(acc, f0, f1, A, L);
  kstep<1, 4, false, true>(acc, f1, f0, A, L);
  kstep<2, 0, false, true>(acc, f0, f1, A, L);
  kstep<3, 0, false, false>(acc, f1, f0, A, L);
}

__device__ __forceinline__ u32 pack2(float a, float b) {
  f32x2 t = {a, b};
  return __builtin_bit_cast(u32, __builtin_convertvector(t, bf16x2));
}
__device__ __forceinline__ float bf_lo(u32 v) { return __builtin_bit_cast(float, v << 16); }
__device__ __forceinline__ float bf_hi(u32 v) { return __builtin_bit_cast(float, v & 0xffff0000u); }

__device__ __forceinline__ void zero8k(u32 l16, u32 l16b, char* pb) {
  const u32x4 z4 = {0u, 0u, 0u, 0u};
  asm volatile("s_nop 4" ::: "memory");
  st16<0>(l16, z4, pb); st16<1024>(l16, z4, pb); st16<2048>(l16, z4, pb); st16<3072>(l16, z4, pb);
  st16<0>(l16b, z4, pb); st16<1024>(l16b, z4, pb); st16<2048>(l16b, z4, pb); st16<3072>(l16b, z4, pb);
}

__device__ __forceinline__ void tile(const Params& P, int tid, int wave, u32 lane, char* smem) {
  const int r = P.rank;
  char* const myws = P.ws[r];
  const int g = r >> 2, b = (r >> 1) & 1, a = r & 1;
  const int bid = blockIdx.x;
  const u64 tstart = now();

  const u32 cnt_off = CNT_OFF + (u32)(bid * 4 + wave) * 4u;
  const u32 call = __builtin_amdgcn_readfirstlane(ld_sys(myws + cnt_off)) + 1u;
  if (lane == 0) st_agt(myws + cnt_off, call);
  const u32 par = call & 1u;
  STAMP(myws, 0);

  int row0, col0, nrem, tskip, j, e, k, rb;
  int pi0 = -1, pi1 = -1;
  bool fin;
  if (tid < N_MAIN_WG) {
    const int xcd = tid & 7, o = tid >> 3, u = xcd >> 1;
    int rbi, cbi;
    if (u < 2) { rbi = o < 24 ? 1 + o % 3 : 0; cbi = o < 24 ? o / 3 : o - 24; }
    else { rbi = o & 3; cbi = o >> 2; }
    rb = u + 4 * rbi; k = cbi * 2 + (xcd & 1);
    j = rb >> 1; e = rb & 1;
    row0 = (r ^ j) * 512 + b * 256 + e * 128; col0 = g * 2048 + (a ? 15 - k : k) * 128;
    nrem = j >= 2 ? 40 : 48; tskip = j >= 2 ? (j - 2) * 8 : 1000;
    fin = j == 0;
    if (j == 0) pi0 = 6 * NS + 15 - k;
    else if (j >= 2 && k >= 16 - NS) pi0 = ((j ^ 1) - 2) * NS + 15 - k;
  } else {
    const int f = tid - N_MAIN_WG;
    const int qd = 1 + f / 32, rem = f & 31;
    e = (rem >> 3) & 1; k = (rem & 7) * 2 + (rem >> 4);
    const int h = b ^ (qd & 1), g2 = g ^ (qd >> 1);
    row0 = r * 512 + h * 256 + e * 128; col0 = g2 * 2048 + (a ? 15 - k : k) * 128;
    nrem = 0; tskip = 1000; j = 0; rb = 0;
    fin = true;
    const int iA = k >= NS ? NOUT1 + (2 * qd - 2) * NOUT + k - NS : -1;
    const int iB = 15 - k >= NS ? NOUT1 + (2 * qd - 1) * NOUT + 15 - k - NS : -1;
    if (k < 8) { pi0 = iA >= 0 ? iA : iB; pi1 = iA >= 0 ? iB : -1; }
    else       { pi0 = iB >= 0 ? iB : iA; pi1 = iB >= 0 ? iA : -1; }
  }

  const u32 sbase = (u32)(uintptr_t)smem;
  const int wm = wave >> 1, wn = wave & 1;
  RdAddr A;
  {
    const u32 h = lane >> 5, m = lane & 31u, f = (lane >> 2) & 3u;
    A.xa[0] = sbase + ((u32)wm * 64u + m) * 64u + 16u * (h ^ f);
    A.xa[1] = sbase + ((u32)wm * 64u + m) * 64u + 16u * ((2u + h) ^ f);
    const u32 blk = (lane >> 4) & 1u, q = (lane & 15u) >> 2, pp = lane & 3u;
#pragma unroll
    for (int nt = 0; nt < 2; ++nt)
#pragma unroll
      for (int t = 0; t < 2; ++t) {
        const u32 row = 8u * h + 4u * t + q;
        const u32 ch = (u32)wn * 8u + (u32)nt * 4u + 2u * blk + (pp & 1u);
        A.wa[nt][t] = sbase + 256u * row + 16u * (ch ^ (((row & 3u) << 2) | ((row >> 2) & 3u))) + 8u * (pp >> 1);
      }
  }

  f32x16 acc[2][2];
#pragma unroll
  for (int i = 0; i < 2; ++i)
#pragma unroll
    for (int k = 0; k < 2; ++k)
#pragma unroll
      for (int z = 0; z < 16; ++z) acc[i][k][z] = 0.f;

  const u32 tile_off = (u32)e * 32768u + (u32)wave * 8192u;
  const u32 l16 = lane * 16u, l16b = lane * 16u + 4096u;
  char* const pin = myws + (size_t)(PIN_OFF + par * PIN_PAR + tile_off);
  const int nph = nrem ? 2 : 1;
  const u32 tin = (u32)(T0 + EARLY + TAU * max(slot_w(k), slot_x(j, e, j == 7 ? 2 : 7)));
  const int addph = fin ? nph - 1 : 0;
  const int plast = fin ? (pi1 >= 0 ? pi1 : pi0) : -1;

  Loader L;
  L.m0base = sbase + (u32)wave * 4096u;
  bool dead = false;
  for (int ph = 0; ph < nph; ++ph) {
    int n, ts;
    if (ph == 0) {
      n = 16; ts = 1000;
      if (wave < 2) {
        L.base = P.x + (size_t)((u32)(row0 + wave * 64) * 1024u);
        L.stride = 64u;
        const u32 c = (lane & 3u) ^ ((lane >> 4) & 3u);
#pragma unroll
        for (int k = 0; k < 4; ++k) L.v[k] = (lane >> 2) * 1024u + c * 16u + (u32)k * (16384u - 1024u);
      } else {
        L.base = P.w + (size_t)((u32)(wave - 2) * 16u * 8192u + (u32)col0 * 2u);
        L.stride = 32u * 8192u;
        const u32 q = lane >> 4;
#pragma unroll
        for (int k = 0; k < 4; ++k) L.v[k] = q * 8192u + (((lane & 15u) ^ ((q << 2) | (u32)k)) * 16u) + (u32)k * (4u * 8192u - 1024u);
      }
    } else {
      n = nrem; ts = tskip;
      STAMP(myws, 1);
      if (wave == 0) {
        const int sA = (int)(lane >> 4), sB = 4 + (int)((lane >> 4) & 1u);
        const int pw = slot_w(k);
        const int pA = max(pw, slot_x(j, e, sA + 2)), pB = max(pw, slot_x(j, e, sB + 2));
        const char* fA = myws + FIN_OFF + (u32)(sA * 32 + pA) * 64u + (lane & 15u) * 4u;
        const char* fB = myws + FIN_OFF + (u32)(sB * 32 + pB) * 64u + (lane & 15u) * 4u;
        const bool skipA = sA + 2 == j, skipB = sB + 2 == j;
        sleep_until(tstart, tin);
        const u64 t0 = now();
        u32 st = 0;
        for (;;) {
          const u32 a0 = ld_sys(fA), a1 = ld_sys(fB);
          const bool ok = (skipA || a0 >= call) && (skipB || a1 >= call);
          if (__builtin_amdgcn_ballot_w64(ok) == ~0ull) break;
          if (give_up(t0, myws, call)) {
            if (lane == 0) st_agt(myws + TMO_OFF, call);
            st = 1;
            break;
          }
          __builtin_amdgcn_s_sleep(4);
        }
        if (lane == 0) *(LDSP u32*)(uintptr_t)(sbase + 4u * STAGE_B) = st;
        __builtin_amdgcn_s_waitcnt(0);
      }
      asm volatile("s_barrier" ::: "memory");
      const bool wgdead = lds_rd(sbase + 4u * STAGE_B) != 0;
      STAMP(myws, 2);
      if (wgdead) { dead = true; break; }
      if (wave < 2) L.base = myws + (size_t)(XS_OFF + par * XS_PAR + (u32)rb * 6u * PIECE + (u32)wave * 4096u);
      else          L.base = myws + (size_t)(WS_OFF + par * XS_PAR + (u32)k * 6u * PIECE + (u32)(wave - 2) * 4096u);
      L.stride = 8192u;
#pragma unroll
      for (int k = 0; k < 4; ++k) L.v[k] = lane * 16u;
    }
    gemm_phase(acc, A, L, n, ts);
    if (ph == nph - 1) STAMP(myws, 3);
    for (int s = 0; s < 2; ++s) {
      const int pi = s == 0 ? pi0 : pi1;
      if (pi < 0 || ph != addph) continue;
      char* const pb = uni(pin + (size_t)((u32)pi * PIECE));
      const bool first = pi == 0 || (pi >= NOUT1 && (pi - NOUT1) % NOUT == 0);
      const u32 wo = PIN_OFF + par * PIN_PAR + tile_off + (u32)(pi - (((lane & 2u) && !first) ? 1 : 0)) * PIECE + (lane & 1u) * 4096u + 4092u;
      sleep_until(tstart, pi >= NOUT1 ? (u32)(T0 + EARLY + TAU * (NIN - 2 + (pi - NOUT1) % NOUT)) : (fin ? 0u : tin - 300u));
      const u64 t0 = now();
      while (!dead) {
        const u32 v = ld_sys(myws + wo);
        if (__builtin_amdgcn_ballot_w64(v != 0u)) break;
        if (give_up(t0, myws, call)) { dead = true; break; }
        __builtin_amdgcn_s_sleep(3);
      }
      if (s == 1 || pi1 < 0) STAMP(myws, 4);
      u32x4 pv[8];
      while (!dead) {
        asm volatile("s_nop 4\n\t"
                     "global_load_dwordx4 %0, %8, %10 sc0 sc1\n\t"
                     "global_load_dwordx4 %1, %8, %10 offset:1024 sc0 sc1\n\t"
                     "global_load_dwordx4 %2, %8, %10 offset:2048 sc0 sc1\n\t"
                     "global_load_dwordx4 %3, %8, %10 offset:3072 sc0 sc1\n\t"
                     "global_load_dwordx4 %4, %9, %10 sc0 sc1\n\t"
                     "global_load_dwordx4 %5, %9, %10 offset:1024 sc0 sc1\n\t"
                     "global_load_dwordx4 %6, %9, %10 offset:2048 sc0 sc1\n\t"
                     "global_load_dwordx4 %7, %9, %10 offset:3072 sc0 sc1\n\t"
                     "s_waitcnt vmcnt(0)"
                     : "=&v"(pv[0]), "=&v"(pv[1]), "=&v"(pv[2]), "=&v"(pv[3]), "=&v"(pv[4]), "=&v"(pv[5]), "=&v"(pv[6]), "=&v"(pv[7])
                     : "v"(l16), "v"(l16b), "s"(pb) : "memory");
        u32 mn = 0xffffffffu;
  #pragma unroll
        for (int i = 0; i < 8; ++i)
  #pragma unroll
          for (int z = 0; z < 4; ++z) mn = min(mn, pv[i][z]);
        if (__builtin_amdgcn_ballot_w64(mn == 0u) == 0ull) break;
        if (give_up(t0, myws, call)) dead = true;
      }
      if (dead) {
        if (lane == 0) st_agt(myws + TMO_OFF, call);
      } else {
  #pragma unroll
        for (int nt = 0; nt < 2; ++nt)
  #pragma unroll
          for (int mt = 0; mt < 2; ++mt)
  #pragma unroll
            for (int sx = 0; sx < 2; ++sx)
  #pragma unroll
              for (int z = 0; z < 4; ++z) {
                const u32 u = pv[(nt * 2 + mt) * 2 + sx][z];
                acc[nt][mt][8 * sx + 2 * z] += bf_lo(u);
                acc[nt][mt][8 * sx + 2 * z + 1] += bf_hi(u);
              }
      }
      if (pi != plast) zero8k(l16, l16b, pb);
    }
  }

  STAMP(myws, 6);
  if (!fin) {
    char* dst = uni(myws + (size_t)(POUT_OFF + (u32)((j - 1) * 16 + k) * PIECE + tile_off));
    char* fl = myws + (size_t)(FLOC_OFF + (u32)((j - 1) * 8 + e * 4 + wave) * 128u + (u32)k * 4u);
    asm volatile("s_nop 4" ::: "memory");
#pragma unroll
    for (int nt = 0; nt < 2; ++nt)
#pragma unroll
      for (int mt = 0; mt < 2; ++mt) {
        u32x4 v0, v1;
#pragma unroll
        for (int z = 0; z < 4; ++z) {
          const u32 t0_ = pack2(acc[nt][mt][2 * z], acc[nt][mt][2 * z + 1]);
          const u32 t1_ = pack2(acc[nt][mt][8 + 2 * z], acc[nt][mt][8 + 2 * z + 1]);
          v0[z] = dead ? 0x7fc07fc0u : (t0_ ? t0_ : 0x8000u);
          v1[z] = dead ? 0x7fc07fc0u : (t1_ ? t1_ : 0x8000u);
        }
        const u32 o = nt ? l16b : l16;
        if (mt == 0) { st16<0>(o, v0, dst); st16<1024>(o, v1, dst); }
        else         { st16<2048>(o, v0, dst); st16<3072>(o, v1, dst); }
      }
    waitv<0>();
    if (lane == 0) st_sys(fl, call);
  } else {
    const u32 h = lane >> 5, m = lane & 31u;
    const u32 tb = sbase + (u32)wave * 8192u;
#pragma unroll
    for (int nt = 0; nt < 2; ++nt)
#pragma unroll
      for (int mt = 0; mt < 2; ++mt)
#pragma unroll
        for (int sx = 0; sx < 2; ++sx) {
          u32x4 v;
#pragma unroll
          for (int z = 0; z < 4; ++z) {
            const float lo = acc[nt][mt][8 * sx + 2 * z], hi = acc[nt][mt][8 * sx + 2 * z + 1];
            v[z] = dead ? 0x7fc07fc0u : pack2(lo < 0.f ? 0.f : lo, hi < 0.f ? 0.f : hi);
          }
          *(volatile LDSP u32x4*)(uintptr_t)(tb + ((u32)mt * 32u + m) * 128u + ((((u32)(nt * 4 + sx * 2) + h) ^ (m & 7u)) * 16u)) = v;
        }
    char* ob = uni(P.out + (size_t)((u32)(row0 - r * 512 + wm * 64) * 8192u + (u32)(col0 + wn * 64) * 2u));
    const u32 rr = lane >> 3, cc = lane & 7u;
    const u32 ra = tb + rr * 128u + ((cc ^ rr) * 16u);
    u32x4 ov[8];
#pragma unroll
    for (int i = 0; i < 8; ++i) ov[i] = *(volatile LDSP u32x4*)(uintptr_t)(ra + (u32)i * 1024u);
#pragma unroll
    for (int i = 0; i < 8; ++i) OST<0>(((u32)i * 8u + rr) * 8192u + cc * 16u, ov[i], ob);
    if (plast >= 0) zero8k(l16, l16b, uni(pin + (size_t)((u32)plast * PIECE)));
  }
  STAMP(myws, 5);
}

__global__ __launch_bounds__(256, 2) void fused_gemm_rs(Params P) {
  extern __shared__ __attribute__((aligned(16))) char smem[];
  const u32 lane = threadIdx.x & 63u;
  const int wave = __builtin_amdgcn_readfirstlane(threadIdx.x >> 6);
  const int bid = blockIdx.x;
  if (bid < N_PUSH_WG) mover(P, wave, lane, smem);
  else if (bid >= 256 && bid < 256 + N_PUSH_WG) {
    for (int i = 0; i < 6; ++i) __builtin_amdgcn_s_sleep(16);
  } else tile(P, bid < 256 ? bid - N_PUSH_WG : bid - 2 * N_PUSH_WG, wave, lane, smem);
}

}

extern "C" void kernel_launch(int rank, int nranks, ncclComm_t comm,
                              hipStream_t stream,
                              void* const* d_in, const int* in_sizes, int n_in,
                              void* d_out, int out_size,
                              const int* in_dtypes, int out_dtype,
                              void* const* d_out_all,
                              void* const* d_ws_all, size_t ws_bytes) {
  if (nranks != NR || n_in < 2 || ws_bytes < (size_t)WS_NEED || (unsigned)rank >= (unsigned)NR) {
    fprintf(stderr, "unsupported configuration\n");
    return;
  }
  Params P;
  P.x = (const char*)d_in[0];
  P.w = (const char*)d_in[1];
  P.out = (char*)d_out;
  for (int i = 0; i < NR; ++i) P.ws[i] = (char*)d_ws_all[i];
  P.rank = rank;
  HIP_CHECK(hipFuncSetAttribute((const void*)fused_gemm_rs, hipFuncAttributeMaxDynamicSharedMemorySize, (int)LDS_BYTES));
  hipLaunchKernelGGL(fused_gemm_rs, dim3(N_WG), dim3(256), LDS_BYTES, stream, P);
  HIP_CHECK(hipGetLastError());
}
```

```cpp
#include <hip/hip_runtime.h>
#include <rccl/rccl.h>
#include <cstdio>
#include <cstdint>

#define HIP_CHECK(expr) do { hipError_t _e = (expr); if (_e != hipSuccess) { \
    fprintf(stderr, "HIP error %s at %s:%d\n", hipGetErrorString(_e), __FILE__, __LINE__); \
    return; } } while (0)

namespace {

typedef unsigned u32;
typedef unsigned long long u64;
typedef u32 u32x4 __attribute__((ext_vector_type(4)));
typedef float f32x2 __attribute__((ext_vector_type(2)));
typedef float f32x16 __attribute__((ext_vector_type(16)));
typedef __bf16 bf16x2 __attribute__((ext_vector_type(2)));
typedef __bf16 bf16x8 __attribute__((ext_vector_type(8)));
typedef short s16x4 __attribute__((ext_vector_type(4)));
typedef short s16x8 __attribute__((ext_vector_type(8)));
#define LDSP __attribute__((address_space(3)))

constexpr int NR = 8;
constexpr u32 PIECE = 65536u;
constexpr u32 MIB = 1u << 20;
#define NS 2
#define OX 14
#define TAU 119
#define T0 100
#define PFG 0u
#define EARLY 100
#define TAU1 100
#define TAU2 0
#define ROT 0
#define STG 15
#define STGD 2
#define PSL 4
#define LSL 0
#define WSL 3
#define RSL 0
#define LAG 1
#define WIN 3
#define GLP "sc0 sc1"
#define MLP "sc0 sc1"
constexpr int NIN = 30;
constexpr int NOUT = 16 - NS;
constexpr int NOUT1 = 16 + 6 * NS;
constexpr u32 XS_OFF = 0;
constexpr u32 XS_PAR = 6u * MIB;
constexpr u32 WS_OFF = 12u * MIB;
constexpr u32 PIN_OFF = 24u * MIB;
constexpr u32 PIN_PAR = 7u * MIB;
constexpr u32 POUT_OFF = 38u * MIB;
constexpr u32 FLG_OFF = 45u * MIB;
constexpr u32 FIN_OFF = FLG_OFF;
constexpr u32 FLOC_OFF = FLG_OFF + 16384u;
constexpr u32 CNT_OFF = FLG_OFF + 32768u;
constexpr u32 TMO_OFF = FLG_OFF + 49152u;
constexpr u32 DUMMY_OFF = FLG_OFF + 65536u;
constexpr u32 DBG_OFF = FLG_OFF + 131072u;
constexpr u32 WS_NEED = FLG_OFF + 262144u;

constexpr int N_PUSH_WG = 56;
constexpr int N_MAIN_WG = 256;
constexpr int N_FIN_WG = 96;
constexpr int N_WG = 2 * N_PUSH_WG + N_MAIN_WG + N_FIN_WG;
constexpr u32 STAGE_B = 16384u;
constexpr u32 LDS_BYTES = 4u * STAGE_B + 1024u;
constexpr u64 DEADLINE = 100000000ull;
constexpr u32 DEAD = 0x7fffffffu;

struct Params {
  const char* x;
  const char* w;
  char* out;
  char* ws[NR];
  int rank;
};

#define SB() __builtin_amdgcn_sched_barrier(0)
template <int N> __device__ __forceinline__ void waitv() { asm volatile("s_waitcnt vmcnt(%0)" :: "i"(N) : "memory"); }

__device__ __forceinline__ u64 now() { return __builtin_amdgcn_s_memrealtime(); }
template <typename T> __device__ __forceinline__ T* uni(T* p) {
  const u64 v = (u64)(uintptr_t)p;
  const u32 lo = __builtin_amdgcn_readfirstlane((u32)v), hi = __builtin_amdgcn_readfirstlane((u32)(v >> 32));
  return (T*)(uintptr_t)(((u64)hi << 32) | lo);
}
__device__ __forceinline__ u32 ld_sys(const void* p) {
  return __hip_atomic_load((const u32*)p, __ATOMIC_RELAXED, __HIP_MEMORY_SCOPE_SYSTEM);
}
__device__ __forceinline__ void st_sys(void* p, u32 v) {
  __hip_atomic_store((u32*)p, v, __ATOMIC_RELAXED, __HIP_MEMORY_SCOPE_SYSTEM);
}
__device__ __forceinline__ void st_agt(void* p, u32 v) {
  __hip_atomic_store((u32*)p, v, __ATOMIC_RELAXED, __HIP_MEMORY_SCOPE_AGENT);
}
__device__ __forceinline__ bool give_up(u64 t0, const char* ws, u32 call) {
  const u64 dt = now() - t0;
  if (!__builtin_amdgcn_readfirstlane((int)(dt > 50000ull))) return false;
  __builtin_amdgcn_s_sleep(64);
  if (__builtin_amdgcn_readfirstlane((int)(dt > DEADLINE))) return true;
  return __builtin_amdgcn_readfirstlane(ld_sys(ws + TMO_OFF)) == call;
}
__device__ __forceinline__ void sleep_until(u64 tstart, u32 t) {
  for (;;) {
    const u32 el = (u32)__builtin_amdgcn_readfirstlane((u32)(now() - tstart));
    if (el >= t) break;
    if (t - el > 40u) __builtin_amdgcn_s_sleep(12); else __builtin_amdgcn_s_sleep(1);
  }
}
template <int IMM> __device__ __forceinline__ void st16(u32 off, u32x4 v, char* base) {
  asm volatile("global_store_dwordx4 %0, %1, %2 offset:%3 sc0 sc1\n\ts_nop 1" :: "v"(off), "v"(v), "s"(base), "i"(IMM) : "memory");
}
template <int IMM> __device__ __forceinline__ void st16p(u32 off, u32x4 v, char* base) {
  asm volatile("global_store_dwordx4 %0, %1, %2 offset:%3\n\ts_nop 1" :: "v"(off), "v"(v), "s"(base), "i"(IMM) : "memory");
}
#define OST st16
__device__ __forceinline__ void dma16x4(u32 lds, const u32 (&o)[4], const char* base) {
  asm volatile("s_mov_b32 m0, %0\n\ts_nop 3\n\t"
               "global_load_lds_dwordx4 %1, %5 " MLP "\n\t"
               "global_load_lds_dwordx4 %2, %5 offset:1024 " MLP "\n\t"
               "global_load_lds_dwordx4 %3, %5 offset:2048 " MLP "\n\t"
               "global_load_lds_dwordx4 %4, %5 offset:3072 " MLP
               :: "s"(lds), "v"(o[0]), "v"(o[1]), "v"(o[2]), "v"(o[3]), "s"(base) : "memory");
}
__device__ __forceinline__ void dma4(u32 lds, u32 off, const char* base) {
  asm volatile("s_mov_b32 m0, %0\n\ts_nop 3\n\t"
               "global_load_lds_dword %1, %2 sc0 sc1"
               :: "s"(lds), "v"(off), "s"(base) : "memory");
}
__device__ __forceinline__ void st4_lane0(u32 off, u32 val, char* base) {
  asm volatile("s_mov_b64 exec, 1\n\ts_nop 3\n\tglobal_store_dword %0, %1, %2 sc0 sc1\n\ts_mov_b64 exec, -1"
               :: "v"(off), "v"(val), "s"(base) : "memory");
}

#define STAMP(ws, idx) do {} while (0)

__device__ __forceinline__ void piece_of(int q, int d, bool& isx, int& idx) {
  const int t = q - NS, u = q - NS - 12;
  int j = (t >> 1) + 2;
  j += j >= d ? 1 : 0;
  j = j == 8 ? 1 : j;
  isx = (q >= NS && u < 0) || (u >= OX && u < OX + 2);
  idx = q < NS ? q : (u < 0 ? 2 * j + (t & 1) : (u < OX ? NS + u : (u < OX + 2 ? u - OX : NS + u - 2)));
}
__device__ __forceinline__ int slot_w(int k) { return k < NS ? k : (k - NS < OX ? 12 + k : 14 + k); }
__device__ __forceinline__ int slot_x(int j, int e, int d) {
  return j == 0 ? NS + 12 + OX + e : NS + 2 * (j == 1 ? 5 : j - 2 - (j > d ? 1 : 0)) + e;
}
__device__ __forceinline__ u32 lds_rd(u32 a) { return __builtin_amdgcn_readfirstlane(*(volatile LDSP u32*)(uintptr_t)a); }
__device__ __forceinline__ void lds_wr(u32 a, u32 v) { *(volatile LDSP u32*)(uintptr_t)a = v; }

__device__ __forceinline__ void mover(const Params& P, int wave, u32 lane, char* smem) {
  __builtin_amdgcn_s_setprio(3);
  const u64 tstart = now();
  const int s = P.rank;
  char* const myws = P.ws[s];
  const int pair = wave >> 1;
  const int d = 1 + ((int)(blockIdx.x >> 3) + 3 * pair) % 7;
  const int p = (blockIdx.x & 7) * 2 + pair;
  const u32 r = (u32)(s ^ d) & (u32)(NR - 1);
  const int nin = d == 1 ? 0 : NIN;
  const int np = d == 1 ? NOUT1 : NIN + NOUT;
  const u32 ring = (u32)(uintptr_t)smem + (u32)pair * 32768u;
  const u32 ctl = (u32)(uintptr_t)smem + 65536u + (u32)pair * 512u;
  const u32 cnt_off = CNT_OFF + (u32)(blockIdx.x * 4 + wave) * 4u;

  if ((wave & 1) == 0) {
    const char* const xin = P.x;
    const char* const win = P.w;
    const int rg = r >> 2, rb_ = (r >> 1) & 1, ra = r & 1;
    u32 xo[4], wo[4], oo[4];
    {
      const u32 xc = (lane & 3u) ^ ((lane >> 4) & 3u), q4 = lane >> 4;
#pragma unroll
      for (int k = 0; k < 4; ++k) {
        xo[k] = ((u32)(p & 1) * 64u + (u32)k * 16u + (lane >> 2)) * 1024u + (u32)(p >> 1) * 64u + xc * 16u - (u32)k * 1024u;
        wo[k] = ((u32)(p >> 1) * 32u + (u32)(p & 1) * 16u + (u32)k * 4u + q4) * 8192u
              + (((lane & 15u) ^ ((q4 << 2) | (u32)k)) * 16u) - (u32)k * 1024u;
        oo[k] = (u32)p * 4096u + lane * 16u;
      }
    }
    u32 poll_off;
    {
      const int i = (int)lane < np - nin ? (int)lane : 0;
      const int jj = d != 1 ? d : (i < 6 * NS ? 2 + i / NS : 1);
      const int kk = d != 1 ? NS + i : (i < 6 * NS ? i % NS : i - 6 * NS);
      poll_off = FLOC_OFF + (u32)((jj - 1) * 8 + (p >> 1)) * 128u + (u32)kk * 4u;
    }
#define ISSUE_LOADS(QQ) do { \
      const int q_ = (QQ); \
      const u32 dst_ = ring + (u32)(q_ & 7) * 4096u; \
      if (q_ < nin) { \
        bool isx_; int ix_; \
        piece_of(q_, d, isx_, ix_); \
        if (isx_) dma16x4(dst_, xo, xin + (size_t)(((r ^ (u32)(ix_ >> 1)) * 512u + (u32)rb_ * 256u + (u32)(ix_ & 1) * 128u) * 1024u + (u32)ra * 512u)); \
        else      dma16x4(dst_, wo, win + (size_t)((u32)ra * 256u * 8192u + (u32)rg * 4096u + (u32)(ra ? 15 - ix_ : ix_) * 256u)); \
      } else { \
        const int i_ = q_ - nin; \
        const int jj_ = d != 1 ? d : (i_ < 6 * NS ? 2 + i_ / NS : 1); \
        const int kk_ = d != 1 ? NS + i_ : (i_ < 6 * NS ? i_ % NS : i_ - 6 * NS); \
        dma16x4(dst_, oo, myws + (size_t)(POUT_OFF + (u32)((jj_ - 1) * 16 + kk_) * PIECE)); \
      } \
    } while (0)

    lds_wr(ctl, 0u);
    lds_wr(ctl + 256u + lane * 4u, 0u);
    asm volatile("s_waitcnt lgkmcnt(0)\n\ts_barrier" ::: "memory");
    dma4(ring + 7u * 4096u, cnt_off, myws);
    int q0 = 0;
    u32 pub = 0;
    if (nin) {
      ISSUE_LOADS(0); ISSUE_LOADS(1); ISSUE_LOADS(2);
      waitv<8>();
      lds_wr(ctl, 1u);
      q0 = 3; pub = 1;
    } else {
      waitv<0>();
    }
    const u32 call = lds_rd(ring + 7u * 4096u) + 1u;
    st4_lane0(cnt_off, call, myws);
    STAMP(myws, 0);

    if (!nin) sleep_until(tstart, (u32)(T0 + (NS + 2) * TAU + 500));
    u64 ready = 0;
    u32 cons = 0;
    bool dead = false;
    u32 dbg1 = 0, dbg2 = 0;
    for (int q = q0; q < np; ++q) {
      bool rdy = true;
      if (q >= nin) {
        if (!((ready >> (q - nin)) & 1ull)) {
          const u32 pv = *(volatile LDSP u32*)(uintptr_t)(ctl + 256u + lane * 4u);
          ready |= __builtin_amdgcn_ballot_w64(pv >= call);
        }
        rdy = (ready >> (q - nin)) & 1ull;
      }
      if (cons + 8u <= (u32)q) cons = lds_rd(ctl + 4u);
      if (!rdy || cons + 8u <= (u32)q) {
        waitv<0>();
        pub = (u32)q;
        lds_wr(ctl, pub);
        const u64 t0 = now();
        for (;;) {
          if (!rdy) {
            const u32 pv = ld_sys(myws + poll_off);
            ready |= __builtin_amdgcn_ballot_w64(pv >= call);
            rdy = (ready >> (q - nin)) & 1ull;
            if (LSL && !rdy) __builtin_amdgcn_s_sleep(LSL);
          } else {
            __builtin_amdgcn_s_sleep(2);
          }
          cons = lds_rd(ctl + 4u);
          if (rdy && cons + 8u > (u32)q) break;
          if (give_up(t0, myws, call)) { dead = true; break; }
        }
        if (dead) break;
      }
      if (q + 4 >= nin) dma4(ctl + 256u, poll_off, myws);
      ISSUE_LOADS(q);
      if (q + 3 >= nin) waitv<10>(); else waitv<8>();
      if (q - 1 > (int)pub) { pub = (u32)(q - 1); lds_wr(ctl, pub); }
      if (q == nin - 1) STAMP(myws, 1);
    }
    waitv<0>();
    if (dead) st4_lane0(TMO_OFF, call, myws);
    lds_wr(ctl, dead ? DEAD : (u32)np);
    STAMP(myws, 3);
    return;
  }

  char* const rws = uni(P.ws[r]);
  lds_wr(ctl + 4u, 0u);
  asm volatile("s_waitcnt lgkmcnt(0)\n\ts_barrier" ::: "memory");
  const u32 call = __builtin_amdgcn_readfirstlane(ld_sys(myws + cnt_off)) + 1u;
  st4_lane0(cnt_off, call, myws);
  STAMP(myws, 0);
  const u32 par = call & 1u;
  const int slot = d - 2;
  const u32 dl = (u32)p * 4096u + lane * 16u;
  const u32 pin0 = PIN_OFF + par * PIN_PAR + (u32)(d == 1 ? 0 : NOUT1 + slot * NOUT) * PIECE;
#define FLAG(QQ) do { \
    const int q_ = (QQ); \
    const bool real_ = q_ >= 0 && q_ < nin; \
    const u32 o_ = real_ ? FIN_OFF + (u32)(slot * 32 + q_) * 64u + (u32)p * 4u : DUMMY_OFF + (u32)(blockIdx.x * 4 + wave) * 128u; \
    st4_lane0(o_, call, uni(real_ ? rws : myws)); \
  } while (0)

  const u32 toff = (u32)(((p + STGD * d) & 15) * STG) >> 1;
  u32 full = 0, tnext = 0;
  u32 dbg1 = 0, dbg2 = 0;
  for (int q = 0; q < np; ++q) {
    {
      const u32 target = d == 1 ? tnext : max(tnext, (u32)q * (u32)TAU + (u32)T0 + toff);
      sleep_until(tstart, target);
    }
    if (full <= (u32)q) {
      full = lds_rd(ctl);
      if (full <= (u32)q) {
        const u64 t0 = now();
        bool drained = !(q > 0 && q < nin + LAG);
        for (;;) {
          __builtin_amdgcn_s_sleep(1);
          full = lds_rd(ctl);
          if (full > (u32)q) break;
          const u64 dt = now() - t0;
          if (!drained && __builtin_amdgcn_readfirstlane((int)(dt > 30ull))) {
            drained = true;
            waitv<0>();
#pragma unroll
            for (int z = LAG; z >= 1; --z) FLAG(q - z);
          }
          if (__builtin_amdgcn_readfirstlane((int)(dt > 2ull * DEADLINE - 1000ull))) { full = DEAD; break; }
        }
      }
      if (full == DEAD) break;
    }
    tnext = (u32)__builtin_amdgcn_readfirstlane((u32)(now() - tstart)) + (u32)(d == 1 ? TAU1 : TAU2);
    const u32 la = ring + (u32)(q & 7) * 4096u + lane * 16u;
    const u32x4 d0 = *(volatile LDSP u32x4*)(uintptr_t)la;
    const u32x4 d1 = *(volatile LDSP u32x4*)(uintptr_t)(la + 1024u);
    const u32x4 d2 = *(volatile LDSP u32x4*)(uintptr_t)(la + 2048u);
    const u32x4 d3 = *(volatile LDSP u32x4*)(uintptr_t)(la + 3072u);
    lds_wr(ctl + 4u, (u32)q + 1u);
    u32 o;
    if (q < nin) {
      bool isx; int ix;
      piece_of(q, d, isx, ix);
      o = (isx ? XS_OFF : WS_OFF) + par * XS_PAR + (u32)(ix * 6 + slot) * PIECE;
    } else {
      o = pin0 + (u32)(q - nin) * PIECE;
    }
    char* const dst = rws + (size_t)o;
    st16<0>(dl, d0, dst);
    st16<1024>(dl, d1, dst);
    st16<2048>(dl, d2, dst);
    st16<3072>(dl, d3, dst);
    if (nin && q < nin + LAG) {
      waitv<5 * LAG>();
      FLAG(q - LAG);
    } else {
      waitv<4 * WIN>();
    }
    if (q == NS + 11) STAMP(myws, 1);
    if (q == nin - 1) STAMP(myws, 2);
    if (q == np - 1) STAMP(myws, 4);
  }
  waitv<0>();
  STAMP(myws, 3);
}

__device__ __forceinline__ void agent(const Params& P, int wave, u32 lane, char* smem) {
  const u64 t0 = now();
  char* const myws = P.ws[P.rank];
  const int xcd = blockIdx.x & 7, w = (int)((blockIdx.x - 256) >> 3) * 4 + wave;
  if (w >= 24) return;
  const int sl = w % 6, qt = w / 6;
  const u32 cnt_off = CNT_OFF + (u32)(blockIdx.x * 4 + wave) * 4u;
  const u32 call = __builtin_amdgcn_readfirstlane(ld_sys(myws + cnt_off)) + 1u;
  if (lane == 0) st_agt(myws + cnt_off, call);
  const u32 par = call & 1u;
  const u32 lb = (u32)(uintptr_t)smem + (u32)wave * 4096u;
  const u32 lo = (u32)qt * 16384u + lane * PFG;
  for (int q = 0; q < NIN; ++q) {
    bool isx; int ix;
    piece_of(q, sl + 2, isx, ix);
    if (isx ? (ix & 3) != (xcd >> 1) : (ix & 1) != (xcd & 1)) continue;
    const char* fp = myws + FIN_OFF + (u32)(sl * 32 + q) * 64u + (u32)qt * 16u + (lane & 3u) * 4u;
    sleep_until(t0, (u32)(T0 + q * TAU + EARLY));
    for (;;) {
      const u32 v = ld_sys(fp);
      if ((__builtin_amdgcn_ballot_w64(v >= call) & 15ull) == 15ull) break;
      if (give_up(t0, myws, call)) return;
      __builtin_amdgcn_s_sleep(2);
    }
    const char* src = myws + (size_t)((isx ? XS_OFF : WS_OFF) + par * XS_PAR + (u32)(ix * 6 + sl) * PIECE);
#pragma unroll
    for (u32 i = 0; i < 16384u / (64u * (PFG ? PFG : 64u)); ++i) dma4(lb + i * 256u, lo + i * 64u * PFG, src);
  }
  waitv<0>();
}

struct Frag {
  bf16x8 x[2][2];
  bf16x8 w[2][2];
};

struct Loader {
  const char* base;
  u32 stride;
  u32 v[4];
  u32 m0base;
};

template <int STAGE> __device__ __forceinline__ void glds_pair(const Loader& L, int half) {
  const u32 m0v = L.m0base + STAGE * STAGE_B;
  if (half == 0)
    asm volatile("s_mov_b32 m0, %0\n\ts_nop 0\n\t"
                 "global_load_lds_dwordx4 %1, %3 " GLP "\n\t"
                 "global_load_lds_dwordx4 %2, %3 offset:1024 " GLP
                 :: "s"(m0v), "v"(L.v[0]), "v"(L.v[1]), "s"(L.base) : "memory");
  else
    asm volatile("s_mov_b32 m0, %0\n\ts_nop 0\n\t"
                 "global_load_lds_dwordx4 %1, %3 offset:2048 " GLP "\n\t"
                 "global_load_lds_dwordx4 %2, %3 offset:3072 " GLP
                 :: "s"(m0v), "v"(L.v[2]), "v"(L.v[3]), "s"(L.base) : "memory");
}

struct RdAddr { u32 xa[2]; u32 wa[2][2]; };

template <int STAGE> __device__ __forceinline__ bf16x8 rd_x(const RdAddr& A, int mt, int ks) {
  return *(const LDSP bf16x8*)(uintptr_t)(A.xa[ks] + (u32)(STAGE * STAGE_B + mt * 2048));
}
template <int STAGE> __device__ __forceinline__ bf16x8 rd_w(const RdAddr& A, int nt, int ks) {
  s16x4 lo = __builtin_amdgcn_ds_read_tr16_b64_v4i16((LDSP s16x4*)(uintptr_t)(A.wa[nt][0] + (u32)(STAGE * STAGE_B + 8192 + ks * 4096)));
  s16x4 hi = __builtin_amdgcn_ds_read_tr16_b64_v4i16((LDSP s16x4*)(uintptr_t)(A.wa[nt][1] + (u32)(STAGE * STAGE_B + 8192 + ks * 4096)));
  s16x8 v = __builtin_shufflevector(lo, hi, 0, 1, 2, 3, 4, 5, 6, 7);
  return __builtin_bit_cast(bf16x8, v);
}
#define MFMA(nt, mt, ks) acc[nt][mt] = __builtin_amdgcn_mfma_f32_32x32x16_bf16(cur.w[nt][ks], cur.x[mt][ks], acc[nt][mt], 0, 0, 0)

template <int S, int VM, bool LOAD, bool READ>
__device__ __forceinline__ void kstep(f32x16 (&acc)[2][2], const Frag& cur, Frag& nxt, const RdAddr& A, Loader& L) {
  constexpr int SN = (S + 1) & 3;
  if (READ) {
    __builtin_amdgcn_s_waitcnt((VM & 15) | (7 << 4) | (0 << 8) | ((VM >> 4) << 14));
    asm volatile("s_barrier" ::: "memory");
  } else {
    __builtin_amdgcn_s_waitcnt(15 | (7 << 4) | (0 << 8) | (3 << 14));
  }
  SB();
  MFMA(0, 0, 0); SB();
  if (READ) { nxt.w[0][0] = rd_w<SN>(A, 0, 0); nxt.x[0][0] = rd_x<SN>(A, 0, 0); } SB();
  MFMA(0, 1, 0); SB();
  if (READ) { nxt.x[1][0] = rd_x<SN>(A, 1, 0); nxt.w[1][0] = rd_w<SN>(A, 1, 0); } SB();
  MFMA(1, 0, 0); SB();
  if (LOAD) glds_pair<S>(L, 0);
  SB();
  MFMA(1, 1, 0); SB();
  if (READ) { nxt.w[0][1] = rd_w<SN>(A, 0, 1); nxt.x[0][1] = rd_x<SN>(A, 0, 1); } SB();
  MFMA(0, 0, 1); SB();
  if (READ) { nxt.x[1][1] = rd_x<SN>(A, 1, 1); nxt.w[1][1] = rd_w<SN>(A, 1, 1); } SB();
  MFMA(0, 1, 1); SB();
  if (LOAD) { glds_pair<S>(L, 1); L.base += L.stride; }
  SB();
  MFMA(1, 0, 1); SB();
  MFMA(1, 1, 1); SB();
}

template <int S> __device__ __forceinline__ void stage_load(Loader& L) {
  glds_pair<S>(L, 0); glds_pair<S>(L, 1); L.base += L.stride;
}

__device__ __forceinline__ void gemm_phase(f32x16 (&acc)[2][2], const RdAddr& A, Loader& L, int n, int u0, int sk, u32& dbgv, int dbg0) {
  Frag f0, f1;
  int u = (u0 >> 1) == sk ? 2 * sk + 2 : u0;
  u = u == 12 ? 0 : u;
  L.base = uni(L.base + (u32)u * 32768u);
  L.stride = __builtin_amdgcn_readfirstlane(L.stride);
  asm volatile("s_nop 4" ::: "memory");
  stage_load<0>(L); stage_load<1>(L); stage_load<2>(L); stage_load<3>(L);
  waitv<12>();
  asm volatile("s_barrier" ::: "memory");
#pragma unroll
  for (int ks = 0; ks < 2; ++ks)
#pragma unroll
    for (int i = 0; i < 2; ++i) { f0.w[i][ks] = rd_w<0>(A, i, ks); f0.x[i][ks] = rd_x<0>(A, i, ks); }
  SB();
  for (int t = 0; t < n - 4; t += 4) {
    {
      int v = u + 1;
      v = v == 2 * sk ? v + 2 : v;
      v = v == 12 ? (sk == 0 ? 2 : 0) : v;
      L.base += (long)(v - u - 1) * 32768;
      u = v;
    }
    kstep<0, 8, true, true>(acc, f0, f1, A, L);
    kstep<1, 8, true, true>(acc, f1, f0, A, L);
    kstep<2, 8, true, true>(acc, f0, f1, A, L);
    kstep<3, 8, true, true>(acc, f1, f0, A, L);
  }
  kstep<0, 8, false, true>(acc, f0, f1, A, L);
  kstep<1, 4, false, true>(acc, f1, f0, A, L);
  kstep<2, 0, false, true>(acc, f0, f1, A, L);
  kstep<3, 0, false, false>(acc, f1, f0, A, L);
}

__device__ __forceinline__ u32 pack2(float a, float b) {
  f32x2 t = {a, b};
  return __builtin_bit_cast(u32, __builtin_convertvector(t, bf16x2));
}
__device__ __forceinline__ float bf_lo(u32 v) { return __builtin_bit_cast(float, v << 16); }
__device__ __forceinline__ float bf_hi(u32 v) { return __builtin_bit_cast(float, v & 0xffff0000u); }

__device__ __forceinline__ void zero8k(u32 l16, u32 l16b, char* pb) {
  const u32x4 z4 = {0u, 0u, 0u, 0u};
  asm volatile("s_nop 4" ::: "memory");
  st16<0>(l16, z4, pb); st16<1024>(l16, z4, pb); st16<2048>(l16, z4, pb); st16<3072>(l16, z4, pb);
  st16<0>(l16b, z4, pb); st16<1024>(l16b, z4, pb); st16<2048>(l16b, z4, pb); st16<3072>(l16b, z4, pb);
}

__device__ __forceinline__ void tile(const Params& P, int tid, int wave, u32 lane, char* smem) {
  const int r = P.rank;
  char* const myws = P.ws[r];
  const int g = r >> 2, b = (r >> 1) & 1, a = r & 1;
  const int bid = blockIdx.x;
  const u64 tstart = now();

  const u32 cnt_off = CNT_OFF + (u32)(bid * 4 + wave) * 4u;
  const u32 call = __builtin_amdgcn_readfirstlane(ld_sys(myws + cnt_off)) + 1u;
  if (lane == 0) st_agt(myws + cnt_off, call);
  const u32 par = call & 1u;
  STAMP(myws, 0);

  int urot = 0;
  int row0, col0, nrem, tskip, j, e, k, rb;
  int pi0 = -1, pi1 = -1;
  bool fin;
  if (tid < N_MAIN_WG) {
    const int xcd = tid & 7, o = tid >> 3, u = xcd >> 1;
    int rbi, cbi;
    if (u < 2) { rbi = o < 24 ? 1 + o % 3 : 0; cbi = o < 24 ? o / 3 : o - 24; }
    else { rbi = o & 3; cbi = o >> 2; }
    rb = u + 4 * rbi; k = cbi * 2 + (xcd & 1);
    j = rb >> 1; e = rb & 1;
    row0 = (r ^ j) * 512 + b * 256 + e * 128; col0 = g * 2048 + (a ? 15 - k : k) * 128;
    nrem = j >= 2 ? 40 : 48; tskip = j >= 2 ? j - 2 : 99;
    urot = ROT == 0 ? 0 : (j == 0 ? (cbi * 3) >> 1 : (u < 2 ? (rbi - 1) * 4 : rbi * 3));
    fin = j == 0;
    if (j == 0) pi0 = 6 * NS + 15 - k;
    else if (j >= 2 && k >= 16 - NS) pi0 = ((j ^ 1) - 2) * NS + 15 - k;
  } else {
    const int f = tid - N_MAIN_WG;
    const int qd = 1 + f / 32, rem = f & 31;
    e = (rem >> 3) & 1; k = (rem & 7) * 2 + (rem >> 4);
    const int h = b ^ (qd & 1), g2 = g ^ (qd >> 1);
    row0 = r * 512 + h * 256 + e * 128; col0 = g2 * 2048 + (a ? 15 - k : k) * 128;
    nrem = 0; tskip = 99; j = 0; rb = 0;
    fin = true;
    const int iA = k >= NS ? NOUT1 + (2 * qd - 2) * NOUT + k - NS : -1;
    const int iB = 15 - k >= NS ? NOUT1 + (2 * qd - 1) * NOUT + 15 - k - NS : -1;
    if (k < 8) { pi0 = iA >= 0 ? iA : iB; pi1 = iA >= 0 ? iB : -1; }
    else       { pi0 = iB >= 0 ? iB : iA; pi1 = iB >= 0 ? iA : -1; }
  }

  const u32 sbase = (u32)(uintptr_t)smem;
  const int wm = wave >> 1, wn = wave & 1;
  RdAddr A;
  {
    const u32 h = lane >> 5, m = lane & 31u, f = (lane >> 2) & 3u;
    A.xa[0] = sbase + ((u32)wm * 64u + m) * 64u + 16u * (h ^ f);
    A.xa[1] = sbase + ((u32)wm * 64u + m) * 64u + 16u * ((2u + h) ^ f);
    const u32 blk = (lane >> 4) & 1u, q = (lane & 15u) >> 2, pp = lane & 3u;
#pragma unroll
    for (int nt = 0; nt < 2; ++nt)
#pragma unroll
      for (int t = 0; t < 2; ++t) {
        const u32 row = 8u * h + 4u * t + q;
        const u32 ch = (u32)wn * 8u + (u32)nt * 4u + 2u * blk + (pp & 1u);
        A.wa[nt][t] = sbase + 256u * row + 16u * (ch ^ (((row & 3u) << 2) | ((row >> 2) & 3u))) + 8u * (pp >> 1);
      }
  }

  f32x16 acc[2][2];
#pragma unroll
  for (int i = 0; i < 2; ++i)
#pragma unroll
    for (int k = 0; k < 2; ++k)
#pragma unroll
      for (int z = 0; z < 16; ++z) acc[i][k][z] = 0.f;

  const u32 tile_off = (u32)e * 32768u + (u32)wave * 8192u;
  const u32 l16 = lane * 16u, l16b = lane * 16u + 4096u;
  char* const pin = myws + (size_t)(PIN_OFF + par * PIN_PAR + tile_off);
  const int nph = nrem ? 2 : 1;
  const u32 tin = (u32)(T0 + EARLY + TAU * max(slot_w(k), slot_x(j, e, j == 7 ? 2 : 7)));
  const int addph = fin ? nph - 1 : 0;
  const int plast = fin ? (pi1 >= 0 ? pi1 : pi0) : -1;

  Loader L;
  L.m0base = sbase + (u32)wave * 4096u;
  bool dead = false;
  u32 dbgv = 0;
  for (int ph = 0; ph < nph; ++ph) {
    int n, ts, u0;
    if (ph == 0) {
      n = 16; ts = 99; u0 = 0;
      if (wave < 2) {
        L.base = P.x + (size_t)((u32)(row0 + wave * 64) * 1024u);
        L.stride = 64u;
        const u32 c = (lane & 3u) ^ ((lane >> 4) & 3u);
#pragma unroll
        for (int k = 0; k < 4; ++k) L.v[k] = (lane >> 2) * 1024u + c * 16u + (u32)k * (16384u - 1024u);
      } else {
        L.base = P.w + (size_t)((u32)(wave - 2) * 16u * 8192u + (u32)col0 * 2u);
        L.stride = 32u * 8192u;
        const u32 q = lane >> 4;
#pragma unroll
        for (int k = 0; k < 4; ++k) L.v[k] = q * 8192u + (((lane & 15u) ^ ((q << 2) | (u32)k)) * 16u) + (u32)k * (4u * 8192u - 1024u);
      }
    } else {
      n = nrem; ts = tskip; u0 = urot;
      STAMP(myws, 1);
      if (wave == 0) {
        const int sA = (int)(lane >> 4), sB = 4 + (int)((lane >> 4) & 1u);
        const int pw = slot_w(k);
        const int pA = max(pw, slot_x(j, e, sA + 2)), pB = max(pw, slot_x(j, e, sB + 2));
        const char* fA = myws + FIN_OFF + (u32)(sA * 32 + pA) * 64u + (lane & 15u) * 4u;
        const char* fB = myws + FIN_OFF + (u32)(sB * 32 + pB) * 64u + (lane & 15u) * 4u;
        const bool skipA = sA + 2 == j, skipB = sB + 2 == j;
        sleep_until(tstart, tin);
        const u64 t0 = now();
        u32 st = 0;
        for (;;) {
          const u32 a0 = ld_sys(fA), a1 = ld_sys(fB);
          const bool ok = (skipA || a0 >= call) && (skipB || a1 >= call);
          if (__builtin_amdgcn_ballot_w64(ok) == ~0ull) break;
          if (give_up(t0, myws, call)) {
            if (lane == 0) st_agt(myws + TMO_OFF, call);
            st = 1;
            break;
          }
          __builtin_amdgcn_s_sleep(PSL);
        }
        if (lane == 0) *(LDSP u32*)(uintptr_t)(sbase + 4u * STAGE_B) = st;
        __builtin_amdgcn_s_waitcnt(0);
      }
      asm volatile("s_barrier" ::: "memory");
      const bool wgdead = lds_rd(sbase + 4u * STAGE_B) != 0;
      STAMP(myws, 2);
      if (wgdead) { dead = true; break; }
      if (wave < 2) L.base = myws + (size_t)(XS_OFF + par * XS_PAR + (u32)rb * 6u * PIECE + (u32)wave * 4096u);
      else          L.base = myws + (size_t)(WS_OFF + par * XS_PAR + (u32)k * 6u * PIECE + (u32)(wave - 2) * 4096u);
      L.stride = 8192u;
#pragma unroll
      for (int k = 0; k < 4; ++k) L.v[k] = lane * 16u;
    }
    gemm_phase(acc, A, L, n, u0, ts, dbgv, ph * 8);
    if (ph == nph - 1) STAMP(myws, 3);
    for (int s = 0; s < 2; ++s) {
      const int pi = s == 0 ? pi0 : pi1;
      if (pi < 0 || ph != addph) continue;
      char* const pb = uni(pin + (size_t)((u32)pi * PIECE));
      const bool first = pi == 0 || (pi >= NOUT1 && (pi - NOUT1) % NOUT == 0);
      const u32 wo = PIN_OFF + par * PIN_PAR + tile_off + (u32)(pi - (((lane & 2u) && !first) ? 1 : 0)) * PIECE + (lane & 1u) * 4096u + 4092u;
      sleep_until(tstart, pi >= NOUT1 ? (u32)(T0 + EARLY + TAU * (NIN - 2 + (pi - NOUT1) % NOUT)) : (fin ? 0u : tin - 300u));
      const u64 t0 = now();
      while (!dead) {
        const u32 v = ld_sys(myws + wo);
        if (__builtin_amdgcn_ballot_w64(v != 0u)) break;
        if (give_up(t0, myws, call)) { dead = true; break; }
        __builtin_amdgcn_s_sleep(WSL);
      }
      if (s == 1 || pi1 < 0) STAMP(myws, 4);
      u32x4 pv[8];
      while (!dead) {
        asm volatile("s_nop 4\n\t"
                     "global_load_dwordx4 %0, %8, %10 sc0 sc1\n\t"
                     "global_load_dwordx4 %1, %8, %10 offset:1024 sc0 sc1\n\t"
                     "global_load_dwordx4 %2, %8, %10 offset:2048 sc0 sc1\n\t"
                     "global_load_dwordx4 %3, %8, %10 offset:3072 sc0 sc1\n\t"
                     "global_load_dwordx4 %4, %9, %10 sc0 sc1\n\t"
                     "global_load_dwordx4 %5, %9, %10 offset:1024 sc0 sc1\n\t"
                     "global_load_dwordx4 %6, %9, %10 offset:2048 sc0 sc1\n\t"
                     "global_load_dwordx4 %7, %9, %10 offset:3072 sc0 sc1\n\t"
                     "s_waitcnt vmcnt(0)"
                     : "=&v"(pv[0]), "=&v"(pv[1]), "=&v"(pv[2]), "=&v"(pv[3]), "=&v"(pv[4]), "=&v"(pv[5]), "=&v"(pv[6]), "=&v"(pv[7])
                     : "v"(l16), "v"(l16b), "s"(pb) : "memory");
        u32 mn = 0xffffffffu;
  #pragma unroll
        for (int i = 0; i < 8; ++i)
  #pragma unroll
          for (int z = 0; z < 4; ++z) mn = min(mn, pv[i][z]);
        if (__builtin_amdgcn_ballot_w64(mn == 0u) == 0ull) break;
        if (give_up(t0, myws, call)) dead = true;
        if (RSL) __builtin_amdgcn_s_sleep(RSL);
      }
      if (dead) {
        if (lane == 0) st_agt(myws + TMO_OFF, call);
      } else {
  #pragma unroll
        for (int nt = 0; nt < 2; ++nt)
  #pragma unroll
          for (int mt = 0; mt < 2; ++mt)
  #pragma unroll
            for (int sx = 0; sx < 2; ++sx)
  #pragma unroll
              for (int z = 0; z < 4; ++z) {
                const u32 u = pv[(nt * 2 + mt) * 2 + sx][z];
                acc[nt][mt][8 * sx + 2 * z] += bf_lo(u);
                acc[nt][mt][8 * sx + 2 * z + 1] += bf_hi(u);
              }
      }
      if (pi != plast) zero8k(l16, l16b, pb);
    }
  }

  STAMP(myws, 6);
  if (!fin) {
    char* dst = uni(myws + (size_t)(POUT_OFF + (u32)((j - 1) * 16 + k) * PIECE + tile_off));
    char* fl = myws + (size_t)(FLOC_OFF + (u32)((j - 1) * 8 + e * 4 + wave) * 128u + (u32)k * 4u);
    asm volatile("s_nop 4" ::: "memory");
#pragma unroll
    for (int nt = 0; nt < 2; ++nt)
#pragma unroll
      for (int mt = 0; mt < 2; ++mt) {
        u32x4 v0, v1;
#pragma unroll
        for (int z = 0; z < 4; ++z) {
          const u32 t0_ = pack2(acc[nt][mt][2 * z], acc[nt][mt][2 * z + 1]);
          const u32 t1_ = pack2(acc[nt][mt][8 + 2 * z], acc[nt][mt][8 + 2 * z + 1]);
          v0[z] = dead ? 0x7fc07fc0u : (t0_ ? t0_ : 0x8000u);
          v1[z] = dead ? 0x7fc07fc0u : (t1_ ? t1_ : 0x8000u);
        }
        const u32 o = nt ? l16b : l16;
        if (mt == 0) { st16<0>(o, v0, dst); st16<1024>(o, v1, dst); }
        else         { st16<2048>(o, v0, dst); st16<3072>(o, v1, dst); }
      }
    waitv<0>();
    if (lane == 0) st_sys(fl, call);
  } else {
    const u32 h = lane >> 5, m = lane & 31u;
    const u32 tb = sbase + (u32)wave * 8192u;
#pragma unroll
    for (int nt = 0; nt < 2; ++nt)
#pragma unroll
      for (int mt = 0; mt < 2; ++mt)
#pragma unroll
        for (int sx = 0; sx < 2; ++sx) {
          u32x4 v;
#pragma unroll
          for (int z = 0; z < 4; ++z) {
            const float lo = acc[nt][mt][8 * sx + 2 * z], hi = acc[nt][mt][8 * sx + 2 * z + 1];
            v[z] = dead ? 0x7fc07fc0u : pack2(lo < 0.f ? 0.f : lo, hi < 0.f ? 0.f : hi);
          }
          *(volatile LDSP u32x4*)(uintptr_t)(tb + ((u32)mt * 32u + m) * 128u + ((((u32)(nt * 4 + sx * 2) + h) ^ (m & 7u)) * 16u)) = v;
        }
    char* ob = uni(P.out + (size_t)((u32)(row0 - r * 512 + wm * 64) * 8192u + (u32)(col0 + wn * 64) * 2u));
    const u32 rr = lane >> 3, cc = lane & 7u;
    const u32 ra = tb + rr * 128u + ((cc ^ rr) * 16u);
    u32x4 ov[8];
#pragma unroll
    for (int i = 0; i < 8; ++i) ov[i] = *(volatile LDSP u32x4*)(uintptr_t)(ra + (u32)i * 1024u);
#pragma unroll
    for (int i = 0; i < 8; ++i) OST<0>(((u32)i * 8u + rr) * 8192u + cc * 16u, ov[i], ob);
    if (plast >= 0) zero8k(l16, l16b, uni(pin + (size_t)((u32)plast * PIECE)));
  }
  STAMP(myws, 5);
}

__global__ __launch_bounds__(256, 2) void fused_gemm_rs(Params P) {
  extern __shared__ __attribute__((aligned(16))) char smem[];
  const u32 lane = threadIdx.x & 63u;
  const int wave = __builtin_amdgcn_readfirstlane(threadIdx.x >> 6);
  const int bid = blockIdx.x;
  if (bid < N_PUSH_WG) mover(P, wave, lane, smem);
  else if (bid >= 256 && bid < 256 + N_PUSH_WG) {
    for (int i = 0; i < 6; ++i) __builtin_amdgcn_s_sleep(16);
  } else tile(P, bid < 256 ? bid - N_PUSH_WG : bid - 2 * N_PUSH_WG, wave, lane, smem);
}

}

extern "C" void kernel_launch(int rank, int nranks, ncclComm_t comm,
                              hipStream_t stream,
                              void* const* d_in, const int* in_sizes, int n_in,
                              void* d_out, int out_size,
                              const int* in_dtypes, int out_dtype,
                              void* const* d_out_all,
                              void* const* d_ws_all, size_t ws_bytes) {
  if (nranks != NR || n_in < 2 || ws_bytes < (size_t)WS_NEED || (unsigned)rank >= (unsigned)NR) {
    fprintf(stderr, "unsupported configuration\n");
    return;
  }
  Params P;
  P.x = (const char*)d_in[0];
  P.w = (const char*)d_in[1];
  P.out = (char*)d_out;
  for (int i = 0; i < NR; ++i) P.ws[i] = (char*)d_ws_all[i];
  P.rank = rank;
  HIP_CHECK(hipFuncSetAttribute((const void*)fused_gemm_rs, hipFuncAttributeMaxDynamicSharedMemorySize, (int)LDS_BYTES));
  hipLaunchKernelGGL(fused_gemm_rs, dim3(N_WG), dim3(256), LDS_BYTES, stream, P);
  HIP_CHECK(hipGetLastError());
}
```

```cpp
#include <hip/hip_runtime.h>
#include <rccl/rccl.h>
#include <cstdio>
#include <cstdint>

#define HIP_CHECK(expr) do { hipError_t _e = (expr); if (_e != hipSuccess) { \
    fprintf(stderr, "HIP error %s at %s:%d\n", hipGetErrorString(_e), __FILE__, __LINE__); \
    return; } } while (0)

namespace {

typedef unsigned u32;
typedef unsigned long long u64;
typedef u32 u32x4 __attribute__((ext_vector_type(4)));
typedef float f32x2 __attribute__((ext_vector_type(2)));
typedef float f32x16 __attribute__((ext_vector_type(16)));
typedef __bf16 bf16x2 __attribute__((ext_vector_type(2)));
typedef __bf16 bf16x8 __attribute__((ext_vector_type(8)));
typedef short s16x4 __attribute__((ext_vector_type(4)));
typedef short s16x8 __attribute__((ext_vector_type(8)));
#define LDSP __attribute__((address_space(3)))

constexpr int NR = 8;
constexpr u32 PIECE = 65536u;
constexpr u32 MIB = 1u << 20;
#define NS 2
#define OX 14
#define TAU 116
#define T0 100
#define EARLY 100
#define TAU1 110
#define TAU2 0
#define STG 14
#define STGD 2
#define PSL 4
#define LSL 0
#define WSL 3
#define RSL 0
#define WOWN 0
#define LAG 1
#define WIN 3
#define GLP "sc0 sc1"
#define MLP "sc0 sc1"
constexpr int NIN = 30;
constexpr int NOUT = 16 - NS;
constexpr int NOUT1 = 16 + 6 * NS;
constexpr u32 XS_OFF = 0;
constexpr u32 XS_PAR = 6u * MIB;
constexpr u32 WS_OFF = 12u * MIB;
constexpr u32 PIN_OFF = 24u * MIB;
constexpr u32 PIN_PAR = 7u * MIB;
constexpr u32 POUT_OFF = 45u * MIB;
constexpr u32 FLG_OFF = 52u * MIB;
constexpr u32 FIN_OFF = FLG_OFF;
constexpr u32 FLOC_OFF = FLG_OFF + 16384u;
constexpr u32 TMO_OFF = FLG_OFF + 49152u;
constexpr u32 DUMMY_OFF = FLG_OFF + 65536u;
constexpr u32 DBG_OFF = FLG_OFF + 131072u;
constexpr u32 CNT_OFF = FLG_OFF + MIB + 262144u;
constexpr u32 WS_NEED = FLG_OFF + 2u * MIB;

constexpr int N_PUSH_WG = 56;
constexpr int N_MAIN_WG = 256;
constexpr int N_FIN_WG = 96;
constexpr int N_WG = 2 * N_PUSH_WG + N_MAIN_WG + N_FIN_WG;
constexpr u32 STAGE_B = 16384u;
constexpr u32 LDS_BYTES = 4u * STAGE_B + 1024u;
constexpr u64 DEADLINE = 100000000ull;
constexpr u32 DEAD = 0x7fffffffu;

struct Params {
  const char* x;
  const char* w;
  char* out;
  char* ws[NR];
  int rank;
};

#define SB() __builtin_amdgcn_sched_barrier(0)
template <int N> __device__ __forceinline__ void waitv() { asm volatile("s_waitcnt vmcnt(%0)" :: "i"(N) : "memory"); }

__device__ __forceinline__ u64 now() { return __builtin_amdgcn_s_memrealtime(); }
template <typename T> __device__ __forceinline__ T* uni(T* p) {
  const u64 v = (u64)(uintptr_t)p;
  const u32 lo = __builtin_amdgcn_readfirstlane((u32)v), hi = __builtin_amdgcn_readfirstlane((u32)(v >> 32));
  return (T*)(uintptr_t)(((u64)hi << 32) | lo);
}
__device__ __forceinline__ u32 ld_sys(const void* p) {
  return __hip_atomic_load((const u32*)p, __ATOMIC_RELAXED, __HIP_MEMORY_SCOPE_SYSTEM);
}
__device__ __forceinline__ void st_sys(void* p, u32 v) {
  __hip_atomic_store((u32*)p, v, __ATOMIC_RELAXED, __HIP_MEMORY_SCOPE_SYSTEM);
}
__device__ __forceinline__ void st_agt(void* p, u32 v) {
  __hip_atomic_store((u32*)p, v, __ATOMIC_RELAXED, __HIP_MEMORY_SCOPE_AGENT);
}
__device__ __forceinline__ bool give_up(u64 t0, const char* ws, u32 call) {
  const u64 dt = now() - t0;
  if (!__builtin_amdgcn_readfirstlane((int)(dt > 50000ull))) return false;
  __builtin_amdgcn_s_sleep(64);
  if (__builtin_amdgcn_readfirstlane((int)(dt > DEADLINE))) return true;
  return __builtin_amdgcn_readfirstlane(ld_sys(ws + TMO_OFF)) == call;
}
__device__ __forceinline__ void sleep_until(u64 tstart, u32 t) {
  for (;;) {
    const u32 el = (u32)__builtin_amdgcn_readfirstlane((u32)(now() - tstart));
    if (el >= t) break;
    if (t - el > 40u) __builtin_amdgcn_s_sleep(12); else __builtin_amdgcn_s_sleep(1);
  }
}
template <int IMM> __device__ __forceinline__ void st16(u32 off, u32x4 v, char* base) {
  asm volatile("global_store_dwordx4 %0, %1, %2 offset:%3 sc0 sc1\n\ts_nop 1" :: "v"(off), "v"(v), "s"(base), "i"(IMM) : "memory");
}
template <int IMM> __device__ __forceinline__ void st16p(u32 off, u32x4 v, char* base) {
  asm volatile("global_store_dwordx4 %0, %1, %2 offset:%3\n\ts_nop 1" :: "v"(off), "v"(v), "s"(base), "i"(IMM) : "memory");
}
#define OST st16
__device__ __forceinline__ void dma16x4(u32 lds, const u32 (&o)[4], const char* base) {
  asm volatile("s_mov_b32 m0, %0\n\ts_nop 3\n\t"
               "global_load_lds_dwordx4 %1, %5 " MLP "\n\t"
               "global_load_lds_dwordx4 %2, %5 offset:1024 " MLP "\n\t"
               "global_load_lds_dwordx4 %3, %5 offset:2048 " MLP "\n\t"
               "global_load_lds_dwordx4 %4, %5 offset:3072 " MLP
               :: "s"(lds), "v"(o[0]), "v"(o[1]), "v"(o[2]), "v"(o[3]), "s"(base) : "memory");
}
__device__ __forceinline__ u32 atom_rd(u32 off, const char* base) {
  u32 r;
  asm volatile("global_atomic_add %0, %1, %2, %3 sc0 sc1\n\ts_waitcnt vmcnt(0)" : "=&v"(r) : "v"(off), "v"(0u), "s"(base) : "memory");
  return r;
}
__device__ __forceinline__ u32 bump_issue(u32 off, const char* base) {
  u32 r;
  asm volatile("s_mov_b64 exec, 1\n\ts_nop 3\n\tglobal_atomic_add %0, %1, %2, %3 sc0 sc1\n\ts_mov_b64 exec, -1"
               : "=&v"(r) : "v"(off), "v"(1u), "s"(base) : "memory");
  return r;
}
template <int N> __device__ __forceinline__ u32 bump_get(u32 r) {
  u32 v;
  asm volatile("s_waitcnt vmcnt(%2)\n\tv_readfirstlane_b32 %0, %1\n\ts_nop 4" : "=s"(v) : "v"(r), "i"(N) : "memory");
  return v + 1u;
}
__device__ __forceinline__ void add1_lane0(u32 off, char* base) {
  asm volatile("s_mov_b64 exec, 1\n\ts_nop 3\n\tglobal_atomic_add %0, %1, %2 sc1\n\ts_mov_b64 exec, -1"
               :: "v"(off), "v"(1u), "s"(base) : "memory");
}
__device__ __forceinline__ void st4_lane0(u32 off, u32 val, char* base) {
  asm volatile("s_mov_b64 exec, 1\n\ts_nop 3\n\tglobal_store_dword %0, %1, %2 sc0 sc1\n\ts_mov_b64 exec, -1"
               :: "v"(off), "v"(val), "s"(base) : "memory");
}

#define STAMP(ws, idx) do {} while (0)

__device__ __forceinline__ void piece_of(int q, int d, bool& isx, int& idx) {
  const int t = q - NS, u = q - NS - 12;
  int j = (t >> 1) + 2;
  j += j >= d ? 1 : 0;
  j = j == 8 ? 1 : j;
  isx = (q >= NS && u < 0) || (u >= OX && u < OX + 2);
  idx = q < NS ? q : (u < 0 ? 2 * j + (t & 1) : (u < OX ? NS + u : (u < OX + 2 ? u - OX : NS + u - 2)));
}
__device__ __forceinline__ int slot_w(int k) { return k < NS ? k : (k - NS < OX ? 12 + k : 14 + k); }
__device__ __forceinline__ int slot_x(int j, int e, int d) {
  return j == 0 ? NS + 12 + OX + e : NS + 2 * (j == 1 ? 5 : j - 2 - (j > d ? 1 : 0)) + e;
}
__device__ __forceinline__ u32 lds_rd(u32 a) { return __builtin_amdgcn_readfirstlane(*(volatile LDSP u32*)(uintptr_t)a); }
__device__ __forceinline__ void lds_wr(u32 a, u32 v) { *(volatile LDSP u32*)(uintptr_t)a = v; }

__device__ __forceinline__ void mover(const Params& P, int wave, u32 lane, char* smem) {
  __builtin_amdgcn_s_setprio(3);
  const u64 tstart = now();
  const int s = P.rank;
  char* const myws = P.ws[s];
  const int pair = wave >> 1;
  const int d = 1 + ((int)(blockIdx.x >> 3) + 3 * pair) % 7;
  const int p = (blockIdx.x & 7) * 2 + pair;
  const u32 r = (u32)(s ^ d) & (u32)(NR - 1);
  const int nin = d == 1 ? 0 : NIN;
  const int np = d == 1 ? NOUT1 : NIN + NOUT;
  const u32 ring = (u32)(uintptr_t)smem + (u32)pair * 32768u;
  const u32 ctl = (u32)(uintptr_t)smem + 65536u + (u32)pair * 512u;
  const u32 cnt = bump_issue(CNT_OFF + (u32)(blockIdx.x * 4 + wave) * 64u, myws);

  if ((wave & 1) == 0) {
    const char* const xin = P.x;
    const char* const win = P.w;
    const int rg = r >> 2, rb_ = (r >> 1) & 1, ra = r & 1;
    u32 xo[4], wo[4], oo[4];
    {
      const u32 xc = (lane & 3u) ^ ((lane >> 4) & 3u), q4 = lane >> 4;
#pragma unroll
      for (int k = 0; k < 4; ++k) {
        xo[k] = ((u32)(p & 1) * 64u + (u32)k * 16u + (lane >> 2)) * 1024u + (u32)(p >> 1) * 64u + xc * 16u - (u32)k * 1024u;
        wo[k] = ((u32)(p >> 1) * 32u + (u32)(p & 1) * 16u + (u32)k * 4u + q4) * 8192u
              + (((lane & 15u) ^ ((q4 << 2) | (u32)k)) * 16u) - (u32)k * 1024u;
        oo[k] = (u32)p * 4096u + lane * 16u;
      }
    }
    u32 poll_off;
    {
      const int i = (int)lane < np - nin ? (int)lane : 0;
      const int jj = d != 1 ? d : (i < 6 * NS ? 2 + i / NS : 1);
      const int kk = d != 1 ? NS + i : (i < 6 * NS ? i % NS : i - 6 * NS);
      poll_off = FLOC_OFF + (u32)((jj - 1) * 8 + (p >> 1)) * 128u + (u32)kk * 4u;
    }
#define ISSUE_LOADS(QQ, II) do { \
      const int q_ = (QQ); \
      const u32 dst_ = ring + (u32)(q_ & 7) * 4096u; \
      if (q_ < nin) { \
        bool isx_; int ix_; \
        piece_of(q_, d, isx_, ix_); \
        if (isx_) dma16x4(dst_, xo, xin + (size_t)(((r ^ (u32)(ix_ >> 1)) * 512u + (u32)rb_ * 256u + (u32)(ix_ & 1) * 128u) * 1024u + (u32)ra * 512u)); \
        else      dma16x4(dst_, wo, win + (size_t)((u32)ra * 256u * 8192u + (u32)rg * 4096u + (u32)(ra ? 15 - ix_ : ix_) * 256u)); \
      } else { \
        const int i_ = (II); \
        const int jj_ = d != 1 ? d : (i_ < 6 * NS ? 2 + i_ / NS : 1); \
        const int kk_ = d != 1 ? NS + i_ : (i_ < 6 * NS ? i_ % NS : i_ - 6 * NS); \
        dma16x4(dst_, oo, myws + (size_t)(POUT_OFF + (u32)((jj_ - 1) * 16 + kk_) * PIECE)); \
      } \
    } while (0)

    int q0 = 0;
    u32 pub = 0, call;
    if (nin) {
      ISSUE_LOADS(0, 0); ISSUE_LOADS(1, 0); ISSUE_LOADS(2, 0);
    }
    lds_wr(ctl, 0u);
    asm volatile("s_waitcnt lgkmcnt(0)\n\ts_barrier" ::: "memory");
    if (nin) {
      call = bump_get<8>(cnt);
      lds_wr(ctl, 1u);
      q0 = 3; pub = 1;
    } else {
      call = bump_get<0>(cnt);
    }
    STAMP(myws, 0);

    if (!nin) sleep_until(tstart, (u32)(T0 + (NS + 2) * TAU + 500));
    u64 ready = 0;
    u64 todo = (1ull << (np - nin)) - 1ull;
    u32 cons = 0;
    bool dead = false;
    u32 dbg1 = 0, dbg2 = 0;
    for (int q = q0; q < np; ++q) {
      bool rdy = q < nin || (ready & todo) != 0ull;
      if (cons + 8u <= (u32)q) cons = lds_rd(ctl + 4u);
      if (!rdy || cons + 8u <= (u32)q) {
        waitv<0>();
        pub = (u32)q;
        lds_wr(ctl, pub);
        const u64 t0 = now();
        for (;;) {
          if (!rdy) {
            u32 pv = 0;
            if ((todo >> lane) & 1ull) pv = atom_rd(poll_off, myws);
            ready |= __builtin_amdgcn_ballot_w64(pv >= call);
            rdy = (ready & todo) != 0ull;
            if (LSL && !rdy) __builtin_amdgcn_s_sleep(LSL);
          } else {
            __builtin_amdgcn_s_sleep(2);
          }
          cons = lds_rd(ctl + 4u);
          if (rdy && cons + 8u > (u32)q) break;
          if (give_up(t0, myws, call)) { dead = true; break; }
        }
        if (dead) break;
      }
      int id = 0;
      if (q >= nin) {
        id = __builtin_ctzll(ready & todo);
        todo &= ~(1ull << id);
        lds_wr(ctl + 32u + (u32)(q & 7) * 4u, (u32)id);
      }
      ISSUE_LOADS(q, id);
      waitv<8>();
      if (q - 1 > (int)pub) { pub = (u32)(q - 1); lds_wr(ctl, pub); }
      if (q == nin - 1) STAMP(myws, 1);
    }
    waitv<0>();
    if (dead) st4_lane0(TMO_OFF, call, myws);
    lds_wr(ctl, dead ? DEAD : (u32)np);
    STAMP(myws, 3);
    return;
  }

  char* const rws = uni(P.ws[r]);
  lds_wr(ctl + 4u, 0u);
  asm volatile("s_waitcnt lgkmcnt(0)\n\ts_barrier" ::: "memory");
  const u32 call = bump_get<0>(cnt);
  STAMP(myws, 0);
  const u32 par = call & 1u;
  const int slot = d - 2;
  const u32 dl = (u32)p * 4096u + lane * 16u;
  const u32 pin0 = PIN_OFF + (call % 3u) * PIN_PAR + (u32)(d == 1 ? 0 : NOUT1 + slot * NOUT) * PIECE;
  int nflag = 0;
#define FLAG(QQ) do { \
    const int q_ = (QQ); \
    const bool real_ = q_ == nflag && q_ < nin; \
    const u32 o_ = real_ ? FIN_OFF + (u32)(slot * 32 + q_) * 64u : DUMMY_OFF + (u32)(blockIdx.x * 4 + wave) * 128u; \
    add1_lane0(o_, uni(real_ ? rws : myws)); \
    nflag += real_ ? 1 : 0; \
  } while (0)

  const u32 toff = (u32)(((p + STGD * d) & 15) * STG) >> 1;
  u32 full = 0, tnext = 0;
  u32 dbg1 = 0, dbg2 = 0;
  for (int q = 0; q < np; ++q) {
    if (d != 1) tnext = max(tnext, (u32)q * (u32)TAU + (u32)T0 + toff);
    sleep_until(tstart, tnext);
    if (full <= (u32)q) {
      full = lds_rd(ctl);
      if (full <= (u32)q) {
        const u64 t0 = now();
        bool drained = !(q > 0 && q < nin + LAG);
        for (;;) {
          __builtin_amdgcn_s_sleep(1);
          full = lds_rd(ctl);
          if (full > (u32)q) break;
          const u64 dt = now() - t0;
          if (!drained && __builtin_amdgcn_readfirstlane((int)(dt > 30ull))) {
            drained = true;
            waitv<0>();
#pragma unroll
            for (int z = LAG; z >= 1; --z) FLAG(q - z);
          }
          if (__builtin_amdgcn_readfirstlane((int)(dt > 2ull * DEADLINE - 1000ull))) { full = DEAD; break; }
        }
        tnext = (u32)__builtin_amdgcn_readfirstlane((u32)(now() - tstart));
      }
      if (full == DEAD) break;
    }
    tnext += (u32)(d == 1 ? TAU1 : TAU2);
    const u32 la = ring + (u32)(q & 7) * 4096u + lane * 16u;
    const u32x4 d0 = *(volatile LDSP u32x4*)(uintptr_t)la;
    const u32x4 d1 = *(volatile LDSP u32x4*)(uintptr_t)(la + 1024u);
    const u32x4 d2 = *(volatile LDSP u32x4*)(uintptr_t)(la + 2048u);
    const u32x4 d3 = *(volatile LDSP u32x4*)(uintptr_t)(la + 3072u);
    const u32 id = lds_rd(ctl + 32u + (u32)(q & 7) * 4u);
    lds_wr(ctl + 4u, (u32)q + 1u);
    u32 o;
    if (q < nin) {
      bool isx; int ix;
      piece_of(q, d, isx, ix);
      o = (isx ? XS_OFF : WS_OFF) + par * XS_PAR + (u32)(ix * 6 + slot) * PIECE;
    } else {
      o = pin0 + min(id, (u32)(np - nin - 1)) * PIECE;
    }
    char* const dst = rws + (size_t)o;
    st16<0>(dl, d0, dst);
    st16<1024>(dl, d1, dst);
    st16<2048>(dl, d2, dst);
    st16<3072>(dl, d3, dst);
    if (nin && q < nin + LAG) {
      waitv<5 * LAG>();
      FLAG(q - LAG);
    } else {
      waitv<4 * WIN>();
    }
    if (q == NS + 11) STAMP(myws, 1);
    if (q == nin - 1) STAMP(myws, 2);
    if (q == np - 1) STAMP(myws, 4);
  }
  waitv<0>();
  STAMP(myws, 3);
}

struct Frag {
  bf16x8 x[2][2];
  bf16x8 w[2][2];
};

struct Loader {
  const char* base;
  u32 stride;
  u32 v[4];
  u32 m0base;
};

template <int STAGE> __device__ __forceinline__ void glds_pair(const Loader& L, int half) {
  const u32 m0v = L.m0base + STAGE * STAGE_B;
  if (half == 0)
    asm volatile("s_mov_b32 m0, %0\n\ts_nop 0\n\t"
                 "global_load_lds_dwordx4 %1, %3 " GLP "\n\t"
                 "global_load_lds_dwordx4 %2, %3 offset:1024 " GLP
                 :: "s"(m0v), "v"(L.v[0]), "v"(L.v[1]), "s"(L.base) : "memory");
  else
    asm volatile("s_mov_b32 m0, %0\n\ts_nop 0\n\t"
                 "global_load_lds_dwordx4 %1, %3 offset:2048 " GLP "\n\t"
                 "global_load_lds_dwordx4 %2, %3 offset:3072 " GLP
                 :: "s"(m0v), "v"(L.v[2]), "v"(L.v[3]), "s"(L.base) : "memory");
}

struct RdAddr { u32 xa[2]; u32 wa[2][2]; };

template <int STAGE> __device__ __forceinline__ bf16x8 rd_x(const RdAddr& A, int mt, int ks) {
  return *(const LDSP bf16x8*)(uintptr_t)(A.xa[ks] + (u32)(STAGE * STAGE_B + mt * 2048));
}
template <int STAGE> __device__ __forceinline__ bf16x8 rd_w(const RdAddr& A, int nt, int ks) {
  s16x4 lo = __builtin_amdgcn_ds_read_tr16_b64_v4i16((LDSP s16x4*)(uintptr_t)(A.wa[nt][0] + (u32)(STAGE * STAGE_B + 8192 + ks * 4096)));
  s16x4 hi = __builtin_amdgcn_ds_read_tr16_b64_v4i16((LDSP s16x4*)(uintptr_t)(A.wa[nt][1] + (u32)(STAGE * STAGE_B + 8192 + ks * 4096)));
  s16x8 v = __builtin_shufflevector(lo, hi, 0, 1, 2, 3, 4, 5, 6, 7);
  return __builtin_bit_cast(bf16x8, v);
}
#define MFMA(nt, mt, ks) acc[nt][mt] = __builtin_amdgcn_mfma_f32_32x32x16_bf16(cur.w[nt][ks], cur.x[mt][ks], acc[nt][mt], 0, 0, 0)

template <int S, int VM, bool LOAD, bool READ>
__device__ __forceinline__ void kstep(f32x16 (&acc)[2][2], const Frag& cur, Frag& nxt, const RdAddr& A, Loader& L) {
  constexpr int SN = (S + 1) & 3;
  if (READ) {
    __builtin_amdgcn_s_waitcnt((VM & 15) | (7 << 4) | (0 << 8) | ((VM >> 4) << 14));
    asm volatile("s_barrier" ::: "memory");
  } else {
    __builtin_amdgcn_s_waitcnt(15 | (7 << 4) | (0 << 8) | (3 << 14));
  }
  SB();
  MFMA(0, 0, 0); SB();
  if (READ) { nxt.w[0][0] = rd_w<SN>(A, 0, 0); nxt.x[0][0] = rd_x<SN>(A, 0, 0); } SB();
  MFMA(0, 1, 0); SB();
  if (READ) { nxt.x[1][0] = rd_x<SN>(A, 1, 0); nxt.w[1][0] = rd_w<SN>(A, 1, 0); } SB();
  MFMA(1, 0, 0); SB();
  if (LOAD) glds_pair<S>(L, 0);
  SB();
  MFMA(1, 1, 0); SB();
  if (READ) { nxt.w[0][1] = rd_w<SN>(A, 0, 1); nxt.x[0][1] = rd_x<SN>(A, 0, 1); } SB();
  MFMA(0, 0, 1); SB();
  if (READ) { nxt.x[1][1] = rd_x<SN>(A, 1, 1); nxt.w[1][1] = rd_w<SN>(A, 1, 1); } SB();
  MFMA(0, 1, 1); SB();
  if (LOAD) { glds_pair<S>(L, 1); L.base += L.stride; }
  SB();
  MFMA(1, 0, 1); SB();
  MFMA(1, 1, 1); SB();
}

template <int S> __device__ __forceinline__ void stage_load(Loader& L) {
  glds_pair<S>(L, 0); glds_pair<S>(L, 1); L.base += L.stride;
}

__device__ __forceinline__ void gemm_phase(f32x16 (&acc)[2][2], const RdAddr& A, Loader& L, int n, int sk, u32& dbgv, int dbg0) {
  Frag f0, f1;
  L.base = uni(L.base + (sk == 0 ? 65536u : 0u));
  L.stride = __builtin_amdgcn_readfirstlane(L.stride);
  asm volatile("s_nop 4" ::: "memory");
  stage_load<0>(L); stage_load<1>(L); stage_load<2>(L); stage_load<3>(L);
  waitv<12>();
  asm volatile("s_barrier" ::: "memory");
#pragma unroll
  for (int ks = 0; ks < 2; ++ks)
#pragma unroll
    for (int i = 0; i < 2; ++i) { f0.w[i][ks] = rd_w<0>(A, i, ks); f0.x[i][ks] = rd_x<0>(A, i, ks); }
  SB();
  for (int t = 0; t < n - 4; t += 4) {
    if (t + 4 == 8 * sk) L.base += 65536;
    kstep<0, 8, true, true>(acc, f0, f1, A, L);
    kstep<1, 8, true, true>(acc, f1, f0, A, L);
    kstep<2, 8, true, true>(acc, f0, f1, A, L);
    kstep<3, 8, true, true>(acc, f1, f0, A, L);
  }
  kstep<0, 8, false, true>(acc, f0, f1, A, L);
  kstep<1, 4, false, true>(acc, f1, f0, A, L);
  kstep<2, 0, false, true>(acc, f0, f1, A, L);
  kstep<3, 0, false, false>(acc, f1, f0, A, L);
}

__device__ __forceinline__ u32 pack2(float a, float b) {
  f32x2 t = {a, b};
  return __builtin_bit_cast(u32, __builtin_convertvector(t, bf16x2));
}
__device__ __forceinline__ float bf_lo(u32 v) { return __builtin_bit_cast(float, v << 16); }
__device__ __forceinline__ float bf_hi(u32 v) { return __builtin_bit_cast(float, v & 0xffff0000u); }

__device__ __forceinline__ void zero8k(u32 l16, u32 l16b, char* pb) {
  const u32x4 z4 = {0u, 0u, 0u, 0u};
  asm volatile("s_nop 4" ::: "memory");
  st16<0>(l16, z4, pb); st16<1024>(l16, z4, pb); st16<2048>(l16, z4, pb); st16<3072>(l16, z4, pb);
  st16<0>(l16b, z4, pb); st16<1024>(l16b, z4, pb); st16<2048>(l16b, z4, pb); st16<3072>(l16b, z4, pb);
}

__device__ __forceinline__ void tile(const Params& P, int tid, int wave, u32 lane, char* smem) {
  const int r = P.rank;
  char* const myws = P.ws[r];
  const int g = r >> 2, b = (r >> 1) & 1, a = r & 1;
  const int bid = blockIdx.x;
  const u64 tstart = now();

  const u32 cnt = bump_issue(CNT_OFF + (u32)(bid * 4 + wave) * 64u, myws);
  STAMP(myws, 0);

  int row0, col0, nrem, tskip, j, e, k, rb;
  int pi0 = -1, pi1 = -1;
  bool fin;
  if (tid < N_MAIN_WG) {
    const int xcd = tid & 7, o = tid >> 3, u = xcd >> 1;
    int rbi, cbi;
    if (u < 2) { rbi = o < 24 ? 1 + o % 3 : 0; cbi = o < 24 ? o / 3 : o - 24; }
    else { rbi = o & 3; cbi = o >> 2; }
    rb = u + 4 * rbi; k = cbi * 2 + (xcd & 1);
    j = rb >> 1; e = rb & 1;
    row0 = (r ^ j) * 512 + b * 256 + e * 128; col0 = g * 2048 + (a ? 15 - k : k) * 128;
    nrem = j >= 2 ? 40 : 48; tskip = j >= 2 ? j - 2 : 99;
    fin = j == 0;
    if (j == 0) pi0 = 6 * NS + 15 - k;
    else if (j >= 2 && k >= 16 - NS) pi0 = ((j ^ 1) - 2) * NS + 15 - k;
  } else {
    const int f = tid - N_MAIN_WG;
    const int qd = 1 + f / 32, rem = f & 31;
    e = (rem >> 3) & 1; k = (rem & 7) * 2 + (rem >> 4);
    const int h = b ^ (qd & 1), g2 = g ^ (qd >> 1);
    row0 = r * 512 + h * 256 + e * 128; col0 = g2 * 2048 + (a ? 15 - k : k) * 128;
    nrem = 0; tskip = 99; j = 0; rb = 0;
    fin = true;
    const int iA = k >= NS ? NOUT1 + (2 * qd - 2) * NOUT + k - NS : -1;
    const int iB = 15 - k >= NS ? NOUT1 + (2 * qd - 1) * NOUT + 15 - k - NS : -1;
    if (k < 8) { pi0 = iA >= 0 ? iA : iB; pi1 = iA >= 0 ? iB : -1; }
    else       { pi0 = iB >= 0 ? iB : iA; pi1 = iB >= 0 ? iA : -1; }
  }

  const u32 sbase = (u32)(uintptr_t)smem;
  const int wm = wave >> 1, wn = wave & 1;
  RdAddr A;
  {
    const u32 h = lane >> 5, m = lane & 31u, f = (lane >> 2) & 3u;
    A.xa[0] = sbase + ((u32)wm * 64u + m) * 64u + 16u * (h ^ f);
    A.xa[1] = sbase + ((u32)wm * 64u + m) * 64u + 16u * ((2u + h) ^ f);
    const u32 blk = (lane >> 4) & 1u, q = (lane & 15u) >> 2, pp = lane & 3u;
#pragma unroll
    for (int nt = 0; nt < 2; ++nt)
#pragma unroll
      for (int t = 0; t < 2; ++t) {
        const u32 row = 8u * h + 4u * t + q;
        const u32 ch = (u32)wn * 8u + (u32)nt * 4u + 2u * blk + (pp & 1u);
        A.wa[nt][t] = sbase + 256u * row + 16u * (ch ^ (((row & 3u) << 2) | ((row >> 2) & 3u))) + 8u * (pp >> 1);
      }
  }

  f32x16 acc[2][2];
#pragma unroll
  for (int i = 0; i < 2; ++i)
#pragma unroll
    for (int k = 0; k < 2; ++k)
#pragma unroll
      for (int z = 0; z < 16; ++z) acc[i][k][z] = 0.f;

  const u32 tile_off = (u32)e * 32768u + (u32)wave * 8192u;
  const u32 l16 = lane * 16u, l16b = lane * 16u + 4096u;
  const int nph = nrem ? 2 : 1;
  const u32 tin = (u32)(T0 + EARLY + TAU * max(slot_w(k), slot_x(j, e, j == 7 ? 2 : 7)));
  const int addph = fin ? nph - 1 : 0;

  Loader L;
  L.m0base = sbase + (u32)wave * 4096u;
  bool dead = false;
  u32 dbgv = 0;
  u32 call = 0;
  for (int ph = 0; ph < nph; ++ph) {
    int n, ts;
    if (ph == 0) {
      n = 16; ts = 99;
      if (wave < 2) {
        L.base = P.x + (size_t)((u32)(row0 + wave * 64) * 1024u);
        L.stride = 64u;
        const u32 c = (lane & 3u) ^ ((lane >> 4) & 3u);
#pragma unroll
        for (int k = 0; k < 4; ++k) L.v[k] = (lane >> 2) * 1024u + c * 16u + (u32)k * (16384u - 1024u);
      } else {
        L.base = P.w + (size_t)((u32)(wave - 2) * 16u * 8192u + (u32)col0 * 2u);
        L.stride = 32u * 8192u;
        const u32 q = lane >> 4;
#pragma unroll
        for (int k = 0; k < 4; ++k) L.v[k] = q * 8192u + (((lane & 15u) ^ ((q << 2) | (u32)k)) * 16u) + (u32)k * (4u * 8192u - 1024u);
      }
    } else {
      n = nrem; ts = tskip;
      STAMP(myws, 1);
      if (wave == 0) {
        const int sl = (int)min(lane, 5u);
        const u32 fo = FIN_OFF + (u32)(sl * 32 + max(slot_w(k), slot_x(j, e, sl + 2))) * 64u;
        bool pend = lane < 6u && sl + 2 != j;
        sleep_until(tstart, tin);
        const u64 t0 = now();
        u32 st = 0;
        for (;;) {
          if (pend) pend = atom_rd(fo, myws) < 16u * call;
          if (__builtin_amdgcn_ballot_w64(pend) == 0ull) break;
          if (give_up(t0, myws, call)) {
            if (lane == 0) st_agt(myws + TMO_OFF, call);
            st = 1;
            break;
          }
          __builtin_amdgcn_s_sleep(PSL);
        }
        if (lane == 0) *(LDSP u32*)(uintptr_t)(sbase + 4u * STAGE_B) = st;
        __builtin_amdgcn_s_waitcnt(0);
      }
      asm volatile("s_barrier" ::: "memory");
      const bool wgdead = lds_rd(sbase + 4u * STAGE_B) != 0;
      STAMP(myws, 2);
      if (wgdead) { dead = true; break; }
      const u32 par = call & 1u;
      if (wave < 2) L.base = myws + (size_t)(XS_OFF + par * XS_PAR + (u32)rb * 6u * PIECE + (u32)wave * 4096u);
      else          L.base = myws + (size_t)(WS_OFF + par * XS_PAR + (u32)k * 6u * PIECE + (u32)(wave - 2) * 4096u);
      L.stride = 8192u;
#pragma unroll
      for (int k = 0; k < 4; ++k) L.v[k] = lane * 16u;
    }
    gemm_phase(acc, A, L, n, ts, dbgv, ph * 8);
    if (ph == nph - 1) STAMP(myws, 3);
    if (ph == 0) {
      call = bump_get<0>(cnt);
      if (pi0 >= 0) {
        char* const pz = myws + (size_t)(PIN_OFF + ((call + 2u) % 3u) * PIN_PAR + tile_off);
        sleep_until(tstart, 900u + (u32)((tid >> 3) & 15) * 100u);
        zero8k(l16, l16b, uni(pz + (size_t)((u32)pi0 * PIECE)));
        if (pi1 >= 0) zero8k(l16, l16b, uni(pz + (size_t)((u32)pi1 * PIECE)));
      }
    }
    const u32 pin = PIN_OFF + (call % 3u) * PIN_PAR + tile_off;
    for (int s = 0; s < 2; ++s) {
      const int pi = s == 0 ? pi0 : pi1;
      if (pi < 0 || ph != addph) continue;
      char* const pb = uni(myws + (size_t)(pin + (u32)pi * PIECE));
      const bool first = pi == 0 || (pi >= NOUT1 && (pi - NOUT1) % NOUT == 0);
      const u32 wo = pin + (u32)(pi - ((!WOWN && (lane & 2u) && !first) ? 1 : 0)) * PIECE + (lane & 1u) * 4096u + 4092u;
      sleep_until(tstart, pi >= NOUT1 ? (u32)(T0 + EARLY + TAU * (NIN - 2 + (pi - NOUT1) % NOUT)) : (fin ? 0u : tin - 300u));
      const u64 t0 = now();
      while (!dead) {
        u32 v = WOWN ? 1u : 0u;
        if (lane < (WOWN ? 2u : 4u)) v = atom_rd(wo, myws);
        if (WOWN ? __builtin_amdgcn_ballot_w64(v == 0u) == 0ull : __builtin_amdgcn_ballot_w64(v != 0u) != 0ull) break;
        if (give_up(t0, myws, call)) { dead = true; break; }
        __builtin_amdgcn_s_sleep(WSL);
      }
      if (s == 1 || pi1 < 0) STAMP(myws, 4);
      u32x4 pv[8];
      while (!dead) {
        asm volatile("s_nop 4\n\t"
                     "global_load_dwordx4 %0, %8, %10 sc0 sc1\n\t"
                     "global_load_dwordx4 %1, %8, %10 offset:1024 sc0 sc1\n\t"
                     "global_load_dwordx4 %2, %8, %10 offset:2048 sc0 sc1\n\t"
                     "global_load_dwordx4 %3, %8, %10 offset:3072 sc0 sc1\n\t"
                     "global_load_dwordx4 %4, %9, %10 sc0 sc1\n\t"
                     "global_load_dwordx4 %5, %9, %10 offset:1024 sc0 sc1\n\t"
                     "global_load_dwordx4 %6, %9, %10 offset:2048 sc0 sc1\n\t"
                     "global_load_dwordx4 %7, %9, %10 offset:3072 sc0 sc1\n\t"
                     "s_waitcnt vmcnt(0)"
                     : "=&v"(pv[0]), "=&v"(pv[1]), "=&v"(pv[2]), "=&v"(pv[3]), "=&v"(pv[4]), "=&v"(pv[5]), "=&v"(pv[6]), "=&v"(pv[7])
                     : "v"(l16), "v"(l16b), "s"(pb) : "memory");
        u32 mn = 0xffffffffu;
  #pragma unroll
        for (int i = 0; i < 8; ++i)
  #pragma unroll
          for (int z = 0; z < 4; ++z) mn = min(mn, pv[i][z]);
        if (__builtin_amdgcn_ballot_w64(mn == 0u) == 0ull) break;
        if (give_up(t0, myws, call)) dead = true;
        if (RSL) __builtin_amdgcn_s_sleep(RSL);
      }
      if (dead) {
        if (lane == 0) st_agt(myws + TMO_OFF, call);
      } else {
  #pragma unroll
        for (int nt = 0; nt < 2; ++nt)
  #pragma unroll
          for (int mt = 0; mt < 2; ++mt)
  #pragma unroll
            for (int sx = 0; sx < 2; ++sx)
  #pragma unroll
              for (int z = 0; z < 4; ++z) {
                const u32 u = pv[(nt * 2 + mt) * 2 + sx][z];
                acc[nt][mt][8 * sx + 2 * z] += bf_lo(u);
                acc[nt][mt][8 * sx + 2 * z + 1] += bf_hi(u);
              }
      }
    }
  }

  STAMP(myws, 6);
  if (!fin) {
    char* dst = uni(myws + (size_t)(POUT_OFF + (u32)((j - 1) * 16 + k) * PIECE + tile_off));
    char* fl = myws + (size_t)(FLOC_OFF + (u32)((j - 1) * 8 + e * 4 + wave) * 128u + (u32)k * 4u);
    asm volatile("s_nop 4" ::: "memory");
#pragma unroll
    for (int nt = 0; nt < 2; ++nt)
#pragma unroll
      for (int mt = 0; mt < 2; ++mt) {
        u32x4 v0, v1;
#pragma unroll
        for (int z = 0; z < 4; ++z) {
          const u32 t0_ = pack2(acc[nt][mt][2 * z], acc[nt][mt][2 * z + 1]);
          const u32 t1_ = pack2(acc[nt][mt][8 + 2 * z], acc[nt][mt][8 + 2 * z + 1]);
          v0[z] = dead ? 0x7fc07fc0u : (t0_ ? t0_ : 0x8000u);
          v1[z] = dead ? 0x7fc07fc0u : (t1_ ? t1_ : 0x8000u);
        }
        const u32 o = nt ? l16b : l16;
        if (mt == 0) { st16<0>(o, v0, dst); st16<1024>(o, v1, dst); }
        else         { st16<2048>(o, v0, dst); st16<3072>(o, v1, dst); }
      }
    waitv<0>();
    if (lane == 0) st_sys(fl, call);
  } else {
    const u32 h = lane >> 5, m = lane & 31u;
    const u32 tb = sbase + (u32)wave * 8192u;
#pragma unroll
    for (int nt = 0; nt < 2; ++nt)
#pragma unroll
      for (int mt = 0; mt < 2; ++mt)
#pragma unroll
        for (int sx = 0; sx < 2; ++sx) {
          u32x4 v;
#pragma unroll
          for (int z = 0; z < 4; ++z) {
            const float lo = acc[nt][mt][8 * sx + 2 * z], hi = acc[nt][mt][8 * sx + 2 * z + 1];
            v[z] = dead ? 0x7fc07fc0u : pack2(lo < 0.f ? 0.f : lo, hi < 0.f ? 0.f : hi);
          }
          *(volatile LDSP u32x4*)(uintptr_t)(tb + ((u32)mt * 32u + m) * 128u + ((((u32)(nt * 4 + sx * 2) + h) ^ (m & 7u)) * 16u)) = v;
        }
    char* ob = uni(P.out + (size_t)((u32)(row0 - r * 512 + wm * 64) * 8192u + (u32)(col0 + wn * 64) * 2u));
    const u32 rr = lane >> 3, cc = lane & 7u;
    const u32 ra = tb + rr * 128u + ((cc ^ rr) * 16u);
    u32x4 ov[8];
#pragma unroll
    for (int i = 0; i < 8; ++i) ov[i] = *(volatile LDSP u32x4*)(uintptr_t)(ra + (u32)i * 1024u);
#pragma unroll
    for (int i = 0; i < 8; ++i) OST<0>(((u32)i * 8u + rr) * 8192u + cc * 16u, ov[i], ob);
  }
  STAMP(myws, 5);
}

__global__ __launch_bounds__(256, 2) void fused_gemm_rs(Params P) {
  extern __shared__ __attribute__((aligned(16))) char smem[];
  const u32 lane = threadIdx.x & 63u;
  const int wave = __builtin_amdgcn_readfirstlane(threadIdx.x >> 6);
  const int bid = blockIdx.x;
  if (bid < N_PUSH_WG) mover(P, wave, lane, smem);
  else if (bid >= 256 && bid < 256 + N_PUSH_WG) {
    for (int i = 0; i < 6; ++i) __builtin_amdgcn_s_sleep(16);
  } else tile(P, bid < 256 ? bid - N_PUSH_WG : bid - 2 * N_PUSH_WG, wave, lane, smem);
}

}

extern "C" void kernel_launch(int rank, int nranks, ncclComm_t comm,
                              hipStream_t stream,
                              void* const* d_in, const int* in_sizes, int n_in,
                              void* d_out, int out_size,
                              const int* in_dtypes, int out_dtype,
                              void* const* d_out_all,
                              void* const* d_ws_all, size_t ws_bytes) {
  if (nranks != NR || n_in < 2 || ws_bytes < (size_t)WS_NEED || (unsigned)rank >= (unsigned)NR) {
    fprintf(stderr, "unsupported configuration\n");
    return;
  }
  Params P;
  P.x = (const char*)d_in[0];
  P.w = (const char*)d_in[1];
  P.out = (char*)d_out;
  for (int i = 0; i < NR; ++i) P.ws[i] = (char*)d_ws_all[i];
  P.rank = rank;
  HIP_CHECK(hipFuncSetAttribute((const void*)fused_gemm_rs, hipFuncAttributeMaxDynamicSharedMemorySize, (int)LDS_BYTES));
  hipLaunchKernelGGL(fused_gemm_rs, dim3(N_WG), dim3(256), LDS_BYTES, stream, P);
  HIP_CHECK(hipGetLastError());
}
```

```cpp
#include <hip/hip_runtime.h>
#include <rccl/rccl.h>
#include <cstdio>
#include <cstdint>

#define HIP_CHECK(expr) do { hipError_t _e = (expr); if (_e != hipSuccess) { \
    fprintf(stderr, "HIP error %s at %s:%d\n", hipGetErrorString(_e), __FILE__, __LINE__); \
    return; } } while (0)

namespace {

typedef unsigned u32;
typedef unsigned long long u64;
typedef u32 u32x4 __attribute__((ext_vector_type(4)));
typedef float f32x2 __attribute__((ext_vector_type(2)));
typedef float f32x16 __attribute__((ext_vector_type(16)));
typedef __bf16 bf16x2 __attribute__((ext_vector_type(2)));
typedef __bf16 bf16x8 __attribute__((ext_vector_type(8)));
typedef short s16x4 __attribute__((ext_vector_type(4)));
typedef short s16x8 __attribute__((ext_vector_type(8)));
#define LDSP __attribute__((address_space(3)))

constexpr int NR = 8;
constexpr u32 PIECE = 65536u;
constexpr u32 MIB = 1u << 20;
#define NS 2
#define OX 14
#define TAU 115
#define T0 50
#define EARLY 100
#define TAU1 110
#define TAU2 0
#define STG 14
#define STGD 2
#define PSL 4
#define LSL 0
#define WSL 3
#define RSL 0
#define WOWN 0
#define ABSG 1
#define LAG 1
#define WIN 3
#define GLP "sc0 sc1"
#define MLP "sc0 sc1"
constexpr int NIN = 30;
constexpr int NOUT = 16 - NS;
constexpr int NOUT1 = 16 + 6 * NS;
constexpr u32 XS_OFF = 0;
constexpr u32 XS_PAR = 6u * MIB;
constexpr u32 WS_OFF = 12u * MIB;
constexpr u32 PIN_OFF = 24u * MIB;
constexpr u32 PIN_PAR = 7u * MIB;
constexpr u32 POUT_OFF = 45u * MIB;
constexpr u32 FLG_OFF = 52u * MIB;
constexpr u32 FIN_OFF = FLG_OFF;
constexpr u32 FLOC_OFF = FLG_OFF + 16384u;
constexpr u32 TMO_OFF = FLG_OFF + 49152u;
constexpr u32 DUMMY_OFF = FLG_OFF + 65536u;
constexpr u32 DBG_OFF = FLG_OFF + 131072u;
constexpr u32 CNT_OFF = FLG_OFF + MIB + 262144u;
constexpr u32 WS_NEED = FLG_OFF + 2u * MIB;

constexpr int N_PUSH_WG = 56;
constexpr int N_MAIN_WG = 256;
constexpr int N_FIN_WG = 96;
constexpr int N_WG = 2 * N_PUSH_WG + N_MAIN_WG + N_FIN_WG;
constexpr u32 STAGE_B = 16384u;
constexpr u32 LDS_BYTES = 4u * STAGE_B + 1024u;
constexpr u64 DEADLINE = 100000000ull;
constexpr u32 DEAD = 0x7fffffffu;

struct Params {
  const char* x;
  const char* w;
  char* out;
  char* ws[NR];
  int rank;
};

#define SB() __builtin_amdgcn_sched_barrier(0)
template <int N> __device__ __forceinline__ void waitv() { asm volatile("s_waitcnt vmcnt(%0)" :: "i"(N) : "memory"); }

__device__ __forceinline__ u64 now() { return __builtin_amdgcn_s_memrealtime(); }
template <typename T> __device__ __forceinline__ T* uni(T* p) {
  const u64 v = (u64)(uintptr_t)p;
  const u32 lo = __builtin_amdgcn_readfirstlane((u32)v), hi = __builtin_amdgcn_readfirstlane((u32)(v >> 32));
  return (T*)(uintptr_t)(((u64)hi << 32) | lo);
}
__device__ __forceinline__ u32 ld_sys(const void* p) {
  return __hip_atomic_load((const u32*)p, __ATOMIC_RELAXED, __HIP_MEMORY_SCOPE_SYSTEM);
}
__device__ __forceinline__ void st_sys(void* p, u32 v) {
  __hip_atomic_store((u32*)p, v, __ATOMIC_RELAXED, __HIP_MEMORY_SCOPE_SYSTEM);
}
__device__ __forceinline__ void st_agt(void* p, u32 v) {
  __hip_atomic_store((u32*)p, v, __ATOMIC_RELAXED, __HIP_MEMORY_SCOPE_AGENT);
}
__device__ __forceinline__ bool give_up(u64 t0, const char* ws, u32 call) {
  const u64 dt = now() - t0;
  if (!__builtin_amdgcn_readfirstlane((int)(dt > 50000ull))) return false;
  __builtin_amdgcn_s_sleep(64);
  if (__builtin_amdgcn_readfirstlane((int)(dt > DEADLINE))) return true;
  return __builtin_amdgcn_readfirstlane(ld_sys(ws + TMO_OFF)) == call;
}
__device__ __forceinline__ void sleep_until(u64 tstart, u32 t) {
  for (;;) {
    const u32 el = (u32)__builtin_amdgcn_readfirstlane((u32)(now() - tstart));
    if (el >= t) break;
    if (t - el > 40u) __builtin_amdgcn_s_sleep(12); else __builtin_amdgcn_s_sleep(1);
  }
}
template <int IMM> __device__ __forceinline__ void st16(u32 off, u32x4 v, char* base) {
  asm volatile("global_store_dwordx4 %0, %1, %2 offset:%3 sc0 sc1\n\ts_nop 1" :: "v"(off), "v"(v), "s"(base), "i"(IMM) : "memory");
}
template <int IMM> __device__ __forceinline__ void st16p(u32 off, u32x4 v, char* base) {
  asm volatile("global_store_dwordx4 %0, %1, %2 offset:%3\n\ts_nop 1" :: "v"(off), "v"(v), "s"(base), "i"(IMM) : "memory");
}
#define OST st16
__device__ __forceinline__ void dma16x4(u32 lds, const u32 (&o)[4], const char* base) {
  asm volatile("s_mov_b32 m0, %0\n\ts_nop 3\n\t"
               "global_load_lds_dwordx4 %1, %5 " MLP "\n\t"
               "global_load_lds_dwordx4 %2, %5 offset:1024 " MLP "\n\t"
               "global_load_lds_dwordx4 %3, %5 offset:2048 " MLP "\n\t"
               "global_load_lds_dwordx4 %4, %5 offset:3072 " MLP
               :: "s"(lds), "v"(o[0]), "v"(o[1]), "v"(o[2]), "v"(o[3]), "s"(base) : "memory");
}
__device__ __forceinline__ u32 atom_rd(u32 off, const char* base) {
  u32 r;
  asm volatile("global_atomic_add %0, %1, %2, %3 sc0 sc1\n\ts_waitcnt vmcnt(0)" : "=&v"(r) : "v"(off), "v"(0u), "s"(base) : "memory");
  return r;
}
__device__ __forceinline__ u32 bump_issue(u32 off, const char* base) {
  u32 r;
  asm volatile("s_mov_b64 exec, 1\n\ts_nop 3\n\tglobal_atomic_add %0, %1, %2, %3 sc0 sc1\n\ts_mov_b64 exec, -1"
               : "=&v"(r) : "v"(off), "v"(1u), "s"(base) : "memory");
  return r;
}
template <int N> __device__ __forceinline__ u32 bump_get(u32 r) {
  u32 v;
  asm volatile("s_waitcnt vmcnt(%2)\n\tv_readfirstlane_b32 %0, %1\n\ts_nop 4" : "=s"(v) : "v"(r), "i"(N) : "memory");
  return v + 1u;
}
__device__ __forceinline__ void add1_lane0(u32 off, char* base) {
  asm volatile("s_mov_b64 exec, 1\n\ts_nop 3\n\tglobal_atomic_add %0, %1, %2 sc1\n\ts_mov_b64 exec, -1"
               :: "v"(off), "v"(1u), "s"(base) : "memory");
}
__device__ __forceinline__ void st4_lane0(u32 off, u32 val, char* base) {
  asm volatile("s_mov_b64 exec, 1\n\ts_nop 3\n\tglobal_store_dword %0, %1, %2 sc0 sc1\n\ts_mov_b64 exec, -1"
               :: "v"(off), "v"(val), "s"(base) : "memory");
}

#define STAMP(ws, idx) do {} while (0)

__device__ __forceinline__ void piece_of(int q, int d, bool& isx, int& idx) {
  const int t = q - NS, u = q - NS - 12;
  int j = (t >> 1) + 2;
  j += j >= d ? 1 : 0;
  j = j == 8 ? 1 : j;
  isx = (q >= NS && u < 0) || (u >= OX && u < OX + 2);
  idx = q < NS ? q : (u < 0 ? 2 * j + (t & 1) : (u < OX ? NS + u : (u < OX + 2 ? u - OX : NS + u - 2)));
}
__device__ __forceinline__ int slot_w(int k) { return k < NS ? k : (k - NS < OX ? 12 + k : 14 + k); }
__device__ __forceinline__ int slot_x(int j, int e, int d) {
  return j == 0 ? NS + 12 + OX + e : NS + 2 * (j == 1 ? 5 : j - 2 - (j > d ? 1 : 0)) + e;
}
__device__ __forceinline__ u32 lds_rd(u32 a) { return __builtin_amdgcn_readfirstlane(*(volatile LDSP u32*)(uintptr_t)a); }
__device__ __forceinline__ void lds_wr(u32 a, u32 v) { *(volatile LDSP u32*)(uintptr_t)a = v; }

__device__ __forceinline__ void mover(const Params& P, int wave, u32 lane, char* smem) {
  __builtin_amdgcn_s_setprio(3);
  const u64 tstart = now();
  const int s = P.rank;
  char* const myws = P.ws[s];
  const int pair = wave >> 1;
  const int d = 1 + ((int)(blockIdx.x >> 3) + 3 * pair) % 7;
  const int p = (blockIdx.x & 7) * 2 + pair;
  const u32 r = (u32)(s ^ d) & (u32)(NR - 1);
  const int nin = d == 1 ? 0 : NIN;
  const int np = d == 1 ? NOUT1 : NIN + NOUT;
  const u32 ring = (u32)(uintptr_t)smem + (u32)pair * 32768u;
  const u32 ctl = (u32)(uintptr_t)smem + 65536u + (u32)pair * 512u;
  const u32 cnt = bump_issue(CNT_OFF + (u32)(blockIdx.x * 4 + wave) * 64u, myws);

  if ((wave & 1) == 0) {
    const char* const xin = P.x;
    const char* const win = P.w;
    const int rg = r >> 2, rb_ = (r >> 1) & 1, ra = r & 1;
    u32 xo[4], wo[4], oo[4];
    {
      const u32 xc = (lane & 3u) ^ ((lane >> 4) & 3u), q4 = lane >> 4;
#pragma unroll
      for (int k = 0; k < 4; ++k) {
        xo[k] = ((u32)(p & 1) * 64u + (u32)k * 16u + (lane >> 2)) * 1024u + (u32)(p >> 1) * 64u + xc * 16u - (u32)k * 1024u;
        wo[k] = ((u32)(p >> 1) * 32u + (u32)(p & 1) * 16u + (u32)k * 4u + q4) * 8192u
              + (((lane & 15u) ^ ((q4 << 2) | (u32)k)) * 16u) - (u32)k * 1024u;
        oo[k] = (u32)p * 4096u + lane * 16u;
      }
    }
    u32 poll_off;
    {
      const int i = (int)lane < np - nin ? (int)lane : 0;
      const int jj = d != 1 ? d : (i < 6 * NS ? 2 + i / NS : 1);
      const int kk = d != 1 ? NS + i : (i < 6 * NS ? i % NS : i - 6 * NS);
      poll_off = FLOC_OFF + (u32)((jj - 1) * 8 + (p >> 1)) * 128u + (u32)kk * 4u;
    }
#define ISSUE_LOADS(QQ, II) do { \
      const int q_ = (QQ); \
      const u32 dst_ = ring + (u32)(q_ & 7) * 4096u; \
      if (q_ < nin) { \
        bool isx_; int ix_; \
        piece_of(q_, d, isx_, ix_); \
        if (isx_) dma16x4(dst_, xo, xin + (size_t)(((r ^ (u32)(ix_ >> 1)) * 512u + (u32)rb_ * 256u + (u32)(ix_ & 1) * 128u) * 1024u + (u32)ra * 512u)); \
        else      dma16x4(dst_, wo, win + (size_t)((u32)ra * 256u * 8192u + (u32)rg * 4096u + (u32)(ra ? 15 - ix_ : ix_) * 256u)); \
      } else { \
        const int i_ = (II); \
        const int jj_ = d != 1 ? d : (i_ < 6 * NS ? 2 + i_ / NS : 1); \
        const int kk_ = d != 1 ? NS + i_ : (i_ < 6 * NS ? i_ % NS : i_ - 6 * NS); \
        dma16x4(dst_, oo, myws + (size_t)(POUT_OFF + (u32)((jj_ - 1) * 16 + kk_) * PIECE)); \
      } \
    } while (0)

    int q0 = 0;
    u32 pub = 0, call;
    if (nin) {
      ISSUE_LOADS(0, 0); ISSUE_LOADS(1, 0); ISSUE_LOADS(2, 0);
    }
    lds_wr(ctl, 0u);
    asm volatile("s_waitcnt lgkmcnt(0)\n\ts_barrier" ::: "memory");
    if (nin) {
      call = bump_get<8>(cnt);
      lds_wr(ctl, 1u);
      q0 = 3; pub = 1;
    } else {
      call = bump_get<0>(cnt);
    }
    STAMP(myws, 0);

    if (!nin) sleep_until(tstart, (u32)(T0 + (NS + 2) * TAU + 500));
    u64 ready = 0;
    u64 todo = (1ull << (np - nin)) - 1ull;
    u32 cons = 0;
    bool dead = false;
    u32 dbg1 = 0, dbg2 = 0;
    for (int q = q0; q < np; ++q) {
      bool rdy = q < nin || (ready & todo) != 0ull;
      if (cons + 8u <= (u32)q) cons = lds_rd(ctl + 4u);
      if (!rdy || cons + 8u <= (u32)q) {
        waitv<0>();
        pub = (u32)q;
        lds_wr(ctl, pub);
        const u64 t0 = now();
        for (;;) {
          if (!rdy) {
            u32 pv = 0;
            if ((todo >> lane) & 1ull) pv = atom_rd(poll_off, myws);
            ready |= __builtin_amdgcn_ballot_w64(pv >= call);
            rdy = (ready & todo) != 0ull;
            if (LSL && !rdy) __builtin_amdgcn_s_sleep(LSL);
          } else {
            __builtin_amdgcn_s_sleep(2);
          }
          cons = lds_rd(ctl + 4u);
          if (rdy && cons + 8u > (u32)q) break;
          if (give_up(t0, myws, call)) { dead = true; break; }
        }
        if (dead) break;
      }
      int id = 0;
      if (q >= nin) {
        id = __builtin_ctzll(ready & todo);
        todo &= ~(1ull << id);
        lds_wr(ctl + 32u + (u32)(q & 7) * 4u, (u32)id);
      }
      ISSUE_LOADS(q, id);
      waitv<8>();
      if (q - 1 > (int)pub) { pub = (u32)(q - 1); lds_wr(ctl, pub); }
      if (q == nin - 1) STAMP(myws, 1);
    }
    waitv<0>();
    if (dead) st4_lane0(TMO_OFF, call, myws);
    lds_wr(ctl, dead ? DEAD : (u32)np);
    STAMP(myws, 3);
    return;
  }

  char* const rws = uni(P.ws[r]);
  lds_wr(ctl + 4u, 0u);
  asm volatile("s_waitcnt lgkmcnt(0)\n\ts_barrier" ::: "memory");
  const u32 call = bump_get<0>(cnt);
  STAMP(myws, 0);
  const u32 par = call & 1u;
  const int slot = d - 2;
  const u32 dl = (u32)p * 4096u + lane * 16u;
  const u32 pin0 = PIN_OFF + (call % 3u) * PIN_PAR + (u32)(d == 1 ? 0 : NOUT1 + slot * NOUT) * PIECE;
  int nflag = 0;
#define FLAG(QQ) do { \
    const int q_ = (QQ); \
    const bool real_ = q_ == nflag && q_ < nin; \
    const u32 o_ = real_ ? FIN_OFF + (u32)(slot * 32 + q_) * 64u : DUMMY_OFF + (u32)(blockIdx.x * 4 + wave) * 128u; \
    add1_lane0(o_, uni(real_ ? rws : myws)); \
    nflag += real_ ? 1 : 0; \
  } while (0)

  const u32 toff = (u32)TAU - 1u - ((u32)__builtin_amdgcn_readfirstlane((u32)tstart) + (u32)(T0 + TAU - 1) - ((u32)(((p + STGD * d) & 15) * TAU) >> 4)) % (u32)TAU;
  u32 full = 0, tnext = 0;
  u32 dbg1 = 0, dbg2 = 0;
  for (int q = 0; q < np; ++q) {
    if (d != 1) tnext = max(tnext, (u32)q * (u32)TAU + (u32)T0 + toff);
    sleep_until(tstart, tnext);
    if (full <= (u32)q) {
      full = lds_rd(ctl);
      if (full <= (u32)q) {
        const u64 t0 = now();
        bool drained = !(q > 0 && q < nin + LAG);
        for (;;) {
          __builtin_amdgcn_s_sleep(1);
          full = lds_rd(ctl);
          if (full > (u32)q) break;
          const u64 dt = now() - t0;
          if (!drained && __builtin_amdgcn_readfirstlane((int)(dt > 30ull))) {
            drained = true;
            waitv<0>();
#pragma unroll
            for (int z = LAG; z >= 1; --z) FLAG(q - z);
          }
          if (__builtin_amdgcn_readfirstlane((int)(dt > 2ull * DEADLINE - 1000ull))) { full = DEAD; break; }
        }
        tnext = (u32)__builtin_amdgcn_readfirstlane((u32)(now() - tstart));
      }
      if (full == DEAD) break;
    }
    tnext += (u32)(d == 1 ? TAU1 : TAU2);
    const u32 la = ring + (u32)(q & 7) * 4096u + lane * 16u;
    const u32x4 d0 = *(volatile LDSP u32x4*)(uintptr_t)la;
    const u32x4 d1 = *(volatile LDSP u32x4*)(uintptr_t)(la + 1024u);
    const u32x4 d2 = *(volatile LDSP u32x4*)(uintptr_t)(la + 2048u);
    const u32x4 d3 = *(volatile LDSP u32x4*)(uintptr_t)(la + 3072u);
    const u32 id = lds_rd(ctl + 32u + (u32)(q & 7) * 4u);
    lds_wr(ctl + 4u, (u32)q + 1u);
    u32 o;
    if (q < nin) {
      bool isx; int ix;
      piece_of(q, d, isx, ix);
      o = (isx ? XS_OFF : WS_OFF) + par * XS_PAR + (u32)(ix * 6 + slot) * PIECE;
    } else {
      o = pin0 + min(id, (u32)(np - nin - 1)) * PIECE;
    }
    char* const dst = rws + (size_t)o;
    st16<0>(dl, d0, dst);
    st16<1024>(dl, d1, dst);
    st16<2048>(dl, d2, dst);
    st16<3072>(dl, d3, dst);
    if (nin && q < nin + LAG) {
      waitv<5 * LAG>();
      FLAG(q - LAG);
    } else {
      waitv<4 * WIN>();
    }
    if (q == NS + 11) STAMP(myws, 1);
    if (q == nin - 1) STAMP(myws, 2);
    if (q == np - 1) STAMP(myws, 4);
  }
  waitv<0>();
  STAMP(myws, 3);
}

struct Frag {
  bf16x8 x[2][2];
  bf16x8 w[2][2];
};

struct Loader {
  const char* base;
  u32 stride;
  u32 v[4];
  u32 m0base;
};

template <int STAGE> __device__ __forceinline__ void glds_pair(const Loader& L, int half) {
  const u32 m0v = L.m0base + STAGE * STAGE_B;
  if (half == 0)
    asm volatile("s_mov_b32 m0, %0\n\ts_nop 0\n\t"
                 "global_load_lds_dwordx4 %1, %3 " GLP "\n\t"
                 "global_load_lds_dwordx4 %2, %3 offset:1024 " GLP
                 :: "s"(m0v), "v"(L.v[0]), "v"(L.v[1]), "s"(L.base) : "memory");
  else
    asm volatile("s_mov_b32 m0, %0\n\ts_nop 0\n\t"
                 "global_load_lds_dwordx4 %1, %3 offset:2048 " GLP "\n\t"
                 "global_load_lds_dwordx4 %2, %3 offset:3072 " GLP
                 :: "s"(m0v), "v"(L.v[2]), "v"(L.v[3]), "s"(L.base) : "memory");
}

struct RdAddr { u32 xa[2]; u32 wa[2][2]; };

template <int STAGE> __device__ __forceinline__ bf16x8 rd_x(const RdAddr& A, int mt, int ks) {
  return *(const LDSP bf16x8*)(uintptr_t)(A.xa[ks] + (u32)(STAGE * STAGE_B + mt * 2048));
}
template <int STAGE> __device__ __forceinline__ bf16x8 rd_w(const RdAddr& A, int nt, int ks) {
  s16x4 lo = __builtin_amdgcn_ds_read_tr16_b64_v4i16((LDSP s16x4*)(uintptr_t)(A.wa[nt][0] + (u32)(STAGE * STAGE_B + 8192 + ks * 4096)));
  s16x4 hi = __builtin_amdgcn_ds_read_tr16_b64_v4i16((LDSP s16x4*)(uintptr_t)(A.wa[nt][1] + (u32)(STAGE * STAGE_B + 8192 + ks * 4096)));
  s16x8 v = __builtin_shufflevector(lo, hi, 0, 1, 2, 3, 4, 5, 6, 7);
  return __builtin_bit_cast(bf16x8, v);
}
#define MFMA(nt, mt, ks) acc[nt][mt] = __builtin_amdgcn_mfma_f32_32x32x16_bf16(cur.w[nt][ks], cur.x[mt][ks], acc[nt][mt], 0, 0, 0)

template <int S, int VM, bool LOAD, bool READ>
__device__ __forceinline__ void kstep(f32x16 (&acc)[2][2], const Frag& cur, Frag& nxt, const RdAddr& A, Loader& L) {
  constexpr int SN = (S + 1) & 3;
  if (READ) {
    __builtin_amdgcn_s_waitcnt((VM & 15) | (7 << 4) | (0 << 8) | ((VM >> 4) << 14));
    asm volatile("s_barrier" ::: "memory");
  } else {
    __builtin_amdgcn_s_waitcnt(15 | (7 << 4) | (0 << 8) | (3 << 14));
  }
  SB();
  MFMA(0, 0, 0); SB();
  if (READ) { nxt.w[0][0] = rd_w<SN>(A, 0, 0); nxt.x[0][0] = rd_x<SN>(A, 0, 0); } SB();
  MFMA(0, 1, 0); SB();
  if (READ) { nxt.x[1][0] = rd_x<SN>(A, 1, 0); nxt.w[1][0] = rd_w<SN>(A, 1, 0); } SB();
  MFMA(1, 0, 0); SB();
  if (LOAD) glds_pair<S>(L, 0);
  SB();
  MFMA(1, 1, 0); SB();
  if (READ) { nxt.w[0][1] = rd_w<SN>(A, 0, 1); nxt.x[0][1] = rd_x<SN>(A, 0, 1); } SB();
  MFMA(0, 0, 1); SB();
  if (READ) { nxt.x[1][1] = rd_x<SN>(A, 1, 1); nxt.w[1][1] = rd_w<SN>(A, 1, 1); } SB();
  MFMA(0, 1, 1); SB();
  if (LOAD) { glds_pair<S>(L, 1); L.base += L.stride; }
  SB();
  MFMA(1, 0, 1); SB();
  MFMA(1, 1, 1); SB();
}

template <int S> __device__ __forceinline__ void stage_load(Loader& L) {
  glds_pair<S>(L, 0); glds_pair<S>(L, 1); L.base += L.stride;
}

__device__ __forceinline__ void gemm_phase(f32x16 (&acc)[2][2], const RdAddr& A, Loader& L, int n, int sk, u32& dbgv, int dbg0) {
  Frag f0, f1;
  L.base = uni(L.base + (sk == 0 ? 65536u : 0u));
  L.stride = __builtin_amdgcn_readfirstlane(L.stride);
  asm volatile("s_nop 4" ::: "memory");
  stage_load<0>(L); stage_load<1>(L); stage_load<2>(L); stage_load<3>(L);
  waitv<12>();
  asm volatile("s_barrier" ::: "memory");
#pragma unroll
  for (int ks = 0; ks < 2; ++ks)
#pragma unroll
    for (int i = 0; i < 2; ++i) { f0.w[i][ks] = rd_w<0>(A, i, ks); f0.x[i][ks] = rd_x<0>(A, i, ks); }
  SB();
  for (int t = 0; t < n - 4; t += 4) {
    if (t + 4 == 8 * sk) L.base += 65536;
    kstep<0, 8, true, true>(acc, f0, f1, A, L);
    kstep<1, 8, true, true>(acc, f1, f0, A, L);
    kstep<2, 8, true, true>(acc, f0, f1, A, L);
    kstep<3, 8, true, true>(acc, f1, f0, A, L);
  }
  kstep<0, 8, false, true>(acc, f0, f1, A, L);
  kstep<1, 4, false, true>(acc, f1, f0, A, L);
  kstep<2, 0, false, true>(acc, f0, f1, A, L);
  kstep<3, 0, false, false>(acc, f1, f0, A, L);
}

__device__ __forceinline__ u32 pack2(float a, float b) {
  f32x2 t = {a, b};
  return __builtin_bit_cast(u32, __builtin_convertvector(t, bf16x2));
}
__device__ __forceinline__ float bf_lo(u32 v) { return __builtin_bit_cast(float, v << 16); }
__device__ __forceinline__ float bf_hi(u32 v) { return __builtin_bit_cast(float, v & 0xffff0000u); }

__device__ __forceinline__ void zero8k(u32 l16, u32 l16b, char* pb) {
  const u32x4 z4 = {0u, 0u, 0u, 0u};
  asm volatile("s_nop 4" ::: "memory");
  st16<0>(l16, z4, pb); st16<1024>(l16, z4, pb); st16<2048>(l16, z4, pb); st16<3072>(l16, z4, pb);
  st16<0>(l16b, z4, pb); st16<1024>(l16b, z4, pb); st16<2048>(l16b, z4, pb); st16<3072>(l16b, z4, pb);
}

__device__ __forceinline__ void tile(const Params& P, int tid, int wave, u32 lane, char* smem) {
  const int r = P.rank;
  char* const myws = P.ws[r];
  const int g = r >> 2, b = (r >> 1) & 1, a = r & 1;
  const int bid = blockIdx.x;
  const u64 tstart = now();

  const u32 cnt = bump_issue(CNT_OFF + (u32)(bid * 4 + wave) * 64u, myws);
  STAMP(myws, 0);

  int row0, col0, nrem, tskip, j, e, k, rb;
  int pi0 = -1, pi1 = -1;
  bool fin;
  if (tid < N_MAIN_WG) {
    const int xcd = tid & 7, o = tid >> 3, u = xcd >> 1;
    int rbi, cbi;
    if (u < 2) { rbi = o < 24 ? 1 + o % 3 : 0; cbi = o < 24 ? o / 3 : o - 24; }
    else { rbi = o & 3; cbi = o >> 2; }
    rb = u + 4 * rbi; k = cbi * 2 + (xcd & 1);
    j = rb >> 1; e = rb & 1;
    row0 = (r ^ j) * 512 + b * 256 + e * 128; col0 = g * 2048 + (a ? 15 - k : k) * 128;
    nrem = j >= 2 ? 40 : 48; tskip = j >= 2 ? j - 2 : 99;
    fin = j == 0;
    if (j == 0) pi0 = 6 * NS + 15 - k;
    else if (j >= 2 && k >= 16 - NS) pi0 = ((j ^ 1) - 2) * NS + 15 - k;
  } else {
    const int f = tid - N_MAIN_WG;
    const int qd = 1 + f / 32, rem = f & 31;
    e = (rem >> 3) & 1; k = (rem & 7) * 2 + (rem >> 4);
    const int h = b ^ (qd & 1), g2 = g ^ (qd >> 1);
    row0 = r * 512 + h * 256 + e * 128; col0 = g2 * 2048 + (a ? 15 - k : k) * 128;
    nrem = 0; tskip = 99; j = 0; rb = 0;
    fin = true;
    const int iA = k >= NS ? NOUT1 + (2 * qd - 2) * NOUT + k - NS : -1;
    const int iB = 15 - k >= NS ? NOUT1 + (2 * qd - 1) * NOUT + 15 - k - NS : -1;
    if (k < 8) { pi0 = iA >= 0 ? iA : iB; pi1 = iA >= 0 ? iB : -1; }
    else       { pi0 = iB >= 0 ? iB : iA; pi1 = iB >= 0 ? iA : -1; }
  }

  const u32 sbase = (u32)(uintptr_t)smem;
  const int wm = wave >> 1, wn = wave & 1;
  RdAddr A;
  {
    const u32 h = lane >> 5, m = lane & 31u, f = (lane >> 2) & 3u;
    A.xa[0] = sbase + ((u32)wm * 64u + m) * 64u + 16u * (h ^ f);
    A.xa[1] = sbase + ((u32)wm * 64u + m) * 64u + 16u * ((2u + h) ^ f);
    const u32 blk = (lane >> 4) & 1u, q = (lane & 15u) >> 2, pp = lane & 3u;
#pragma unroll
    for (int nt = 0; nt < 2; ++nt)
#pragma unroll
      for (int t = 0; t < 2; ++t) {
        const u32 row = 8u * h + 4u * t + q;
        const u32 ch = (u32)wn * 8u + (u32)nt * 4u + 2u * blk + (pp & 1u);
        A.wa[nt][t] = sbase + 256u * row + 16u * (ch ^ (((row & 3u) << 2) | ((row >> 2) & 3u))) + 8u * (pp >> 1);
      }
  }

  f32x16 acc[2][2];
#pragma unroll
  for (int i = 0; i < 2; ++i)
#pragma unroll
    for (int k = 0; k < 2; ++k)
#pragma unroll
      for (int z = 0; z < 16; ++z) acc[i][k][z] = 0.f;

  const u32 tile_off = (u32)e * 32768u + (u32)wave * 8192u;
  const u32 l16 = lane * 16u, l16b = lane * 16u + 4096u;
  const int nph = nrem ? 2 : 1;
  const u32 tin = (u32)(T0 + EARLY + TAU * max(slot_w(k), slot_x(j, e, j == 7 ? 2 : 7)));
  const int addph = fin ? nph - 1 : 0;

  Loader L;
  L.m0base = sbase + (u32)wave * 4096u;
  bool dead = false;
  u32 dbgv = 0;
  u32 call = 0;
  for (int ph = 0; ph < nph; ++ph) {
    int n, ts;
    if (ph == 0) {
      n = 16; ts = 99;
      if (wave < 2) {
        L.base = P.x + (size_t)((u32)(row0 + wave * 64) * 1024u);
        L.stride = 64u;
        const u32 c = (lane & 3u) ^ ((lane >> 4) & 3u);
#pragma unroll
        for (int k = 0; k < 4; ++k) L.v[k] = (lane >> 2) * 1024u + c * 16u + (u32)k * (16384u - 1024u);
      } else {
        L.base = P.w + (size_t)((u32)(wave - 2) * 16u * 8192u + (u32)col0 * 2u);
        L.stride = 32u * 8192u;
        const u32 q = lane >> 4;
#pragma unroll
        for (int k = 0; k < 4; ++k) L.v[k] = q * 8192u + (((lane & 15u) ^ ((q << 2) | (u32)k)) * 16u) + (u32)k * (4u * 8192u - 1024u);
      }
    } else {
      n = nrem; ts = tskip;
      STAMP(myws, 1);
      if (wave == 0) {
        const int sl = (int)min(lane, 5u);
        const u32 fo = FIN_OFF + (u32)(sl * 32 + max(slot_w(k), slot_x(j, e, sl + 2))) * 64u;
        bool pend = lane < 6u && sl + 2 != j;
        sleep_until(tstart, tin);
        const u64 t0 = now();
        u32 st = 0;
        for (;;) {
          if (pend) pend = atom_rd(fo, myws) < 16u * call;
          if (__builtin_amdgcn_ballot_w64(pend) == 0ull) break;
          if (give_up(t0, myws, call)) {
            if (lane == 0) st_agt(myws + TMO_OFF, call);
            st = 1;
            break;
          }
          __builtin_amdgcn_s_sleep(PSL);
        }
        if (lane == 0) *(LDSP u32*)(uintptr_t)(sbase + 4u * STAGE_B) = st;
        __builtin_amdgcn_s_waitcnt(0);
      }
      asm volatile("s_barrier" ::: "memory");
      const bool wgdead = lds_rd(sbase + 4u * STAGE_B) != 0;
      STAMP(myws, 2);
      if (wgdead) { dead = true; break; }
      const u32 par = call & 1u;
      if (wave < 2) L.base = myws + (size_t)(XS_OFF + par * XS_PAR + (u32)rb * 6u * PIECE + (u32)wave * 4096u);
      else          L.base = myws + (size_t)(WS_OFF + par * XS_PAR + (u32)k * 6u * PIECE + (u32)(wave - 2) * 4096u);
      L.stride = 8192u;
#pragma unroll
      for (int k = 0; k < 4; ++k) L.v[k] = lane * 16u;
    }
    gemm_phase(acc, A, L, n, ts, dbgv, ph * 8);
    if (ph == nph - 1) STAMP(myws, 3);
    if (ph == 0) {
      call = bump_get<0>(cnt);
      if (pi0 >= 0) {
        char* const pz = myws + (size_t)(PIN_OFF + ((call + 2u) % 3u) * PIN_PAR + tile_off);
        sleep_until(tstart, 900u + (u32)((tid >> 3) & 15) * 100u);
        zero8k(l16, l16b, uni(pz + (size_t)((u32)pi0 * PIECE)));
        if (pi1 >= 0) zero8k(l16, l16b, uni(pz + (size_t)((u32)pi1 * PIECE)));
      }
    }
    const u32 pin = PIN_OFF + (call % 3u) * PIN_PAR + tile_off;
    for (int s = 0; s < 2; ++s) {
      const int pi = s == 0 ? pi0 : pi1;
      if (pi < 0 || ph != addph) continue;
      char* const pb = uni(myws + (size_t)(pin + (u32)pi * PIECE));
      const bool first = pi == 0 || (pi >= NOUT1 && (pi - NOUT1) % NOUT == 0);
      const u32 wo = pin + (u32)(pi - ((!WOWN && (lane & 2u) && !first) ? 1 : 0)) * PIECE + (lane & 1u) * 4096u + 4092u;
      sleep_until(tstart, pi >= NOUT1 ? (u32)(T0 + EARLY + TAU * (NIN - 2 + (pi - NOUT1) % NOUT)) : (fin ? 0u : tin - 300u));
      const u64 t0 = now();
      while (!dead) {
        u32 v = WOWN ? 1u : 0u;
        if (lane < (WOWN ? 2u : 4u)) v = atom_rd(wo, myws);
        if (WOWN ? __builtin_amdgcn_ballot_w64(v == 0u) == 0ull : __builtin_amdgcn_ballot_w64(v != 0u) != 0ull) break;
        if (give_up(t0, myws, call)) { dead = true; break; }
        __builtin_amdgcn_s_sleep(WSL);
      }
      if (s == 1 || pi1 < 0) STAMP(myws, 4);
      u32x4 pv[8];
      while (!dead) {
        asm volatile("s_nop 4\n\t"
                     "global_load_dwordx4 %0, %8, %10 sc0 sc1\n\t"
                     "global_load_dwordx4 %1, %8, %10 offset:1024 sc0 sc1\n\t"
                     "global_load_dwordx4 %2, %8, %10 offset:2048 sc0 sc1\n\t"
                     "global_load_dwordx4 %3, %8, %10 offset:3072 sc0 sc1\n\t"
                     "global_load_dwordx4 %4, %9, %10 sc0 sc1\n\t"
                     "global_load_dwordx4 %5, %9, %10 offset:1024 sc0 sc1\n\t"
                     "global_load_dwordx4 %6, %9, %10 offset:2048 sc0 sc1\n\t"
                     "global_load_dwordx4 %7, %9, %10 offset:3072 sc0 sc1\n\t"
                     "s_waitcnt vmcnt(0)"
                     : "=&v"(pv[0]), "=&v"(pv[1]), "=&v"(pv[2]), "=&v"(pv[3]), "=&v"(pv[4]), "=&v"(pv[5]), "=&v"(pv[6]), "=&v"(pv[7])
                     : "v"(l16), "v"(l16b), "s"(pb) : "memory");
        u32 mn = 0xffffffffu;
  #pragma unroll
        for (int i = 0; i < 8; ++i)
  #pragma unroll
          for (int z = 0; z < 4; ++z) mn = min(mn, pv[i][z]);
        if (__builtin_amdgcn_ballot_w64(mn == 0u) == 0ull) break;
        if (give_up(t0, myws, call)) dead = true;
        if (RSL) __builtin_amdgcn_s_sleep(RSL);
      }
      if (dead) {
        if (lane == 0) st_agt(myws + TMO_OFF, call);
      } else {
  #pragma unroll
        for (int nt = 0; nt < 2; ++nt)
  #pragma unroll
          for (int mt = 0; mt < 2; ++mt)
  #pragma unroll
            for (int sx = 0; sx < 2; ++sx)
  #pragma unroll
              for (int z = 0; z < 4; ++z) {
                const u32 u = pv[(nt * 2 + mt) * 2 + sx][z];
                acc[nt][mt][8 * sx + 2 * z] += bf_lo(u);
                acc[nt][mt][8 * sx + 2 * z + 1] += bf_hi(u);
              }
      }
    }
  }

  STAMP(myws, 6);
  if (!fin) {
    char* dst = uni(myws + (size_t)(POUT_OFF + (u32)((j - 1) * 16 + k) * PIECE + tile_off));
    char* fl = myws + (size_t)(FLOC_OFF + (u32)((j - 1) * 8 + e * 4 + wave) * 128u + (u32)k * 4u);
    asm volatile("s_nop 4" ::: "memory");
#pragma unroll
    for (int nt = 0; nt < 2; ++nt)
#pragma unroll
      for (int mt = 0; mt < 2; ++mt) {
        u32x4 v0, v1;
#pragma unroll
        for (int z = 0; z < 4; ++z) {
          const u32 t0_ = pack2(acc[nt][mt][2 * z], acc[nt][mt][2 * z + 1]);
          const u32 t1_ = pack2(acc[nt][mt][8 + 2 * z], acc[nt][mt][8 + 2 * z + 1]);
          v0[z] = dead ? 0x7fc07fc0u : (t0_ ? t0_ : 0x8000u);
          v1[z] = dead ? 0x7fc07fc0u : (t1_ ? t1_ : 0x8000u);
        }
        const u32 o = nt ? l16b : l16;
        if (mt == 0) { st16<0>(o, v0, dst); st16<1024>(o, v1, dst); }
        else         { st16<2048>(o, v0, dst); st16<3072>(o, v1, dst); }
      }
    waitv<0>();
    if (lane == 0) st_sys(fl, call);
  } else {
    const u32 h = lane >> 5, m = lane & 31u;
    const u32 tb = sbase + (u32)wave * 8192u;
#pragma unroll
    for (int nt = 0; nt < 2; ++nt)
#pragma unroll
      for (int mt = 0; mt < 2; ++mt)
#pragma unroll
        for (int sx = 0; sx < 2; ++sx) {
          u32x4 v;
#pragma unroll
          for (int z = 0; z < 4; ++z) {
            const float lo = acc[nt][mt][8 * sx + 2 * z], hi = acc[nt][mt][8 * sx + 2 * z + 1];
            v[z] = dead ? 0x7fc07fc0u : pack2(lo < 0.f ? 0.f : lo, hi < 0.f ? 0.f : hi);
          }
          *(volatile LDSP u32x4*)(uintptr_t)(tb + ((u32)mt * 32u + m) * 128u + ((((u32)(nt * 4 + sx * 2) + h) ^ (m & 7u)) * 16u)) = v;
        }
    char* ob = uni(P.out + (size_t)((u32)(row0 - r * 512 + wm * 64) * 8192u + (u32)(col0 + wn * 64) * 2u));
    const u32 rr = lane >> 3, cc = lane & 7u;
    const u32 ra = tb + rr * 128u + ((cc ^ rr) * 16u);
    u32x4 ov[8];
#pragma unroll
    for (int i = 0; i < 8; ++i) ov[i] = *(volatile LDSP u32x4*)(uintptr_t)(ra + (u32)i * 1024u);
#pragma unroll
    for (int i = 0; i < 8; ++i) OST<0>(((u32)i * 8u + rr) * 8192u + cc * 16u, ov[i], ob);
  }
  STAMP(myws, 5);
}

__global__ __launch_bounds__(256, 2) void fused_gemm_rs(Params P) {
  extern __shared__ __attribute__((aligned(16))) char smem[];
  const u32 lane = threadIdx.x & 63u;
  const int wave = __builtin_amdgcn_readfirstlane(threadIdx.x >> 6);
  const int bid = blockIdx.x;
  if (bid < N_PUSH_WG) mover(P, wave, lane, smem);
  else if (bid >= 256 && bid < 256 + N_PUSH_WG) {
    for (int i = 0; i < 6; ++i) __builtin_amdgcn_s_sleep(16);
  } else tile(P, bid < 256 ? bid - N_PUSH_WG : bid - 2 * N_PUSH_WG, wave, lane, smem);
}

}

extern "C" void kernel_launch(int rank, int nranks, ncclComm_t comm,
                              hipStream_t stream,
                              void* const* d_in, const int* in_sizes, int n_in,
                              void* d_out, int out_size,
                              const int* in_dtypes, int out_dtype,
                              void* const* d_out_all,
                              void* const* d_ws_all, size_t ws_bytes) {
  if (nranks != NR || n_in < 2 || ws_bytes < (size_t)WS_NEED || (unsigned)rank >= (unsigned)NR) {
    fprintf(stderr, "unsupported configuration\n");
    return;
  }
  Params P;
  P.x = (const char*)d_in[0];
  P.w = (const char*)d_in[1];
  P.out = (char*)d_out;
  for (int i = 0; i < NR; ++i) P.ws[i] = (char*)d_ws_all[i];
  P.rank = rank;
  HIP_CHECK(hipFuncSetAttribute((const void*)fused_gemm_rs, hipFuncAttributeMaxDynamicSharedMemorySize, (int)LDS_BYTES));
  hipLaunchKernelGGL(fused_gemm_rs, dim3(N_WG), dim3(256), LDS_BYTES, stream, P);
  HIP_CHECK(hipGetLastError());
}
```

```cpp
#include <hip/hip_runtime.h>
#include <rccl/rccl.h>
#include <cstdio>
#include <cstdint>

#define HIP_CHECK(expr) do { hipError_t _e = (expr); if (_e != hipSuccess) { \
    fprintf(stderr, "HIP error %s at %s:%d\n", hipGetErrorString(_e), __FILE__, __LINE__); \
    return; } } while (0)

namespace {

typedef unsigned u32;
typedef unsigned long long u64;
typedef u32 u32x4 __attribute__((ext_vector_type(4)));
typedef float f32x2 __attribute__((ext_vector_type(2)));
typedef float f32x16 __attribute__((ext_vector_type(16)));
typedef __bf16 bf16x2 __attribute__((ext_vector_type(2)));
typedef __bf16 bf16x8 __attribute__((ext_vector_type(8)));
typedef short s16x4 __attribute__((ext_vector_type(4)));
typedef short s16x8 __attribute__((ext_vector_type(8)));
#define LDSP __attribute__((address_space(3)))

constexpr int NR = 8;
constexpr u32 PIECE = 65536u;
constexpr u32 MIB = 1u << 20;
#define NS 2
#define OX 14
#define TAU 115
#define T0 50
#define EARLY 100
#define TAU1 110
#define TAU2 0
#define STGD 2
#define PSL 4
#define LSL 0
#define WSL 3
#define RSL 0
#define WOWN 0
#define ALN 1
#define ALO 99
#define ALT 40
#define PW 1
#define PWN 5
#define PWS 0
#define PWD 0
#define G1 0
#define LAG 1
#define WIN 3
#define GLP "sc0 sc1"
#define MLP "sc0 sc1"
constexpr int NIN = 30;
constexpr int NOUT = 16 - NS;
constexpr int NOUT1 = 16 + 6 * NS;
constexpr u32 XS_OFF = 0;
constexpr u32 XS_PAR = 6u * MIB;
constexpr u32 WS_OFF = 12u * MIB;
constexpr u32 PIN_OFF = 24u * MIB;
constexpr u32 PIN_PAR = 7u * MIB;
constexpr u32 POUT_OFF = 45u * MIB;
constexpr u32 FLG_OFF = 52u * MIB;
constexpr u32 FIN_OFF = FLG_OFF;
constexpr u32 FLOC_OFF = FLG_OFF + 16384u;
constexpr u32 TMO_OFF = FLG_OFF + 49152u;
constexpr u32 DUMMY_OFF = FLG_OFF + 65536u;
constexpr u32 DBG_OFF = FLG_OFF + 131072u;
constexpr u32 CNT_OFF = FLG_OFF + MIB + 262144u;
constexpr u32 PRB_OFF = 0;
constexpr u32 WS_NEED = FLG_OFF + 2u * MIB;

constexpr int N_PUSH_WG = 56;
constexpr int N_MAIN_WG = 256;
constexpr int N_FIN_WG = 96;
constexpr int N_WG = 2 * N_PUSH_WG + N_MAIN_WG + N_FIN_WG;
constexpr u32 STAGE_B = 16384u;
constexpr u32 LDS_BYTES = 4u * STAGE_B + 1024u;
constexpr u64 DEADLINE = 100000000ull;
constexpr u32 DEAD = 0x7fffffffu;

struct Params {
  const char* x;
  const char* w;
  char* out;
  char* ws[NR];
  int rank;
};

#define SB() __builtin_amdgcn_sched_barrier(0)
template <int N> __device__ __forceinline__ void waitv() { asm volatile("s_waitcnt vmcnt(%0)" :: "i"(N) : "memory"); }

__device__ __forceinline__ u64 now() { return __builtin_amdgcn_s_memrealtime(); }
template <typename T> __device__ __forceinline__ T* uni(T* p) {
  const u64 v = (u64)(uintptr_t)p;
  const u32 lo = __builtin_amdgcn_readfirstlane((u32)v), hi = __builtin_amdgcn_readfirstlane((u32)(v >> 32));
  return (T*)(uintptr_t)(((u64)hi << 32) | lo);
}
__device__ __forceinline__ u32 ld_sys(const void* p) {
  return __hip_atomic_load((const u32*)p, __ATOMIC_RELAXED, __HIP_MEMORY_SCOPE_SYSTEM);
}
__device__ __forceinline__ void st_sys(void* p, u32 v) {
  __hip_atomic_store((u32*)p, v, __ATOMIC_RELAXED, __HIP_MEMORY_SCOPE_SYSTEM);
}
__device__ __forceinline__ void st_agt(void* p, u32 v) {
  __hip_atomic_store((u32*)p, v, __ATOMIC_RELAXED, __HIP_MEMORY_SCOPE_AGENT);
}
__device__ __forceinline__ bool give_up(u64 t0, const char* ws, u32 call) {
  const u64 dt = now() - t0;
  if (!__builtin_amdgcn_readfirstlane((int)(dt > 50000ull))) return false;
  __builtin_amdgcn_s_sleep(64);
  if (__builtin_amdgcn_readfirstlane((int)(dt > DEADLINE))) return true;
  return __builtin_amdgcn_readfirstlane(ld_sys(ws + TMO_OFF)) == call;
}
__device__ __forceinline__ void sleep_until(u64 tstart, u32 t) {
  for (;;) {
    const u32 el = (u32)__builtin_amdgcn_readfirstlane((u32)(now() - tstart));
    if (el >= t) break;
    if (t - el > 40u) __builtin_amdgcn_s_sleep(12); else __builtin_amdgcn_s_sleep(1);
  }
}
template <int IMM> __device__ __forceinline__ void st16(u32 off, u32x4 v, char* base) {
  asm volatile("global_store_dwordx4 %0, %1, %2 offset:%3 sc0 sc1\n\ts_nop 1" :: "v"(off), "v"(v), "s"(base), "i"(IMM) : "memory");
}
template <int IMM> __device__ __forceinline__ void st16p(u32 off, u32x4 v, char* base) {
  asm volatile("global_store_dwordx4 %0, %1, %2 offset:%3\n\ts_nop 1" :: "v"(off), "v"(v), "s"(base), "i"(IMM) : "memory");
}
#define OST st16
__device__ __forceinline__ void dma16x4(u32 lds, const u32 (&o)[4], const char* base) {
  asm volatile("s_mov_b32 m0, %0\n\ts_nop 3\n\t"
               "global_load_lds_dwordx4 %1, %5 " MLP "\n\t"
               "global_load_lds_dwordx4 %2, %5 offset:1024 " MLP "\n\t"
               "global_load_lds_dwordx4 %3, %5 offset:2048 " MLP "\n\t"
               "global_load_lds_dwordx4 %4, %5 offset:3072 " MLP
               :: "s"(lds), "v"(o[0]), "v"(o[1]), "v"(o[2]), "v"(o[3]), "s"(base) : "memory");
}
__device__ __forceinline__ void probe(u32 off, char* base) {
  asm volatile("s_nop 4\n\tglobal_atomic_add %0, %1, %2 sc1" :: "v"(off), "v"(0u), "s"(base) : "memory");
}
__device__ __forceinline__ u32 atom_rd(u32 off, const char* base) {
  u32 r;
  asm volatile("global_atomic_add %0, %1, %2, %3 sc0 sc1\n\ts_waitcnt vmcnt(0)" : "=&v"(r) : "v"(off), "v"(0u), "s"(base) : "memory");
  return r;
}
__device__ __forceinline__ u32 bump_issue(u32 off, const char* base) {
  u32 r;
  asm volatile("s_mov_b64 exec, 1\n\ts_nop 3\n\tglobal_atomic_add %0, %1, %2, %3 sc0 sc1\n\ts_mov_b64 exec, -1"
               : "=&v"(r) : "v"(off), "v"(1u), "s"(base) : "memory");
  return r;
}
template <int N> __device__ __forceinline__ u32 bump_get(u32 r) {
  u32 v;
  asm volatile("s_waitcnt vmcnt(%2)\n\tv_readfirstlane_b32 %0, %1\n\ts_nop 4" : "=s"(v) : "v"(r), "i"(N) : "memory");
  return v + 1u;
}
__device__ __forceinline__ void add1_lane0(u32 off, char* base) {
  asm volatile("s_mov_b64 exec, 1\n\ts_nop 3\n\tglobal_atomic_add %0, %1, %2 sc1\n\ts_mov_b64 exec, -1"
               :: "v"(off), "v"(1u), "s"(base) : "memory");
}
__device__ __forceinline__ void st4_lane0(u32 off, u32 val, char* base) {
  asm volatile("s_mov_b64 exec, 1\n\ts_nop 3\n\tglobal_store_dword %0, %1, %2 sc0 sc1\n\ts_mov_b64 exec, -1"
               :: "v"(off), "v"(val), "s"(base) : "memory");
}

#define STAMP(ws, idx) do {} while (0)

__device__ __forceinline__ void piece_of(int q, int d, bool& isx, int& idx) {
  const int t = q - NS, u = q - NS - 12;
  int j = (t >> 1) + 2;
  j += j >= d ? 1 : 0;
  j = j == 8 ? 1 : j;
  isx = (q >= NS && u < 0) || (u >= OX && u < OX + 2);
  idx = q < NS ? q : (u < 0 ? 2 * j + (t & 1) : (u < OX ? NS + u : (u < OX + 2 ? u - OX : NS + u - 2)));
}
__device__ __forceinline__ int slot_w(int k) { return k < NS ? k : (k - NS < OX ? 12 + k : 14 + k); }
__device__ __forceinline__ int slot_x(int j, int e, int d) {
  return j == 0 ? NS + 12 + OX + e : NS + 2 * (j == 1 ? 5 : j - 2 - (j > d ? 1 : 0)) + e;
}
__device__ __forceinline__ u32 lds_rd(u32 a) { return __builtin_amdgcn_readfirstlane(*(volatile LDSP u32*)(uintptr_t)a); }
__device__ __forceinline__ void lds_wr(u32 a, u32 v) { *(volatile LDSP u32*)(uintptr_t)a = v; }

__device__ __forceinline__ void mover(const Params& P, int wave, u32 lane, char* smem) {
  __builtin_amdgcn_s_setprio(3);
  const u64 tstart = now();
  const int s = P.rank;
  char* const myws = P.ws[s];
  const int pair = wave >> 1;
  const int d = 1 + ((int)(blockIdx.x >> 3) + 3 * pair) % 7;
  const int p = (blockIdx.x & 7) * 2 + pair;
  const u32 r = (u32)(s ^ d) & (u32)(NR - 1);
  const int nin = d == 1 ? 0 : NIN;
  const int np = d == 1 ? NOUT1 : NIN + NOUT;
  const u32 ring = (u32)(uintptr_t)smem + (u32)pair * 32768u;
  const u32 ctl = (u32)(uintptr_t)smem + 65536u + (u32)pair * 512u;
  const u32 cnt = bump_issue(CNT_OFF + (u32)(blockIdx.x * 4 + wave) * 64u, myws);

  if ((wave & 1) == 0) {
    const char* const xin = P.x;
    const char* const win = P.w;
    const int rg = r >> 2, rb_ = (r >> 1) & 1, ra = r & 1;
    u32 xo[4], wo[4], oo[4];
    {
      const u32 xc = (lane & 3u) ^ ((lane >> 4) & 3u), q4 = lane >> 4;
#pragma unroll
      for (int k = 0; k < 4; ++k) {
        xo[k] = ((u32)(p & 1) * 64u + (u32)k * 16u + (lane >> 2)) * 1024u + (u32)(p >> 1) * 64u + xc * 16u - (u32)k * 1024u;
        wo[k] = ((u32)(p >> 1) * 32u + (u32)(p & 1) * 16u + (u32)k * 4u + q4) * 8192u
              + (((lane & 15u) ^ ((q4 << 2) | (u32)k)) * 16u) - (u32)k * 1024u;
        oo[k] = (u32)p * 4096u + lane * 16u;
      }
    }
    u32 poll_off;
    {
      const int i = (int)lane < np - nin ? (int)lane : 0;
      const int jj = d != 1 ? d : (i < 6 * NS ? 2 + i / NS : 1);
      const int kk = d != 1 ? NS + i : (i < 6 * NS ? i % NS : i - 6 * NS);
      poll_off = FLOC_OFF + (u32)((jj - 1) * 8 + (p >> 1)) * 128u + (u32)kk * 4u;
    }
#define ISSUE_LOADS(QQ, II) do { \
      const int q_ = (QQ); \
      const u32 dst_ = ring + (u32)(q_ & 7) * 4096u; \
      if (q_ < nin) { \
        bool isx_; int ix_; \
        piece_of(q_, d, isx_, ix_); \
        if (isx_) dma16x4(dst_, xo, xin + (size_t)(((r ^ (u32)(ix_ >> 1)) * 512u + (u32)rb_ * 256u + (u32)(ix_ & 1) * 128u) * 1024u + (u32)ra * 512u)); \
        else      dma16x4(dst_, wo, win + (size_t)((u32)ra * 256u * 8192u + (u32)rg * 4096u + (u32)(ra ? 15 - ix_ : ix_) * 256u)); \
      } else { \
        const int i_ = (II); \
        const int jj_ = d != 1 ? d : (i_ < 6 * NS ? 2 + i_ / NS : 1); \
        const int kk_ = d != 1 ? NS + i_ : (i_ < 6 * NS ? i_ % NS : i_ - 6 * NS); \
        dma16x4(dst_, oo, myws + (size_t)(POUT_OFF + (u32)((jj_ - 1) * 16 + kk_) * PIECE)); \
      } \
    } while (0)

    int q0 = 0;
    u32 pub = 0, call;
    if (nin) {
      ISSUE_LOADS(0, 0); ISSUE_LOADS(1, 0); ISSUE_LOADS(2, 0);
    }
    lds_wr(ctl, 0u);
    asm volatile("s_waitcnt lgkmcnt(0)\n\ts_barrier" ::: "memory");
    if (nin) {
      call = bump_get<8>(cnt);
      lds_wr(ctl, 1u);
      q0 = 3; pub = 1;
    } else {
      call = bump_get<0>(cnt);
    }
    STAMP(myws, 0);

    if (!nin) sleep_until(tstart, (u32)(T0 + (NS + 2) * TAU + 500));
    u64 ready = 0;
    u64 todo = (1ull << (np - nin)) - 1ull;
    u32 cons = 0;
    bool dead = false;
    u32 dbg1 = 0, dbg2 = 0;
    for (int q = q0; q < np; ++q) {
      bool rdy = q < nin || (ready & todo) != 0ull;
      if (cons + 8u <= (u32)q) cons = lds_rd(ctl + 4u);
      if (!rdy || cons + 8u <= (u32)q) {
        waitv<0>();
        pub = (u32)q;
        lds_wr(ctl, pub);
        const u64 t0 = now();
        for (;;) {
          if (!rdy) {
            u32 pv = 0;
            if ((todo >> lane) & 1ull) pv = atom_rd(poll_off, myws);
            ready |= __builtin_amdgcn_ballot_w64(pv >= call);
            rdy = (ready & todo) != 0ull;
            if (LSL && !rdy) __builtin_amdgcn_s_sleep(LSL);
          } else {
            __builtin_amdgcn_s_sleep(2);
          }
          cons = lds_rd(ctl + 4u);
          if (rdy && cons + 8u > (u32)q) break;
          if (give_up(t0, myws, call)) { dead = true; break; }
        }
        if (dead) break;
      }
      int id = 0;
      if (q >= nin) {
        id = __builtin_ctzll(ready & todo);
        todo &= ~(1ull << id);
        lds_wr(ctl + 32u + (u32)(q & 7) * 4u, (u32)id);
      }
      ISSUE_LOADS(q, id);
      waitv<8>();
      if (q - 1 > (int)pub) { pub = (u32)(q - 1); lds_wr(ctl, pub); }
      if (q == nin - 1) STAMP(myws, 1);
    }
    waitv<0>();
    if (dead) st4_lane0(TMO_OFF, call, myws);
    lds_wr(ctl, dead ? DEAD : (u32)np);
    STAMP(myws, 3);
    return;
  }

  char* const rws = uni(P.ws[r]);
  if (PWS) __builtin_amdgcn_s_sleep(PWS);
  if ((PW & 1) && lane < (u32)PWN) probe(lane * (FLG_OFF / (PWN - 1) & ~4095u) + 32768u + (u32)(d * 16 + p) * 128u, rws);
  lds_wr(ctl + 4u, 0u);
  asm volatile("s_waitcnt lgkmcnt(0)\n\ts_barrier" ::: "memory");
  const u32 call = bump_get<(PW & 1)>(cnt);
  STAMP(myws, 0);
  const u32 par = call & 1u;
  const int slot = d - 2;
  const u32 dl = (u32)p * 4096u + lane * 16u;
  const u32 pin0 = PIN_OFF + (call % 3u) * PIN_PAR + (u32)(d == 1 ? 0 : NOUT1 + slot * NOUT) * PIECE;
  int nflag = 0;
#define FLAG(QQ) do { \
    const int q_ = (QQ); \
    const bool real_ = q_ == nflag && q_ < nin; \
    const u32 o_ = real_ ? FIN_OFF + (u32)(slot * 32 + q_) * 64u : DUMMY_OFF + (u32)(blockIdx.x * 4 + wave) * 128u; \
    add1_lane0(o_, uni(real_ ? rws : myws)); \
    nflag += real_ ? 1 : 0; \
  } while (0)

  const u32 toff = (u32)TAU - 1u - ((u32)__builtin_amdgcn_readfirstlane((u32)tstart) + (u32)(T0 + TAU - 1) - ((u32)(((p + STGD * d) & 15) * TAU) >> 4)) % (u32)TAU;
  u32 full = 0, tnext = 0, tgrid = (u32)T0 + toff;
  u32 dbg1 = 0, dbg2 = 0;
  for (int q = 0; q < np; ++q) {
    if (d != 1 || G1) tnext = max(tnext, tgrid);
    sleep_until(tstart, tnext);
    if (full <= (u32)q) {
      full = lds_rd(ctl);
      if (full <= (u32)q) {
        const u64 t0 = now();
        bool drained = !(q > 0 && q < nin + LAG);
        for (;;) {
          __builtin_amdgcn_s_sleep(1);
          full = lds_rd(ctl);
          if (full > (u32)q) break;
          const u64 dt = now() - t0;
          if (!drained && __builtin_amdgcn_readfirstlane((int)(dt > 30ull))) {
            drained = true;
            waitv<0>();
#pragma unroll
            for (int z = LAG; z >= 1; --z) FLAG(q - z);
          }
          if (__builtin_amdgcn_readfirstlane((int)(dt > 2ull * DEADLINE - 1000ull))) { full = DEAD; break; }
        }
        tnext = (u32)__builtin_amdgcn_readfirstlane((u32)(now() - tstart));
      }
      if (full == DEAD) break;
    }
    if ((d != 1 || G1) && (q < ALN || q >= nin + ALO)) {
      const u32 el = (u32)__builtin_amdgcn_readfirstlane((u32)(now() - tstart));
      if (el > tgrid + (u32)ALT) {
        tgrid += (el - tgrid - (u32)ALT + (u32)TAU - 1u) / (u32)TAU * (u32)TAU;
        sleep_until(tstart, tgrid);
        tnext = tgrid;
      }
    }
    tgrid += (u32)TAU;
    tnext += (u32)(d == 1 && !G1 ? TAU1 : TAU2);
    const u32 la = ring + (u32)(q & 7) * 4096u + lane * 16u;
    const u32x4 d0 = *(volatile LDSP u32x4*)(uintptr_t)la;
    const u32x4 d1 = *(volatile LDSP u32x4*)(uintptr_t)(la + 1024u);
    const u32x4 d2 = *(volatile LDSP u32x4*)(uintptr_t)(la + 2048u);
    const u32x4 d3 = *(volatile LDSP u32x4*)(uintptr_t)(la + 3072u);
    const u32 id = lds_rd(ctl + 32u + (u32)(q & 7) * 4u);
    lds_wr(ctl + 4u, (u32)q + 1u);
    u32 o;
    if (q < nin) {
      bool isx; int ix;
      piece_of(q, d, isx, ix);
      o = (isx ? XS_OFF : WS_OFF) + par * XS_PAR + (u32)(ix * 6 + slot) * PIECE;
    } else {
      o = pin0 + min(id, (u32)(np - nin - 1)) * PIECE;
    }
    char* const dst = rws + (size_t)o;
    st16<0>(dl, d0, dst);
    st16<1024>(dl, d1, dst);
    st16<2048>(dl, d2, dst);
    st16<3072>(dl, d3, dst);
    if (nin && q < nin + LAG) {
      waitv<5 * LAG>();
      FLAG(q - LAG);
    } else {
      waitv<4 * WIN>();
    }
    if (q == NS + 11) STAMP(myws, 1);
    if (q == nin - 1) STAMP(myws, 2);
    if (q == np - 1) STAMP(myws, 4);
  }
  waitv<0>();
  STAMP(myws, 3);
}

struct Frag {
  bf16x8 x[2][2];
  bf16x8 w[2][2];
};

struct Loader {
  const char* base;
  u32 stride;
  u32 v[4];
  u32 m0base;
};

template <int STAGE> __device__ __forceinline__ void glds_pair(const Loader& L, int half) {
  const u32 m0v = L.m0base + STAGE * STAGE_B;
  if (half == 0)
    asm volatile("s_mov_b32 m0, %0\n\ts_nop 0\n\t"
                 "global_load_lds_dwordx4 %1, %3 " GLP "\n\t"
                 "global_load_lds_dwordx4 %2, %3 offset:1024 " GLP
                 :: "s"(m0v), "v"(L.v[0]), "v"(L.v[1]), "s"(L.base) : "memory");
  else
    asm volatile("s_mov_b32 m0, %0\n\ts_nop 0\n\t"
                 "global_load_lds_dwordx4 %1, %3 offset:2048 " GLP "\n\t"
                 "global_load_lds_dwordx4 %2, %3 offset:3072 " GLP
                 :: "s"(m0v), "v"(L.v[2]), "v"(L.v[3]), "s"(L.base) : "memory");
}

struct RdAddr { u32 xa[2]; u32 wa[2][2]; };

template <int STAGE> __device__ __forceinline__ bf16x8 rd_x(const RdAddr& A, int mt, int ks) {
  return *(const LDSP bf16x8*)(uintptr_t)(A.xa[ks] + (u32)(STAGE * STAGE_B + mt * 2048));
}
template <int STAGE> __device__ __forceinline__ bf16x8 rd_w(const RdAddr& A, int nt, int ks) {
  s16x4 lo = __builtin_amdgcn_ds_read_tr16_b64_v4i16((LDSP s16x4*)(uintptr_t)(A.wa[nt][0] + (u32)(STAGE * STAGE_B + 8192 + ks * 4096)));
  s16x4 hi = __builtin_amdgcn_ds_read_tr16_b64_v4i16((LDSP s16x4*)(uintptr_t)(A.wa[nt][1] + (u32)(STAGE * STAGE_B + 8192 + ks * 4096)));
  s16x8 v = __builtin_shufflevector(lo, hi, 0, 1, 2, 3, 4, 5, 6, 7);
  return __builtin_bit_cast(bf16x8, v);
}
#define MFMA(nt, mt, ks) acc[nt][mt] = __builtin_amdgcn_mfma_f32_32x32x16_bf16(cur.w[nt][ks], cur.x[mt][ks], acc[nt][mt], 0, 0, 0)

template <int S, int VM, bool LOAD, bool READ>
__device__ __forceinline__ void kstep(f32x16 (&acc)[2][2], const Frag& cur, Frag& nxt, const RdAddr& A, Loader& L) {
  constexpr int SN = (S + 1) & 3;
  if (READ) {
    __builtin_amdgcn_s_waitcnt((VM & 15) | (7 << 4) | (0 << 8) | ((VM >> 4) << 14));
    asm volatile("s_barrier" ::: "memory");
  } else {
    __builtin_amdgcn_s_waitcnt(15 | (7 << 4) | (0 << 8) | (3 << 14));
  }
  SB();
  MFMA(0, 0, 0); SB();
  if (READ) { nxt.w[0][0] = rd_w<SN>(A, 0, 0); nxt.x[0][0] = rd_x<SN>(A, 0, 0); } SB();
  MFMA(0, 1, 0); SB();
  if (READ) { nxt.x[1][0] = rd_x<SN>(A, 1, 0); nxt.w[1][0] = rd_w<SN>(A, 1, 0); } SB();
  MFMA(1, 0, 0); SB();
  if (LOAD) glds_pair<S>(L, 0);
  SB();
  MFMA(1, 1, 0); SB();
  if (READ) { nxt.w[0][1] = rd_w<SN>(A, 0, 1); nxt.x[0][1] = rd_x<SN>(A, 0, 1); } SB();
  MFMA(0, 0, 1); SB();
  if (READ) { nxt.x[1][1] = rd_x<SN>(A, 1, 1); nxt.w[1][1] = rd_w<SN>(A, 1, 1); } SB();
  MFMA(0, 1, 1); SB();
  if (LOAD) { glds_pair<S>(L, 1); L.base += L.stride; }
  SB();
  MFMA(1, 0, 1); SB();
  MFMA(1, 1, 1); SB();
}

template <int S> __device__ __forceinline__ void stage_load(Loader& L) {
  glds_pair<S>(L, 0); glds_pair<S>(L, 1); L.base += L.stride;
}

__device__ __forceinline__ void gemm_phase(f32x16 (&acc)[2][2], const RdAddr& A, Loader& L, int n, int sk, u32& dbgv, int dbg0) {
  Frag f0, f1;
  L.base = uni(L.base + (sk == 0 ? 65536u : 0u));
  L.stride = __builtin_amdgcn_readfirstlane(L.stride);
  asm volatile("s_nop 4" ::: "memory");
  stage_load<0>(L); stage_load<1>(L); stage_load<2>(L); stage_load<3>(L);
  waitv<12>();
  asm volatile("s_barrier" ::: "memory");
#pragma unroll
  for (int ks = 0; ks < 2; ++ks)
#pragma unroll
    for (int i = 0; i < 2; ++i) { f0.w[i][ks] = rd_w<0>(A, i, ks); f0.x[i][ks] = rd_x<0>(A, i, ks); }
  SB();
  for (int t = 0; t < n - 4; t += 4) {
    if (t + 4 == 8 * sk) L.base += 65536;
    kstep<0, 8, true, true>(acc, f0, f1, A, L);
    kstep<1, 8, true, true>(acc, f1, f0, A, L);
    kstep<2, 8, true, true>(acc, f0, f1, A, L);
    kstep<3, 8, true, true>(acc, f1, f0, A, L);
  }
  kstep<0, 8, false, true>(acc, f0, f1, A, L);
  kstep<1, 4, false, true>(acc, f1, f0, A, L);
  kstep<2, 0, false, true>(acc, f0, f1, A, L);
  kstep<3, 0, false, false>(acc, f1, f0, A, L);
}

__device__ __forceinline__ u32 pack2(float a, float b) {
  f32x2 t = {a, b};
  return __builtin_bit_cast(u32, __builtin_convertvector(t, bf16x2));
}
__device__ __forceinline__ float bf_lo(u32 v) { return __builtin_bit_cast(float, v << 16); }
__device__ __forceinline__ float bf_hi(u32 v) { return __builtin_bit_cast(float, v & 0xffff0000u); }

__device__ __forceinline__ void zero8k(u32 l16, u32 l16b, char* pb) {
  const u32x4 z4 = {0u, 0u, 0u, 0u};
  asm volatile("s_nop 4" ::: "memory");
  st16<0>(l16, z4, pb); st16<1024>(l16, z4, pb); st16<2048>(l16, z4, pb); st16<3072>(l16, z4, pb);
  st16<0>(l16b, z4, pb); st16<1024>(l16b, z4, pb); st16<2048>(l16b, z4, pb); st16<3072>(l16b, z4, pb);
}

__device__ __forceinline__ void tile(const Params& P, int tid, int wave, u32 lane, char* smem) {
  const int r = P.rank;
  char* const myws = P.ws[r];
  const int g = r >> 2, b = (r >> 1) & 1, a = r & 1;
  const int bid = blockIdx.x;
  const u64 tstart = now();

  const u32 cnt = bump_issue(CNT_OFF + (u32)(bid * 4 + wave) * 64u, myws);
  STAMP(myws, 0);

  int row0, col0, nrem, tskip, j, e, k, rb;
  int pi0 = -1, pi1 = -1;
  bool fin;
  if (tid < N_MAIN_WG) {
    const int xcd = tid & 7, o = tid >> 3, u = xcd >> 1;
    int rbi, cbi;
    if (u < 2) { rbi = o < 24 ? 1 + o % 3 : 0; cbi = o < 24 ? o / 3 : o - 24; }
    else { rbi = o & 3; cbi = o >> 2; }
    rb = u + 4 * rbi; k = cbi * 2 + (xcd & 1);
    j = rb >> 1; e = rb & 1;
    row0 = (r ^ j) * 512 + b * 256 + e * 128; col0 = g * 2048 + (a ? 15 - k : k) * 128;
    nrem = j >= 2 ? 40 : 48; tskip = j >= 2 ? j - 2 : 99;
    fin = j == 0;
    if (j == 0) pi0 = 6 * NS + 15 - k;
    else if (j >= 2 && k >= 16 - NS) pi0 = ((j ^ 1) - 2) * NS + 15 - k;
  } else {
    const int f = tid - N_MAIN_WG;
    const int qd = 1 + f / 32, rem = f & 31;
    e = (rem >> 3) & 1; k = (rem & 7) * 2 + (rem >> 4);
    const int h = b ^ (qd & 1), g2 = g ^ (qd >> 1);
    row0 = r * 512 + h * 256 + e * 128; col0 = g2 * 2048 + (a ? 15 - k : k) * 128;
    nrem = 0; tskip = 99; j = 0; rb = 0;
    fin = true;
    const int iA = k >= NS ? NOUT1 + (2 * qd - 2) * NOUT + k - NS : -1;
    const int iB = 15 - k >= NS ? NOUT1 + (2 * qd - 1) * NOUT + 15 - k - NS : -1;
    if (k < 8) { pi0 = iA >= 0 ? iA : iB; pi1 = iA >= 0 ? iB : -1; }
    else       { pi0 = iB >= 0 ? iB : iA; pi1 = iB >= 0 ? iA : -1; }
  }

  const u32 sbase = (u32)(uintptr_t)smem;
  const int wm = wave >> 1, wn = wave & 1;
  RdAddr A;
  {
    const u32 h = lane >> 5, m = lane & 31u, f = (lane >> 2) & 3u;
    A.xa[0] = sbase + ((u32)wm * 64u + m) * 64u + 16u * (h ^ f);
    A.xa[1] = sbase + ((u32)wm * 64u + m) * 64u + 16u * ((2u + h) ^ f);
    const u32 blk = (lane >> 4) & 1u, q = (lane & 15u) >> 2, pp = lane & 3u;
#pragma unroll
    for (int nt = 0; nt < 2; ++nt)
#pragma unroll
      for (int t = 0; t < 2; ++t) {
        const u32 row = 8u * h + 4u * t + q;
        const u32 ch = (u32)wn * 8u + (u32)nt * 4u + 2u * blk + (pp & 1u);
        A.wa[nt][t] = sbase + 256u * row + 16u * (ch ^ (((row & 3u) << 2) | ((row >> 2) & 3u))) + 8u * (pp >> 1);
      }
  }

  f32x16 acc[2][2];
#pragma unroll
  for (int i = 0; i < 2; ++i)
#pragma unroll
    for (int k = 0; k < 2; ++k)
#pragma unroll
      for (int z = 0; z < 16; ++z) acc[i][k][z] = 0.f;

  const u32 tile_off = (u32)e * 32768u + (u32)wave * 8192u;
  const u32 l16 = lane * 16u, l16b = lane * 16u + 4096u;
  const int nph = nrem ? 2 : 1;
  const u32 tin = (u32)(T0 + EARLY + TAU * max(slot_w(k), slot_x(j, e, j == 7 ? 2 : 7)));
  const int addph = fin ? nph - 1 : 0;

  Loader L;
  L.m0base = sbase + (u32)wave * 4096u;
  bool dead = false;
  u32 dbgv = 0;
  u32 call = 0;
  for (int ph = 0; ph < nph; ++ph) {
    int n, ts;
    if (ph == 0) {
      n = 16; ts = 99;
      if (wave < 2) {
        L.base = P.x + (size_t)((u32)(row0 + wave * 64) * 1024u);
        L.stride = 64u;
        const u32 c = (lane & 3u) ^ ((lane >> 4) & 3u);
#pragma unroll
        for (int k = 0; k < 4; ++k) L.v[k] = (lane >> 2) * 1024u + c * 16u + (u32)k * (16384u - 1024u);
      } else {
        L.base = P.w + (size_t)((u32)(wave - 2) * 16u * 8192u + (u32)col0 * 2u);
        L.stride = 32u * 8192u;
        const u32 q = lane >> 4;
#pragma unroll
        for (int k = 0; k < 4; ++k) L.v[k] = q * 8192u + (((lane & 15u) ^ ((q << 2) | (u32)k)) * 16u) + (u32)k * (4u * 8192u - 1024u);
      }
    } else {
      n = nrem; ts = tskip;
      STAMP(myws, 1);
      if (wave == 0) {
        const int sl = (int)min(lane, 5u);
        const u32 fo = FIN_OFF + (u32)(sl * 32 + max(slot_w(k), slot_x(j, e, sl + 2))) * 64u;
        bool pend = lane < 6u && sl + 2 != j;
        sleep_until(tstart, tin);
        const u64 t0 = now();
        u32 st = 0;
        for (;;) {
          if (pend) pend = atom_rd(fo, myws) < 16u * call;
          if (__builtin_amdgcn_ballot_w64(pend) == 0ull) break;
          if (give_up(t0, myws, call)) {
            if (lane == 0) st_agt(myws + TMO_OFF, call);
            st = 1;
            break;
          }
          __builtin_amdgcn_s_sleep(PSL);
        }
        if (lane == 0) *(LDSP u32*)(uintptr_t)(sbase + 4u * STAGE_B) = st;
        __builtin_amdgcn_s_waitcnt(0);
      }
      asm volatile("s_barrier" ::: "memory");
      const bool wgdead = lds_rd(sbase + 4u * STAGE_B) != 0;
      STAMP(myws, 2);
      if (wgdead) { dead = true; break; }
      const u32 par = call & 1u;
      if (wave < 2) L.base = myws + (size_t)(XS_OFF + par * XS_PAR + (u32)rb * 6u * PIECE + (u32)wave * 4096u);
      else          L.base = myws + (size_t)(WS_OFF + par * XS_PAR + (u32)k * 6u * PIECE + (u32)(wave - 2) * 4096u);
      L.stride = 8192u;
#pragma unroll
      for (int k = 0; k < 4; ++k) L.v[k] = lane * 16u;
    }
    gemm_phase(acc, A, L, n, ts, dbgv, ph * 8);
    if (ph == nph - 1) STAMP(myws, 3);
    if (ph == 0) {
      call = bump_get<0>(cnt);
      if (pi0 >= 0) {
        char* const pz = myws + (size_t)(PIN_OFF + ((call + 2u) % 3u) * PIN_PAR + tile_off);
        sleep_until(tstart, 900u + (u32)((tid >> 3) & 15) * 100u);
        zero8k(l16, l16b, uni(pz + (size_t)((u32)pi0 * PIECE)));
        if (pi1 >= 0) zero8k(l16, l16b, uni(pz + (size_t)((u32)pi1 * PIECE)));
      }
    }
    const u32 pin = PIN_OFF + (call % 3u) * PIN_PAR + tile_off;
    for (int s = 0; s < 2; ++s) {
      const int pi = s == 0 ? pi0 : pi1;
      if (pi < 0 || ph != addph) continue;
      char* const pb = uni(myws + (size_t)(pin + (u32)pi * PIECE));
      const bool first = pi == 0 || (pi >= NOUT1 && (pi - NOUT1) % NOUT == 0);
      const u32 wo = pin + (u32)(pi - ((!WOWN && (lane & 2u) && !first) ? 1 : 0)) * PIECE + (lane & 1u) * 4096u + 4092u;
      sleep_until(tstart, pi >= NOUT1 ? (u32)(T0 + EARLY + TAU * (NIN - 2 + (pi - NOUT1) % NOUT)) : (fin ? 0u : tin - 300u));
      const u64 t0 = now();
      while (!dead) {
        u32 v = WOWN ? 1u : 0u;
        if (lane < (WOWN ? 2u : 4u)) v = atom_rd(wo, myws);
        if (WOWN ? __builtin_amdgcn_ballot_w64(v == 0u) == 0ull : __builtin_amdgcn_ballot_w64(v != 0u) != 0ull) break;
        if (give_up(t0, myws, call)) { dead = true; break; }
        __builtin_amdgcn_s_sleep(WSL);
      }
      if (s == 1 || pi1 < 0) STAMP(myws, 4);
      u32x4 pv[8];
      while (!dead) {
        asm volatile("s_nop 4\n\t"
                     "global_load_dwordx4 %0, %8, %10 sc0 sc1\n\t"
                     "global_load_dwordx4 %1, %8, %10 offset:1024 sc0 sc1\n\t"
                     "global_load_dwordx4 %2, %8, %10 offset:2048 sc0 sc1\n\t"
                     "global_load_dwordx4 %3, %8, %10 offset:3072 sc0 sc1\n\t"
                     "global_load_dwordx4 %4, %9, %10 sc0 sc1\n\t"
                     "global_load_dwordx4 %5, %9, %10 offset:1024 sc0 sc1\n\t"
                     "global_load_dwordx4 %6, %9, %10 offset:2048 sc0 sc1\n\t"
                     "global_load_dwordx4 %7, %9, %10 offset:3072 sc0 sc1\n\t"
                     "s_waitcnt vmcnt(0)"
                     : "=&v"(pv[0]), "=&v"(pv[1]), "=&v"(pv[2]), "=&v"(pv[3]), "=&v"(pv[4]), "=&v"(pv[5]), "=&v"(pv[6]), "=&v"(pv[7])
                     : "v"(l16), "v"(l16b), "s"(pb) : "memory");
        u32 mn = 0xffffffffu;
  #pragma unroll
        for (int i = 0; i < 8; ++i)
  #pragma unroll
          for (int z = 0; z < 4; ++z) mn = min(mn, pv[i][z]);
        if (__builtin_amdgcn_ballot_w64(mn == 0u) == 0ull) break;
        if (give_up(t0, myws, call)) dead = true;
        if (RSL) __builtin_amdgcn_s_sleep(RSL);
      }
      if (dead) {
        if (lane == 0) st_agt(myws + TMO_OFF, call);
      } else {
  #pragma unroll
        for (int nt = 0; nt < 2; ++nt)
  #pragma unroll
          for (int mt = 0; mt < 2; ++mt)
  #pragma unroll
            for (int sx = 0; sx < 2; ++sx)
  #pragma unroll
              for (int z = 0; z < 4; ++z) {
                const u32 u = pv[(nt * 2 + mt) * 2 + sx][z];
                acc[nt][mt][8 * sx + 2 * z] += bf_lo(u);
                acc[nt][mt][8 * sx + 2 * z + 1] += bf_hi(u);
              }
      }
    }
  }

  STAMP(myws, 6);
  if (!fin) {
    char* dst = uni(myws + (size_t)(POUT_OFF + (u32)((j - 1) * 16 + k) * PIECE + tile_off));
    char* fl = myws + (size_t)(FLOC_OFF + (u32)((j - 1) * 8 + e * 4 + wave) * 128u + (u32)k * 4u);
    asm volatile("s_nop 4" ::: "memory");
#pragma unroll
    for (int nt = 0; nt < 2; ++nt)
#pragma unroll
      for (int mt = 0; mt < 2; ++mt) {
        u32x4 v0, v1;
#pragma unroll
        for (int z = 0; z < 4; ++z) {
          const u32 t0_ = pack2(acc[nt][mt][2 * z], acc[nt][mt][2 * z + 1]);
          const u32 t1_ = pack2(acc[nt][mt][8 + 2 * z], acc[nt][mt][8 + 2 * z + 1]);
          v0[z] = dead ? 0x7fc07fc0u : (t0_ ? t0_ : 0x8000u);
          v1[z] = dead ? 0x7fc07fc0u : (t1_ ? t1_ : 0x8000u);
        }
        const u32 o = nt ? l16b : l16;
        if (mt == 0) { st16<0>(o, v0, dst); st16<1024>(o, v1, dst); }
        else         { st16<2048>(o, v0, dst); st16<3072>(o, v1, dst); }
      }
    waitv<0>();
    if (lane == 0) st_sys(fl, call);
  } else {
    const u32 h = lane >> 5, m = lane & 31u;
    const u32 tb = sbase + (u32)wave * 8192u;
#pragma unroll
    for (int nt = 0; nt < 2; ++nt)
#pragma unroll
      for (int mt = 0; mt < 2; ++mt)
#pragma unroll
        for (int sx = 0; sx < 2; ++sx) {
          u32x4 v;
#pragma unroll
          for (int z = 0; z < 4; ++z) {
            const float lo = acc[nt][mt][8 * sx + 2 * z], hi = acc[nt][mt][8 * sx + 2 * z + 1];
            v[z] = dead ? 0x7fc07fc0u : pack2(lo < 0.f ? 0.f : lo, hi < 0.f ? 0.f : hi);
          }
          *(volatile LDSP u32x4*)(uintptr_t)(tb + ((u32)mt * 32u + m) * 128u + ((((u32)(nt * 4 + sx * 2) + h) ^ (m & 7u)) * 16u)) = v;
        }
    char* ob = uni(P.out + (size_t)((u32)(row0 - r * 512 + wm * 64) * 8192u + (u32)(col0 + wn * 64) * 2u));
    const u32 rr = lane >> 3, cc = lane & 7u;
    const u32 ra = tb + rr * 128u + ((cc ^ rr) * 16u);
    u32x4 ov[8];
#pragma unroll
    for (int i = 0; i < 8; ++i) ov[i] = *(volatile LDSP u32x4*)(uintptr_t)(ra + (u32)i * 1024u);
#pragma unroll
    for (int i = 0; i < 8; ++i) OST<0>(((u32)i * 8u + rr) * 8192u + cc * 16u, ov[i], ob);
  }
  STAMP(myws, 5);
}

__global__ __launch_bounds__(256, 2) void fused_gemm_rs(Params P) {
  extern __shared__ __attribute__((aligned(16))) char smem[];
  const u32 lane = threadIdx.x & 63u;
  const int wave = __builtin_amdgcn_readfirstlane(threadIdx.x >> 6);
  const int bid = blockIdx.x;
  if (bid < N_PUSH_WG) mover(P, wave, lane, smem);
  else if (bid >= 256 && bid < 256 + N_PUSH_WG) {
    if (PW & 2) {
      STAMP(P.ws[P.rank], 0);
      const u32 pr = (u32)(P.rank ^ (wave == 0 ? 1 + ((bid - 256) >> 3) : 0)) & (u32)(NR - 1);
      if (PWD) __builtin_amdgcn_s_sleep(PWD);
      if ((wave == 0 || (wave == 1 && bid < 264)) && lane < 27u) probe(lane * (2u * MIB) + PRB_OFF, uni(P.ws[pr]));
      if (wave == 2 && bid < 264 && lane < 5u) probe(lane * (MIB - 4u), uni(P.out));
      waitv<0>();
      STAMP(P.ws[P.rank], 1);
    }
    for (int i = 0; i < 6; ++i) __builtin_amdgcn_s_sleep(16);
  } else tile(P, bid < 256 ? bid - N_PUSH_WG : bid - 2 * N_PUSH_WG, wave, lane, smem);
}

}

extern "C" void kernel_launch(int rank, int nranks, ncclComm_t comm,
                              hipStream_t stream,
                              void* const* d_in, const int* in_sizes, int n_in,
                              void* d_out, int out_size,
                              const int* in_dtypes, int out_dtype,
                              void* const* d_out_all,
                              void* const* d_ws_all, size_t ws_bytes) {
  if (nranks != NR || n_in < 2 || ws_bytes < (size_t)WS_NEED || (unsigned)rank >= (unsigned)NR) {
    fprintf(stderr, "unsupported configuration\n");
    return;
  }
  Params P;
  P.x = (const char*)d_in[0];
  P.w = (const char*)d_in[1];
  P.out = (char*)d_out;
  for (int i = 0; i < NR; ++i) P.ws[i] = (char*)d_ws_all[i];
  P.rank = rank;
  HIP_CHECK(hipFuncSetAttribute((const void*)fused_gemm_rs, hipFuncAttributeMaxDynamicSharedMemorySize, (int)LDS_BYTES));
  hipLaunchKernelGGL(fused_gemm_rs, dim3(N_WG), dim3(256), LDS_BYTES, stream, P);
  HIP_CHECK(hipGetLastError());
}
```

```cpp
#include <hip/hip_runtime.h>
#include <rccl/rccl.h>
#include <cstdio>
#include <cstdint>

#define HIP_CHECK(expr) do { hipError_t _e = (expr); if (_e != hipSuccess) { \
    fprintf(stderr, "HIP error %s at %s:%d\n", hipGetErrorString(_e), __FILE__, __LINE__); \
    return; } } while (0)

namespace {

typedef unsigned u32;
typedef unsigned long long u64;
typedef u32 u32x4 __attribute__((ext_vector_type(4)));
typedef float f32x2 __attribute__((ext_vector_type(2)));
typedef float f32x16 __attribute__((ext_vector_type(16)));
typedef __bf16 bf16x2 __attribute__((ext_vector_type(2)));
typedef __bf16 bf16x8 __attribute__((ext_vector_type(8)));
typedef short s16x4 __attribute__((ext_vector_type(4)));
typedef short s16x8 __attribute__((ext_vector_type(8)));
#define LDSP __attribute__((address_space(3)))
#define GLBP __attribute__((address_space(1)))

constexpr int NR = 8;
constexpr u32 PIECE = 65536u;
constexpr u32 MIB = 1u << 20;
#define NS 2
#define OX 14
#define TAU 115
#define T0 20
#define EARLY 150
#define TAU1 110
#define TAU2 0
#define STGD 2
#define PSL 4
#define LSL 0
#define WSL 3
#define RSL 0
#define WOWN 0
#define ALN 1
#define ALO 99
#define ALT 25
#define PWN 1
#define DIR 0
#define DSL 4
#define SWP 14
#define CPF 19968
#define CPB 56
#define PD0 0
#define PD1 0
#define PDF 0
#define GSH 0
#define LPT 400
#define ROT 48
#define PWIN 64
#define PWHO 1
#define PQ1 9
#define PQ2 14
#define G1 0
#define LAG 1
#define WIN 3
#define GLP "sc0 sc1"
#define MLP "sc0 sc1"
constexpr int NIN = 30;
constexpr int NOUT = 16 - NS;
constexpr int NOUT1 = 16 + 6 * NS;
constexpr u32 IN_SRC = 4u * MIB;
constexpr u32 PIN_OFF = 24u * MIB;
constexpr u32 PIN_PAR = 8u * MIB;
constexpr u32 POUT_OFF = 48u * MIB;
constexpr u32 FLG_OFF = 55u * MIB;
constexpr u32 FLOC_OFF = FLG_OFF + 16384u;
constexpr u32 TMO_OFF = FLG_OFF + 49152u;
constexpr u32 DUMMY_OFF = FLG_OFF + 65536u;
constexpr u32 DBG_OFF = FLG_OFF + 131072u;
constexpr u32 CNT_OFF = FLG_OFF + MIB + 262144u;
constexpr u32 WS_NEED = FLG_OFF + 2u * MIB;

constexpr int N_PUSH_WG = 56;
constexpr int N_MAIN_WG = 256;
constexpr int N_FIN_WG = 96;
constexpr int N_WG = 2 * N_PUSH_WG + N_MAIN_WG + N_FIN_WG;
constexpr u32 STAGE_B = 16384u;
constexpr u32 LDS_BYTES = 4u * STAGE_B + 1024u;
constexpr u64 DEADLINE = 100000000ull;
constexpr u32 DEAD = 0x7fffffffu;

struct Params {
  const char* x;
  const char* w;
  char* my;
  char* out;
  int rank;
  int pad;
  char* peer[NR];
};
struct Args { const char* x; const char* w; char* my; char* out; char* peer; int rank; u32 t1; };

#define SB() __builtin_amdgcn_sched_barrier(0)
template <int N> __device__ __forceinline__ void waitv() { asm volatile("s_waitcnt vmcnt(%0)" :: "i"(N) : "memory"); }

__device__ __forceinline__ u64 now() { return __builtin_amdgcn_s_memrealtime(); }
template <typename T> __device__ __forceinline__ T* uni(T* p) {
  const u64 v = (u64)(uintptr_t)p;
  const u32 lo = __builtin_amdgcn_readfirstlane((u32)v), hi = __builtin_amdgcn_readfirstlane((u32)(v >> 32));
  return (T*)(uintptr_t)(((u64)hi << 32) | lo);
}
__device__ __forceinline__ u32 ld_sys(const void* p) {
  return __hip_atomic_load((const GLBP u32*)(uintptr_t)p, __ATOMIC_RELAXED, __HIP_MEMORY_SCOPE_SYSTEM);
}
__device__ __forceinline__ void st_sys(void* p, u32 v) {
  __hip_atomic_store((GLBP u32*)(uintptr_t)p, v, __ATOMIC_RELAXED, __HIP_MEMORY_SCOPE_SYSTEM);
}
__device__ __forceinline__ void st_agt(void* p, u32 v) {
  __hip_atomic_store((GLBP u32*)(uintptr_t)p, v, __ATOMIC_RELAXED, __HIP_MEMORY_SCOPE_AGENT);
}
__device__ __forceinline__ bool give_up(u64 t0, const char* ws, u32 call) {
  const u64 dt = now() - t0;
  if (!__builtin_amdgcn_readfirstlane((int)(dt > 50000ull))) return false;
  __builtin_amdgcn_s_sleep(64);
  if (__builtin_amdgcn_readfirstlane((int)(dt > DEADLINE))) return true;
  return __builtin_amdgcn_readfirstlane(ld_sys(ws + TMO_OFF)) == call;
}
__device__ __forceinline__ void sleep_until(u64 tstart, u32 t) {
  for (;;) {
    const u32 el = (u32)__builtin_amdgcn_readfirstlane((u32)(now() - tstart));
    if (el >= t) break;
    if (t - el > 40u) __builtin_amdgcn_s_sleep(12); else __builtin_amdgcn_s_sleep(1);
  }
}
template <int IMM> __device__ __forceinline__ void st16(u32 off, u32x4 v, char* base) {
  asm volatile("global_store_dwordx4 %0, %1, %2 offset:%3 sc0 sc1\n\ts_nop 1" :: "v"(off), "v"(v), "s"(base), "i"(IMM) : "memory");
}
template <int IMM> __device__ __forceinline__ void st16p(u32 off, u32x4 v, char* base) {
  asm volatile("global_store_dwordx4 %0, %1, %2 offset:%3\n\ts_nop 1" :: "v"(off), "v"(v), "s"(base), "i"(IMM) : "memory");
}
#define OST st16
__device__ __forceinline__ void dma16x4(u32 lds, const u32 (&o)[4], const char* base) {
  asm volatile("s_mov_b32 m0, %0\n\ts_nop 3\n\t"
               "global_load_lds_dwordx4 %1, %5 " MLP "\n\t"
               "global_load_lds_dwordx4 %2, %5 offset:1024 " MLP "\n\t"
               "global_load_lds_dwordx4 %3, %5 offset:2048 " MLP "\n\t"
               "global_load_lds_dwordx4 %4, %5 offset:3072 " MLP
               :: "s"(lds), "v"(o[0]), "v"(o[1]), "v"(o[2]), "v"(o[3]), "s"(base) : "memory");
}
__device__ __forceinline__ void probe(u32 off, char* base) {
  asm volatile("s_nop 4\n\tglobal_atomic_add %0, %1, %2 sc1" :: "v"(off), "v"(0u), "s"(base) : "memory");
}
__device__ __forceinline__ void probe0(u32 off, char* base) {
  asm volatile("s_mov_b64 exec, 1\n\ts_nop 3\n\tglobal_atomic_add %0, %1, %2 sc1\n\ts_mov_b64 exec, -1"
               :: "v"(off), "v"(0u), "s"(base) : "memory");
}
__device__ __forceinline__ u32 atom_rd(u32 off, const char* base) {
  u32 r;
  asm volatile("global_atomic_add %0, %1, %2, %3 sc0 sc1\n\ts_waitcnt vmcnt(0)" : "=&v"(r) : "v"(off), "v"(0u), "s"(base) : "memory");
  return r;
}
__device__ __forceinline__ u32 bump_issue(u32 off, const char* base) {
  u32 r;
  asm volatile("s_mov_b64 exec, 1\n\ts_nop 3\n\tglobal_atomic_add %0, %1, %2, %3 sc0 sc1\n\ts_mov_b64 exec, -1"
               : "=&v"(r) : "v"(off), "v"(1u), "s"(base) : "memory");
  return r;
}
template <int N> __device__ __forceinline__ u32 bump_get(u32 r) {
  u32 v;
  asm volatile("s_waitcnt vmcnt(%2)\n\tv_readfirstlane_b32 %0, %1\n\ts_nop 4" : "=s"(v) : "v"(r), "i"(N) : "memory");
  return v + 1u;
}
__device__ __forceinline__ void add1_lane0(u32 off, char* base) {
  asm volatile("s_mov_b64 exec, 1\n\ts_nop 3\n\tglobal_atomic_add %0, %1, %2 sc1\n\ts_mov_b64 exec, -1"
               :: "v"(off), "v"(1u), "s"(base) : "memory");
}
__device__ __forceinline__ void st4_lane0(u32 off, u32 val, char* base) {
  asm volatile("s_mov_b64 exec, 1\n\ts_nop 3\n\tglobal_store_dword %0, %1, %2 sc0 sc1\n\ts_mov_b64 exec, -1"
               :: "v"(off), "v"(val), "s"(base) : "memory");
}

#define STAMP(ws, idx) do {} while (0)
#define STAMPV(ws, idx, v) do {} while (0)

__device__ __forceinline__ void piece_of(int q, int d, bool& isx, int& idx) {
  const int t = q - NS, u = q - NS - 12;
  int j = (t >> 1) + 2;
  j += j >= d ? 1 : 0;
  j = j == 8 ? 1 : j;
  isx = (q >= NS && u < 0) || (u >= OX && u < OX + 2);
  idx = q < NS ? q : (u < 0 ? 2 * j + (t & 1) : (u < OX ? NS + u : (u < OX + 2 ? u - OX : NS + u - 2)));
}
__device__ __forceinline__ int slot_w(int k) { return k < NS ? k : (k - NS < OX ? 12 + k : 14 + k); }
__device__ __forceinline__ int slot_x(int j, int e, int d) {
  return j == 0 ? NS + 12 + OX + e : NS + 2 * (j == 1 ? 5 : j - 2 - (j > d ? 1 : 0)) + e;
}
__device__ __forceinline__ int cont_w(int k) { return k < NS ? k : 14 + k; }
__device__ __forceinline__ int cont_x(int rb) { return rb < 2 ? 30 + rb : (rb < 4 ? NS + 10 + rb : NS - 4 + rb); }
__device__ __forceinline__ u32 fin_off(int sl, int q) { return (u32)sl * IN_SRC + (u32)(NS + 2 * sl) * 2u * PIECE + (u32)q * 64u; }
__device__ __forceinline__ u32 lds_rd(u32 a) { return __builtin_amdgcn_readfirstlane(*(volatile LDSP u32*)(uintptr_t)a); }
__device__ __forceinline__ void lds_wr(u32 a, u32 v) { *(volatile LDSP u32*)(uintptr_t)a = v; }

__device__ __forceinline__ void mover(const Args& P, int d, int wave, u32 lane, char* smem, u64 tstart) {
  __builtin_amdgcn_s_setprio(3);
  const int s = P.rank;
  char* const myws = P.my;
  const int pair = wave >> 1;
  const int p = (blockIdx.x & 7) * 2 + pair;
  const u32 r = (u32)(s ^ d) & (u32)(NR - 1);
  const int nin = d == 1 ? 0 : NIN;
  const int np = d == 1 ? (DIR ? 16 : NOUT1) : NIN + NOUT;
  const int nid = d == 1 ? NOUT1 : NOUT;
  const u32 ring = (u32)(uintptr_t)smem + (u32)pair * 32768u;
  const u32 ctl = (u32)(uintptr_t)smem + 65536u + (u32)pair * 512u;
  const char* const xin = P.x;
  const char* const win = P.w;
  const int rg = r >> 2, rb_ = (r >> 1) & 1, ra = r & 1;
  u32 wo[4];
#pragma unroll
  for (int k = 0; k < 4; ++k) {
    const u32 q4 = lane >> 4;
    wo[k] = ((u32)p * 16u + (u32)k * 4u + q4) * 8192u + (((lane & 15u) ^ ((q4 << 2) | (u32)k)) * 16u) - (u32)k * 1024u;
  }
#define W_BASE(IX) (win + (size_t)((u32)ra * 256u * 8192u + (u32)rg * 4096u + (u32)(ra ? 15 - (IX) : (IX)) * 256u))

  if ((wave & 1) == 0) {
    const u32 cnt = bump_issue(CNT_OFF + (u32)(blockIdx.x * 4 + wave) * 64u, myws);
    u32 xo[4], oo[4];
    {
      const u32 xc = (lane & 3u) ^ ((lane >> 4) & 3u);
#pragma unroll
      for (int k = 0; k < 4; ++k) {
        xo[k] = ((u32)(p & 1) * 64u + (u32)k * 16u + (lane >> 2)) * 1024u + (u32)(p >> 1) * 64u + xc * 16u - (u32)k * 1024u;
        oo[k] = (u32)p * 4096u + lane * 16u;
      }
    }
    u32 poll_off;
    {
      const int i0 = (int)lane < nid ? (int)lane : 0;
      const bool sw = d == 1 && i0 < SWP;
      const int i = sw ? (i0 & ~1) | (p >> 3) : i0, ps = sw ? ((i0 & 1) << 3) | (p & 7) : p;
      const int jj = d != 1 ? d : (i < 6 * NS ? 2 + i / NS : 1);
      const int kk = d != 1 ? NS + i : (i < 6 * NS ? i % NS : i - 6 * NS);
      poll_off = FLOC_OFF + (u32)((jj - 1) * 8 + (ps >> 1)) * 128u + (u32)kk * 4u;
    }
#define ISSUE_LOADS(QQ, II) do { \
      const int q_ = (QQ); \
      const u32 dst_ = ring + (u32)(q_ & 7) * 4096u; \
      if (q_ < nin) { \
        bool isx_; int ix_; \
        piece_of(q_, d, isx_, ix_); \
        if (isx_) dma16x4(dst_, xo, xin + (size_t)(((r ^ (u32)(ix_ >> 1)) * 512u + (u32)rb_ * 256u + (u32)(ix_ & 1) * 128u) * 1024u + (u32)ra * 512u)); \
        else      dma16x4(dst_, wo, W_BASE(ix_)); \
      } else { \
        const bool sw_ = d == 1 && (II) < SWP; \
        const int i_ = sw_ ? ((II) & ~1) | (p >> 3) : (II); \
        const int jj_ = d != 1 ? d : (i_ < 6 * NS ? 2 + i_ / NS : 1); \
        const int kk_ = d != 1 ? NS + i_ : (i_ < 6 * NS ? i_ % NS : i_ - 6 * NS); \
        dma16x4(dst_, oo, myws + (size_t)(POUT_OFF + (u32)((jj_ - 1) * 16 + kk_) * PIECE + (u32)(sw_ ? (((II) & 1) - (p >> 3)) * 32768 : 0))); \
      } \
    } while (0)

    int q0 = 0;
    u32 pub = 0, call;
    u32 dbg1 = 0, dbg2 = 0;
    if (nin) {
      ISSUE_LOADS(1, 0); ISSUE_LOADS(2, 0); ISSUE_LOADS(3, 0);
    }
    if (nin) {
      call = bump_get<8>(cnt);
      lds_wr(ctl, 2u);
      q0 = 4; pub = 2;
    } else {
      call = bump_get<0>(cnt);
    }

    if (!nin) sleep_until(tstart, (u32)(T0 + (NS + (DIR ? 12 : 2)) * TAU + 500));
    u64 ready = 0;
    u64 todo = ((1ull << (np - nin)) - 1ull) << (nid - (np - nin));
    u32 cons = 0;
    bool dead = false;
    for (int q = q0; q < np; ++q) {
      bool rdy = q < nin || (ready & todo) != 0ull;
      if (cons + 8u <= (u32)q) cons = lds_rd(ctl + 4u);
      if (!rdy || cons + 8u <= (u32)q) {
        waitv<0>();
        pub = (u32)q;
        lds_wr(ctl, pub);
        const u64 t0 = now();
        u32 win = 64u;
        for (;;) {
          if (!rdy) {
            u32 pv = 0;
            const u32 below = __builtin_amdgcn_mbcnt_hi((u32)(todo >> 32), __builtin_amdgcn_mbcnt_lo((u32)todo, 0u));
            if (((todo >> lane) & 1ull) && below < win) pv = atom_rd(poll_off, myws);
            win = (u32)PWIN;
            ready |= __builtin_amdgcn_ballot_w64(pv >= call);
            rdy = (ready & todo) != 0ull;
            if (LSL && !rdy) __builtin_amdgcn_s_sleep(LSL);
          } else {
            __builtin_amdgcn_s_sleep(2);
          }
          cons = lds_rd(ctl + 4u);
          if (rdy && cons + 8u > (u32)q) break;
          if (give_up(t0, myws, call)) { dead = true; break; }
        }
        if (dead) break;
      }
      int id = 0;
      if (q >= nin) {
        id = __builtin_ctzll(ready & todo);
        todo &= ~(1ull << id);
        lds_wr(ctl + 32u + (u32)(q & 7) * 4u, (u32)id);
      }
      ISSUE_LOADS(q, id);
      waitv<8>();
      if (PWHO == 1 && nin && q == PQ1) probe0((u32)(d - 2) * IN_SRC + 2u * MIB + (u32)p * 4096u, P.peer);
      if (PWHO == 1 && nin && q == PQ2) probe0(PIN_OFF + (call % 3u) * PIN_PAR + (u32)(32 + (d - 2) * 16 + NS) * PIECE + (u32)p * 4096u, P.peer);
      if (q - 1 > (int)pub) { pub = (u32)(q - 1); lds_wr(ctl, pub); }
      if (q == nin - 1) STAMP(myws, 1);
    }
    waitv<0>();
    if (dead) st4_lane0(TMO_OFF, call, myws);
    lds_wr(ctl, dead ? DEAD : (u32)np);
    STAMP(myws, 3);
    STAMPV(myws, 0, tstart);
    return;
  }

  char* const rws = P.peer;
  u32x4 f0, f1, f2, f3;
  asm volatile("s_nop 4\n\t"
               "global_load_dwordx4 %0, %4, %8 " MLP "\n\t"
               "global_load_dwordx4 %1, %5, %8 offset:1024 " MLP "\n\t"
               "global_load_dwordx4 %2, %6, %8 offset:2048 " MLP "\n\t"
               "global_load_dwordx4 %3, %7, %8 offset:3072 " MLP
               : "=&v"(f0), "=&v"(f1), "=&v"(f2), "=&v"(f3)
               : "v"(wo[0]), "v"(wo[1]), "v"(wo[2]), "v"(wo[3]), "s"(W_BASE(0)) : "memory");
  const u32 cnt = bump_issue(CNT_OFF + (u32)(blockIdx.x * 4 + wave) * 64u, myws);
  const int slot = d - 2;
  if (lane < (u32)PWN) probe(fin_off(max(slot, 0), 64 + p * 2) + lane * 2u * MIB, rws);
  u32 dbg1 = 0, dbg2 = 0;
  u32 tgrid = (u32)(T0 + TAU - 1) - ((u32)__builtin_amdgcn_readfirstlane((u32)tstart) + (u32)(T0 + TAU - 1) - ((u32)(((p + STGD * d) & 15) * TAU) >> 4)) % (u32)TAU;
  const u32 call = bump_get<1>(cnt);
  asm volatile("" : "+v"(f0), "+v"(f1), "+v"(f2), "+v"(f3));
  const u32 in0 = (u32)slot * IN_SRC + (call & 1u) * PIECE;
  const u32 dl = (u32)p * 4096u + lane * 16u;
  const u32 pin0 = PIN_OFF + (call % 3u) * PIN_PAR + (u32)(d == 1 ? 0 : 32 + slot * 16 + NS) * PIECE;
  int nflag = 0;
#define FLAG(QQ) do { \
    const int q_ = (QQ); \
    const bool real_ = q_ == nflag && q_ < nin; \
    const u32 o_ = real_ ? fin_off(slot, q_) : DUMMY_OFF + (u32)(blockIdx.x * 4 + wave) * 128u; \
    add1_lane0(o_, uni(real_ ? rws : myws)); \
    nflag += real_ ? 1 : 0; \
  } while (0)

#define SLIP() do { \
    const u32 el_ = (u32)__builtin_amdgcn_readfirstlane((u32)(now() - tstart)); \
    if (el_ > tgrid + (u32)ALT) tgrid += (el_ - tgrid - (u32)ALT + (u32)TAU - 1u) / (u32)TAU * (u32)TAU; \
    sleep_until(tstart, tgrid); \
  } while (0)
  if (nin) {
    char* const dst = rws + (size_t)in0;
    SLIP();
    st16<0>(dl, f0, dst);
    st16<1024>(dl, f1, dst);
    st16<2048>(dl, f2, dst);
    st16<3072>(dl, f3, dst);
    FLAG(-1);
    tgrid += (u32)(TAU - GSH);
  } else {
    sleep_until(tstart, 600u);
    probe0(pin0 + (u32)p * 4096u, rws);
    probe0(pin0 + (u32)(NOUT1 - 1) * PIECE + (u32)p * 4096u, rws);
  }
  u32 full = 0, tnext = 0;
  for (int q = nin ? 1 : 0; q < np; ++q) {
    if (d != 1 || G1) tnext = max(tnext, tgrid);
    sleep_until(tstart, tnext);
    if (full <= (u32)q) {
      full = lds_rd(ctl);
      if (full <= (u32)q) {
        const u64 t0 = now();
        bool drained = !(q > 0 && q < nin + LAG);
        for (;;) {
          __builtin_amdgcn_s_sleep(1);
          full = lds_rd(ctl);
          if (full > (u32)q) break;
          const u64 dt = now() - t0;
          if (!drained && __builtin_amdgcn_readfirstlane((int)(dt > 30ull))) {
            drained = true;
            waitv<0>();
#pragma unroll
            for (int z = LAG; z >= 1; --z) FLAG(q - z);
          }
          if (__builtin_amdgcn_readfirstlane((int)(dt > 2ull * DEADLINE - 1000ull))) { full = DEAD; break; }
        }
        tnext = (u32)__builtin_amdgcn_readfirstlane((u32)(now() - tstart));
      }
      if (full == DEAD) break;
    }
    if ((d != 1 || G1) && (q < ALN || q >= nin + ALO)) { SLIP(); tnext = tgrid; }
    tgrid += (u32)TAU;
    tnext += (u32)(d == 1 && !G1 ? TAU1 : TAU2);
    const u32 la = ring + (u32)(q & 7) * 4096u + lane * 16u;
    const u32x4 d0 = *(volatile LDSP u32x4*)(uintptr_t)la;
    const u32x4 d1 = *(volatile LDSP u32x4*)(uintptr_t)(la + 1024u);
    const u32x4 d2 = *(volatile LDSP u32x4*)(uintptr_t)(la + 2048u);
    const u32x4 d3 = *(volatile LDSP u32x4*)(uintptr_t)(la + 3072u);
    const u32 id = lds_rd(ctl + 32u + (u32)(q & 7) * 4u);
    lds_wr(ctl + 4u, (u32)q + 1u);
    u32 o;
    if (q < nin) {
      bool isx; int ix;
      piece_of(q, d, isx, ix);
      o = in0 + (u32)(isx ? cont_x(ix) : cont_w(ix)) * 2u * PIECE;
    } else {
      o = pin0 + min(id, (u32)(nid - 1)) * PIECE;
      if (d == 1 && id < (u32)SWP) o += (u32)(((p >> 3) - (int)(id & 1u)) * 32768);
    }
    char* const dst = rws + (size_t)o;
    st16<0>(dl, d0, dst);
    st16<1024>(dl, d1, dst);
    st16<2048>(dl, d2, dst);
    st16<3072>(dl, d3, dst);
    if (nin && q < nin + LAG) {
      waitv<5 * LAG>();
      FLAG(q - LAG);
      if (PWHO == 0 && q == PQ1) probe0((u32)slot * IN_SRC + 2u * MIB + (u32)p * 4096u, rws);
      if (PWHO == 0 && q == PQ2) probe0(pin0 + (u32)p * 4096u, rws);
    } else {
      waitv<4 * WIN>();
    }
    if (q == NS + 11) STAMP(myws, 1);
    if (q == nin - 1) STAMP(myws, 2);
    if (q == np - 1) STAMP(myws, 4);
  }
  waitv<0>();
  STAMP(myws, 3);
  STAMPV(myws, 0, tstart);
}

struct Frag {
  bf16x8 x[2][2];
  bf16x8 w[2][2];
};

struct Loader {
  const char* base;
  u32 stride;
  u32 v[4];
  u32 m0base;
};

template <int STAGE> __device__ __forceinline__ void glds_pair(const Loader& L, int half) {
  const u32 m0v = L.m0base + STAGE * STAGE_B;
  if (half == 0)
    asm volatile("s_mov_b32 m0, %0\n\ts_nop 0\n\t"
                 "global_load_lds_dwordx4 %1, %3 " GLP "\n\t"
                 "global_load_lds_dwordx4 %2, %3 offset:1024 " GLP
                 :: "s"(m0v), "v"(L.v[0]), "v"(L.v[1]), "s"(L.base) : "memory");
  else
    asm volatile("s_mov_b32 m0, %0\n\ts_nop 0\n\t"
                 "global_load_lds_dwordx4 %1, %3 offset:2048 " GLP "\n\t"
                 "global_load_lds_dwordx4 %2, %3 offset:3072 " GLP
                 :: "s"(m0v), "v"(L.v[2]), "v"(L.v[3]), "s"(L.base) : "memory");
}

struct RdAddr { u32 xa[2]; u32 wa[2][2]; };

template <int STAGE> __device__ __forceinline__ bf16x8 rd_x(const RdAddr& A, int mt, int ks) {
  return *(const LDSP bf16x8*)(uintptr_t)(A.xa[ks] + (u32)(STAGE * STAGE_B + mt * 2048));
}
template <int STAGE> __device__ __forceinline__ bf16x8 rd_w(const RdAddr& A, int nt, int ks) {
  s16x4 lo = __builtin_amdgcn_ds_read_tr16_b64_v4i16((LDSP s16x4*)(uintptr_t)(A.wa[nt][0] + (u32)(STAGE * STAGE_B + 8192 + ks * 4096)));
  s16x4 hi = __builtin_amdgcn_ds_read_tr16_b64_v4i16((LDSP s16x4*)(uintptr_t)(A.wa[nt][1] + (u32)(STAGE * STAGE_B + 8192 + ks * 4096)));
  s16x8 v = __builtin_shufflevector(lo, hi, 0, 1, 2, 3, 4, 5, 6, 7);
  return __builtin_bit_cast(bf16x8, v);
}
#define MFMA(nt, mt, ks) acc[nt][mt] = __builtin_amdgcn_mfma_f32_32x32x16_bf16(cur.w[nt][ks], cur.x[mt][ks], acc[nt][mt], 0, 0, 0)

template <int S, int VM, bool LOAD, bool READ>
__device__ __forceinline__ void kstep(f32x16 (&acc)[2][2], const Frag& cur, Frag& nxt, const RdAddr& A, Loader& L) {
  constexpr int SN = (S + 1) & 3;
  if (READ) {
    __builtin_amdgcn_s_waitcnt((VM & 15) | (7 << 4) | (0 << 8) | ((VM >> 4) << 14));
    asm volatile("s_barrier" ::: "memory");
  } else {
    __builtin_amdgcn_s_waitcnt(15 | (7 << 4) | (0 << 8) | (3 << 14));
  }
  SB();
  MFMA(0, 0, 0); SB();
  if (READ) { nxt.w[0][0] = rd_w<SN>(A, 0, 0); nxt.x[0][0] = rd_x<SN>(A, 0, 0); } SB();
  MFMA(0, 1, 0); SB();
  if (READ) { nxt.x[1][0] = rd_x<SN>(A, 1, 0); nxt.w[1][0] = rd_w<SN>(A, 1, 0); } SB();
  MFMA(1, 0, 0); SB();
  if (LOAD) glds_pair<S>(L, 0);
  SB();
  MFMA(1, 1, 0); SB();
  if (READ) { nxt.w[0][1] = rd_w<SN>(A, 0, 1); nxt.x[0][1] = rd_x<SN>(A, 0, 1); } SB();
  MFMA(0, 0, 1); SB();
  if (READ) { nxt.x[1][1] = rd_x<SN>(A, 1, 1); nxt.w[1][1] = rd_w<SN>(A, 1, 1); } SB();
  MFMA(0, 1, 1); SB();
  if (LOAD) { glds_pair<S>(L, 1); L.base += L.stride; }
  SB();
  MFMA(1, 0, 1); SB();
  MFMA(1, 1, 1); SB();
}

template <int S> __device__ __forceinline__ void stage_load(Loader& L) {
  glds_pair<S>(L, 0); glds_pair<S>(L, 1); L.base += L.stride;
}

__device__ __forceinline__ void gemm_phase(f32x16 (&acc)[2][2], const RdAddr& A, Loader& L, int n, int sk, u32 jump, u32& dbgv, int dbg0) {
  Frag f0, f1;
  L.base = uni(L.base + (sk == 0 ? IN_SRC : 0u));
  L.stride = __builtin_amdgcn_readfirstlane(L.stride);
  asm volatile("s_nop 4" ::: "memory");
  stage_load<0>(L); stage_load<1>(L); stage_load<2>(L); stage_load<3>(L);
  waitv<12>();
  asm volatile("s_barrier" ::: "memory");
#pragma unroll
  for (int ks = 0; ks < 2; ++ks)
#pragma unroll
    for (int i = 0; i < 2; ++i) { f0.w[i][ks] = rd_w<0>(A, i, ks); f0.x[i][ks] = rd_x<0>(A, i, ks); }
  SB();
  for (int t = 0; t < n - 4; t += 4) {
    L.base += (t & 4 ? jump : 0u) + (t + 4 == 8 * sk ? IN_SRC : 0u);
    kstep<0, 8, true, true>(acc, f0, f1, A, L);
    kstep<1, 8, true, true>(acc, f1, f0, A, L);
    kstep<2, 8, true, true>(acc, f0, f1, A, L);
    kstep<3, 8, true, true>(acc, f1, f0, A, L);
  }
  kstep<0, 8, false, true>(acc, f0, f1, A, L);
  kstep<1, 4, false, true>(acc, f1, f0, A, L);
  kstep<2, 0, false, true>(acc, f0, f1, A, L);
  kstep<3, 0, false, false>(acc, f1, f0, A, L);
}

__device__ __forceinline__ u32 pack2(float a, float b) {
  f32x2 t = {a, b};
  return __builtin_bit_cast(u32, __builtin_convertvector(t, bf16x2));
}
__device__ __forceinline__ float bf_lo(u32 v) { return __builtin_bit_cast(float, v << 16); }
__device__ __forceinline__ float bf_hi(u32 v) { return __builtin_bit_cast(float, v & 0xffff0000u); }

__device__ __forceinline__ void zero8k(u32 l16, u32 l16b, char* pb) {
  const u32x4 z4 = {0u, 0u, 0u, 0u};
  asm volatile("s_nop 4" ::: "memory");
  st16<0>(l16, z4, pb); st16<1024>(l16, z4, pb); st16<2048>(l16, z4, pb); st16<3072>(l16, z4, pb);
  st16<0>(l16b, z4, pb); st16<1024>(l16b, z4, pb); st16<2048>(l16b, z4, pb); st16<3072>(l16b, z4, pb);
}

__device__ __forceinline__ void tile(const Args& P, int tid, int wave, u32 lane, char* smem, u64 tstart) {
  const int r = P.rank;
  char* const myws = P.my;
  const int g = r >> 2, b = (r >> 1) & 1, a = r & 1;
  const int bid = blockIdx.x;

  const u32 cnt = bump_issue(CNT_OFF + (u32)(bid * 4 + wave) * 64u, myws);

  int row0, col0, nrem, tskip, j, e, k, rb;
  int pi0 = -1, pi1 = -1;
  bool fin;
  if (tid < N_MAIN_WG) {
    const int xcd = tid & 7, o = tid >> 3, u = xcd >> 1;
    int rbi, cbi;
    if (u < 2) { rbi = o < 24 ? 1 + o % 3 : 0; cbi = o < 24 ? o / 3 : o - 24; }
    else { rbi = o & 3; cbi = o >> 2; }
    rb = u + 4 * rbi; k = cbi * 2 + (xcd & 1);
    j = rb >> 1; e = rb & 1;
    row0 = (r ^ j) * 512 + b * 256 + e * 128; col0 = g * 2048 + (a ? 15 - k : k) * 128;
    nrem = j >= 2 ? 40 : 48; tskip = j >= 2 ? j - 2 : 99;
    fin = j == 0;
    if (j == 0) pi0 = 6 * NS + 15 - k;
    else if (j >= 2 && k >= 16 - NS) pi0 = ((j ^ 1) - 2) * NS + 15 - k;
  } else {
    const int f = tid - N_MAIN_WG;
    const int qd = 1 + f / 32, rem = f & 31;
    e = (rem >> 3) & 1; k = (rem & 7) * 2 + (rem >> 4);
    const int h = b ^ (qd & 1), g2 = g ^ (qd >> 1);
    row0 = r * 512 + h * 256 + e * 128; col0 = g2 * 2048 + (a ? 15 - k : k) * 128;
    nrem = 0; tskip = 99; j = 0; rb = 0;
    fin = true;
    const int iA = k >= NS ? 32 + (2 * qd - 2) * 16 + k : -1;
    const int iB = 15 - k >= NS ? 32 + (2 * qd - 1) * 16 + 15 - k : -1;
    if (k < 8) { pi0 = iA >= 0 ? iA : iB; pi1 = iA >= 0 ? iB : -1; }
    else       { pi0 = iB >= 0 ? iB : iA; pi1 = iB >= 0 ? iA : -1; }
  }

  const u32 sbase = (u32)(uintptr_t)smem;
  const int wm = wave >> 1, wn = wave & 1;
  RdAddr A;
  {
    const u32 h = lane >> 5, m = lane & 31u, f = (lane >> 2) & 3u;
    A.xa[0] = sbase + ((u32)wm * 64u + m) * 64u + 16u * (h ^ f);
    A.xa[1] = sbase + ((u32)wm * 64u + m) * 64u + 16u * ((2u + h) ^ f);
    const u32 blk = (lane >> 4) & 1u, q = (lane & 15u) >> 2, pp = lane & 3u;
#pragma unroll
    for (int nt = 0; nt < 2; ++nt)
#pragma unroll
      for (int t = 0; t < 2; ++t) {
        const u32 row = 8u * h + 4u * t + q;
        const u32 ch = (u32)wn * 8u + (u32)nt * 4u + 2u * blk + (pp & 1u);
        A.wa[nt][t] = sbase + 256u * row + 16u * (ch ^ (((row & 3u) << 2) | ((row >> 2) & 3u))) + 8u * (pp >> 1);
      }
  }

  f32x16 acc[2][2];
#pragma unroll
  for (int i = 0; i < 2; ++i)
#pragma unroll
    for (int k = 0; k < 2; ++k)
#pragma unroll
      for (int z = 0; z < 16; ++z) acc[i][k][z] = 0.f;

  const u32 tile_off = (u32)e * 32768u + (u32)wave * 8192u;
  const u32 l16 = lane * 16u, l16b = lane * 16u + 4096u;
  const int nph = nrem ? 2 : 1;
  const u32 tin = (u32)(T0 + EARLY + TAU * max(slot_w(k), slot_x(j, e, j == 7 ? 2 : 7)));
  const int addph = fin ? nph - 1 : 0;

  if (LPT && (bid >> 3) == 25 && wave == 0) {
    sleep_until(tstart, (u32)LPT);
    if (lane < 28u) probe(lane * 2u * MIB + MIB + (u32)(bid & 7) * 128u, myws);
  }
  if (PD0 && (fin || k >= NS)) sleep_until(tstart, (u32)(PD0 + k * PD1 + (fin ? PDF : 0)));
  Loader L;
  L.m0base = sbase + (u32)wave * 4096u;
  bool dead = false;
  u32 dbgv = 0;
  u32 call = 0;
  for (int ph = 0; ph < nph; ++ph) {
    int n, ts;
    if (ph == 0) {
      n = 16; ts = 99;
      if (wave < 2) {
        L.base = P.x + (size_t)((u32)(row0 + wave * 64) * 1024u);
        L.stride = 64u;
        const u32 c = (lane & 3u) ^ ((lane >> 4) & 3u);
#pragma unroll
        for (int k = 0; k < 4; ++k) L.v[k] = (lane >> 2) * 1024u + c * 16u + (u32)k * (16384u - 1024u);
      } else {
        L.base = P.w + (size_t)((u32)(wave - 2) * 16u * 8192u + (u32)col0 * 2u);
        L.stride = 32u * 8192u;
        const u32 q = lane >> 4;
#pragma unroll
        for (int k = 0; k < 4; ++k) L.v[k] = q * 8192u + (((lane & 15u) ^ ((q << 2) | (u32)k)) * 16u) + (u32)k * (4u * 8192u - 1024u);
      }
    } else {
      n = nrem; ts = tskip;
      STAMP(myws, 1);
      if (wave == 0) {
        const int sl = (int)min(lane, 5u);
        const u32 fo = fin_off(sl, max(slot_w(k), slot_x(j, e, sl + 2)));
        bool pend = lane < 6u && sl + 2 != j;
        sleep_until(tstart, tin);
        const u64 t0 = now();
        u32 st = 0;
        for (;;) {
          if (pend) pend = atom_rd(fo, myws) < 16u * call;
          if (__builtin_amdgcn_ballot_w64(pend) == 0ull) break;
          if (give_up(t0, myws, call)) {
            if (lane == 0) st_agt(myws + TMO_OFF, call);
            st = 1;
            break;
          }
          __builtin_amdgcn_s_sleep(PSL);
        }
        if (lane == 0) *(LDSP u32*)(uintptr_t)(sbase + 4u * STAGE_B) = st;
        __builtin_amdgcn_s_waitcnt(0);
      }
      asm volatile("s_barrier" ::: "memory");
      const bool wgdead = lds_rd(sbase + 4u * STAGE_B) != 0;
      STAMP(myws, 2);
      if (wgdead) { dead = true; break; }
      const u32 par = call & 1u;
      L.base = myws + (size_t)(((u32)(wave < 2 ? cont_x(rb) : cont_w(k)) * 2u + par) * PIECE + (u32)(wave & 1) * 4096u);
      L.stride = 8192u;
#pragma unroll
      for (int k = 0; k < 4; ++k) L.v[k] = lane * 16u;
    }
    gemm_phase(acc, A, L, n, ts, ph ? IN_SRC - PIECE : 0u, dbgv, ph * 8);
    if (ph == nph - 1) STAMP(myws, 3);
    if (ph == 0) {
      call = bump_get<0>(cnt);
      if (pi0 >= 0) {
        char* const pz = myws + (size_t)(PIN_OFF + ((call + 2u) % 3u) * PIN_PAR + tile_off);
        sleep_until(tstart, 900u + (u32)((tid >> 3) & 15) * 100u);
        zero8k(l16, l16b, uni(pz + (size_t)((u32)pi0 * PIECE)));
        if (pi1 >= 0) zero8k(l16, l16b, uni(pz + (size_t)((u32)pi1 * PIECE)));
      }
    }
    const u32 pin = PIN_OFF + (call % 3u) * PIN_PAR + tile_off;
    for (int s = 0; s < 2; ++s) {
      const int pi = s == 0 ? pi0 : pi1;
      if (pi < 0 || ph != addph) continue;
      char* const pb = uni(myws + (size_t)(pin + (u32)pi * PIECE));
      const bool first = pi == 0 || (pi >= NOUT1 && (pi & 15) == NS);
      const u32 wo = pin + (u32)(pi - ((!WOWN && (lane & 2u) && !first) ? 1 : 0)) * PIECE + (lane & 1u) * 4096u + 4092u;
      sleep_until(tstart, pi >= NOUT1 ? (u32)(T0 + EARLY + TAU * (NIN - 2 + (pi & 15) - NS)) : (fin ? 0u : tin - 300u));
      const u64 t0 = now();
      while (!dead) {
        u32 v = WOWN ? 1u : 0u;
        if (lane < (WOWN ? 2u : 4u)) v = atom_rd(wo, myws);
        if (WOWN ? __builtin_amdgcn_ballot_w64(v == 0u) == 0ull : __builtin_amdgcn_ballot_w64(v != 0u) != 0ull) break;
        if (give_up(t0, myws, call)) { dead = true; break; }
        __builtin_amdgcn_s_sleep(WSL);
      }
      if (s == 1 || pi1 < 0) STAMP(myws, 4);
      u32x4 pv[8];
      while (!dead) {
        asm volatile("s_nop 4\n\t"
                     "global_load_dwordx4 %0, %8, %10 sc0 sc1\n\t"
                     "global_load_dwordx4 %1, %8, %10 offset:1024 sc0 sc1\n\t"
                     "global_load_dwordx4 %2, %8, %10 offset:2048 sc0 sc1\n\t"
                     "global_load_dwordx4 %3, %8, %10 offset:3072 sc0 sc1\n\t"
                     "global_load_dwordx4 %4, %9, %10 sc0 sc1\n\t"
                     "global_load_dwordx4 %5, %9, %10 offset:1024 sc0 sc1\n\t"
                     "global_load_dwordx4 %6, %9, %10 offset:2048 sc0 sc1\n\t"
                     "global_load_dwordx4 %7, %9, %10 offset:3072 sc0 sc1\n\t"
                     "s_waitcnt vmcnt(0)"
                     : "=&v"(pv[0]), "=&v"(pv[1]), "=&v"(pv[2]), "=&v"(pv[3]), "=&v"(pv[4]), "=&v"(pv[5]), "=&v"(pv[6]), "=&v"(pv[7])
                     : "v"(l16), "v"(l16b), "s"(pb) : "memory");
        u32 mn = 0xffffffffu;
  #pragma unroll
        for (int i = 0; i < 8; ++i)
  #pragma unroll
          for (int z = 0; z < 4; ++z) mn = min(mn, pv[i][z]);
        if (__builtin_amdgcn_ballot_w64(mn == 0u) == 0ull) break;
        if (give_up(t0, myws, call)) dead = true;
        if (RSL) __builtin_amdgcn_s_sleep(RSL);
      }
      if (dead) {
        if (lane == 0) st_agt(myws + TMO_OFF, call);
      } else {
  #pragma unroll
        for (int nt = 0; nt < 2; ++nt)
  #pragma unroll
          for (int mt = 0; mt < 2; ++mt)
  #pragma unroll
            for (int sx = 0; sx < 2; ++sx)
  #pragma unroll
              for (int z = 0; z < 4; ++z) {
                const u32 u = pv[(nt * 2 + mt) * 2 + sx][z];
                acc[nt][mt][8 * sx + 2 * z] += bf_lo(u);
                acc[nt][mt][8 * sx + 2 * z + 1] += bf_hi(u);
              }
      }
    }
  }

  STAMP(myws, 6);
  if (!fin) {
    const bool dir = DIR && j >= 2 && k < NS;
    char* dst = uni(dir ? P.peer + (size_t)(PIN_OFF + (call % 3u) * PIN_PAR + (u32)((j - 2) * NS + k) * PIECE + tile_off)
                        : myws + (size_t)(POUT_OFF + (u32)((j - 1) * 16 + k) * PIECE + tile_off));
    char* fl = myws + (size_t)(FLOC_OFF + (u32)((j - 1) * 8 + e * 4 + wave) * 128u + (u32)k * 4u);
    asm volatile("s_nop 4" ::: "memory");
#pragma unroll
    for (int nt = 0; nt < 2; ++nt)
#pragma unroll
      for (int mt = 0; mt < 2; ++mt) {
        u32x4 v0, v1;
#pragma unroll
        for (int z = 0; z < 4; ++z) {
          const u32 t0_ = pack2(acc[nt][mt][2 * z], acc[nt][mt][2 * z + 1]);
          const u32 t1_ = pack2(acc[nt][mt][8 + 2 * z], acc[nt][mt][8 + 2 * z + 1]);
          v0[z] = dead ? 0x7fc07fc0u : (t0_ ? t0_ : 0x8000u);
          v1[z] = dead ? 0x7fc07fc0u : (t1_ ? t1_ : 0x8000u);
        }
        const u32 o = nt ? l16b : l16;
        if (mt == 0) st16<0>(o, v0, dst); else st16<2048>(o, v0, dst);
        if (dir) __builtin_amdgcn_s_sleep(DSL);
        if (mt == 0) st16<1024>(o, v1, dst); else st16<3072>(o, v1, dst);
        if (dir) __builtin_amdgcn_s_sleep(DSL);
      }
    waitv<0>();
    if (lane == 0) st_sys(fl, call);
  } else {
    const u32 h = lane >> 5, m = lane & 31u;
    const u32 tb = sbase + (u32)wave * 8192u;
#pragma unroll
    for (int nt = 0; nt < 2; ++nt)
#pragma unroll
      for (int mt = 0; mt < 2; ++mt)
#pragma unroll
        for (int sx = 0; sx < 2; ++sx) {
          u32x4 v;
#pragma unroll
          for (int z = 0; z < 4; ++z) {
            const float lo = acc[nt][mt][8 * sx + 2 * z], hi = acc[nt][mt][8 * sx + 2 * z + 1];
            v[z] = dead ? 0x7fc07fc0u : pack2(lo < 0.f ? 0.f : lo, hi < 0.f ? 0.f : hi);
          }
          *(volatile LDSP u32x4*)(uintptr_t)(tb + ((u32)mt * 32u + m) * 128u + ((((u32)(nt * 4 + sx * 2) + h) ^ (m & 7u)) * 16u)) = v;
        }
    char* ob = uni(P.out + (size_t)((u32)(row0 - r * 512 + wm * 64) * 8192u + (u32)(col0 + wn * 64) * 2u));
    const u32 rr = lane >> 3, cc = lane & 7u;
    const u32 ra = tb + rr * 128u + ((cc ^ rr) * 16u);
    u32x4 ov[8];
#pragma unroll
    for (int i = 0; i < 8; ++i) ov[i] = *(volatile LDSP u32x4*)(uintptr_t)(ra + (u32)i * 1024u);
#pragma unroll
    for (int i = 0; i < 8; ++i) OST<0>(((u32)i * 8u + rr) * 8192u + cc * 16u, ov[i], ob);
  }
  STAMP(myws, 5);
  STAMPV(myws, 0, tstart);
}

__global__ __launch_bounds__(256, 2) void fused_gemm_rs(Params P) {
  extern __shared__ __attribute__((aligned(16))) char smem[];
  const u32 lane = threadIdx.x & 63u;
  const int wave = __builtin_amdgcn_readfirstlane(threadIdx.x >> 6);
  const int bid = blockIdx.x;
  const int d = bid < N_PUSH_WG ? 1 + ((bid >> 3) + 3 * (wave >> 1)) % 7 : 1;
  if (CPF && (u32)(bid - CPB) < 16u && wave < 3) {
    u64 pc;
    asm volatile("s_getpc_b64 %0\n\ts_mov_b32 m0, %1\n\ts_nop 3\n\tglobal_load_lds_dword %2, %0"
                 : "=&s"(pc) : "s"((u32)(uintptr_t)smem + (CPB < N_PUSH_WG ? 0u : 66048u)), "v"(min(threadIdx.x * 128u, (u32)CPF - 128u)) : "memory");
  }
  typedef u64 u64x4 __attribute__((ext_vector_type(4)));
  u64x4 a4;
  u64 tstart, pe;
  int rk;
  asm volatile("s_load_dwordx8 %0, %4, 0x0\n\t"
               "s_load_dwordx2 %1, %4, %5 offset:0x28\n\t"
               "s_load_dword %2, %4, 0x20\n\t"
               "s_memrealtime %3\n\t"
               "ds_write_b32 %6, %7\n\t"
               "s_waitcnt lgkmcnt(0)\n\t"
               "s_barrier"
               : "=&s"(a4), "=&s"(pe), "=&s"(rk), "=&s"(tstart)
               : "s"(__builtin_amdgcn_kernarg_segment_ptr()), "s"(d * 8),
                 "v"((u32)(uintptr_t)smem + 65536u + (threadIdx.x & 255u) * 4u), "v"(0u) : "memory");
  u32 t1 = 0;
  const Args A = {(const char*)a4[0], (const char*)a4[1], (char*)a4[2], (char*)a4[3], (char*)pe, rk, t1};
  if (bid < N_PUSH_WG) mover(A, d, wave, lane, smem, tstart);
  else if (bid >= 256 && bid < 256 + N_PUSH_WG) {
    for (int i = 0; i < 6; ++i) __builtin_amdgcn_s_sleep(16);
  } else tile(A, bid < 256 - ROT ? bid - N_PUSH_WG + ROT : (bid < 256 ? bid - (256 - ROT) : bid - 2 * N_PUSH_WG), wave, lane, smem, tstart);
}

}

extern "C" void kernel_launch(int rank, int nranks, ncclComm_t comm,
                              hipStream_t stream,
                              void* const* d_in, const int* in_sizes, int n_in,
                              void* d_out, int out_size,
                              const int* in_dtypes, int out_dtype,
                              void* const* d_out_all,
                              void* const* d_ws_all, size_t ws_bytes) {
  if (nranks != NR || n_in < 2 || ws_bytes < (size_t)WS_NEED || (unsigned)rank >= (unsigned)NR) {
    fprintf(stderr, "unsupported configuration\n");
    return;
  }
  Params P;
  P.x = (const char*)d_in[0];
  P.w = (const char*)d_in[1];
  P.out = (char*)d_out;
  P.my = (char*)d_ws_all[rank];
  for (int i = 0; i < NR; ++i) P.peer[i] = (char*)d_ws_all[rank ^ i];
  P.rank = rank;
  P.pad = 0;
  HIP_CHECK(hipFuncSetAttribute((const void*)fused_gemm_rs, hipFuncAttributeMaxDynamicSharedMemorySize, (int)LDS_BYTES));
  hipLaunchKernelGGL(fused_gemm_rs, dim3(N_WG), dim3(256), LDS_BYTES, stream, P);
  HIP_CHECK(hipGetLastError());
}
```

```cpp
#include <hip/hip_runtime.h>
#include <rccl/rccl.h>
#include <cstdio>
#include <cstdint>

#define HIP_CHECK(expr) do { hipError_t _e = (expr); if (_e != hipSuccess) { \
    fprintf(stderr, "HIP error %s at %s:%d\n", hipGetErrorString(_e), __FILE__, __LINE__); \
    return; } } while (0)

namespace {

typedef unsigned u32;
typedef unsigned long long u64;
typedef u32 u32x4 __attribute__((ext_vector_type(4)));
typedef float f32x2 __attribute__((ext_vector_type(2)));
typedef float f32x16 __attribute__((ext_vector_type(16)));
typedef __bf16 bf16x2 __attribute__((ext_vector_type(2)));
typedef __bf16 bf16x8 __attribute__((ext_vector_type(8)));
typedef short s16x4 __attribute__((ext_vector_type(4)));
typedef short s16x8 __attribute__((ext_vector_type(8)));
#define LDSP __attribute__((address_space(3)))
#define GLBP __attribute__((address_space(1)))

constexpr int NR = 8;
constexpr u32 PIECE = 65536u;
constexpr u32 MIB = 1u << 20;
#define NS 2
#define OX 14
#define TAU 112
#define T0 20
#define QA 0
#define DA 0
#define QB 20
#define DB 1
#define TQ(q) (T0 + TAU * (q) - DA * ((q) < QA ? (q) : QA) - DB * ((q) > QB ? (q) - QB : 0))
#define EARLY 150
#define TAU1 110
#define TAU2 0
#define STGD 2
#define PSL 4
#define LSL 0
#define WSL 3
#define RSL 0
#define WOWN 0
#define ALN 1
#define ALO 99
#define ALT 25
#define PWN 1
#define DIR 0
#define DSL 4
#define SWP 14
#define CPF 20992
#define CPB 56
#define PD0 0
#define PD1 0
#define PDF 0
#define GSH 0
#define LPT 400
#define ROT 48
#define PWIN 64
#define PWHO 1
#define PQ1 9
#define PQ2 14
#define PROT 0
#define XR 32
#define ZT0 900
#define ZDT 100
#define RPT 400
#define RPB 25
#define XCT 0
#define L1T 500
#define SXT 6
#define PRE 1
#define G1 0
#define LAG 1
#define WIN 3
#define GLP "sc0 sc1"
#define MLP "sc0 sc1"
constexpr int NIN = 30;
constexpr int NOUT = 16 - NS;
constexpr int NOUT1 = 16 + 6 * NS;
constexpr u32 IN_SRC = 4u * MIB;
constexpr u32 PIN_OFF = 24u * MIB;
constexpr u32 PIN_PAR = 8u * MIB;
constexpr u32 POUT_OFF = 48u * MIB;
constexpr u32 FLG_OFF = 55u * MIB;
constexpr u32 FLOC_OFF = FLG_OFF + 16384u;
constexpr u32 TMO_OFF = FLG_OFF + 49152u;
constexpr u32 DUMMY_OFF = FLG_OFF + 65536u;
constexpr u32 DBG_OFF = FLG_OFF + 131072u;
constexpr u32 CNT_OFF = FLG_OFF + MIB + 262144u;
constexpr u32 WS_NEED = FLG_OFF + 2u * MIB;

constexpr int N_PUSH_WG = 56;
constexpr int N_MAIN_WG = 256;
constexpr int N_FIN_WG = 96;
constexpr int N_WG = 2 * N_PUSH_WG + N_MAIN_WG + N_FIN_WG;
constexpr u32 STAGE_B = 16384u;
constexpr u32 LDS_BYTES = 4u * STAGE_B + 1024u;
constexpr u64 DEADLINE = 100000000ull;
constexpr u32 DEAD = 0x7fffffffu;

struct Params {
  const char* x;
  const char* w;
  char* my;
  char* out;
  int rank;
  int pad;
  char* peer[NR];
};
struct Args { const char* x; const char* w; char* my; char* out; char* peer; int rank; u32 t1; };

#define SB() __builtin_amdgcn_sched_barrier(0)
template <int N> __device__ __forceinline__ void waitv() { asm volatile("s_waitcnt vmcnt(%0)" :: "i"(N) : "memory"); }

__device__ __forceinline__ u64 now() { return __builtin_amdgcn_s_memrealtime(); }
template <typename T> __device__ __forceinline__ T* uni(T* p) {
  const u64 v = (u64)(uintptr_t)p;
  const u32 lo = __builtin_amdgcn_readfirstlane((u32)v), hi = __builtin_amdgcn_readfirstlane((u32)(v >> 32));
  return (T*)(uintptr_t)(((u64)hi << 32) | lo);
}
__device__ __forceinline__ u32 ld_sys(const void* p) {
  return __hip_atomic_load((const GLBP u32*)(uintptr_t)p, __ATOMIC_RELAXED, __HIP_MEMORY_SCOPE_SYSTEM);
}
__device__ __forceinline__ void st_sys(void* p, u32 v) {
  __hip_atomic_store((GLBP u32*)(uintptr_t)p, v, __ATOMIC_RELAXED, __HIP_MEMORY_SCOPE_SYSTEM);
}
__device__ __forceinline__ void st_agt(void* p, u32 v) {
  __hip_atomic_store((GLBP u32*)(uintptr_t)p, v, __ATOMIC_RELAXED, __HIP_MEMORY_SCOPE_AGENT);
}
__device__ __forceinline__ bool give_up(u64 t0, const char* ws, u32 call) {
  const u64 dt = now() - t0;
  if (!__builtin_amdgcn_readfirstlane((int)(dt > 50000ull))) return false;
  __builtin_amdgcn_s_sleep(64);
  if (__builtin_amdgcn_readfirstlane((int)(dt > DEADLINE))) return true;
  return __builtin_amdgcn_readfirstlane(ld_sys(ws + TMO_OFF)) == call;
}
__device__ __forceinline__ void sleep_until(u64 tstart, u32 t) {
  for (;;) {
    const u32 el = (u32)__builtin_amdgcn_readfirstlane((u32)(now() - tstart));
    if (el >= t) break;
    if (t - el > 40u) __builtin_amdgcn_s_sleep(12); else __builtin_amdgcn_s_sleep(1);
  }
}
__device__ __forceinline__ void sleep_exact(u64 tstart, u32 t) {
  for (;;) {
    const u32 el = (u32)__builtin_amdgcn_readfirstlane((u32)(now() - tstart));
    if (el >= t) break;
    if (t - el > 60u) __builtin_amdgcn_s_sleep(12);
    else if (t - el > 20u) __builtin_amdgcn_s_sleep(3);
    else if (t - el > (u32)SXT) __builtin_amdgcn_s_sleep(1);
  }
}
template <int IMM> __device__ __forceinline__ void st16(u32 off, u32x4 v, char* base) {
  asm volatile("global_store_dwordx4 %0, %1, %2 offset:%3 sc0 sc1\n\ts_nop 1" :: "v"(off), "v"(v), "s"(base), "i"(IMM) : "memory");
}
template <int IMM> __device__ __forceinline__ void st16p(u32 off, u32x4 v, char* base) {
  asm volatile("global_store_dwordx4 %0, %1, %2 offset:%3\n\ts_nop 1" :: "v"(off), "v"(v), "s"(base), "i"(IMM) : "memory");
}
#define OST st16
__device__ __forceinline__ void dma16x4(u32 lds, const u32 (&o)[4], const char* base) {
  asm volatile("s_mov_b32 m0, %0\n\ts_nop 3\n\t"
               "global_load_lds_dwordx4 %1, %5 " MLP "\n\t"
               "global_load_lds_dwordx4 %2, %5 offset:1024 " MLP "\n\t"
               "global_load_lds_dwordx4 %3, %5 offset:2048 " MLP "\n\t"
               "global_load_lds_dwordx4 %4, %5 offset:3072 " MLP
               :: "s"(lds), "v"(o[0]), "v"(o[1]), "v"(o[2]), "v"(o[3]), "s"(base) : "memory");
}
__device__ __forceinline__ void probe(u32 off, char* base) {
  asm volatile("s_nop 4\n\tglobal_atomic_add %0, %1, %2 sc1" :: "v"(off), "v"(0u), "s"(base) : "memory");
}
__device__ __forceinline__ void probe0(u32 off, char* base) {
  asm volatile("s_mov_b64 exec, 1\n\ts_nop 3\n\tglobal_atomic_add %0, %1, %2 sc1\n\ts_mov_b64 exec, -1"
               :: "v"(off), "v"(0u), "s"(base) : "memory");
}
__device__ __forceinline__ u32 atom_rd(u32 off, const char* base) {
  u32 r;
  asm volatile("global_atomic_add %0, %1, %2, %3 sc0 sc1\n\ts_waitcnt vmcnt(0)" : "=&v"(r) : "v"(off), "v"(0u), "s"(base) : "memory");
  return r;
}
__device__ __forceinline__ u32 bump_issue(u32 off, const char* base) {
  u32 r;
  asm volatile("s_mov_b64 exec, 1\n\ts_nop 3\n\tglobal_atomic_add %0, %1, %2, %3 sc0 sc1\n\ts_mov_b64 exec, -1"
               : "=&v"(r) : "v"(off), "v"(1u), "s"(base) : "memory");
  return r;
}
template <int N> __device__ __forceinline__ u32 bump_get(u32 r) {
  u32 v;
  asm volatile("s_waitcnt vmcnt(%2)\n\tv_readfirstlane_b32 %0, %1\n\ts_nop 4" : "=s"(v) : "v"(r), "i"(N) : "memory");
  return v + 1u;
}
__device__ __forceinline__ void add1_lane0(u32 off, char* base) {
  asm volatile("s_mov_b64 exec, 1\n\ts_nop 3\n\tglobal_atomic_add %0, %1, %2 sc1\n\ts_mov_b64 exec, -1"
               :: "v"(off), "v"(1u), "s"(base) : "memory");
}
__device__ __forceinline__ void st4_lane0(u32 off, u32 val, char* base) {
  asm volatile("s_mov_b64 exec, 1\n\ts_nop 3\n\tglobal_store_dword %0, %1, %2 sc0 sc1\n\ts_mov_b64 exec, -1"
               :: "v"(off), "v"(val), "s"(base) : "memory");
}

#define STAMP(ws, idx) do {} while (0)
#define STAMPV(ws, idx, v) do {} while (0)

__device__ __forceinline__ void piece_of(int q, int d, bool& isx, int& idx) {
  const int t = q - NS, u = q - NS - 12;
  int j = (t >> 1) + 2;
  j += j >= d ? 1 : 0;
  j = j == 8 ? 1 : j;
  isx = (q >= NS && u < 0) || (u >= OX && u < OX + 2);
  idx = q < NS ? q : (u < 0 ? 2 * j + (t & 1) : (u < OX ? NS + u : (u < OX + 2 ? u - OX : NS + u - 2)));
}
__device__ __forceinline__ int slot_w(int k) { return k < NS ? k : (k - NS < OX ? 12 + k : 14 + k); }
__device__ __forceinline__ int slot_x(int j, int e, int d) {
  return j == 0 ? NS + 12 + OX + e : NS + 2 * (j == 1 ? 5 : j - 2 - (j > d ? 1 : 0)) + e;
}
__device__ __forceinline__ int cont_w(int k) { return k < NS ? k : 14 + k; }
__device__ __forceinline__ int cont_x(int rb) { return rb < 2 ? 30 + rb : (rb < 4 ? NS + 10 + rb : NS - 4 + rb); }
__device__ __forceinline__ u32 fin_off(int sl, int q) { return (u32)sl * IN_SRC + (u32)(NS + 2 * sl) * 2u * PIECE + (u32)q * 64u; }
__device__ __forceinline__ u32 lds_rd(u32 a) { return __builtin_amdgcn_readfirstlane(*(volatile LDSP u32*)(uintptr_t)a); }
__device__ __forceinline__ void lds_wr(u32 a, u32 v) { *(volatile LDSP u32*)(uintptr_t)a = v; }

__device__ __forceinline__ void mover(const Args& P, int d, int wave, u32 lane, char* smem, u64 tstart) {
  __builtin_amdgcn_s_setprio(3);
  const int s = P.rank;
  char* const myws = P.my;
  const int pair = wave >> 1;
  const int p = ((blockIdx.x + PROT) & 7) * 2 + pair;
  const u32 r = (u32)(s ^ d) & (u32)(NR - 1);
  const int nin = d == 1 ? 0 : NIN;
  const int np = d == 1 ? (DIR ? 16 : NOUT1) : NIN + NOUT;
  const int nid = d == 1 ? NOUT1 : NOUT;
  const u32 ring = (u32)(uintptr_t)smem + (u32)pair * 32768u;
  const u32 ctl = (u32)(uintptr_t)smem + 65536u + (u32)pair * 512u;
  const char* const xin = P.x;
  const char* const win = P.w;
  const int rg = r >> 2, rb_ = (r >> 1) & 1, ra = r & 1;
  u32 wo[4];
#pragma unroll
  for (int k = 0; k < 4; ++k) {
    const u32 q4 = lane >> 4;
    wo[k] = ((u32)p * 16u + (u32)k * 4u + q4) * 8192u + (((lane & 15u) ^ ((q4 << 2) | (u32)k)) * 16u) - (u32)k * 1024u;
  }
  constexpr u32 XC = 256u / XR;
  const u32 xrow = ((u32)p % (128u / XR)) * XR, xks = (u32)p / (128u / XR) * (XC / 4u);
#define W_BASE(IX) (win + (size_t)((u32)ra * 256u * 8192u + (u32)rg * 4096u + (u32)(ra ? 15 - (IX) : (IX)) * 256u))

  if ((wave & 1) == 0) {
    const u32 cnt = bump_issue(CNT_OFF + (u32)(blockIdx.x * 4 + wave) * 64u, myws);
    u32 xo[4], oo[4];
    {
#pragma unroll
      for (int k = 0; k < 4; ++k) {
        const u32 c = (u32)k * 64u + lane;
        xo[k] = (xrow + c / XC) * 1024u + xks * 64u + (c % XC) * 16u - (u32)k * 1024u;
        oo[k] = (u32)p * 4096u + lane * 16u;
      }
    }
    u32 poll_off;
    {
      const int i0 = (int)lane < nid ? (int)lane : 0;
      const bool sw = d == 1 && i0 < SWP;
      const int i = sw ? (i0 & ~1) | (p >> 3) : i0, ps = sw ? ((i0 & 1) << 3) | (p & 7) : p;
      const int jj = d != 1 ? d : (i < 6 * NS ? 2 + i / NS : 1);
      const int kk = d != 1 ? NS + i : (i < 6 * NS ? i % NS : i - 6 * NS);
      poll_off = FLOC_OFF + (u32)((jj - 1) * 8 + (ps >> 1)) * 128u + (u32)kk * 4u;
    }
#define ISSUE_LOADS(QQ, II) do { \
      const int q_ = (QQ); \
      const u32 dst_ = ring + (u32)(q_ & 7) * 4096u; \
      if (q_ < nin) { \
        bool isx_; int ix_; \
        piece_of(q_, d, isx_, ix_); \
        if (isx_) dma16x4(dst_, xo, xin + (size_t)(((r ^ (u32)(ix_ >> 1)) * 512u + (u32)rb_ * 256u + (u32)(ix_ & 1) * 128u) * 1024u + (u32)ra * 512u)); \
        else      dma16x4(dst_, wo, W_BASE(ix_)); \
      } else { \
        const bool sw_ = d == 1 && (II) < SWP; \
        const int i_ = sw_ ? ((II) & ~1) | (p >> 3) : (II); \
        const int jj_ = d != 1 ? d : (i_ < 6 * NS ? 2 + i_ / NS : 1); \
        const int kk_ = d != 1 ? NS + i_ : (i_ < 6 * NS ? i_ % NS : i_ - 6 * NS); \
        dma16x4(dst_, oo, myws + (size_t)(POUT_OFF + (u32)((jj_ - 1) * 16 + kk_) * PIECE + (u32)(sw_ ? (((II) & 1) - (p >> 3)) * 32768 : 0))); \
      } \
    } while (0)

    int q0 = 0;
    u32 pub = 0, call;
    u32 dbg1 = 0, dbg2 = 0;
    if (nin) {
      ISSUE_LOADS(1, 0); ISSUE_LOADS(2, 0); ISSUE_LOADS(3, 0);
    }
    if (nin) {
      call = bump_get<8>(cnt);
      lds_wr(ctl, 2u);
      q0 = 4; pub = 2;
    } else {
      call = bump_get<0>(cnt);
    }

    if (!nin) sleep_until(tstart, (u32)(TQ(NS + (DIR ? 12 : 2)) + L1T));
    u64 ready = 0;
    u64 todo = ((1ull << (np - nin)) - 1ull) << (nid - (np - nin));
    u32 cons = 0;
    bool dead = false;
    for (int q = q0; q < np; ++q) {
      bool rdy = q < nin || (ready & todo) != 0ull;
      if (cons + 8u <= (u32)q) cons = lds_rd(ctl + 4u);
      if (!rdy || cons + 8u <= (u32)q) {
        waitv<0>();
        pub = (u32)q;
        lds_wr(ctl, pub);
        const u64 t0 = now();
        u32 win = 64u;
        for (;;) {
          if (!rdy) {
            u32 pv = 0;
            const u32 below = __builtin_amdgcn_mbcnt_hi((u32)(todo >> 32), __builtin_amdgcn_mbcnt_lo((u32)todo, 0u));
            if (((todo >> lane) & 1ull) && below < win) pv = atom_rd(poll_off, myws);
            win = (u32)PWIN;
            ready |= __builtin_amdgcn_ballot_w64(pv >= call);
            rdy = (ready & todo) != 0ull;
            if (LSL && !rdy) __builtin_amdgcn_s_sleep(LSL);
          } else {
            __builtin_amdgcn_s_sleep(2);
          }
          cons = lds_rd(ctl + 4u);
          if (rdy && cons + 8u > (u32)q) break;
          if (give_up(t0, myws, call)) { dead = true; break; }
        }
        if (dead) break;
      }
      int id = 0;
      if (q >= nin) {
        id = __builtin_ctzll(ready & todo);
        todo &= ~(1ull << id);
        lds_wr(ctl + 32u + (u32)(q & 7) * 4u, (u32)id);
      }
      ISSUE_LOADS(q, id);
      waitv<8>();
      if (PWHO == 1 && nin && q == PQ1) probe0((u32)(d - 2) * IN_SRC + 2u * MIB + (u32)p * 4096u, P.peer);
      if (PWHO == 1 && nin && q == PQ2) probe0(PIN_OFF + (call % 3u) * PIN_PAR + (u32)(32 + (d - 2) * 16 + NS) * PIECE + (u32)p * 4096u, P.peer);
      if (q - 1 > (int)pub) { pub = (u32)(q - 1); lds_wr(ctl, pub); }
      if (q == nin - 1) STAMP(myws, 1);
    }
    waitv<0>();
    if (dead) st4_lane0(TMO_OFF, call, myws);
    lds_wr(ctl, dead ? DEAD : (u32)np);
    STAMP(myws, 3);
    STAMPV(myws, 0, tstart);
    return;
  }

  char* const rws = P.peer;
  u32x4 f0, f1, f2, f3;
  asm volatile("s_nop 4\n\t"
               "global_load_dwordx4 %0, %4, %8 " MLP "\n\t"
               "global_load_dwordx4 %1, %5, %8 offset:1024 " MLP "\n\t"
               "global_load_dwordx4 %2, %6, %8 offset:2048 " MLP "\n\t"
               "global_load_dwordx4 %3, %7, %8 offset:3072 " MLP
               : "=&v"(f0), "=&v"(f1), "=&v"(f2), "=&v"(f3)
               : "v"(wo[0]), "v"(wo[1]), "v"(wo[2]), "v"(wo[3]), "s"(W_BASE(0)) : "memory");
  const u32 cnt = bump_issue(CNT_OFF + (u32)(blockIdx.x * 4 + wave) * 64u, myws);
  const int slot = d - 2;
  if (lane < (u32)PWN) probe(fin_off(max(slot, 0), 64 + p * 2) + lane * 2u * MIB, rws);
  u32 dbg1 = 0, dbg2 = 0;
  u32 tgrid = (u32)(T0 + TAU - 1) - ((u32)__builtin_amdgcn_readfirstlane((u32)tstart) + (u32)(T0 + TAU - 1) - ((u32)(((p + STGD * d) & 15) * TAU) >> 4)) % (u32)TAU;
  const u32 call = bump_get<1>(cnt);
  asm volatile("" : "+v"(f0), "+v"(f1), "+v"(f2), "+v"(f3));
  const u32 in0 = (u32)slot * IN_SRC + (call & 1u) * PIECE;
  const u32 dl = (u32)p * 4096u + lane * 16u;
  u32 xl[4], xd[4];
#pragma unroll
  for (int k = 0; k < 4; ++k) {
    const u32 c = (u32)k * 64u + lane, t = c / (4u * XR), row = (c >> 2) % XR;
    xl[k] = (row * XC + t * 4u + ((c ^ ((xrow + row) >> 2)) & 3u)) * 16u;
    xd[k] = XCT ? dl : (xks + t) * 8192u + (xrow + row) * 64u + (c & 3u) * 16u - (u32)k * 1024u;
  }
  const u32 pin0 = PIN_OFF + (call % 3u) * PIN_PAR + (u32)(d == 1 ? 0 : 32 + slot * 16 + NS) * PIECE;
  int nflag = 0;
#define FLAG(QQ) do { \
    const int q_ = (QQ); \
    const bool real_ = q_ == nflag && q_ < nin; \
    const u32 o_ = real_ ? fin_off(slot, q_) : DUMMY_OFF + (u32)(blockIdx.x * 4 + wave) * 128u; \
    add1_lane0(o_, uni(real_ ? rws : myws)); \
    nflag += real_ ? 1 : 0; \
  } while (0)

#define SLIP() do { \
    const u32 el_ = (u32)__builtin_amdgcn_readfirstlane((u32)(now() - tstart)); \
    if (el_ > tgrid + (u32)ALT) tgrid += (el_ - tgrid - (u32)ALT + (u32)TAU - 1u) / (u32)TAU * (u32)TAU; \
    sleep_exact(tstart, tgrid); \
  } while (0)
  if (nin) {
    char* const dst = rws + (size_t)in0;
    SLIP();
    st16<0>(dl, f0, dst);
    st16<1024>(dl, f1, dst);
    st16<2048>(dl, f2, dst);
    st16<3072>(dl, f3, dst);
    FLAG(-1);
    tgrid += (u32)(TQ(1) - TQ(0) - GSH);
  } else {
    sleep_until(tstart, 600u);
    probe0(pin0 + (u32)p * 4096u, rws);
    probe0(pin0 + (u32)(NOUT1 - 1) * PIECE + (u32)p * 4096u, rws);
  }
  u32 full = 0, tnext = 0;
  for (int q = nin ? 1 : 0; q < np; ++q) {
    bool isx; int ix;
    piece_of(q, d, isx, ix);
    isx = isx && q < nin;
    if (d != 1 || G1) tnext = max(tnext, tgrid);
    if (!PRE) sleep_until(tstart, tnext);
    if (full <= (u32)q) {
      full = lds_rd(ctl);
      if (full <= (u32)q) {
        bool drained = !(q > 0 && q < nin + LAG);
        u64 el;
        for (;;) {
          __builtin_amdgcn_s_sleep(1);
          full = lds_rd(ctl);
          el = now() - tstart;
          if (full > (u32)q) break;
          if (!drained && __builtin_amdgcn_readfirstlane((int)(el > (u64)(tnext + 30u)))) {
            drained = true;
            waitv<0>();
#pragma unroll
            for (int z = LAG; z >= 1; --z) FLAG(q - z);
          }
          if (__builtin_amdgcn_readfirstlane((int)(el > 2ull * DEADLINE - 1000ull))) { full = DEAD; break; }
        }
        tnext = max(tnext, (u32)__builtin_amdgcn_readfirstlane((u32)el));
      }
      if (full == DEAD) break;
    }
    const u32 la = ring + (u32)(q & 7) * 4096u;
    const u32x4 d0 = *(volatile LDSP u32x4*)(uintptr_t)(la + (isx ? xl[0] : lane * 16u));
    const u32x4 d1 = *(volatile LDSP u32x4*)(uintptr_t)(la + (isx ? xl[1] : lane * 16u + 1024u));
    const u32x4 d2 = *(volatile LDSP u32x4*)(uintptr_t)(la + (isx ? xl[2] : lane * 16u + 2048u));
    const u32x4 d3 = *(volatile LDSP u32x4*)(uintptr_t)(la + (isx ? xl[3] : lane * 16u + 3072u));
    const u32 id = lds_rd(ctl + 32u + (u32)(q & 7) * 4u);
    lds_wr(ctl + 4u, (u32)q + 1u);
    u32 o;
    if (q < nin) {
      o = in0 + (u32)(isx ? cont_x(ix) : cont_w(ix)) * 2u * PIECE;
    } else {
      o = pin0 + min(id, (u32)(nid - 1)) * PIECE;
      if (d == 1 && id < (u32)SWP) o += (u32)(((p >> 3) - (int)(id & 1u)) * 32768);
    }
    char* const dst = rws + (size_t)o;
    if (PRE) sleep_exact(tstart, tnext);
    tgrid += (u32)(TAU - (q < QA ? DA : 0) - (q >= QB ? DB : 0));
    tnext += (u32)(d == 1 && !G1 ? TAU1 : TAU2);
    st16<0>(isx ? xd[0] : dl, d0, dst);
    st16<1024>(isx ? xd[1] : dl, d1, dst);
    st16<2048>(isx ? xd[2] : dl, d2, dst);
    st16<3072>(isx ? xd[3] : dl, d3, dst);
    if (nin && q < nin + LAG) {
      waitv<5 * LAG>();
      FLAG(q - LAG);
      if (PWHO == 0 && q == PQ1) probe0((u32)slot * IN_SRC + 2u * MIB + (u32)p * 4096u, rws);
      if (PWHO == 0 && q == PQ2) probe0(pin0 + (u32)p * 4096u, rws);
    } else {
      waitv<4 * WIN>();
    }
    if (q == NS + 11) STAMP(myws, 1);
    if (q == nin - 1) STAMP(myws, 2);
    if (q == np - 1) STAMP(myws, 4);
  }
  waitv<0>();
  STAMP(myws, 3);
  STAMPV(myws, 0, tstart);
}

struct Frag {
  bf16x8 x[2][2];
  bf16x8 w[2][2];
};

struct Loader {
  const char* base;
  u32 stride[2];
  u32 v[4];
  u32 m0base;
};

template <int STAGE> __device__ __forceinline__ void glds_pair(const Loader& L, int half) {
  const u32 m0v = L.m0base + STAGE * STAGE_B;
  if (half == 0)
    asm volatile("s_mov_b32 m0, %0\n\ts_nop 0\n\t"
                 "global_load_lds_dwordx4 %1, %3 " GLP "\n\t"
                 "global_load_lds_dwordx4 %2, %3 offset:1024 " GLP
                 :: "s"(m0v), "v"(L.v[0]), "v"(L.v[1]), "s"(L.base) : "memory");
  else
    asm volatile("s_mov_b32 m0, %0\n\ts_nop 0\n\t"
                 "global_load_lds_dwordx4 %1, %3 offset:2048 " GLP "\n\t"
                 "global_load_lds_dwordx4 %2, %3 offset:3072 " GLP
                 :: "s"(m0v), "v"(L.v[2]), "v"(L.v[3]), "s"(L.base) : "memory");
}

struct RdAddr { u32 xa[2]; u32 wa[2][2]; };

template <int STAGE> __device__ __forceinline__ bf16x8 rd_x(const RdAddr& A, int mt, int ks) {
  return *(const LDSP bf16x8*)(uintptr_t)(A.xa[ks] + (u32)(STAGE * STAGE_B + mt * 2048));
}
template <int STAGE> __device__ __forceinline__ bf16x8 rd_w(const RdAddr& A, int nt, int ks) {
  s16x4 lo = __builtin_amdgcn_ds_read_tr16_b64_v4i16((LDSP s16x4*)(uintptr_t)(A.wa[nt][0] + (u32)(STAGE * STAGE_B + 8192 + ks * 4096)));
  s16x4 hi = __builtin_amdgcn_ds_read_tr16_b64_v4i16((LDSP s16x4*)(uintptr_t)(A.wa[nt][1] + (u32)(STAGE * STAGE_B + 8192 + ks * 4096)));
  s16x8 v = __builtin_shufflevector(lo, hi, 0, 1, 2, 3, 4, 5, 6, 7);
  return __builtin_bit_cast(bf16x8, v);
}
#define MFMA(nt, mt, ks) acc[nt][mt] = __builtin_amdgcn_mfma_f32_32x32x16_bf16(cur.w[nt][ks], cur.x[mt][ks], acc[nt][mt], 0, 0, 0)

template <int S, int VM, bool LOAD, bool READ>
__device__ __forceinline__ void kstep(f32x16 (&acc)[2][2], const Frag& cur, Frag& nxt, const RdAddr& A, Loader& L) {
  constexpr int SN = (S + 1) & 3;
  if (READ) {
    __builtin_amdgcn_s_waitcnt((VM & 15) | (7 << 4) | (0 << 8) | ((VM >> 4) << 14));
    asm volatile("s_barrier" ::: "memory");
  } else {
    __builtin_amdgcn_s_waitcnt(15 | (7 << 4) | (0 << 8) | (3 << 14));
  }
  SB();
  MFMA(0, 0, 0); SB();
  if (READ) { nxt.w[0][0] = rd_w<SN>(A, 0, 0); nxt.x[0][0] = rd_x<SN>(A, 0, 0); } SB();
  MFMA(0, 1, 0); SB();
  if (READ) { nxt.x[1][0] = rd_x<SN>(A, 1, 0); nxt.w[1][0] = rd_w<SN>(A, 1, 0); } SB();
  MFMA(1, 0, 0); SB();
  if (LOAD) glds_pair<S>(L, 0);
  SB();
  MFMA(1, 1, 0); SB();
  if (READ) { nxt.w[0][1] = rd_w<SN>(A, 0, 1); nxt.x[0][1] = rd_x<SN>(A, 0, 1); } SB();
  MFMA(0, 0, 1); SB();
  if (READ) { nxt.x[1][1] = rd_x<SN>(A, 1, 1); nxt.w[1][1] = rd_w<SN>(A, 1, 1); } SB();
  MFMA(0, 1, 1); SB();
  if (LOAD) { glds_pair<S>(L, 1); L.base += L.stride[S & 1]; }
  SB();
  MFMA(1, 0, 1); SB();
  MFMA(1, 1, 1); SB();
}

template <int S> __device__ __forceinline__ void stage_load(Loader& L) {
  glds_pair<S>(L, 0); glds_pair<S>(L, 1); L.base += L.stride[S & 1];
}

__device__ __forceinline__ void gemm_phase(f32x16 (&acc)[2][2], const RdAddr& A, Loader& L, int n, int sk, u32 jump, u32& dbgv, int dbg0) {
  Frag f0, f1;
  L.base = uni(L.base + (sk == 0 ? IN_SRC : 0u));
  L.stride[0] = __builtin_amdgcn_readfirstlane(L.stride[0]);
  L.stride[1] = __builtin_amdgcn_readfirstlane(L.stride[1]);
  asm volatile("s_nop 4" ::: "memory");
  stage_load<0>(L); stage_load<1>(L); stage_load<2>(L); stage_load<3>(L);
  waitv<12>();
  asm volatile("s_barrier" ::: "memory");
#pragma unroll
  for (int ks = 0; ks < 2; ++ks)
#pragma unroll
    for (int i = 0; i < 2; ++i) { f0.w[i][ks] = rd_w<0>(A, i, ks); f0.x[i][ks] = rd_x<0>(A, i, ks); }
  SB();
  for (int t = 0; t < n - 4; t += 4) {
    L.base += (t & 4 ? jump : 0u) + (t + 4 == 8 * sk ? IN_SRC : 0u);
    kstep<0, 8, true, true>(acc, f0, f1, A, L);
    kstep<1, 8, true, true>(acc, f1, f0, A, L);
    kstep<2, 8, true, true>(acc, f0, f1, A, L);
    kstep<3, 8, true, true>(acc, f1, f0, A, L);
  }
  kstep<0, 8, false, true>(acc, f0, f1, A, L);
  kstep<1, 4, false, true>(acc, f1, f0, A, L);
  kstep<2, 0, false, true>(acc, f0, f1, A, L);
  kstep<3, 0, false, false>(acc, f1, f0, A, L);
}

__device__ __forceinline__ u32 pack2(float a, float b) {
  f32x2 t = {a, b};
  return __builtin_bit_cast(u32, __builtin_convertvector(t, bf16x2));
}
__device__ __forceinline__ float bf_lo(u32 v) { return __builtin_bit_cast(float, v << 16); }
__device__ __forceinline__ float bf_hi(u32 v) { return __builtin_bit_cast(float, v & 0xffff0000u); }

__device__ __forceinline__ void zero8k(u32 l16, u32 l16b, char* pb) {
  const u32x4 z4 = {0u, 0u, 0u, 0u};
  asm volatile("s_nop 4" ::: "memory");
  st16<0>(l16, z4, pb); st16<1024>(l16, z4, pb); st16<2048>(l16, z4, pb); st16<3072>(l16, z4, pb);
  st16<0>(l16b, z4, pb); st16<1024>(l16b, z4, pb); st16<2048>(l16b, z4, pb); st16<3072>(l16b, z4, pb);
}

__device__ __forceinline__ void tile(const Args& P, int tid, int wave, u32 lane, char* smem, u64 tstart) {
  const int r = P.rank;
  char* const myws = P.my;
  const int g = r >> 2, b = (r >> 1) & 1, a = r & 1;
  const int bid = blockIdx.x;

  const u32 cnt = bump_issue(CNT_OFF + (u32)(bid * 4 + wave) * 64u, myws);

  int row0, col0, nrem, tskip, j, e, k, rb;
  int pi0 = -1, pi1 = -1;
  bool fin;
  if (tid < N_MAIN_WG) {
    const int xcd = tid & 7, o = tid >> 3, u = xcd >> 1;
    int rbi, cbi;
    if (u < 2) { rbi = o < 24 ? 1 + o % 3 : 0; cbi = o < 24 ? o / 3 : o - 24; }
    else { rbi = o & 3; cbi = o >> 2; }
    rb = u + 4 * rbi; k = cbi * 2 + (xcd & 1);
    j = rb >> 1; e = rb & 1;
    row0 = (r ^ j) * 512 + b * 256 + e * 128; col0 = g * 2048 + (a ? 15 - k : k) * 128;
    nrem = j >= 2 ? 40 : 48; tskip = j >= 2 ? j - 2 : 99;
    fin = j == 0;
    if (j == 0) pi0 = 6 * NS + 15 - k;
    else if (j >= 2 && k >= 16 - NS) pi0 = ((j ^ 1) - 2) * NS + 15 - k;
  } else {
    const int f = tid - N_MAIN_WG;
    const int qd = 1 + f / 32, rem = f & 31;
    e = (rem >> 3) & 1; k = (rem & 7) * 2 + (rem >> 4);
    const int h = b ^ (qd & 1), g2 = g ^ (qd >> 1);
    row0 = r * 512 + h * 256 + e * 128; col0 = g2 * 2048 + (a ? 15 - k : k) * 128;
    nrem = 0; tskip = 99; j = 0; rb = 0;
    fin = true;
    const int iA = k >= NS ? 32 + (2 * qd - 2) * 16 + k : -1;
    const int iB = 15 - k >= NS ? 32 + (2 * qd - 1) * 16 + 15 - k : -1;
    if (k < 8) { pi0 = iA >= 0 ? iA : iB; pi1 = iA >= 0 ? iB : -1; }
    else       { pi0 = iB >= 0 ? iB : iA; pi1 = iB >= 0 ? iA : -1; }
  }

  const u32 sbase = (u32)(uintptr_t)smem;
  const int wm = wave >> 1, wn = wave & 1;
  RdAddr A;
  {
    const u32 h = lane >> 5, m = lane & 31u, f = (lane >> 2) & 3u;
    A.xa[0] = sbase + ((u32)wm * 64u + m) * 64u + 16u * (h ^ f);
    A.xa[1] = sbase + ((u32)wm * 64u + m) * 64u + 16u * ((2u + h) ^ f);
    const u32 blk = (lane >> 4) & 1u, q = (lane & 15u) >> 2, pp = lane & 3u;
#pragma unroll
    for (int nt = 0; nt < 2; ++nt)
#pragma unroll
      for (int t = 0; t < 2; ++t) {
        const u32 row = 8u * h + 4u * t + q;
        const u32 ch = (u32)wn * 8u + (u32)nt * 4u + 2u * blk + (pp & 1u);
        A.wa[nt][t] = sbase + 256u * row + 16u * (ch ^ (((row & 3u) << 2) | ((row >> 2) & 3u))) + 8u * (pp >> 1);
      }
  }

  f32x16 acc[2][2];
#pragma unroll
  for (int i = 0; i < 2; ++i)
#pragma unroll
    for (int k = 0; k < 2; ++k)
#pragma unroll
      for (int z = 0; z < 16; ++z) acc[i][k][z] = 0.f;

  const u32 tile_off = (u32)e * 32768u + (u32)wave * 8192u;
  const u32 l16 = lane * 16u, l16b = lane * 16u + 4096u;
  const int nph = nrem ? 2 : 1;
  const int qin = max(slot_w(k), slot_x(j, e, j == 7 ? 2 : 7));
  const u32 tin = (u32)(EARLY + TQ(qin));
  const int addph = fin ? nph - 1 : 0;

  if (LPT && (bid >> 3) == 25 && wave == 0) {
    sleep_until(tstart, (u32)LPT);
    if (lane < 28u) probe(lane * 2u * MIB + MIB + (u32)(bid & 7) * 128u, myws);
  }
  if (RPT && (bid >> 3) == RPB && wave == 1) {
    const u32 dd = 1u + lane % 7u;
    const u64 pp = ((const GLBP u64*)(uintptr_t)__builtin_amdgcn_kernarg_segment_ptr())[5u + dd];
    const u32 cl = bump_get<0>(cnt);
    const u32 o = (lane < 7u && dd > 1u ? (dd - 2u) * IN_SRC + 2u * MIB : PIN_OFF + (cl % 3u) * PIN_PAR + (dd > 1u ? dd * 16u : 30u) * PIECE) + (u32)(bid & 7) * 128u;
    sleep_until(tstart, (u32)RPT);
    if (lane < 14u) asm volatile("global_atomic_add %0, %1, off sc1" :: "v"(pp + o), "v"(0u) : "memory");
  }
  if (PD0 && (fin || k >= NS)) sleep_until(tstart, (u32)(PD0 + k * PD1 + (fin ? PDF : 0)));
  Loader L;
  L.m0base = sbase + (u32)wave * 4096u;
  bool dead = false;
  u32 dbgv = 0;
  u32 call = 0;
  for (int ph = 0; ph < nph; ++ph) {
    int n, ts;
    if (ph == 0) {
      n = 16; ts = 99;
      if (wave < 2) {
        L.base = P.x + (size_t)((u32)(row0 + wave * 64) * 1024u);
        L.stride[0] = L.stride[1] = 64u;
        const u32 c = (lane & 3u) ^ ((lane >> 4) & 3u);
#pragma unroll
        for (int k = 0; k < 4; ++k) L.v[k] = (lane >> 2) * 1024u + c * 16u + (u32)k * (16384u - 1024u);
      } else {
        L.base = P.w + (size_t)((u32)(wave - 2) * 16u * 8192u + (u32)col0 * 2u);
        L.stride[0] = L.stride[1] = 32u * 8192u;
        const u32 q = lane >> 4;
#pragma unroll
        for (int k = 0; k < 4; ++k) L.v[k] = q * 8192u + (((lane & 15u) ^ ((q << 2) | (u32)k)) * 16u) + (u32)k * (4u * 8192u - 1024u);
      }
    } else {
      n = nrem; ts = tskip;
      STAMP(myws, 1);
      if (wave == 0) {
        const int sl = (int)min(lane, 5u);
        const u32 fo = fin_off(sl, max(slot_w(k), slot_x(j, e, sl + 2)));
        bool pend = lane < 6u && sl + 2 != j;
        sleep_until(tstart, tin);
        const u64 t0 = now();
        u32 st = 0;
        for (;;) {
          if (pend) pend = atom_rd(fo, myws) < 16u * call;
          if (__builtin_amdgcn_ballot_w64(pend) == 0ull) break;
          if (give_up(t0, myws, call)) {
            if (lane == 0) st_agt(myws + TMO_OFF, call);
            st = 1;
            break;
          }
          __builtin_amdgcn_s_sleep(PSL);
        }
        if (lane == 0) *(LDSP u32*)(uintptr_t)(sbase + 4u * STAGE_B) = st;
        __builtin_amdgcn_s_waitcnt(0);
      }
      asm volatile("s_barrier" ::: "memory");
      const bool wgdead = lds_rd(sbase + 4u * STAGE_B) != 0;
      STAMP(myws, 2);
      if (wgdead) { dead = true; break; }
      const u32 par = call & 1u;
      static_assert(!XCT || XR == 32, "layout of the x pieces");
      const bool xc = XCT && wave < 2;
      L.base = myws + (size_t)(((u32)(wave < 2 ? cont_x(rb) : cont_w(k)) * 2u + par) * PIECE + (u32)(wave & 1) * (xc ? 8192u : 4096u));
      L.stride[0] = xc ? 2048u : 8192u;
      L.stride[1] = xc ? 14336u : 8192u;
#pragma unroll
      for (int k = 0; k < 4; ++k) L.v[k] = lane * 16u + (xc && k >= 2 ? 2048u : 0u);
    }
    gemm_phase(acc, A, L, n, ts, ph ? IN_SRC - PIECE : 0u, dbgv, ph * 8);
    if (ph == nph - 1) STAMP(myws, 3);
    if (ph == 0) {
      call = bump_get<0>(cnt);
      if (ZT0 && pi0 >= 0) {
        char* const pz = myws + (size_t)(PIN_OFF + ((call + 2u) % 3u) * PIN_PAR + tile_off);
        sleep_until(tstart, (u32)ZT0 + (u32)((tid >> 3) & 15) * (u32)ZDT);
        zero8k(l16, l16b, uni(pz + (size_t)((u32)pi0 * PIECE)));
        if (pi1 >= 0) zero8k(l16, l16b, uni(pz + (size_t)((u32)pi1 * PIECE)));
      }
    }
    const u32 pin = PIN_OFF + (call % 3u) * PIN_PAR + tile_off;
    for (int s = 0; s < 2; ++s) {
      const int pi = s == 0 ? pi0 : pi1;
      if (pi < 0 || ph != addph) continue;
      char* const pb = uni(myws + (size_t)(pin + (u32)pi * PIECE));
      const bool first = pi == 0 || (pi >= NOUT1 && (pi & 15) == NS);
      const u32 wo = pin + (u32)(pi - ((!WOWN && (lane & 2u) && !first) ? 1 : 0)) * PIECE + (lane & 1u) * 4096u + 4092u;
      sleep_until(tstart, pi >= NOUT1 ? (u32)(EARLY + TQ(NIN - 2 + (pi & 15) - NS)) : (fin ? 0u : tin - 300u));
      const u64 t0 = now();
      while (!dead) {
        u32 v = WOWN ? 1u : 0u;
        if (lane < (WOWN ? 2u : 4u)) v = atom_rd(wo, myws);
        if (WOWN ? __builtin_amdgcn_ballot_w64(v == 0u) == 0ull : __builtin_amdgcn_ballot_w64(v != 0u) != 0ull) break;
        if (give_up(t0, myws, call)) { dead = true; break; }
        __builtin_amdgcn_s_sleep(WSL);
      }
      if (s == 1 || pi1 < 0) STAMP(myws, 4);
      u32x4 pv[8];
      while (!dead) {
        asm volatile("s_nop 4\n\t"
                     "global_load_dwordx4 %0, %8, %10 sc0 sc1\n\t"
                     "global_load_dwordx4 %1, %8, %10 offset:1024 sc0 sc1\n\t"
                     "global_load_dwordx4 %2, %8, %10 offset:2048 sc0 sc1\n\t"
                     "global_load_dwordx4 %3, %8, %10 offset:3072 sc0 sc1\n\t"
                     "global_load_dwordx4 %4, %9, %10 sc0 sc1\n\t"
                     "global_load_dwordx4 %5, %9, %10 offset:1024 sc0 sc1\n\t"
                     "global_load_dwordx4 %6, %9, %10 offset:2048 sc0 sc1\n\t"
                     "global_load_dwordx4 %7, %9, %10 offset:3072 sc0 sc1\n\t"
                     "s_waitcnt vmcnt(0)"
                     : "=&v"(pv[0]), "=&v"(pv[1]), "=&v"(pv[2]), "=&v"(pv[3]), "=&v"(pv[4]), "=&v"(pv[5]), "=&v"(pv[6]), "=&v"(pv[7])
                     : "v"(l16), "v"(l16b), "s"(pb) : "memory");
        u32 mn = 0xffffffffu;
  #pragma unroll
        for (int i = 0; i < 8; ++i)
  #pragma unroll
          for (int z = 0; z < 4; ++z) mn = min(mn, pv[i][z]);
        if (__builtin_amdgcn_ballot_w64(mn == 0u) == 0ull) break;
        if (give_up(t0, myws, call)) dead = true;
        if (RSL) __builtin_amdgcn_s_sleep(RSL);
      }
      if (dead) {
        if (lane == 0) st_agt(myws + TMO_OFF, call);
      } else {
  #pragma unroll
        for (int nt = 0; nt < 2; ++nt)
  #pragma unroll
          for (int mt = 0; mt < 2; ++mt)
  #pragma unroll
            for (int sx = 0; sx < 2; ++sx)
  #pragma unroll
              for (int z = 0; z < 4; ++z) {
                const u32 u = pv[(nt * 2 + mt) * 2 + sx][z];
                acc[nt][mt][8 * sx + 2 * z] += bf_lo(u);
                acc[nt][mt][8 * sx + 2 * z + 1] += bf_hi(u);
              }
      }
    }
  }

  STAMP(myws, 6);
  if (!fin) {
    const bool dir = DIR && j >= 2 && k < NS;
    char* dst = uni(dir ? P.peer + (size_t)(PIN_OFF + (call % 3u) * PIN_PAR + (u32)((j - 2) * NS + k) * PIECE + tile_off)
                        : myws + (size_t)(POUT_OFF + (u32)((j - 1) * 16 + k) * PIECE + tile_off));
    char* fl = myws + (size_t)(FLOC_OFF + (u32)((j - 1) * 8 + e * 4 + wave) * 128u + (u32)k * 4u);
    asm volatile("s_nop 4" ::: "memory");
#pragma unroll
    for (int nt = 0; nt < 2; ++nt)
#pragma unroll
      for (int mt = 0; mt < 2; ++mt) {
        u32x4 v0, v1;
#pragma unroll
        for (int z = 0; z < 4; ++z) {
          const u32 t0_ = pack2(acc[nt][mt][2 * z], acc[nt][mt][2 * z + 1]);
          const u32 t1_ = pack2(acc[nt][mt][8 + 2 * z], acc[nt][mt][8 + 2 * z + 1]);
          v0[z] = dead ? 0x7fc07fc0u : (t0_ ? t0_ : 0x8000u);
          v1[z] = dead ? 0x7fc07fc0u : (t1_ ? t1_ : 0x8000u);
        }
        const u32 o = nt ? l16b : l16;
        if (mt == 0) st16<0>(o, v0, dst); else st16<2048>(o, v0, dst);
        if (dir) __builtin_amdgcn_s_sleep(DSL);
        if (mt == 0) st16<1024>(o, v1, dst); else st16<3072>(o, v1, dst);
        if (dir) __builtin_amdgcn_s_sleep(DSL);
      }
    waitv<0>();
    if (lane == 0) st_sys(fl, call);
  } else {
    const u32 h = lane >> 5, m = lane & 31u;
    const u32 tb = sbase + (u32)wave * 8192u;
#pragma unroll
    for (int nt = 0; nt < 2; ++nt)
#pragma unroll
      for (int mt = 0; mt < 2; ++mt)
#pragma unroll
        for (int sx = 0; sx < 2; ++sx) {
          u32x4 v;
#pragma unroll
          for (int z = 0; z < 4; ++z) {
            const float lo = acc[nt][mt][8 * sx + 2 * z], hi = acc[nt][mt][8 * sx + 2 * z + 1];
            v[z] = dead ? 0x7fc07fc0u : pack2(lo < 0.f ? 0.f : lo, hi < 0.f ? 0.f : hi);
          }
          *(volatile LDSP u32x4*)(uintptr_t)(tb + ((u32)mt * 32u + m) * 128u + ((((u32)(nt * 4 + sx * 2) + h) ^ (m & 7u)) * 16u)) = v;
        }
    char* ob = uni(P.out + (size_t)((u32)(row0 - r * 512 + wm * 64) * 8192u + (u32)(col0 + wn * 64) * 2u));
    const u32 rr = lane >> 3, cc = lane & 7u;
    const u32 ra = tb + rr * 128u + ((cc ^ rr) * 16u);
    u32x4 ov[8];
#pragma unroll
    for (int i = 0; i < 8; ++i) ov[i] = *(volatile LDSP u32x4*)(uintptr_t)(ra + (u32)i * 1024u);
#pragma unroll
    for (int i = 0; i < 8; ++i) OST<0>(((u32)i * 8u + rr) * 8192u + cc * 16u, ov[i], ob);
  }
  STAMP(myws, 5);
  STAMPV(myws, 0, tstart);
}

__global__ __launch_bounds__(256, 2) void fused_gemm_rs(Params P) {
  extern __shared__ __attribute__((aligned(16))) char smem[];
  const u32 lane = threadIdx.x & 63u;
  const int wave = __builtin_amdgcn_readfirstlane(threadIdx.x >> 6);
  const int bid = blockIdx.x;
  const int d = bid < N_PUSH_WG ? 1 + ((bid >> 3) + 3 * (wave >> 1)) % 7 : 1;
  if (CPF && (u32)(bid - CPB) < 16u && wave < 3) {
    u64 pc;
    asm volatile("s_getpc_b64 %0\n\ts_mov_b32 m0, %1\n\ts_nop 3\n\tglobal_load_lds_dword %2, %0"
                 : "=&s"(pc) : "s"((u32)(uintptr_t)smem + (CPB < N_PUSH_WG ? 0u : 66048u)), "v"(min(threadIdx.x * 128u, (u32)CPF - 128u)) : "memory");
  }
  typedef u64 u64x4 __attribute__((ext_vector_type(4)));
  u64x4 a4;
  u64 tstart, pe;
  int rk;
  asm volatile("s_load_dwordx8 %0, %4, 0x0\n\t"
               "s_load_dwordx2 %1, %4, %5 offset:0x28\n\t"
               "s_load_dword %2, %4, 0x20\n\t"
               "s_memrealtime %3\n\t"
               "ds_write_b32 %6, %7\n\t"
               "s_waitcnt lgkmcnt(0)\n\t"
               "s_barrier"
               : "=&s"(a4), "=&s"(pe), "=&s"(rk), "=&s"(tstart)
               : "s"(__builtin_amdgcn_kernarg_segment_ptr()), "s"(d * 8),
                 "v"((u32)(uintptr_t)smem + 65536u + (threadIdx.x & 255u) * 4u), "v"(0u) : "memory");
  u32 t1 = 0;
  const Args A = {(const char*)a4[0], (const char*)a4[1], (char*)a4[2], (char*)a4[3], (char*)pe, rk, t1};
  if (bid < N_PUSH_WG) mover(A, d, wave, lane, smem, tstart);
  else if (bid >= 256 && bid < 256 + N_PUSH_WG) {
    for (int i = 0; i < 6; ++i) __builtin_amdgcn_s_sleep(16);
  } else tile(A, bid < 256 - ROT ? bid - N_PUSH_WG + ROT : (bid < 256 ? bid - (256 - ROT) : bid - 2 * N_PUSH_WG), wave, lane, smem, tstart);
}

}

extern "C" void kernel_launch(int rank, int nranks, ncclComm_t comm,
                              hipStream_t stream,
                              void* const* d_in, const int* in_sizes, int n_in,
                              void* d_out, int out_size,
                              const int* in_dtypes, int out_dtype,
                              void* const* d_out_all,
                              void* const* d_ws_all, size_t ws_bytes) {
  if (nranks != NR || n_in < 2 || ws_bytes < (size_t)WS_NEED || (unsigned)rank >= (unsigned)NR) {
    fprintf(stderr, "unsupported configuration\n");
    return;
  }
  Params P;
  P.x = (const char*)d_in[0];
  P.w = (const char*)d_in[1];
  P.out = (char*)d_out;
  P.my = (char*)d_ws_all[rank];
  for (int i = 0; i < NR; ++i) P.peer[i] = (char*)d_ws_all[rank ^ i];
  P.rank = rank;
  P.pad = 0;
  HIP_CHECK(hipFuncSetAttribute((const void*)fused_gemm_rs, hipFuncAttributeMaxDynamicSharedMemorySize, (int)LDS_BYTES));
  hipLaunchKernelGGL(fused_gemm_rs, dim3(N_WG), dim3(256), LDS_BYTES, stream, P);
  HIP_CHECK(hipGetLastError());
}
```

```cpp
#include <hip/hip_runtime.h>
#include <rccl/rccl.h>
#include <cstdio>
#include <cstdint>

#define HIP_CHECK(expr) do { hipError_t _e = (expr); if (_e != hipSuccess) { \
    fprintf(stderr, "HIP error %s at %s:%d\n", hipGetErrorString(_e), __FILE__, __LINE__); \
    return; } } while (0)

namespace {

typedef unsigned u32;
typedef unsigned long long u64;
typedef u32 u32x4 __attribute__((ext_vector_type(4)));
typedef float f32x2 __attribute__((ext_vector_type(2)));
typedef float f32x16 __attribute__((ext_vector_type(16)));
typedef __bf16 bf16x2 __attribute__((ext_vector_type(2)));
typedef __bf16 bf16x8 __attribute__((ext_vector_type(8)));
typedef short s16x4 __attribute__((ext_vector_type(4)));
typedef short s16x8 __attribute__((ext_vector_type(8)));
#define LDSP __attribute__((address_space(3)))
#define GLBP __attribute__((address_space(1)))

constexpr int NR = 8;
constexpr u32 PIECE = 65536u;
constexpr u32 MIB = 1u << 20;
#define NS 2
#define OX 14
#define TAU 112
#define T0 20
#define QA 0
#define DA 0
#define QB 20
#define DB 1
#define TQ(q) (T0 + TAU * (q) - DA * ((q) < QA ? (q) : QA) - DB * ((q) > QB ? (q) - QB : 0))
#define EARLY 150
#define TAU1 110
#define TAU2 0
#define STGD 2
#define PSL 4
#define LSL 0
#define WSL 3
#define RSL 0
#define WOWN 0
#define ALN 1
#define ALO 99
#define ALT 25
#define PWN 1
#define DIR 0
#define DSL 4
#define SWP 14
#define CPF 17664
#define CPB 56
#define PD0 0
#define PD1 0
#define PDF 0
#define GSH -18
#define LPT 400
#define ROT 48
#define PWIN 64
#define PWHO 1
#define PQ1 9
#define PQ2 14
#define PROT 0
#define XR 32
#define ZT0 900
#define ZDT 100
#define RPT 400
#define RPB 25
#define XCT 0
#define L1T 500
#define SXT 6
#define PRE 1
#define WMK 1ull
#define G1 0
#define LAG 1
#define WIN 3
#define GLP "sc0 sc1"
#define MLP "sc0 sc1"
constexpr int NIN = 30;
constexpr int NOUT = 16 - NS;
constexpr int NOUT1 = 16 + 6 * NS;
constexpr u32 IN_SRC = 4u * MIB;
constexpr u32 PIN_OFF = 24u * MIB;
constexpr u32 PIN_PAR = 8u * MIB;
constexpr u32 POUT_OFF = 48u * MIB;
constexpr u32 FLG_OFF = 55u * MIB;
constexpr u32 FLOC_OFF = FLG_OFF + 16384u;
constexpr u32 TMO_OFF = FLG_OFF + 49152u;
constexpr u32 DUMMY_OFF = FLG_OFF + 65536u;
constexpr u32 DBG_OFF = FLG_OFF + 131072u;
constexpr u32 CNT_OFF = FLG_OFF + MIB + 262144u;
constexpr u32 WS_NEED = FLG_OFF + 2u * MIB;

constexpr int N_PUSH_WG = 56;
constexpr int N_MAIN_WG = 256;
constexpr int N_FIN_WG = 96;
constexpr int N_WG = 2 * N_PUSH_WG + N_MAIN_WG + N_FIN_WG;
constexpr u32 STAGE_B = 16384u;
constexpr u32 LDS_BYTES = 4u * STAGE_B + 1024u;
constexpr u64 DEADLINE = 100000000ull;
constexpr u32 DEAD = 0x7fffffffu;

struct Params {
  const char* x;
  const char* w;
  char* my;
  char* out;
  int rank;
  int pad;
  char* peer[NR];
};
struct Args { const char* x; const char* w; char* my; char* out; char* peer; int rank; u32 t1; };

#define SB() __builtin_amdgcn_sched_barrier(0)
template <int N> __device__ __forceinline__ void waitv() { asm volatile("s_waitcnt vmcnt(%0)" :: "i"(N) : "memory"); }

__device__ __forceinline__ u64 now() { return __builtin_amdgcn_s_memrealtime(); }
template <typename T> __device__ __forceinline__ T* uni(T* p) {
  const u64 v = (u64)(uintptr_t)p;
  const u32 lo = __builtin_amdgcn_readfirstlane((u32)v), hi = __builtin_amdgcn_readfirstlane((u32)(v >> 32));
  return (T*)(uintptr_t)(((u64)hi << 32) | lo);
}
__device__ __forceinline__ u32 ld_sys(const void* p) {
  return __hip_atomic_load((const GLBP u32*)(uintptr_t)p, __ATOMIC_RELAXED, __HIP_MEMORY_SCOPE_SYSTEM);
}
__device__ __forceinline__ void st_sys(void* p, u32 v) {
  __hip_atomic_store((GLBP u32*)(uintptr_t)p, v, __ATOMIC_RELAXED, __HIP_MEMORY_SCOPE_SYSTEM);
}
__device__ __forceinline__ void st_agt(void* p, u32 v) {
  __hip_atomic_store((GLBP u32*)(uintptr_t)p, v, __ATOMIC_RELAXED, __HIP_MEMORY_SCOPE_AGENT);
}
__device__ __forceinline__ bool give_up(u64 t0, const char* ws, u32 call) {
  const u64 dt = now() - t0;
  if (!__builtin_amdgcn_readfirstlane((int)(dt > 50000ull))) return false;
  __builtin_amdgcn_s_sleep(64);
  if (__builtin_amdgcn_readfirstlane((int)(dt > DEADLINE))) return true;
  return __builtin_amdgcn_readfirstlane(ld_sys(ws + TMO_OFF)) == call;
}
__device__ __forceinline__ void sleep_until(u64 tstart, u32 t) {
  for (;;) {
    const u32 el = (u32)__builtin_amdgcn_readfirstlane((u32)(now() - tstart));
    if (el >= t) break;
    if (t - el > 40u) __builtin_amdgcn_s_sleep(12); else __builtin_amdgcn_s_sleep(1);
  }
}
__device__ __forceinline__ void sleep_exact(u64 tstart, u32 t) {
  for (;;) {
    const u32 el = (u32)__builtin_amdgcn_readfirstlane((u32)(now() - tstart));
    if (el >= t) break;
    if (t - el > 60u) __builtin_amdgcn_s_sleep(12);
    else if (t - el > 20u) __builtin_amdgcn_s_sleep(3);
    else if (t - el > (u32)SXT) __builtin_amdgcn_s_sleep(1);
  }
}
template <int IMM> __device__ __forceinline__ void st16(u32 off, u32x4 v, char* base) {
  asm volatile("global_store_dwordx4 %0, %1, %2 offset:%3 sc0 sc1\n\ts_nop 1" :: "v"(off), "v"(v), "s"(base), "i"(IMM) : "memory");
}
template <int IMM> __device__ __forceinline__ void st16p(u32 off, u32x4 v, char* base) {
  asm volatile("global_store_dwordx4 %0, %1, %2 offset:%3\n\ts_nop 1" :: "v"(off), "v"(v), "s"(base), "i"(IMM) : "memory");
}
#define OST st16
__device__ __forceinline__ void dma16x4(u32 lds, const u32 (&o)[4], const char* base) {
  asm volatile("s_mov_b32 m0, %0\n\ts_nop 3\n\t"
               "global_load_lds_dwordx4 %1, %5 " MLP "\n\t"
               "global_load_lds_dwordx4 %2, %5 offset:1024 " MLP "\n\t"
               "global_load_lds_dwordx4 %3, %5 offset:2048 " MLP "\n\t"
               "global_load_lds_dwordx4 %4, %5 offset:3072 " MLP
               :: "s"(lds), "v"(o[0]), "v"(o[1]), "v"(o[2]), "v"(o[3]), "s"(base) : "memory");
}
__device__ __forceinline__ void probe(u32 off, char* base) {
  asm volatile("s_nop 4\n\tglobal_atomic_add %0, %1, %2 sc1" :: "v"(off), "v"(0u), "s"(base) : "memory");
}
__device__ __forceinline__ void probe0(u32 off, char* base) {
  asm volatile("s_mov_b64 exec, 1\n\ts_nop 3\n\tglobal_atomic_add %0, %1, %2 sc1\n\ts_mov_b64 exec, -1"
               :: "v"(off), "v"(0u), "s"(base) : "memory");
}
__device__ __forceinline__ u32 atom_rd(u32 off, const char* base) {
  u32 r;
  asm volatile("global_atomic_add %0, %1, %2, %3 sc0 sc1\n\ts_waitcnt vmcnt(0)" : "=&v"(r) : "v"(off), "v"(0u), "s"(base) : "memory");
  return r;
}
__device__ __forceinline__ u32 bump_issue(u32 off, const char* base) {
  u32 r;
  asm volatile("s_mov_b64 exec, 1\n\ts_nop 3\n\tglobal_atomic_add %0, %1, %2, %3 sc0 sc1\n\ts_mov_b64 exec, -1"
               : "=&v"(r) : "v"(off), "v"(1u), "s"(base) : "memory");
  return r;
}
template <int N> __device__ __forceinline__ u32 bump_get(u32 r) {
  u32 v;
  asm volatile("s_waitcnt vmcnt(%2)\n\tv_readfirstlane_b32 %0, %1\n\ts_nop 4" : "=s"(v) : "v"(r), "i"(N) : "memory");
  return v + 1u;
}
__device__ __forceinline__ void add1_lane0(u32 off, char* base) {
  asm volatile("s_mov_b64 exec, 1\n\ts_nop 3\n\tglobal_atomic_add %0, %1, %2 sc1\n\ts_mov_b64 exec, -1"
               :: "v"(off), "v"(1u), "s"(base) : "memory");
}
__device__ __forceinline__ void st4_lane0(u32 off, u32 val, char* base) {
  asm volatile("s_mov_b64 exec, 1\n\ts_nop 3\n\tglobal_store_dword %0, %1, %2 sc0 sc1\n\ts_mov_b64 exec, -1"
               :: "v"(off), "v"(val), "s"(base) : "memory");
}

#define STAMP(ws, idx) do {} while (0)
#define STAMPV(ws, idx, v) do {} while (0)

__device__ __forceinline__ void piece_of(int q, int d, bool& isx, int& idx) {
  const int t = q - NS, u = q - NS - 12;
  int j = (t >> 1) + 2;
  j += j >= d ? 1 : 0;
  j = j == 8 ? 1 : j;
  isx = (q >= NS && u < 0) || (u >= OX && u < OX + 2);
  idx = q < NS ? q : (u < 0 ? 2 * j + (t & 1) : (u < OX ? NS + u : (u < OX + 2 ? u - OX : NS + u - 2)));
}
__device__ __forceinline__ int slot_w(int k) { return k < NS ? k : (k - NS < OX ? 12 + k : 14 + k); }
__device__ __forceinline__ int slot_x(int j, int e, int d) {
  return j == 0 ? NS + 12 + OX + e : NS + 2 * (j == 1 ? 5 : j - 2 - (j > d ? 1 : 0)) + e;
}
__device__ __forceinline__ int cont_w(int k) { return k < NS ? k : 14 + k; }
__device__ __forceinline__ int cont_x(int rb) { return rb < 2 ? 30 + rb : (rb < 4 ? NS + 10 + rb : NS - 4 + rb); }
__device__ __forceinline__ u32 fin_off(int sl, int q) { return (u32)sl * IN_SRC + (u32)(NS + 2 * sl) * 2u * PIECE + (u32)q * 64u; }
__device__ __forceinline__ u32 lds_rd(u32 a) { return __builtin_amdgcn_readfirstlane(*(volatile LDSP u32*)(uintptr_t)a); }
__device__ __forceinline__ void lds_wr(u32 a, u32 v) { *(volatile LDSP u32*)(uintptr_t)a = v; }

__device__ __forceinline__ void mover(const Args& P, int d, int wave, u32 lane, char* smem, u64 tstart) {
  __builtin_amdgcn_s_setprio(3);
  const int s = P.rank;
  char* const myws = P.my;
  const int pair = wave >> 1;
  const int p = ((blockIdx.x + PROT) & 7) * 2 + pair;
  const u32 r = (u32)(s ^ d) & (u32)(NR - 1);
  const int nin = d == 1 ? 0 : NIN;
  const int np = d == 1 ? (DIR ? 16 : NOUT1) : NIN + NOUT;
  const int nid = d == 1 ? NOUT1 : NOUT;
  const u32 ring = (u32)(uintptr_t)smem + (u32)pair * 32768u;
  const u32 ctl = (u32)(uintptr_t)smem + 65536u + (u32)pair * 512u;
  const char* const xin = P.x;
  const char* const win = P.w;
  const int rg = r >> 2, rb_ = (r >> 1) & 1, ra = r & 1;
  u32 wo[4];
#pragma unroll
  for (int k = 0; k < 4; ++k) {
    const u32 q4 = lane >> 4;
    wo[k] = ((u32)p * 16u + (u32)k * 4u + q4) * 8192u + (((lane & 15u) ^ ((q4 << 2) | (u32)k)) * 16u) - (u32)k * 1024u;
  }
  constexpr u32 XC = 256u / XR;
  const u32 xrow = ((u32)p % (128u / XR)) * XR, xks = (u32)p / (128u / XR) * (XC / 4u);
#define W_BASE(IX) (win + (size_t)((u32)ra * 256u * 8192u + (u32)rg * 4096u + (u32)(ra ? 15 - (IX) : (IX)) * 256u))

  if ((wave & 1) == 0) {
    const u32 cnt = bump_issue(CNT_OFF + (u32)(blockIdx.x * 4 + wave) * 64u, myws);
    u32 xo[4], oo[4];
    {
#pragma unroll
      for (int k = 0; k < 4; ++k) {
        const u32 c = (u32)k * 64u + lane;
        xo[k] = (xrow + c / XC) * 1024u + xks * 64u + (c % XC) * 16u - (u32)k * 1024u;
        oo[k] = (u32)p * 4096u + lane * 16u;
      }
    }
    u32 poll_off;
    {
      const int i0 = (int)lane < nid ? (int)lane : 0;
      const bool sw = d == 1 && i0 < SWP;
      const int i = sw ? (i0 & ~1) | (p >> 3) : i0, ps = sw ? ((i0 & 1) << 3) | (p & 7) : p;
      const int jj = d != 1 ? d : (i < 6 * NS ? 2 + i / NS : 1);
      const int kk = d != 1 ? NS + i : (i < 6 * NS ? i % NS : i - 6 * NS);
      poll_off = FLOC_OFF + (u32)((jj - 1) * 8 + (ps >> 1)) * 128u + (u32)kk * 4u;
    }
#define ISSUE_LOADS(QQ, II) do { \
      const int q_ = (QQ); \
      const u32 dst_ = ring + (u32)(q_ & 7) * 4096u; \
      if (q_ < nin) { \
        bool isx_; int ix_; \
        piece_of(q_, d, isx_, ix_); \
        if (isx_) dma16x4(dst_, xo, xin + (size_t)(((r ^ (u32)(ix_ >> 1)) * 512u + (u32)rb_ * 256u + (u32)(ix_ & 1) * 128u) * 1024u + (u32)ra * 512u)); \
        else      dma16x4(dst_, wo, W_BASE(ix_)); \
      } else { \
        const bool sw_ = d == 1 && (II) < SWP; \
        const int i_ = sw_ ? ((II) & ~1) | (p >> 3) : (II); \
        const int jj_ = d != 1 ? d : (i_ < 6 * NS ? 2 + i_ / NS : 1); \
        const int kk_ = d != 1 ? NS + i_ : (i_ < 6 * NS ? i_ % NS : i_ - 6 * NS); \
        dma16x4(dst_, oo, myws + (size_t)(POUT_OFF + (u32)((jj_ - 1) * 16 + kk_) * PIECE + (u32)(sw_ ? (((II) & 1) - (p >> 3)) * 32768 : 0))); \
      } \
    } while (0)

    int q0 = 0;
    u32 pub = 0, call;
    u32 dbg1 = 0, dbg2 = 0;
    if (nin) {
      ISSUE_LOADS(1, 0); ISSUE_LOADS(2, 0); ISSUE_LOADS(3, 0);
    }
    if (nin) {
      call = bump_get<8>(cnt);
      lds_wr(ctl, 2u);
      q0 = 4; pub = 2;
    } else {
      call = bump_get<0>(cnt);
    }

    if (!nin) sleep_until(tstart, (u32)(TQ(NS + (DIR ? 12 : 2)) + L1T));
    u64 ready = 0;
    u64 todo = ((1ull << (np - nin)) - 1ull) << (nid - (np - nin));
    u32 cons = 0;
    bool dead = false;
    for (int q = q0; q < np; ++q) {
      bool rdy = q < nin || (ready & todo) != 0ull;
      if (cons + 8u <= (u32)q) cons = lds_rd(ctl + 4u);
      if (!rdy || cons + 8u <= (u32)q) {
        waitv<0>();
        pub = (u32)q;
        lds_wr(ctl, pub);
        const u64 t0 = now();
        u32 win = 64u;
        for (;;) {
          if (!rdy) {
            u32 pv = 0;
            const u32 below = __builtin_amdgcn_mbcnt_hi((u32)(todo >> 32), __builtin_amdgcn_mbcnt_lo((u32)todo, 0u));
            if (((todo >> lane) & 1ull) && below < win) pv = atom_rd(poll_off, myws);
            win = (u32)PWIN;
            ready |= __builtin_amdgcn_ballot_w64(pv >= call);
            rdy = (ready & todo) != 0ull;
            if (LSL && !rdy) __builtin_amdgcn_s_sleep(LSL);
          } else {
            __builtin_amdgcn_s_sleep(2);
          }
          cons = lds_rd(ctl + 4u);
          if (rdy && cons + 8u > (u32)q) break;
          if (give_up(t0, myws, call)) { dead = true; break; }
        }
        if (dead) break;
      }
      int id = 0;
      if (q >= nin) {
        id = __builtin_ctzll(ready & todo);
        todo &= ~(1ull << id);
        lds_wr(ctl + 32u + (u32)(q & 7) * 4u, (u32)id);
      }
      ISSUE_LOADS(q, id);
      waitv<8>();
      if (PWHO == 1 && nin && q == PQ1) probe0((u32)(d - 2) * IN_SRC + 2u * MIB + (u32)p * 4096u, P.peer);
      if (PWHO == 1 && nin && q == PQ2) probe0(PIN_OFF + (call % 3u) * PIN_PAR + (u32)(32 + (d - 2) * 16 + NS) * PIECE + (u32)p * 4096u, P.peer);
      if (q - 1 > (int)pub) { pub = (u32)(q - 1); lds_wr(ctl, pub); }
      if (q == nin - 1) STAMP(myws, 1);
    }
    waitv<0>();
    if (dead) st4_lane0(TMO_OFF, call, myws);
    lds_wr(ctl, dead ? DEAD : (u32)np);
    STAMP(myws, 3);
    STAMPV(myws, 0, tstart);
    return;
  }

  char* const rws = P.peer;
  u32x4 f0, f1, f2, f3;
  asm volatile("s_nop 4\n\t"
               "global_load_dwordx4 %0, %4, %8 " MLP "\n\t"
               "global_load_dwordx4 %1, %5, %8 offset:1024 " MLP "\n\t"
               "global_load_dwordx4 %2, %6, %8 offset:2048 " MLP "\n\t"
               "global_load_dwordx4 %3, %7, %8 offset:3072 " MLP
               : "=&v"(f0), "=&v"(f1), "=&v"(f2), "=&v"(f3)
               : "v"(wo[0]), "v"(wo[1]), "v"(wo[2]), "v"(wo[3]), "s"(W_BASE(0)) : "memory");
  const u32 cnt = bump_issue(CNT_OFF + (u32)(blockIdx.x * 4 + wave) * 64u, myws);
  const int slot = d - 2;
  if (lane < (u32)PWN) probe(fin_off(max(slot, 0), 64 + p * 2) + lane * 2u * MIB, rws);
  u32 dbg1 = 0, dbg2 = 0;
  u32 tgrid = (u32)(T0 + TAU - 1) - ((u32)__builtin_amdgcn_readfirstlane((u32)tstart) + (u32)(T0 + TAU - 1) - ((u32)(((p + STGD * d) & 15) * TAU) >> 4)) % (u32)TAU;
  const u32 call = bump_get<1>(cnt);
  asm volatile("" : "+v"(f0), "+v"(f1), "+v"(f2), "+v"(f3));
  const u32 in0 = (u32)slot * IN_SRC + (call & 1u) * PIECE;
  const u32 dl = (u32)p * 4096u + lane * 16u;
  u32 xl[4], xd[4];
#pragma unroll
  for (int k = 0; k < 4; ++k) {
    const u32 c = (u32)k * 64u + lane, t = c / (4u * XR), row = (c >> 2) % XR;
    xl[k] = (row * XC + t * 4u + ((c ^ ((xrow + row) >> 2)) & 3u)) * 16u;
    xd[k] = XCT ? dl : (xks + t) * 8192u + (xrow + row) * 64u + (c & 3u) * 16u - (u32)k * 1024u;
  }
  const u32 pin0 = PIN_OFF + (call % 3u) * PIN_PAR + (u32)(d == 1 ? 0 : 32 + slot * 16 + NS) * PIECE;
  int nflag = 0;
#define FLAG(QQ) do { \
    const int q_ = (QQ); \
    const bool real_ = q_ == nflag && q_ < nin; \
    const u32 o_ = real_ ? fin_off(slot, q_) : DUMMY_OFF + (u32)(blockIdx.x * 4 + wave) * 128u; \
    add1_lane0(o_, uni(real_ ? rws : myws)); \
    nflag += real_ ? 1 : 0; \
  } while (0)

#define SLIP() do { \
    const u32 el_ = (u32)__builtin_amdgcn_readfirstlane((u32)(now() - tstart)); \
    if (el_ > tgrid + (u32)ALT) tgrid += (el_ - tgrid - (u32)ALT + (u32)TAU - 1u) / (u32)TAU * (u32)TAU; \
    sleep_exact(tstart, tgrid); \
  } while (0)
  if (nin) {
    char* const dst = rws + (size_t)in0;
    SLIP();
    st16<0>(dl, f0, dst);
    st16<1024>(dl, f1, dst);
    st16<2048>(dl, f2, dst);
    st16<3072>(dl, f3, dst);
    FLAG(-1);
    tgrid += (u32)(TQ(1) - TQ(0) - GSH);
  } else {
    sleep_until(tstart, 600u);
    probe0(pin0 + (u32)p * 4096u, rws);
    probe0(pin0 + (u32)(NOUT1 - 1) * PIECE + (u32)p * 4096u, rws);
  }
  u32 full = 0, tnext = 0;
  for (int q = nin ? 1 : 0; q < np; ++q) {
    bool isx; int ix;
    piece_of(q, d, isx, ix);
    isx = isx && q < nin;
    if (d != 1 || G1) tnext = max(tnext, tgrid);
    if (!PRE) sleep_until(tstart, tnext);
    if (full <= (u32)q) {
      full = lds_rd(ctl);
      if (full <= (u32)q) {
        bool drained = !(q > 0 && q < nin + LAG);
        u64 el;
        for (;;) {
          __builtin_amdgcn_s_sleep(1);
          full = lds_rd(ctl);
          el = now() - tstart;
          if (full > (u32)q) break;
          if (!drained && __builtin_amdgcn_readfirstlane((int)(el > (u64)(tnext + 30u)))) {
            drained = true;
            waitv<0>();
#pragma unroll
            for (int z = LAG; z >= 1; --z) FLAG(q - z);
          }
          if (__builtin_amdgcn_readfirstlane((int)(el > 2ull * DEADLINE - 1000ull))) { full = DEAD; break; }
        }
        tnext = max(tnext, (u32)__builtin_amdgcn_readfirstlane((u32)el));
      }
      if (full == DEAD) break;
    }
    const u32 la = ring + (u32)(q & 7) * 4096u;
    const u32x4 d0 = *(volatile LDSP u32x4*)(uintptr_t)(la + (isx ? xl[0] : lane * 16u));
    const u32x4 d1 = *(volatile LDSP u32x4*)(uintptr_t)(la + (isx ? xl[1] : lane * 16u + 1024u));
    const u32x4 d2 = *(volatile LDSP u32x4*)(uintptr_t)(la + (isx ? xl[2] : lane * 16u + 2048u));
    const u32x4 d3 = *(volatile LDSP u32x4*)(uintptr_t)(la + (isx ? xl[3] : lane * 16u + 3072u));
    const u32 id = lds_rd(ctl + 32u + (u32)(q & 7) * 4u);
    lds_wr(ctl + 4u, (u32)q + 1u);
    u32 o;
    if (q < nin) {
      o = in0 + (u32)(isx ? cont_x(ix) : cont_w(ix)) * 2u * PIECE;
    } else {
      o = pin0 + min(id, (u32)(nid - 1)) * PIECE;
      if (d == 1 && id < (u32)SWP) o += (u32)(((p >> 3) - (int)(id & 1u)) * 32768);
    }
    char* const dst = rws + (size_t)o;
    if (PRE) sleep_exact(tstart, tnext);
    tgrid += (u32)(TAU - (q < QA ? DA : 0) - (q >= QB ? DB : 0));
    tnext += (u32)(d == 1 && !G1 ? TAU1 : TAU2);
    st16<0>(isx ? xd[0] : dl, d0, dst);
    st16<1024>(isx ? xd[1] : dl, d1, dst);
    st16<2048>(isx ? xd[2] : dl, d2, dst);
    st16<3072>(isx ? xd[3] : dl, d3, dst);
    if (nin && q < nin + LAG) {
      waitv<5 * LAG>();
      FLAG(q - LAG);
      if (PWHO == 0 && q == PQ1) probe0((u32)slot * IN_SRC + 2u * MIB + (u32)p * 4096u, rws);
      if (PWHO == 0 && q == PQ2) probe0(pin0 + (u32)p * 4096u, rws);
    } else {
      waitv<4 * WIN>();
    }
    if (q == NS + 11) STAMP(myws, 1);
    if (q == nin - 1) STAMP(myws, 2);
    if (q == np - 1) STAMP(myws, 4);
  }
  waitv<0>();
  STAMP(myws, 3);
  STAMPV(myws, 0, tstart);
}

struct Frag {
  bf16x8 x[2][2];
  bf16x8 w[2][2];
};

struct Loader {
  const char* base;
  u32 stride[2];
  u32 v[4];
  u32 m0base;
};

template <int STAGE> __device__ __forceinline__ void glds_pair(const Loader& L, int half) {
  const u32 m0v = L.m0base + STAGE * STAGE_B;
  if (half == 0)
    asm volatile("s_mov_b32 m0, %0\n\ts_nop 0\n\t"
                 "global_load_lds_dwordx4 %1, %3 " GLP "\n\t"
                 "global_load_lds_dwordx4 %2, %3 offset:1024 " GLP
                 :: "s"(m0v), "v"(L.v[0]), "v"(L.v[1]), "s"(L.base) : "memory");
  else
    asm volatile("s_mov_b32 m0, %0\n\ts_nop 0\n\t"
                 "global_load_lds_dwordx4 %1, %3 offset:2048 " GLP "\n\t"
                 "global_load_lds_dwordx4 %2, %3 offset:3072 " GLP
                 :: "s"(m0v), "v"(L.v[2]), "v"(L.v[3]), "s"(L.base) : "memory");
}

struct RdAddr { u32 xa[2]; u32 wa[2][2]; };

template <int STAGE> __device__ __forceinline__ bf16x8 rd_x(const RdAddr& A, int mt, int ks) {
  return *(const LDSP bf16x8*)(uintptr_t)(A.xa[ks] + (u32)(STAGE * STAGE_B + mt * 2048));
}
template <int STAGE> __device__ __forceinline__ bf16x8 rd_w(const RdAddr& A, int nt, int ks) {
  s16x4 lo = __builtin_amdgcn_ds_read_tr16_b64_v4i16((LDSP s16x4*)(uintptr_t)(A.wa[nt][0] + (u32)(STAGE * STAGE_B + 8192 + ks * 4096)));
  s16x4 hi = __builtin_amdgcn_ds_read_tr16_b64_v4i16((LDSP s16x4*)(uintptr_t)(A.wa[nt][1] + (u32)(STAGE * STAGE_B + 8192 + ks * 4096)));
  s16x8 v = __builtin_shufflevector(lo, hi, 0, 1, 2, 3, 4, 5, 6, 7);
  return __builtin_bit_cast(bf16x8, v);
}
#define MFMA(nt, mt, ks) acc[nt][mt] = __builtin_amdgcn_mfma_f32_32x32x16_bf16(cur.w[nt][ks], cur.x[mt][ks], acc[nt][mt], 0, 0, 0)

template <int S, int VM, bool LOAD, bool READ>
__device__ __forceinline__ void kstep(f32x16 (&acc)[2][2], const Frag& cur, Frag& nxt, const RdAddr& A, Loader& L) {
  constexpr int SN = (S + 1) & 3;
  if (READ) {
    __builtin_amdgcn_s_waitcnt((VM & 15) | (7 << 4) | (0 << 8) | ((VM >> 4) << 14));
    asm volatile("s_barrier" ::: "memory");
  } else {
    __builtin_amdgcn_s_waitcnt(15 | (7 << 4) | (0 << 8) | (3 << 14));
  }
  SB();
  MFMA(0, 0, 0); SB();
  if (READ) { nxt.w[0][0] = rd_w<SN>(A, 0, 0); nxt.x[0][0] = rd_x<SN>(A, 0, 0); } SB();
  MFMA(0, 1, 0); SB();
  if (READ) { nxt.x[1][0] = rd_x<SN>(A, 1, 0); nxt.w[1][0] = rd_w<SN>(A, 1, 0); } SB();
  MFMA(1, 0, 0); SB();
  if (LOAD) glds_pair<S>(L, 0);
  SB();
  MFMA(1, 1, 0); SB();
  if (READ) { nxt.w[0][1] = rd_w<SN>(A, 0, 1); nxt.x[0][1] = rd_x<SN>(A, 0, 1); } SB();
  MFMA(0, 0, 1); SB();
  if (READ) { nxt.x[1][1] = rd_x<SN>(A, 1, 1); nxt.w[1][1] = rd_w<SN>(A, 1, 1); } SB();
  MFMA(0, 1, 1); SB();
  if (LOAD) { glds_pair<S>(L, 1); L.base += L.stride[S & 1]; }
  SB();
  MFMA(1, 0, 1); SB();
  MFMA(1, 1, 1); SB();
}

template <int S> __device__ __forceinline__ void stage_load(Loader& L) {
  glds_pair<S>(L, 0); glds_pair<S>(L, 1); L.base += L.stride[S & 1];
}

__device__ __forceinline__ void gemm_phase(f32x16 (&acc)[2][2], const RdAddr& A, Loader& L, int n, int sk, u32 jump, u32& dbgv, int dbg0) {
  Frag f0, f1;
  L.base = uni(L.base + (sk == 0 ? IN_SRC : 0u));
  L.stride[0] = __builtin_amdgcn_readfirstlane(L.stride[0]);
  L.stride[1] = __builtin_amdgcn_readfirstlane(L.stride[1]);
  asm volatile("s_nop 4" ::: "memory");
  stage_load<0>(L); stage_load<1>(L); stage_load<2>(L); stage_load<3>(L);
  waitv<12>();
  asm volatile("s_barrier" ::: "memory");
#pragma unroll
  for (int ks = 0; ks < 2; ++ks)
#pragma unroll
    for (int i = 0; i < 2; ++i) { f0.w[i][ks] = rd_w<0>(A, i, ks); f0.x[i][ks] = rd_x<0>(A, i, ks); }
  SB();
  for (int t = 0; t < n - 4; t += 4) {
    L.base += (t & 4 ? jump : 0u) + (t + 4 == 8 * sk ? IN_SRC : 0u);
    kstep<0, 8, true, true>(acc, f0, f1, A, L);
    kstep<1, 8, true, true>(acc, f1, f0, A, L);
    kstep<2, 8, true, true>(acc, f0, f1, A, L);
    kstep<3, 8, true, true>(acc, f1, f0, A, L);
  }
  kstep<0, 8, false, true>(acc, f0, f1, A, L);
  kstep<1, 4, false, true>(acc, f1, f0, A, L);
  kstep<2, 0, false, true>(acc, f0, f1, A, L);
  kstep<3, 0, false, false>(acc, f1, f0, A, L);
}

__device__ __forceinline__ u32 pack2(float a, float b) {
  f32x2 t = {a, b};
  return __builtin_bit_cast(u32, __builtin_convertvector(t, bf16x2));
}
__device__ __forceinline__ u32 relu2(u32 v) {
  typedef short s16x2 __attribute__((ext_vector_type(2)));
  return __builtin_bit_cast(u32, __builtin_elementwise_max(__builtin_bit_cast(s16x2, v), s16x2{0, 0}));
}
__device__ __forceinline__ float bf_lo(u32 v) { return __builtin_bit_cast(float, v << 16); }
__device__ __forceinline__ float bf_hi(u32 v) { return __builtin_bit_cast(float, v & 0xffff0000u); }

__device__ __forceinline__ void zero8k(u32 l16, u32 l16b, char* pb) {
  const u32x4 z4 = {0u, 0u, 0u, 0u};
  asm volatile("s_nop 4" ::: "memory");
  st16<0>(l16, z4, pb); st16<1024>(l16, z4, pb); st16<2048>(l16, z4, pb); st16<3072>(l16, z4, pb);
  st16<0>(l16b, z4, pb); st16<1024>(l16b, z4, pb); st16<2048>(l16b, z4, pb); st16<3072>(l16b, z4, pb);
}

__device__ __forceinline__ void tile(const Args& P, int tid, int wave, u32 lane, char* smem, u64 tstart) {
  const int r = P.rank;
  char* const myws = P.my;
  const int g = r >> 2, b = (r >> 1) & 1, a = r & 1;
  const int bid = blockIdx.x;

  const u32 cnt = bump_issue(CNT_OFF + (u32)(bid * 4 + wave) * 64u, myws);

  int row0, col0, nrem, tskip, j, e, k, rb;
  int pi0 = -1, pi1 = -1;
  bool fin;
  if (tid < N_MAIN_WG) {
    const int xcd = tid & 7, o = tid >> 3, u = xcd >> 1;
    int rbi, cbi;
    if (u < 2) { rbi = o < 24 ? 1 + o % 3 : 0; cbi = o < 24 ? o / 3 : o - 24; }
    else { rbi = o & 3; cbi = o >> 2; }
    rb = u + 4 * rbi; k = cbi * 2 + (xcd & 1);
    j = rb >> 1; e = rb & 1;
    row0 = (r ^ j) * 512 + b * 256 + e * 128; col0 = g * 2048 + (a ? 15 - k : k) * 128;
    nrem = j >= 2 ? 40 : 48; tskip = j >= 2 ? j - 2 : 99;
    fin = j == 0;
    if (j == 0) pi1 = 6 * NS + 15 - k;
    else if (j >= 2 && k >= 16 - NS) pi0 = ((j ^ 1) - 2) * NS + 15 - k;
  } else {
    const int f = tid - N_MAIN_WG;
    const int qd = 1 + f / 32, rem = f & 31;
    e = (rem >> 3) & 1; k = (rem & 7) * 2 + (rem >> 4);
    const int h = b ^ (qd & 1), g2 = g ^ (qd >> 1);
    row0 = r * 512 + h * 256 + e * 128; col0 = g2 * 2048 + (a ? 15 - k : k) * 128;
    nrem = 0; tskip = 99; j = 0; rb = 0;
    fin = true;
    const int iA = k >= NS ? 32 + (2 * qd - 2) * 16 + k : -1;
    const int iB = 15 - k >= NS ? 32 + (2 * qd - 1) * 16 + 15 - k : -1;
    pi0 = k < 8 ? iA : iB;
    pi1 = k < 8 ? iB : iA;
  }

  const u32 sbase = (u32)(uintptr_t)smem;
  const int wm = wave >> 1, wn = wave & 1;
  RdAddr A;
  {
    const u32 h = lane >> 5, m = lane & 31u, f = (lane >> 2) & 3u;
    A.xa[0] = sbase + ((u32)wm * 64u + m) * 64u + 16u * (h ^ f);
    A.xa[1] = sbase + ((u32)wm * 64u + m) * 64u + 16u * ((2u + h) ^ f);
    const u32 blk = (lane >> 4) & 1u, q = (lane & 15u) >> 2, pp = lane & 3u;
#pragma unroll
    for (int nt = 0; nt < 2; ++nt)
#pragma unroll
      for (int t = 0; t < 2; ++t) {
        const u32 row = 8u * h + 4u * t + q;
        const u32 ch = (u32)wn * 8u + (u32)nt * 4u + 2u * blk + (pp & 1u);
        A.wa[nt][t] = sbase + 256u * row + 16u * (ch ^ (((row & 3u) << 2) | ((row >> 2) & 3u))) + 8u * (pp >> 1);
      }
  }

  f32x16 acc[2][2];
#pragma unroll
  for (int i = 0; i < 2; ++i)
#pragma unroll
    for (int k = 0; k < 2; ++k)
#pragma unroll
      for (int z = 0; z < 16; ++z) acc[i][k][z] = 0.f;

  const u32 tile_off = (u32)e * 32768u + (u32)wave * 8192u;
  const u32 l16 = lane * 16u, l16b = lane * 16u + 4096u;
  const u32 tw = sbase + (u32)wave * 8192u + (lane & 31u) * 128u + (((lane >> 5) ^ (lane & 7u)) * 16u);
  const u32 tr = sbase + (u32)wave * 8192u + (lane >> 3) * 128u + (((lane & 7u) ^ (lane >> 3)) * 16u);
  const u32 oo = (lane >> 3) * 8192u + (lane & 7u) * 16u;
  char* const ob = uni(P.out + (size_t)((u32)((row0 & 511) + wm * 64) * 8192u + (u32)(col0 + wn * 64) * 2u));
  const int nph = nrem ? 2 : 1;
  const int qin = max(slot_w(k), slot_x(j, e, j == 7 ? 2 : 7));
  const u32 tin = (u32)(EARLY + TQ(qin));
  const int addph = fin ? nph - 1 : 0;

  if (LPT && (bid >> 3) == 25 && wave == 0) {
    sleep_until(tstart, (u32)LPT);
    if (lane < 28u) probe(lane * 2u * MIB + MIB + (u32)(bid & 7) * 128u, myws);
  }
  if (RPT && (bid >> 3) == RPB && wave == 1) {
    const u32 dd = 1u + lane % 7u;
    const u64 pp = ((const GLBP u64*)(uintptr_t)__builtin_amdgcn_kernarg_segment_ptr())[5u + dd];
    const u32 cl = bump_get<0>(cnt);
    const u32 o = (lane < 7u && dd > 1u ? (dd - 2u) * IN_SRC + 2u * MIB : PIN_OFF + (cl % 3u) * PIN_PAR + (dd > 1u ? dd * 16u : 30u) * PIECE) + (u32)(bid & 7) * 128u;
    sleep_until(tstart, (u32)RPT);
    if (lane < 14u) asm volatile("global_atomic_add %0, %1, off sc1" :: "v"(pp + o), "v"(0u) : "memory");
  }
  if (PD0 && (fin || k >= NS)) sleep_until(tstart, (u32)(PD0 + k * PD1 + (fin ? PDF : 0)));
  Loader L;
  L.m0base = sbase + (u32)wave * 4096u;
  bool dead = false;
  u32 dbgv = 0;
  u32 call = 0;
  for (int ph = 0; ph < nph; ++ph) {
    int n, ts;
    bool skip = false;
    if (ph == 0) {
      n = 16; ts = 99;
      if (wave < 2) {
        L.base = P.x + (size_t)((u32)(row0 + wave * 64) * 1024u);
        L.stride[0] = L.stride[1] = 64u;
        const u32 c = (lane & 3u) ^ ((lane >> 4) & 3u);
#pragma unroll
        for (int k = 0; k < 4; ++k) L.v[k] = (lane >> 2) * 1024u + c * 16u + (u32)k * (16384u - 1024u);
      } else {
        L.base = P.w + (size_t)((u32)(wave - 2) * 16u * 8192u + (u32)col0 * 2u);
        L.stride[0] = L.stride[1] = 32u * 8192u;
        const u32 q = lane >> 4;
#pragma unroll
        for (int k = 0; k < 4; ++k) L.v[k] = q * 8192u + (((lane & 15u) ^ ((q << 2) | (u32)k)) * 16u) + (u32)k * (4u * 8192u - 1024u);
      }
    } else {
      n = nrem; ts = tskip;
      STAMP(myws, 1);
      if (wave == 0) {
        const int sl = (int)min(lane, 5u);
        const u32 fo = fin_off(sl, max(slot_w(k), slot_x(j, e, sl + 2)));
        bool pend = lane < 6u && sl + 2 != j;
        sleep_until(tstart, tin);
        const u64 t0 = now();
        u32 st = 0;
        for (;;) {
          if (pend) pend = atom_rd(fo, myws) < 16u * call;
          if (__builtin_amdgcn_ballot_w64(pend) == 0ull) break;
          if (give_up(t0, myws, call)) {
            if (lane == 0) st_agt(myws + TMO_OFF, call);
            st = 1;
            break;
          }
          __builtin_amdgcn_s_sleep(PSL);
        }
        if (lane == 0) *(LDSP u32*)(uintptr_t)(sbase + 4u * STAGE_B) = st;
        __builtin_amdgcn_s_waitcnt(0);
      }
      asm volatile("s_barrier" ::: "memory");
      const bool wgdead = lds_rd(sbase + 4u * STAGE_B) != 0;
      STAMP(myws, 2);
      dead = dead || wgdead;
      skip = wgdead;
      const u32 par = call & 1u;
      static_assert(!XCT || XR == 32, "layout of the x pieces");
      const bool xc = XCT && wave < 2;
      L.base = myws + (size_t)(((u32)(wave < 2 ? cont_x(rb) : cont_w(k)) * 2u + par) * PIECE + (u32)(wave & 1) * (xc ? 8192u : 4096u));
      L.stride[0] = xc ? 2048u : 8192u;
      L.stride[1] = xc ? 14336u : 8192u;
#pragma unroll
      for (int k = 0; k < 4; ++k) L.v[k] = lane * 16u + (xc && k >= 2 ? 2048u : 0u);
    }
    if (!skip) gemm_phase(acc, A, L, n, ts, ph ? IN_SRC - PIECE : 0u, dbgv, ph * 8);
    if (ph == nph - 1) STAMP(myws, 3);
    if (ph == 0) {
      call = bump_get<0>(cnt);
      if (ZT0 && max(pi0, pi1) >= 0) {
        char* const pz = myws + (size_t)(PIN_OFF + ((call + 2u) % 3u) * PIN_PAR + tile_off);
        sleep_until(tstart, (u32)ZT0 + (u32)((tid >> 3) & 15) * (u32)ZDT);
        if (pi0 >= 0) zero8k(l16, l16b, uni(pz + (size_t)((u32)pi0 * PIECE)));
        if (pi1 >= 0) zero8k(l16, l16b, uni(pz + (size_t)((u32)pi1 * PIECE)));
      }
    }
    const u32 pin = PIN_OFF + (call % 3u) * PIN_PAR + tile_off;
#pragma unroll 1
    for (int s = 0; s < 2; ++s) {
      const int pi = ph != addph ? -1 : (s == 0 ? pi0 : pi1);
      if (pi < 0 && !(fin && ph == 0 && s == 0)) continue;
      u32x4 pv[8];
#pragma unroll
      for (int i = 0; i < 8; ++i) pv[i] = u32x4{0u, 0u, 0u, 0u};
      if (pi >= 0) {
        char* const pb = uni(myws + (size_t)(pin + (u32)pi * PIECE));
        const bool first = pi == 0 || (pi >= NOUT1 && (pi & 15) == NS);
        const u32 wo = pin + (u32)(pi - ((!WOWN && (lane & 2u) && !first) ? 1 : 0)) * PIECE + (lane & 1u) * 4096u + 4092u;
        sleep_until(tstart, pi >= NOUT1 ? (u32)(EARLY + TQ(NIN - 2 + (pi & 15) - NS)) : (fin ? 0u : tin - 300u));
        const u64 t0 = now();
        while (!dead) {
          u32 v = WOWN ? 1u : 0u;
          if (lane < (WOWN ? 2u : 4u)) v = atom_rd(wo, myws);
          if (WOWN ? __builtin_amdgcn_ballot_w64(v == 0u) == 0ull : __builtin_amdgcn_ballot_w64(v != 0u) != 0ull) break;
          if (give_up(t0, myws, call)) { dead = true; break; }
          __builtin_amdgcn_s_sleep(WSL);
        }
        if (s == 1) STAMP(myws, 4);
        for (;;) {
          if (__builtin_expect(dead, 0)) {
            if (lane == 0) st_agt(myws + TMO_OFF, call);
#pragma unroll
            for (int i = 0; i < 8; ++i) pv[i] = u32x4{0x7fc07fc0u, 0x7fc07fc0u, 0x7fc07fc0u, 0x7fc07fc0u};
            break;
          }
          asm volatile("s_nop 4\n\t"
                       "global_load_dwordx4 %0, %8, %10 sc0 sc1\n\t"
                       "global_load_dwordx4 %1, %8, %10 offset:1024 sc0 sc1\n\t"
                       "global_load_dwordx4 %2, %8, %10 offset:2048 sc0 sc1\n\t"
                       "global_load_dwordx4 %3, %8, %10 offset:3072 sc0 sc1\n\t"
                       "global_load_dwordx4 %4, %9, %10 sc0 sc1\n\t"
                       "global_load_dwordx4 %5, %9, %10 offset:1024 sc0 sc1\n\t"
                       "global_load_dwordx4 %6, %9, %10 offset:2048 sc0 sc1\n\t"
                       "global_load_dwordx4 %7, %9, %10 offset:3072 sc0 sc1"
                       : "=&v"(pv[0]), "=&v"(pv[1]), "=&v"(pv[2]), "=&v"(pv[3]), "=&v"(pv[4]), "=&v"(pv[5]), "=&v"(pv[6]), "=&v"(pv[7])
                       : "v"(l16), "v"(l16b), "s"(pb) : "memory");
          const bool late = __builtin_amdgcn_readfirstlane((int)(now() - t0 > 50000ull)) != 0;
          asm volatile("s_waitcnt vmcnt(0)"
                       : "+v"(pv[0]), "+v"(pv[1]), "+v"(pv[2]), "+v"(pv[3]), "+v"(pv[4]), "+v"(pv[5]), "+v"(pv[6]), "+v"(pv[7]) :: "memory");
          u32 mn = 0xffffffffu;
#pragma unroll
          for (int i = 0; i < 8; ++i)
#pragma unroll
            for (int z = 0; z < 4; ++z) mn = min(mn, pv[i][z]);
          if (__builtin_amdgcn_ballot_w64(mn == 0u) == 0ull) break;
          if (__builtin_expect(late, 0)) dead = give_up(t0, myws, call);
        }
      }
#pragma unroll
      for (int nt = 0; nt < 2; ++nt)
#pragma unroll
        for (int mt = 0; mt < 2; ++mt)
#pragma unroll
          for (int sx = 0; sx < 2; ++sx)
#pragma unroll
            for (int z = 0; z < 4; ++z) {
              const u32 u = pv[(nt * 2 + mt) * 2 + sx][z];
              acc[nt][mt][8 * sx + 2 * z] += bf_lo(u);
              acc[nt][mt][8 * sx + 2 * z + 1] += bf_hi(u);
            }
      if (!fin) continue;
      if (s == 1) STAMP(myws, 6);
#pragma unroll
      for (int nt = 0; nt < 2; ++nt)
#pragma unroll
        for (int mt = 0; mt < 2; ++mt)
#pragma unroll
          for (int sx = 0; sx < 2; ++sx) {
            u32x4 v;
#pragma unroll
            for (int z = 0; z < 4; ++z) v[z] = relu2(pack2(acc[nt][mt][8 * sx + 2 * z], acc[nt][mt][8 * sx + 2 * z + 1]));
            *(volatile LDSP u32x4*)(uintptr_t)((tw ^ (u32)((nt * 4 + sx * 2) * 16)) + (u32)mt * 4096u) = v;
          }
      u32 o1, o2, o3, o4, o5, o6, o7;
      asm volatile("ds_read_b128 %0, %15\n\t"
                   "ds_read_b128 %1, %15 offset:1024\n\t"
                   "ds_read_b128 %2, %15 offset:2048\n\t"
                   "ds_read_b128 %3, %15 offset:3072\n\t"
                   "ds_read_b128 %4, %15 offset:4096\n\t"
                   "ds_read_b128 %5, %15 offset:5120\n\t"
                   "ds_read_b128 %6, %15 offset:6144\n\t"
                   "ds_read_b128 %7, %15 offset:7168\n\t"
                   "v_add_u32 %8, 0x10000, %16\n\t"
                   "v_add_u32 %9, 0x20000, %16\n\t"
                   "v_add_u32 %10, 0x30000, %16\n\t"
                   "v_add_u32 %11, 0x40000, %16\n\t"
                   "v_add_u32 %12, 0x50000, %16\n\t"
                   "v_add_u32 %13, 0x60000, %16\n\t"
                   "v_add_u32 %14, 0x70000, %16\n\t"
                   "s_mov_b64 exec, %18\n\t"
                   "s_waitcnt lgkmcnt(7)\n\tglobal_store_dwordx4 %16, %0, %17 sc0 sc1\n\t"
                   "s_waitcnt lgkmcnt(6)\n\tglobal_store_dwordx4 %8, %1, %17 sc0 sc1\n\t"
                   "s_waitcnt lgkmcnt(5)\n\tglobal_store_dwordx4 %9, %2, %17 sc0 sc1\n\t"
                   "s_waitcnt lgkmcnt(4)\n\tglobal_store_dwordx4 %10, %3, %17 sc0 sc1\n\t"
                   "s_waitcnt lgkmcnt(3)\n\tglobal_store_dwordx4 %11, %4, %17 sc0 sc1\n\t"
                   "s_waitcnt lgkmcnt(2)\n\tglobal_store_dwordx4 %12, %5, %17 sc0 sc1\n\t"
                   "s_waitcnt lgkmcnt(1)\n\tglobal_store_dwordx4 %13, %6, %17 sc0 sc1\n\t"
                   "s_waitcnt lgkmcnt(0)\n\tglobal_store_dwordx4 %14, %7, %17 sc0 sc1\n\t"
                   "s_cmp_lg_u64 %18, -1\n\t"
                   "s_cbranch_scc1 1f\n\t"
                   "s_endpgm\n"
                   "1:\n\t"
                   "s_mov_b64 exec, -1\n\t"
                   "s_nop 1"
                   : "=&v"(pv[0]), "=&v"(pv[1]), "=&v"(pv[2]), "=&v"(pv[3]), "=&v"(pv[4]), "=&v"(pv[5]), "=&v"(pv[6]), "=&v"(pv[7]),
                     "=&v"(o1), "=&v"(o2), "=&v"(o3), "=&v"(o4), "=&v"(o5), "=&v"(o6), "=&v"(o7)
                   : "v"(tr), "v"(oo), "s"(ob), "s"(s == 1 ? ~0ull : (u64)WMK) : "memory", "scc");
      if (s == 0) waitv<0>();
    }
  }

  if (!fin) {
    STAMP(myws, 6);
    const bool dir = DIR && j >= 2 && k < NS;
    char* dst = uni(dir ? P.peer + (size_t)(PIN_OFF + (call % 3u) * PIN_PAR + (u32)((j - 2) * NS + k) * PIECE + tile_off)
                        : myws + (size_t)(POUT_OFF + (u32)((j - 1) * 16 + k) * PIECE + tile_off));
    char* fl = myws + (size_t)(FLOC_OFF + (u32)((j - 1) * 8 + e * 4 + wave) * 128u + (u32)k * 4u);
    u32 least = 1u;
    if (__builtin_expect(dead, 0)) {
      least = 0x7fc07fc0u;
#pragma unroll
      for (int i = 0; i < 64; ++i) acc[i >> 5][(i >> 4) & 1][i & 15] = 0.f;
    }
    asm volatile("s_nop 4" ::: "memory");
#pragma unroll
    for (int nt = 0; nt < 2; ++nt)
#pragma unroll
      for (int mt = 0; mt < 2; ++mt) {
        u32x4 v0, v1;
#pragma unroll
        for (int z = 0; z < 4; ++z) {
          v0[z] = max(pack2(acc[nt][mt][2 * z], acc[nt][mt][2 * z + 1]), least);
          v1[z] = max(pack2(acc[nt][mt][8 + 2 * z], acc[nt][mt][8 + 2 * z + 1]), least);
        }
        const u32 o = nt ? l16b : l16;
        if (mt == 0) st16<0>(o, v0, dst); else st16<2048>(o, v0, dst);
        if (dir) __builtin_amdgcn_s_sleep(DSL);
        if (mt == 0) st16<1024>(o, v1, dst); else st16<3072>(o, v1, dst);
        if (dir) __builtin_amdgcn_s_sleep(DSL);
      }
    waitv<0>();
    if (lane == 0) st_sys(fl, call);
  }
  STAMP(myws, 5);
  STAMPV(myws, 0, tstart);
}

__global__ __launch_bounds__(256, 2) void fused_gemm_rs(Params P) {
  extern __shared__ __attribute__((aligned(16))) char smem[];
  const u32 lane = threadIdx.x & 63u;
  const int wave = __builtin_amdgcn_readfirstlane(threadIdx.x >> 6);
  const int bid = blockIdx.x;
  const int d = bid < N_PUSH_WG ? 1 + ((bid >> 3) + 3 * (wave >> 1)) % 7 : 1;
  if (CPF && (u32)(bid - CPB) < 16u && wave < 3) {
    u64 pc;
    asm volatile("s_getpc_b64 %0\n\ts_mov_b32 m0, %1\n\ts_nop 3\n\tglobal_load_lds_dword %2, %0"
                 : "=&s"(pc) : "s"((u32)(uintptr_t)smem + (CPB < N_PUSH_WG ? 0u : 66048u)), "v"(min(threadIdx.x * 128u, (u32)CPF - 128u)) : "memory");
  }
  typedef u64 u64x4 __attribute__((ext_vector_type(4)));
  u64x4 a4;
  u64 tstart, pe;
  int rk;
  asm volatile("s_load_dwordx8 %0, %4, 0x0\n\t"
               "s_load_dwordx2 %1, %4, %5 offset:0x28\n\t"
               "s_load_dword %2, %4, 0x20\n\t"
               "s_memrealtime %3\n\t"
               "ds_write_b32 %6, %7\n\t"
               "s_waitcnt lgkmcnt(0)\n\t"
               "s_barrier"
               : "=&s"(a4), "=&s"(pe), "=&s"(rk), "=&s"(tstart)
               : "s"(__builtin_amdgcn_kernarg_segment_ptr()), "s"(d * 8),
                 "v"((u32)(uintptr_t)smem + 65536u + (threadIdx.x & 255u) * 4u), "v"(0u) : "memory");
  u32 t1 = 0;
  const Args A = {(const char*)a4[0], (const char*)a4[1], (char*)a4[2], (char*)a4[3], (char*)pe, rk, t1};
  if (bid < N_PUSH_WG) mover(A, d, wave, lane, smem, tstart);
  else if (bid >= 256 && bid < 256 + N_PUSH_WG) {
    for (int i = 0; i < 6; ++i) __builtin_amdgcn_s_sleep(16);
  } else tile(A, bid < 256 - ROT ? bid - N_PUSH_WG + ROT : (bid < 256 ? bid - (256 - ROT) : bid - 2 * N_PUSH_WG), wave, lane, smem, tstart);
}

}

extern "C" void kernel_launch(int rank, int nranks, ncclComm_t comm,
                              hipStream_t stream,
                              void* const* d_in, const int* in_sizes, int n_in,
                              void* d_out, int out_size,
                              const int* in_dtypes, int out_dtype,
                              void* const* d_out_all,
                              void* const* d_ws_all, size_t ws_bytes) {
  if (nranks != NR || n_in < 2 || ws_bytes < (size_t)WS_NEED || (unsigned)rank >= (unsigned)NR) {
    fprintf(stderr, "unsupported configuration\n");
    return;
  }
  Params P;
  P.x = (const char*)d_in[0];
  P.w = (const char*)d_in[1];
  P.out = (char*)d_out;
  P.my = (char*)d_ws_all[rank];
  for (int i = 0; i < NR; ++i) P.peer[i] = (char*)d_ws_all[rank ^ i];
  P.rank = rank;
  P.pad = 0;
  HIP_CHECK(hipFuncSetAttribute((const void*)fused_gemm_rs, hipFuncAttributeMaxDynamicSharedMemorySize, (int)LDS_BYTES));
  hipLaunchKernelGGL(fused_gemm_rs, dim3(N_WG), dim3(256), LDS_BYTES, stream, P);
  HIP_CHECK(hipGetLastError());
}
```

```cpp
#include <hip/hip_runtime.h>
#include <rccl/rccl.h>
#include <cstdio>
#include <cstdint>

#define HIP_CHECK(expr) do { hipError_t _e = (expr); if (_e != hipSuccess) { \
    fprintf(stderr, "HIP error %s at %s:%d\n", hipGetErrorString(_e), __FILE__, __LINE__); \
    return; } } while (0)

namespace {

typedef unsigned u32;
typedef unsigned long long u64;
typedef u32 u32x4 __attribute__((ext_vector_type(4)));
typedef u32 u32x2 __attribute__((ext_vector_type(2)));
typedef u32 u32x16 __attribute__((ext_vector_type(16)));
typedef float f32x2 __attribute__((ext_vector_type(2)));
typedef float f32x16 __attribute__((ext_vector_type(16)));
typedef __bf16 bf16x2 __attribute__((ext_vector_type(2)));
typedef __bf16 bf16x8 __attribute__((ext_vector_type(8)));
typedef short s16x4 __attribute__((ext_vector_type(4)));
typedef short s16x8 __attribute__((ext_vector_type(8)));
#define LDSP __attribute__((address_space(3)))
#define GLBP __attribute__((address_space(1)))

constexpr int NR = 8;
constexpr u32 PIECE = 65536u;
constexpr u32 MIB = 1u << 20;
#define NS 2
#define OX 14
#define TAU 112
#define T0 20
#define QA 0
#define DA 0
#define QB 20
#define DB 1
#define QC 99
#define DC 0
#define TQ(q) (T0 + TAU * (q) - DA * ((q) < QA ? (q) : QA) - DB * ((q) > QB ? (q) - QB : 0))
#define EARLY 150
#define TAU1 110
#define TAU2 0
#define STGD 2
#define PSL 4
#define LSL 0
#define WSL 3
#define RSL 0
#define WOWN 0
#define ALN 1
#define ALO 99
#define ALT 25
#define PWN 1
#define DIR 0
#define DSL 4
#define SWP 14
#define CPF 13824
#define CPB 56
#define PD0 0
#define PD1 0
#define PDF 0
#define GSH -18
#define ORG 1
#define OC 95
#define OG 18
#define OSP 7
#define ORD 1
#define OD 20
#define LPT 400
#define ROT 48
#define PWIN 64
#define PWHO 1
#define PQ1 9
#define PQ2 14
#define PROT 0
#define XR 32
#define ZT0 900
#define ZDT 100
#define RPT 400
#define RPB 25
#define XCT 0
#define L1T 500
#define SXT 6
#define PRE 1
#define WMK 1ull
#define G1 0
#define LAG 1
#define WIN 3
#define GLP "sc0 sc1"
#define MLP "sc0 sc1"
constexpr int NIN = 30;
constexpr int NOUT = 16 - NS;
constexpr int NOUT1 = 16 + 6 * NS;
constexpr u32 IN_SRC = 4u * MIB;
constexpr u32 PIN_OFF = 24u * MIB;
constexpr u32 PIN_PAR = 8u * MIB;
constexpr u32 POUT_OFF = 48u * MIB;
constexpr u32 FLG_OFF = 55u * MIB;
constexpr u32 FLOC_OFF = FLG_OFF + 16384u;
constexpr u32 TMO_OFF = FLG_OFF + 49152u;
constexpr u32 LNK_OFF = FLG_OFF + MIB + 262144u + 131072u;
constexpr u32 DUMMY_OFF = FLG_OFF + 65536u;
constexpr u32 DBG_OFF = FLG_OFF + 131072u;
constexpr u32 CNT_OFF = FLG_OFF + MIB + 262144u;
constexpr u32 WS_NEED = FLG_OFF + 2u * MIB;

constexpr int N_PUSH_WG = 56;
constexpr int N_MAIN_WG = 256;
constexpr int N_FIN_WG = 96;
constexpr int N_WG = 2 * N_PUSH_WG + N_MAIN_WG + N_FIN_WG;
constexpr u32 STAGE_B = 16384u;
constexpr u32 LDS_BYTES = 4u * STAGE_B + 1024u;
constexpr u64 DEADLINE = 100000000ull;
constexpr u32 DEAD = 0x7fffffffu;

struct Params {
  const char* x;
  const char* w;
  char* my;
  char* out;
  int rank;
  int pad[3];
  struct { char* peer; const char* w0; } lk[NR];
};
struct Args { const char* x; const char* w; char* my; char* out; char* peer; int rank; u32 tis; u32x16 f16; u32x2 co; };

#define SB() __builtin_amdgcn_sched_barrier(0)
template <int N> __device__ __forceinline__ void waitv() { asm volatile("s_waitcnt vmcnt(%0)" :: "i"(N) : "memory"); }

__device__ __forceinline__ u64 now() { return __builtin_amdgcn_s_memrealtime(); }
template <typename T> __device__ __forceinline__ T* uni(T* p) {
  const u64 v = (u64)(uintptr_t)p;
  const u32 lo = __builtin_amdgcn_readfirstlane((u32)v), hi = __builtin_amdgcn_readfirstlane((u32)(v >> 32));
  return (T*)(uintptr_t)(((u64)hi << 32) | lo);
}
__device__ __forceinline__ u32 ld_sys(const void* p) {
  return __hip_atomic_load((const GLBP u32*)(uintptr_t)p, __ATOMIC_RELAXED, __HIP_MEMORY_SCOPE_SYSTEM);
}
__device__ __forceinline__ void st_sys(void* p, u32 v) {
  __hip_atomic_store((GLBP u32*)(uintptr_t)p, v, __ATOMIC_RELAXED, __HIP_MEMORY_SCOPE_SYSTEM);
}
__device__ __forceinline__ void st_agt(void* p, u32 v) {
  __hip_atomic_store((GLBP u32*)(uintptr_t)p, v, __ATOMIC_RELAXED, __HIP_MEMORY_SCOPE_AGENT);
}
__device__ __forceinline__ bool give_up(u64 t0, const char* ws, u32 call) {
  const u64 dt = now() - t0;
  if (!__builtin_amdgcn_readfirstlane((int)(dt > 50000ull))) return false;
  __builtin_amdgcn_s_sleep(64);
  if (__builtin_amdgcn_readfirstlane((int)(dt > DEADLINE))) return true;
  return __builtin_amdgcn_readfirstlane(ld_sys(ws + TMO_OFF)) == call;
}
__device__ __forceinline__ void sleep_until(u64 tstart, u32 t) {
  for (;;) {
    const u32 el = (u32)__builtin_amdgcn_readfirstlane((u32)(now() - tstart));
    if (el >= t) break;
    if (t - el > 40u) __builtin_amdgcn_s_sleep(12); else __builtin_amdgcn_s_sleep(1);
  }
}
__device__ __forceinline__ void sleep_exact(u64 tstart, u32 t) {
  for (;;) {
    const u32 el = (u32)__builtin_amdgcn_readfirstlane((u32)(now() - tstart));
    if (el >= t) break;
    if (t - el > 60u) __builtin_amdgcn_s_sleep(12);
    else if (t - el > 20u) __builtin_amdgcn_s_sleep(3);
    else if (t - el > (u32)SXT) __builtin_amdgcn_s_sleep(1);
  }
}
template <int IMM> __device__ __forceinline__ void st16(u32 off, u32x4 v, char* base) {
  asm volatile("global_store_dwordx4 %0, %1, %2 offset:%3 sc0 sc1\n\ts_nop 1" :: "v"(off), "v"(v), "s"(base), "i"(IMM) : "memory");
}
template <int IMM> __device__ __forceinline__ void st16p(u32 off, u32x4 v, char* base) {
  asm volatile("global_store_dwordx4 %0, %1, %2 offset:%3\n\ts_nop 1" :: "v"(off), "v"(v), "s"(base), "i"(IMM) : "memory");
}
#define OST st16
__device__ __forceinline__ void dma16x4(u32 lds, const u32 (&o)[4], const char* base) {
  asm volatile("s_mov_b32 m0, %0\n\ts_nop 3\n\t"
               "global_load_lds_dwordx4 %1, %5 " MLP "\n\t"
               "global_load_lds_dwordx4 %2, %5 offset:1024 " MLP "\n\t"
               "global_load_lds_dwordx4 %3, %5 offset:2048 " MLP "\n\t"
               "global_load_lds_dwordx4 %4, %5 offset:3072 " MLP
               :: "s"(lds), "v"(o[0]), "v"(o[1]), "v"(o[2]), "v"(o[3]), "s"(base) : "memory");
}
__device__ __forceinline__ void probe(u32 off, char* base) {
  asm volatile("s_nop 4\n\tglobal_atomic_add %0, %1, %2 sc1" :: "v"(off), "v"(0u), "s"(base) : "memory");
}
__device__ __forceinline__ void probe0(u32 off, char* base) {
  asm volatile("s_mov_b64 exec, 1\n\ts_nop 3\n\tglobal_atomic_add %0, %1, %2 sc1\n\ts_mov_b64 exec, -1"
               :: "v"(off), "v"(0u), "s"(base) : "memory");
}
__device__ __forceinline__ u32 atom_rd(u32 off, const char* base) {
  u32 r;
  asm volatile("global_atomic_add %0, %1, %2, %3 sc0 sc1\n\ts_waitcnt vmcnt(0)" : "=&v"(r) : "v"(off), "v"(0u), "s"(base) : "memory");
  return r;
}
__device__ __forceinline__ u32 bump_issue(u32 off, const char* base) {
  u32 r;
  asm volatile("s_mov_b64 exec, 1\n\ts_nop 3\n\tglobal_atomic_add %0, %1, %2, %3 sc0 sc1\n\ts_mov_b64 exec, -1"
               : "=&v"(r) : "v"(off), "v"(1u), "s"(base) : "memory");
  return r;
}
template <int N> __device__ __forceinline__ u32 bump_get(u32 r) {
  u32 v;
  asm volatile("s_waitcnt vmcnt(%2)\n\tv_readfirstlane_b32 %0, %1\n\ts_nop 4" : "=s"(v) : "v"(r), "i"(N) : "memory");
  return v + 1u;
}
__device__ __forceinline__ void add1_lane0(u32 off, char* base) {
  asm volatile("s_mov_b64 exec, 1\n\ts_nop 3\n\tglobal_atomic_add %0, %1, %2 sc1\n\ts_mov_b64 exec, -1"
               :: "v"(off), "v"(1u), "s"(base) : "memory");
}
__device__ __forceinline__ void st4_lane0(u32 off, u32 val, char* base) {
  asm volatile("s_mov_b64 exec, 1\n\ts_nop 3\n\tglobal_store_dword %0, %1, %2 sc0 sc1\n\ts_mov_b64 exec, -1"
               :: "v"(off), "v"(val), "s"(base) : "memory");
}

#define STAMP(ws, idx) do {} while (0)
#define STAMPV(ws, idx, v) do {} while (0)

__device__ __forceinline__ void piece_of(int q, int d, bool& isx, int& idx) {
  const int t = q - NS, u = q - NS - 12;
  int j = (t >> 1) + 2;
  j += j >= d ? 1 : 0;
  j = j == 8 ? 1 : j;
  isx = (q >= NS && u < 0) || (u >= OX && u < OX + 2);
  idx = q < NS ? q : (u < 0 ? 2 * j + (t & 1) : (u < OX ? NS + u : (u < OX + 2 ? u - OX : NS + u - 2)));
}
__device__ __forceinline__ int slot_w(int k) { return k < NS ? k : (k - NS < OX ? 12 + k : 14 + k); }
__device__ __forceinline__ int slot_x(int j, int e, int d) {
  return j == 0 ? NS + 12 + OX + e : NS + 2 * (j == 1 ? 5 : j - 2 - (j > d ? 1 : 0)) + e;
}
__device__ __forceinline__ int cont_w(int k) { return k < NS ? k : 14 + k; }
__device__ __forceinline__ int cont_x(int rb) { return rb < 2 ? 30 + rb : (rb < 4 ? NS + 10 + rb : NS - 4 + rb); }
__device__ __forceinline__ u32 fin_off(int sl, int q) { return (u32)sl * IN_SRC + (u32)(NS + 2 * sl) * 2u * PIECE + (u32)q * 64u; }
__device__ __forceinline__ u32 lds_rd(u32 a) { return __builtin_amdgcn_readfirstlane(*(volatile LDSP u32*)(uintptr_t)a); }
__device__ __forceinline__ void lds_wr(u32 a, u32 v) { *(volatile LDSP u32*)(uintptr_t)a = v; }

__device__ __forceinline__ void mover(const Args& P, int d, int wave, u32 lane, char* smem, u64 tstart) {
  const int s = P.rank;
  char* const myws = P.my;
  const int pair = wave >> 1;
  const int p = ((blockIdx.x + PROT) & 7) * 2 + pair;
  const u32 r = (u32)(s ^ d) & (u32)(NR - 1);
  const int nin = d == 1 ? 0 : NIN;
  const int np = d == 1 ? (DIR ? 16 : NOUT1) : NIN + NOUT;
  const int nid = d == 1 ? NOUT1 : NOUT;
  const u32 ring = (u32)(uintptr_t)smem + (u32)pair * 32768u;
  const u32 ctl = (u32)(uintptr_t)smem + 65536u + (u32)pair * 512u;
  const char* const xin = P.x;
  const char* const win = P.w;
  const int rg = r >> 2, rb_ = (r >> 1) & 1, ra = r & 1;
  u32 wo[4];
#pragma unroll
  for (int k = 0; k < 4; ++k) {
    const u32 q4 = lane >> 4;
    wo[k] = ((u32)p * 16u + (u32)k * 4u + q4) * 8192u + (((lane & 15u) ^ ((q4 << 2) | (u32)k)) * 16u) - (u32)k * 1024u;
  }
  constexpr u32 XC = 256u / XR;
  const u32 xrow = ((u32)p % (128u / XR)) * XR, xks = (u32)p / (128u / XR) * (XC / 4u);
#define W_BASE(IX) (win + (size_t)((u32)ra * 256u * 8192u + (u32)rg * 4096u + (u32)(ra ? 15 - (IX) : (IX)) * 256u))

  if ((wave & 1) == 0) {
    const u32 cnt = bump_issue(CNT_OFF + (u32)(blockIdx.x * 4 + wave) * 64u, myws);
    u32 xo[4], oo[4];
    {
#pragma unroll
      for (int k = 0; k < 4; ++k) {
        const u32 c = (u32)k * 64u + lane;
        xo[k] = (xrow + c / XC) * 1024u + xks * 64u + (c % XC) * 16u - (u32)k * 1024u;
        oo[k] = (u32)p * 4096u + lane * 16u;
      }
    }
    u32 poll_off;
    {
      const int i0 = (int)lane < nid ? (int)lane : 0;
      const bool sw = d == 1 && i0 < SWP;
      const int i = sw ? (i0 & ~1) | (p >> 3) : i0, ps = sw ? ((i0 & 1) << 3) | (p & 7) : p;
      const int jj = d != 1 ? d : (i < 6 * NS ? 2 + i / NS : 1);
      const int kk = d != 1 ? NS + i : (i < 6 * NS ? i % NS : i - 6 * NS);
      poll_off = FLOC_OFF + (u32)((jj - 1) * 8 + (ps >> 1)) * 128u + (u32)kk * 4u;
    }
#define ISSUE_LOADS(QQ, II) do { \
      const int q_ = (QQ); \
      const u32 dst_ = ring + (u32)(q_ & 7) * 4096u; \
      if (q_ < nin) { \
        bool isx_; int ix_; \
        piece_of(q_, d, isx_, ix_); \
        if (isx_) dma16x4(dst_, xo, xin + (size_t)(((r ^ (u32)(ix_ >> 1)) * 512u + (u32)rb_ * 256u + (u32)(ix_ & 1) * 128u) * 1024u + (u32)ra * 512u)); \
        else      dma16x4(dst_, wo, W_BASE(ix_)); \
      } else { \
        const bool sw_ = d == 1 && (II) < SWP; \
        const int i_ = sw_ ? ((II) & ~1) | (p >> 3) : (II); \
        const int jj_ = d != 1 ? d : (i_ < 6 * NS ? 2 + i_ / NS : 1); \
        const int kk_ = d != 1 ? NS + i_ : (i_ < 6 * NS ? i_ % NS : i_ - 6 * NS); \
        dma16x4(dst_, oo, myws + (size_t)(POUT_OFF + (u32)((jj_ - 1) * 16 + kk_) * PIECE + (u32)(sw_ ? (((II) & 1) - (p >> 3)) * 32768 : 0))); \
      } \
    } while (0)

    int q0 = 0;
    u32 pub = 0, call;
    u32 dbg1 = 0, dbg2 = 0;
    if (nin) {
      ISSUE_LOADS(1, 0); ISSUE_LOADS(2, 0); ISSUE_LOADS(3, 0);
    }
    if (nin) {
      call = bump_get<8>(cnt);
      lds_wr(ctl, 2u);
      q0 = 4; pub = 2;
    } else {
      call = bump_get<0>(cnt);
    }

    if (!nin) sleep_until(tstart, (u32)(TQ(NS + (DIR ? 12 : 2)) + L1T));
    u64 ready = 0;
    u64 todo = ((1ull << (np - nin)) - 1ull) << (nid - (np - nin));
    u32 cons = 0;
    bool dead = false;
    for (int q = q0; q < np; ++q) {
      bool rdy = q < nin || (ready & todo) != 0ull;
      if (cons + 8u <= (u32)q) cons = lds_rd(ctl + 4u);
      if (!rdy || cons + 8u <= (u32)q) {
        waitv<0>();
        pub = (u32)q;
        lds_wr(ctl, pub);
        const u64 t0 = now();
        u32 win = 64u;
        for (;;) {
          if (!rdy) {
            u32 pv = 0;
            const u32 below = __builtin_amdgcn_mbcnt_hi((u32)(todo >> 32), __builtin_amdgcn_mbcnt_lo((u32)todo, 0u));
            if (((todo >> lane) & 1ull) && below < win) pv = atom_rd(poll_off, myws);
            win = (u32)PWIN;
            ready |= __builtin_amdgcn_ballot_w64(pv >= call);
            rdy = (ready & todo) != 0ull;
            if (LSL && !rdy) __builtin_amdgcn_s_sleep(LSL);
          } else {
            __builtin_amdgcn_s_sleep(2);
          }
          cons = lds_rd(ctl + 4u);
          if (rdy && cons + 8u > (u32)q) break;
          if (give_up(t0, myws, call)) { dead = true; break; }
        }
        if (dead) break;
      }
      int id = 0;
      if (q >= nin) {
        id = __builtin_ctzll(ready & todo);
        todo &= ~(1ull << id);
        lds_wr(ctl + 32u + (u32)(q & 7) * 4u, (u32)id);
      }
      ISSUE_LOADS(q, id);
      waitv<8>();
      if (PWHO == 1 && nin && q == PQ1) probe0((u32)(d - 2) * IN_SRC + 2u * MIB + (u32)p * 4096u, P.peer);
      if (PWHO == 1 && nin && q == PQ2) probe0(PIN_OFF + (call % 3u) * PIN_PAR + (u32)(32 + (d - 2) * 16 + NS) * PIECE + (u32)p * 4096u, P.peer);
      if (q - 1 > (int)pub) { pub = (u32)(q - 1); lds_wr(ctl, pub); }
      if (q == nin - 1) STAMP(myws, 1);
    }
    waitv<0>();
    if (dead) st4_lane0(TMO_OFF, call, myws);
    lds_wr(ctl, dead ? DEAD : (u32)np);
    STAMP(myws, 3);
    STAMPV(myws, 0, tstart);
    return;
  }

  char* const rws = P.peer;
  const int slot = d - 2;
  const u32 dl = (u32)p * 4096u + lane * 16u;
  u32 xl[4], xd[4];
#pragma unroll
  for (int k = 0; k < 4; ++k) {
    const u32 c = (u32)k * 64u + lane, t = c / (4u * XR), row = (c >> 2) % XR;
    xl[k] = (row * XC + t * 4u + ((c ^ ((xrow + row) >> 2)) & 3u)) * 16u;
    xd[k] = XCT ? dl : (xks + t) * 8192u + (xrow + row) * 64u + (c & 3u) * 16u - (u32)k * 1024u;
  }
  u32 tick, ts0;
  int to;
  {
    u32 tg, tm;
    asm volatile("" : "+v"(xl[0]), "+v"(xl[1]), "+v"(xl[2]), "+v"(xl[3]), "+v"(xd[0]), "+v"(xd[1]), "+v"(xd[2]), "+v"(xd[3]));
    asm volatile("s_waitcnt vmcnt(1)\n\t"
                 "v_readfirstlane_b32 %[tick], v160\n\t"
                 "v_readfirstlane_b32 %[tg], v161\n\t"
                 "s_memrealtime s[58:59]\n\t"
                 "s_sub_i32 %[tm], %[tg], %[tis]\n\t"
                 "s_addk_i32 %[tm], 500\n\t"
                 "s_cmpk_gt_u32 %[tm], 1000\n\t"
                 "s_cselect_b32 %[tg], %[tis], %[tg]\n\t"
                 "s_addk_i32 %[tg], %[oc]\n\t"
                 "s_sub_i32 %[to], %[tg], %[tst]\n\t"
                 "s_and_b32 %[tm], %[tick], 15\n\t"
                 "s_mul_i32 %[tm], %[tm], %[osp]\n\t"
                 "s_add_i32 %[tg], %[tg], %[tm]\n\t"
                 "s_bitcmp0_b32 %[tick], 4\n\t"
                 "s_cselect_b32 %[tm], 0x10000, 0\n\t"
                 "v_add_u32 v160, %[tm], %[dl]\n\t"
                 "s_cmp_eq_u32 %[d], 1\n\t"
                 "s_cbranch_scc1 5f\n"
                 "4:\n\t"
                 "s_waitcnt lgkmcnt(0)\n\t"
                 "s_sub_i32 %[tm], s58, %[tg]\n\t"
                 "s_mov_b32 %[ts0], s58\n\t"
                 "s_memrealtime s[58:59]\n\t"
                 "s_cmp_lt_i32 %[tm], 0\n\t"
                 "s_cbranch_scc1 4b\n\t"
                 "global_store_dwordx4 v160, v[144:147], %[dst] sc0 sc1\n\t"
                 "global_store_dwordx4 v160, v[148:151], %[dst] offset:1024 sc0 sc1\n\t"
                 "global_store_dwordx4 v160, v[152:155], %[dst] offset:2048 sc0 sc1\n\t"
                 "global_store_dwordx4 v160, v[156:159], %[dst] offset:3072 sc0 sc1\n"
                 "5:\n\t"
                 "s_waitcnt lgkmcnt(0)"
                 : [tick] "=&s"(tick), [ts0] "=&s"(ts0), [to] "=&s"(to), [tg] "=&s"(tg), [tm] "=&s"(tm)
                 : "{v[144:159]}"(P.f16), "{v[160:161]}"(P.co), [tis] "s"(P.tis), [tst] "s"((u32)tstart), [d] "s"(d),
                   [dl] "v"(dl), [dst] "s"(rws + (size_t)((u32)max(slot, 0) * IN_SRC)), [oc] "i"(OC), [osp] "i"(OSP)
                 : "memory", "scc", "s58", "s59");
  }
  const u32 call = (tick >> 4) + 1u;
  u32 dbg1 = 0, dbg2 = 0;
  const u32 in0 = (u32)slot * IN_SRC + (call & 1u) * PIECE;
  const u32 pin0 = PIN_OFF + (call % 3u) * PIN_PAR + (u32)(d == 1 ? 0 : 32 + slot * 16 + NS) * PIECE;
  int nflag = 0;
#define FLAG(QQ) do { \
    const int q_ = (QQ); \
    const bool real_ = q_ == nflag && q_ < nin; \
    const u32 o_ = real_ ? fin_off(slot, q_) : DUMMY_OFF + (u32)(blockIdx.x * 4 + wave) * 128u; \
    add1_lane0(o_, uni(real_ ? rws : myws)); \
    nflag += real_ ? 1 : 0; \
  } while (0)

  u32 tgrid = 0, tord = 0;
  if (nin) {
    FLAG(-1);
    const u32 tb = (u32)max(to + TAU + OG, (int)(ts0 - (u32)tstart) + OD) + (u32)(TAU - 1);
    tgrid = tb - ((u32)tstart + tb - ((u32)(((p + STGD * d) & 15) * TAU) >> 4)) % (u32)TAU;
    tord = ORD > 1 ? (u32)max(to + TAU + OG + (int)(tick & 15u) * OSP, 0) - tgrid : 0u;
  } else {
    sleep_until(tstart, 600u);
    probe0(pin0 + (u32)p * 4096u, rws);
    probe0(pin0 + (u32)(NOUT1 - 1) * PIECE + (u32)p * 4096u, rws);
  }
  u32 full = 0, tnext = 0;
  for (int q = nin ? 1 : 0; q < np; ++q) {
    bool isx; int ix;
    piece_of(q, d, isx, ix);
    isx = isx && q < nin;
    if (d != 1 || G1) tnext = max(tnext, tgrid + (q < ORD ? tord : 0u));
    if (!PRE) sleep_until(tstart, tnext);
    if (full <= (u32)q) {
      full = lds_rd(ctl);
      if (full <= (u32)q) {
        bool drained = !(q > 0 && q < nin + LAG);
        u64 el;
        for (;;) {
          __builtin_amdgcn_s_sleep(1);
          full = lds_rd(ctl);
          el = now() - tstart;
          if (full > (u32)q) break;
          if (!drained && __builtin_amdgcn_readfirstlane((int)(el > (u64)(tnext + 30u)))) {
            drained = true;
            waitv<0>();
#pragma unroll
            for (int z = LAG; z >= 1; --z) FLAG(q - z);
          }
          if (__builtin_amdgcn_readfirstlane((int)(el > 2ull * DEADLINE - 1000ull))) { full = DEAD; break; }
        }
        tnext = max(tnext, (u32)__builtin_amdgcn_readfirstlane((u32)el));
      }
      if (full == DEAD) break;
    }
    const u32 la = ring + (u32)(q & 7) * 4096u;
    const u32x4 d0 = *(volatile LDSP u32x4*)(uintptr_t)(la + (isx ? xl[0] : lane * 16u));
    const u32x4 d1 = *(volatile LDSP u32x4*)(uintptr_t)(la + (isx ? xl[1] : lane * 16u + 1024u));
    const u32x4 d2 = *(volatile LDSP u32x4*)(uintptr_t)(la + (isx ? xl[2] : lane * 16u + 2048u));
    const u32x4 d3 = *(volatile LDSP u32x4*)(uintptr_t)(la + (isx ? xl[3] : lane * 16u + 3072u));
    const u32 id = lds_rd(ctl + 32u + (u32)(q & 7) * 4u);
    lds_wr(ctl + 4u, (u32)q + 1u);
    u32 o;
    if (q < nin) {
      o = in0 + (u32)(isx ? cont_x(ix) : cont_w(ix)) * 2u * PIECE;
    } else {
      o = pin0 + min(id, (u32)(nid - 1)) * PIECE;
      if (d == 1 && id < (u32)SWP) o += (u32)(((p >> 3) - (int)(id & 1u)) * 32768);
    }
    char* const dst = rws + (size_t)o;
    if (PRE) sleep_exact(tstart, tnext);
    tgrid += (u32)(TAU - (q < QA ? DA : 0) - (q >= QB ? DB : 0) - (q >= QC ? DC : 0));
    tnext += (u32)(d == 1 && !G1 ? TAU1 : TAU2);
    st16<0>(isx ? xd[0] : dl, d0, dst);
    st16<1024>(isx ? xd[1] : dl, d1, dst);
    st16<2048>(isx ? xd[2] : dl, d2, dst);
    st16<3072>(isx ? xd[3] : dl, d3, dst);
    if (nin && q < nin + LAG) {
      waitv<5 * LAG>();
      FLAG(q - LAG);
      if (PWHO == 0 && q == PQ1) probe0((u32)slot * IN_SRC + 2u * MIB + (u32)p * 4096u, rws);
      if (PWHO == 0 && q == PQ2) probe0(pin0 + (u32)p * 4096u, rws);
    } else {
      waitv<4 * WIN>();
    }
    if (q == NS + 11) STAMP(myws, 1);
    if (q == nin - 1) STAMP(myws, 2);
    if (q == np - 1) STAMP(myws, 4);
  }
  if ((tick & 15u) == 0u) st4_lane0(LNK_OFF + (u32)d * 128u + 4u, 0u, myws);
  waitv<0>();
  STAMP(myws, 3);
  STAMPV(myws, 0, tstart);
}

struct Frag {
  bf16x8 x[2][2];
  bf16x8 w[2][2];
};

struct Loader {
  const char* base;
  u32 stride[2];
  u32 v[4];
  u32 m0base;
};

template <int STAGE> __device__ __forceinline__ void glds_pair(const Loader& L, int half) {
  const u32 m0v = L.m0base + STAGE * STAGE_B;
  if (half == 0)
    asm volatile("s_mov_b32 m0, %0\n\ts_nop 0\n\t"
                 "global_load_lds_dwordx4 %1, %3 " GLP "\n\t"
                 "global_load_lds_dwordx4 %2, %3 offset:1024 " GLP
                 :: "s"(m0v), "v"(L.v[0]), "v"(L.v[1]), "s"(L.base) : "memory");
  else
    asm volatile("s_mov_b32 m0, %0\n\ts_nop 0\n\t"
                 "global_load_lds_dwordx4 %1, %3 offset:2048 " GLP "\n\t"
                 "global_load_lds_dwordx4 %2, %3 offset:3072 " GLP
                 :: "s"(m0v), "v"(L.v[2]), "v"(L.v[3]), "s"(L.base) : "memory");
}

struct RdAddr { u32 xa[2]; u32 wa[2][2]; };

template <int STAGE> __device__ __forceinline__ bf16x8 rd_x(const RdAddr& A, int mt, int ks) {
  return *(const LDSP bf16x8*)(uintptr_t)(A.xa[ks] + (u32)(STAGE * STAGE_B + mt * 2048));
}
template <int STAGE> __device__ __forceinline__ bf16x8 rd_w(const RdAddr& A, int nt, int ks) {
  s16x4 lo = __builtin_amdgcn_ds_read_tr16_b64_v4i16((LDSP s16x4*)(uintptr_t)(A.wa[nt][0] + (u32)(STAGE * STAGE_B + 8192 + ks * 4096)));
  s16x4 hi = __builtin_amdgcn_ds_read_tr16_b64_v4i16((LDSP s16x4*)(uintptr_t)(A.wa[nt][1] + (u32)(STAGE * STAGE_B + 8192 + ks * 4096)));
  s16x8 v = __builtin_shufflevector(lo, hi, 0, 1, 2, 3, 4, 5, 6, 7);
  return __builtin_bit_cast(bf16x8, v);
}
#define MFMA(nt, mt, ks) acc[nt][mt] = __builtin_amdgcn_mfma_f32_32x32x16_bf16(cur.w[nt][ks], cur.x[mt][ks], acc[nt][mt], 0, 0, 0)

template <int S, int VM, bool LOAD, bool READ>
__device__ __forceinline__ void kstep(f32x16 (&acc)[2][2], const Frag& cur, Frag& nxt, const RdAddr& A, Loader& L) {
  constexpr int SN = (S + 1) & 3;
  if (READ) {
    __builtin_amdgcn_s_waitcnt((VM & 15) | (7 << 4) | (0 << 8) | ((VM >> 4) << 14));
    asm volatile("s_barrier" ::: "memory");
  } else {
    __builtin_amdgcn_s_waitcnt(15 | (7 << 4) | (0 << 8) | (3 << 14));
  }
  SB();
  MFMA(0, 0, 0); SB();
  if (READ) { nxt.w[0][0] = rd_w<SN>(A, 0, 0); nxt.x[0][0] = rd_x<SN>(A, 0, 0); } SB();
  MFMA(0, 1, 0); SB();
  if (READ) { nxt.x[1][0] = rd_x<SN>(A, 1, 0); nxt.w[1][0] = rd_w<SN>(A, 1, 0); } SB();
  MFMA(1, 0, 0); SB();
  if (LOAD) glds_pair<S>(L, 0);
  SB();
  MFMA(1, 1, 0); SB();
  if (READ) { nxt.w[0][1] = rd_w<SN>(A, 0, 1); nxt.x[0][1] = rd_x<SN>(A, 0, 1); } SB();
  MFMA(0, 0, 1); SB();
  if (READ) { nxt.x[1][1] = rd_x<SN>(A, 1, 1); nxt.w[1][1] = rd_w<SN>(A, 1, 1); } SB();
  MFMA(0, 1, 1); SB();
  if (LOAD) { glds_pair<S>(L, 1); L.base += L.stride[S & 1]; }
  SB();
  MFMA(1, 0, 1); SB();
  MFMA(1, 1, 1); SB();
}

template <int S> __device__ __forceinline__ void stage_load(Loader& L) {
  glds_pair<S>(L, 0); glds_pair<S>(L, 1); L.base += L.stride[S & 1];
}

__device__ __forceinline__ void gemm_phase(f32x16 (&acc)[2][2], const RdAddr& A, Loader& L, int n, int sk, u32 jump, u32& dbgv, int dbg0) {
  Frag f0, f1;
  L.base = uni(L.base + (sk == 0 ? IN_SRC : 0u));
  L.stride[0] = __builtin_amdgcn_readfirstlane(L.stride[0]);
  L.stride[1] = __builtin_amdgcn_readfirstlane(L.stride[1]);
  asm volatile("s_nop 4" ::: "memory");
  stage_load<0>(L); stage_load<1>(L); stage_load<2>(L); stage_load<3>(L);
  waitv<12>();
  asm volatile("s_barrier" ::: "memory");
#pragma unroll
  for (int ks = 0; ks < 2; ++ks)
#pragma unroll
    for (int i = 0; i < 2; ++i) { f0.w[i][ks] = rd_w<0>(A, i, ks); f0.x[i][ks] = rd_x<0>(A, i, ks); }
  SB();
  for (int t = 0; t < n - 4; t += 4) {
    L.base += (t & 4 ? jump : 0u) + (t + 4 == 8 * sk ? IN_SRC : 0u);
    kstep<0, 8, true, true>(acc, f0, f1, A, L);
    kstep<1, 8, true, true>(acc, f1, f0, A, L);
    kstep<2, 8, true, true>(acc, f0, f1, A, L);
    kstep<3, 8, true, true>(acc, f1, f0, A, L);
  }
  kstep<0, 8, false, true>(acc, f0, f1, A, L);
  kstep<1, 4, false, true>(acc, f1, f0, A, L);
  kstep<2, 0, false, true>(acc, f0, f1, A, L);
  kstep<3, 0, false, false>(acc, f1, f0, A, L);
}

__device__ __forceinline__ u32 pack2(float a, float b) {
  f32x2 t = {a, b};
  return __builtin_bit_cast(u32, __builtin_convertvector(t, bf16x2));
}
__device__ __forceinline__ u32 relu2(u32 v) {
  typedef short s16x2 __attribute__((ext_vector_type(2)));
  return __builtin_bit_cast(u32, __builtin_elementwise_max(__builtin_bit_cast(s16x2, v), s16x2{0, 0}));
}
__device__ __forceinline__ float bf_lo(u32 v) { return __builtin_bit_cast(float, v << 16); }
__device__ __forceinline__ float bf_hi(u32 v) { return __builtin_bit_cast(float, v & 0xffff0000u); }

__device__ __forceinline__ void zero8k(u32 l16, u32 l16b, char* pb) {
  const u32x4 z4 = {0u, 0u, 0u, 0u};
  asm volatile("s_nop 4" ::: "memory");
  st16<0>(l16, z4, pb); st16<1024>(l16, z4, pb); st16<2048>(l16, z4, pb); st16<3072>(l16, z4, pb);
  st16<0>(l16b, z4, pb); st16<1024>(l16b, z4, pb); st16<2048>(l16b, z4, pb); st16<3072>(l16b, z4, pb);
}

__device__ __forceinline__ void tile(const Args& P, int tid, int wave, u32 lane, char* smem, u64 tstart) {
  const int r = P.rank;
  char* const myws = P.my;
  const int g = r >> 2, b = (r >> 1) & 1, a = r & 1;
  const int bid = blockIdx.x;

  const u32 cnt = bump_issue(CNT_OFF + (u32)(bid * 4 + wave) * 64u, myws);

  int row0, col0, nrem, tskip, j, e, k, rb;
  int pi0 = -1, pi1 = -1;
  bool fin;
  if (tid < N_MAIN_WG) {
    const int xcd = tid & 7, o = tid >> 3, u = xcd >> 1;
    int rbi, cbi;
    if (u < 2) { rbi = o < 24 ? 1 + o % 3 : 0; cbi = o < 24 ? o / 3 : o - 24; }
    else { rbi = o & 3; cbi = o >> 2; }
    rb = u + 4 * rbi; k = cbi * 2 + (xcd & 1);
    j = rb >> 1; e = rb & 1;
    row0 = (r ^ j) * 512 + b * 256 + e * 128; col0 = g * 2048 + (a ? 15 - k : k) * 128;
    nrem = j >= 2 ? 40 : 48; tskip = j >= 2 ? j - 2 : 99;
    fin = j == 0;
    if (j == 0) pi1 = 6 * NS + 15 - k;
    else if (j >= 2 && k >= 16 - NS) pi0 = ((j ^ 1) - 2) * NS + 15 - k;
  } else {
    const int f = tid - N_MAIN_WG;
    const int qd = 1 + f / 32, rem = f & 31;
    e = (rem >> 3) & 1; k = (rem & 7) * 2 + (rem >> 4);
    const int h = b ^ (qd & 1), g2 = g ^ (qd >> 1);
    row0 = r * 512 + h * 256 + e * 128; col0 = g2 * 2048 + (a ? 15 - k : k) * 128;
    nrem = 0; tskip = 99; j = 0; rb = 0;
    fin = true;
    const int iA = k >= NS ? 32 + (2 * qd - 2) * 16 + k : -1;
    const int iB = 15 - k >= NS ? 32 + (2 * qd - 1) * 16 + 15 - k : -1;
    pi0 = k < 8 ? iA : iB;
    pi1 = k < 8 ? iB : iA;
  }

  const u32 sbase = (u32)(uintptr_t)smem;
  const int wm = wave >> 1, wn = wave & 1;
  RdAddr A;
  {
    const u32 h = lane >> 5, m = lane & 31u, f = (lane >> 2) & 3u;
    A.xa[0] = sbase + ((u32)wm * 64u + m) * 64u + 16u * (h ^ f);
    A.xa[1] = sbase + ((u32)wm * 64u + m) * 64u + 16u * ((2u + h) ^ f);
    const u32 blk = (lane >> 4) & 1u, q = (lane & 15u) >> 2, pp = lane & 3u;
#pragma unroll
    for (int nt = 0; nt < 2; ++nt)
#pragma unroll
      for (int t = 0; t < 2; ++t) {
        const u32 row = 8u * h + 4u * t + q;
        const u32 ch = (u32)wn * 8u + (u32)nt * 4u + 2u * blk + (pp & 1u);
        A.wa[nt][t] = sbase + 256u * row + 16u * (ch ^ (((row & 3u) << 2) | ((row >> 2) & 3u))) + 8u * (pp >> 1);
      }
  }

  f32x16 acc[2][2];
#pragma unroll
  for (int i = 0; i < 2; ++i)
#pragma unroll
    for (int k = 0; k < 2; ++k)
#pragma unroll
      for (int z = 0; z < 16; ++z) acc[i][k][z] = 0.f;

  const u32 tile_off = (u32)e * 32768u + (u32)wave * 8192u;
  const u32 l16 = lane * 16u, l16b = lane * 16u + 4096u;
  const u32 tw = sbase + (u32)wave * 8192u + (lane & 31u) * 128u + (((lane >> 5) ^ (lane & 7u)) * 16u);
  const u32 tr = sbase + (u32)wave * 8192u + (lane >> 3) * 128u + (((lane & 7u) ^ (lane >> 3)) * 16u);
  const u32 oo = (lane >> 3) * 8192u + (lane & 7u) * 16u;
  char* const ob = uni(P.out + (size_t)((u32)((row0 & 511) + wm * 64) * 8192u + (u32)(col0 + wn * 64) * 2u));
  const int nph = nrem ? 2 : 1;
  const int qin = max(slot_w(k), slot_x(j, e, j == 7 ? 2 : 7));
  const u32 tin = (u32)(EARLY + TQ(qin));
  const int addph = fin ? nph - 1 : 0;

  if (LPT && (bid >> 3) == 25 && wave == 0) {
    sleep_until(tstart, (u32)LPT);
    if (lane < 28u) probe(lane * 2u * MIB + MIB + (u32)(bid & 7) * 128u, myws);
  }
  if (RPT && (bid >> 3) == RPB && wave == 1) {
    const u32 dd = 1u + lane % 7u;
    const u64 pp = ((const GLBP u64*)(uintptr_t)__builtin_amdgcn_kernarg_segment_ptr())[6u + 2u * dd];
    const u32 cl = bump_get<0>(cnt);
    const u32 o = (lane < 7u && dd > 1u ? (dd - 2u) * IN_SRC + 2u * MIB : PIN_OFF + (cl % 3u) * PIN_PAR + (dd > 1u ? dd * 16u : 30u) * PIECE) + (u32)(bid & 7) * 128u;
    sleep_until(tstart, (u32)RPT);
    if (lane < 14u) asm volatile("global_atomic_add %0, %1, off sc1" :: "v"(pp + o), "v"(0u) : "memory");
  }
  if (PD0 && (fin || k >= NS)) sleep_until(tstart, (u32)(PD0 + k * PD1 + (fin ? PDF : 0)));
  Loader L;
  L.m0base = sbase + (u32)wave * 4096u;
  bool dead = false;
  u32 dbgv = 0;
  u32 call = 0;
  for (int ph = 0; ph < nph; ++ph) {
    int n, ts;
    bool skip = false;
    if (ph == 0) {
      n = 16; ts = 99;
      if (wave < 2) {
        L.base = P.x + (size_t)((u32)(row0 + wave * 64) * 1024u);
        L.stride[0] = L.stride[1] = 64u;
        const u32 c = (lane & 3u) ^ ((lane >> 4) & 3u);
#pragma unroll
        for (int k = 0; k < 4; ++k) L.v[k] = (lane >> 2) * 1024u + c * 16u + (u32)k * (16384u - 1024u);
      } else {
        L.base = P.w + (size_t)((u32)(wave - 2) * 16u * 8192u + (u32)col0 * 2u);
        L.stride[0] = L.stride[1] = 32u * 8192u;
        const u32 q = lane >> 4;
#pragma unroll
        for (int k = 0; k < 4; ++k) L.v[k] = q * 8192u + (((lane & 15u) ^ ((q << 2) | (u32)k)) * 16u) + (u32)k * (4u * 8192u - 1024u);
      }
    } else {
      n = nrem; ts = tskip;
      STAMP(myws, 1);
      if (wave == 0) {
        const int sl = (int)min(lane, 5u);
        const u32 fo = fin_off(sl, max(slot_w(k), slot_x(j, e, sl + 2)));
        bool pend = lane < 6u && sl + 2 != j;
        sleep_until(tstart, tin);
        const u64 t0 = now();
        u32 st = 0;
        for (;;) {
          if (pend) pend = atom_rd(fo, myws) < 16u * call;
          if (__builtin_amdgcn_ballot_w64(pend) == 0ull) break;
          if (give_up(t0, myws, call)) {
            if (lane == 0) st_agt(myws + TMO_OFF, call);
            st = 1;
            break;
          }
          __builtin_amdgcn_s_sleep(PSL);
        }
        if (lane == 0) *(LDSP u32*)(uintptr_t)(sbase + 4u * STAGE_B) = st;
        __builtin_amdgcn_s_waitcnt(0);
      }
      asm volatile("s_barrier" ::: "memory");
      const bool wgdead = lds_rd(sbase + 4u * STAGE_B) != 0;
      STAMP(myws, 2);
      dead = dead || wgdead;
      skip = wgdead;
      const u32 par = call & 1u;
      static_assert(!XCT || XR == 32, "layout of the x pieces");
      const bool xc = XCT && wave < 2;
      L.base = myws + (size_t)(((u32)(wave < 2 ? cont_x(rb) : cont_w(k)) * 2u + par) * PIECE + (u32)(wave & 1) * (xc ? 8192u : 4096u));
      L.stride[0] = xc ? 2048u : 8192u;
      L.stride[1] = xc ? 14336u : 8192u;
#pragma unroll
      for (int k = 0; k < 4; ++k) L.v[k] = lane * 16u + (xc && k >= 2 ? 2048u : 0u);
    }
    if (!skip) gemm_phase(acc, A, L, n, ts, ph ? IN_SRC - PIECE : 0u, dbgv, ph * 8);
    if (ph == nph - 1) STAMP(myws, 3);
    if (ph == 0) {
      call = bump_get<0>(cnt);
      if (ZT0 && max(pi0, pi1) >= 0) {
        char* const pz = myws + (size_t)(PIN_OFF + ((call + 2u) % 3u) * PIN_PAR + tile_off);
        sleep_until(tstart, (u32)ZT0 + (u32)((tid >> 3) & 15) * (u32)ZDT);
        if (pi0 >= 0) zero8k(l16, l16b, uni(pz + (size_t)((u32)pi0 * PIECE)));
        if (pi1 >= 0) zero8k(l16, l16b, uni(pz + (size_t)((u32)pi1 * PIECE)));
      }
    }
    const u32 pin = PIN_OFF + (call % 3u) * PIN_PAR + tile_off;
#pragma unroll 1
    for (int s = 0; s < 2; ++s) {
      const int pi = ph != addph ? -1 : (s == 0 ? pi0 : pi1);
      if (pi < 0 && !(fin && ph == 0 && s == 0)) continue;
      u32x4 pv[8];
#pragma unroll
      for (int i = 0; i < 8; ++i) pv[i] = u32x4{0u, 0u, 0u, 0u};
      if (pi >= 0) {
        char* const pb = uni(myws + (size_t)(pin + (u32)pi * PIECE));
        const bool first = pi == 0 || (pi >= NOUT1 && (pi & 15) == NS);
        const u32 wo = pin + (u32)(pi - ((!WOWN && (lane & 2u) && !first) ? 1 : 0)) * PIECE + (lane & 1u) * 4096u + 4092u;
        sleep_until(tstart, pi >= NOUT1 ? (u32)(EARLY + TQ(NIN - 2 + (pi & 15) - NS)) : (fin ? 0u : tin - 300u));
        const u64 t0 = now();
        while (!dead) {
          u32 v = WOWN ? 1u : 0u;
          if (lane < (WOWN ? 2u : 4u)) v = atom_rd(wo, myws);
          if (WOWN ? __builtin_amdgcn_ballot_w64(v == 0u) == 0ull : __builtin_amdgcn_ballot_w64(v != 0u) != 0ull) break;
          if (give_up(t0, myws, call)) { dead = true; break; }
          __builtin_amdgcn_s_sleep(WSL);
        }
        if (s == 1) STAMP(myws, 4);
        for (;;) {
          if (__builtin_expect(dead, 0)) {
            if (lane == 0) st_agt(myws + TMO_OFF, call);
#pragma unroll
            for (int i = 0; i < 8; ++i) pv[i] = u32x4{0x7fc07fc0u, 0x7fc07fc0u, 0x7fc07fc0u, 0x7fc07fc0u};
            break;
          }
          asm volatile("s_nop 4\n\t"
                       "global_load_dwordx4 %0, %8, %10 sc0 sc1\n\t"
                       "global_load_dwordx4 %1, %8, %10 offset:1024 sc0 sc1\n\t"
                       "global_load_dwordx4 %2, %8, %10 offset:2048 sc0 sc1\n\t"
                       "global_load_dwordx4 %3, %8, %10 offset:3072 sc0 sc1\n\t"
                       "global_load_dwordx4 %4, %9, %10 sc0 sc1\n\t"
                       "global_load_dwordx4 %5, %9, %10 offset:1024 sc0 sc1\n\t"
                       "global_load_dwordx4 %6, %9, %10 offset:2048 sc0 sc1\n\t"
                       "global_load_dwordx4 %7, %9, %10 offset:3072 sc0 sc1"
                       : "=&v"(pv[0]), "=&v"(pv[1]), "=&v"(pv[2]), "=&v"(pv[3]), "=&v"(pv[4]), "=&v"(pv[5]), "=&v"(pv[6]), "=&v"(pv[7])
                       : "v"(l16), "v"(l16b), "s"(pb) : "memory");
          const bool late = __builtin_amdgcn_readfirstlane((int)(now() - t0 > 50000ull)) != 0;
          asm volatile("s_waitcnt vmcnt(0)"
                       : "+v"(pv[0]), "+v"(pv[1]), "+v"(pv[2]), "+v"(pv[3]), "+v"(pv[4]), "+v"(pv[5]), "+v"(pv[6]), "+v"(pv[7]) :: "memory");
          u32 mn = 0xffffffffu;
#pragma unroll
          for (int i = 0; i < 8; ++i)
#pragma unroll
            for (int z = 0; z < 4; ++z) mn = min(mn, pv[i][z]);
          if (__builtin_amdgcn_ballot_w64(mn == 0u) == 0ull) break;
          if (__builtin_expect(late, 0)) dead = give_up(t0, myws, call);
        }
      }
#pragma unroll
      for (int nt = 0; nt < 2; ++nt)
#pragma unroll
        for (int mt = 0; mt < 2; ++mt)
#pragma unroll
          for (int sx = 0; sx < 2; ++sx)
#pragma unroll
            for (int z = 0; z < 4; ++z) {
              const u32 u = pv[(nt * 2 + mt) * 2 + sx][z];
              acc[nt][mt][8 * sx + 2 * z] += bf_lo(u);
              acc[nt][mt][8 * sx + 2 * z + 1] += bf_hi(u);
            }
      if (!fin) continue;
      if (s == 1) STAMP(myws, 6);
#pragma unroll
      for (int nt = 0; nt < 2; ++nt)
#pragma unroll
        for (int mt = 0; mt < 2; ++mt)
#pragma unroll
          for (int sx = 0; sx < 2; ++sx) {
            u32x4 v;
#pragma unroll
            for (int z = 0; z < 4; ++z) v[z] = relu2(pack2(acc[nt][mt][8 * sx + 2 * z], acc[nt][mt][8 * sx + 2 * z + 1]));
            *(volatile LDSP u32x4*)(uintptr_t)((tw ^ (u32)((nt * 4 + sx * 2) * 16)) + (u32)mt * 4096u) = v;
          }
      u32 o1, o2, o3, o4, o5, o6, o7;
      asm volatile("ds_read_b128 %0, %15\n\t"
                   "ds_read_b128 %1, %15 offset:1024\n\t"
                   "ds_read_b128 %2, %15 offset:2048\n\t"
                   "ds_read_b128 %3, %15 offset:3072\n\t"
                   "ds_read_b128 %4, %15 offset:4096\n\t"
                   "ds_read_b128 %5, %15 offset:5120\n\t"
                   "ds_read_b128 %6, %15 offset:6144\n\t"
                   "ds_read_b128 %7, %15 offset:7168\n\t"
                   "v_add_u32 %8, 0x10000, %16\n\t"
                   "v_add_u32 %9, 0x20000, %16\n\t"
                   "v_add_u32 %10, 0x30000, %16\n\t"
                   "v_add_u32 %11, 0x40000, %16\n\t"
                   "v_add_u32 %12, 0x50000, %16\n\t"
                   "v_add_u32 %13, 0x60000, %16\n\t"
                   "v_add_u32 %14, 0x70000, %16\n\t"
                   "s_mov_b64 exec, %18\n\t"
                   "s_waitcnt lgkmcnt(7)\n\tglobal_store_dwordx4 %16, %0, %17 sc0 sc1\n\t"
                   "s_waitcnt lgkmcnt(6)\n\tglobal_store_dwordx4 %8, %1, %17 sc0 sc1\n\t"
                   "s_waitcnt lgkmcnt(5)\n\tglobal_store_dwordx4 %9, %2, %17 sc0 sc1\n\t"
                   "s_waitcnt lgkmcnt(4)\n\tglobal_store_dwordx4 %10, %3, %17 sc0 sc1\n\t"
                   "s_waitcnt lgkmcnt(3)\n\tglobal_store_dwordx4 %11, %4, %17 sc0 sc1\n\t"
                   "s_waitcnt lgkmcnt(2)\n\tglobal_store_dwordx4 %12, %5, %17 sc0 sc1\n\t"
                   "s_waitcnt lgkmcnt(1)\n\tglobal_store_dwordx4 %13, %6, %17 sc0 sc1\n\t"
                   "s_waitcnt lgkmcnt(0)\n\tglobal_store_dwordx4 %14, %7, %17 sc0 sc1\n\t"
                   "s_cmp_lg_u64 %18, -1\n\t"
                   "s_cbranch_scc1 1f\n\t"
                   "s_endpgm\n"
                   "1:\n\t"
                   "s_mov_b64 exec, -1\n\t"
                   "s_nop 1"
                   : "=&v"(pv[0]), "=&v"(pv[1]), "=&v"(pv[2]), "=&v"(pv[3]), "=&v"(pv[4]), "=&v"(pv[5]), "=&v"(pv[6]), "=&v"(pv[7]),
                     "=&v"(o1), "=&v"(o2), "=&v"(o3), "=&v"(o4), "=&v"(o5), "=&v"(o6), "=&v"(o7)
                   : "v"(tr), "v"(oo), "s"(ob), "s"(s == 1 ? ~0ull : (u64)WMK) : "memory", "scc");
      if (s == 0) waitv<0>();
    }
  }

  if (!fin) {
    STAMP(myws, 6);
    const bool dir = DIR && j >= 2 && k < NS;
    char* dst = uni(dir ? P.peer + (size_t)(PIN_OFF + (call % 3u) * PIN_PAR + (u32)((j - 2) * NS + k) * PIECE + tile_off)
                        : myws + (size_t)(POUT_OFF + (u32)((j - 1) * 16 + k) * PIECE + tile_off));
    char* fl = myws + (size_t)(FLOC_OFF + (u32)((j - 1) * 8 + e * 4 + wave) * 128u + (u32)k * 4u);
    u32 least = 1u;
    if (__builtin_expect(dead, 0)) {
      least = 0x7fc07fc0u;
#pragma unroll
      for (int i = 0; i < 64; ++i) acc[i >> 5][(i >> 4) & 1][i & 15] = 0.f;
    }
    asm volatile("s_nop 4" ::: "memory");
#pragma unroll
    for (int nt = 0; nt < 2; ++nt)
#pragma unroll
      for (int mt = 0; mt < 2; ++mt) {
        u32x4 v0, v1;
#pragma unroll
        for (int z = 0; z < 4; ++z) {
          v0[z] = max(pack2(acc[nt][mt][2 * z], acc[nt][mt][2 * z + 1]), least);
          v1[z] = max(pack2(acc[nt][mt][8 + 2 * z], acc[nt][mt][8 + 2 * z + 1]), least);
        }
        const u32 o = nt ? l16b : l16;
        if (mt == 0) st16<0>(o, v0, dst); else st16<2048>(o, v0, dst);
        if (dir) __builtin_amdgcn_s_sleep(DSL);
        if (mt == 0) st16<1024>(o, v1, dst); else st16<3072>(o, v1, dst);
        if (dir) __builtin_amdgcn_s_sleep(DSL);
      }
    waitv<0>();
    if (lane == 0) st_sys(fl, call);
  }
  STAMP(myws, 5);
  STAMPV(myws, 0, tstart);
}

__global__ __launch_bounds__(256, 2) void fused_gemm_rs(Params P) {
  extern __shared__ __attribute__((aligned(16))) char smem[];
  const int bid = blockIdx.x;
  typedef u64 u64x4 __attribute__((ext_vector_type(4)));
  u64x4 a4;
  u64 tstart, pe;
  int rk, d;
  u32 tis, t0;
  u32x16 f16;
  u32x2 co;
  static_assert(N_PUSH_WG == 56 && PROT == 0 && PWN == 1 && CPB >= N_PUSH_WG, "entry code");
  asm volatile("v_readfirstlane_b32 s46, %[tid]\n\t"
               "s_load_dwordx8 s[36:43], %[ka], 0x0\n\t"
               "s_lshr_b32 s47, %[bid], 3\n\t"
               "s_cmp_gt_u32 %[bid], 55\n\t"
               "s_cbranch_scc1 2f\n\t"
               "s_lshr_b32 s48, s46, 7\n\t"
               "s_mul_i32 s49, s48, 3\n\t"
               "s_add_i32 s47, s47, s49\n\t"
               "s_add_i32 s49, s47, -7\n\t"
               "s_min_u32 s47, s47, s49\n\t"
               "s_lshl_b32 s49, s47, 4\n\t"
               "s_load_dwordx4 s[52:55], %[ka], s49 offset:0x40\n\t"
               "s_memrealtime s[44:45]\n\t"
               "s_load_dword s62, %[ka], 0x20\n\t"
               "s_setprio 3\n\t"
               "s_add_i32 s63, s47, 1\n\t"
               "v_lshlrev_b32 v1, 2, %[tid]\n\t"
               "v_mov_b32 v27, 0\n\t"
               "v_add_u32 v1, %[ctl], v1\n\t"
               "s_bitcmp1_b32 s46, 6\n\t"
               "ds_write_b32 v1, v27\n\t"
               "s_cbranch_scc0 1f\n\t"
               "s_and_b32 s49, %[bid], 7\n\t"
               "s_lshl1_add_u32 s48, s49, s48\n\t"
               "v_bfe_u32 v1, %[tid], 4, 2\n\t"
               "v_and_b32 v2, 15, %[tid]\n\t"
               "v_lshlrev_b32 v3, 2, v1\n\t"
               "v_lshlrev_b32 v1, 13, v1\n\t"
               "v_xor_b32 v2, v3, v2\n\t"
               "v_lshl_or_b32 v1, s48, 17, v1\n\t"
               "v_lshl_or_b32 v20, v2, 4, v1\n\t"
               "v_xor_b32 v21, 16, v20\n\t"
               "v_xor_b32 v22, 32, v20\n\t"
               "v_xor_b32 v23, 48, v20\n\t"
               "v_add_u32 v21, 0x7c00, v21\n\t"
               "v_add_u32 v22, 0xf800, v22\n\t"
               "v_add_u32 v23, 0x17400, v23\n\t"
               "s_lshl_b32 s49, s63, 7\n\t"
               "s_add_i32 s49, s49, %[lnk]\n\t"
               "v_mov_b32 v28, s49\n\t"
               "v_mov_b32 v25, 1\n\t"
               "s_waitcnt lgkmcnt(0)\n\t"
               "s_memrealtime s[58:59]\n\t"
               "global_load_dwordx4 v[144:147], v20, s[54:55] " MLP "\n\t"
               "global_load_dwordx4 v[148:151], v21, s[54:55] offset:1024 " MLP "\n\t"
               "global_load_dwordx4 v[152:155], v22, s[54:55] offset:2048 " MLP "\n\t"
               "global_load_dwordx4 v[156:159], v23, s[54:55] offset:3072 " MLP "\n\t"
               "s_mov_b64 exec, 1\n\t"
               "global_atomic_add v160, v28, v25, s[40:41] sc0 sc1\n\t"
               "s_add_i32 s50, s47, -1\n\t"
               "s_max_i32 s50, s50, 0\n\t"
               "s_mul_i32 s51, s50, %[fsl]\n\t"
               "s_lshl_b32 s49, s48, 7\n\t"
               "s_add_i32 s51, s51, s49\n\t"
               "s_add_i32 s51, s51, %[f0]\n\t"
               "v_mov_b32 v29, s51\n\t"
               "s_waitcnt lgkmcnt(0)\n\t"
               "s_or_b32 s66, s58, 1\n\t"
               "v_mov_b32 v26, s66\n\t"
               "global_atomic_cmpswap v161, v28, v[26:27], s[40:41] offset:4 sc0 sc1\n\t"
               "global_atomic_add v29, v27, s[52:53] sc1\n\t"
               "s_mov_b64 exec, -1\n"
               "1:\n\t"
               "s_waitcnt lgkmcnt(0)\n\t"
               "s_barrier\n\t"
               "s_branch 5f\n"
               "2:\n\t"
               "s_memrealtime s[44:45]\n\t"
               "s_load_dword s62, %[ka], 0x20\n\t"
               "s_load_dwordx2 s[52:53], %[ka], 0x40\n\t"
               "s_mov_b32 s63, 1\n\t"
               "s_add_i32 s47, %[bid], %[cpb]\n\t"
               "s_cmp_lt_u32 s47, 16\n\t"
               "s_cbranch_scc0 5f\n\t"
               "s_cmpk_lt_u32 s46, 0xc0\n\t"
               "s_cbranch_scc0 5f\n\t"
               "s_getpc_b64 s[56:57]\n\t"
               "s_mov_b32 m0, %[cpl]\n\t"
               "v_lshlrev_b32 v1, 7, %[tid]\n\t"
               "v_min_u32 v1, %[cpf], v1\n\t"
               "global_load_lds_dword v1, s[56:57]\n"
               "5:\n\t"
               "s_waitcnt lgkmcnt(0)"
               : "={s[36:43]}"(a4), "={s[44:45]}"(tstart), "={s46}"(t0), "={s[52:53]}"(pe), "={s62}"(rk), "={s63}"(d),
                 "={s66}"(tis)
               : [ka] "s"(__builtin_amdgcn_kernarg_segment_ptr()), [bid] "s"(bid), [tid] "v"(threadIdx.x),
                 [ctl] "i"(65536), [lnk] "i"(LNK_OFF), [fsl] "i"(IN_SRC + 4u * PIECE), [f0] "i"(NS * 2u * PIECE + 4096u),
                 [cpb] "i"(-CPB), [cpl] "i"(66048), [cpf] "i"(CPF - 128)
               : "memory", "scc", "s47", "s48", "s49", "s50", "s51", "s54", "s55", "s56", "s57", "s58", "s59",
                 "v1", "v2", "v3", "v20", "v21", "v22", "v23", "v25", "v26", "v27", "v28", "v29");
  asm volatile("" : "={v[144:159]}"(f16), "={v[160:161]}"(co));
  const u32 lane = threadIdx.x & 63u;
  const int wave = (int)(t0 >> 6);
  const Args A = {(const char*)a4[0], (const char*)a4[1], (char*)a4[2], (char*)a4[3], (char*)pe, rk, tis, f16, co};
  if (bid < N_PUSH_WG) mover(A, d, wave, lane, smem, tstart);
  else if (bid >= 256 && bid < 256 + N_PUSH_WG) {
    for (int i = 0; i < 6; ++i) __builtin_amdgcn_s_sleep(16);
  } else tile(A, bid < 256 - ROT ? bid - N_PUSH_WG + ROT : (bid < 256 ? bid - (256 - ROT) : bid - 2 * N_PUSH_WG), wave, lane, smem, tstart);
}

}

extern "C" void kernel_launch(int rank, int nranks, ncclComm_t comm,
                              hipStream_t stream,
                              void* const* d_in, const int* in_sizes, int n_in,
                              void* d_out, int out_size,
                              const int* in_dtypes, int out_dtype,
                              void* const* d_out_all,
                              void* const* d_ws_all, size_t ws_bytes) {
  if (nranks != NR || n_in < 2 || ws_bytes < (size_t)WS_NEED || (unsigned)rank >= (unsigned)NR) {
    fprintf(stderr, "unsupported configuration\n");
    return;
  }
  Params P;
  P.x = (const char*)d_in[0];
  P.w = (const char*)d_in[1];
  P.out = (char*)d_out;
  P.my = (char*)d_ws_all[rank];
  for (int i = 0; i < NR; ++i) {
    const int r = rank ^ i;
    P.lk[i].peer = (char*)d_ws_all[r];
    P.lk[i].w0 = P.w + (size_t)((r & 1) * (256 * 8192 + 15 * 256) + (r >> 2) * 4096);
  }
  P.rank = rank;
  P.pad[0] = P.pad[1] = P.pad[2] = 0;
  HIP_CHECK(hipFuncSetAttribute((const void*)fused_gemm_rs, hipFuncAttributeMaxDynamicSharedMemorySize, (int)LDS_BYTES));
  hipLaunchKernelGGL(fused_gemm_rs, dim3(N_WG), dim3(256), LDS_BYTES, stream, P);
  HIP_CHECK(hipGetLastError());
}
```

```cpp
#include <hip/hip_runtime.h>
#include <rccl/rccl.h>
#include <cstdio>
#include <cstdint>

#define HIP_CHECK(expr) do { hipError_t _e = (expr); if (_e != hipSuccess) { \
    fprintf(stderr, "HIP error %s at %s:%d\n", hipGetErrorString(_e), __FILE__, __LINE__); \
    return; } } while (0)

namespace {

typedef unsigned u32;
typedef unsigned long long u64;
typedef u32 u32x4 __attribute__((ext_vector_type(4)));
typedef u32 u32x2 __attribute__((ext_vector_type(2)));
typedef u32 u32x16 __attribute__((ext_vector_type(16)));
typedef float f32x2 __attribute__((ext_vector_type(2)));
typedef float f32x16 __attribute__((ext_vector_type(16)));
typedef __bf16 bf16x2 __attribute__((ext_vector_type(2)));
typedef __bf16 bf16x8 __attribute__((ext_vector_type(8)));
typedef short s16x4 __attribute__((ext_vector_type(4)));
typedef short s16x8 __attribute__((ext_vector_type(8)));
#define LDSP __attribute__((address_space(3)))
#define GLBP __attribute__((address_space(1)))

constexpr int NR = 8;
constexpr u32 PIECE = 65536u;
constexpr u32 MIB = 1u << 20;
#define NS 2
#define OX 14
#define TAU 112
#define T0 20
#define QA 0
#define DA 0
#define QB 6
#define DB 1
#define QC 99
#define DC 0
#define TQ(q) (T0 + TAU * (q) - DA * ((q) < QA ? (q) : QA) - DB * ((q) > QB ? (q) - QB : 0))
#define FQ 99
#define FM 1
#define FLQ(q) ((q) >= FQ ? min((q) | FM, NIN - 1) : (q))
#define AG0 4
#define AG1 28
#define QE 16
#define DE 2
#define SHD 5
#define SHT 0
#define SHQ 2
#define SHN 1
#define SHR 1
#define SHP 0
#define EARLY 150
#define TAU1 110
#define TAU2 0
#define STGD 2
#define PSL 4
#define LSL 0
#define WSL 3
#define RSL 0
#define WOWN 0
#define ALN 1
#define ALO 99
#define ALT 25
#define PWN 1
#define DIR 0
#define DSL 4
#define SWP 14
#define CPF 14592
#define CPB 56
#define PD0 0
#define PD1 0
#define PDF 0
#define GSH -18
#define ORG 1
#define OC 95
#define OG 18
#define OSP 7
#define ORD 1
#define OD 20
#define LPT 400
#define ROT 48
#define PWIN 64
#define PWHO 1
#define PQ1 9
#define PQ2 14
#define PROT 0
#define XR 32
#define ZT0 900
#define ZDT 100
#define RPT 400
#define RPB 25
#define XCT 0
#define L1T 500
#define SXT 6
#define PRE 1
#define WMK 1ull
#define G1 0
#define LAG 1
#define WIN 3
#define GLP "sc0 sc1"
#define MLP "sc0 sc1"
constexpr int NIN = 30;
constexpr int NOUT = 16 - NS;
constexpr int NOUT1 = 16 + 6 * NS;
constexpr u32 IN_SRC = 4u * MIB;
constexpr u32 PIN_OFF = 24u * MIB;
constexpr u32 PIN_PAR = 8u * MIB;
constexpr u32 POUT_OFF = 48u * MIB;
constexpr u32 FLG_OFF = 55u * MIB;
constexpr u32 FLOC_OFF = FLG_OFF + 16384u;
constexpr u32 TMO_OFF = FLG_OFF + 49152u;
constexpr u32 LNK_OFF = FLG_OFF + MIB + 262144u + 131072u;
constexpr u32 DUMMY_OFF = FLG_OFF + 65536u;
constexpr u32 DBG_OFF = FLG_OFF + 131072u;
constexpr u32 CNT_OFF = FLG_OFF + MIB + 262144u;
constexpr u32 WS_NEED = FLG_OFF + 2u * MIB;

constexpr int N_PUSH_WG = 56;
constexpr int N_MAIN_WG = 256;
constexpr int N_FIN_WG = 96;
constexpr int N_WG = 2 * N_PUSH_WG + N_MAIN_WG + N_FIN_WG;
constexpr u32 STAGE_B = 16384u;
constexpr u32 LDS_BYTES = 4u * STAGE_B + 1024u;
constexpr u64 DEADLINE = 100000000ull;
constexpr u32 DEAD = 0x7fffffffu;

struct Params {
  const char* x;
  const char* w;
  char* my;
  char* out;
  int rank;
  int pad[3];
  struct { char* peer; const char* w0; } lk[NR];
};
struct Args { const char* x; const char* w; char* my; char* out; char* peer; int rank; u32 tis; u32x16 f16; u32x2 co; };

#define SB() __builtin_amdgcn_sched_barrier(0)
template <int N> __device__ __forceinline__ void waitv() { asm volatile("s_waitcnt vmcnt(%0)" :: "i"(N) : "memory"); }

__device__ __forceinline__ u64 now() { return __builtin_amdgcn_s_memrealtime(); }
template <typename T> __device__ __forceinline__ T* uni(T* p) {
  const u64 v = (u64)(uintptr_t)p;
  const u32 lo = __builtin_amdgcn_readfirstlane((u32)v), hi = __builtin_amdgcn_readfirstlane((u32)(v >> 32));
  return (T*)(uintptr_t)(((u64)hi << 32) | lo);
}
__device__ __forceinline__ u32 ld_sys(const void* p) {
  return __hip_atomic_load((const GLBP u32*)(uintptr_t)p, __ATOMIC_RELAXED, __HIP_MEMORY_SCOPE_SYSTEM);
}
__device__ __forceinline__ void st_sys(void* p, u32 v) {
  __hip_atomic_store((GLBP u32*)(uintptr_t)p, v, __ATOMIC_RELAXED, __HIP_MEMORY_SCOPE_SYSTEM);
}
__device__ __forceinline__ void st_agt(void* p, u32 v) {
  __hip_atomic_store((GLBP u32*)(uintptr_t)p, v, __ATOMIC_RELAXED, __HIP_MEMORY_SCOPE_AGENT);
}
__device__ __forceinline__ bool give_up(u64 t0, const char* ws, u32 call) {
  const u64 dt = now() - t0;
  if (!__builtin_amdgcn_readfirstlane((int)(dt > 50000ull))) return false;
  __builtin_amdgcn_s_sleep(64);
  if (__builtin_amdgcn_readfirstlane((int)(dt > DEADLINE))) return true;
  return __builtin_amdgcn_readfirstlane(ld_sys(ws + TMO_OFF)) == call;
}
__device__ __forceinline__ void sleep_until(u64 tstart, u32 t) {
  for (;;) {
    const u32 el = (u32)__builtin_amdgcn_readfirstlane((u32)(now() - tstart));
    if (el >= t) break;
    if (t - el > 40u) __builtin_amdgcn_s_sleep(12); else __builtin_amdgcn_s_sleep(1);
  }
}
__device__ __forceinline__ void sleep_exact(u64 tstart, u32 t) {
  for (;;) {
    const u32 el = (u32)__builtin_amdgcn_readfirstlane((u32)(now() - tstart));
    if (el >= t) break;
    if (t - el > 60u) __builtin_amdgcn_s_sleep(12);
    else if (t - el > 20u) __builtin_amdgcn_s_sleep(3);
    else if (t - el > (u32)SXT) __builtin_amdgcn_s_sleep(1);
  }
}
template <int IMM> __device__ __forceinline__ void st16(u32 off, u32x4 v, char* base) {
  asm volatile("global_store_dwordx4 %0, %1, %2 offset:%3 sc0 sc1\n\ts_nop 1" :: "v"(off), "v"(v), "s"(base), "i"(IMM) : "memory");
}
template <int IMM> __device__ __forceinline__ void st16p(u32 off, u32x4 v, char* base) {
  asm volatile("global_store_dwordx4 %0, %1, %2 offset:%3\n\ts_nop 1" :: "v"(off), "v"(v), "s"(base), "i"(IMM) : "memory");
}
#define OST st16
__device__ __forceinline__ void dma16x4(u32 lds, const u32 (&o)[4], const char* base) {
  asm volatile("s_mov_b32 m0, %0\n\ts_nop 3\n\t"
               "global_load_lds_dwordx4 %1, %5 " MLP "\n\t"
               "global_load_lds_dwordx4 %2, %5 offset:1024 " MLP "\n\t"
               "global_load_lds_dwordx4 %3, %5 offset:2048 " MLP "\n\t"
               "global_load_lds_dwordx4 %4, %5 offset:3072 " MLP
               :: "s"(lds), "v"(o[0]), "v"(o[1]), "v"(o[2]), "v"(o[3]), "s"(base) : "memory");
}
__device__ __forceinline__ void probe(u32 off, char* base) {
  asm volatile("s_nop 4\n\tglobal_atomic_add %0, %1, %2 sc1" :: "v"(off), "v"(0u), "s"(base) : "memory");
}
__device__ __forceinline__ void probe0(u32 off, char* base) {
  asm volatile("s_mov_b64 exec, 1\n\ts_nop 3\n\tglobal_atomic_add %0, %1, %2 sc1\n\ts_mov_b64 exec, -1"
               :: "v"(off), "v"(0u), "s"(base) : "memory");
}
__device__ __forceinline__ u32 atom_rd(u32 off, const char* base) {
  u32 r;
  asm volatile("global_atomic_add %0, %1, %2, %3 sc0 sc1\n\ts_waitcnt vmcnt(0)" : "=&v"(r) : "v"(off), "v"(0u), "s"(base) : "memory");
  return r;
}
__device__ __forceinline__ u32 bump_issue(u32 off, const char* base) {
  u32 r;
  asm volatile("s_mov_b64 exec, 1\n\ts_nop 3\n\tglobal_atomic_add %0, %1, %2, %3 sc0 sc1\n\ts_mov_b64 exec, -1"
               : "=&v"(r) : "v"(off), "v"(1u), "s"(base) : "memory");
  return r;
}
template <int N> __device__ __forceinline__ u32 bump_get(u32 r) {
  u32 v;
  asm volatile("s_waitcnt vmcnt(%2)\n\tv_readfirstlane_b32 %0, %1\n\ts_nop 4" : "=s"(v) : "v"(r), "i"(N) : "memory");
  return v + 1u;
}
__device__ __forceinline__ void add1_lane0(u32 off, char* base) {
  asm volatile("s_mov_b64 exec, 1\n\ts_nop 3\n\tglobal_atomic_add %0, %1, %2 sc1\n\ts_mov_b64 exec, -1"
               :: "v"(off), "v"(1u), "s"(base) : "memory");
}
__device__ __forceinline__ void st4_lane0(u32 off, u32 val, char* base) {
  asm volatile("s_mov_b64 exec, 1\n\ts_nop 3\n\tglobal_store_dword %0, %1, %2 sc0 sc1\n\ts_mov_b64 exec, -1"
               :: "v"(off), "v"(val), "s"(base) : "memory");
}

#define STAMP(ws, idx) do {} while (0)
#define STAMPV(ws, idx, v) do {} while (0)

__device__ __forceinline__ void piece_of(int q, int d, bool& isx, int& idx) {
  const int t = q - NS, u = q - NS - 12;
  int j = (t >> 1) + 2;
  j += j >= d ? 1 : 0;
  j = j == 8 ? 1 : j;
  isx = (q >= NS && u < 0) || (u >= OX && u < OX + 2);
  idx = q < NS ? q : (u < 0 ? 2 * j + (t & 1) : (u < OX ? NS + u : (u < OX + 2 ? u - OX : NS + u - 2)));
}
__device__ __forceinline__ int slot_w(int k) { return k < NS ? k : (k - NS < OX ? 12 + k : 14 + k); }
__device__ __forceinline__ int slot_x(int j, int e, int d) {
  return j == 0 ? NS + 12 + OX + e : NS + 2 * (j == 1 ? 5 : j - 2 - (j > d ? 1 : 0)) + e;
}
__device__ __forceinline__ int cont_w(int k) { return k < NS ? k : 14 + k; }
__device__ __forceinline__ int cont_x(int rb) { return rb < 2 ? 30 + rb : (rb < 4 ? NS + 10 + rb : NS - 4 + rb); }
__device__ __forceinline__ u32 fin_off(int sl, int q) { return (u32)sl * IN_SRC + (u32)(NS + 2 * sl) * 2u * PIECE + (u32)q * 64u; }
__device__ __forceinline__ u32 lds_rd(u32 a) { return __builtin_amdgcn_readfirstlane(*(volatile LDSP u32*)(uintptr_t)a); }
__device__ __forceinline__ void lds_wr(u32 a, u32 v) { *(volatile LDSP u32*)(uintptr_t)a = v; }

__device__ __forceinline__ void mover(const Args& P, int d, int wave, u32 lane, char* smem, u64 tstart) {
  const int s = P.rank;
  char* const myws = P.my;
  const int pair = wave >> 1;
  const int p = ((blockIdx.x + PROT) & 7) * 2 + pair;
  const u32 r = (u32)(s ^ d) & (u32)(NR - 1);
  const int nin = d == 1 ? 0 : NIN;
  const int np = d == 1 ? (DIR ? 16 : NOUT1) : NIN + NOUT;
  const int nid = d == 1 ? NOUT1 : NOUT;
  const u32 ring = (u32)(uintptr_t)smem + (u32)pair * 32768u;
  const u32 ctl = (u32)(uintptr_t)smem + 65536u + (u32)pair * 512u;
  const char* const xin = P.x;
  const char* const win = P.w;
  const int rg = r >> 2, rb_ = (r >> 1) & 1, ra = r & 1;
  u32 wo[4];
#pragma unroll
  for (int k = 0; k < 4; ++k) {
    const u32 q4 = lane >> 4;
    wo[k] = ((u32)p * 16u + (u32)k * 4u + q4) * 8192u + (((lane & 15u) ^ ((q4 << 2) | (u32)k)) * 16u) - (u32)k * 1024u;
  }
  constexpr u32 XC = 256u / XR;
  const u32 xrow = ((u32)p % (128u / XR)) * XR, xks = (u32)p / (128u / XR) * (XC / 4u);
#define W_BASE(IX) (win + (size_t)((u32)ra * 256u * 8192u + (u32)rg * 4096u + (u32)(ra ? 15 - (IX) : (IX)) * 256u))

  if ((wave & 1) == 0) {
    const u32 cnt = bump_issue(CNT_OFF + (u32)(blockIdx.x * 4 + wave) * 64u, myws);
    u32 xo[4], oo[4];
    {
#pragma unroll
      for (int k = 0; k < 4; ++k) {
        const u32 c = (u32)k * 64u + lane;
        xo[k] = (xrow + c / XC) * 1024u + xks * 64u + (c % XC) * 16u - (u32)k * 1024u;
        oo[k] = (u32)p * 4096u + lane * 16u;
      }
    }
    u32 poll_off;
    {
      const int i0 = (int)lane < nid ? (int)lane : 0;
      const bool sw = d == 1 && i0 < SWP;
      const int i = sw ? (i0 & ~1) | (p >> 3) : i0, ps = sw ? ((i0 & 1) << 3) | (p & 7) : p;
      const int jj = d != 1 ? d : (i < 6 * NS ? 2 + i / NS : 1);
      const int kk = d != 1 ? NS + i : (i < 6 * NS ? i % NS : i - 6 * NS);
      poll_off = FLOC_OFF + (u32)((jj - 1) * 8 + (ps >> 1)) * 128u + (u32)kk * 4u;
    }
#define ISSUE_LOADS(QQ, II) do { \
      const int q_ = (QQ); \
      const u32 dst_ = ring + (u32)(q_ & 7) * 4096u; \
      if (q_ < nin) { \
        bool isx_; int ix_; \
        piece_of(q_, d, isx_, ix_); \
        if (isx_) dma16x4(dst_, xo, xin + (size_t)(((r ^ (u32)(ix_ >> 1)) * 512u + (u32)rb_ * 256u + (u32)(ix_ & 1) * 128u) * 1024u + (u32)ra * 512u)); \
        else      dma16x4(dst_, wo, W_BASE(ix_)); \
      } else { \
        const bool sw_ = d == 1 && (II) < SWP; \
        const int i_ = sw_ ? ((II) & ~1) | (p >> 3) : (II); \
        const int jj_ = d != 1 ? d : (i_ < 6 * NS ? 2 + i_ / NS : 1); \
        const int kk_ = d != 1 ? NS + i_ : (i_ < 6 * NS ? i_ % NS : i_ - 6 * NS); \
        dma16x4(dst_, oo, myws + (size_t)(POUT_OFF + (u32)((jj_ - 1) * 16 + kk_) * PIECE + (u32)(sw_ ? (((II) & 1) - (p >> 3)) * 32768 : 0))); \
      } \
    } while (0)

    int q0 = 0;
    u32 pub = 0, call;
    u32 dbg1 = 0, dbg2 = 0;
    if (nin) {
      ISSUE_LOADS(1, 0); ISSUE_LOADS(2, 0); ISSUE_LOADS(3, 0);
    }
    if (nin) {
      call = bump_get<8>(cnt);
      lds_wr(ctl, 2u);
      q0 = 4; pub = 2;
    } else {
      call = bump_get<0>(cnt);
    }

    if (!nin) sleep_until(tstart, (u32)(TQ(NS + (DIR ? 12 : 2)) + L1T));
    u64 ready = 0;
    u64 todo = ((1ull << (np - nin)) - 1ull) << (nid - (np - nin));
    u32 cons = 0;
    bool dead = false;
    for (int q = q0; q < np; ++q) {
      bool rdy = q < nin || (ready & todo) != 0ull;
      if (cons + 8u <= (u32)q) cons = lds_rd(ctl + 4u);
      if (!rdy || cons + 8u <= (u32)q) {
        waitv<0>();
        pub = (u32)q;
        lds_wr(ctl, pub);
        const u64 t0 = now();
        u32 win = 64u;
        for (;;) {
          if (!rdy) {
            u32 pv = 0;
            const u32 below = __builtin_amdgcn_mbcnt_hi((u32)(todo >> 32), __builtin_amdgcn_mbcnt_lo((u32)todo, 0u));
            if (((todo >> lane) & 1ull) && below < win) pv = atom_rd(poll_off, myws);
            win = (u32)PWIN;
            ready |= __builtin_amdgcn_ballot_w64(pv >= call);
            rdy = (ready & todo) != 0ull;
            if (LSL && !rdy) __builtin_amdgcn_s_sleep(LSL);
          } else {
            __builtin_amdgcn_s_sleep(2);
          }
          cons = lds_rd(ctl + 4u);
          if (rdy && cons + 8u > (u32)q) break;
          if (give_up(t0, myws, call)) { dead = true; break; }
        }
        if (dead) break;
      }
      int id = 0;
      if (q >= nin) {
        id = __builtin_ctzll(ready & todo);
        todo &= ~(1ull << id);
        lds_wr(ctl + 32u + (u32)(q & 7) * 4u, (u32)id);
      }
      ISSUE_LOADS(q, id);
      waitv<8>();
      if (PWHO == 1 && nin && q == PQ1) probe0((u32)(d - 2) * IN_SRC + 2u * MIB + (u32)p * 4096u, P.peer);
      if (PWHO == 1 && nin && q == PQ2) probe0(PIN_OFF + (call % 3u) * PIN_PAR + (u32)(32 + (d - 2) * 16 + NS) * PIECE + (u32)p * 4096u, P.peer);
      if (q - 1 > (int)pub) { pub = (u32)(q - 1); lds_wr(ctl, pub); }
      if (q == nin - 1) STAMP(myws, 1);
    }
    waitv<0>();
    if (dead) st4_lane0(TMO_OFF, call, myws);
    lds_wr(ctl, dead ? DEAD : (u32)np);
    STAMP(myws, 3);
    STAMPV(myws, 0, tstart);
    return;
  }

  char* const rws = P.peer;
  const int slot = d - 2;
  const u32 dl = (u32)p * 4096u + lane * 16u;
  u32 xl[4], xd[4];
#pragma unroll
  for (int k = 0; k < 4; ++k) {
    const u32 c = (u32)k * 64u + lane, t = c / (4u * XR), row = (c >> 2) % XR;
    xl[k] = (row * XC + t * 4u + ((c ^ ((xrow + row) >> 2)) & 3u)) * 16u;
    xd[k] = XCT ? dl : (xks + t) * 8192u + (xrow + row) * 64u + (c & 3u) * 16u - (u32)k * 1024u;
  }
  u32 tick, ts0;
  int to;
  {
    u32 tg, tm;
    asm volatile("" : "+v"(xl[0]), "+v"(xl[1]), "+v"(xl[2]), "+v"(xl[3]), "+v"(xd[0]), "+v"(xd[1]), "+v"(xd[2]), "+v"(xd[3]));
    asm volatile("s_waitcnt vmcnt(1)\n\t"
                 "v_readfirstlane_b32 %[tick], v160\n\t"
                 "v_readfirstlane_b32 %[tg], v161\n\t"
                 "s_memrealtime s[58:59]\n\t"
                 "s_sub_i32 %[tm], %[tg], %[tis]\n\t"
                 "s_addk_i32 %[tm], 500\n\t"
                 "s_cmpk_gt_u32 %[tm], 1000\n\t"
                 "s_cselect_b32 %[tg], %[tis], %[tg]\n\t"
                 "s_addk_i32 %[tg], %[oc]\n\t"
                 "s_sub_i32 %[to], %[tg], %[tst]\n\t"
                 "s_and_b32 %[tm], %[tick], 15\n\t"
                 "s_mul_i32 %[tm], %[tm], %[osp]\n\t"
                 "s_add_i32 %[tg], %[tg], %[tm]\n\t"
                 "s_bitcmp0_b32 %[tick], 4\n\t"
                 "s_cselect_b32 %[tm], 0x10000, 0\n\t"
                 "v_add_u32 v160, %[tm], %[dl]\n\t"
                 "s_cmp_eq_u32 %[d], 1\n\t"
                 "s_cbranch_scc1 5f\n"
                 "4:\n\t"
                 "s_waitcnt lgkmcnt(0)\n\t"
                 "s_sub_i32 %[tm], s58, %[tg]\n\t"
                 "s_mov_b32 %[ts0], s58\n\t"
                 "s_memrealtime s[58:59]\n\t"
                 "s_cmp_lt_i32 %[tm], 0\n\t"
                 "s_cbranch_scc1 4b\n\t"
                 "global_store_dwordx4 v160, v[144:147], %[dst] sc0 sc1\n\t"
                 "global_store_dwordx4 v160, v[148:151], %[dst] offset:1024 sc0 sc1\n\t"
                 "global_store_dwordx4 v160, v[152:155], %[dst] offset:2048 sc0 sc1\n\t"
                 "global_store_dwordx4 v160, v[156:159], %[dst] offset:3072 sc0 sc1\n"
                 "5:\n\t"
                 "s_waitcnt lgkmcnt(0)"
                 : [tick] "=&s"(tick), [ts0] "=&s"(ts0), [to] "=&s"(to), [tg] "=&s"(tg), [tm] "=&s"(tm)
                 : "{v[144:159]}"(P.f16), "{v[160:161]}"(P.co), [tis] "s"(P.tis), [tst] "s"((u32)tstart), [d] "s"(d),
                   [dl] "v"(dl), [dst] "s"(rws + (size_t)((u32)max(slot, 0) * IN_SRC)), [oc] "i"(OC), [osp] "i"(OSP)
                 : "memory", "scc", "s58", "s59");
  }
  const u32 call = (tick >> 4) + 1u;
  u32 dbg1 = 0, dbg2 = 0;
  const u32 in0 = (u32)slot * IN_SRC + (call & 1u) * PIECE;
  const u32 pin0 = PIN_OFF + (call % 3u) * PIN_PAR + (u32)(d == 1 ? 0 : 32 + slot * 16 + NS) * PIECE;
  int nflag = 0;
  int pend = -1;
#define FLAG(QQ) do { \
    const int q_ = (QQ); \
    if (pend >= 0) { \
      u32 v_; \
      asm volatile("v_readfirstlane_b32 %0, v200" : "=s"(v_) :: "memory"); \
      if (v_ == 16u * call - 1u) st4_lane0(fin_off(slot, pend), 16u * call, rws); \
      pend = -1; \
    } \
    if (q_ == nflag && q_ < nin) { \
      if (q_ >= AG0 && q_ < AG1) { \
        asm volatile("s_mov_b64 exec, 1\n\ts_nop 3\n\tglobal_atomic_add v200, %0, %1, %2 sc0 sc1\n\ts_mov_b64 exec, -1" \
                     :: "v"(FLG_OFF + (u32)(d * 32 + q_) * 64u), "v"(1u), "s"(myws) : "memory", "v200"); \
        pend = q_; \
      } else { \
        add1_lane0(fin_off(slot, q_), rws); \
      } \
      ++nflag; \
    } \
  } while (0)

  u32 tgrid = 0, tord = 0;
  if (nin) {
    FLAG(-1);
    const u32 tb = (u32)max(to + TAU + OG, (int)(ts0 - (u32)tstart) + OD) + (u32)(TAU - 1);
    tgrid = tb - ((u32)tstart + tb - ((u32)(((p + STGD * d) & 15) * TAU) >> 4)) % (u32)TAU;
    tord = ORD > 1 ? (u32)max(to + TAU + OG + (int)(tick & 15u) * OSP, 0) - tgrid : 0u;
  } else {
    sleep_until(tstart, 600u);
    probe0(pin0 + (u32)p * 4096u, rws);
    probe0(pin0 + (u32)(NOUT1 - 1) * PIECE + (u32)p * 4096u, rws);
  }
  u32 full = 0, tnext = 0, tsh = 0;
  for (int q = nin ? 1 : 0; q < np; ++q) {
    bool isx; int ix;
    piece_of(q, d, isx, ix);
    isx = isx && q < nin;
    if (d != 1 || G1) tnext = max(tnext, tgrid + (q < ORD ? tord : 0u));
    if (!PRE) sleep_until(tstart, tnext);
    if (full <= (u32)q) {
      full = lds_rd(ctl);
      if (full <= (u32)q) {
        bool drained = !(q > 0 && q < nin + LAG);
        u64 el;
        for (;;) {
          __builtin_amdgcn_s_sleep(1);
          full = lds_rd(ctl);
          el = now() - tstart;
          if (full > (u32)q) break;
          if (!drained && __builtin_amdgcn_readfirstlane((int)(el > (u64)(tnext + 30u)))) {
            drained = true;
            waitv<0>();
#pragma unroll
            for (int z = LAG; z >= 1; --z) FLAG(q - z);
            if (AG0) { waitv<0>(); FLAG(-1); }
          }
          if (__builtin_amdgcn_readfirstlane((int)(el > 2ull * DEADLINE - 1000ull))) { full = DEAD; break; }
        }
        tnext = max(tnext, (u32)__builtin_amdgcn_readfirstlane((u32)el));
      }
      if (full == DEAD) break;
    }
    const u32 la = ring + (u32)(q & 7) * 4096u;
    const u32x4 d0 = *(volatile LDSP u32x4*)(uintptr_t)(la + (isx ? xl[0] : lane * 16u));
    const u32x4 d1 = *(volatile LDSP u32x4*)(uintptr_t)(la + (isx ? xl[1] : lane * 16u + 1024u));
    const u32x4 d2 = *(volatile LDSP u32x4*)(uintptr_t)(la + (isx ? xl[2] : lane * 16u + 2048u));
    const u32x4 d3 = *(volatile LDSP u32x4*)(uintptr_t)(la + (isx ? xl[3] : lane * 16u + 3072u));
    const u32 id = lds_rd(ctl + 32u + (u32)(q & 7) * 4u);
    lds_wr(ctl + 4u, (u32)q + 1u);
    u32 o;
    if (q < nin) {
      o = in0 + (u32)(isx ? cont_x(ix) : cont_w(ix)) * 2u * PIECE;
    } else {
      o = pin0 + min(id, (u32)(nid - 1)) * PIECE;
      if (d == 1 && id < (u32)SWP) o += (u32)(((p >> 3) - (int)(id & 1u)) * 32768);
    }
    char* const dst = rws + (size_t)o;
    if (SHD && !SHP && nin && q >= SHQ) {
      sleep_exact(tstart, tnext - (u32)SHT);
      u32 vm;
      asm volatile("s_getreg_b32 %0, hwreg(HW_REG_IB_STS, 0, 4)" : "=s"(vm) :: "memory");
      if (vm > 1u) { tgrid += (u32)SHD; tsh += (u32)SHD; tnext += SHN ? (u32)SHD : 0u; }
      else { const u32 b = min(tsh, (u32)SHR); tgrid -= b; tsh -= b; }
    }
    if (PRE) sleep_exact(tstart, tnext);
    const u32 tsend = tnext;
    tgrid += (u32)(TAU - (q < QA ? DA : 0) - (q >= QB ? DB : 0) - (q >= QC ? DC : 0) - (q >= QE ? ((q + 1) * DE >> 2) - (q * DE >> 2) : 0));
    tnext += (u32)(d == 1 && !G1 ? TAU1 : TAU2);
    st16<0>(isx ? xd[0] : dl, d0, dst);
    st16<1024>(isx ? xd[1] : dl, d1, dst);
    st16<2048>(isx ? xd[2] : dl, d2, dst);
    st16<3072>(isx ? xd[3] : dl, d3, dst);
    if (nin && q < nin + LAG) {
      waitv<AG0 ? 4 : 5 * LAG>();
      FLAG(q - LAG);
      if (PWHO == 0 && q == PQ1) probe0((u32)slot * IN_SRC + 2u * MIB + (u32)p * 4096u, rws);
      if (PWHO == 0 && q == PQ2) probe0(pin0 + (u32)p * 4096u, rws);
    } else {
      waitv<4 * WIN>();
    }
    if (SHD && SHP && nin && q >= SHQ) {
      bool cg;
      if (q < nin + LAG) {
        cg = (u32)__builtin_amdgcn_readfirstlane((u32)(now() - tstart)) - tsend > (u32)SHP;
      } else {
        sleep_exact(tstart, tsend + (u32)SHP);
        u32 vm;
        asm volatile("s_getreg_b32 %0, hwreg(HW_REG_IB_STS, 0, 4)" : "=s"(vm) :: "memory");
        cg = vm > 4u;
      }
      if (cg) { tgrid += (u32)SHD; tsh += (u32)SHD; }
      else { const u32 b = min(tsh, (u32)SHR); tgrid -= b; tsh -= b; }
    }
    if (q == NS + 11) STAMP(myws, 1);
    if (q == nin - 1) STAMP(myws, 2);
    if (q == np - 1) STAMP(myws, 4);
  }
  if ((tick & 15u) == 0u) st4_lane0(LNK_OFF + (u32)d * 128u + 4u, 0u, myws);
  waitv<0>();
  STAMP(myws, 3);
  STAMPV(myws, 0, tstart);
}

struct Frag {
  bf16x8 x[2][2];
  bf16x8 w[2][2];
};

struct Loader {
  const char* base;
  u32 stride[2];
  u32 v[4];
  u32 m0base;
};

template <int STAGE> __device__ __forceinline__ void glds_pair(const Loader& L, int half) {
  const u32 m0v = L.m0base + STAGE * STAGE_B;
  if (half == 0)
    asm volatile("s_mov_b32 m0, %0\n\ts_nop 0\n\t"
                 "global_load_lds_dwordx4 %1, %3 " GLP "\n\t"
                 "global_load_lds_dwordx4 %2, %3 offset:1024 " GLP
                 :: "s"(m0v), "v"(L.v[0]), "v"(L.v[1]), "s"(L.base) : "memory");
  else
    asm volatile("s_mov_b32 m0, %0\n\ts_nop 0\n\t"
                 "global_load_lds_dwordx4 %1, %3 offset:2048 " GLP "\n\t"
                 "global_load_lds_dwordx4 %2, %3 offset:3072 " GLP
                 :: "s"(m0v), "v"(L.v[2]), "v"(L.v[3]), "s"(L.base) : "memory");
}

struct RdAddr { u32 xa[2]; u32 wa[2][2]; };

template <int STAGE> __device__ __forceinline__ bf16x8 rd_x(const RdAddr& A, int mt, int ks) {
  return *(const LDSP bf16x8*)(uintptr_t)(A.xa[ks] + (u32)(STAGE * STAGE_B + mt * 2048));
}
template <int STAGE> __device__ __forceinline__ bf16x8 rd_w(const RdAddr& A, int nt, int ks) {
  s16x4 lo = __builtin_amdgcn_ds_read_tr16_b64_v4i16((LDSP s16x4*)(uintptr_t)(A.wa[nt][0] + (u32)(STAGE * STAGE_B + 8192 + ks * 4096)));
  s16x4 hi = __builtin_amdgcn_ds_read_tr16_b64_v4i16((LDSP s16x4*)(uintptr_t)(A.wa[nt][1] + (u32)(STAGE * STAGE_B + 8192 + ks * 4096)));
  s16x8 v = __builtin_shufflevector(lo, hi, 0, 1, 2, 3, 4, 5, 6, 7);
  return __builtin_bit_cast(bf16x8, v);
}
#define MFMA(nt, mt, ks) acc[nt][mt] = __builtin_amdgcn_mfma_f32_32x32x16_bf16(cur.w[nt][ks], cur.x[mt][ks], acc[nt][mt], 0, 0, 0)

template <int S, int VM, bool LOAD, bool READ>
__device__ __forceinline__ void kstep(f32x16 (&acc)[2][2], const Frag& cur, Frag& nxt, const RdAddr& A, Loader& L) {
  constexpr int SN = (S + 1) & 3;
  if (READ) {
    __builtin_amdgcn_s_waitcnt((VM & 15) | (7 << 4) | (0 << 8) | ((VM >> 4) << 14));
    asm volatile("s_barrier" ::: "memory");
  } else {
    __builtin_amdgcn_s_waitcnt(15 | (7 << 4) | (0 << 8) | (3 << 14));
  }
  SB();
  MFMA(0, 0, 0); SB();
  if (READ) { nxt.w[0][0] = rd_w<SN>(A, 0, 0); nxt.x[0][0] = rd_x<SN>(A, 0, 0); } SB();
  MFMA(0, 1, 0); SB();
  if (READ) { nxt.x[1][0] = rd_x<SN>(A, 1, 0); nxt.w[1][0] = rd_w<SN>(A, 1, 0); } SB();
  MFMA(1, 0, 0); SB();
  if (LOAD) glds_pair<S>(L, 0);
  SB();
  MFMA(1, 1, 0); SB();
  if (READ) { nxt.w[0][1] = rd_w<SN>(A, 0, 1); nxt.x[0][1] = rd_x<SN>(A, 0, 1); } SB();
  MFMA(0, 0, 1); SB();
  if (READ) { nxt.x[1][1] = rd_x<SN>(A, 1, 1); nxt.w[1][1] = rd_w<SN>(A, 1, 1); } SB();
  MFMA(0, 1, 1); SB();
  if (LOAD) { glds_pair<S>(L, 1); L.base += L.stride[S & 1]; }
  SB();
  MFMA(1, 0, 1); SB();
  MFMA(1, 1, 1); SB();
}

template <int S> __device__ __forceinline__ void stage_load(Loader& L) {
  glds_pair<S>(L, 0); glds_pair<S>(L, 1); L.base += L.stride[S & 1];
}

__device__ __forceinline__ void gemm_phase(f32x16 (&acc)[2][2], const RdAddr& A, Loader& L, int n, int sk, u32 jump, u32& dbgv, int dbg0) {
  Frag f0, f1;
  L.base = uni(L.base + (sk == 0 ? IN_SRC : 0u));
  L.stride[0] = __builtin_amdgcn_readfirstlane(L.stride[0]);
  L.stride[1] = __builtin_amdgcn_readfirstlane(L.stride[1]);
  asm volatile("s_nop 4" ::: "memory");
  stage_load<0>(L); stage_load<1>(L); stage_load<2>(L); stage_load<3>(L);
  waitv<12>();
  asm volatile("s_barrier" ::: "memory");
#pragma unroll
  for (int ks = 0; ks < 2; ++ks)
#pragma unroll
    for (int i = 0; i < 2; ++i) { f0.w[i][ks] = rd_w<0>(A, i, ks); f0.x[i][ks] = rd_x<0>(A, i, ks); }
  SB();
  for (int t = 0; t < n - 4; t += 4) {
    L.base += (t & 4 ? jump : 0u) + (t + 4 == 8 * sk ? IN_SRC : 0u);
    kstep<0, 8, true, true>(acc, f0, f1, A, L);
    kstep<1, 8, true, true>(acc, f1, f0, A, L);
    kstep<2, 8, true, true>(acc, f0, f1, A, L);
    kstep<3, 8, true, true>(acc, f1, f0, A, L);
  }
  kstep<0, 8, false, true>(acc, f0, f1, A, L);
  kstep<1, 4, false, true>(acc, f1, f0, A, L);
  kstep<2, 0, false, true>(acc, f0, f1, A, L);
  kstep<3, 0, false, false>(acc, f1, f0, A, L);
}

__device__ __forceinline__ u32 pack2(float a, float b) {
  f32x2 t = {a, b};
  return __builtin_bit_cast(u32, __builtin_convertvector(t, bf16x2));
}
__device__ __forceinline__ u32 relu2(u32 v) {
  typedef short s16x2 __attribute__((ext_vector_type(2)));
  return __builtin_bit_cast(u32, __builtin_elementwise_max(__builtin_bit_cast(s16x2, v), s16x2{0, 0}));
}
__device__ __forceinline__ float bf_lo(u32 v) { return __builtin_bit_cast(float, v << 16); }
__device__ __forceinline__ float bf_hi(u32 v) { return __builtin_bit_cast(float, v & 0xffff0000u); }

__device__ __forceinline__ void zero8k(u32 l16, u32 l16b, char* pb) {
  const u32x4 z4 = {0u, 0u, 0u, 0u};
  asm volatile("s_nop 4" ::: "memory");
  st16<0>(l16, z4, pb); st16<1024>(l16, z4, pb); st16<2048>(l16, z4, pb); st16<3072>(l16, z4, pb);
  st16<0>(l16b, z4, pb); st16<1024>(l16b, z4, pb); st16<2048>(l16b, z4, pb); st16<3072>(l16b, z4, pb);
}

__device__ __forceinline__ void tile(const Args& P, int tid, int wave, u32 lane, char* smem, u64 tstart) {
  const int r = P.rank;
  char* const myws = P.my;
  const int g = r >> 2, b = (r >> 1) & 1, a = r & 1;
  const int bid = blockIdx.x;

  const u32 cnt = bump_issue(CNT_OFF + (u32)(bid * 4 + wave) * 64u, myws);

  int row0, col0, nrem, tskip, j, e, k, rb;
  int pi0 = -1, pi1 = -1;
  bool fin;
  if (tid < N_MAIN_WG) {
    const int xcd = tid & 7, o = tid >> 3, u = xcd >> 1;
    int rbi, cbi;
    if (u < 2) { rbi = o < 24 ? 1 + o % 3 : 0; cbi = o < 24 ? o / 3 : o - 24; }
    else { rbi = o & 3; cbi = o >> 2; }
    rb = u + 4 * rbi; k = cbi * 2 + (xcd & 1);
    j = rb >> 1; e = rb & 1;
    row0 = (r ^ j) * 512 + b * 256 + e * 128; col0 = g * 2048 + (a ? 15 - k : k) * 128;
    nrem = j >= 2 ? 40 : 48; tskip = j >= 2 ? j - 2 : 99;
    fin = j == 0;
    if (j == 0) pi1 = 6 * NS + 15 - k;
    else if (j >= 2 && k >= 16 - NS) pi0 = ((j ^ 1) - 2) * NS + 15 - k;
  } else {
    const int f = tid - N_MAIN_WG;
    const int qd = 1 + f / 32, rem = f & 31;
    e = (rem >> 3) & 1; k = (rem & 7) * 2 + (rem >> 4);
    const int h = b ^ (qd & 1), g2 = g ^ (qd >> 1);
    row0 = r * 512 + h * 256 + e * 128; col0 = g2 * 2048 + (a ? 15 - k : k) * 128;
    nrem = 0; tskip = 99; j = 0; rb = 0;
    fin = true;
    const int iA = k >= NS ? 32 + (2 * qd - 2) * 16 + k : -1;
    const int iB = 15 - k >= NS ? 32 + (2 * qd - 1) * 16 + 15 - k : -1;
    pi0 = k < 8 ? iA : iB;
    pi1 = k < 8 ? iB : iA;
  }

  const u32 sbase = (u32)(uintptr_t)smem;
  const int wm = wave >> 1, wn = wave & 1;
  RdAddr A;
  {
    const u32 h = lane >> 5, m = lane & 31u, f = (lane >> 2) & 3u;
    A.xa[0] = sbase + ((u32)wm * 64u + m) * 64u + 16u * (h ^ f);
    A.xa[1] = sbase + ((u32)wm * 64u + m) * 64u + 16u * ((2u + h) ^ f);
    const u32 blk = (lane >> 4) & 1u, q = (lane & 15u) >> 2, pp = lane & 3u;
#pragma unroll
    for (int nt = 0; nt < 2; ++nt)
#pragma unroll
      for (int t = 0; t < 2; ++t) {
        const u32 row = 8u * h + 4u * t + q;
        const u32 ch = (u32)wn * 8u + (u32)nt * 4u + 2u * blk + (pp & 1u);
        A.wa[nt][t] = sbase + 256u * row + 16u * (ch ^ (((row & 3u) << 2) | ((row >> 2) & 3u))) + 8u * (pp >> 1);
      }
  }

  f32x16 acc[2][2];
#pragma unroll
  for (int i = 0; i < 2; ++i)
#pragma unroll
    for (int k = 0; k < 2; ++k)
#pragma unroll
      for (int z = 0; z < 16; ++z) acc[i][k][z] = 0.f;

  const u32 tile_off = (u32)e * 32768u + (u32)wave * 8192u;
  const u32 l16 = lane * 16u, l16b = lane * 16u + 4096u;
  const u32 tw = sbase + (u32)wave * 8192u + (lane & 31u) * 128u + (((lane >> 5) ^ (lane & 7u)) * 16u);
  const u32 tr = sbase + (u32)wave * 8192u + (lane >> 3) * 128u + (((lane & 7u) ^ (lane >> 3)) * 16u);
  const u32 oo = (lane >> 3) * 8192u + (lane & 7u) * 16u;
  char* const ob = uni(P.out + (size_t)((u32)((row0 & 511) + wm * 64) * 8192u + (u32)(col0 + wn * 64) * 2u));
  const int nph = nrem ? 2 : 1;
  const int qin = FLQ(max(slot_w(k), slot_x(j, e, j == 7 ? 2 : 7)));
  const u32 tin = (u32)(EARLY + TQ(qin));
  const int addph = fin ? nph - 1 : 0;

  if (LPT && (bid >> 3) == 25 && wave == 0) {
    sleep_until(tstart, (u32)LPT);
    if (lane < 28u) probe(lane * 2u * MIB + MIB + (u32)(bid & 7) * 128u, myws);
  }
  if (RPT && (bid >> 3) == RPB && wave == 1) {
    const u32 dd = 1u + lane % 7u;
    const u64 pp = ((const GLBP u64*)(uintptr_t)__builtin_amdgcn_kernarg_segment_ptr())[6u + 2u * dd];
    const u32 cl = bump_get<0>(cnt);
    const u32 o = (lane < 7u && dd > 1u ? (dd - 2u) * IN_SRC + 2u * MIB : PIN_OFF + (cl % 3u) * PIN_PAR + (dd > 1u ? dd * 16u : 30u) * PIECE) + (u32)(bid & 7) * 128u;
    sleep_until(tstart, (u32)RPT);
    if (lane < 14u) asm volatile("global_atomic_add %0, %1, off sc1" :: "v"(pp + o), "v"(0u) : "memory");
  }
  if (PD0 && (fin || k >= NS)) sleep_until(tstart, (u32)(PD0 + k * PD1 + (fin ? PDF : 0)));
  Loader L;
  L.m0base = sbase + (u32)wave * 4096u;
  bool dead = false;
  u32 dbgv = 0;
  u32 call = 0;
  for (int ph = 0; ph < nph; ++ph) {
    int n, ts;
    bool skip = false;
    if (ph == 0) {
      n = 16; ts = 99;
      if (wave < 2) {
        L.base = P.x + (size_t)((u32)(row0 + wave * 64) * 1024u);
        L.stride[0] = L.stride[1] = 64u;
        const u32 c = (lane & 3u) ^ ((lane >> 4) & 3u);
#pragma unroll
        for (int k = 0; k < 4; ++k) L.v[k] = (lane >> 2) * 1024u + c * 16u + (u32)k * (16384u - 1024u);
      } else {
        L.base = P.w + (size_t)((u32)(wave - 2) * 16u * 8192u + (u32)col0 * 2u);
        L.stride[0] = L.stride[1] = 32u * 8192u;
        const u32 q = lane >> 4;
#pragma unroll
        for (int k = 0; k < 4; ++k) L.v[k] = q * 8192u + (((lane & 15u) ^ ((q << 2) | (u32)k)) * 16u) + (u32)k * (4u * 8192u - 1024u);
      }
    } else {
      n = nrem; ts = tskip;
      STAMP(myws, 1);
      if (wave == 0) {
        const int sl = (int)min(lane, 5u);
        const u32 fo = fin_off(sl, FLQ(max(slot_w(k), slot_x(j, e, sl + 2))));
        bool pend = lane < 6u && sl + 2 != j;
        sleep_until(tstart, tin);
        const u64 t0 = now();
        u32 st = 0;
        for (;;) {
          if (pend) pend = atom_rd(fo, myws) < 16u * call;
          if (__builtin_amdgcn_ballot_w64(pend) == 0ull) break;
          if (give_up(t0, myws, call)) {
            if (lane == 0) st_agt(myws + TMO_OFF, call);
            st = 1;
            break;
          }
          __builtin_amdgcn_s_sleep(PSL);
        }
        if (lane == 0) *(LDSP u32*)(uintptr_t)(sbase + 4u * STAGE_B) = st;
        __builtin_amdgcn_s_waitcnt(0);
      }
      asm volatile("s_barrier" ::: "memory");
      const bool wgdead = lds_rd(sbase + 4u * STAGE_B) != 0;
      STAMP(myws, 2);
      dead = dead || wgdead;
      skip = wgdead;
      const u32 par = call & 1u;
      static_assert(!XCT || XR == 32, "layout of the x pieces");
      const bool xc = XCT && wave < 2;
      L.base = myws + (size_t)(((u32)(wave < 2 ? cont_x(rb) : cont_w(k)) * 2u + par) * PIECE + (u32)(wave & 1) * (xc ? 8192u : 4096u));
      L.stride[0] = xc ? 2048u : 8192u;
      L.stride[1] = xc ? 14336u : 8192u;
#pragma unroll
      for (int k = 0; k < 4; ++k) L.v[k] = lane * 16u + (xc && k >= 2 ? 2048u : 0u);
    }
    if (!skip) gemm_phase(acc, A, L, n, ts, ph ? IN_SRC - PIECE : 0u, dbgv, ph * 8);
    if (ph == nph - 1) STAMP(myws, 3);
    if (ph == 0) {
      call = bump_get<0>(cnt);
      if (ZT0 && max(pi0, pi1) >= 0) {
        char* const pz = myws + (size_t)(PIN_OFF + ((call + 2u) % 3u) * PIN_PAR + tile_off);
        sleep_until(tstart, (u32)ZT0 + (u32)((tid >> 3) & 15) * (u32)ZDT);
        if (pi0 >= 0) zero8k(l16, l16b, uni(pz + (size_t)((u32)pi0 * PIECE)));
        if (pi1 >= 0) zero8k(l16, l16b, uni(pz + (size_t)((u32)pi1 * PIECE)));
      }
    }
    const u32 pin = PIN_OFF + (call % 3u) * PIN_PAR + tile_off;
#pragma unroll 1
    for (int s = 0; s < 2; ++s) {
      const int pi = ph != addph ? -1 : (s == 0 ? pi0 : pi1);
      if (pi < 0 && !(fin && ph == 0 && s == 0)) continue;
      u32x4 pv[8];
#pragma unroll
      for (int i = 0; i < 8; ++i) pv[i] = u32x4{0u, 0u, 0u, 0u};
      if (pi >= 0) {
        char* const pb = uni(myws + (size_t)(pin + (u32)pi * PIECE));
        const bool first = pi == 0 || (pi >= NOUT1 && (pi & 15) == NS);
        const u32 wo = pin + (u32)(pi - ((!WOWN && (lane & 2u) && !first) ? 1 : 0)) * PIECE + (lane & 1u) * 4096u + 4092u;
        sleep_until(tstart, pi >= NOUT1 ? (u32)(EARLY + TQ(NIN - 2 + (pi & 15) - NS)) : (fin ? 0u : tin - 300u));
        const u64 t0 = now();
        while (!dead) {
          u32 v = WOWN ? 1u : 0u;
          if (lane < (WOWN ? 2u : 4u)) v = atom_rd(wo, myws);
          if (WOWN ? __builtin_amdgcn_ballot_w64(v == 0u) == 0ull : __builtin_amdgcn_ballot_w64(v != 0u) != 0ull) break;
          if (give_up(t0, myws, call)) { dead = true; break; }
          __builtin_amdgcn_s_sleep(WSL);
        }
        if (s == 1) STAMP(myws, 4);
        for (;;) {
          if (__builtin_expect(dead, 0)) {
            if (lane == 0) st_agt(myws + TMO_OFF, call);
#pragma unroll
            for (int i = 0; i < 8; ++i) pv[i] = u32x4{0x7fc07fc0u, 0x7fc07fc0u, 0x7fc07fc0u, 0x7fc07fc0u};
            break;
          }
          asm volatile("s_nop 4\n\t"
                       "global_load_dwordx4 %0, %8, %10 sc0 sc1\n\t"
                       "global_load_dwordx4 %1, %8, %10 offset:1024 sc0 sc1\n\t"
                       "global_load_dwordx4 %2, %8, %10 offset:2048 sc0 sc1\n\t"
                       "global_load_dwordx4 %3, %8, %10 offset:3072 sc0 sc1\n\t"
                       "global_load_dwordx4 %4, %9, %10 sc0 sc1\n\t"
                       "global_load_dwordx4 %5, %9, %10 offset:1024 sc0 sc1\n\t"
                       "global_load_dwordx4 %6, %9, %10 offset:2048 sc0 sc1\n\t"
                       "global_load_dwordx4 %7, %9, %10 offset:3072 sc0 sc1"
                       : "=&v"(pv[0]), "=&v"(pv[1]), "=&v"(pv[2]), "=&v"(pv[3]), "=&v"(pv[4]), "=&v"(pv[5]), "=&v"(pv[6]), "=&v"(pv[7])
                       : "v"(l16), "v"(l16b), "s"(pb) : "memory");
          const bool late = __builtin_amdgcn_readfirstlane((int)(now() - t0 > 50000ull)) != 0;
          asm volatile("s_waitcnt vmcnt(0)"
                       : "+v"(pv[0]), "+v"(pv[1]), "+v"(pv[2]), "+v"(pv[3]), "+v"(pv[4]), "+v"(pv[5]), "+v"(pv[6]), "+v"(pv[7]) :: "memory");
          u32 mn = 0xffffffffu;
#pragma unroll
          for (int i = 0; i < 8; ++i)
#pragma unroll
            for (int z = 0; z < 4; ++z) mn = min(mn, pv[i][z]);
          if (__builtin_amdgcn_ballot_w64(mn == 0u) == 0ull) break;
          if (__builtin_expect(late, 0)) dead = give_up(t0, myws, call);
        }
      }
#pragma unroll
      for (int nt = 0; nt < 2; ++nt)
#pragma unroll
        for (int mt = 0; mt < 2; ++mt)
#pragma unroll
          for (int sx = 0; sx < 2; ++sx)
#pragma unroll
            for (int z = 0; z < 4; ++z) {
              const u32 u = pv[(nt * 2 + mt) * 2 + sx][z];
              acc[nt][mt][8 * sx + 2 * z] += bf_lo(u);
              acc[nt][mt][8 * sx + 2 * z + 1] += bf_hi(u);
            }
      if (!fin) continue;
      if (s == 1) STAMP(myws, 6);
#pragma unroll
      for (int nt = 0; nt < 2; ++nt)
#pragma unroll
        for (int mt = 0; mt < 2; ++mt)
#pragma unroll
          for (int sx = 0; sx < 2; ++sx) {
            u32x4 v;
#pragma unroll
            for (int z = 0; z < 4; ++z) v[z] = relu2(pack2(acc[nt][mt][8 * sx + 2 * z], acc[nt][mt][8 * sx + 2 * z + 1]));
            *(volatile LDSP u32x4*)(uintptr_t)((tw ^ (u32)((nt * 4 + sx * 2) * 16)) + (u32)mt * 4096u) = v;
          }
      u32 o1, o2, o3, o4, o5, o6, o7;
      asm volatile("ds_read_b128 %0, %15\n\t"
                   "ds_read_b128 %1, %15 offset:1024\n\t"
                   "ds_read_b128 %2, %15 offset:2048\n\t"
                   "ds_read_b128 %3, %15 offset:3072\n\t"
                   "ds_read_b128 %4, %15 offset:4096\n\t"
                   "ds_read_b128 %5, %15 offset:5120\n\t"
                   "ds_read_b128 %6, %15 offset:6144\n\t"
                   "ds_read_b128 %7, %15 offset:7168\n\t"
                   "v_add_u32 %8, 0x10000, %16\n\t"
                   "v_add_u32 %9, 0x20000, %16\n\t"
                   "v_add_u32 %10, 0x30000, %16\n\t"
                   "v_add_u32 %11, 0x40000, %16\n\t"
                   "v_add_u32 %12, 0x50000, %16\n\t"
                   "v_add_u32 %13, 0x60000, %16\n\t"
                   "v_add_u32 %14, 0x70000, %16\n\t"
                   "s_mov_b64 exec, %18\n\t"
                   "s_waitcnt lgkmcnt(7)\n\tglobal_store_dwordx4 %16, %0, %17 sc0 sc1\n\t"
                   "s_waitcnt lgkmcnt(6)\n\tglobal_store_dwordx4 %8, %1, %17 sc0 sc1\n\t"
                   "s_waitcnt lgkmcnt(5)\n\tglobal_store_dwordx4 %9, %2, %17 sc0 sc1\n\t"
                   "s_waitcnt lgkmcnt(4)\n\tglobal_store_dwordx4 %10, %3, %17 sc0 sc1\n\t"
                   "s_waitcnt lgkmcnt(3)\n\tglobal_store_dwordx4 %11, %4, %17 sc0 sc1\n\t"
                   "s_waitcnt lgkmcnt(2)\n\tglobal_store_dwordx4 %12, %5, %17 sc0 sc1\n\t"
                   "s_waitcnt lgkmcnt(1)\n\tglobal_store_dwordx4 %13, %6, %17 sc0 sc1\n\t"
                   "s_waitcnt lgkmcnt(0)\n\tglobal_store_dwordx4 %14, %7, %17 sc0 sc1\n\t"
                   "s_cmp_lg_u64 %18, -1\n\t"
                   "s_cbranch_scc1 1f\n\t"
                   "s_endpgm\n"
                   "1:\n\t"
                   "s_mov_b64 exec, -1\n\t"
                   "s_nop 1"
                   : "=&v"(pv[0]), "=&v"(pv[1]), "=&v"(pv[2]), "=&v"(pv[3]), "=&v"(pv[4]), "=&v"(pv[5]), "=&v"(pv[6]), "=&v"(pv[7]),
                     "=&v"(o1), "=&v"(o2), "=&v"(o3), "=&v"(o4), "=&v"(o5), "=&v"(o6), "=&v"(o7)
                   : "v"(tr), "v"(oo), "s"(ob), "s"(s == 1 ? ~0ull : (u64)WMK) : "memory", "scc");
      if (s == 0) waitv<0>();
    }
  }

  if (!fin) {
    STAMP(myws, 6);
    const bool dir = DIR && j >= 2 && k < NS;
    char* dst = uni(dir ? P.peer + (size_t)(PIN_OFF + (call % 3u) * PIN_PAR + (u32)((j - 2) * NS + k) * PIECE + tile_off)
                        : myws + (size_t)(POUT_OFF + (u32)((j - 1) * 16 + k) * PIECE + tile_off));
    char* fl = myws + (size_t)(FLOC_OFF + (u32)((j - 1) * 8 + e * 4 + wave) * 128u + (u32)k * 4u);
    u32 least = 1u;
    if (__builtin_expect(dead, 0)) {
      least = 0x7fc07fc0u;
#pragma unroll
      for (int i = 0; i < 64; ++i) acc[i >> 5][(i >> 4) & 1][i & 15] = 0.f;
    }
    asm volatile("s_nop 4" ::: "memory");
#pragma unroll
    for (int nt = 0; nt < 2; ++nt)
#pragma unroll
      for (int mt = 0; mt < 2; ++mt) {
        u32x4 v0, v1;
#pragma unroll
        for (int z = 0; z < 4; ++z) {
          v0[z] = max(pack2(acc[nt][mt][2 * z], acc[nt][mt][2 * z + 1]), least);
          v1[z] = max(pack2(acc[nt][mt][8 + 2 * z], acc[nt][mt][8 + 2 * z + 1]), least);
        }
        const u32 o = nt ? l16b : l16;
        if (mt == 0) st16<0>(o, v0, dst); else st16<2048>(o, v0, dst);
        if (dir) __builtin_amdgcn_s_sleep(DSL);
        if (mt == 0) st16<1024>(o, v1, dst); else st16<3072>(o, v1, dst);
        if (dir) __builtin_amdgcn_s_sleep(DSL);
      }
    waitv<0>();
    if (lane == 0) st_sys(fl, call);
  }
  STAMP(myws, 5);
  STAMPV(myws, 0, tstart);
}

__global__ __launch_bounds__(256, 2) void fused_gemm_rs(Params P) {
  extern __shared__ __attribute__((aligned(16))) char smem[];
  const int bid = blockIdx.x;
  typedef u64 u64x4 __attribute__((ext_vector_type(4)));
  u64x4 a4;
  u64 tstart, pe;
  int rk, d;
  u32 tis, t0;
  u32x16 f16;
  u32x2 co;
  static_assert(N_PUSH_WG == 56 && PROT == 0 && PWN == 1 && CPB >= N_PUSH_WG, "entry code");
  asm volatile("v_readfirstlane_b32 s46, %[tid]\n\t"
               "s_load_dwordx8 s[36:43], %[ka], 0x0\n\t"
               "s_lshr_b32 s47, %[bid], 3\n\t"
               "s_cmp_gt_u32 %[bid], 55\n\t"
               "s_cbranch_scc1 2f\n\t"
               "s_lshr_b32 s48, s46, 7\n\t"
               "s_mul_i32 s49, s48, 3\n\t"
               "s_add_i32 s47, s47, s49\n\t"
               "s_add_i32 s49, s47, -7\n\t"
               "s_min_u32 s47, s47, s49\n\t"
               "s_lshl_b32 s49, s47, 4\n\t"
               "s_load_dwordx4 s[52:55], %[ka], s49 offset:0x40\n\t"
               "s_memrealtime s[44:45]\n\t"
               "s_load_dword s62, %[ka], 0x20\n\t"
               "s_setprio 3\n\t"
               "s_add_i32 s63, s47, 1\n\t"
               "v_lshlrev_b32 v1, 2, %[tid]\n\t"
               "v_mov_b32 v27, 0\n\t"
               "v_add_u32 v1, %[ctl], v1\n\t"
               "s_bitcmp1_b32 s46, 6\n\t"
               "ds_write_b32 v1, v27\n\t"
               "s_cbranch_scc0 1f\n\t"
               "s_and_b32 s49, %[bid], 7\n\t"
               "s_lshl1_add_u32 s48, s49, s48\n\t"
               "v_bfe_u32 v1, %[tid], 4, 2\n\t"
               "v_and_b32 v2, 15, %[tid]\n\t"
               "v_lshlrev_b32 v3, 2, v1\n\t"
               "v_lshlrev_b32 v1, 13, v1\n\t"
               "v_xor_b32 v2, v3, v2\n\t"
               "v_lshl_or_b32 v1, s48, 17, v1\n\t"
               "v_lshl_or_b32 v20, v2, 4, v1\n\t"
               "v_xor_b32 v21, 16, v20\n\t"
               "v_xor_b32 v22, 32, v20\n\t"
               "v_xor_b32 v23, 48, v20\n\t"
               "v_add_u32 v21, 0x7c00, v21\n\t"
               "v_add_u32 v22, 0xf800, v22\n\t"
               "v_add_u32 v23, 0x17400, v23\n\t"
               "s_lshl_b32 s49, s63, 7\n\t"
               "s_add_i32 s49, s49, %[lnk]\n\t"
               "v_mov_b32 v28, s49\n\t"
               "v_mov_b32 v25, 1\n\t"
               "s_waitcnt lgkmcnt(0)\n\t"
               "s_memrealtime s[58:59]\n\t"
               "global_load_dwordx4 v[144:147], v20, s[54:55] " MLP "\n\t"
               "global_load_dwordx4 v[148:151], v21, s[54:55] offset:1024 " MLP "\n\t"
               "global_load_dwordx4 v[152:155], v22, s[54:55] offset:2048 " MLP "\n\t"
               "global_load_dwordx4 v[156:159], v23, s[54:55] offset:3072 " MLP "\n\t"
               "s_mov_b64 exec, 1\n\t"
               "global_atomic_add v160, v28, v25, s[40:41] sc0 sc1\n\t"
               "s_add_i32 s50, s47, -1\n\t"
               "s_max_i32 s50, s50, 0\n\t"
               "s_mul_i32 s51, s50, %[fsl]\n\t"
               "s_lshl_b32 s49, s48, 7\n\t"
               "s_add_i32 s51, s51, s49\n\t"
               "s_add_i32 s51, s51, %[f0]\n\t"
               "v_mov_b32 v29, s51\n\t"
               "s_waitcnt lgkmcnt(0)\n\t"
               "s_or_b32 s66, s58, 1\n\t"
               "v_mov_b32 v26, s66\n\t"
               "global_atomic_cmpswap v161, v28, v[26:27], s[40:41] offset:4 sc0 sc1\n\t"
               "global_atomic_add v29, v27, s[52:53] sc1\n\t"
               "s_mov_b64 exec, -1\n"
               "1:\n\t"
               "s_waitcnt lgkmcnt(0)\n\t"
               "s_barrier\n\t"
               "s_branch 5f\n"
               "2:\n\t"
               "s_memrealtime s[44:45]\n\t"
               "s_load_dword s62, %[ka], 0x20\n\t"
               "s_load_dwordx2 s[52:53], %[ka], 0x40\n\t"
               "s_mov_b32 s63, 1\n\t"
               "s_add_i32 s47, %[bid], %[cpb]\n\t"
               "s_cmp_lt_u32 s47, 16\n\t"
               "s_cbranch_scc0 5f\n\t"
               "s_cmpk_lt_u32 s46, 0xc0\n\t"
               "s_cbranch_scc0 5f\n\t"
               "s_getpc_b64 s[56:57]\n\t"
               "s_mov_b32 m0, %[cpl]\n\t"
               "v_lshlrev_b32 v1, 7, %[tid]\n\t"
               "v_min_u32 v1, %[cpf], v1\n\t"
               "global_load_lds_dword v1, s[56:57]\n"
               "5:\n\t"
               "s_waitcnt lgkmcnt(0)"
               : "={s[36:43]}"(a4), "={s[44:45]}"(tstart), "={s46}"(t0), "={s[52:53]}"(pe), "={s62}"(rk), "={s63}"(d),
                 "={s66}"(tis)
               : [ka] "s"(__builtin_amdgcn_kernarg_segment_ptr()), [bid] "s"(bid), [tid] "v"(threadIdx.x),
                 [ctl] "i"(65536), [lnk] "i"(LNK_OFF), [fsl] "i"(IN_SRC + 4u * PIECE), [f0] "i"(NS * 2u * PIECE + 4096u),
                 [cpb] "i"(-CPB), [cpl] "i"(66048), [cpf] "i"(CPF - 128)
               : "memory", "scc", "s47", "s48", "s49", "s50", "s51", "s54", "s55", "s56", "s57", "s58", "s59",
                 "v1", "v2", "v3", "v20", "v21", "v22", "v23", "v25", "v26", "v27", "v28", "v29");
  asm volatile("" : "={v[144:159]}"(f16), "={v[160:161]}"(co));
  const u32 lane = threadIdx.x & 63u;
  const int wave = (int)(t0 >> 6);
  const Args A = {(const char*)a4[0], (const char*)a4[1], (char*)a4[2], (char*)a4[3], (char*)pe, rk, tis, f16, co};
  if (bid < N_PUSH_WG) mover(A, d, wave, lane, smem, tstart);
  else if (bid >= 256 && bid < 256 + N_PUSH_WG) {
    for (int i = 0; i < 6; ++i) __builtin_amdgcn_s_sleep(16);
  } else tile(A, bid < 256 - ROT ? bid - N_PUSH_WG + ROT : (bid < 256 ? bid - (256 - ROT) : bid - 2 * N_PUSH_WG), wave, lane, smem, tstart);
}

}

extern "C" void kernel_launch(int rank, int nranks, ncclComm_t comm,
                              hipStream_t stream,
                              void* const* d_in, const int* in_sizes, int n_in,
                              void* d_out, int out_size,
                              const int* in_dtypes, int out_dtype,
                              void* const* d_out_all,
                              void* const* d_ws_all, size_t ws_bytes) {
  if (nranks != NR || n_in < 2 || ws_bytes < (size_t)WS_NEED || (unsigned)rank >= (unsigned)NR) {
    fprintf(stderr, "unsupported configuration\n");
    return;
  }
  Params P;
  P.x = (const char*)d_in[0];
  P.w = (const char*)d_in[1];
  P.out = (char*)d_out;
  P.my = (char*)d_ws_all[rank];
  for (int i = 0; i < NR; ++i) {
    const int r = rank ^ i;
    P.lk[i].peer = (char*)d_ws_all[r];
    P.lk[i].w0 = P.w + (size_t)((r & 1) * (256 * 8192 + 15 * 256) + (r >> 2) * 4096);
  }
  P.rank = rank;
  P.pad[0] = P.pad[1] = P.pad[2] = 0;
  HIP_CHECK(hipFuncSetAttribute((const void*)fused_gemm_rs, hipFuncAttributeMaxDynamicSharedMemorySize, (int)LDS_BYTES));
  hipLaunchKernelGGL(fused_gemm_rs, dim3(N_WG), dim3(256), LDS_BYTES, stream, P);
  HIP_CHECK(hipGetLastError());
}
```

```cpp
#include <hip/hip_runtime.h>
#include <rccl/rccl.h>
#include <cstdio>
#include <cstdint>

#define HIP_CHECK(expr) do { hipError_t _e = (expr); if (_e != hipSuccess) { \
    fprintf(stderr, "HIP error %s at %s:%d\n", hipGetErrorString(_e), __FILE__, __LINE__); \
    return; } } while (0)

namespace {

typedef unsigned u32;
typedef unsigned long long u64;
typedef u32 u32x4 __attribute__((ext_vector_type(4)));
typedef u32 u32x2 __attribute__((ext_vector_type(2)));
typedef u32 u32x16 __attribute__((ext_vector_type(16)));
typedef float f32x2 __attribute__((ext_vector_type(2)));
typedef float f32x16 __attribute__((ext_vector_type(16)));
typedef __bf16 bf16x2 __attribute__((ext_vector_type(2)));
typedef __bf16 bf16x8 __attribute__((ext_vector_type(8)));
typedef short s16x4 __attribute__((ext_vector_type(4)));
typedef short s16x8 __attribute__((ext_vector_type(8)));
#define LDSP __attribute__((address_space(3)))
#define GLBP __attribute__((address_space(1)))

constexpr int NR = 8;
constexpr u32 PIECE = 65536u;
constexpr u32 MIB = 1u << 20;
#define NS 2
#define OX 14
#define TAU 112
#define T0 20
#define QA 0
#define DA 0
#define QB 20
#define DB 1
#define QC 99
#define DC 0
#define TQ(q) (T0 + TAU * (q) - DA * ((q) < QA ? (q) : QA) - DB * ((q) > QB ? (q) - QB : 0))
#define EARLY 150
#define TAU1 110
#define TAU2 0
#define STGD 2
#define PSL 4
#define LSL 0
#define WSL 3
#define RSL 0
#define WOWN 0
#define ALN 1
#define ALO 99
#define ALT 25
#define PWN 1
#define DIR 0
#define DSL 4
#define SWP 14
#define CPF 13824
#define CPB 56
#define PD0 0
#define PD1 0
#define PDF 0
#define GSH -18
#define ORG 1
#define OC 40
#define OG 8
#define OSP 7
#define ORD 1
#define OD 20
#define LPT 400
#define ROT 48
#define PWIN 64
#define PWHO 1
#define PQ1 9
#define PQ2 14
#define PROT 0
#define XR 32
#define ZT0 900
#define ZDT 100
#define RPT 400
#define RPB 25
#define XCT 0
#define L1T 500
#define SXT 6
#define PRE 1
#define WMK 1ull
#define G1 0
#define LAG 1
#define WIN 3
#define GLP "sc0 sc1"
#define MLP "sc0 sc1"
constexpr int NIN = 30;
constexpr int NOUT = 16 - NS;
constexpr int NOUT1 = 16 + 6 * NS;
constexpr u32 IN_SRC = 4u * MIB;
constexpr u32 PIN_OFF = 24u * MIB;
constexpr u32 PIN_PAR = 8u * MIB;
constexpr u32 POUT_OFF = 48u * MIB;
constexpr u32 FLG_OFF = 55u * MIB;
constexpr u32 FLOC_OFF = FLG_OFF + 16384u;
constexpr u32 TMO_OFF = FLG_OFF + 49152u;
constexpr u32 LNK_OFF = FLG_OFF + MIB + 262144u + 131072u;
constexpr u32 DUMMY_OFF = FLG_OFF + 65536u;
constexpr u32 DBG_OFF = FLG_OFF + 131072u;
constexpr u32 CNT_OFF = FLG_OFF + MIB + 262144u;
constexpr u32 WS_NEED = FLG_OFF + 2u * MIB;

constexpr int N_PUSH_WG = 56;
constexpr int N_MAIN_WG = 256;
constexpr int N_FIN_WG = 96;
constexpr int N_WG = 2 * N_PUSH_WG + N_MAIN_WG + N_FIN_WG;
constexpr u32 STAGE_B = 16384u;
constexpr u32 LDS_BYTES = 4u * STAGE_B + 1024u;
constexpr u64 DEADLINE = 100000000ull;
constexpr u32 DEAD = 0x7fffffffu;

struct Params {
  const char* x;
  const char* w;
  char* my;
  char* out;
  int rank;
  int pad[3];
  struct { char* peer; const char* w0; } lk[NR];
};
struct Args { const char* x; const char* w; char* my; char* out; char* peer; int rank; u32 tis; u32x16 f16; u32x2 co; };

#define SB() __builtin_amdgcn_sched_barrier(0)
template <int N> __device__ __forceinline__ void waitv() { asm volatile("s_waitcnt vmcnt(%0)" :: "i"(N) : "memory"); }

__device__ __forceinline__ u64 now() { return __builtin_amdgcn_s_memrealtime(); }
template <typename T> __device__ __forceinline__ T* uni(T* p) {
  const u64 v = (u64)(uintptr_t)p;
  const u32 lo = __builtin_amdgcn_readfirstlane((u32)v), hi = __builtin_amdgcn_readfirstlane((u32)(v >> 32));
  return (T*)(uintptr_t)(((u64)hi << 32) | lo);
}
__device__ __forceinline__ u32 ld_sys(const void* p) {
  return __hip_atomic_load((const GLBP u32*)(uintptr_t)p, __ATOMIC_RELAXED, __HIP_MEMORY_SCOPE_SYSTEM);
}
__device__ __forceinline__ void st_sys(void* p, u32 v) {
  __hip_atomic_store((GLBP u32*)(uintptr_t)p, v, __ATOMIC_RELAXED, __HIP_MEMORY_SCOPE_SYSTEM);
}
__device__ __forceinline__ void st_agt(void* p, u32 v) {
  __hip_atomic_store((GLBP u32*)(uintptr_t)p, v, __ATOMIC_RELAXED, __HIP_MEMORY_SCOPE_AGENT);
}
__device__ __forceinline__ bool give_up(u64 t0, const char* ws, u32 call) {
  const u64 dt = now() - t0;
  if (!__builtin_amdgcn_readfirstlane((int)(dt > 50000ull))) return false;
  __builtin_amdgcn_s_sleep(64);
  if (__builtin_amdgcn_readfirstlane((int)(dt > DEADLINE))) return true;
  return __builtin_amdgcn_readfirstlane(ld_sys(ws + TMO_OFF)) == call;
}
__device__ __forceinline__ void sleep_until(u64 tstart, u32 t) {
  for (;;) {
    const u32 el = (u32)__builtin_amdgcn_readfirstlane((u32)(now() - tstart));
    if (el >= t) break;
    if (t - el > 40u) __builtin_amdgcn_s_sleep(12); else __builtin_amdgcn_s_sleep(1);
  }
}
__device__ __forceinline__ void sleep_exact(u64 tstart, u32 t) {
  for (;;) {
    const u32 el = (u32)__builtin_amdgcn_readfirstlane((u32)(now() - tstart));
    if (el >= t) break;
    if (t - el > 60u) __builtin_amdgcn_s_sleep(12);
    else if (t - el > 20u) __builtin_amdgcn_s_sleep(3);
    else if (t - el > (u32)SXT) __builtin_amdgcn_s_sleep(1);
  }
}
template <int IMM> __device__ __forceinline__ void st16(u32 off, u32x4 v, char* base) {
  asm volatile("global_store_dwordx4 %0, %1, %2 offset:%3 sc0 sc1\n\ts_nop 1" :: "v"(off), "v"(v), "s"(base), "i"(IMM) : "memory");
}
template <int IMM> __device__ __forceinline__ void st16p(u32 off, u32x4 v, char* base) {
  asm volatile("global_store_dwordx4 %0, %1, %2 offset:%3\n\ts_nop 1" :: "v"(off), "v"(v), "s"(base), "i"(IMM) : "memory");
}
#define OST st16
__device__ __forceinline__ void dma16x4(u32 lds, const u32 (&o)[4], const char* base) {
  asm volatile("s_mov_b32 m0, %0\n\ts_nop 3\n\t"
               "global_load_lds_dwordx4 %1, %5 " MLP "\n\t"
               "global_load_lds_dwordx4 %2, %5 offset:1024 " MLP "\n\t"
               "global_load_lds_dwordx4 %3, %5 offset:2048 " MLP "\n\t"
               "global_load_lds_dwordx4 %4, %5 offset:3072 " MLP
               :: "s"(lds), "v"(o[0]), "v"(o[1]), "v"(o[2]), "v"(o[3]), "s"(base) : "memory");
}
__device__ __forceinline__ void probe(u32 off, char* base) {
  asm volatile("s_nop 4\n\tglobal_atomic_add %0, %1, %2 sc1" :: "v"(off), "v"(0u), "s"(base) : "memory");
}
__device__ __forceinline__ void probe0(u32 off, char* base) {
  asm volatile("s_mov_b64 exec, 1\n\ts_nop 3\n\tglobal_atomic_add %0, %1, %2 sc1\n\ts_mov_b64 exec, -1"
               :: "v"(off), "v"(0u), "s"(base) : "memory");
}
__device__ __forceinline__ u32 atom_rd(u32 off, const char* base) {
  u32 r;
  asm volatile("global_atomic_add %0, %1, %2, %3 sc0 sc1\n\ts_waitcnt vmcnt(0)" : "=&v"(r) : "v"(off), "v"(0u), "s"(base) : "memory");
  return r;
}
__device__ __forceinline__ u32 bump_issue(u32 off, const char* base) {
  u32 r;
  asm volatile("s_mov_b64 exec, 1\n\ts_nop 3\n\tglobal_atomic_add %0, %1, %2, %3 sc0 sc1\n\ts_mov_b64 exec, -1"
               : "=&v"(r) : "v"(off), "v"(1u), "s"(base) : "memory");
  return r;
}
template <int N> __device__ __forceinline__ u32 bump_get(u32 r) {
  u32 v;
  asm volatile("s_waitcnt vmcnt(%2)\n\tv_readfirstlane_b32 %0, %1\n\ts_nop 4" : "=s"(v) : "v"(r), "i"(N) : "memory");
  return v + 1u;
}
__device__ __forceinline__ void add1_lane0(u32 off, char* base) {
  asm volatile("s_mov_b64 exec, 1\n\ts_nop 3\n\tglobal_atomic_add %0, %1, %2 sc1\n\ts_mov_b64 exec, -1"
               :: "v"(off), "v"(1u), "s"(base) : "memory");
}
__device__ __forceinline__ void st4_lane0(u32 off, u32 val, char* base) {
  asm volatile("s_mov_b64 exec, 1\n\ts_nop 3\n\tglobal_store_dword %0, %1, %2 sc0 sc1\n\ts_mov_b64 exec, -1"
               :: "v"(off), "v"(val), "s"(base) : "memory");
}

#define STAMP(ws, idx) do {} while (0)
#define STAMPV(ws, idx, v) do {} while (0)

__device__ __forceinline__ void piece_of(int q, int d, bool& isx, int& idx) {
  const int t = q - NS, u = q - NS - 12;
  int j = (t >> 1) + 2;
  j += j >= d ? 1 : 0;
  j = j == 8 ? 1 : j;
  isx = (q >= NS && u < 0) || (u >= OX && u < OX + 2);
  idx = q < NS ? q : (u < 0 ? 2 * j + (t & 1) : (u < OX ? NS + u : (u < OX + 2 ? u - OX : NS + u - 2)));
}
__device__ __forceinline__ int slot_w(int k) { return k < NS ? k : (k - NS < OX ? 12 + k : 14 + k); }
__device__ __forceinline__ int slot_x(int j, int e, int d) {
  return j == 0 ? NS + 12 + OX + e : NS + 2 * (j == 1 ? 5 : j - 2 - (j > d ? 1 : 0)) + e;
}
__device__ __forceinline__ int cont_w(int k) { return k < NS ? k : 14 + k; }
__device__ __forceinline__ int cont_x(int rb) { return rb < 2 ? 30 + rb : (rb < 4 ? NS + 10 + rb : NS - 4 + rb); }
__device__ __forceinline__ u32 fin_off(int sl, int q) { return (u32)sl * IN_SRC + (u32)(NS + 2 * sl) * 2u * PIECE + (u32)q * 64u; }
__device__ __forceinline__ u32 lds_rd(u32 a) { return __builtin_amdgcn_readfirstlane(*(volatile LDSP u32*)(uintptr_t)a); }
__device__ __forceinline__ void lds_wr(u32 a, u32 v) { *(volatile LDSP u32*)(uintptr_t)a = v; }

__device__ __forceinline__ void mover(const Args& P, int d, int wave, u32 lane, char* smem, u64 tstart) {
  const int s = P.rank;
  char* const myws = P.my;
  const int pair = wave >> 1;
  const int p = ((blockIdx.x + PROT) & 7) * 2 + pair;
  const u32 r = (u32)(s ^ d) & (u32)(NR - 1);
  const int nin = d == 1 ? 0 : NIN;
  const int np = d == 1 ? (DIR ? 16 : NOUT1) : NIN + NOUT;
  const int nid = d == 1 ? NOUT1 : NOUT;
  const u32 ring = (u32)(uintptr_t)smem + (u32)pair * 32768u;
  const u32 ctl = (u32)(uintptr_t)smem + 65536u + (u32)pair * 512u;
  const char* const xin = P.x;
  const char* const win = P.w;
  const int rg = r >> 2, rb_ = (r >> 1) & 1, ra = r & 1;
  u32 wo[4];
#pragma unroll
  for (int k = 0; k < 4; ++k) {
    const u32 q4 = lane >> 4;
    wo[k] = ((u32)p * 16u + (u32)k * 4u + q4) * 8192u + (((lane & 15u) ^ ((q4 << 2) | (u32)k)) * 16u) - (u32)k * 1024u;
  }
  constexpr u32 XC = 256u / XR;
  const u32 xrow = ((u32)p % (128u / XR)) * XR, xks = (u32)p / (128u / XR) * (XC / 4u);
#define W_BASE(IX) (win + (size_t)((u32)ra * 256u * 8192u + (u32)rg * 4096u + (u32)(ra ? 15 - (IX) : (IX)) * 256u))

  if ((wave & 1) == 0) {
    const u32 cnt = bump_issue(CNT_OFF + (u32)(blockIdx.x * 4 + wave) * 64u, myws);
    u32 xo[4], oo[4];
    {
#pragma unroll
      for (int k = 0; k < 4; ++k) {
        const u32 c = (u32)k * 64u + lane;
        xo[k] = (xrow + c / XC) * 1024u + xks * 64u + (c % XC) * 16u - (u32)k * 1024u;
        oo[k] = (u32)p * 4096u + lane * 16u;
      }
    }
    u32 poll_off;
    {
      const int i0 = (int)lane < nid ? (int)lane : 0;
      const bool sw = d == 1 && i0 < SWP;
      const int i = sw ? (i0 & ~1) | (p >> 3) : i0, ps = sw ? ((i0 & 1) << 3) | (p & 7) : p;
      const int jj = d != 1 ? d : (i < 6 * NS ? 2 + i / NS : 1);
      const int kk = d != 1 ? NS + i : (i < 6 * NS ? i % NS : i - 6 * NS);
      poll_off = FLOC_OFF + (u32)((jj - 1) * 8 + (ps >> 1)) * 128u + (u32)kk * 4u;
    }
#define ISSUE_LOADS(QQ, II) do { \
      const int q_ = (QQ); \
      const u32 dst_ = ring + (u32)(q_ & 7) * 4096u; \
      if (q_ < nin) { \
        bool isx_; int ix_; \
        piece_of(q_, d, isx_, ix_); \
        if (isx_) dma16x4(dst_, xo, xin + (size_t)(((r ^ (u32)(ix_ >> 1)) * 512u + (u32)rb_ * 256u + (u32)(ix_ & 1) * 128u) * 1024u + (u32)ra * 512u)); \
        else      dma16x4(dst_, wo, W_BASE(ix_)); \
      } else { \
        const bool sw_ = d == 1 && (II) < SWP; \
        const int i_ = sw_ ? ((II) & ~1) | (p >> 3) : (II); \
        const int jj_ = d != 1 ? d : (i_ < 6 * NS ? 2 + i_ / NS : 1); \
        const int kk_ = d != 1 ? NS + i_ : (i_ < 6 * NS ? i_ % NS : i_ - 6 * NS); \
        dma16x4(dst_, oo, myws + (size_t)(POUT_OFF + (u32)((jj_ - 1) * 16 + kk_) * PIECE + (u32)(sw_ ? (((II) & 1) - (p >> 3)) * 32768 : 0))); \
      } \
    } while (0)

    int q0 = 0;
    u32 pub = 0, call;
    u32 dbg1 = 0, dbg2 = 0;
    if (nin) {
      ISSUE_LOADS(1, 0); ISSUE_LOADS(2, 0); ISSUE_LOADS(3, 0);
    }
    if (nin) {
      call = bump_get<8>(cnt);
      lds_wr(ctl, 2u);
      q0 = 4; pub = 2;
    } else {
      call = bump_get<0>(cnt);
    }

    if (!nin) sleep_until(tstart, (u32)(TQ(NS + (DIR ? 12 : 2)) + L1T));
    u64 ready = 0;
    u64 todo = ((1ull << (np - nin)) - 1ull) << (nid - (np - nin));
    u32 cons = 0;
    bool dead = false;
    for (int q = q0; q < np; ++q) {
      bool rdy = q < nin || (ready & todo) != 0ull;
      if (cons + 8u <= (u32)q) cons = lds_rd(ctl + 4u);
      if (!rdy || cons + 8u <= (u32)q) {
        waitv<0>();
        pub = (u32)q;
        lds_wr(ctl, pub);
        const u64 t0 = now();
        u32 win = 64u;
        for (;;) {
          if (!rdy) {
            u32 pv = 0;
            const u32 below = __builtin_amdgcn_mbcnt_hi((u32)(todo >> 32), __builtin_amdgcn_mbcnt_lo((u32)todo, 0u));
            if (((todo >> lane) & 1ull) && below < win) pv = atom_rd(poll_off, myws);
            win = (u32)PWIN;
            ready |= __builtin_amdgcn_ballot_w64(pv >= call);
            rdy = (ready & todo) != 0ull;
            if (LSL && !rdy) __builtin_amdgcn_s_sleep(LSL);
          } else {
            __builtin_amdgcn_s_sleep(2);
          }
          cons = lds_rd(ctl + 4u);
          if (rdy && cons + 8u > (u32)q) break;
          if (give_up(t0, myws, call)) { dead = true; break; }
        }
        if (dead) break;
      }
      int id = 0;
      if (q >= nin) {
        id = __builtin_ctzll(ready & todo);
        todo &= ~(1ull << id);
        lds_wr(ctl + 32u + (u32)(q & 7) * 4u, (u32)id);
      }
      ISSUE_LOADS(q, id);
      waitv<8>();
      if (PWHO == 1 && nin && q == PQ1) probe0((u32)(d - 2) * IN_SRC + 2u * MIB + (u32)p * 4096u, P.peer);
      if (PWHO == 1 && nin && q == PQ2) probe0(PIN_OFF + (call % 3u) * PIN_PAR + (u32)(32 + (d - 2) * 16 + NS) * PIECE + (u32)p * 4096u, P.peer);
      if (q - 1 > (int)pub) { pub = (u32)(q - 1); lds_wr(ctl, pub); }
      if (q == nin - 1) STAMP(myws, 1);
    }
    waitv<0>();
    if (dead) st4_lane0(TMO_OFF, call, myws);
    lds_wr(ctl, dead ? DEAD : (u32)np);
    STAMP(myws, 3);
    STAMPV(myws, 0, tstart);
    return;
  }

  char* const rws = P.peer;
  const int slot = d - 2;
  const u32 dl = (u32)p * 4096u + lane * 16u;
  u32 xl[4], xd[4];
#pragma unroll
  for (int k = 0; k < 4; ++k) {
    const u32 c = (u32)k * 64u + lane, t = c / (4u * XR), row = (c >> 2) % XR;
    xl[k] = (row * XC + t * 4u + ((c ^ ((xrow + row) >> 2)) & 3u)) * 16u;
    xd[k] = XCT ? dl : (xks + t) * 8192u + (xrow + row) * 64u + (c & 3u) * 16u - (u32)k * 1024u;
  }
  u32 tick, ts0;
  int to;
  {
    u32 tg, tm;
    asm volatile("" : "+v"(xl[0]), "+v"(xl[1]), "+v"(xl[2]), "+v"(xl[3]), "+v"(xd[0]), "+v"(xd[1]), "+v"(xd[2]), "+v"(xd[3]));
    asm volatile("s_waitcnt vmcnt(1)\n\t"
                 "v_readfirstlane_b32 %[tick], v160\n\t"
                 "v_readfirstlane_b32 %[tg], v161\n\t"
                 "s_memrealtime s[58:59]\n\t"
                 "s_sub_i32 %[tm], %[tg], %[tis]\n\t"
                 "s_addk_i32 %[tm], 500\n\t"
                 "s_cmpk_gt_u32 %[tm], 1000\n\t"
                 "s_cselect_b32 %[tg], %[tis], %[tg]\n\t"
                 "s_addk_i32 %[tg], %[oc]\n\t"
                 "s_sub_i32 %[to], %[tg], %[tst]\n\t"
                 "s_and_b32 %[tm], %[tick], 15\n\t"
                 "s_mul_i32 %[tm], %[tm], %[osp]\n\t"
                 "s_add_i32 %[tg], %[tg], %[tm]\n\t"
                 "s_bitcmp0_b32 %[tick], 4\n\t"
                 "s_cselect_b32 %[tm], 0x10000, 0\n\t"
                 "v_add_u32 v160, %[tm], %[dl]\n\t"
                 "s_cmp_eq_u32 %[d], 1\n\t"
                 "s_cbranch_scc1 5f\n"
                 "4:\n\t"
                 "s_waitcnt lgkmcnt(0)\n\t"
                 "s_sub_i32 %[tm], s58, %[tg]\n\t"
                 "s_mov_b32 %[ts0], s58\n\t"
                 "s_memrealtime s[58:59]\n\t"
                 "s_cmp_lt_i32 %[tm], 0\n\t"
                 "s_cbranch_scc1 4b\n\t"
                 "global_store_dwordx4 v160, v[144:147], %[dst] sc0 sc1\n\t"
                 "global_store_dwordx4 v160, v[148:151], %[dst] offset:1024 sc0 sc1\n\t"
                 "global_store_dwordx4 v160, v[152:155], %[dst] offset:2048 sc0 sc1\n\t"
                 "global_store_dwordx4 v160, v[156:159], %[dst] offset:3072 sc0 sc1\n"
                 "5:\n\t"
                 "s_waitcnt lgkmcnt(0)"
                 : [tick] "=&s"(tick), [ts0] "=&s"(ts0), [to] "=&s"(to), [tg] "=&s"(tg), [tm] "=&s"(tm)
                 : "{v[144:159]}"(P.f16), "{v[160:161]}"(P.co), [tis] "s"(P.tis), [tst] "s"((u32)tstart), [d] "s"(d),
                   [dl] "v"(dl), [dst] "s"(rws + (size_t)((u32)max(slot, 0) * IN_SRC)), [oc] "i"(OC), [osp] "i"(OSP)
                 : "memory", "scc", "s58", "s59");
  }
  const u32 call = (tick >> 4) + 1u;
  u32 dbg1 = 0, dbg2 = 0;
  const u32 in0 = (u32)slot * IN_SRC + (call & 1u) * PIECE;
  const u32 pin0 = PIN_OFF + (call % 3u) * PIN_PAR + (u32)(d == 1 ? 0 : 32 + slot * 16 + NS) * PIECE;
  int nflag = 0;
#define FLAG(QQ) do { \
    const int q_ = (QQ); \
    const bool real_ = q_ == nflag && q_ < nin; \
    const u32 o_ = real_ ? fin_off(slot, q_) : DUMMY_OFF + (u32)(blockIdx.x * 4 + wave) * 128u; \
    add1_lane0(o_, uni(real_ ? rws : myws)); \
    nflag += real_ ? 1 : 0; \
  } while (0)

  u32 tgrid = 0, tord = 0;
  if (nin) {
    FLAG(-1);
    const u32 tb = (u32)max(to + TAU + OG, (int)(ts0 - (u32)tstart) + OD) + (u32)(TAU - 1);
    tgrid = tb - ((u32)tstart + tb - ((u32)(((p + STGD * d) & 15) * TAU) >> 4)) % (u32)TAU;
    tord = ORD > 1 ? (u32)max(to + TAU + OG + (int)(tick & 15u) * OSP, 0) - tgrid : 0u;
  } else {
    sleep_until(tstart, 600u);
    probe0(pin0 + (u32)p * 4096u, rws);
    probe0(pin0 + (u32)(NOUT1 - 1) * PIECE + (u32)p * 4096u, rws);
  }
  u32 full = 0, tnext = 0;
  for (int q = nin ? 1 : 0; q < np; ++q) {
    bool isx; int ix;
    piece_of(q, d, isx, ix);
    isx = isx && q < nin;
    if (d != 1 || G1) tnext = max(tnext, tgrid + (q < ORD ? tord : 0u));
    if (!PRE) sleep_until(tstart, tnext);
    if (full <= (u32)q) {
      full = lds_rd(ctl);
      if (full <= (u32)q) {
        bool drained = !(q > 0 && q < nin + LAG);
        u64 el;
        for (;;) {
          __builtin_amdgcn_s_sleep(1);
          full = lds_rd(ctl);
          el = now() - tstart;
          if (full > (u32)q) break;
          if (!drained && __builtin_amdgcn_readfirstlane((int)(el > (u64)(tnext + 30u)))) {
            drained = true;
            waitv<0>();
#pragma unroll
            for (int z = LAG; z >= 1; --z) FLAG(q - z);
          }
          if (__builtin_amdgcn_readfirstlane((int)(el > 2ull * DEADLINE - 1000ull))) { full = DEAD; break; }
        }
        tnext = max(tnext, (u32)__builtin_amdgcn_readfirstlane((u32)el));
      }
      if (full == DEAD) break;
    }
    const u32 la = ring + (u32)(q & 7) * 4096u;
    const u32x4 d0 = *(volatile LDSP u32x4*)(uintptr_t)(la + (isx ? xl[0] : lane * 16u));
    const u32x4 d1 = *(volatile LDSP u32x4*)(uintptr_t)(la + (isx ? xl[1] : lane * 16u + 1024u));
    const u32x4 d2 = *(volatile LDSP u32x4*)(uintptr_t)(la + (isx ? xl[2] : lane * 16u + 2048u));
    const u32x4 d3 = *(volatile LDSP u32x4*)(uintptr_t)(la + (isx ? xl[3] : lane * 16u + 3072u));
    const u32 id = lds_rd(ctl + 32u + (u32)(q & 7) * 4u);
    lds_wr(ctl + 4u, (u32)q + 1u);
    u32 o;
    if (q < nin) {
      o = in0 + (u32)(isx ? cont_x(ix) : cont_w(ix)) * 2u * PIECE;
    } else {
      o = pin0 + min(id, (u32)(nid - 1)) * PIECE;
      if (d == 1 && id < (u32)SWP) o += (u32)(((p >> 3) - (int)(id & 1u)) * 32768);
    }
    char* const dst = rws + (size_t)o;
    if (PRE) sleep_exact(tstart, tnext);
    tgrid += (u32)(TAU - (q < QA ? DA : 0) - (q >= QB ? DB : 0) - (q >= QC ? DC : 0));
    tnext += (u32)(d == 1 && !G1 ? TAU1 : TAU2);
    st16<0>(isx ? xd[0] : dl, d0, dst);
    st16<1024>(isx ? xd[1] : dl, d1, dst);
    st16<2048>(isx ? xd[2] : dl, d2, dst);
    st16<3072>(isx ? xd[3] : dl, d3, dst);
    if (nin && q < nin + LAG) {
      waitv<5 * LAG>();
      FLAG(q - LAG);
      if (PWHO == 0 && q == PQ1) probe0((u32)slot * IN_SRC + 2u * MIB + (u32)p * 4096u, rws);
      if (PWHO == 0 && q == PQ2) probe0(pin0 + (u32)p * 4096u, rws);
    } else {
      waitv<4 * WIN>();
    }
    if (q == NS + 11) STAMP(myws, 1);
    if (q == nin - 1) STAMP(myws, 2);
    if (q == np - 1) STAMP(myws, 4);
  }
  if ((tick & 15u) == 0u) st4_lane0(LNK_OFF + (u32)d * 128u + 4u, 0u, myws);
  waitv<0>();
  STAMP(myws, 3);
  STAMPV(myws, 0, tstart);
}

struct Frag {
  bf16x8 x[2][2];
  bf16x8 w[2][2];
};

struct Loader {
  const char* base;
  u32 stride[2];
  u32 v[4];
  u32 m0base;
};

template <int STAGE> __device__ __forceinline__ void glds_pair(const Loader& L, int half) {
  const u32 m0v = L.m0base + STAGE * STAGE_B;
  if (half == 0)
    asm volatile("s_mov_b32 m0, %0\n\ts_nop 0\n\t"
                 "global_load_lds_dwordx4 %1, %3 " GLP "\n\t"
                 "global_load_lds_dwordx4 %2, %3 offset:1024 " GLP
                 :: "s"(m0v), "v"(L.v[0]), "v"(L.v[1]), "s"(L.base) : "memory");
  else
    asm volatile("s_mov_b32 m0, %0\n\ts_nop 0\n\t"
                 "global_load_lds_dwordx4 %1, %3 offset:2048 " GLP "\n\t"
                 "global_load_lds_dwordx4 %2, %3 offset:3072 " GLP
                 :: "s"(m0v), "v"(L.v[2]), "v"(L.v[3]), "s"(L.base) : "memory");
}

struct RdAddr { u32 xa[2]; u32 wa[2][2]; };

template <int STAGE> __device__ __forceinline__ bf16x8 rd_x(const RdAddr& A, int mt, int ks) {
  return *(const LDSP bf16x8*)(uintptr_t)(A.xa[ks] + (u32)(STAGE * STAGE_B + mt * 2048));
}
template <int STAGE> __device__ __forceinline__ bf16x8 rd_w(const RdAddr& A, int nt, int ks) {
  s16x4 lo = __builtin_amdgcn_ds_read_tr16_b64_v4i16((LDSP s16x4*)(uintptr_t)(A.wa[nt][0] + (u32)(STAGE * STAGE_B + 8192 + ks * 4096)));
  s16x4 hi = __builtin_amdgcn_ds_read_tr16_b64_v4i16((LDSP s16x4*)(uintptr_t)(A.wa[nt][1] + (u32)(STAGE * STAGE_B + 8192 + ks * 4096)));
  s16x8 v = __builtin_shufflevector(lo, hi, 0, 1, 2, 3, 4, 5, 6, 7);
  return __builtin_bit_cast(bf16x8, v);
}
#define MFMA(nt, mt, ks) acc[nt][mt] = __builtin_amdgcn_mfma_f32_32x32x16_bf16(cur.w[nt][ks], cur.x[mt][ks], acc[nt][mt], 0, 0, 0)

template <int S, int VM, bool LOAD, bool READ>
__device__ __forceinline__ void kstep(f32x16 (&acc)[2][2], const Frag& cur, Frag& nxt, const RdAddr& A, Loader& L) {
  constexpr int SN = (S + 1) & 3;
  if (READ) {
    __builtin_amdgcn_s_waitcnt((VM & 15) | (7 << 4) | (0 << 8) | ((VM >> 4) << 14));
    asm volatile("s_barrier" ::: "memory");
  } else {
    __builtin_amdgcn_s_waitcnt(15 | (7 << 4) | (0 << 8) | (3 << 14));
  }
  SB();
  MFMA(0, 0, 0); SB();
  if (READ) { nxt.w[0][0] = rd_w<SN>(A, 0, 0); nxt.x[0][0] = rd_x<SN>(A, 0, 0); } SB();
  MFMA(0, 1, 0); SB();
  if (READ) { nxt.x[1][0] = rd_x<SN>(A, 1, 0); nxt.w[1][0] = rd_w<SN>(A, 1, 0); } SB();
  MFMA(1, 0, 0); SB();
  if (LOAD) glds_pair<S>(L, 0);
  SB();
  MFMA(1, 1, 0); SB();
  if (READ) { nxt.w[0][1] = rd_w<SN>(A, 0, 1); nxt.x[0][1] = rd_x<SN>(A, 0, 1); } SB();
  MFMA(0, 0, 1); SB();
  if (READ) { nxt.x[1][1] = rd_x<SN>(A, 1, 1); nxt.w[1][1] = rd_w<SN>(A, 1, 1); } SB();
  MFMA(0, 1, 1); SB();
  if (LOAD) { glds_pair<S>(L, 1); L.base += L.stride[S & 1]; }
  SB();
  MFMA(1, 0, 1); SB();
  MFMA(1, 1, 1); SB();
}

template <int S> __device__ __forceinline__ void stage_load(Loader& L) {
  glds_pair<S>(L, 0); glds_pair<S>(L, 1); L.base += L.stride[S & 1];
}

__device__ __forceinline__ void gemm_phase(f32x16 (&acc)[2][2], const RdAddr& A, Loader& L, int n, int sk, u32 jump, u32& dbgv, int dbg0) {
  Frag f0, f1;
  L.base = uni(L.base + (sk == 0 ? IN_SRC : 0u));
  L.stride[0] = __builtin_amdgcn_readfirstlane(L.stride[0]);
  L.stride[1] = __builtin_amdgcn_readfirstlane(L.stride[1]);
  asm volatile("s_nop 4" ::: "memory");
  stage_load<0>(L); stage_load<1>(L); stage_load<2>(L); stage_load<3>(L);
  waitv<12>();
  asm volatile("s_barrier" ::: "memory");
#pragma unroll
  for (int ks = 0; ks < 2; ++ks)
#pragma unroll
    for (int i = 0; i < 2; ++i) { f0.w[i][ks] = rd_w<0>(A, i, ks); f0.x[i][ks] = rd_x<0>(A, i, ks); }
  SB();
  for (int t = 0; t < n - 4; t += 4) {
    L.base += (t & 4 ? jump : 0u) + (t + 4 == 8 * sk ? IN_SRC : 0u);
    kstep<0, 8, true, true>(acc, f0, f1, A, L);
    kstep<1, 8, true, true>(acc, f1, f0, A, L);
    kstep<2, 8, true, true>(acc, f0, f1, A, L);
    kstep<3, 8, true, true>(acc, f1, f0, A, L);
  }
  kstep<0, 8, false, true>(acc, f0, f1, A, L);
  kstep<1, 4, false, true>(acc, f1, f0, A, L);
  kstep<2, 0, false, true>(acc, f0, f1, A, L);
  kstep<3, 0, false, false>(acc, f1, f0, A, L);
}

__device__ __forceinline__ u32 pack2(float a, float b) {
  f32x2 t = {a, b};
  return __builtin_bit_cast(u32, __builtin_convertvector(t, bf16x2));
}
__device__ __forceinline__ u32 relu2(u32 v) {
  typedef short s16x2 __attribute__((ext_vector_type(2)));
  return __builtin_bit_cast(u32, __builtin_elementwise_max(__builtin_bit_cast(s16x2, v), s16x2{0, 0}));
}
__device__ __forceinline__ float bf_lo(u32 v) { return __builtin_bit_cast(float, v << 16); }
__device__ __forceinline__ float bf_hi(u32 v) { return __builtin_bit_cast(float, v & 0xffff0000u); }

__device__ __forceinline__ void zero8k(u32 l16, u32 l16b, char* pb) {
  const u32x4 z4 = {0u, 0u, 0u, 0u};
  asm volatile("s_nop 4" ::: "memory");
  st16<0>(l16, z4, pb); st16<1024>(l16, z4, pb); st16<2048>(l16, z4, pb); st16<3072>(l16, z4, pb);
  st16<0>(l16b, z4, pb); st16<1024>(l16b, z4, pb); st16<2048>(l16b, z4, pb); st16<3072>(l16b, z4, pb);
}

__device__ __forceinline__ void tile(const Args& P, int tid, int wave, u32 lane, char* smem, u64 tstart) {
  const int r = P.rank;
  char* const myws = P.my;
  const int g = r >> 2, b = (r >> 1) & 1, a = r & 1;
  const int bid = blockIdx.x;

  const u32 cnt = bump_issue(CNT_OFF + (u32)(bid * 4 + wave) * 64u, myws);

  int row0, col0, nrem, tskip, j, e, k, rb;
  int pi0 = -1, pi1 = -1;
  bool fin;
  if (tid < N_MAIN_WG) {
    const int xcd = tid & 7, o = tid >> 3, u = xcd >> 1;
    int rbi, cbi;
    if (u < 2) { rbi = o < 24 ? 1 + o % 3 : 0; cbi = o < 24 ? o / 3 : o - 24; }
    else { rbi = o & 3; cbi = o >> 2; }
    rb = u + 4 * rbi; k = cbi * 2 + (xcd & 1);
    j = rb >> 1; e = rb & 1;
    row0 = (r ^ j) * 512 + b * 256 + e * 128; col0 = g * 2048 + (a ? 15 - k : k) * 128;
    nrem = j >= 2 ? 40 : 48; tskip = j >= 2 ? j - 2 : 99;
    fin = j == 0;
    if (j == 0) pi1 = 6 * NS + 15 - k;
    else if (j >= 2 && k >= 16 - NS) pi0 = ((j ^ 1) - 2) * NS + 15 - k;
  } else {
    const int f = tid - N_MAIN_WG;
    const int qd = 1 + f / 32, rem = f & 31;
    e = (rem >> 3) & 1; k = (rem & 7) * 2 + (rem >> 4);
    const int h = b ^ (qd & 1), g2 = g ^ (qd >> 1);
    row0 = r * 512 + h * 256 + e * 128; col0 = g2 * 2048 + (a ? 15 - k : k) * 128;
    nrem = 0; tskip = 99; j = 0; rb = 0;
    fin = true;
    const int iA = k >= NS ? 32 + (2 * qd - 2) * 16 + k : -1;
    const int iB = 15 - k >= NS ? 32 + (2 * qd - 1) * 16 + 15 - k : -1;
    pi0 = k < 8 ? iA : iB;
    pi1 = k < 8 ? iB : iA;
  }

  const u32 sbase = (u32)(uintptr_t)smem;
  const int wm = wave >> 1, wn = wave & 1;
  RdAddr A;
  {
    const u32 h = lane >> 5, m = lane & 31u, f = (lane >> 2) & 3u;
    A.xa[0] = sbase + ((u32)wm * 64u + m) * 64u + 16u * (h ^ f);
    A.xa[1] = sbase + ((u32)wm * 64u + m) * 64u + 16u * ((2u + h) ^ f);
    const u32 blk = (lane >> 4) & 1u, q = (lane & 15u) >> 2, pp = lane & 3u;
#pragma unroll
    for (int nt = 0; nt < 2; ++nt)
#pragma unroll
      for (int t = 0; t < 2; ++t) {
        const u32 row = 8u * h + 4u * t + q;
        const u32 ch = (u32)wn * 8u + (u32)nt * 4u + 2u * blk + (pp & 1u);
        A.wa[nt][t] = sbase + 256u * row + 16u * (ch ^ (((row & 3u) << 2) | ((row >> 2) & 3u))) + 8u * (pp >> 1);
      }
  }

  f32x16 acc[2][2];
#pragma unroll
  for (int i = 0; i < 2; ++i)
#pragma unroll
    for (int k = 0; k < 2; ++k)
#pragma unroll
      for (int z = 0; z < 16; ++z) acc[i][k][z] = 0.f;

  const u32 tile_off = (u32)e * 32768u + (u32)wave * 8192u;
  const u32 l16 = lane * 16u, l16b = lane * 16u + 4096u;
  const u32 tw = sbase + (u32)wave * 8192u + (lane & 31u) * 128u + (((lane >> 5) ^ (lane & 7u)) * 16u);
  const u32 tr = sbase + (u32)wave * 8192u + (lane >> 3) * 128u + (((lane & 7u) ^ (lane >> 3)) * 16u);
  const u32 oo = (lane >> 3) * 8192u + (lane & 7u) * 16u;
  char* const ob = uni(P.out + (size_t)((u32)((row0 & 511) + wm * 64) * 8192u + (u32)(col0 + wn * 64) * 2u));
  const int nph = nrem ? 2 : 1;
  const int qin = max(slot_w(k), slot_x(j, e, j == 7 ? 2 : 7));
  const u32 tin = (u32)(EARLY + TQ(qin));
  const int addph = fin ? nph - 1 : 0;

  if (LPT && (bid >> 3) == 25 && wave == 0) {
    sleep_until(tstart, (u32)LPT);
    if (lane < 28u) probe(lane * 2u * MIB + MIB + (u32)(bid & 7) * 128u, myws);
  }
  if (RPT && (bid >> 3) == RPB && wave == 1) {
    const u32 dd = 1u + lane % 7u;
    const u64 pp = ((const GLBP u64*)(uintptr_t)__builtin_amdgcn_kernarg_segment_ptr())[6u + 2u * dd];
    const u32 cl = bump_get<0>(cnt);
    const u32 o = (lane < 7u && dd > 1u ? (dd - 2u) * IN_SRC + 2u * MIB : PIN_OFF + (cl % 3u) * PIN_PAR + (dd > 1u ? dd * 16u : 30u) * PIECE) + (u32)(bid & 7) * 128u;
    sleep_until(tstart, (u32)RPT);
    if (lane < 14u) asm volatile("global_atomic_add %0, %1, off sc1" :: "v"(pp + o), "v"(0u) : "memory");
  }
  if (PD0 && (fin || k >= NS)) sleep_until(tstart, (u32)(PD0 + k * PD1 + (fin ? PDF : 0)));
  Loader L;
  L.m0base = sbase + (u32)wave * 4096u;
  bool dead = false;
  u32 dbgv = 0;
  u32 call = 0;
  for (int ph = 0; ph < nph; ++ph) {
    int n, ts;
    bool skip = false;
    if (ph == 0) {
      n = 16; ts = 99;
      if (wave < 2) {
        L.base = P.x + (size_t)((u32)(row0 + wave * 64) * 1024u);
        L.stride[0] = L.stride[1] = 64u;
        const u32 c = (lane & 3u) ^ ((lane >> 4) & 3u);
#pragma unroll
        for (int k = 0; k < 4; ++k) L.v[k] = (lane >> 2) * 1024u + c * 16u + (u32)k * (16384u - 1024u);
      } else {
        L.base = P.w + (size_t)((u32)(wave - 2) * 16u * 8192u + (u32)col0 * 2u);
        L.stride[0] = L.stride[1] = 32u * 8192u;
        const u32 q = lane >> 4;
#pragma unroll
        for (int k = 0; k < 4; ++k) L.v[k] = q * 8192u + (((lane & 15u) ^ ((q << 2) | (u32)k)) * 16u) + (u32)k * (4u * 8192u - 1024u);
      }
    } else {
      n = nrem; ts = tskip;
      STAMP(myws, 1);
      if (wave == 0) {
        const int sl = (int)min(lane, 5u);
        const u32 fo = fin_off(sl, max(slot_w(k), slot_x(j, e, sl + 2)));
        bool pend = lane < 6u && sl + 2 != j;
        sleep_until(tstart, tin);
        const u64 t0 = now();
        u32 st = 0;
        for (;;) {
          if (pend) pend = atom_rd(fo, myws) < 16u * call;
          if (__builtin_amdgcn_ballot_w64(pend) == 0ull) break;
          if (give_up(t0, myws, call)) {
            if (lane == 0) st_agt(myws + TMO_OFF, call);
            st = 1;
            break;
          }
          __builtin_amdgcn_s_sleep(PSL);
        }
        if (lane == 0) *(LDSP u32*)(uintptr_t)(sbase + 4u * STAGE_B) = st;
        __builtin_amdgcn_s_waitcnt(0);
      }
      asm volatile("s_barrier" ::: "memory");
      const bool wgdead = lds_rd(sbase + 4u * STAGE_B) != 0;
      STAMP(myws, 2);
      dead = dead || wgdead;
      skip = wgdead;
      const u32 par = call & 1u;
      static_assert(!XCT || XR == 32, "layout of the x pieces");
      const bool xc = XCT && wave < 2;
      L.base = myws + (size_t)(((u32)(wave < 2 ? cont_x(rb) : cont_w(k)) * 2u + par) * PIECE + (u32)(wave & 1) * (xc ? 8192u : 4096u));
      L.stride[0] = xc ? 2048u : 8192u;
      L.stride[1] = xc ? 14336u : 8192u;
#pragma unroll
      for (int k = 0; k < 4; ++k) L.v[k] = lane * 16u + (xc && k >= 2 ? 2048u : 0u);
    }
    if (!skip) gemm_phase(acc, A, L, n, ts, ph ? IN_SRC - PIECE : 0u, dbgv, ph * 8);
    if (ph == nph - 1) STAMP(myws, 3);
    if (ph == 0) {
      call = bump_get<0>(cnt);
      if (ZT0 && max(pi0, pi1) >= 0) {
        char* const pz = myws + (size_t)(PIN_OFF + ((call + 2u) % 3u) * PIN_PAR + tile_off);
        sleep_until(tstart, (u32)ZT0 + (u32)((tid >> 3) & 15) * (u32)ZDT);
        if (pi0 >= 0) zero8k(l16, l16b, uni(pz + (size_t)((u32)pi0 * PIECE)));
        if (pi1 >= 0) zero8k(l16, l16b, uni(pz + (size_t)((u32)pi1 * PIECE)));
      }
    }
    const u32 pin = PIN_OFF + (call % 3u) * PIN_PAR + tile_off;
#pragma unroll 1
    for (int s = 0; s < 2; ++s) {
      const int pi = ph != addph ? -1 : (s == 0 ? pi0 : pi1);
      if (pi < 0 && !(fin && ph == 0 && s == 0)) continue;
      u32x4 pv[8];
#pragma unroll
      for (int i = 0; i < 8; ++i) pv[i] = u32x4{0u, 0u, 0u, 0u};
      if (pi >= 0) {
        char* const pb = uni(myws + (size_t)(pin + (u32)pi * PIECE));
        const bool first = pi == 0 || (pi >= NOUT1 && (pi & 15) == NS);
        const u32 wo = pin + (u32)(pi - ((!WOWN && (lane & 2u) && !first) ? 1 : 0)) * PIECE + (lane & 1u) * 4096u + 4092u;
        sleep_until(tstart, pi >= NOUT1 ? (u32)(EARLY + TQ(NIN - 2 + (pi & 15) - NS)) : (fin ? 0u : tin - 300u));
        const u64 t0 = now();
        while (!dead) {
          u32 v = WOWN ? 1u : 0u;
          if (lane < (WOWN ? 2u : 4u)) v = atom_rd(wo, myws);
          if (WOWN ? __builtin_amdgcn_ballot_w64(v == 0u) == 0ull : __builtin_amdgcn_ballot_w64(v != 0u) != 0ull) break;
          if (give_up(t0, myws, call)) { dead = true; break; }
          __builtin_amdgcn_s_sleep(WSL);
        }
        if (s == 1) STAMP(myws, 4);
        for (;;) {
          if (__builtin_expect(dead, 0)) {
            if (lane == 0) st_agt(myws + TMO_OFF, call);
#pragma unroll
            for (int i = 0; i < 8; ++i) pv[i] = u32x4{0x7fc07fc0u, 0x7fc07fc0u, 0x7fc07fc0u, 0x7fc07fc0u};
            break;
          }
          asm volatile("s_nop 4\n\t"
                       "global_load_dwordx4 %0, %8, %10 sc0 sc1\n\t"
                       "global_load_dwordx4 %1, %8, %10 offset:1024 sc0 sc1\n\t"
                       "global_load_dwordx4 %2, %8, %10 offset:2048 sc0 sc1\n\t"
                       "global_load_dwordx4 %3, %8, %10 offset:3072 sc0 sc1\n\t"
                       "global_load_dwordx4 %4, %9, %10 sc0 sc1\n\t"
                       "global_load_dwordx4 %5, %9, %10 offset:1024 sc0 sc1\n\t"
                       "global_load_dwordx4 %6, %9, %10 offset:2048 sc0 sc1\n\t"
                       "global_load_dwordx4 %7, %9, %10 offset:3072 sc0 sc1"
                       : "=&v"(pv[0]), "=&v"(pv[1]), "=&v"(pv[2]), "=&v"(pv[3]), "=&v"(pv[4]), "=&v"(pv[5]), "=&v"(pv[6]), "=&v"(pv[7])
                       : "v"(l16), "v"(l16b), "s"(pb) : "memory");
          const bool late = __builtin_amdgcn_readfirstlane((int)(now() - t0 > 50000ull)) != 0;
          asm volatile("s_waitcnt vmcnt(0)"
                       : "+v"(pv[0]), "+v"(pv[1]), "+v"(pv[2]), "+v"(pv[3]), "+v"(pv[4]), "+v"(pv[5]), "+v"(pv[6]), "+v"(pv[7]) :: "memory");
          u32 mn = 0xffffffffu;
#pragma unroll
          for (int i = 0; i < 8; ++i)
#pragma unroll
            for (int z = 0; z < 4; ++z) mn = min(mn, pv[i][z]);
          if (__builtin_amdgcn_ballot_w64(mn == 0u) == 0ull) break;
          if (__builtin_expect(late, 0)) dead = give_up(t0, myws, call);
        }
      }
#pragma unroll
      for (int nt = 0; nt < 2; ++nt)
#pragma unroll
        for (int mt = 0; mt < 2; ++mt)
#pragma unroll
          for (int sx = 0; sx < 2; ++sx)
#pragma unroll
            for (int z = 0; z < 4; ++z) {
              const u32 u = pv[(nt * 2 + mt) * 2 + sx][z];
              acc[nt][mt][8 * sx + 2 * z] += bf_lo(u);
              acc[nt][mt][8 * sx + 2 * z + 1] += bf_hi(u);
            }
      if (!fin) continue;
      if (s == 1) STAMP(myws, 6);
#pragma unroll
      for (int nt = 0; nt < 2; ++nt)
#pragma unroll
        for (int mt = 0; mt < 2; ++mt)
#pragma unroll
          for (int sx = 0; sx < 2; ++sx) {
            u32x4 v;
#pragma unroll
            for (int z = 0; z < 4; ++z) v[z] = relu2(pack2(acc[nt][mt][8 * sx + 2 * z], acc[nt][mt][8 * sx + 2 * z + 1]));
            *(volatile LDSP u32x4*)(uintptr_t)((tw ^ (u32)((nt * 4 + sx * 2) * 16)) + (u32)mt * 4096u) = v;
          }
      u32 o1, o2, o3, o4, o5, o6, o7;
      asm volatile("ds_read_b128 %0, %15\n\t"
                   "ds_read_b128 %1, %15 offset:1024\n\t"
                   "ds_read_b128 %2, %15 offset:2048\n\t"
                   "ds_read_b128 %3, %15 offset:3072\n\t"
                   "ds_read_b128 %4, %15 offset:4096\n\t"
                   "ds_read_b128 %5, %15 offset:5120\n\t"
                   "ds_read_b128 %6, %15 offset:6144\n\t"
                   "ds_read_b128 %7, %15 offset:7168\n\t"
                   "v_add_u32 %8, 0x10000, %16\n\t"
                   "v_add_u32 %9, 0x20000, %16\n\t"
                   "v_add_u32 %10, 0x30000, %16\n\t"
                   "v_add_u32 %11, 0x40000, %16\n\t"
                   "v_add_u32 %12, 0x50000, %16\n\t"
                   "v_add_u32 %13, 0x60000, %16\n\t"
                   "v_add_u32 %14, 0x70000, %16\n\t"
                   "s_mov_b64 exec, %18\n\t"
                   "s_waitcnt lgkmcnt(7)\n\tglobal_store_dwordx4 %16, %0, %17 sc0 sc1\n\t"
                   "s_waitcnt lgkmcnt(6)\n\tglobal_store_dwordx4 %8, %1, %17 sc0 sc1\n\t"
                   "s_waitcnt lgkmcnt(5)\n\tglobal_store_dwordx4 %9, %2, %17 sc0 sc1\n\t"
                   "s_waitcnt lgkmcnt(4)\n\tglobal_store_dwordx4 %10, %3, %17 sc0 sc1\n\t"
                   "s_waitcnt lgkmcnt(3)\n\tglobal_store_dwordx4 %11, %4, %17 sc0 sc1\n\t"
                   "s_waitcnt lgkmcnt(2)\n\tglobal_store_dwordx4 %12, %5, %17 sc0 sc1\n\t"
                   "s_waitcnt lgkmcnt(1)\n\tglobal_store_dwordx4 %13, %6, %17 sc0 sc1\n\t"
                   "s_waitcnt lgkmcnt(0)\n\tglobal_store_dwordx4 %14, %7, %17 sc0 sc1\n\t"
                   "s_cmp_lg_u64 %18, -1\n\t"
                   "s_cbranch_scc1 1f\n\t"
                   "s_endpgm\n"
                   "1:\n\t"
                   "s_mov_b64 exec, -1\n\t"
                   "s_nop 1"
                   : "=&v"(pv[0]), "=&v"(pv[1]), "=&v"(pv[2]), "=&v"(pv[3]), "=&v"(pv[4]), "=&v"(pv[5]), "=&v"(pv[6]), "=&v"(pv[7]),
                     "=&v"(o1), "=&v"(o2), "=&v"(o3), "=&v"(o4), "=&v"(o5), "=&v"(o6), "=&v"(o7)
                   : "v"(tr), "v"(oo), "s"(ob), "s"(s == 1 ? ~0ull : (u64)WMK) : "memory", "scc");
      if (s == 0) waitv<0>();
    }
  }

  if (!fin) {
    STAMP(myws, 6);
    const bool dir = DIR && j >= 2 && k < NS;
    char* dst = uni(dir ? P.peer + (size_t)(PIN_OFF + (call % 3u) * PIN_PAR + (u32)((j - 2) * NS + k) * PIECE + tile_off)
                        : myws + (size_t)(POUT_OFF + (u32)((j - 1) * 16 + k) * PIECE + tile_off));
    char* fl = myws + (size_t)(FLOC_OFF + (u32)((j - 1) * 8 + e * 4 + wave) * 128u + (u32)k * 4u);
    u32 least = 1u;
    if (__builtin_expect(dead, 0)) {
      least = 0x7fc07fc0u;
#pragma unroll
      for (int i = 0; i < 64; ++i) acc[i >> 5][(i >> 4) & 1][i & 15] = 0.f;
    }
    asm volatile("s_nop 4" ::: "memory");
#pragma unroll
    for (int nt = 0; nt < 2; ++nt)
#pragma unroll
      for (int mt = 0; mt < 2; ++mt) {
        u32x4 v0, v1;
#pragma unroll
        for (int z = 0; z < 4; ++z) {
          v0[z] = max(pack2(acc[nt][mt][2 * z], acc[nt][mt][2 * z + 1]), least);
          v1[z] = max(pack2(acc[nt][mt][8 + 2 * z], acc[nt][mt][8 + 2 * z + 1]), least);
        }
        const u32 o = nt ? l16b : l16;
        if (mt == 0) st16<0>(o, v0, dst); else st16<2048>(o, v0, dst);
        if (dir) __builtin_amdgcn_s_sleep(DSL);
        if (mt == 0) st16<1024>(o, v1, dst); else st16<3072>(o, v1, dst);
        if (dir) __builtin_amdgcn_s_sleep(DSL);
      }
    waitv<0>();
    if (lane == 0) st_sys(fl, call);
  }
  STAMP(myws, 5);
  STAMPV(myws, 0, tstart);
}

__global__ __launch_bounds__(256, 2) void fused_gemm_rs(Params P) {
  extern __shared__ __attribute__((aligned(16))) char smem[];
  const int bid = blockIdx.x;
  typedef u64 u64x4 __attribute__((ext_vector_type(4)));
  u64x4 a4;
  u64 tstart, pe;
  int rk, d;
  u32 tis, t0;
  u32x16 f16;
  u32x2 co;
  static_assert(N_PUSH_WG == 56 && PROT == 0 && PWN == 1 && CPB >= N_PUSH_WG, "entry code");
  asm volatile("v_readfirstlane_b32 s46, %[tid]\n\t"
               "s_load_dwordx8 s[36:43], %[ka], 0x0\n\t"
               "s_lshr_b32 s47, %[bid], 3\n\t"
               "s_cmp_gt_u32 %[bid], 55\n\t"
               "s_cbranch_scc1 2f\n\t"
               "s_lshr_b32 s48, s46, 7\n\t"
               "s_mul_i32 s49, s48, 3\n\t"
               "s_add_i32 s47, s47, s49\n\t"
               "s_add_i32 s49, s47, -7\n\t"
               "s_min_u32 s47, s47, s49\n\t"
               "s_lshl_b32 s49, s47, 4\n\t"
               "s_load_dwordx4 s[52:55], %[ka], s49 offset:0x40\n\t"
               "s_memrealtime s[44:45]\n\t"
               "s_load_dword s62, %[ka], 0x20\n\t"
               "s_setprio 3\n\t"
               "s_add_i32 s63, s47, 1\n\t"
               "v_lshlrev_b32 v1, 2, %[tid]\n\t"
               "v_mov_b32 v27, 0\n\t"
               "v_add_u32 v1, %[ctl], v1\n\t"
               "s_bitcmp1_b32 s46, 6\n\t"
               "ds_write_b32 v1, v27\n\t"
               "s_cbranch_scc0 1f\n\t"
               "s_and_b32 s49, %[bid], 7\n\t"
               "s_lshl1_add_u32 s48, s49, s48\n\t"
               "v_bfe_u32 v1, %[tid], 4, 2\n\t"
               "v_and_b32 v2, 15, %[tid]\n\t"
               "v_lshlrev_b32 v3, 2, v1\n\t"
               "v_lshlrev_b32 v1, 13, v1\n\t"
               "v_xor_b32 v2, v3, v2\n\t"
               "v_lshl_or_b32 v1, s48, 17, v1\n\t"
               "v_lshl_or_b32 v20, v2, 4, v1\n\t"
               "v_xor_b32 v21, 16, v20\n\t"
               "v_xor_b32 v22, 32, v20\n\t"
               "v_xor_b32 v23, 48, v20\n\t"
               "v_add_u32 v21, 0x7c00, v21\n\t"
               "v_add_u32 v22, 0xf800, v22\n\t"
               "v_add_u32 v23, 0x17400, v23\n\t"
               "s_lshl_b32 s49, s63, 7\n\t"
               "s_add_i32 s49, s49, %[lnk]\n\t"
               "v_mov_b32 v28, s49\n\t"
               "v_mov_b32 v25, 1\n\t"
               "s_waitcnt lgkmcnt(0)\n\t"
               "s_memrealtime s[58:59]\n\t"
               "global_load_dwordx4 v[144:147], v20, s[54:55] " MLP "\n\t"
               "global_load_dwordx4 v[148:151], v21, s[54:55] offset:1024 " MLP "\n\t"
               "global_load_dwordx4 v[152:155], v22, s[54:55] offset:2048 " MLP "\n\t"
               "global_load_dwordx4 v[156:159], v23, s[54:55] offset:3072 " MLP "\n\t"
               "s_mov_b64 exec, 1\n\t"
               "global_atomic_add v160, v28, v25, s[40:41] sc0 sc1\n\t"
               "s_add_i32 s50, s47, -1\n\t"
               "s_max_i32 s50, s50, 0\n\t"
               "s_mul_i32 s51, s50, %[fsl]\n\t"
               "s_lshl_b32 s49, s48, 7\n\t"
               "s_add_i32 s51, s51, s49\n\t"
               "s_add_i32 s51, s51, %[f0]\n\t"
               "v_mov_b32 v29, s51\n\t"
               "s_waitcnt lgkmcnt(0)\n\t"
               "s_or_b32 s66, s58, 1\n\t"
               "v_mov_b32 v26, s66\n\t"
               "global_atomic_cmpswap v161, v28, v[26:27], s[40:41] offset:4 sc0 sc1\n\t"
               "global_atomic_add v29, v27, s[52:53] sc1\n\t"
               "s_mov_b64 exec, -1\n"
               "1:\n\t"
               "s_waitcnt lgkmcnt(0)\n\t"
               "s_barrier\n\t"
               "s_branch 5f\n"
               "2:\n\t"
               "s_memrealtime s[44:45]\n\t"
               "s_load_dword s62, %[ka], 0x20\n\t"
               "s_load_dwordx2 s[52:53], %[ka], 0x40\n\t"
               "s_mov_b32 s63, 1\n\t"
               "s_add_i32 s47, %[bid], %[cpb]\n\t"
               "s_cmp_lt_u32 s47, 16\n\t"
               "s_cbranch_scc0 5f\n\t"
               "s_cmpk_lt_u32 s46, 0xc0\n\t"
               "s_cbranch_scc0 5f\n\t"
               "s_getpc_b64 s[56:57]\n\t"
               "s_mov_b32 m0, %[cpl]\n\t"
               "v_lshlrev_b32 v1, 7, %[tid]\n\t"
               "v_min_u32 v1, %[cpf], v1\n\t"
               "global_load_lds_dword v1, s[56:57]\n"
               "5:\n\t"
               "s_waitcnt lgkmcnt(0)"
               : "={s[36:43]}"(a4), "={s[44:45]}"(tstart), "={s46}"(t0), "={s[52:53]}"(pe), "={s62}"(rk), "={s63}"(d),
                 "={s66}"(tis)
               : [ka] "s"(__builtin_amdgcn_kernarg_segment_ptr()), [bid] "s"(bid), [tid] "v"(threadIdx.x),
                 [ctl] "i"(65536), [lnk] "i"(LNK_OFF), [fsl] "i"(IN_SRC + 4u * PIECE), [f0] "i"(NS * 2u * PIECE + 4096u),
                 [cpb] "i"(-CPB), [cpl] "i"(66048), [cpf] "i"(CPF - 128)
               : "memory", "scc", "s47", "s48", "s49", "s50", "s51", "s54", "s55", "s56", "s57", "s58", "s59",
                 "v1", "v2", "v3", "v20", "v21", "v22", "v23", "v25", "v26", "v27", "v28", "v29");
  asm volatile("" : "={v[144:159]}"(f16), "={v[160:161]}"(co));
  const u32 lane = threadIdx.x & 63u;
  const int wave = (int)(t0 >> 6);
  const Args A = {(const char*)a4[0], (const char*)a4[1], (char*)a4[2], (char*)a4[3], (char*)pe, rk, tis, f16, co};
  if (bid < N_PUSH_WG) mover(A, d, wave, lane, smem, tstart);
  else if (bid >= 256 && bid < 256 + N_PUSH_WG) {
    for (int i = 0; i < 6; ++i) __builtin_amdgcn_s_sleep(16);
  } else tile(A, bid < 256 - ROT ? bid - N_PUSH_WG + ROT : (bid < 256 ? bid - (256 - ROT) : bid - 2 * N_PUSH_WG), wave, lane, smem, tstart);
}

}

extern "C" void kernel_launch(int rank, int nranks, ncclComm_t comm,
                              hipStream_t stream,
                              void* const* d_in, const int* in_sizes, int n_in,
                              void* d_out, int out_size,
                              const int* in_dtypes, int out_dtype,
                              void* const* d_out_all,
                              void* const* d_ws_all, size_t ws_bytes) {
  if (nranks != NR || n_in < 2 || ws_bytes < (size_t)WS_NEED || (unsigned)rank >= (unsigned)NR) {
    fprintf(stderr, "unsupported configuration\n");
    return;
  }
  Params P;
  P.x = (const char*)d_in[0];
  P.w = (const char*)d_in[1];
  P.out = (char*)d_out;
  P.my = (char*)d_ws_all[rank];
  for (int i = 0; i < NR; ++i) {
    const int r = rank ^ i;
    P.lk[i].peer = (char*)d_ws_all[r];
    P.lk[i].w0 = P.w + (size_t)((r & 1) * (256 * 8192 + 15 * 256) + (r >> 2) * 4096);
  }
  P.rank = rank;
  P.pad[0] = P.pad[1] = P.pad[2] = 0;
  HIP_CHECK(hipFuncSetAttribute((const void*)fused_gemm_rs, hipFuncAttributeMaxDynamicSharedMemorySize, (int)LDS_BYTES));
  hipLaunchKernelGGL(fused_gemm_rs, dim3(N_WG), dim3(256), LDS_BYTES, stream, P);
  HIP_CHECK(hipGetLastError());
}
```

```cpp
#include <hip/hip_runtime.h>
#include <rccl/rccl.h>
#include <cstdio>
#include <cstdint>

#define HIP_CHECK(expr) do { hipError_t _e = (expr); if (_e != hipSuccess) { \
    fprintf(stderr, "HIP error %s at %s:%d\n", hipGetErrorString(_e), __FILE__, __LINE__); \
    return; } } while (0)

namespace {

typedef unsigned u32;
typedef unsigned long long u64;
typedef u32 u32x4 __attribute__((ext_vector_type(4)));
typedef u32 u32x2 __attribute__((ext_vector_type(2)));
typedef u32 u32x16 __attribute__((ext_vector_type(16)));
typedef float f32x2 __attribute__((ext_vector_type(2)));
typedef float f32x16 __attribute__((ext_vector_type(16)));
typedef __bf16 bf16x2 __attribute__((ext_vector_type(2)));
typedef __bf16 bf16x8 __attribute__((ext_vector_type(8)));
typedef short s16x4 __attribute__((ext_vector_type(4)));
typedef short s16x8 __attribute__((ext_vector_type(8)));
#define LDSP __attribute__((address_space(3)))
#define GLBP __attribute__((address_space(1)))

constexpr int NR = 8;
constexpr u32 PIECE = 65536u;
constexpr u32 MIB = 1u << 20;
#define NS 2
#define OX 14
#define TAU 112
#define T0 20
#define QA 0
#define DA 0
#define QB 20
#define DB 1
#define QC 99
#define DC 0
#define TQ(q) (T0 + TAU * (q) - DA * ((q) < QA ? (q) : QA) - DB * ((q) > QB ? (q) - QB : 0))
#define EARLY 150
#define TAU1 110
#define TAU2 0
#define STGD 2
#define PSL 4
#define LSL 0
#define WSL 3
#define RSL 0
#define WOWN 0
#define ALN 1
#define ALO 99
#define ALT 25
#define PWN 1
#define DIR 0
#define DSL 4
#define SWP 14
#define CPF 14080
#define CPB 56
#define PD0 0
#define PD1 0
#define PDF 0
#define GSH -18
#define ORG 1
#define OC 34
#define OG 10
#define OSP 7
#define ORD 1
#define OD 20
#define CLT 84
#define CKT 44
#define CLD 68
#define LPT 400
#define ROT 48
#define PWIN 64
#define PWHO 1
#define PQ1 9
#define PQ2 14
#define PROT 0
#define XR 32
#define ZT0 900
#define ZDT 100
#define RPT 400
#define RPB 25
#define XCT 0
#define L1T 500
#define SXT 6
#define PRE 1
#define WMK 1ull
#define G1 0
#define LAG 1
#define WIN 3
#define GLP "sc0 sc1"
#define MLP "sc0 sc1"
constexpr int NIN = 30;
constexpr int NOUT = 16 - NS;
constexpr int NOUT1 = 16 + 6 * NS;
constexpr u32 IN_SRC = 4u * MIB;
constexpr u32 PIN_OFF = 24u * MIB;
constexpr u32 PIN_PAR = 8u * MIB;
constexpr u32 POUT_OFF = 48u * MIB;
constexpr u32 FLG_OFF = 55u * MIB;
constexpr u32 FLOC_OFF = FLG_OFF + 16384u;
constexpr u32 TMO_OFF = FLG_OFF + 49152u;
constexpr u32 LNK_OFF = FLG_OFF + MIB + 262144u + 131072u;
constexpr u32 DUMMY_OFF = FLG_OFF + 65536u;
constexpr u32 DBG_OFF = FLG_OFF + 131072u;
constexpr u32 CNT_OFF = FLG_OFF + MIB + 262144u;
constexpr u32 WS_NEED = FLG_OFF + 2u * MIB;

constexpr int N_PUSH_WG = 56;
constexpr int N_MAIN_WG = 256;
constexpr int N_FIN_WG = 96;
constexpr int N_WG = 2 * N_PUSH_WG + N_MAIN_WG + N_FIN_WG;
constexpr u32 STAGE_B = 16384u;
constexpr u32 LDS_BYTES = 4u * STAGE_B + 1024u;
constexpr u64 DEADLINE = 100000000ull;
constexpr u32 DEAD = 0x7fffffffu;

struct Params {
  const char* x;
  const char* w;
  char* my;
  char* out;
  int rank;
  int pad[3];
  struct { char* peer; const char* w0; } lk[NR];
};
struct Args { const char* x; const char* w; char* my; char* out; char* peer; int rank; u32 tis; u32x16 f16; u32x2 co; };

#define SB() __builtin_amdgcn_sched_barrier(0)
template <int N> __device__ __forceinline__ void waitv() { asm volatile("s_waitcnt vmcnt(%0)" :: "i"(N) : "memory"); }

__device__ __forceinline__ u64 now() { return __builtin_amdgcn_s_memrealtime(); }
template <typename T> __device__ __forceinline__ T* uni(T* p) {
  const u64 v = (u64)(uintptr_t)p;
  const u32 lo = __builtin_amdgcn_readfirstlane((u32)v), hi = __builtin_amdgcn_readfirstlane((u32)(v >> 32));
  return (T*)(uintptr_t)(((u64)hi << 32) | lo);
}
__device__ __forceinline__ u32 ld_sys(const void* p) {
  return __hip_atomic_load((const GLBP u32*)(uintptr_t)p, __ATOMIC_RELAXED, __HIP_MEMORY_SCOPE_SYSTEM);
}
__device__ __forceinline__ void st_sys(void* p, u32 v) {
  __hip_atomic_store((GLBP u32*)(uintptr_t)p, v, __ATOMIC_RELAXED, __HIP_MEMORY_SCOPE_SYSTEM);
}
__device__ __forceinline__ void st_agt(void* p, u32 v) {
  __hip_atomic_store((GLBP u32*)(uintptr_t)p, v, __ATOMIC_RELAXED, __HIP_MEMORY_SCOPE_AGENT);
}
__device__ __forceinline__ bool give_up(u64 t0, const char* ws, u32 call) {
  const u64 dt = now() - t0;
  if (!__builtin_amdgcn_readfirstlane((int)(dt > 50000ull))) return false;
  __builtin_amdgcn_s_sleep(64);
  if (__builtin_amdgcn_readfirstlane((int)(dt > DEADLINE))) return true;
  return __builtin_amdgcn_readfirstlane(ld_sys(ws + TMO_OFF)) == call;
}
__device__ __forceinline__ void sleep_until(u64 tstart, u32 t) {
  for (;;) {
    const u32 el = (u32)__builtin_amdgcn_readfirstlane((u32)(now() - tstart));
    if (el >= t) break;
    if (t - el > 40u) __builtin_amdgcn_s_sleep(12); else __builtin_amdgcn_s_sleep(1);
  }
}
__device__ __forceinline__ void sleep_exact(u64 tstart, u32 t) {
  for (;;) {
    const u32 el = (u32)__builtin_amdgcn_readfirstlane((u32)(now() - tstart));
    if (el >= t) break;
    if (t - el > 60u) __builtin_amdgcn_s_sleep(12);
    else if (t - el > 20u) __builtin_amdgcn_s_sleep(3);
    else if (t - el > (u32)SXT) __builtin_amdgcn_s_sleep(1);
  }
}
template <int IMM> __device__ __forceinline__ void st16(u32 off, u32x4 v, char* base) {
  asm volatile("global_store_dwordx4 %0, %1, %2 offset:%3 sc0 sc1\n\ts_nop 1" :: "v"(off), "v"(v), "s"(base), "i"(IMM) : "memory");
}
template <int IMM> __device__ __forceinline__ void st16p(u32 off, u32x4 v, char* base) {
  asm volatile("global_store_dwordx4 %0, %1, %2 offset:%3\n\ts_nop 1" :: "v"(off), "v"(v), "s"(base), "i"(IMM) : "memory");
}
#define OST st16
__device__ __forceinline__ void dma16x4(u32 lds, const u32 (&o)[4], const char* base) {
  asm volatile("s_mov_b32 m0, %0\n\ts_nop 3\n\t"
               "global_load_lds_dwordx4 %1, %5 " MLP "\n\t"
               "global_load_lds_dwordx4 %2, %5 offset:1024 " MLP "\n\t"
               "global_load_lds_dwordx4 %3, %5 offset:2048 " MLP "\n\t"
               "global_load_lds_dwordx4 %4, %5 offset:3072 " MLP
               :: "s"(lds), "v"(o[0]), "v"(o[1]), "v"(o[2]), "v"(o[3]), "s"(base) : "memory");
}
__device__ __forceinline__ void probe(u32 off, char* base) {
  asm volatile("s_nop 4\n\tglobal_atomic_add %0, %1, %2 sc1" :: "v"(off), "v"(0u), "s"(base) : "memory");
}
__device__ __forceinline__ void probe0(u32 off, char* base) {
  asm volatile("s_mov_b64 exec, 1\n\ts_nop 3\n\tglobal_atomic_add %0, %1, %2 sc1\n\ts_mov_b64 exec, -1"
               :: "v"(off), "v"(0u), "s"(base) : "memory");
}
__device__ __forceinline__ u32 atom_rd(u32 off, const char* base) {
  u32 r;
  asm volatile("global_atomic_add %0, %1, %2, %3 sc0 sc1\n\ts_waitcnt vmcnt(0)" : "=&v"(r) : "v"(off), "v"(0u), "s"(base) : "memory");
  return r;
}
__device__ __forceinline__ u32 bump_issue(u32 off, const char* base) {
  u32 r;
  asm volatile("s_mov_b64 exec, 1\n\ts_nop 3\n\tglobal_atomic_add %0, %1, %2, %3 sc0 sc1\n\ts_mov_b64 exec, -1"
               : "=&v"(r) : "v"(off), "v"(1u), "s"(base) : "memory");
  return r;
}
template <int N> __device__ __forceinline__ u32 bump_get(u32 r) {
  u32 v;
  asm volatile("s_waitcnt vmcnt(%2)\n\tv_readfirstlane_b32 %0, %1\n\ts_nop 4" : "=s"(v) : "v"(r), "i"(N) : "memory");
  return v + 1u;
}
__device__ __forceinline__ void add1_lane0(u32 off, char* base) {
  asm volatile("s_mov_b64 exec, 1\n\ts_nop 3\n\tglobal_atomic_add %0, %1, %2 sc1\n\ts_mov_b64 exec, -1"
               :: "v"(off), "v"(1u), "s"(base) : "memory");
}
__device__ __forceinline__ void st4_lane0(u32 off, u32 val, char* base) {
  asm volatile("s_mov_b64 exec, 1\n\ts_nop 3\n\tglobal_store_dword %0, %1, %2 sc0 sc1\n\ts_mov_b64 exec, -1"
               :: "v"(off), "v"(val), "s"(base) : "memory");
}

#define STAMP(ws, idx) do {} while (0)
#define STAMPV(ws, idx, v) do {} while (0)

__device__ __forceinline__ void piece_of(int q, int d, bool& isx, int& idx) {
  const int t = q - NS, u = q - NS - 12;
  int j = (t >> 1) + 2;
  j += j >= d ? 1 : 0;
  j = j == 8 ? 1 : j;
  isx = (q >= NS && u < 0) || (u >= OX && u < OX + 2);
  idx = q < NS ? q : (u < 0 ? 2 * j + (t & 1) : (u < OX ? NS + u : (u < OX + 2 ? u - OX : NS + u - 2)));
}
__device__ __forceinline__ int slot_w(int k) { return k < NS ? k : (k - NS < OX ? 12 + k : 14 + k); }
__device__ __forceinline__ int slot_x(int j, int e, int d) {
  return j == 0 ? NS + 12 + OX + e : NS + 2 * (j == 1 ? 5 : j - 2 - (j > d ? 1 : 0)) + e;
}
__device__ __forceinline__ int cont_w(int k) { return k < NS ? k : 14 + k; }
__device__ __forceinline__ int cont_x(int rb) { return rb < 2 ? 30 + rb : (rb < 4 ? NS + 10 + rb : NS - 4 + rb); }
__device__ __forceinline__ u32 fin_off(int sl, int q) { return (u32)sl * IN_SRC + (u32)(NS + 2 * sl) * 2u * PIECE + (u32)q * 64u; }
__device__ __forceinline__ u32 lds_rd(u32 a) { return __builtin_amdgcn_readfirstlane(*(volatile LDSP u32*)(uintptr_t)a); }
__device__ __forceinline__ void lds_wr(u32 a, u32 v) { *(volatile LDSP u32*)(uintptr_t)a = v; }

__device__ __forceinline__ void mover(const Args& P, int d, int wave, u32 lane, char* smem, u64 tstart) {
  const int s = P.rank;
  char* const myws = P.my;
  const int pair = wave >> 1;
  const int p = ((blockIdx.x + PROT) & 7) * 2 + pair;
  const u32 r = (u32)(s ^ d) & (u32)(NR - 1);
  const int nin = d == 1 ? 0 : NIN;
  const int np = d == 1 ? (DIR ? 16 : NOUT1) : NIN + NOUT;
  const int nid = d == 1 ? NOUT1 : NOUT;
  const u32 ring = (u32)(uintptr_t)smem + (u32)pair * 32768u;
  const u32 ctl = (u32)(uintptr_t)smem + 65536u + (u32)pair * 512u;
  const char* const xin = P.x;
  const char* const win = P.w;
  const int rg = r >> 2, rb_ = (r >> 1) & 1, ra = r & 1;
  u32 wo[4];
#pragma unroll
  for (int k = 0; k < 4; ++k) {
    const u32 q4 = lane >> 4;
    wo[k] = ((u32)p * 16u + (u32)k * 4u + q4) * 8192u + (((lane & 15u) ^ ((q4 << 2) | (u32)k)) * 16u) - (u32)k * 1024u;
  }
  constexpr u32 XC = 256u / XR;
  const u32 xrow = ((u32)p % (128u / XR)) * XR, xks = (u32)p / (128u / XR) * (XC / 4u);
#define W_BASE(IX) (win + (size_t)((u32)ra * 256u * 8192u + (u32)rg * 4096u + (u32)(ra ? 15 - (IX) : (IX)) * 256u))

  if ((wave & 1) == 0) {
    const u32 cnt = bump_issue(CNT_OFF + (u32)(blockIdx.x * 4 + wave) * 64u, myws);
    u32 xo[4], oo[4];
    {
#pragma unroll
      for (int k = 0; k < 4; ++k) {
        const u32 c = (u32)k * 64u + lane;
        xo[k] = (xrow + c / XC) * 1024u + xks * 64u + (c % XC) * 16u - (u32)k * 1024u;
        oo[k] = (u32)p * 4096u + lane * 16u;
      }
    }
    u32 poll_off;
    {
      const int i0 = (int)lane < nid ? (int)lane : 0;
      const bool sw = d == 1 && i0 < SWP;
      const int i = sw ? (i0 & ~1) | (p >> 3) : i0, ps = sw ? ((i0 & 1) << 3) | (p & 7) : p;
      const int jj = d != 1 ? d : (i < 6 * NS ? 2 + i / NS : 1);
      const int kk = d != 1 ? NS + i : (i < 6 * NS ? i % NS : i - 6 * NS);
      poll_off = FLOC_OFF + (u32)((jj - 1) * 8 + (ps >> 1)) * 128u + (u32)kk * 4u;
    }
#define ISSUE_LOADS(QQ, II) do { \
      const int q_ = (QQ); \
      const u32 dst_ = ring + (u32)(q_ & 7) * 4096u; \
      if (q_ < nin) { \
        bool isx_; int ix_; \
        piece_of(q_, d, isx_, ix_); \
        if (isx_) dma16x4(dst_, xo, xin + (size_t)(((r ^ (u32)(ix_ >> 1)) * 512u + (u32)rb_ * 256u + (u32)(ix_ & 1) * 128u) * 1024u + (u32)ra * 512u)); \
        else      dma16x4(dst_, wo, W_BASE(ix_)); \
      } else { \
        const bool sw_ = d == 1 && (II) < SWP; \
        const int i_ = sw_ ? ((II) & ~1) | (p >> 3) : (II); \
        const int jj_ = d != 1 ? d : (i_ < 6 * NS ? 2 + i_ / NS : 1); \
        const int kk_ = d != 1 ? NS + i_ : (i_ < 6 * NS ? i_ % NS : i_ - 6 * NS); \
        dma16x4(dst_, oo, myws + (size_t)(POUT_OFF + (u32)((jj_ - 1) * 16 + kk_) * PIECE + (u32)(sw_ ? (((II) & 1) - (p >> 3)) * 32768 : 0))); \
      } \
    } while (0)

    int q0 = 0;
    u32 pub = 0, call;
    u32 dbg1 = 0, dbg2 = 0;
    if (nin) {
      ISSUE_LOADS(1, 0); ISSUE_LOADS(2, 0); ISSUE_LOADS(3, 0);
    }
    if (nin) {
      call = bump_get<8>(cnt);
      lds_wr(ctl, 2u);
      q0 = 4; pub = 2;
    } else {
      call = bump_get<0>(cnt);
    }

    if (!nin) sleep_until(tstart, (u32)(TQ(NS + (DIR ? 12 : 2)) + L1T));
    u64 ready = 0;
    u64 todo = ((1ull << (np - nin)) - 1ull) << (nid - (np - nin));
    u32 cons = 0;
    bool dead = false;
    for (int q = q0; q < np; ++q) {
      bool rdy = q < nin || (ready & todo) != 0ull;
      if (cons + 8u <= (u32)q) cons = lds_rd(ctl + 4u);
      if (!rdy || cons + 8u <= (u32)q) {
        waitv<0>();
        pub = (u32)q;
        lds_wr(ctl, pub);
        const u64 t0 = now();
        u32 win = 64u;
        for (;;) {
          if (!rdy) {
            u32 pv = 0;
            const u32 below = __builtin_amdgcn_mbcnt_hi((u32)(todo >> 32), __builtin_amdgcn_mbcnt_lo((u32)todo, 0u));
            if (((todo >> lane) & 1ull) && below < win) pv = atom_rd(poll_off, myws);
            win = (u32)PWIN;
            ready |= __builtin_amdgcn_ballot_w64(pv >= call);
            rdy = (ready & todo) != 0ull;
            if (LSL && !rdy) __builtin_amdgcn_s_sleep(LSL);
          } else {
            __builtin_amdgcn_s_sleep(2);
          }
          cons = lds_rd(ctl + 4u);
          if (rdy && cons + 8u > (u32)q) break;
          if (give_up(t0, myws, call)) { dead = true; break; }
        }
        if (dead) break;
      }
      int id = 0;
      if (q >= nin) {
        id = __builtin_ctzll(ready & todo);
        todo &= ~(1ull << id);
        lds_wr(ctl + 32u + (u32)(q & 7) * 4u, (u32)id);
      }
      ISSUE_LOADS(q, id);
      waitv<8>();
      if (PWHO == 1 && nin && q == PQ1) probe0((u32)(d - 2) * IN_SRC + 2u * MIB + (u32)p * 4096u, P.peer);
      if (PWHO == 1 && nin && q == PQ2) probe0(PIN_OFF + (call % 3u) * PIN_PAR + (u32)(32 + (d - 2) * 16 + NS) * PIECE + (u32)p * 4096u, P.peer);
      if (q - 1 > (int)pub) { pub = (u32)(q - 1); lds_wr(ctl, pub); }
      if (q == nin - 1) STAMP(myws, 1);
    }
    waitv<0>();
    if (dead) st4_lane0(TMO_OFF, call, myws);
    lds_wr(ctl, dead ? DEAD : (u32)np);
    STAMP(myws, 3);
    STAMPV(myws, 0, tstart);
    return;
  }

  char* const rws = P.peer;
  const int slot = d - 2;
  const u32 dl = (u32)p * 4096u + lane * 16u;
  u32 xl[4], xd[4];
#pragma unroll
  for (int k = 0; k < 4; ++k) {
    const u32 c = (u32)k * 64u + lane, t = c / (4u * XR), row = (c >> 2) % XR;
    xl[k] = (row * XC + t * 4u + ((c ^ ((xrow + row) >> 2)) & 3u)) * 16u;
    xd[k] = XCT ? dl : (xks + t) * 8192u + (xrow + row) * 64u + (c & 3u) * 16u - (u32)k * 1024u;
  }
  u32 tick, ts0, ta;
  int to;
  {
    u32 tg, tm;
    asm volatile("" : "+v"(xl[0]), "+v"(xl[1]), "+v"(xl[2]), "+v"(xl[3]), "+v"(xd[0]), "+v"(xd[1]), "+v"(xd[2]), "+v"(xd[3]));
    asm volatile("s_waitcnt vmcnt(1)\n\t"
                 "v_readfirstlane_b32 %[tick], v160\n\t"
                 "v_readfirstlane_b32 %[tg], v161\n\t"
                 "s_memrealtime s[58:59]\n\t"
                 "s_sub_i32 %[tm], %[tg], %[tis]\n\t"
                 "s_addk_i32 %[tm], 500\n\t"
                 "s_cmpk_gt_u32 %[tm], 1000\n\t"
                 "s_cselect_b32 %[tg], %[tis], %[tg]\n\t"
                 "s_addk_i32 %[tg], %[oc]\n\t"
                 "s_sub_i32 %[to], %[tg], %[tst]\n\t"
                 "s_and_b32 %[tm], %[tick], 15\n\t"
                 "s_mul_i32 %[tm], %[tm], %[osp]\n\t"
                 "s_add_i32 %[tg], %[tg], %[tm]\n\t"
                 "s_bitcmp0_b32 %[tick], 4\n\t"
                 "s_cselect_b32 %[tm], 0x10000, 0\n\t"
                 "v_add_u32 v160, %[tm], %[dl]\n\t"
                 "s_cmp_eq_u32 %[d], 1\n\t"
                 "s_waitcnt lgkmcnt(0)\n\t"
                 "s_mov_b32 %[ta], s58\n\t"
                 "s_cbranch_scc1 5f\n"
                 "4:\n\t"
                 "s_waitcnt lgkmcnt(0)\n\t"
                 "s_sub_i32 %[tm], s58, %[tg]\n\t"
                 "s_mov_b32 %[ts0], s58\n\t"
                 "s_memrealtime s[58:59]\n\t"
                 "s_cmp_lt_i32 %[tm], 0\n\t"
                 "s_cbranch_scc1 4b\n\t"
                 "global_store_dwordx4 v160, v[144:147], %[dst] sc0 sc1\n\t"
                 "global_store_dwordx4 v160, v[148:151], %[dst] offset:1024 sc0 sc1\n\t"
                 "global_store_dwordx4 v160, v[152:155], %[dst] offset:2048 sc0 sc1\n\t"
                 "global_store_dwordx4 v160, v[156:159], %[dst] offset:3072 sc0 sc1\n"
                 "5:\n\t"
                 "s_waitcnt lgkmcnt(0)"
                 : [tick] "=&s"(tick), [ts0] "=&s"(ts0), [ta] "=&s"(ta), [to] "=&s"(to), [tg] "=&s"(tg), [tm] "=&s"(tm)
                 : "{v[144:159]}"(P.f16), "{v[160:161]}"(P.co), [tis] "s"(P.tis), [tst] "s"((u32)tstart), [d] "s"(d),
                   [dl] "v"(dl), [dst] "s"(rws + (size_t)((u32)max(slot, 0) * IN_SRC)), [oc] "i"(OC), [osp] "i"(OSP)
                 : "memory", "scc", "s58", "s59");
  }
  const u32 call = (tick >> 4) + 1u;
  u32 dbg1 = 0, dbg2 = 0;
  const u32 in0 = (u32)slot * IN_SRC + (call & 1u) * PIECE;
  const u32 pin0 = PIN_OFF + (call % 3u) * PIN_PAR + (u32)(d == 1 ? 0 : 32 + slot * 16 + NS) * PIECE;
  int nflag = 0;
#define FLAG(QQ) do { \
    const int q_ = (QQ); \
    const bool real_ = q_ == nflag && q_ < nin; \
    const u32 o_ = real_ ? fin_off(slot, q_) : DUMMY_OFF + (u32)(blockIdx.x * 4 + wave) * 128u; \
    add1_lane0(o_, uni(real_ ? rws : myws)); \
    nflag += real_ ? 1 : 0; \
  } while (0)

  u32 tgrid = 0, tord = 0;
  if (nin) {
    FLAG(-1);
    const int cold = ta - P.tis > (u32)CLT || P.tis - (u32)tstart > (u32)CKT ? CLD : 0;
    const u32 tb = (u32)max(to + TAU + OG + cold, (int)(ts0 - (u32)tstart) + OD) + (u32)(TAU - 1);
    tgrid = tb - ((u32)tstart + tb - ((u32)(((p + STGD * d) & 15) * TAU) >> 4)) % (u32)TAU;
    tord = ORD > 1 ? (u32)max(to + TAU + OG + (int)(tick & 15u) * OSP, 0) - tgrid : 0u;
  } else {
    sleep_until(tstart, 600u);
    probe0(pin0 + (u32)p * 4096u, rws);
    probe0(pin0 + (u32)(NOUT1 - 1) * PIECE + (u32)p * 4096u, rws);
  }
  u32 full = 0, tnext = 0;
  for (int q = nin ? 1 : 0; q < np; ++q) {
    bool isx; int ix;
    piece_of(q, d, isx, ix);
    isx = isx && q < nin;
    if (d != 1 || G1) tnext = max(tnext, tgrid + (q < ORD ? tord : 0u));
    if (!PRE) sleep_until(tstart, tnext);
    if (full <= (u32)q) {
      full = lds_rd(ctl);
      if (full <= (u32)q) {
        bool drained = !(q > 0 && q < nin + LAG);
        u64 el;
        for (;;) {
          __builtin_amdgcn_s_sleep(1);
          full = lds_rd(ctl);
          el = now() - tstart;
          if (full > (u32)q) break;
          if (!drained && __builtin_amdgcn_readfirstlane((int)(el > (u64)(tnext + 30u)))) {
            drained = true;
            waitv<0>();
#pragma unroll
            for (int z = LAG; z >= 1; --z) FLAG(q - z);
          }
          if (__builtin_amdgcn_readfirstlane((int)(el > 2ull * DEADLINE - 1000ull))) { full = DEAD; break; }
        }
        tnext = max(tnext, (u32)__builtin_amdgcn_readfirstlane((u32)el));
      }
      if (full == DEAD) break;
    }
    const u32 la = ring + (u32)(q & 7) * 4096u;
    const u32x4 d0 = *(volatile LDSP u32x4*)(uintptr_t)(la + (isx ? xl[0] : lane * 16u));
    const u32x4 d1 = *(volatile LDSP u32x4*)(uintptr_t)(la + (isx ? xl[1] : lane * 16u + 1024u));
    const u32x4 d2 = *(volatile LDSP u32x4*)(uintptr_t)(la + (isx ? xl[2] : lane * 16u + 2048u));
    const u32x4 d3 = *(volatile LDSP u32x4*)(uintptr_t)(la + (isx ? xl[3] : lane * 16u + 3072u));
    const u32 id = lds_rd(ctl + 32u + (u32)(q & 7) * 4u);
    lds_wr(ctl + 4u, (u32)q + 1u);
    u32 o;
    if (q < nin) {
      o = in0 + (u32)(isx ? cont_x(ix) : cont_w(ix)) * 2u * PIECE;
    } else {
      o = pin0 + min(id, (u32)(nid - 1)) * PIECE;
      if (d == 1 && id < (u32)SWP) o += (u32)(((p >> 3) - (int)(id & 1u)) * 32768);
    }
    char* const dst = rws + (size_t)o;
    if (PRE) sleep_exact(tstart, tnext);
    tgrid += (u32)(TAU - (q < QA ? DA : 0) - (q >= QB ? DB : 0) - (q >= QC ? DC : 0));
    tnext += (u32)(d == 1 && !G1 ? TAU1 : TAU2);
    st16<0>(isx ? xd[0] : dl, d0, dst);
    st16<1024>(isx ? xd[1] : dl, d1, dst);
    st16<2048>(isx ? xd[2] : dl, d2, dst);
    st16<3072>(isx ? xd[3] : dl, d3, dst);
    if (nin && q < nin + LAG) {
      waitv<5 * LAG>();
      FLAG(q - LAG);
      if (PWHO == 0 && q == PQ1) probe0((u32)slot * IN_SRC + 2u * MIB + (u32)p * 4096u, rws);
      if (PWHO == 0 && q == PQ2) probe0(pin0 + (u32)p * 4096u, rws);
    } else {
      waitv<4 * WIN>();
    }
    if (q == NS + 11) STAMP(myws, 1);
    if (q == nin - 1) STAMP(myws, 2);
    if (q == np - 1) STAMP(myws, 4);
  }
  if ((tick & 15u) == 0u) st4_lane0(LNK_OFF + (u32)d * 128u + 4u, 0u, myws);
  waitv<0>();
  STAMP(myws, 3);
  STAMPV(myws, 0, tstart);
}

struct Frag {
  bf16x8 x[2][2];
  bf16x8 w[2][2];
};

struct Loader {
  const char* base;
  u32 stride[2];
  u32 v[4];
  u32 m0base;
};

template <int STAGE> __device__ __forceinline__ void glds_pair(const Loader& L, int half) {
  const u32 m0v = L.m0base + STAGE * STAGE_B;
  if (half == 0)
    asm volatile("s_mov_b32 m0, %0\n\ts_nop 0\n\t"
                 "global_load_lds_dwordx4 %1, %3 " GLP "\n\t"
                 "global_load_lds_dwordx4 %2, %3 offset:1024 " GLP
                 :: "s"(m0v), "v"(L.v[0]), "v"(L.v[1]), "s"(L.base) : "memory");
  else
    asm volatile("s_mov_b32 m0, %0\n\ts_nop 0\n\t"
                 "global_load_lds_dwordx4 %1, %3 offset:2048 " GLP "\n\t"
                 "global_load_lds_dwordx4 %2, %3 offset:3072 " GLP
                 :: "s"(m0v), "v"(L.v[2]), "v"(L.v[3]), "s"(L.base) : "memory");
}

struct RdAddr { u32 xa[2]; u32 wa[2][2]; };

template <int STAGE> __device__ __forceinline__ bf16x8 rd_x(const RdAddr& A, int mt, int ks) {
  return *(const LDSP bf16x8*)(uintptr_t)(A.xa[ks] + (u32)(STAGE * STAGE_B + mt * 2048));
}
template <int STAGE> __device__ __forceinline__ bf16x8 rd_w(const RdAddr& A, int nt, int ks) {
  s16x4 lo = __builtin_amdgcn_ds_read_tr16_b64_v4i16((LDSP s16x4*)(uintptr_t)(A.wa[nt][0] + (u32)(STAGE * STAGE_B + 8192 + ks * 4096)));
  s16x4 hi = __builtin_amdgcn_ds_read_tr16_b64_v4i16((LDSP s16x4*)(uintptr_t)(A.wa[nt][1] + (u32)(STAGE * STAGE_B + 8192 + ks * 4096)));
  s16x8 v = __builtin_shufflevector(lo, hi, 0, 1, 2, 3, 4, 5, 6, 7);
  return __builtin_bit_cast(bf16x8, v);
}
#define MFMA(nt, mt, ks) acc[nt][mt] = __builtin_amdgcn_mfma_f32_32x32x16_bf16(cur.w[nt][ks], cur.x[mt][ks], acc[nt][mt], 0, 0, 0)

template <int S, int VM, bool LOAD, bool READ>
__device__ __forceinline__ void kstep(f32x16 (&acc)[2][2], const Frag& cur, Frag& nxt, const RdAddr& A, Loader& L) {
  constexpr int SN = (S + 1) & 3;
  if (READ) {
    __builtin_amdgcn_s_waitcnt((VM & 15) | (7 << 4) | (0 << 8) | ((VM >> 4) << 14));
    asm volatile("s_barrier" ::: "memory");
  } else {
    __builtin_amdgcn_s_waitcnt(15 | (7 << 4) | (0 << 8) | (3 << 14));
  }
  SB();
  MFMA(0, 0, 0); SB();
  if (READ) { nxt.w[0][0] = rd_w<SN>(A, 0, 0); nxt.x[0][0] = rd_x<SN>(A, 0, 0); } SB();
  MFMA(0, 1, 0); SB();
  if (READ) { nxt.x[1][0] = rd_x<SN>(A, 1, 0); nxt.w[1][0] = rd_w<SN>(A, 1, 0); } SB();
  MFMA(1, 0, 0); SB();
  if (LOAD) glds_pair<S>(L, 0);
  SB();
  MFMA(1, 1, 0); SB();
  if (READ) { nxt.w[0][1] = rd_w<SN>(A, 0, 1); nxt.x[0][1] = rd_x<SN>(A, 0, 1); } SB();
  MFMA(0, 0, 1); SB();
  if (READ) { nxt.x[1][1] = rd_x<SN>(A, 1, 1); nxt.w[1][1] = rd_w<SN>(A, 1, 1); } SB();
  MFMA(0, 1, 1); SB();
  if (LOAD) { glds_pair<S>(L, 1); L.base += L.stride[S & 1]; }
  SB();
  MFMA(1, 0, 1); SB();
  MFMA(1, 1, 1); SB();
}

template <int S> __device__ __forceinline__ void stage_load(Loader& L) {
  glds_pair<S>(L, 0); glds_pair<S>(L, 1); L.base += L.stride[S & 1];
}

__device__ __forceinline__ void gemm_phase(f32x16 (&acc)[2][2], const RdAddr& A, Loader& L, int n, int sk, u32 jump, u32& dbgv, int dbg0) {
  Frag f0, f1;
  L.base = uni(L.base + (sk == 0 ? IN_SRC : 0u));
  L.stride[0] = __builtin_amdgcn_readfirstlane(L.stride[0]);
  L.stride[1] = __builtin_amdgcn_readfirstlane(L.stride[1]);
  asm volatile("s_nop 4" ::: "memory");
  stage_load<0>(L); stage_load<1>(L); stage_load<2>(L); stage_load<3>(L);
  waitv<12>();
  asm volatile("s_barrier" ::: "memory");
#pragma unroll
  for (int ks = 0; ks < 2; ++ks)
#pragma unroll
    for (int i = 0; i < 2; ++i) { f0.w[i][ks] = rd_w<0>(A, i, ks); f0.x[i][ks] = rd_x<0>(A, i, ks); }
  SB();
  for (int t = 0; t < n - 4; t += 4) {
    L.base += (t & 4 ? jump : 0u) + (t + 4 == 8 * sk ? IN_SRC : 0u);
    kstep<0, 8, true, true>(acc, f0, f1, A, L);
    kstep<1, 8, true, true>(acc, f1, f0, A, L);
    kstep<2, 8, true, true>(acc, f0, f1, A, L);
    kstep<3, 8, true, true>(acc, f1, f0, A, L);
  }
  kstep<0, 8, false, true>(acc, f0, f1, A, L);
  kstep<1, 4, false, true>(acc, f1, f0, A, L);
  kstep<2, 0, false, true>(acc, f0, f1, A, L);
  kstep<3, 0, false, false>(acc, f1, f0, A, L);
}

__device__ __forceinline__ u32 pack2(float a, float b) {
  f32x2 t = {a, b};
  return __builtin_bit_cast(u32, __builtin_convertvector(t, bf16x2));
}
__device__ __forceinline__ u32 relu2(u32 v) {
  typedef short s16x2 __attribute__((ext_vector_type(2)));
  return __builtin_bit_cast(u32, __builtin_elementwise_max(__builtin_bit_cast(s16x2, v), s16x2{0, 0}));
}
__device__ __forceinline__ float bf_lo(u32 v) { return __builtin_bit_cast(float, v << 16); }
__device__ __forceinline__ float bf_hi(u32 v) { return __builtin_bit_cast(float, v & 0xffff0000u); }

__device__ __forceinline__ void zero8k(u32 l16, u32 l16b, char* pb) {
  const u32x4 z4 = {0u, 0u, 0u, 0u};
  asm volatile("s_nop 4" ::: "memory");
  st16<0>(l16, z4, pb); st16<1024>(l16, z4, pb); st16<2048>(l16, z4, pb); st16<3072>(l16, z4, pb);
  st16<0>(l16b, z4, pb); st16<1024>(l16b, z4, pb); st16<2048>(l16b, z4, pb); st16<3072>(l16b, z4, pb);
}

__device__ __forceinline__ void tile(const Args& P, int tid, int wave, u32 lane, char* smem, u64 tstart) {
  const int r = P.rank;
  char* const myws = P.my;
  const int g = r >> 2, b = (r >> 1) & 1, a = r & 1;
  const int bid = blockIdx.x;

  const u32 cnt = bump_issue(CNT_OFF + (u32)(bid * 4 + wave) * 64u, myws);

  int row0, col0, nrem, tskip, j, e, k, rb;
  int pi0 = -1, pi1 = -1;
  bool fin;
  if (tid < N_MAIN_WG) {
    const int xcd = tid & 7, o = tid >> 3, u = xcd >> 1;
    int rbi, cbi;
    if (u < 2) { rbi = o < 24 ? 1 + o % 3 : 0; cbi = o < 24 ? o / 3 : o - 24; }
    else { rbi = o & 3; cbi = o >> 2; }
    rb = u + 4 * rbi; k = cbi * 2 + (xcd & 1);
    j = rb >> 1; e = rb & 1;
    row0 = (r ^ j) * 512 + b * 256 + e * 128; col0 = g * 2048 + (a ? 15 - k : k) * 128;
    nrem = j >= 2 ? 40 : 48; tskip = j >= 2 ? j - 2 : 99;
    fin = j == 0;
    if (j == 0) pi1 = 6 * NS + 15 - k;
    else if (j >= 2 && k >= 16 - NS) pi0 = ((j ^ 1) - 2) * NS + 15 - k;
  } else {
    const int f = tid - N_MAIN_WG;
    const int qd = 1 + f / 32, rem = f & 31;
    e = (rem >> 3) & 1; k = (rem & 7) * 2 + (rem >> 4);
    const int h = b ^ (qd & 1), g2 = g ^ (qd >> 1);
    row0 = r * 512 + h * 256 + e * 128; col0 = g2 * 2048 + (a ? 15 - k : k) * 128;
    nrem = 0; tskip = 99; j = 0; rb = 0;
    fin = true;
    const int iA = k >= NS ? 32 + (2 * qd - 2) * 16 + k : -1;
    const int iB = 15 - k >= NS ? 32 + (2 * qd - 1) * 16 + 15 - k : -1;
    pi0 = k < 8 ? iA : iB;
    pi1 = k < 8 ? iB : iA;
  }

  const u32 sbase = (u32)(uintptr_t)smem;
  const int wm = wave >> 1, wn = wave & 1;
  RdAddr A;
  {
    const u32 h = lane >> 5, m = lane & 31u, f = (lane >> 2) & 3u;
    A.xa[0] = sbase + ((u32)wm * 64u + m) * 64u + 16u * (h ^ f);
    A.xa[1] = sbase + ((u32)wm * 64u + m) * 64u + 16u * ((2u + h) ^ f);
    const u32 blk = (lane >> 4) & 1u, q = (lane & 15u) >> 2, pp = lane & 3u;
#pragma unroll
    for (int nt = 0; nt < 2; ++nt)
#pragma unroll
      for (int t = 0; t < 2; ++t) {
        const u32 row = 8u * h + 4u * t + q;
        const u32 ch = (u32)wn * 8u + (u32)nt * 4u + 2u * blk + (pp & 1u);
        A.wa[nt][t] = sbase + 256u * row + 16u * (ch ^ (((row & 3u) << 2) | ((row >> 2) & 3u))) + 8u * (pp >> 1);
      }
  }

  f32x16 acc[2][2];
#pragma unroll
  for (int i = 0; i < 2; ++i)
#pragma unroll
    for (int k = 0; k < 2; ++k)
#pragma unroll
      for (int z = 0; z < 16; ++z) acc[i][k][z] = 0.f;

  const u32 tile_off = (u32)e * 32768u + (u32)wave * 8192u;
  const u32 l16 = lane * 16u, l16b = lane * 16u + 4096u;
  const u32 tw = sbase + (u32)wave * 8192u + (lane & 31u) * 128u + (((lane >> 5) ^ (lane & 7u)) * 16u);
  const u32 tr = sbase + (u32)wave * 8192u + (lane >> 3) * 128u + (((lane & 7u) ^ (lane >> 3)) * 16u);
  const u32 oo = (lane >> 3) * 8192u + (lane & 7u) * 16u;
  char* const ob = uni(P.out + (size_t)((u32)((row0 & 511) + wm * 64) * 8192u + (u32)(col0 + wn * 64) * 2u));
  const int nph = nrem ? 2 : 1;
  const int qin = max(slot_w(k), slot_x(j, e, j == 7 ? 2 : 7));
  const u32 tin = (u32)(EARLY + TQ(qin));
  const int addph = fin ? nph - 1 : 0;

  if (LPT && (bid >> 3) == 25 && wave == 0) {
    sleep_until(tstart, (u32)LPT);
    if (lane < 28u) probe(lane * 2u * MIB + MIB + (u32)(bid & 7) * 128u, myws);
  }
  if (RPT && (bid >> 3) == RPB && wave == 1) {
    const u32 dd = 1u + lane % 7u;
    const u64 pp = ((const GLBP u64*)(uintptr_t)__builtin_amdgcn_kernarg_segment_ptr())[6u + 2u * dd];
    const u32 cl = bump_get<0>(cnt);
    const u32 o = (lane < 7u && dd > 1u ? (dd - 2u) * IN_SRC + 2u * MIB : PIN_OFF + (cl % 3u) * PIN_PAR + (dd > 1u ? dd * 16u : 30u) * PIECE) + (u32)(bid & 7) * 128u;
    sleep_until(tstart, (u32)RPT);
    if (lane < 14u) asm volatile("global_atomic_add %0, %1, off sc1" :: "v"(pp + o), "v"(0u) : "memory");
  }
  if (PD0 && (fin || k >= NS)) sleep_until(tstart, (u32)(PD0 + k * PD1 + (fin ? PDF : 0)));
  Loader L;
  L.m0base = sbase + (u32)wave * 4096u;
  bool dead = false;
  u32 dbgv = 0;
  u32 call = 0;
  for (int ph = 0; ph < nph; ++ph) {
    int n, ts;
    bool skip = false;
    if (ph == 0) {
      n = 16; ts = 99;
      if (wave < 2) {
        L.base = P.x + (size_t)((u32)(row0 + wave * 64) * 1024u);
        L.stride[0] = L.stride[1] = 64u;
        const u32 c = (lane & 3u) ^ ((lane >> 4) & 3u);
#pragma unroll
        for (int k = 0; k < 4; ++k) L.v[k] = (lane >> 2) * 1024u + c * 16u + (u32)k * (16384u - 1024u);
      } else {
        L.base = P.w + (size_t)((u32)(wave - 2) * 16u * 8192u + (u32)col0 * 2u);
        L.stride[0] = L.stride[1] = 32u * 8192u;
        const u32 q = lane >> 4;
#pragma unroll
        for (int k = 0; k < 4; ++k) L.v[k] = q * 8192u + (((lane & 15u) ^ ((q << 2) | (u32)k)) * 16u) + (u32)k * (4u * 8192u - 1024u);
      }
    } else {
      n = nrem; ts = tskip;
      STAMP(myws, 1);
      if (wave == 0) {
        const int sl = (int)min(lane, 5u);
        const u32 fo = fin_off(sl, max(slot_w(k), slot_x(j, e, sl + 2)));
        bool pend = lane < 6u && sl + 2 != j;
        sleep_until(tstart, tin);
        const u64 t0 = now();
        u32 st = 0;
        for (;;) {
          if (pend) pend = atom_rd(fo, myws) < 16u * call;
          if (__builtin_amdgcn_ballot_w64(pend) == 0ull) break;
          if (give_up(t0, myws, call)) {
            if (lane == 0) st_agt(myws + TMO_OFF, call);
            st = 1;
            break;
          }
          __builtin_amdgcn_s_sleep(PSL);
        }
        if (lane == 0) *(LDSP u32*)(uintptr_t)(sbase + 4u * STAGE_B) = st;
        __builtin_amdgcn_s_waitcnt(0);
      }
      asm volatile("s_barrier" ::: "memory");
      const bool wgdead = lds_rd(sbase + 4u * STAGE_B) != 0;
      STAMP(myws, 2);
      dead = dead || wgdead;
      skip = wgdead;
      const u32 par = call & 1u;
      static_assert(!XCT || XR == 32, "layout of the x pieces");
      const bool xc = XCT && wave < 2;
      L.base = myws + (size_t)(((u32)(wave < 2 ? cont_x(rb) : cont_w(k)) * 2u + par) * PIECE + (u32)(wave & 1) * (xc ? 8192u : 4096u));
      L.stride[0] = xc ? 2048u : 8192u;
      L.stride[1] = xc ? 14336u : 8192u;
#pragma unroll
      for (int k = 0; k < 4; ++k) L.v[k] = lane * 16u + (xc && k >= 2 ? 2048u : 0u);
    }
    if (!skip) gemm_phase(acc, A, L, n, ts, ph ? IN_SRC - PIECE : 0u, dbgv, ph * 8);
    if (ph == nph - 1) STAMP(myws, 3);
    if (ph == 0) {
      call = bump_get<0>(cnt);
      if (ZT0 && max(pi0, pi1) >= 0) {
        char* const pz = myws + (size_t)(PIN_OFF + ((call + 2u) % 3u) * PIN_PAR + tile_off);
        sleep_until(tstart, (u32)ZT0 + (u32)((tid >> 3) & 15) * (u32)ZDT);
        if (pi0 >= 0) zero8k(l16, l16b, uni(pz + (size_t)((u32)pi0 * PIECE)));
        if (pi1 >= 0) zero8k(l16, l16b, uni(pz + (size_t)((u32)pi1 * PIECE)));
      }
    }
    const u32 pin = PIN_OFF + (call % 3u) * PIN_PAR + tile_off;
#pragma unroll 1
    for (int s = 0; s < 2; ++s) {
      const int pi = ph != addph ? -1 : (s == 0 ? pi0 : pi1);
      if (pi < 0 && !(fin && ph == 0 && s == 0)) continue;
      u32x4 pv[8];
#pragma unroll
      for (int i = 0; i < 8; ++i) pv[i] = u32x4{0u, 0u, 0u, 0u};
      if (pi >= 0) {
        char* const pb = uni(myws + (size_t)(pin + (u32)pi * PIECE));
        const bool first = pi == 0 || (pi >= NOUT1 && (pi & 15) == NS);
        const u32 wo = pin + (u32)(pi - ((!WOWN && (lane & 2u) && !first) ? 1 : 0)) * PIECE + (lane & 1u) * 4096u + 4092u;
        sleep_until(tstart, pi >= NOUT1 ? (u32)(EARLY + TQ(NIN - 2 + (pi & 15) - NS)) : (fin ? 0u : tin - 300u));
        const u64 t0 = now();
        while (!dead) {
          u32 v = WOWN ? 1u : 0u;
          if (lane < (WOWN ? 2u : 4u)) v = atom_rd(wo, myws);
          if (WOWN ? __builtin_amdgcn_ballot_w64(v == 0u) == 0ull : __builtin_amdgcn_ballot_w64(v != 0u) != 0ull) break;
          if (give_up(t0, myws, call)) { dead = true; break; }
          __builtin_amdgcn_s_sleep(WSL);
        }
        if (s == 1) STAMP(myws, 4);
        for (;;) {
          if (__builtin_expect(dead, 0)) {
            if (lane == 0) st_agt(myws + TMO_OFF, call);
#pragma unroll
            for (int i = 0; i < 8; ++i) pv[i] = u32x4{0x7fc07fc0u, 0x7fc07fc0u, 0x7fc07fc0u, 0x7fc07fc0u};
            break;
          }
          asm volatile("s_nop 4\n\t"
                       "global_load_dwordx4 %0, %8, %10 sc0 sc1\n\t"
                       "global_load_dwordx4 %1, %8, %10 offset:1024 sc0 sc1\n\t"
                       "global_load_dwordx4 %2, %8, %10 offset:2048 sc0 sc1\n\t"
                       "global_load_dwordx4 %3, %8, %10 offset:3072 sc0 sc1\n\t"
                       "global_load_dwordx4 %4, %9, %10 sc0 sc1\n\t"
                       "global_load_dwordx4 %5, %9, %10 offset:1024 sc0 sc1\n\t"
                       "global_load_dwordx4 %6, %9, %10 offset:2048 sc0 sc1\n\t"
                       "global_load_dwordx4 %7, %9, %10 offset:3072 sc0 sc1"
                       : "=&v"(pv[0]), "=&v"(pv[1]), "=&v"(pv[2]), "=&v"(pv[3]), "=&v"(pv[4]), "=&v"(pv[5]), "=&v"(pv[6]), "=&v"(pv[7])
                       : "v"(l16), "v"(l16b), "s"(pb) : "memory");
          const bool late = __builtin_amdgcn_readfirstlane((int)(now() - t0 > 50000ull)) != 0;
          asm volatile("s_waitcnt vmcnt(0)"
                       : "+v"(pv[0]), "+v"(pv[1]), "+v"(pv[2]), "+v"(pv[3]), "+v"(pv[4]), "+v"(pv[5]), "+v"(pv[6]), "+v"(pv[7]) :: "memory");
          u32 mn = 0xffffffffu;
#pragma unroll
          for (int i = 0; i < 8; ++i)
#pragma unroll
            for (int z = 0; z < 4; ++z) mn = min(mn, pv[i][z]);
          if (__builtin_amdgcn_ballot_w64(mn == 0u) == 0ull) break;
          if (__builtin_expect(late, 0)) dead = give_up(t0, myws, call);
        }
      }
#pragma unroll
      for (int nt = 0; nt < 2; ++nt)
#pragma unroll
        for (int mt = 0; mt < 2; ++mt)
#pragma unroll
          for (int sx = 0; sx < 2; ++sx)
#pragma unroll
            for (int z = 0; z < 4; ++z) {
              const u32 u = pv[(nt * 2 + mt) * 2 + sx][z];
              acc[nt][mt][8 * sx + 2 * z] += bf_lo(u);
              acc[nt][mt][8 * sx + 2 * z + 1] += bf_hi(u);
            }
      if (!fin) continue;
      if (s == 1) STAMP(myws, 6);
#pragma unroll
      for (int nt = 0; nt < 2; ++nt)
#pragma unroll
        for (int mt = 0; mt < 2; ++mt)
#pragma unroll
          for (int sx = 0; sx < 2; ++sx) {
            u32x4 v;
#pragma unroll
            for (int z = 0; z < 4; ++z) v[z] = relu2(pack2(acc[nt][mt][8 * sx + 2 * z], acc[nt][mt][8 * sx + 2 * z + 1]));
            *(volatile LDSP u32x4*)(uintptr_t)((tw ^ (u32)((nt * 4 + sx * 2) * 16)) + (u32)mt * 4096u) = v;
          }
      u32 o1, o2, o3, o4, o5, o6, o7;
      asm volatile("ds_read_b128 %0, %15\n\t"
                   "ds_read_b128 %1, %15 offset:1024\n\t"
                   "ds_read_b128 %2, %15 offset:2048\n\t"
                   "ds_read_b128 %3, %15 offset:3072\n\t"
                   "ds_read_b128 %4, %15 offset:4096\n\t"
                   "ds_read_b128 %5, %15 offset:5120\n\t"
                   "ds_read_b128 %6, %15 offset:6144\n\t"
                   "ds_read_b128 %7, %15 offset:7168\n\t"
                   "v_add_u32 %8, 0x10000, %16\n\t"
                   "v_add_u32 %9, 0x20000, %16\n\t"
                   "v_add_u32 %10, 0x30000, %16\n\t"
                   "v_add_u32 %11, 0x40000, %16\n\t"
                   "v_add_u32 %12, 0x50000, %16\n\t"
                   "v_add_u32 %13, 0x60000, %16\n\t"
                   "v_add_u32 %14, 0x70000, %16\n\t"
                   "s_mov_b64 exec, %18\n\t"
                   "s_waitcnt lgkmcnt(7)\n\tglobal_store_dwordx4 %16, %0, %17 sc0 sc1\n\t"
                   "s_waitcnt lgkmcnt(6)\n\tglobal_store_dwordx4 %8, %1, %17 sc0 sc1\n\t"
                   "s_waitcnt lgkmcnt(5)\n\tglobal_store_dwordx4 %9, %2, %17 sc0 sc1\n\t"
                   "s_waitcnt lgkmcnt(4)\n\tglobal_store_dwordx4 %10, %3, %17 sc0 sc1\n\t"
                   "s_waitcnt lgkmcnt(3)\n\tglobal_store_dwordx4 %11, %4, %17 sc0 sc1\n\t"
                   "s_waitcnt lgkmcnt(2)\n\tglobal_store_dwordx4 %12, %5, %17 sc0 sc1\n\t"
                   "s_waitcnt lgkmcnt(1)\n\tglobal_store_dwordx4 %13, %6, %17 sc0 sc1\n\t"
                   "s_waitcnt lgkmcnt(0)\n\tglobal_store_dwordx4 %14, %7, %17 sc0 sc1\n\t"
                   "s_cmp_lg_u64 %18, -1\n\t"
                   "s_cbranch_scc1 1f\n\t"
                   "s_endpgm\n"
                   "1:\n\t"
                   "s_mov_b64 exec, -1\n\t"
                   "s_nop 1"
                   : "=&v"(pv[0]), "=&v"(pv[1]), "=&v"(pv[2]), "=&v"(pv[3]), "=&v"(pv[4]), "=&v"(pv[5]), "=&v"(pv[6]), "=&v"(pv[7]),
                     "=&v"(o1), "=&v"(o2), "=&v"(o3), "=&v"(o4), "=&v"(o5), "=&v"(o6), "=&v"(o7)
                   : "v"(tr), "v"(oo), "s"(ob), "s"(s == 1 ? ~0ull : (u64)WMK) : "memory", "scc");
      if (s == 0) waitv<0>();
    }
  }

  if (!fin) {
    STAMP(myws, 6);
    const bool dir = DIR && j >= 2 && k < NS;
    char* dst = uni(dir ? P.peer + (size_t)(PIN_OFF + (call % 3u) * PIN_PAR + (u32)((j - 2) * NS + k) * PIECE + tile_off)
                        : myws + (size_t)(POUT_OFF + (u32)((j - 1) * 16 + k) * PIECE + tile_off));
    char* fl = myws + (size_t)(FLOC_OFF + (u32)((j - 1) * 8 + e * 4 + wave) * 128u + (u32)k * 4u);
    u32 least = 1u;
    if (__builtin_expect(dead, 0)) {
      least = 0x7fc07fc0u;
#pragma unroll
      for (int i = 0; i < 64; ++i) acc[i >> 5][(i >> 4) & 1][i & 15] = 0.f;
    }
    asm volatile("s_nop 4" ::: "memory");
#pragma unroll
    for (int nt = 0; nt < 2; ++nt)
#pragma unroll
      for (int mt = 0; mt < 2; ++mt) {
        u32x4 v0, v1;
#pragma unroll
        for (int z = 0; z < 4; ++z) {
          v0[z] = max(pack2(acc[nt][mt][2 * z], acc[nt][mt][2 * z + 1]), least);
          v1[z] = max(pack2(acc[nt][mt][8 + 2 * z], acc[nt][mt][8 + 2 * z + 1]), least);
        }
        const u32 o = nt ? l16b : l16;
        if (mt == 0) st16<0>(o, v0, dst); else st16<2048>(o, v0, dst);
        if (dir) __builtin_amdgcn_s_sleep(DSL);
        if (mt == 0) st16<1024>(o, v1, dst); else st16<3072>(o, v1, dst);
        if (dir) __builtin_amdgcn_s_sleep(DSL);
      }
    waitv<0>();
    if (lane == 0) st_sys(fl, call);
  }
  STAMP(myws, 5);
  STAMPV(myws, 0, tstart);
}

__global__ __launch_bounds__(256, 2) void fused_gemm_rs(Params P) {
  extern __shared__ __attribute__((aligned(16))) char smem[];
  const int bid = blockIdx.x;
  typedef u64 u64x4 __attribute__((ext_vector_type(4)));
  u64x4 a4;
  u64 tstart, pe;
  int rk, d;
  u32 tis, t0;
  u32x16 f16;
  u32x2 co;
  static_assert(N_PUSH_WG == 56 && PROT == 0 && PWN == 1 && CPB >= N_PUSH_WG, "entry code");
  asm volatile("v_readfirstlane_b32 s46, %[tid]\n\t"
               "s_load_dwordx8 s[36:43], %[ka], 0x0\n\t"
               "s_lshr_b32 s47, %[bid], 3\n\t"
               "s_cmp_gt_u32 %[bid], 55\n\t"
               "s_cbranch_scc1 2f\n\t"
               "s_lshr_b32 s48, s46, 7\n\t"
               "s_mul_i32 s49, s48, 3\n\t"
               "s_add_i32 s47, s47, s49\n\t"
               "s_add_i32 s49, s47, -7\n\t"
               "s_min_u32 s47, s47, s49\n\t"
               "s_lshl_b32 s49, s47, 4\n\t"
               "s_load_dwordx4 s[52:55], %[ka], s49 offset:0x40\n\t"
               "s_memrealtime s[44:45]\n\t"
               "s_load_dword s62, %[ka], 0x20\n\t"
               "s_setprio 3\n\t"
               "s_add_i32 s63, s47, 1\n\t"
               "v_lshlrev_b32 v1, 2, %[tid]\n\t"
               "v_mov_b32 v27, 0\n\t"
               "v_add_u32 v1, %[ctl], v1\n\t"
               "s_bitcmp1_b32 s46, 6\n\t"
               "ds_write_b32 v1, v27\n\t"
               "s_cbranch_scc0 1f\n\t"
               "s_and_b32 s49, %[bid], 7\n\t"
               "s_lshl1_add_u32 s48, s49, s48\n\t"
               "v_bfe_u32 v1, %[tid], 4, 2\n\t"
               "v_and_b32 v2, 15, %[tid]\n\t"
               "v_lshlrev_b32 v3, 2, v1\n\t"
               "v_lshlrev_b32 v1, 13, v1\n\t"
               "v_xor_b32 v2, v3, v2\n\t"
               "v_lshl_or_b32 v1, s48, 17, v1\n\t"
               "v_lshl_or_b32 v20, v2, 4, v1\n\t"
               "v_xor_b32 v21, 16, v20\n\t"
               "v_xor_b32 v22, 32, v20\n\t"
               "v_xor_b32 v23, 48, v20\n\t"
               "v_add_u32 v21, 0x7c00, v21\n\t"
               "v_add_u32 v22, 0xf800, v22\n\t"
               "v_add_u32 v23, 0x17400, v23\n\t"
               "s_lshl_b32 s49, s63, 7\n\t"
               "s_add_i32 s49, s49, %[lnk]\n\t"
               "v_mov_b32 v28, s49\n\t"
               "v_mov_b32 v25, 1\n\t"
               "s_waitcnt lgkmcnt(0)\n\t"
               "s_memrealtime s[58:59]\n\t"
               "global_load_dwordx4 v[144:147], v20, s[54:55] " MLP "\n\t"
               "global_load_dwordx4 v[148:151], v21, s[54:55] offset:1024 " MLP "\n\t"
               "global_load_dwordx4 v[152:155], v22, s[54:55] offset:2048 " MLP "\n\t"
               "global_load_dwordx4 v[156:159], v23, s[54:55] offset:3072 " MLP "\n\t"
               "s_mov_b64 exec, 1\n\t"
               "global_atomic_add v160, v28, v25, s[40:41] sc0 sc1\n\t"
               "s_add_i32 s50, s47, -1\n\t"
               "s_max_i32 s50, s50, 0\n\t"
               "s_mul_i32 s51, s50, %[fsl]\n\t"
               "s_lshl_b32 s49, s48, 7\n\t"
               "s_add_i32 s51, s51, s49\n\t"
               "s_add_i32 s51, s51, %[f0]\n\t"
               "v_mov_b32 v29, s51\n\t"
               "s_waitcnt lgkmcnt(0)\n\t"
               "s_or_b32 s66, s58, 1\n\t"
               "v_mov_b32 v26, s66\n\t"
               "global_atomic_cmpswap v161, v28, v[26:27], s[40:41] offset:4 sc0 sc1\n\t"
               "global_atomic_add v29, v27, s[52:53] sc1\n\t"
               "s_mov_b64 exec, -1\n"
               "1:\n\t"
               "s_waitcnt lgkmcnt(0)\n\t"
               "s_barrier\n\t"
               "s_branch 5f\n"
               "2:\n\t"
               "s_memrealtime s[44:45]\n\t"
               "s_load_dword s62, %[ka], 0x20\n\t"
               "s_load_dwordx2 s[52:53], %[ka], 0x40\n\t"
               "s_mov_b32 s63, 1\n\t"
               "s_add_i32 s47, %[bid], %[cpb]\n\t"
               "s_cmp_lt_u32 s47, 16\n\t"
               "s_cbranch_scc0 5f\n\t"
               "s_cmpk_lt_u32 s46, 0xc0\n\t"
               "s_cbranch_scc0 5f\n\t"
               "s_getpc_b64 s[56:57]\n\t"
               "s_mov_b32 m0, %[cpl]\n\t"
               "v_lshlrev_b32 v1, 7, %[tid]\n\t"
               "v_min_u32 v1, %[cpf], v1\n\t"
               "global_load_lds_dword v1, s[56:57]\n"
               "5:\n\t"
               "s_waitcnt lgkmcnt(0)"
               : "={s[36:43]}"(a4), "={s[44:45]}"(tstart), "={s46}"(t0), "={s[52:53]}"(pe), "={s62}"(rk), "={s63}"(d),
                 "={s66}"(tis)
               : [ka] "s"(__builtin_amdgcn_kernarg_segment_ptr()), [bid] "s"(bid), [tid] "v"(threadIdx.x),
                 [ctl] "i"(65536), [lnk] "i"(LNK_OFF), [fsl] "i"(IN_SRC + 4u * PIECE), [f0] "i"(NS * 2u * PIECE + 4096u),
                 [cpb] "i"(-CPB), [cpl] "i"(66048), [cpf] "i"(CPF - 128)
               : "memory", "scc", "s47", "s48", "s49", "s50", "s51", "s54", "s55", "s56", "s57", "s58", "s59",
                 "v1", "v2", "v3", "v20", "v21", "v22", "v23", "v25", "v26", "v27", "v28", "v29");
  asm volatile("" : "={v[144:159]}"(f16), "={v[160:161]}"(co));
  const u32 lane = threadIdx.x & 63u;
  const int wave = (int)(t0 >> 6);
  const Args A = {(const char*)a4[0], (const char*)a4[1], (char*)a4[2], (char*)a4[3], (char*)pe, rk, tis, f16, co};
  if (bid < N_PUSH_WG) mover(A, d, wave, lane, smem, tstart);
  else if (bid >= 256 && bid < 256 + N_PUSH_WG) {
    for (int i = 0; i < 6; ++i) __builtin_amdgcn_s_sleep(16);
  } else tile(A, bid < 256 - ROT ? bid - N_PUSH_WG + ROT : (bid < 256 ? bid - (256 - ROT) : bid - 2 * N_PUSH_WG), wave, lane, smem, tstart);
}

}

extern "C" void kernel_launch(int rank, int nranks, ncclComm_t comm,
                              hipStream_t stream,
                              void* const* d_in, const int* in_sizes, int n_in,
                              void* d_out, int out_size,
                              const int* in_dtypes, int out_dtype,
                              void* const* d_out_all,
                              void* const* d_ws_all, size_t ws_bytes) {
  if (nranks != NR || n_in < 2 || ws_bytes < (size_t)WS_NEED || (unsigned)rank >= (unsigned)NR) {
    fprintf(stderr, "unsupported configuration\n");
    return;
  }
  Params P;
  P.x = (const char*)d_in[0];
  P.w = (const char*)d_in[1];
  P.out = (char*)d_out;
  P.my = (char*)d_ws_all[rank];
  for (int i = 0; i < NR; ++i) {
    const int r = rank ^ i;
    P.lk[i].peer = (char*)d_ws_all[r];
    P.lk[i].w0 = P.w + (size_t)((r & 1) * (256 * 8192 + 15 * 256) + (r >> 2) * 4096);
  }
  P.rank = rank;
  P.pad[0] = P.pad[1] = P.pad[2] = 0;
  HIP_CHECK(hipFuncSetAttribute((const void*)fused_gemm_rs, hipFuncAttributeMaxDynamicSharedMemorySize, (int)LDS_BYTES));
  hipLaunchKernelGGL(fused_gemm_rs, dim3(N_WG), dim3(256), LDS_BYTES, stream, P);
  HIP_CHECK(hipGetLastError());
}
```

```cpp
#include <hip/hip_runtime.h>
#include <rccl/rccl.h>
#include <cstdio>
#include <cstdint>

#define HIP_CHECK(expr) do { hipError_t _e = (expr); if (_e != hipSuccess) { \
    fprintf(stderr, "HIP error %s at %s:%d\n", hipGetErrorString(_e), __FILE__, __LINE__); \
    return; } } while (0)

namespace {

typedef unsigned u32;
typedef unsigned long long u64;
typedef u32 u32x4 __attribute__((ext_vector_type(4)));
typedef u32 u32x2 __attribute__((ext_vector_type(2)));
typedef u32 u32x16 __attribute__((ext_vector_type(16)));
typedef float f32x2 __attribute__((ext_vector_type(2)));
typedef float f32x16 __attribute__((ext_vector_type(16)));
typedef __bf16 bf16x2 __attribute__((ext_vector_type(2)));
typedef __bf16 bf16x8 __attribute__((ext_vector_type(8)));
typedef short s16x4 __attribute__((ext_vector_type(4)));
typedef short s16x8 __attribute__((ext_vector_type(8)));
#define LDSP __attribute__((address_space(3)))
#define GLBP __attribute__((address_space(1)))

constexpr int NR = 8;
constexpr u32 PIECE = 65536u;
constexpr u32 MIB = 1u << 20;
#define NS 2
#define NX 4
#define PF 0
#define SUB 1
#define OX 14
#define TAU 112
#define T0 20
#define QA 0
#define DA 0
#define QB 20
#define DB 1
#define QC 99
#define DC 0
#define TQ(q) (T0 + TAU * (q) - DA * ((q) < QA ? (q) : QA) - DB * ((q) > QB ? (q) - QB : 0))
#define EARLY 150
#define TAU1 111
#define TAU2 0
#define STGD 2
#define PSL 4
#define LSL 0
#define WSL 3
#define RSL 0
#define WOWN 0
#define ALN 1
#define ALO 99
#define ALT 25
#define PWN 1
#define DIR 0
#define DSL 4
#define SWP 14
#define CPF 15488
#define CPB 56
#define PD0 400
#define PD1 1400
#define PDL 104
#define WE 1
#define PJ1 1
#define OVP 1
#define SPR 2
#define PDF 0
#define GSH -18
#define ORG 1
#define OC 34
#define OG 10
#define OSP 7
#define ORD 1
#define OD 20
#define CLT 84
#define CKT 44
#define CLD 68
#define LPT 400
#define ROT 48
#define PWIN 64
#define PWHO 1
#define PQ1 9
#define PQ2 14
#define PROT 0
#define XR 32
#define ZT0 900
#define ZDT 100
#define RPT 400
#define RPB 25
#define XCT 0
#define L1T 500
#define SXT 6
#define PRE 1
#define WMK 1ull
#define G1 0
#define G1M 0
#define G1Q 7
#define PFD 0
#define LAG 1
#define WIN 3
#define GLP "sc0 sc1"
#define MLP "sc0 sc1"
constexpr int NIN = 30;
constexpr int NOUT = 16 - NS;
constexpr int NOUT1 = 16 + 6 * NS;
constexpr int NXT = (12 * NX + 15) / 16;
constexpr int NXL = 12 * NX - 16 * (NXT - 1);
static_assert(NOUT1 + NXT <= 32, "receive buffer of link 1");
constexpr int XLA = NX ? (TAU1 * (16 - NXL) + 8 * (14 + NXT)) / (16 * (14 + NXT)) : 0;
constexpr int XLB = NX ? (TAU1 * NXL + 8 * (13 + NXT)) / (16 * (13 + NXT)) : 0;
constexpr u32 IN_SRC = 4u * MIB;
constexpr u32 PIN_OFF = 24u * MIB;
constexpr u32 PIN_PAR = 8u * MIB;
constexpr u32 POUT_OFF = 48u * MIB;
constexpr u32 FLG_OFF = 55u * MIB;
constexpr u32 FLOC_OFF = FLG_OFF + 16384u;
constexpr u32 TMO_OFF = FLG_OFF + 49152u;
constexpr u32 LNK_OFF = FLG_OFF + MIB + 262144u + 131072u;
constexpr u32 DUMMY_OFF = FLG_OFF + 65536u;
constexpr u32 DBG_OFF = FLG_OFF + 131072u;
constexpr u32 CNT_OFF = FLG_OFF + MIB + 262144u;
constexpr u32 WS_NEED = FLG_OFF + 2u * MIB;

constexpr int N_PUSH_WG = 56;
constexpr int N_MAIN_WG = 256;
constexpr int N_FIN_WG = 96;
constexpr int N_WG = 2 * N_PUSH_WG + N_MAIN_WG + N_FIN_WG;
constexpr u32 STAGE_B = 16384u;
constexpr u32 LDS_BYTES = 4u * STAGE_B + 1024u;
constexpr u64 DEADLINE = 100000000ull;
constexpr u32 DEAD = 0x7fffffffu;

struct Params {
  const char* x;
  const char* w;
  char* my;
  char* out;
  int rank;
  int pad[3];
  struct { char* peer; const char* w0; } lk[NR];
};
struct Args { const char* x; const char* w; char* my; char* out; char* peer; int rank; u32 tis; u32x16 f16; u32x2 co; };

#define SB() __builtin_amdgcn_sched_barrier(0)
template <int N> __device__ __forceinline__ void waitv() { asm volatile("s_waitcnt vmcnt(%0)" :: "i"(N) : "memory"); }

__device__ __forceinline__ u64 now() { return __builtin_amdgcn_s_memrealtime(); }
template <typename T> __device__ __forceinline__ T* uni(T* p) {
  const u64 v = (u64)(uintptr_t)p;
  const u32 lo = __builtin_amdgcn_readfirstlane((u32)v), hi = __builtin_amdgcn_readfirstlane((u32)(v >> 32));
  return (T*)(uintptr_t)(((u64)hi << 32) | lo);
}
__device__ __forceinline__ u32 ld_sys(const void* p) {
  return __hip_atomic_load((const GLBP u32*)(uintptr_t)p, __ATOMIC_RELAXED, __HIP_MEMORY_SCOPE_SYSTEM);
}
__device__ __forceinline__ void st_sys(void* p, u32 v) {
  __hip_atomic_store((GLBP u32*)(uintptr_t)p, v, __ATOMIC_RELAXED, __HIP_MEMORY_SCOPE_SYSTEM);
}
__device__ __forceinline__ void st_agt(void* p, u32 v) {
  __hip_atomic_store((GLBP u32*)(uintptr_t)p, v, __ATOMIC_RELAXED, __HIP_MEMORY_SCOPE_AGENT);
}
__device__ __forceinline__ bool give_up(u64 t0, const char* ws, u32 call) {
  const u64 dt = now() - t0;
  if (!__builtin_amdgcn_readfirstlane((int)(dt > 50000ull))) return false;
  __builtin_amdgcn_s_sleep(64);
  if (__builtin_amdgcn_readfirstlane((int)(dt > DEADLINE))) return true;
  return __builtin_amdgcn_readfirstlane(ld_sys(ws + TMO_OFF)) == call;
}
__device__ __forceinline__ void sleep_until(u64 tstart, u32 t) {
  for (;;) {
    const u32 el = (u32)__builtin_amdgcn_readfirstlane((u32)(now() - tstart));
    if (el >= t) break;
    if (t - el > 40u) __builtin_amdgcn_s_sleep(12); else __builtin_amdgcn_s_sleep(1);
  }
}
__device__ __forceinline__ void sleep_exact(u64 tstart, u32 t) {
  for (;;) {
    const u32 el = (u32)__builtin_amdgcn_readfirstlane((u32)(now() - tstart));
    if (el >= t) break;
    if (t - el > 60u) __builtin_amdgcn_s_sleep(12);
    else if (t - el > 20u) __builtin_amdgcn_s_sleep(3);
    else if (t - el > (u32)SXT) __builtin_amdgcn_s_sleep(1);
  }
}
template <int IMM> __device__ __forceinline__ void st16(u32 off, u32x4 v, char* base) {
  asm volatile("global_store_dwordx4 %0, %1, %2 offset:%3 sc0 sc1\n\ts_nop 1" :: "v"(off), "v"(v), "s"(base), "i"(IMM) : "memory");
}
template <int IMM> __device__ __forceinline__ void st16p(u32 off, u32x4 v, char* base) {
  asm volatile("global_store_dwordx4 %0, %1, %2 offset:%3\n\ts_nop 1" :: "v"(off), "v"(v), "s"(base), "i"(IMM) : "memory");
}
#define OST st16
__device__ __forceinline__ void dma16x4(u32 lds, const u32 (&o)[4], const char* base) {
  asm volatile("s_mov_b32 m0, %0\n\ts_nop 3\n\t"
               "global_load_lds_dwordx4 %1, %5 " MLP "\n\t"
               "global_load_lds_dwordx4 %2, %5 offset:1024 " MLP "\n\t"
               "global_load_lds_dwordx4 %3, %5 offset:2048 " MLP "\n\t"
               "global_load_lds_dwordx4 %4, %5 offset:3072 " MLP
               :: "s"(lds), "v"(o[0]), "v"(o[1]), "v"(o[2]), "v"(o[3]), "s"(base) : "memory");
}
__device__ __forceinline__ void probe(u32 off, char* base) {
  asm volatile("s_nop 4\n\tglobal_atomic_add %0, %1, %2 sc1" :: "v"(off), "v"(0u), "s"(base) : "memory");
}
__device__ __forceinline__ void probe0(u32 off, char* base) {
  asm volatile("s_mov_b64 exec, 1\n\ts_nop 3\n\tglobal_atomic_add %0, %1, %2 sc1\n\ts_mov_b64 exec, -1"
               :: "v"(off), "v"(0u), "s"(base) : "memory");
}
__device__ __forceinline__ u32 atom_rd(u32 off, const char* base) {
  u32 r;
  asm volatile("global_atomic_add %0, %1, %2, %3 sc0 sc1\n\ts_waitcnt vmcnt(0)" : "=&v"(r) : "v"(off), "v"(0u), "s"(base) : "memory");
  return r;
}
__device__ __forceinline__ u32 bump_issue(u32 off, const char* base) {
  u32 r;
  asm volatile("s_mov_b64 exec, 1\n\ts_nop 3\n\tglobal_atomic_add %0, %1, %2, %3 sc0 sc1\n\ts_mov_b64 exec, -1"
               : "=&v"(r) : "v"(off), "v"(1u), "s"(base) : "memory");
  return r;
}
template <int N> __device__ __forceinline__ u32 bump_get(u32 r) {
  u32 v;
  asm volatile("s_waitcnt vmcnt(%2)\n\tv_readfirstlane_b32 %0, %1\n\ts_nop 4" : "=s"(v) : "v"(r), "i"(N) : "memory");
  return v + 1u;
}
__device__ __forceinline__ void add1_lane0(u32 off, char* base) {
  asm volatile("s_mov_b64 exec, 1\n\ts_nop 3\n\tglobal_atomic_add %0, %1, %2 sc1\n\ts_mov_b64 exec, -1"
               :: "v"(off), "v"(1u), "s"(base) : "memory");
}
__device__ __forceinline__ void st4_lane0(u32 off, u32 val, char* base) {
  asm volatile("s_mov_b64 exec, 1\n\ts_nop 3\n\tglobal_store_dword %0, %1, %2 sc0 sc1\n\ts_mov_b64 exec, -1"
               :: "v"(off), "v"(val), "s"(base) : "memory");
}

#define STAMP(ws, idx) do {} while (0)
#define STAMPV(ws, idx, v) do {} while (0)

__device__ __forceinline__ void piece_of(int q, int d, bool& isx, int& idx) {
  if (WE) q = q == NS + 10 ? NS + 12 : ((u32)(q - NS - 11) < 2u ? q - 1 : q);
  const int t = q - NS, u = q - NS - 12;
  int j = (t >> 1) + 2;
  j += j >= d ? 1 : 0;
  j = j == 8 ? 1 : j;
  isx = (q >= NS && u < 0) || (u >= OX && u < OX + 2);
  idx = q < NS ? q : (u < 0 ? 2 * j + (t & 1) : (u < OX ? NS + u : (u < OX + 2 ? u - OX : NS + u - 2)));
}
__device__ __forceinline__ int slot_w(int k) { return k < NS ? k : (WE && k == NS ? 10 + k : (k - NS < OX ? 12 + k : 14 + k)); }
__device__ __forceinline__ int slot_x(int j, int e, int d) {
  return j == 0 ? NS + 12 + OX + e : (j == 1 ? NS + 10 + WE : NS + 2 * (j - 2 - (j > d ? 1 : 0))) + e;
}
__device__ __forceinline__ int cont_w(int k) { return k < NS ? k : 14 + k; }
__device__ __forceinline__ int cont_x(int rb) { return rb < 2 ? 30 + rb : (rb < 4 ? NS + 10 + rb : NS - 4 + rb); }
__device__ __forceinline__ u32 fin_off(int sl, int q) { return (u32)sl * IN_SRC + (u32)(NS + 2 * sl) * 2u * PIECE + (u32)q * 64u; }
__device__ __forceinline__ u32 lds_rd(u32 a) { return __builtin_amdgcn_readfirstlane(*(volatile LDSP u32*)(uintptr_t)a); }
__device__ __forceinline__ void lds_wr(u32 a, u32 v) { *(volatile LDSP u32*)(uintptr_t)a = v; }

__device__ __forceinline__ void mover(const Args& P, int d, int wave, u32 lane, char* smem, u64 tstart) {
  const int s = P.rank;
  char* const myws = P.my;
  const int pair = wave >> 1;
  const int p = ((blockIdx.x + PROT) & 7) * 2 + pair;
  const u32 r = (u32)(s ^ d) & (u32)(NR - 1);
  const int nin = d == 1 ? 0 : NIN;
  const bool less = NX && (d == 1 ? p >= NXL : p < 2 * NX);
  const int nid = d == 1 ? NOUT1 + NXT : NOUT;
  const int np = nin + nid - (less ? 1 : 0);
  const u32 ring = (u32)(uintptr_t)smem + (u32)pair * 32768u;
  const u32 ctl = (u32)(uintptr_t)smem + 65536u + (u32)pair * 512u;
  const char* const xin = P.x;
  const char* const win = P.w;
  const int rg = r >> 2, rb_ = (r >> 1) & 1, ra = r & 1;
  u32 wo[4];
#pragma unroll
  for (int k = 0; k < 4; ++k) {
    const u32 q4 = lane >> 4;
    wo[k] = ((u32)p * 16u + (u32)k * 4u + q4) * 8192u + (((lane & 15u) ^ ((q4 << 2) | (u32)k)) * 16u) - (u32)k * 1024u;
  }
  constexpr u32 XC = 256u / XR;
  const u32 xrow = ((u32)p % (128u / XR)) * XR, xks = (u32)p / (128u / XR) * (XC / 4u);
#define W_BASE(IX) (win + (size_t)((u32)ra * 256u * 8192u + (u32)rg * 4096u + (u32)(ra ? 15 - (IX) : (IX)) * 256u))

  if ((wave & 1) == 0) {
    const u32 cnt = bump_issue(CNT_OFF + (u32)(blockIdx.x * 4 + wave) * 64u, myws);
    u32 xo[4], oo[4];
    {
#pragma unroll
      for (int k = 0; k < 4; ++k) {
        const u32 c = (u32)k * 64u + lane;
        xo[k] = (xrow + c / XC) * 1024u + xks * 64u + (c % XC) * 16u - (u32)k * 1024u;
        oo[k] = lane * 16u;
      }
    }
    u32 poll_off, src_off = 0, dst_off;
#define ISSUE_LOADS(QQ, II) do { \
      const int q_ = (QQ); \
      const u32 dst_ = ring + (u32)(q_ & 7) * 4096u; \
      if (q_ < nin) { \
        bool isx_; int ix_; \
        piece_of(q_, d, isx_, ix_); \
        if (isx_) dma16x4(dst_, xo, xin + (size_t)(((r ^ (u32)(ix_ >> 1)) * 512u + (u32)rb_ * 256u + (u32)(ix_ & 1) * 128u) * 1024u + (u32)ra * 512u)); \
        else      dma16x4(dst_, wo, W_BASE(ix_)); \
      } else { \
        dma16x4(dst_, oo, myws + (size_t)(u32)__builtin_amdgcn_readlane((int)src_off, (II))); \
      } \
    } while (0)

    int q0 = 0;
    u32 pub = 0, call;
    u32 dbg1 = 0, dbg2 = 0;
    if (nin) {
      ISSUE_LOADS(1, 0); ISSUE_LOADS(2, 0); ISSUE_LOADS(3, 0);
    }
    if (nin) {
      call = bump_get<8>(cnt);
      lds_wr(ctl, 2u);
      q0 = 4; pub = 2;
    } else {
      call = bump_get<0>(cnt);
    }

    {
      u32 ln = lane;
      asm volatile("" : "+v"(ln));
      const u32 i0 = ln < (u32)nid ? ln : 0u, up = (u32)p;
      const u32 nsw = d == 1 ? SWP : 0, nex = d == 1 ? NXT : 0;
      const bool sw = i0 < nsw, ex = !sw && i0 < nsw + nex;
      const u32 z = min((i0 - nsw) * 16u + up, (u32)(12 * NX - 1)), zj = z / (u32)(NX ? 2 * NX : 1);
      const u32 i = sw ? (i0 & ~1u) | (up >> 3) : (ex ? i0 + (u32)(NOUT1 - SWP) : i0 - nex);
      const u32 pd = sw ? ((i0 & 1u) << 3) | (up & 7u) : up;
      const u32 ps = ex ? z - zj * (u32)(2 * NX) : pd;
      const u32 jk = d != 1 ? (u32)((d - 1) * 16 + NS) + i
                            : (ex ? zj * 16u + (u32)(16 + NS) : (i < 6u * NS ? 16u + i / NS * 16u + i % NS : i - 6u * NS));
      poll_off = FLOC_OFF + ((jk >> 4) * 8u + (ps >> 1)) * 128u + (jk & 15u) * 4u;
      src_off = POUT_OFF + jk * PIECE + ps * 4096u;
      dst_off = i * PIECE + pd * 4096u;
    }
    if (!nin) sleep_until(tstart, (u32)(TQ(NS + 2) + L1T));
    u64 ready = 0;
    u64 todo = ((1ull << nid) - 1ull) & ~(less ? 1ull << (d == 1 ? SWP + NXT - 1 : 0) : 0ull);
    u32 cons = 0;
    bool dead = false;
    for (int q = q0; q < np; ++q) {
      bool rdy = q < nin || (ready & todo) != 0ull;
      if (cons + 8u <= (u32)q) cons = lds_rd(ctl + 4u);
      if (!rdy || cons + 8u <= (u32)q) {
        if (!OVP || rdy) waitv<0>();
        pub = (u32)q;
        if (!OVP || rdy) lds_wr(ctl, pub);
        const u64 t0 = now();
        for (;;) {
          if (!rdy) {
            u32 pv = 0;
            if (OVP) {
              asm volatile("s_mov_b64 exec, %4\n\ts_nop 3\n\t"
                           "global_atomic_add %0, %1, %2, %3 sc0 sc1\n\t"
                           "s_mov_b64 exec, -1\n\t"
                           "s_waitcnt vmcnt(1)\n\t"
                           "ds_write_b32 %5, %6\n\t"
                           "s_waitcnt vmcnt(0)"
                           : "+&v"(pv) : "v"(poll_off), "v"(0u), "s"(myws), "s"(todo), "v"(ctl), "v"(pub) : "memory");
            } else if ((todo >> lane) & 1ull) pv = atom_rd(poll_off, myws);
            ready |= __builtin_amdgcn_ballot_w64(pv >= call);
            rdy = (ready & todo) != 0ull;
            if (LSL && !rdy) __builtin_amdgcn_s_sleep(LSL);
          } else {
            __builtin_amdgcn_s_sleep(2);
          }
          cons = lds_rd(ctl + 4u);
          if (rdy && cons + 8u > (u32)q) break;
          if (give_up(t0, myws, call)) { dead = true; break; }
        }
        if (dead) break;
      }
      int id = 0;
      if (q >= nin) {
        id = __builtin_ctzll(ready & todo);
        todo &= ~(1ull << id);
        lds_wr(ctl + 32u + (u32)(q & 7) * 4u, (u32)__builtin_amdgcn_readlane((int)dst_off, id));
      }
      ISSUE_LOADS(q, id);
      waitv<8>();
      if (PWHO == 1 && nin && q == PQ1) probe0((u32)(d - 2) * IN_SRC + 2u * MIB + (u32)p * 4096u, P.peer);
      if (PWHO == 1 && nin && q == PQ2) probe0(PIN_OFF + (call % 3u) * PIN_PAR + (u32)(32 + (d - 2) * 16 + NS) * PIECE + (u32)p * 4096u, P.peer);
      if (q - 1 > (int)pub) { pub = (u32)(q - 1); lds_wr(ctl, pub); }
      if (q == nin - 1) STAMP(myws, 1);
    }
    waitv<0>();
    if (dead) st4_lane0(TMO_OFF, call, myws);
    lds_wr(ctl, dead ? DEAD : (u32)np);
    STAMP(myws, 3);
    STAMPV(myws, 0, tstart);
    return;
  }

  char* const rws = P.peer;
  const int slot = d - 2;
  const u32 dl = (u32)p * 4096u + lane * 16u;
  u32 xl[4], xd[4];
#pragma unroll
  for (int k = 0; k < 4; ++k) {
    const u32 c = (u32)k * 64u + lane, t = c / (4u * XR), row = (c >> 2) % XR;
    xl[k] = (row * XC + t * 4u + ((c ^ ((xrow + row) >> 2)) & 3u)) * 16u;
    xd[k] = XCT ? dl : (xks + t) * 8192u + (xrow + row) * 64u + (c & 3u) * 16u - (u32)k * 1024u;
  }
  u32 tick, ts0, ta;
  int to;
  {
    u32 tg, tm;
    asm volatile("" : "+v"(xl[0]), "+v"(xl[1]), "+v"(xl[2]), "+v"(xl[3]), "+v"(xd[0]), "+v"(xd[1]), "+v"(xd[2]), "+v"(xd[3]));
    asm volatile("s_waitcnt vmcnt(1)\n\t"
                 "v_readfirstlane_b32 %[tick], v160\n\t"
                 "v_readfirstlane_b32 %[tg], v161\n\t"
                 "s_memrealtime s[58:59]\n\t"
                 "s_sub_i32 %[tm], %[tg], %[tis]\n\t"
                 "s_addk_i32 %[tm], 500\n\t"
                 "s_cmpk_gt_u32 %[tm], 1000\n\t"
                 "s_cselect_b32 %[tg], %[tis], %[tg]\n\t"
                 "s_addk_i32 %[tg], %[oc]\n\t"
                 "s_sub_i32 %[to], %[tg], %[tst]\n\t"
                 "s_and_b32 %[tm], %[tick], 15\n\t"
                 "s_mul_i32 %[tm], %[tm], %[osp]\n\t"
                 "s_add_i32 %[tg], %[tg], %[tm]\n\t"
                 "s_bitcmp0_b32 %[tick], 4\n\t"
                 "s_cselect_b32 %[tm], 0x10000, 0\n\t"
                 "v_add_u32 v160, %[tm], %[dl]\n\t"
                 "s_cmp_eq_u32 %[d], 1\n\t"
                 "s_waitcnt lgkmcnt(0)\n\t"
                 "s_mov_b32 %[ta], s58\n\t"
                 "s_cbranch_scc1 5f\n"
                 "4:\n\t"
                 "s_waitcnt lgkmcnt(0)\n\t"
                 "s_sub_i32 %[tm], s58, %[tg]\n\t"
                 "s_mov_b32 %[ts0], s58\n\t"
                 "s_memrealtime s[58:59]\n\t"
                 "s_cmp_lt_i32 %[tm], 0\n\t"
                 "s_cbranch_scc1 4b\n\t"
                 "global_store_dwordx4 v160, v[144:147], %[dst] sc0 sc1\n\t"
                 "global_store_dwordx4 v160, v[148:151], %[dst] offset:1024 sc0 sc1\n\t"
                 "global_store_dwordx4 v160, v[152:155], %[dst] offset:2048 sc0 sc1\n\t"
                 "global_store_dwordx4 v160, v[156:159], %[dst] offset:3072 sc0 sc1\n"
                 "5:\n\t"
                 "s_waitcnt lgkmcnt(0)"
                 : [tick] "=&s"(tick), [ts0] "=&s"(ts0), [ta] "=&s"(ta), [to] "=&s"(to), [tg] "=&s"(tg), [tm] "=&s"(tm)
                 : "{v[144:159]}"(P.f16), "{v[160:161]}"(P.co), [tis] "s"(P.tis), [tst] "s"((u32)tstart), [d] "s"(d),
                   [dl] "v"(dl), [dst] "s"(rws + (size_t)((u32)max(slot, 0) * IN_SRC)), [oc] "i"(OC), [osp] "i"(OSP)
                 : "memory", "scc", "s58", "s59");
  }
  const u32 call = (tick >> 4) + 1u;
  u32 dbg1 = 0, dbg2 = 0;
  const u32 in0 = (u32)slot * IN_SRC + (call & 1u) * PIECE;
  const u32 pin0 = PIN_OFF + (call % 3u) * PIN_PAR + (u32)(d == 1 ? 0 : 32 + slot * 16 + NS) * PIECE;
  int nflag = 0;
#define FLAG(QQ) do { \
    const int q_ = (QQ); \
    const bool real_ = q_ == nflag && q_ < nin; \
    const u32 o_ = real_ ? fin_off(slot, q_) : DUMMY_OFF + (u32)(blockIdx.x * 4 + wave) * 128u; \
    add1_lane0(o_, uni(real_ ? rws : myws)); \
    nflag += real_ ? 1 : 0; \
  } while (0)

  u32 tgrid = 0, tord = 0;
  int qsw = 0;
  if (nin) {
    FLAG(-1);
    const int cold = ta - P.tis > (u32)CLT || P.tis - (u32)tstart > (u32)CKT ? CLD : 0;
    const u32 tb = (u32)max(to + TAU + OG + cold, (int)(ts0 - (u32)tstart) + OD) + (u32)(TAU - 1);
    tgrid = tb - ((u32)tstart + tb - ((u32)(((p + STGD * d) & 15) * TAU) >> 4)) % (u32)TAU;
    qsw = (int)tgrid - (to + TAU + OG + cold) < ((p + 1) * TAU >> 4) ? NIN + 1 : NIN;
    tord = ORD > 1 ? (u32)max(to + TAU + OG + (int)(tick & 15u) * OSP, 0) - tgrid : 0u;
  } else {
    sleep_until(tstart, 600u);
    probe0(pin0 + (u32)p * 4096u, rws);
    probe0(pin0 + (u32)(NOUT1 + NXT - 1) * PIECE + (u32)p * 4096u, rws);
  }
  u32 full = 0, tnext = 0;
  for (int q = nin ? 1 : 0; q < np; ++q) {
    bool isx; int ix;
    piece_of(q, d, isx, ix);
    isx = isx && q < nin;
    if (d != 1 || G1) tnext = max(tnext, tgrid + (q < ORD ? tord : 0u));
    if (!PRE) sleep_until(tstart, tnext);
    if (full <= (u32)q) {
      full = lds_rd(ctl);
      if (full <= (u32)q) {
        bool drained = !(q > 0 && q < nin + LAG);
        u64 el;
        for (;;) {
          __builtin_amdgcn_s_sleep(1);
          full = lds_rd(ctl);
          el = now() - tstart;
          if (full > (u32)q) break;
          if (!drained && __builtin_amdgcn_readfirstlane((int)(el > (u64)(tnext + 30u)))) {
            drained = true;
            waitv<0>();
#pragma unroll
            for (int z = LAG; z >= 1; --z) FLAG(q - z);
          }
          if (__builtin_amdgcn_readfirstlane((int)(el > 2ull * DEADLINE - 1000ull))) { full = DEAD; break; }
        }
        tnext = max(tnext, (u32)__builtin_amdgcn_readfirstlane((u32)el));
      }
      if (full == DEAD) break;
    }
    const u32 la = ring + (u32)(q & 7) * 4096u;
    const u32x4 d0 = *(volatile LDSP u32x4*)(uintptr_t)(la + (isx ? xl[0] : lane * 16u));
    const u32x4 d1 = *(volatile LDSP u32x4*)(uintptr_t)(la + (isx ? xl[1] : lane * 16u + 1024u));
    const u32x4 d2 = *(volatile LDSP u32x4*)(uintptr_t)(la + (isx ? xl[2] : lane * 16u + 2048u));
    const u32x4 d3 = *(volatile LDSP u32x4*)(uintptr_t)(la + (isx ? xl[3] : lane * 16u + 3072u));
    const u32 id = lds_rd(ctl + 32u + (u32)(q & 7) * 4u);
    lds_wr(ctl + 4u, (u32)q + 1u);
    u32 o;
    if (q < nin) {
      o = in0 + (u32)(isx ? cont_x(ix) : cont_w(ix)) * 2u * PIECE;
    } else {
      o = pin0 + min(id, (u32)nid * PIECE - 4096u) - (u32)p * 4096u;
    }
    char* const dst = rws + (size_t)o;
    if (G1 && q == 0) {
      const u32 tb = tnext + (u32)(G1M + TAU1 - 1);
      tnext = tgrid = tb - ((u32)tstart + tb - ((u32)(p * TAU1) >> 4)) % (u32)TAU1;
    }
    if (PRE) sleep_exact(tstart, tnext);
    if (G1 && d == 1) tgrid += q < G1Q && lds_rd(ctl) > (u32)q + 1u ? 0u : (u32)TAU1;
    else tgrid += (u32)(TAU - (q < QA ? DA : 0) - (q >= QB ? DB : 0) - (q >= QC ? DC : 0));
    if (NX && q == qsw && d != 1) tgrid += (u32)((less ? TAU - DB : 0) - (2 * NX * TAU >> 4));
    tnext += (u32)(d == 1 && !G1 ? TAU1 + (q < SWP ? 0 : (less ? XLB : -XLA)) : TAU2);
    st16<0>(isx ? xd[0] : dl, d0, dst);
    st16<1024>(isx ? xd[1] : dl, d1, dst);
    st16<2048>(isx ? xd[2] : dl, d2, dst);
    st16<3072>(isx ? xd[3] : dl, d3, dst);
    if (nin && q < nin + LAG) {
      waitv<5 * LAG>();
      FLAG(q - LAG);
      if (PWHO == 0 && q == PQ1) probe0((u32)slot * IN_SRC + 2u * MIB + (u32)p * 4096u, rws);
      if (PWHO == 0 && q == PQ2) probe0(pin0 + (u32)p * 4096u, rws);
    } else {
      waitv<4 * WIN>();
    }
    if (q == NS + 11) STAMP(myws, 1);
    if (q == nin - 1) STAMP(myws, 2);
    if (q == np - 1) STAMP(myws, 4);
  }
  if ((tick & 15u) == 0u) st4_lane0(LNK_OFF + (u32)d * 128u + 4u, 0u, myws);
  waitv<0>();
  STAMP(myws, 3);
  STAMPV(myws, 0, tstart);
}

struct Frag {
  bf16x8 x[2][2];
  bf16x8 w[2][2];
};

struct Loader {
  const char* base;
  u32 stride[2];
  u32 v[4];
  u32 m0base;
};

template <int STAGE> __device__ __forceinline__ void glds_pair(const Loader& L, int half) {
  const u32 m0v = L.m0base + STAGE * STAGE_B;
  if (half == 0)
    asm volatile("s_mov_b32 m0, %0\n\ts_nop 0\n\t"
                 "global_load_lds_dwordx4 %1, %3 " GLP "\n\t"
                 "global_load_lds_dwordx4 %2, %3 offset:1024 " GLP
                 :: "s"(m0v), "v"(L.v[0]), "v"(L.v[1]), "s"(L.base) : "memory");
  else
    asm volatile("s_mov_b32 m0, %0\n\ts_nop 0\n\t"
                 "global_load_lds_dwordx4 %1, %3 offset:2048 " GLP "\n\t"
                 "global_load_lds_dwordx4 %2, %3 offset:3072 " GLP
                 :: "s"(m0v), "v"(L.v[2]), "v"(L.v[3]), "s"(L.base) : "memory");
}

struct RdAddr { u32 xa[2]; u32 wa[2][2]; };

template <int STAGE> __device__ __forceinline__ bf16x8 rd_x(const RdAddr& A, int mt, int ks) {
  return *(const LDSP bf16x8*)(uintptr_t)(A.xa[ks] + (u32)(STAGE * STAGE_B + mt * 2048));
}
template <int STAGE> __device__ __forceinline__ bf16x8 rd_w(const RdAddr& A, int nt, int ks) {
  s16x4 lo = __builtin_amdgcn_ds_read_tr16_b64_v4i16((LDSP s16x4*)(uintptr_t)(A.wa[nt][0] + (u32)(STAGE * STAGE_B + 8192 + ks * 4096)));
  s16x4 hi = __builtin_amdgcn_ds_read_tr16_b64_v4i16((LDSP s16x4*)(uintptr_t)(A.wa[nt][1] + (u32)(STAGE * STAGE_B + 8192 + ks * 4096)));
  s16x8 v = __builtin_shufflevector(lo, hi, 0, 1, 2, 3, 4, 5, 6, 7);
  return __builtin_bit_cast(bf16x8, v);
}
#define MFMA(nt, mt, ks) acc[nt][mt] = __builtin_amdgcn_mfma_f32_32x32x16_bf16(cur.w[nt][ks], cur.x[mt][ks], acc[nt][mt], 0, 0, 0)

template <int S, int VM, bool LOAD, bool READ>
__device__ __forceinline__ void kstep(f32x16 (&acc)[2][2], const Frag& cur, Frag& nxt, const RdAddr& A, Loader& L) {
  constexpr int SN = (S + 1) & 3;
  if (READ) {
    __builtin_amdgcn_s_waitcnt((VM & 15) | (7 << 4) | (0 << 8) | ((VM >> 4) << 14));
    asm volatile("s_barrier" ::: "memory");
  } else {
    __builtin_amdgcn_s_waitcnt(15 | (7 << 4) | (0 << 8) | (3 << 14));
  }
  SB();
  MFMA(0, 0, 0); SB();
  if (READ) { nxt.w[0][0] = rd_w<SN>(A, 0, 0); nxt.x[0][0] = rd_x<SN>(A, 0, 0); } SB();
  MFMA(0, 1, 0); SB();
  if (READ) { nxt.x[1][0] = rd_x<SN>(A, 1, 0); nxt.w[1][0] = rd_w<SN>(A, 1, 0); } SB();
  MFMA(1, 0, 0); SB();
  if (LOAD) glds_pair<S>(L, 0);
  SB();
  MFMA(1, 1, 0); SB();
  if (READ) { nxt.w[0][1] = rd_w<SN>(A, 0, 1); nxt.x[0][1] = rd_x<SN>(A, 0, 1); } SB();
  MFMA(0, 0, 1); SB();
  if (READ) { nxt.x[1][1] = rd_x<SN>(A, 1, 1); nxt.w[1][1] = rd_w<SN>(A, 1, 1); } SB();
  MFMA(0, 1, 1); SB();
  if (LOAD) { glds_pair<S>(L, 1); L.base += L.stride[S & 1]; }
  SB();
  MFMA(1, 0, 1); SB();
  MFMA(1, 1, 1); SB();
}

template <int S> __device__ __forceinline__ void stage_load(Loader& L) {
  glds_pair<S>(L, 0); glds_pair<S>(L, 1); L.base += L.stride[S & 1];
}

__device__ __forceinline__ void gemm_phase(f32x16 (&acc)[2][2], const RdAddr& A, Loader& L, int n, int sk, int jm, u32 jump, u32& dbgv, int dbg0) {
  Frag f0, f1;
  L.base = uni(L.base + (sk == 0 ? IN_SRC : 0u));
  L.stride[0] = __builtin_amdgcn_readfirstlane(L.stride[0]);
  L.stride[1] = __builtin_amdgcn_readfirstlane(L.stride[1]);
  asm volatile("s_nop 4" ::: "memory");
  stage_load<0>(L); stage_load<1>(L); stage_load<2>(L); stage_load<3>(L);
  waitv<12>();
  asm volatile("s_barrier" ::: "memory");
#pragma unroll
  for (int ks = 0; ks < 2; ++ks)
#pragma unroll
    for (int i = 0; i < 2; ++i) { f0.w[i][ks] = rd_w<0>(A, i, ks); f0.x[i][ks] = rd_x<0>(A, i, ks); }
  SB();
  for (int t = 0; t < n - 4; t += 4) {
    L.base += ((t | jm) & 4 ? jump : 0u) + (t + 4 == sk ? IN_SRC : 0u);
    kstep<0, 8, true, true>(acc, f0, f1, A, L);
    kstep<1, 8, true, true>(acc, f1, f0, A, L);
    kstep<2, 8, true, true>(acc, f0, f1, A, L);
    kstep<3, 8, true, true>(acc, f1, f0, A, L);
  }
  kstep<0, 8, false, true>(acc, f0, f1, A, L);
  kstep<1, 4, false, true>(acc, f1, f0, A, L);
  kstep<2, 0, false, true>(acc, f0, f1, A, L);
  kstep<3, 0, false, false>(acc, f1, f0, A, L);
}

__device__ __forceinline__ u32 pack2(float a, float b) {
  f32x2 t = {a, b};
  return __builtin_bit_cast(u32, __builtin_convertvector(t, bf16x2));
}
__device__ __forceinline__ u32 relu2(u32 v) {
  typedef short s16x2 __attribute__((ext_vector_type(2)));
  return __builtin_bit_cast(u32, __builtin_elementwise_max(__builtin_bit_cast(s16x2, v), s16x2{0, 0}));
}
__device__ __forceinline__ float bf_lo(u32 v) { return __builtin_bit_cast(float, v << 16); }
__device__ __forceinline__ float bf_hi(u32 v) { return __builtin_bit_cast(float, v & 0xffff0000u); }

__device__ __forceinline__ void zero8k(u32 l16, u32 l16b, char* pb) {
  const u32x4 z4 = {0u, 0u, 0u, 0u};
  asm volatile("s_nop 4" ::: "memory");
  st16<0>(l16, z4, pb); st16<1024>(l16, z4, pb); st16<2048>(l16, z4, pb); st16<3072>(l16, z4, pb);
  st16<0>(l16b, z4, pb); st16<1024>(l16b, z4, pb); st16<2048>(l16b, z4, pb); st16<3072>(l16b, z4, pb);
}

__device__ __forceinline__ void tile(const Args& P, int tid, int wave, u32 lane, char* smem, u64 tstart) {
  const int r = P.rank;
  char* const myws = P.my;
  const int g = r >> 2, b = (r >> 1) & 1, a = r & 1;
  const int bid = blockIdx.x;

  if (PF && (tid >> 6) == PF) {
    const int h = ((tid >> 3) & 7) * 4 + wave, u = (tid & 7) >> 1, kf = tid & 1;
    const char* const pb = h < 16 ? P.x + (size_t)((u32)((r ^ (2 + (u >> 1))) * 512 + b * 256 + (u & 1) * 128 + (h & 1) * 64) * 1024u + (u32)(h >> 1) * 128u)
                                  : P.w + (size_t)((u32)((h >> 1) - 8) * 64u * 8192u + (u32)(g * 16 + (a ? 15 - kf : kf)) * 256u + (u32)(h & 1) * 128u);
    if (PFD) sleep_until(tstart, PFD);
    asm volatile("s_mov_b32 m0, %0\n\ts_nop 3\n\tglobal_load_lds_dword %1, %2"
                 :: "s"((u32)(uintptr_t)smem + 66048u), "v"(lane << (h < 16 ? 10 : 13)), "s"(pb) : "memory");
  }
  const u32 cnt = bump_issue(CNT_OFF + (u32)(bid * 4 + wave) * 64u, myws);

  if (LPT && (bid >> 3) == 25 && wave == 0) {
    sleep_until(tstart, (u32)LPT);
    if (lane < 28u) probe(lane * 2u * MIB + MIB + (u32)(bid & 7) * 128u, myws);
  }
  if (RPT && (bid >> 3) == RPB && wave == 1) {
    const u32 dd = 1u + lane % 7u;
    const u64 pp = ((const GLBP u64*)(uintptr_t)__builtin_amdgcn_kernarg_segment_ptr())[6u + 2u * dd];
    const u32 cl = bump_get<0>(cnt);
    const u32 o = (lane < 7u && dd > 1u ? (dd - 2u) * IN_SRC + 2u * MIB : PIN_OFF + (cl % 3u) * PIN_PAR + (dd > 1u ? dd * 16u : 30u) * PIECE) + (u32)(bid & 7) * 128u;
    sleep_until(tstart, (u32)RPT);
    if (lane < 14u) asm volatile("global_atomic_add %0, %1, off sc1" :: "v"(pp + o), "v"(0u) : "memory");
  }
  static_assert(!SUB || NS == 2, "first tiles");
  bool dead = false;
  u32 dbgv = 0;
  int ra = P.rank;
#pragma unroll 1
  for (int job = 0; job < 2; ++job) {
  asm volatile("" : "+s"(tid), "+s"(wave), "+s"(ra), "+v"(lane));
  const int r = ra, g = r >> 2, b = (r >> 1) & 1, a = r & 1;
  const int hi = tid < N_MAIN_WG ? (tid >> 3) * 3 : (tid - N_MAIN_WG) >> 3;
  const int sq = tid < N_MAIN_WG ? 0 : 1 + hi % 3;
  const bool sub = SUB && job == 0 && hi < ((tid & 4) ? 12 : 9);
  int row0, col0, nrem, tskip, j, e, k, rb;
  int pi0 = -1, pi1 = -1;
  bool fin;
  u32 dz = 0;
  if (tid < N_MAIN_WG || sub) {
    const int xcd = tid & 7, o = hi / 3, u = xcd >> 1;
    int rbi, cbi;
    if (u < 2) { rbi = o < 24 ? 1 + o % 3 : 0; cbi = o < 24 ? o / 3 : o - 24; }
    else { rbi = o & 3; cbi = o >> 2; }
    rb = u + 4 * rbi; k = cbi * 2 + (xcd & 1);
    j = rb >> 1; e = rb & 1;
    row0 = (r ^ j) * 512 + b * 256 + e * 128; col0 = g * 2048 + (a ? 15 - k : k) * 128;
    nrem = (j >= 2 ? 40 : 48) >> (sub ? 1 : 0); tskip = j >= 2 ? (j - 2) * (sub ? 4 : 8) : 99;
    fin = j == 0;
    if (j == 0) pi1 = 6 * NS + 15 - k;
    else if (j >= 2 && k >= 16 - NS) pi0 = ((j ^ 1) - 2) * NS + 15 - k;
    else if (j >= 2 && k == 15 - NS && e * 4 + wave < NX) { pi0 = NOUT1; dz = (u32)(((j ^ 1) - 2) * NX) * 8192u; }
  } else {
    const int f = tid - N_MAIN_WG;
    const int qd = 1 + f / 32, rem = f & 31;
    e = (rem >> 3) & 1; k = (rem & 7) * 2 + (rem >> 4);
    const int h = b ^ (qd & 1), g2 = g ^ (qd >> 1);
    row0 = r * 512 + h * 256 + e * 128; col0 = g2 * 2048 + (a ? 15 - k : k) * 128;
    nrem = 0; tskip = 99; j = 0; rb = 0;
    fin = true;
    const int ns = e * 4 + wave < NX ? NS + 1 : NS;
    const int iA = k >= ns ? 32 + (2 * qd - 2) * 16 + k : -1;
    const int iB = 15 - k >= ns ? 32 + (2 * qd - 1) * 16 + 15 - k : -1;
    pi0 = k < 8 ? iA : iB;
    pi1 = k < 8 ? iB : iA;
  }

  const u32 sbase = (u32)(uintptr_t)smem;
  const int wm = wave >> 1, wn = wave & 1;
  const u32 hb = sub ? ((lane >> 3) ^ (lane >> 5)) & 1u : 0u;
  RdAddr A;
  {
    const u32 h = lane >> 5, m = lane & 31u, f = (lane >> 2) & 3u;
    const u32 xr = (u32)(sub ? wave : wm * 2) * 32u + m;
    A.xa[0] = sbase + xr * 64u + 16u * (h ^ f);
    A.xa[1] = sbase + xr * 64u + 16u * ((2u + h) ^ f);
    const u32 blk = (lane >> 4) & 1u, q = (lane & 15u) >> 2, pp = lane & 3u;
#pragma unroll
    for (int nt = 0; nt < 2; ++nt)
#pragma unroll
      for (int t = 0; t < 2; ++t) {
        const u32 row = 8u * h + 4u * t + q;
        const u32 ch = ((u32)(sub ? wm * 12 : wn * 8) ^ ((u32)nt * 4u)) + 2u * blk + (pp & 1u);
        A.wa[nt][t] = sbase + 256u * row + 16u * (ch ^ (((row & 3u) << 2) | ((row >> 2) & 3u))) + 8u * (pp >> 1);
      }
  }

  f32x16 acc[2][2];
#pragma unroll
  for (int i = 0; i < 2; ++i)
#pragma unroll
    for (int k = 0; k < 2; ++k)
#pragma unroll
      for (int z = 0; z < 16; ++z) acc[i][k][z] = 0.f;

  const u32 tile_off = (u32)e * 32768u + (sub ? (u32)(sq * 4 + wave) * 2048u : (u32)wave * 8192u);
  const u32 l16 = lane * 16u, l16b = lane * 16u + 4096u;
  const u32 tw = sbase + (u32)wave * 8192u + (lane & 31u) * 128u + (((lane >> 5) ^ (lane & 7u)) * 16u);
  const u32 tr = sbase + (u32)wave * 8192u + (lane >> 3) * 128u + (((lane & 7u) ^ (lane >> 3)) * 16u);
  const u32 oo = (lane >> 3) * 8192u + (lane & 7u) * 16u;
  char* const ob = uni(P.out + (size_t)((u32)((row0 & 511) + wm * 64) * 8192u + (u32)(col0 + wn * 64) * 2u));
  const int nph = nrem ? 2 : 1;
  const int qin = max(slot_w(k), slot_x(j, e, j == 7 ? 2 : 7));
  const u32 tin = (u32)(EARLY + TQ(qin));
  const int addph = fin ? nph - 1 : 0;

  if (SPR) { if (sub || (PJ1 && j == 1)) __builtin_amdgcn_s_setprio(SPR); else __builtin_amdgcn_s_setprio(0); }
  if (PD0 && !sub && (u32)(tid - PDL) < (u32)(200 - PDL)) {
    const int h1 = (tid - 104) >> 3;
    sleep_until(tstart, (u32)(PD1 && (u32)(h1 - ((tid & 4) ? 3 : 0)) < 3u ? PD1 : PD0));
  }
  Loader L;
  L.m0base = sbase + (u32)wave * 4096u;
  u32 call = 0;
  for (int ph = 0; ph < nph; ++ph) {
    int n, ts;
    bool skip = false;
    if (ph == 0) {
      n = sub ? 8 : 16; ts = 99;
      if (wave < 2) {
        L.base = P.x + (size_t)((u32)row0 * 1024u + (sub ? (u32)(sq >> 1) * 65536u + (u32)wave * 64u : (u32)wave * 65536u));
        L.stride[0] = L.stride[1] = sub ? 128u : 64u;
        const u32 c = (lane & 3u) ^ ((lane >> 4) & 3u);
#pragma unroll
        for (int k = 0; k < 4; ++k) L.v[k] = (lane >> 2) * 1024u + c * 16u + (u32)k * (16384u - 1024u);
      } else {
        L.base = P.w + (size_t)((u32)(wave - 2) * 16u * 8192u + (u32)col0 * 2u + (sub ? (u32)(sq & 1) * 128u : 0u));
        L.stride[0] = L.stride[1] = sub ? 64u * 8192u : 32u * 8192u;
        const u32 q = lane >> 4;
#pragma unroll
        for (int k = 0; k < 4; ++k)
          L.v[k] = q * 8192u + (((lane & 15u) ^ ((q << 2) | (u32)k)) * 16u) + (u32)k * (4u * 8192u - 1024u) + hb * (32u * 8192u - 128u);
      }
    } else {
      n = nrem; ts = tskip;
      STAMP(myws, 1);
      if (wave == 0) {
        const int sl = (int)min(lane, 5u);
        const u32 fo = fin_off(sl, max(slot_w(k), slot_x(j, e, sl + 2)));
        bool pend = lane < 6u && sl + 2 != j;
        sleep_until(tstart, tin);
        const u64 t0 = now();
        u32 st = 0;
        for (;;) {
          if (pend) pend = atom_rd(fo, myws) < 16u * call;
          if (__builtin_amdgcn_ballot_w64(pend) == 0ull) break;
          if (give_up(t0, myws, call)) {
            if (lane == 0) st_agt(myws + TMO_OFF, call);
            st = 1;
            break;
          }
          __builtin_amdgcn_s_sleep(PSL);
        }
        if (lane == 0) *(LDSP u32*)(uintptr_t)(sbase + 4u * STAGE_B) = st;
        __builtin_amdgcn_s_waitcnt(0);
      }
      asm volatile("s_barrier" ::: "memory");
      const bool wgdead = lds_rd(sbase + 4u * STAGE_B) != 0;
      STAMP(myws, 2);
      dead = dead || wgdead;
      skip = wgdead;
      const u32 par = call & 1u;
      static_assert(!XCT, "layout of the x pieces");
      L.base = myws + (size_t)(((u32)(wave < 2 ? cont_x(rb) : cont_w(k)) * 2u + par) * PIECE
                               + (u32)(sub && wave < 2 ? (sq >> 1) + wave * 2 : wave & 1) * 4096u);
      L.stride[0] = L.stride[1] = sub ? 16384u : 8192u;
#pragma unroll
      for (int k = 0; k < 4; ++k) L.v[k] = sub && wave >= 2 ? (l16 ^ ((hb ^ (u32)(sq & 1)) << 7)) + hb * 8192u : l16;
    }
    if (!skip) gemm_phase(acc, A, L, n, ts, sub ? 4 : 0, ph ? IN_SRC - PIECE : 0u, dbgv, ph * 8);
    if (ph == nph - 1) STAMP(myws, 3);
    if (ph == 0) {
      call = bump_get<0>(cnt);
      if (ZT0 && max(pi0, pi1) >= 0) {
        char* const pz = myws + (size_t)(PIN_OFF + ((call + 2u) % 3u) * PIN_PAR + tile_off + dz);
        sleep_until(tstart, (u32)ZT0 + (u32)((tid >> 3) & 15) * (u32)ZDT);
        if (pi0 >= 0) zero8k(l16, l16b, uni(pz + (size_t)((u32)pi0 * PIECE)));
        if (pi1 >= 0) zero8k(l16, l16b, uni(pz + (size_t)((u32)pi1 * PIECE)));
      }
    }
    const u32 pin = PIN_OFF + (call % 3u) * PIN_PAR + tile_off + dz;
#pragma unroll 1
    for (int s = 0; s < 2; ++s) {
      const int pi = ph != addph ? -1 : (s == 0 ? pi0 : pi1);
      if (pi < 0 && !(fin && ph == 0 && s == 0)) continue;
      u32x4 pv[8];
#pragma unroll
      for (int i = 0; i < 8; ++i) pv[i] = u32x4{0u, 0u, 0u, 0u};
      if (pi >= 0) {
        char* const pb = uni(myws + (size_t)(pin + (u32)pi * PIECE));
        const bool first = pi == 0 || (pi >= 32 && (pi & 15) == NS);
        const u32 wo = pin + (u32)(pi - ((!WOWN && (lane & 2u) && !first) ? 1 : 0)) * PIECE + (lane & 1u) * 4096u + 4092u;
        sleep_until(tstart, pi >= 32 ? (u32)(EARLY + TQ(NIN - 2 + (pi & 15) - NS)) : (fin ? 0u : tin - 300u));
        const u64 t0 = now();
        while (!dead) {
          u32 v = WOWN ? 1u : 0u;
          if (lane < (WOWN ? 2u : 4u)) v = atom_rd(wo, myws);
          if (WOWN ? __builtin_amdgcn_ballot_w64(v == 0u) == 0ull : __builtin_amdgcn_ballot_w64(v != 0u) != 0ull) break;
          if (give_up(t0, myws, call)) { dead = true; break; }
          __builtin_amdgcn_s_sleep(WSL);
        }
        if (s == 1) STAMP(myws, 4);
        for (;;) {
          if (__builtin_expect(dead, 0)) {
            if (lane == 0) st_agt(myws + TMO_OFF, call);
#pragma unroll
            for (int i = 0; i < 8; ++i) pv[i] = u32x4{0x7fc07fc0u, 0x7fc07fc0u, 0x7fc07fc0u, 0x7fc07fc0u};
            break;
          }
          asm volatile("s_nop 4\n\t"
                       "global_load_dwordx4 %0, %8, %10 sc0 sc1\n\t"
                       "global_load_dwordx4 %1, %8, %10 offset:1024 sc0 sc1\n\t"
                       "global_load_dwordx4 %2, %8, %10 offset:2048 sc0 sc1\n\t"
                       "global_load_dwordx4 %3, %8, %10 offset:3072 sc0 sc1\n\t"
                       "global_load_dwordx4 %4, %9, %10 sc0 sc1\n\t"
                       "global_load_dwordx4 %5, %9, %10 offset:1024 sc0 sc1\n\t"
                       "global_load_dwordx4 %6, %9, %10 offset:2048 sc0 sc1\n\t"
                       "global_load_dwordx4 %7, %9, %10 offset:3072 sc0 sc1"
                       : "=&v"(pv[0]), "=&v"(pv[1]), "=&v"(pv[2]), "=&v"(pv[3]), "=&v"(pv[4]), "=&v"(pv[5]), "=&v"(pv[6]), "=&v"(pv[7])
                       : "v"(l16), "v"(l16b), "s"(pb) : "memory");
          const bool late = __builtin_amdgcn_readfirstlane((int)(now() - t0 > 50000ull)) != 0;
          asm volatile("s_waitcnt vmcnt(0)"
                       : "+v"(pv[0]), "+v"(pv[1]), "+v"(pv[2]), "+v"(pv[3]), "+v"(pv[4]), "+v"(pv[5]), "+v"(pv[6]), "+v"(pv[7]) :: "memory");
          u32 mn = 0xffffffffu;
#pragma unroll
          for (int i = 0; i < 8; ++i)
#pragma unroll
            for (int z = 0; z < 4; ++z) mn = min(mn, pv[i][z]);
          if (__builtin_amdgcn_ballot_w64(mn == 0u) == 0ull) break;
          if (__builtin_expect(late, 0)) dead = give_up(t0, myws, call);
        }
      }
#pragma unroll
      for (int nt = 0; nt < 2; ++nt)
#pragma unroll
        for (int mt = 0; mt < 2; ++mt)
#pragma unroll
          for (int sx = 0; sx < 2; ++sx)
#pragma unroll
            for (int z = 0; z < 4; ++z) {
              const u32 u = pv[(nt * 2 + mt) * 2 + sx][z];
              acc[nt][mt][8 * sx + 2 * z] += bf_lo(u);
              acc[nt][mt][8 * sx + 2 * z + 1] += bf_hi(u);
            }
      if (!fin) continue;
      if (s == 1) STAMP(myws, 6);
#pragma unroll
      for (int nt = 0; nt < 2; ++nt)
#pragma unroll
        for (int mt = 0; mt < 2; ++mt)
#pragma unroll
          for (int sx = 0; sx < 2; ++sx) {
            u32x4 v;
#pragma unroll
            for (int z = 0; z < 4; ++z) v[z] = relu2(pack2(acc[nt][mt][8 * sx + 2 * z], acc[nt][mt][8 * sx + 2 * z + 1]));
            *(volatile LDSP u32x4*)(uintptr_t)((tw ^ (u32)((nt * 4 + sx * 2) * 16)) + (u32)mt * 4096u) = v;
          }
      u32 o1, o2, o3, o4, o5, o6, o7;
      asm volatile("ds_read_b128 %0, %15\n\t"
                   "ds_read_b128 %1, %15 offset:1024\n\t"
                   "ds_read_b128 %2, %15 offset:2048\n\t"
                   "ds_read_b128 %3, %15 offset:3072\n\t"
                   "ds_read_b128 %4, %15 offset:4096\n\t"
                   "ds_read_b128 %5, %15 offset:5120\n\t"
                   "ds_read_b128 %6, %15 offset:6144\n\t"
                   "ds_read_b128 %7, %15 offset:7168\n\t"
                   "v_add_u32 %8, 0x10000, %16\n\t"
                   "v_add_u32 %9, 0x20000, %16\n\t"
                   "v_add_u32 %10, 0x30000, %16\n\t"
                   "v_add_u32 %11, 0x40000, %16\n\t"
                   "v_add_u32 %12, 0x50000, %16\n\t"
                   "v_add_u32 %13, 0x60000, %16\n\t"
                   "v_add_u32 %14, 0x70000, %16\n\t"
                   "s_mov_b64 exec, %18\n\t"
                   "s_waitcnt lgkmcnt(7)\n\tglobal_store_dwordx4 %16, %0, %17 sc0 sc1\n\t"
                   "s_waitcnt lgkmcnt(6)\n\tglobal_store_dwordx4 %8, %1, %17 sc0 sc1\n\t"
                   "s_waitcnt lgkmcnt(5)\n\tglobal_store_dwordx4 %9, %2, %17 sc0 sc1\n\t"
                   "s_waitcnt lgkmcnt(4)\n\tglobal_store_dwordx4 %10, %3, %17 sc0 sc1\n\t"
                   "s_waitcnt lgkmcnt(3)\n\tglobal_store_dwordx4 %11, %4, %17 sc0 sc1\n\t"
                   "s_waitcnt lgkmcnt(2)\n\tglobal_store_dwordx4 %12, %5, %17 sc0 sc1\n\t"
                   "s_waitcnt lgkmcnt(1)\n\tglobal_store_dwordx4 %13, %6, %17 sc0 sc1\n\t"
                   "s_waitcnt lgkmcnt(0)\n\tglobal_store_dwordx4 %14, %7, %17 sc0 sc1\n\t"
                   "s_cmp_lg_u64 %18, -1\n\t"
                   "s_cbranch_scc1 1f\n\t"
                   "s_endpgm\n"
                   "1:\n\t"
                   "s_mov_b64 exec, -1\n\t"
                   "s_nop 1"
                   : "=&v"(pv[0]), "=&v"(pv[1]), "=&v"(pv[2]), "=&v"(pv[3]), "=&v"(pv[4]), "=&v"(pv[5]), "=&v"(pv[6]), "=&v"(pv[7]),
                     "=&v"(o1), "=&v"(o2), "=&v"(o3), "=&v"(o4), "=&v"(o5), "=&v"(o6), "=&v"(o7)
                   : "v"(tr), "v"(oo), "s"(ob), "s"(s == 1 ? ~0ull : (u64)WMK) : "memory", "scc");
      if (s == 0) waitv<0>();
    }
  }

  if (!fin) {
    STAMP(myws, 6);
    const bool dir = DIR && j >= 2 && k < NS;
    char* dst = uni(dir ? P.peer + (size_t)(PIN_OFF + (call % 3u) * PIN_PAR + (u32)((j - 2) * NS + k) * PIECE + tile_off)
                        : myws + (size_t)(POUT_OFF + (u32)((j - 1) * 16 + k) * PIECE + tile_off));
    char* fl = myws + (size_t)(FLOC_OFF + (u32)((j - 1) * 8 + e * 4 + (sub ? sq : wave)) * 128u + (u32)k * 4u);
    if (sub) {
      typedef float f32x4 __attribute__((ext_vector_type(4)));
#pragma unroll
      for (int i = 0; i < 4; ++i)
        *(volatile LDSP f32x4*)(uintptr_t)(L.m0base + l16 + (u32)i * 1024u) =
            f32x4{acc[1][0][4 * i], acc[1][0][4 * i + 1], acc[1][0][4 * i + 2], acc[1][0][4 * i + 3]};
      __builtin_amdgcn_s_waitcnt(15 | (7 << 4) | (0 << 8) | (3 << 14));
      asm volatile("s_barrier" ::: "memory");
#pragma unroll
      for (int i = 0; i < 4; ++i) {
        const f32x4 t = *(volatile LDSP f32x4*)(uintptr_t)((sbase + (u32)(wave ^ 2) * 4096u) + l16 + (u32)i * 1024u);
#pragma unroll
        for (int z = 0; z < 4; ++z) acc[0][0][4 * i + z] += t[z];
      }
    }
    u32 least = 1u;
    if (__builtin_expect(dead, 0)) {
      least = 0x7fc07fc0u;
#pragma unroll
      for (int i = 0; i < 64; ++i) acc[i >> 5][(i >> 4) & 1][i & 15] = 0.f;
    }
    asm volatile("s_nop 4" ::: "memory");
#pragma unroll
    for (int nt = 0; nt < 2; ++nt)
#pragma unroll
      for (int mt = 0; mt < 2; ++mt) {
        if (sub && nt + mt) continue;
        u32x4 v0, v1;
#pragma unroll
        for (int z = 0; z < 4; ++z) {
          v0[z] = max(pack2(acc[nt][mt][2 * z], acc[nt][mt][2 * z + 1]), least);
          v1[z] = max(pack2(acc[nt][mt][8 + 2 * z], acc[nt][mt][8 + 2 * z + 1]), least);
        }
        const u32 o = nt ? l16b : l16;
        if (mt == 0) st16<0>(o, v0, dst); else st16<2048>(o, v0, dst);
        if (dir) __builtin_amdgcn_s_sleep(DSL);
        if (mt == 0) st16<1024>(o, v1, dst); else st16<3072>(o, v1, dst);
        if (dir) __builtin_amdgcn_s_sleep(DSL);
      }
    waitv<0>();
    if (sub) asm volatile("s_barrier" ::: "memory");
    if (lane == 0) st_sys(fl, call);
  }
  if (tid < N_MAIN_WG || !sub) break;
  }
  STAMP(myws, 5);
  STAMPV(myws, 0, tstart);
}

__global__ __launch_bounds__(256, 2) void fused_gemm_rs(Params P) {
  extern __shared__ __attribute__((aligned(16))) char smem[];
  const int bid = blockIdx.x;
  typedef u64 u64x4 __attribute__((ext_vector_type(4)));
  u64x4 a4;
  u64 tstart, pe;
  int rk, d;
  u32 tis, t0;
  u32x16 f16;
  u32x2 co;
  static_assert(N_PUSH_WG == 56 && PROT == 0 && PWN == 1 && CPB >= N_PUSH_WG, "entry code");
  asm volatile("v_readfirstlane_b32 s46, %[tid]\n\t"
               "s_load_dwordx8 s[36:43], %[ka], 0x0\n\t"
               "s_lshr_b32 s47, %[bid], 3\n\t"
               "s_cmp_gt_u32 %[bid], 55\n\t"
               "s_cbranch_scc1 2f\n\t"
               "s_lshr_b32 s48, s46, 7\n\t"
               "s_mul_i32 s49, s48, 3\n\t"
               "s_add_i32 s47, s47, s49\n\t"
               "s_add_i32 s49, s47, -7\n\t"
               "s_min_u32 s47, s47, s49\n\t"
               "s_lshl_b32 s49, s47, 4\n\t"
               "s_load_dwordx4 s[52:55], %[ka], s49 offset:0x40\n\t"
               "s_memrealtime s[44:45]\n\t"
               "s_load_dword s62, %[ka], 0x20\n\t"
               "s_setprio 3\n\t"
               "s_add_i32 s63, s47, 1\n\t"
               "v_lshlrev_b32 v1, 2, %[tid]\n\t"
               "v_mov_b32 v27, 0\n\t"
               "v_add_u32 v1, %[ctl], v1\n\t"
               "s_bitcmp1_b32 s46, 6\n\t"
               "ds_write_b32 v1, v27\n\t"
               "s_cbranch_scc0 1f\n\t"
               "s_and_b32 s49, %[bid], 7\n\t"
               "s_lshl1_add_u32 s48, s49, s48\n\t"
               "v_bfe_u32 v1, %[tid], 4, 2\n\t"
               "v_and_b32 v2, 15, %[tid]\n\t"
               "v_lshlrev_b32 v3, 2, v1\n\t"
               "v_lshlrev_b32 v1, 13, v1\n\t"
               "v_xor_b32 v2, v3, v2\n\t"
               "v_lshl_or_b32 v1, s48, 17, v1\n\t"
               "v_lshl_or_b32 v20, v2, 4, v1\n\t"
               "v_xor_b32 v21, 16, v20\n\t"
               "v_xor_b32 v22, 32, v20\n\t"
               "v_xor_b32 v23, 48, v20\n\t"
               "v_add_u32 v21, 0x7c00, v21\n\t"
               "v_add_u32 v22, 0xf800, v22\n\t"
               "v_add_u32 v23, 0x17400, v23\n\t"
               "s_lshl_b32 s49, s63, 7\n\t"
               "s_add_i32 s49, s49, %[lnk]\n\t"
               "v_mov_b32 v28, s49\n\t"
               "v_mov_b32 v25, 1\n\t"
               "s_waitcnt lgkmcnt(0)\n\t"
               "s_memrealtime s[58:59]\n\t"
               "global_load_dwordx4 v[144:147], v20, s[54:55] " MLP "\n\t"
               "global_load_dwordx4 v[148:151], v21, s[54:55] offset:1024 " MLP "\n\t"
               "global_load_dwordx4 v[152:155], v22, s[54:55] offset:2048 " MLP "\n\t"
               "global_load_dwordx4 v[156:159], v23, s[54:55] offset:3072 " MLP "\n\t"
               "s_mov_b64 exec, 1\n\t"
               "global_atomic_add v160, v28, v25, s[40:41] sc0 sc1\n\t"
               "s_add_i32 s50, s47, -1\n\t"
               "s_max_i32 s50, s50, 0\n\t"
               "s_mul_i32 s51, s50, %[fsl]\n\t"
               "s_lshl_b32 s49, s48, 7\n\t"
               "s_add_i32 s51, s51, s49\n\t"
               "s_add_i32 s51, s51, %[f0]\n\t"
               "v_mov_b32 v29, s51\n\t"
               "s_waitcnt lgkmcnt(0)\n\t"
               "s_or_b32 s66, s58, 1\n\t"
               "v_mov_b32 v26, s66\n\t"
               "global_atomic_cmpswap v161, v28, v[26:27], s[40:41] offset:4 sc0 sc1\n\t"
               "global_atomic_add v29, v27, s[52:53] sc1\n\t"
               "s_mov_b64 exec, -1\n"
               "1:\n\t"
               "s_waitcnt lgkmcnt(0)\n\t"
               "s_barrier\n\t"
               "s_branch 5f\n"
               "2:\n\t"
               "s_memrealtime s[44:45]\n\t"
               "s_load_dword s62, %[ka], 0x20\n\t"
               "s_load_dwordx2 s[52:53], %[ka], 0x40\n\t"
               "s_mov_b32 s63, 1\n\t"
               "s_add_i32 s47, %[bid], %[cpb]\n\t"
               "s_cmp_lt_u32 s47, 16\n\t"
               "s_cbranch_scc0 5f\n\t"
               "s_cmpk_lt_u32 s46, 0xc0\n\t"
               "s_cbranch_scc0 5f\n\t"
               "s_getpc_b64 s[56:57]\n\t"
               "s_mov_b32 m0, %[cpl]\n\t"
               "v_lshlrev_b32 v1, 7, %[tid]\n\t"
               "v_min_u32 v1, %[cpf], v1\n\t"
               "global_load_lds_dword v1, s[56:57]\n"
               "5:\n\t"
               "s_waitcnt lgkmcnt(0)"
               : "={s[36:43]}"(a4), "={s[44:45]}"(tstart), "={s46}"(t0), "={s[52:53]}"(pe), "={s62}"(rk), "={s63}"(d),
                 "={s66}"(tis)
               : [ka] "s"(__builtin_amdgcn_kernarg_segment_ptr()), [bid] "s"(bid), [tid] "v"(threadIdx.x),
                 [ctl] "i"(65536), [lnk] "i"(LNK_OFF), [fsl] "i"(IN_SRC + 4u * PIECE), [f0] "i"(NS * 2u * PIECE + 4096u),
                 [cpb] "i"(-CPB), [cpl] "i"(66048), [cpf] "i"(CPF - 128)
               : "memory", "scc", "s47", "s48", "s49", "s50", "s51", "s54", "s55", "s56", "s57", "s58", "s59",
                 "v1", "v2", "v3", "v20", "v21", "v22", "v23", "v25", "v26", "v27", "v28", "v29");
  asm volatile("" : "={v[144:159]}"(f16), "={v[160:161]}"(co));
  const u32 lane = threadIdx.x & 63u;
  const int wave = (int)(t0 >> 6);
  const Args A = {(const char*)a4[0], (const char*)a4[1], (char*)a4[2], (char*)a4[3], (char*)pe, rk, tis, f16, co};
  if (bid < N_PUSH_WG) mover(A, d, wave, lane, smem, tstart);
  else if (bid >= 256 && bid < 256 + N_PUSH_WG) {
    for (int i = 0; i < 6; ++i) __builtin_amdgcn_s_sleep(16);
  } else tile(A, bid < 256 - ROT ? bid - N_PUSH_WG + ROT : (bid < 256 ? bid - (256 - ROT) : bid - 2 * N_PUSH_WG), wave, lane, smem, tstart);
}

}

extern "C" void kernel_launch(int rank, int nranks, ncclComm_t comm,
                              hipStream_t stream,
                              void* const* d_in, const int* in_sizes, int n_in,
                              void* d_out, int out_size,
                              const int* in_dtypes, int out_dtype,
                              void* const* d_out_all,
                              void* const* d_ws_all, size_t ws_bytes) {
  if (nranks != NR || n_in < 2 || ws_bytes < (size_t)WS_NEED || (unsigned)rank >= (unsigned)NR) {
    fprintf(stderr, "unsupported configuration\n");
    return;
  }
  Params P;
  P.x = (const char*)d_in[0];
  P.w = (const char*)d_in[1];
  P.out = (char*)d_out;
  P.my = (char*)d_ws_all[rank];
  for (int i = 0; i < NR; ++i) {
    const int r = rank ^ i;
    P.lk[i].peer = (char*)d_ws_all[r];
    P.lk[i].w0 = P.w + (size_t)((r & 1) * (256 * 8192 + 15 * 256) + (r >> 2) * 4096);
  }
  P.rank = rank;
  P.pad[0] = P.pad[1] = P.pad[2] = 0;
  HIP_CHECK(hipFuncSetAttribute((const void*)fused_gemm_rs, hipFuncAttributeMaxDynamicSharedMemorySize, (int)LDS_BYTES));
  hipLaunchKernelGGL(fused_gemm_rs, dim3(N_WG), dim3(256), LDS_BYTES, stream, P);
  HIP_CHECK(hipGetLastError());
}
```

```cpp
#include <hip/hip_runtime.h>
#include <rccl/rccl.h>
#include <cstdio>
#include <cstdint>

#define HIP_CHECK(expr) do { hipError_t _e = (expr); if (_e != hipSuccess) { \
    fprintf(stderr, "HIP error %s at %s:%d\n", hipGetErrorString(_e), __FILE__, __LINE__); \
    return; } } while (0)

namespace {

typedef unsigned u32;
typedef unsigned long long u64;
typedef u32 u32x4 __attribute__((ext_vector_type(4)));
typedef u32 u32x2 __attribute__((ext_vector_type(2)));
typedef u32 u32x16 __attribute__((ext_vector_type(16)));
typedef float f32x2 __attribute__((ext_vector_type(2)));
typedef float f32x16 __attribute__((ext_vector_type(16)));
typedef __bf16 bf16x2 __attribute__((ext_vector_type(2)));
typedef __bf16 bf16x8 __attribute__((ext_vector_type(8)));
typedef short s16x4 __attribute__((ext_vector_type(4)));
typedef short s16x8 __attribute__((ext_vector_type(8)));
#define LDSP __attribute__((address_space(3)))
#define GLBP __attribute__((address_space(1)))

constexpr int NR = 8;
constexpr u32 PIECE = 65536u;
constexpr u32 MIB = 1u << 20;
#define NS 2
#define NX 4
#define PF 0
#define SUB 1
#define OX 14
#define TAU 112
#define T0 20
#define QA 0
#define DA 0
#define QB 20
#define DB 1
#define QC 99
#define DC 0
#define TQ(q) (T0 + TAU * (q) - DA * ((q) < QA ? (q) : QA) - DB * ((q) > QB ? (q) - QB : 0))
#define EARLY 150
#define TAU1 111
#define TAU2 108
#define STGD 2
#define PSL 4
#define LSL 16
#define WSL 8
#define RSL 0
#define WOWN 0
#define ALN 1
#define ALO 99
#define ALT 25
#define PWN 1
#define DIR 0
#define DSL 4
#define SWP 14
#define CPF 15616
#define CPB 56
#define PD0 400
#define PD1 1400
#define LZ 13
#define LZ1 4
#define LZD 200
#define LZS 24
#define PSL2 0
#define LSQ 2
#define HRM 1
#define EIN 350
#define PAT 200
#define RSLOT 0
#define PDL 104
#define WE 1
#define PJ1 1
#define OVP 1
#define SPR 2
#define PDF 0
#define GSH -18
#define ORG 1
#define OC 34
#define OG 10
#define OSP 7
#define ORD 1
#define OD 20
#define CLT 84
#define CKT 44
#define CLD 68
#define LPT 400
#define ROT 48
#define PWIN 64
#define PWHO 1
#define PQ1 9
#define PQ2 14
#define PROT 0
#define XR 32
#define ZT0 900
#define ZDT 100
#define RPT 400
#define RPB 25
#define XCT 0
#define L1T 500
#define SXT 6
#define PRE 1
#define WMK 1ull
#define G1 0
#define G1M 0
#define G1Q 7
#define PFD 0
#define LAG 1
#define WIN 3
#define GLP "sc0 sc1"
#define MLP "sc0 sc1"
constexpr int NIN = 30;
constexpr int NOUT = 16 - NS;
constexpr int NOUT1 = 16 + 6 * NS;
constexpr int NXT = (12 * NX + 15) / 16;
constexpr int NXL = 12 * NX - 16 * (NXT - 1);
static_assert(NOUT1 + NXT <= 32, "receive buffer of link 1");
constexpr int XLA = NX ? (TAU1 * (16 - NXL) + 8 * (14 + NXT)) / (16 * (14 + NXT)) : 0;
constexpr int XLB = NX ? (TAU1 * NXL + 8 * (13 + NXT)) / (16 * (13 + NXT)) : 0;
constexpr u32 IN_SRC = 4u * MIB;
constexpr u32 PIN_OFF = 24u * MIB;
constexpr u32 PIN_PAR = 8u * MIB;
constexpr u32 POUT_OFF = 48u * MIB;
constexpr u32 FLG_OFF = 55u * MIB;
constexpr u32 FLOC_OFF = FLG_OFF + 16384u;
constexpr u32 TMO_OFF = FLG_OFF + 49152u;
constexpr u32 LNK_OFF = FLG_OFF + MIB + 262144u + 131072u;
constexpr u32 DUMMY_OFF = FLG_OFF + 65536u;
constexpr u32 DBG_OFF = FLG_OFF + 131072u;
constexpr u32 CNT_OFF = FLG_OFF + MIB + 262144u;
constexpr u32 WS_NEED = FLG_OFF + 2u * MIB;

constexpr int N_PUSH_WG = 56;
constexpr int N_MAIN_WG = 256;
constexpr int N_FIN_WG = 96;
constexpr int N_WG = 2 * N_PUSH_WG + N_MAIN_WG + N_FIN_WG;
constexpr u32 STAGE_B = 16384u;
constexpr u32 LDS_BYTES = 4u * STAGE_B + 1024u;
constexpr u64 DEADLINE = 100000000ull;
constexpr u32 DEAD = 0x7fffffffu;

struct Params {
  const char* x;
  const char* w;
  char* my;
  char* out;
  int rank;
  int pad[3];
  struct { char* peer; const char* w0; } lk[NR];
};
struct Args { const char* x; const char* w; char* my; char* out; char* peer; int rank; u32 tis; u32x16 f16; u32x2 co; };

#define SB() __builtin_amdgcn_sched_barrier(0)
template <int N> __device__ __forceinline__ void waitv() { asm volatile("s_waitcnt vmcnt(%0)" :: "i"(N) : "memory"); }

__device__ __forceinline__ u64 now() { return __builtin_amdgcn_s_memrealtime(); }
template <typename T> __device__ __forceinline__ T* uni(T* p) {
  const u64 v = (u64)(uintptr_t)p;
  const u32 lo = __builtin_amdgcn_readfirstlane((u32)v), hi = __builtin_amdgcn_readfirstlane((u32)(v >> 32));
  return (T*)(uintptr_t)(((u64)hi << 32) | lo);
}
__device__ __forceinline__ u32 ld_sys(const void* p) {
  return __hip_atomic_load((const GLBP u32*)(uintptr_t)p, __ATOMIC_RELAXED, __HIP_MEMORY_SCOPE_SYSTEM);
}
__device__ __forceinline__ void st_sys(void* p, u32 v) {
  __hip_atomic_store((GLBP u32*)(uintptr_t)p, v, __ATOMIC_RELAXED, __HIP_MEMORY_SCOPE_SYSTEM);
}
__device__ __forceinline__ void st_agt(void* p, u32 v) {
  __hip_atomic_store((GLBP u32*)(uintptr_t)p, v, __ATOMIC_RELAXED, __HIP_MEMORY_SCOPE_AGENT);
}
__device__ __forceinline__ bool give_up(u64 t0, const char* ws, u32 call) {
  const u64 dt = now() - t0;
  if (!__builtin_amdgcn_readfirstlane((int)(dt > 50000ull))) return false;
  __builtin_amdgcn_s_sleep(64);
  if (__builtin_amdgcn_readfirstlane((int)(dt > DEADLINE))) return true;
  return __builtin_amdgcn_readfirstlane(ld_sys(ws + TMO_OFF)) == call;
}
__device__ __forceinline__ void sleep_until(u64 tstart, u32 t) {
  for (;;) {
    const u32 el = (u32)__builtin_amdgcn_readfirstlane((u32)(now() - tstart));
    if (el >= t) break;
    if (t - el > 40u) __builtin_amdgcn_s_sleep(12); else __builtin_amdgcn_s_sleep(1);
  }
}
__device__ __forceinline__ void sleep_exact(u64 tstart, u32 t) {
  for (;;) {
    const u32 el = (u32)__builtin_amdgcn_readfirstlane((u32)(now() - tstart));
    if (el >= t) break;
    if (t - el > 60u) __builtin_amdgcn_s_sleep(12);
    else if (t - el > 20u) __builtin_amdgcn_s_sleep(3);
    else if (t - el > (u32)SXT) __builtin_amdgcn_s_sleep(1);
  }
}
template <int IMM> __device__ __forceinline__ void st16(u32 off, u32x4 v, char* base) {
  asm volatile("global_store_dwordx4 %0, %1, %2 offset:%3 sc0 sc1\n\ts_nop 1" :: "v"(off), "v"(v), "s"(base), "i"(IMM) : "memory");
}
template <int IMM> __device__ __forceinline__ void st16p(u32 off, u32x4 v, char* base) {
  asm volatile("global_store_dwordx4 %0, %1, %2 offset:%3\n\ts_nop 1" :: "v"(off), "v"(v), "s"(base), "i"(IMM) : "memory");
}
#define OST st16
__device__ __forceinline__ void dma16x4(u32 lds, const u32 (&o)[4], const char* base) {
  asm volatile("s_mov_b32 m0, %0\n\ts_nop 3\n\t"
               "global_load_lds_dwordx4 %1, %5 " MLP "\n\t"
               "global_load_lds_dwordx4 %2, %5 offset:1024 " MLP "\n\t"
               "global_load_lds_dwordx4 %3, %5 offset:2048 " MLP "\n\t"
               "global_load_lds_dwordx4 %4, %5 offset:3072 " MLP
               :: "s"(lds), "v"(o[0]), "v"(o[1]), "v"(o[2]), "v"(o[3]), "s"(base) : "memory");
}
__device__ __forceinline__ void probe(u32 off, char* base) {
  asm volatile("s_nop 4\n\tglobal_atomic_add %0, %1, %2 sc1" :: "v"(off), "v"(0u), "s"(base) : "memory");
}
__device__ __forceinline__ void probe0(u32 off, char* base) {
  asm volatile("s_mov_b64 exec, 1\n\ts_nop 3\n\tglobal_atomic_add %0, %1, %2 sc1\n\ts_mov_b64 exec, -1"
               :: "v"(off), "v"(0u), "s"(base) : "memory");
}
__device__ __forceinline__ u32 atom_rd(u32 off, const char* base) {
  u32 r;
  asm volatile("global_atomic_add %0, %1, %2, %3 sc0 sc1\n\ts_waitcnt vmcnt(0)" : "=&v"(r) : "v"(off), "v"(0u), "s"(base) : "memory");
  return r;
}
__device__ __forceinline__ u32 bump_issue(u32 off, const char* base) {
  u32 r;
  asm volatile("s_mov_b64 exec, 1\n\ts_nop 3\n\tglobal_atomic_add %0, %1, %2, %3 sc0 sc1\n\ts_mov_b64 exec, -1"
               : "=&v"(r) : "v"(off), "v"(1u), "s"(base) : "memory");
  return r;
}
template <int N> __device__ __forceinline__ u32 bump_get(u32 r) {
  u32 v;
  asm volatile("s_waitcnt vmcnt(%2)\n\tv_readfirstlane_b32 %0, %1\n\ts_nop 4" : "=s"(v) : "v"(r), "i"(N) : "memory");
  return v + 1u;
}
__device__ __forceinline__ void add1_lane0(u32 off, char* base) {
  asm volatile("s_mov_b64 exec, 1\n\ts_nop 3\n\tglobal_atomic_add %0, %1, %2 sc1\n\ts_mov_b64 exec, -1"
               :: "v"(off), "v"(1u), "s"(base) : "memory");
}
__device__ __forceinline__ void st4_lane0(u32 off, u32 val, char* base) {
  asm volatile("s_mov_b64 exec, 1\n\ts_nop 3\n\tglobal_store_dword %0, %1, %2 sc0 sc1\n\ts_mov_b64 exec, -1"
               :: "v"(off), "v"(val), "s"(base) : "memory");
}

#define STAMP(ws, idx) do {} while (0)
#define STAMPV(ws, idx, v) do {} while (0)

__device__ __forceinline__ void piece_of(int q, int d, bool& isx, int& idx) {
  if (WE) q = q == NS + 10 ? NS + 12 : ((u32)(q - NS - 11) < 2u ? q - 1 : q);
  const int t = q - NS, u = q - NS - 12;
  int j = (t >> 1) + 2;
  j += j >= d ? 1 : 0;
  j = j == 8 ? 1 : j;
  isx = (q >= NS && u < 0) || (u >= OX && u < OX + 2);
  idx = q < NS ? q : (u < 0 ? 2 * j + (t & 1) : (u < OX ? NS + u : (u < OX + 2 ? u - OX : NS + u - 2)));
}
__device__ __forceinline__ int slot_w(int k) { return k < NS ? k : (WE && k == NS ? 10 + k : (k - NS < OX ? 12 + k : 14 + k)); }
__device__ __forceinline__ int slot_x(int j, int e, int d) {
  return j == 0 ? NS + 12 + OX + e : (j == 1 ? NS + 10 + WE : NS + 2 * (j - 2 - (j > d ? 1 : 0))) + e;
}
__device__ __forceinline__ int cont_w(int k) { return k < NS ? k : 14 + k; }
__device__ __forceinline__ int cont_x(int rb) { return rb < 2 ? 30 + rb : (rb < 4 ? NS + 10 + rb : NS - 4 + rb); }
__device__ __forceinline__ u32 fin_off(int sl, int q) { return (u32)sl * IN_SRC + (u32)(NS + 2 * sl) * 2u * PIECE + (u32)q * 64u; }
__device__ __forceinline__ u32 lds_rd(u32 a) { return __builtin_amdgcn_readfirstlane(*(volatile LDSP u32*)(uintptr_t)a); }
__device__ __forceinline__ void lds_wr(u32 a, u32 v) { *(volatile LDSP u32*)(uintptr_t)a = v; }

__device__ __forceinline__ void mover(const Args& P, int d, int wave, u32 lane, char* smem, u64 tstart) {
  const int s = P.rank;
  char* const myws = P.my;
  const int pair = wave >> 1;
  const int p = ((blockIdx.x + PROT) & 7) * 2 + pair;
  const u32 r = (u32)(s ^ d) & (u32)(NR - 1);
  const int nin = d == 1 ? 0 : NIN;
  const bool less = NX && (d == 1 ? p >= NXL : p < 2 * NX);
  const int nid = d == 1 ? NOUT1 + NXT : NOUT;
  const int np = nin + nid - (less ? 1 : 0);
  const u32 ring = (u32)(uintptr_t)smem + (u32)pair * 32768u;
  const u32 ctl = (u32)(uintptr_t)smem + 65536u + (u32)pair * 512u;
  const char* const xin = P.x;
  const char* const win = P.w;
  const int rg = r >> 2, rb_ = (r >> 1) & 1, ra = r & 1;
  u32 wo[4];
#pragma unroll
  for (int k = 0; k < 4; ++k) {
    const u32 q4 = lane >> 4;
    wo[k] = ((u32)p * 16u + (u32)k * 4u + q4) * 8192u + (((lane & 15u) ^ ((q4 << 2) | (u32)k)) * 16u) - (u32)k * 1024u;
  }
  constexpr u32 XC = 256u / XR;
  const u32 xrow = ((u32)p % (128u / XR)) * XR, xks = (u32)p / (128u / XR) * (XC / 4u);
#define W_BASE(IX) (win + (size_t)((u32)ra * 256u * 8192u + (u32)rg * 4096u + (u32)(ra ? 15 - (IX) : (IX)) * 256u))

  if ((wave & 1) == 0) {
    const u32 cnt = bump_issue(CNT_OFF + (u32)(blockIdx.x * 4 + wave) * 64u, myws);
    u32 xo[4], oo[4];
    {
#pragma unroll
      for (int k = 0; k < 4; ++k) {
        const u32 c = (u32)k * 64u + lane;
        xo[k] = (xrow + c / XC) * 1024u + xks * 64u + (c % XC) * 16u - (u32)k * 1024u;
        oo[k] = lane * 16u;
      }
    }
    u32 poll_off, src_off = 0, dst_off;
#define ISSUE_LOADS(QQ, II) do { \
      const int q_ = (QQ); \
      const u32 dst_ = ring + (u32)(q_ & 7) * 4096u; \
      if (q_ < nin) { \
        bool isx_; int ix_; \
        piece_of(q_, d, isx_, ix_); \
        if (isx_) dma16x4(dst_, xo, xin + (size_t)(((r ^ (u32)(ix_ >> 1)) * 512u + (u32)rb_ * 256u + (u32)(ix_ & 1) * 128u) * 1024u + (u32)ra * 512u)); \
        else      dma16x4(dst_, wo, W_BASE(ix_)); \
      } else { \
        dma16x4(dst_, oo, myws + (size_t)(u32)__builtin_amdgcn_readlane((int)src_off, (II))); \
      } \
    } while (0)

    int q0 = 0;
    u32 pub = 0, call;
    u32 dbg1 = 0, dbg2 = 0;
    if (nin) {
      ISSUE_LOADS(1, 0); ISSUE_LOADS(2, 0); ISSUE_LOADS(3, 0);
    }
    if (nin) {
      call = bump_get<8>(cnt);
      lds_wr(ctl, 2u);
      q0 = 4; pub = 2;
    } else {
      call = bump_get<0>(cnt);
    }

    {
      u32 ln = lane;
      asm volatile("" : "+v"(ln));
      const u32 i0 = ln < (u32)nid ? ln : 0u, up = (u32)p;
      const u32 nsw = d == 1 ? SWP : 0, nex = d == 1 ? NXT : 0;
      const bool sw = i0 < nsw, ex = !sw && i0 < nsw + nex;
      const u32 z = min((i0 - nsw) * 16u + up, (u32)(12 * NX - 1)), zj = z / (u32)(NX ? 2 * NX : 1);
      const u32 i = sw ? (i0 & ~1u) | (up >> 3) : (ex ? i0 + (u32)(NOUT1 - SWP) : i0 - nex);
      const u32 pd = sw ? ((i0 & 1u) << 3) | (up & 7u) : up;
      const u32 ps = ex ? z - zj * (u32)(2 * NX) : pd;
      const u32 jk = d != 1 ? (u32)((d - 1) * 16 + NS) + i
                            : (ex ? zj * 16u + (u32)(16 + NS) : (i < 6u * NS ? 16u + i / NS * 16u + i % NS : i - 6u * NS));
      poll_off = FLOC_OFF + ((jk >> 4) * 8u + (ps >> 1)) * 128u + (jk & 15u) * 4u;
      src_off = POUT_OFF + jk * PIECE + ps * 4096u;
      dst_off = i * PIECE + pd * 4096u;
    }
    if (!nin) sleep_until(tstart, (u32)(TQ(NS + 2) + L1T));
    u64 ready = 0;
    u64 todo = ((1ull << nid) - 1ull) & ~(less ? 1ull << (d == 1 ? SWP + NXT - 1 : 0) : 0ull);
    u32 cons = 0;
    bool dead = false;
    for (int q = q0; q < np; ++q) {
      bool rdy = q < nin || (ready & todo) != 0ull;
      if (cons + 8u <= (u32)q) cons = lds_rd(ctl + 4u);
      if (!rdy || cons + 8u <= (u32)q) {
        if (!OVP || rdy) waitv<0>();
        pub = (u32)q;
        if (!OVP || rdy) lds_wr(ctl, pub);
        const u64 t0 = now();
        for (;;) {
          if (!rdy) {
            u32 pv = 0;
            if (OVP) {
              asm volatile("s_mov_b64 exec, %4\n\ts_nop 3\n\t"
                           "global_atomic_add %0, %1, %2, %3 sc0 sc1\n\t"
                           "s_mov_b64 exec, -1\n\t"
                           "s_waitcnt vmcnt(1)\n\t"
                           "ds_write_b32 %5, %6\n\t"
                           "s_waitcnt vmcnt(0)"
                           : "+&v"(pv) : "v"(poll_off), "v"(0u), "s"(myws), "s"(todo), "v"(ctl), "v"(pub) : "memory");
            } else if ((todo >> lane) & 1ull) pv = atom_rd(poll_off, myws);
            ready |= __builtin_amdgcn_ballot_w64(pv >= call);
            rdy = (ready & todo) != 0ull;
            if (LSL && !rdy && nin && cons + (u32)LSQ <= (u32)q) __builtin_amdgcn_s_sleep(LSL);
          } else {
            __builtin_amdgcn_s_sleep(2);
          }
          cons = lds_rd(ctl + 4u);
          if (rdy && cons + 8u > (u32)q) break;
          if (give_up(t0, myws, call)) { dead = true; break; }
        }
        if (dead) break;
      }
      int id = 0;
      if (q >= nin) {
        id = __builtin_ctzll(ready & todo);
        todo &= ~(1ull << id);
        lds_wr(ctl + 32u + (u32)(q & 7) * 4u, (u32)__builtin_amdgcn_readlane((int)dst_off, id));
      }
      ISSUE_LOADS(q, id);
      waitv<8>();
      if (PWHO == 1 && nin && q == PQ1) probe0((u32)(d - 2) * IN_SRC + 2u * MIB + (u32)p * 4096u, P.peer);
      if (PWHO == 1 && nin && q == PQ2) probe0(PIN_OFF + (call % 3u) * PIN_PAR + (u32)(32 + (d - 2) * 16 + NS) * PIECE + (u32)p * 4096u, P.peer);
      if (q - 1 > (int)pub) { pub = (u32)(q - 1); lds_wr(ctl, pub); }
      if (q == nin - 1) STAMP(myws, 1);
    }
    waitv<0>();
    if (dead) st4_lane0(TMO_OFF, call, myws);
    lds_wr(ctl, dead ? DEAD : (u32)np);
    STAMP(myws, 3);
    STAMPV(myws, 0, tstart);
    return;
  }

  char* const rws = P.peer;
  const int slot = d - 2;
  const u32 dl = (u32)p * 4096u + lane * 16u;
  u32 xl[4], xd[4];
#pragma unroll
  for (int k = 0; k < 4; ++k) {
    const u32 c = (u32)k * 64u + lane, t = c / (4u * XR), row = (c >> 2) % XR;
    xl[k] = (row * XC + t * 4u + ((c ^ ((xrow + row) >> 2)) & 3u)) * 16u;
    xd[k] = XCT ? dl : (xks + t) * 8192u + (xrow + row) * 64u + (c & 3u) * 16u - (u32)k * 1024u;
  }
  u32 tick, ts0, ta;
  int to;
  {
    u32 tg, tm;
    asm volatile("" : "+v"(xl[0]), "+v"(xl[1]), "+v"(xl[2]), "+v"(xl[3]), "+v"(xd[0]), "+v"(xd[1]), "+v"(xd[2]), "+v"(xd[3]));
    asm volatile("s_waitcnt vmcnt(1)\n\t"
                 "v_readfirstlane_b32 %[tick], v160\n\t"
                 "v_readfirstlane_b32 %[tg], v161\n\t"
                 "s_memrealtime s[58:59]\n\t"
                 "s_sub_i32 %[tm], %[tg], %[tis]\n\t"
                 "s_addk_i32 %[tm], 500\n\t"
                 "s_cmpk_gt_u32 %[tm], 1000\n\t"
                 "s_cselect_b32 %[tg], %[tis], %[tg]\n\t"
                 "s_addk_i32 %[tg], %[oc]\n\t"
                 "s_sub_i32 %[to], %[tg], %[tst]\n\t"
                 "s_and_b32 %[tm], %[tick], 15\n\t"
                 "s_mul_i32 %[tm], %[tm], %[osp]\n\t"
                 "s_add_i32 %[tg], %[tg], %[tm]\n\t"
                 "s_bitcmp0_b32 %[tick], 4\n\t"
                 "s_cselect_b32 %[tm], 0x10000, 0\n\t"
                 "v_add_u32 v160, %[tm], %[dl]\n\t"
                 "s_cmp_eq_u32 %[d], 1\n\t"
                 "s_waitcnt lgkmcnt(0)\n\t"
                 "s_mov_b32 %[ta], s58\n\t"
                 "s_cbranch_scc1 5f\n"
                 "4:\n\t"
                 "s_waitcnt lgkmcnt(0)\n\t"
                 "s_sub_i32 %[tm], s58, %[tg]\n\t"
                 "s_mov_b32 %[ts0], s58\n\t"
                 "s_memrealtime s[58:59]\n\t"
                 "s_cmp_lt_i32 %[tm], 0\n\t"
                 "s_cbranch_scc1 4b\n\t"
                 "global_store_dwordx4 v160, v[144:147], %[dst] sc0 sc1\n\t"
                 "global_store_dwordx4 v160, v[148:151], %[dst] offset:1024 sc0 sc1\n\t"
                 "global_store_dwordx4 v160, v[152:155], %[dst] offset:2048 sc0 sc1\n\t"
                 "global_store_dwordx4 v160, v[156:159], %[dst] offset:3072 sc0 sc1\n"
                 "5:\n\t"
                 "s_waitcnt lgkmcnt(0)"
                 : [tick] "=&s"(tick), [ts0] "=&s"(ts0), [ta] "=&s"(ta), [to] "=&s"(to), [tg] "=&s"(tg), [tm] "=&s"(tm)
                 : "{v[144:159]}"(P.f16), "{v[160:161]}"(P.co), [tis] "s"(P.tis), [tst] "s"((u32)tstart), [d] "s"(d),
                   [dl] "v"(dl), [dst] "s"(rws + (size_t)((u32)max(slot, 0) * IN_SRC)), [oc] "i"(OC), [osp] "i"(OSP)
                 : "memory", "scc", "s58", "s59");
  }
  const u32 call = (tick >> 4) + 1u;
  u32 dbg1 = 0, dbg2 = 0;
  const u32 in0 = (u32)slot * IN_SRC + (call & 1u) * PIECE;
  const u32 pin0 = PIN_OFF + (call % 3u) * PIN_PAR + (u32)(d == 1 ? 0 : 32 + slot * 16 + NS) * PIECE;
  int nflag = 0;
#define FLAG(QQ) do { \
    const int q_ = (QQ); \
    const bool real_ = q_ == nflag && q_ < nin; \
    const u32 o_ = real_ ? fin_off(slot, q_) : DUMMY_OFF + (u32)(blockIdx.x * 4 + wave) * 128u; \
    add1_lane0(o_, uni(real_ ? rws : myws)); \
    nflag += real_ ? 1 : 0; \
  } while (0)

  u32 tgrid = 0, tord = 0;
  int qsw = 0;
  if (nin) {
    FLAG(-1);
    const int cold = ta - P.tis > (u32)CLT || P.tis - (u32)tstart > (u32)CKT ? CLD : 0;
    const u32 tb = (u32)max(to + TAU + OG + cold, (int)(ts0 - (u32)tstart) + OD) + (u32)(TAU - 1);
    tgrid = tb - ((u32)tstart + tb - ((u32)(((p + STGD * d) & 15) * TAU) >> 4)) % (u32)TAU;
    qsw = (int)tgrid - (to + TAU + OG + cold) < ((p + 1) * TAU >> 4) ? NIN + 1 : NIN;
    tord = ORD > 1 ? (u32)max(to + TAU + OG + (int)(tick & 15u) * OSP, 0) - tgrid : 0u;
  } else {
    sleep_until(tstart, 600u);
    probe0(pin0 + (u32)p * 4096u, rws);
    probe0(pin0 + (u32)(NOUT1 + NXT - 1) * PIECE + (u32)p * 4096u, rws);
  }
  u32 full = 0, tnext = 0;
  for (int q = nin ? 1 : 0; q < np; ++q) {
    bool isx; int ix;
    piece_of(q, d, isx, ix);
    isx = isx && q < nin;
    if (d != 1 || G1) tnext = max(tnext, tgrid + (q < ORD ? tord : 0u));
    if (!PRE) sleep_until(tstart, tnext);
    if (full <= (u32)q) {
      full = lds_rd(ctl);
      if (full <= (u32)q) {
        bool drained = !(q > 0 && q < nin + LAG);
        u64 el;
        for (;;) {
          __builtin_amdgcn_s_sleep(1);
          full = lds_rd(ctl);
          el = now() - tstart;
          if (full > (u32)q) break;
          if (!drained && __builtin_amdgcn_readfirstlane((int)(el > (u64)(tnext + 30u)))) {
            drained = true;
            waitv<0>();
#pragma unroll
            for (int z = LAG; z >= 1; --z) FLAG(q - z);
          }
          if (__builtin_amdgcn_readfirstlane((int)(el > 2ull * DEADLINE - 1000ull))) { full = DEAD; break; }
        }
        tnext = max(tnext, (u32)__builtin_amdgcn_readfirstlane((u32)el));
        if (RSLOT && q >= nin && nin) {
          while (tnext > tgrid + (u32)RSLOT) tgrid += (u32)(TAU - DB);
          tnext = max(tnext, tgrid);
        }
      }
      if (full == DEAD) break;
    }
    const u32 la = ring + (u32)(q & 7) * 4096u;
    const u32x4 d0 = *(volatile LDSP u32x4*)(uintptr_t)(la + (isx ? xl[0] : lane * 16u));
    const u32x4 d1 = *(volatile LDSP u32x4*)(uintptr_t)(la + (isx ? xl[1] : lane * 16u + 1024u));
    const u32x4 d2 = *(volatile LDSP u32x4*)(uintptr_t)(la + (isx ? xl[2] : lane * 16u + 2048u));
    const u32x4 d3 = *(volatile LDSP u32x4*)(uintptr_t)(la + (isx ? xl[3] : lane * 16u + 3072u));
    const u32 id = lds_rd(ctl + 32u + (u32)(q & 7) * 4u);
    lds_wr(ctl + 4u, (u32)q + 1u);
    u32 o;
    if (q < nin) {
      o = in0 + (u32)(isx ? cont_x(ix) : cont_w(ix)) * 2u * PIECE;
    } else {
      o = pin0 + min(id, (u32)nid * PIECE - 4096u) - (u32)p * 4096u;
    }
    char* const dst = rws + (size_t)o;
    if (G1 && q == 0) {
      const u32 tb = tnext + (u32)(G1M + TAU1 - 1);
      tnext = tgrid = tb - ((u32)tstart + tb - ((u32)(p * TAU1) >> 4)) % (u32)TAU1;
    }
    if (PRE) sleep_exact(tstart, tnext);
    if (G1 && d == 1) tgrid += q < G1Q && lds_rd(ctl) > (u32)q + 1u ? 0u : (u32)TAU1;
    else tgrid += (u32)(TAU - (q < QA ? DA : 0) - (q >= QB ? DB : 0) - (q >= QC ? DC : 0));
    if (NX && q == qsw && d != 1) tgrid += (u32)((less ? TAU - DB : 0) - (2 * NX * TAU >> 4));
    tnext += (u32)(d == 1 && !G1 ? TAU1 + (q < SWP ? 0 : (less ? XLB : -XLA)) : (q > NIN + 1 ? TAU2 : 0));
    st16<0>(isx ? xd[0] : dl, d0, dst);
    st16<1024>(isx ? xd[1] : dl, d1, dst);
    st16<2048>(isx ? xd[2] : dl, d2, dst);
    st16<3072>(isx ? xd[3] : dl, d3, dst);
    if (nin && q < nin + LAG) {
      waitv<5 * LAG>();
      FLAG(q - LAG);
      if (PWHO == 0 && q == PQ1) probe0((u32)slot * IN_SRC + 2u * MIB + (u32)p * 4096u, rws);
      if (PWHO == 0 && q == PQ2) probe0(pin0 + (u32)p * 4096u, rws);
    } else {
      waitv<4 * WIN>();
    }
    if (q == NS + 11) STAMP(myws, 1);
    if (q == nin - 1) STAMP(myws, 2);
    if (q == np - 1) STAMP(myws, 4);
  }
  if ((tick & 15u) == 0u) st4_lane0(LNK_OFF + (u32)d * 128u + 4u, 0u, myws);
  waitv<0>();
  STAMP(myws, 3);
  STAMPV(myws, 0, tstart);
}

struct Frag {
  bf16x8 x[2][2];
  bf16x8 w[2][2];
};

struct Loader {
  const char* base;
  u32 stride[2];
  u32 v[4];
  u32 m0base;
};

template <int STAGE> __device__ __forceinline__ void glds_pair(const Loader& L, int half) {
  const u32 m0v = L.m0base + STAGE * STAGE_B;
  if (half == 0)
    asm volatile("s_mov_b32 m0, %0\n\ts_nop 0\n\t"
                 "global_load_lds_dwordx4 %1, %3 " GLP "\n\t"
                 "global_load_lds_dwordx4 %2, %3 offset:1024 " GLP
                 :: "s"(m0v), "v"(L.v[0]), "v"(L.v[1]), "s"(L.base) : "memory");
  else
    asm volatile("s_mov_b32 m0, %0\n\ts_nop 0\n\t"
                 "global_load_lds_dwordx4 %1, %3 offset:2048 " GLP "\n\t"
                 "global_load_lds_dwordx4 %2, %3 offset:3072 " GLP
                 :: "s"(m0v), "v"(L.v[2]), "v"(L.v[3]), "s"(L.base) : "memory");
}

struct RdAddr { u32 xa[2]; u32 wa[2][2]; };

template <int STAGE> __device__ __forceinline__ bf16x8 rd_x(const RdAddr& A, int mt, int ks) {
  return *(const LDSP bf16x8*)(uintptr_t)(A.xa[ks] + (u32)(STAGE * STAGE_B + mt * 2048));
}
template <int STAGE> __device__ __forceinline__ bf16x8 rd_w(const RdAddr& A, int nt, int ks) {
  s16x4 lo = __builtin_amdgcn_ds_read_tr16_b64_v4i16((LDSP s16x4*)(uintptr_t)(A.wa[nt][0] + (u32)(STAGE * STAGE_B + 8192 + ks * 4096)));
  s16x4 hi = __builtin_amdgcn_ds_read_tr16_b64_v4i16((LDSP s16x4*)(uintptr_t)(A.wa[nt][1] + (u32)(STAGE * STAGE_B + 8192 + ks * 4096)));
  s16x8 v = __builtin_shufflevector(lo, hi, 0, 1, 2, 3, 4, 5, 6, 7);
  return __builtin_bit_cast(bf16x8, v);
}
#define MFMA(nt, mt, ks) acc[nt][mt] = __builtin_amdgcn_mfma_f32_32x32x16_bf16(cur.w[nt][ks], cur.x[mt][ks], acc[nt][mt], 0, 0, 0)

template <int S, int VM, bool LOAD, bool READ>
__device__ __forceinline__ void kstep(f32x16 (&acc)[2][2], const Frag& cur, Frag& nxt, const RdAddr& A, Loader& L) {
  constexpr int SN = (S + 1) & 3;
  if (READ) {
    __builtin_amdgcn_s_waitcnt((VM & 15) | (7 << 4) | (0 << 8) | ((VM >> 4) << 14));
    asm volatile("s_barrier" ::: "memory");
  } else {
    __builtin_amdgcn_s_waitcnt(15 | (7 << 4) | (0 << 8) | (3 << 14));
  }
  SB();
  MFMA(0, 0, 0); SB();
  if (READ) { nxt.w[0][0] = rd_w<SN>(A, 0, 0); nxt.x[0][0] = rd_x<SN>(A, 0, 0); } SB();
  MFMA(0, 1, 0); SB();
  if (READ) { nxt.x[1][0] = rd_x<SN>(A, 1, 0); nxt.w[1][0] = rd_w<SN>(A, 1, 0); } SB();
  MFMA(1, 0, 0); SB();
  if (LOAD) glds_pair<S>(L, 0);
  SB();
  MFMA(1, 1, 0); SB();
  if (READ) { nxt.w[0][1] = rd_w<SN>(A, 0, 1); nxt.x[0][1] = rd_x<SN>(A, 0, 1); } SB();
  MFMA(0, 0, 1); SB();
  if (READ) { nxt.x[1][1] = rd_x<SN>(A, 1, 1); nxt.w[1][1] = rd_w<SN>(A, 1, 1); } SB();
  MFMA(0, 1, 1); SB();
  if (LOAD) { glds_pair<S>(L, 1); L.base += L.stride[S & 1]; }
  SB();
  MFMA(1, 0, 1); SB();
  MFMA(1, 1, 1); SB();
}

template <int S> __device__ __forceinline__ void stage_load(Loader& L) {
  glds_pair<S>(L, 0); glds_pair<S>(L, 1); L.base += L.stride[S & 1];
}

__device__ __forceinline__ void gemm_phase(f32x16 (&acc)[2][2], const RdAddr& A, Loader& L, int n, int sk, int jm, u32 jump, u32& dbgv, int dbg0) {
  Frag f0, f1;
  L.base = uni(L.base + (sk == 0 ? IN_SRC : 0u));
  L.stride[0] = __builtin_amdgcn_readfirstlane(L.stride[0]);
  L.stride[1] = __builtin_amdgcn_readfirstlane(L.stride[1]);
  asm volatile("s_nop 4" ::: "memory");
  stage_load<0>(L); stage_load<1>(L); stage_load<2>(L); stage_load<3>(L);
  waitv<12>();
  asm volatile("s_barrier" ::: "memory");
#pragma unroll
  for (int ks = 0; ks < 2; ++ks)
#pragma unroll
    for (int i = 0; i < 2; ++i) { f0.w[i][ks] = rd_w<0>(A, i, ks); f0.x[i][ks] = rd_x<0>(A, i, ks); }
  SB();
  for (int t = 0; t < n - 4; t += 4) {
    L.base += ((t | jm) & 4 ? jump : 0u) + (t + 4 == sk ? IN_SRC : 0u);
    kstep<0, 8, true, true>(acc, f0, f1, A, L);
    kstep<1, 8, true, true>(acc, f1, f0, A, L);
    kstep<2, 8, true, true>(acc, f0, f1, A, L);
    kstep<3, 8, true, true>(acc, f1, f0, A, L);
  }
  kstep<0, 8, false, true>(acc, f0, f1, A, L);
  kstep<1, 4, false, true>(acc, f1, f0, A, L);
  kstep<2, 0, false, true>(acc, f0, f1, A, L);
  kstep<3, 0, false, false>(acc, f1, f0, A, L);
}

__device__ __forceinline__ u32 pack2(float a, float b) {
  f32x2 t = {a, b};
  return __builtin_bit_cast(u32, __builtin_convertvector(t, bf16x2));
}
__device__ __forceinline__ u32 relu2(u32 v) {
  typedef short s16x2 __attribute__((ext_vector_type(2)));
  return __builtin_bit_cast(u32, __builtin_elementwise_max(__builtin_bit_cast(s16x2, v), s16x2{0, 0}));
}
__device__ __forceinline__ float bf_lo(u32 v) { return __builtin_bit_cast(float, v << 16); }
__device__ __forceinline__ float bf_hi(u32 v) { return __builtin_bit_cast(float, v & 0xffff0000u); }

__device__ __forceinline__ void zero8k(u32 l16, u32 l16b, char* pb) {
  const u32x4 z4 = {0u, 0u, 0u, 0u};
  asm volatile("s_nop 4" ::: "memory");
  st16<0>(l16, z4, pb); st16<1024>(l16, z4, pb); st16<2048>(l16, z4, pb); st16<3072>(l16, z4, pb);
  st16<0>(l16b, z4, pb); st16<1024>(l16b, z4, pb); st16<2048>(l16b, z4, pb); st16<3072>(l16b, z4, pb);
}

__device__ __forceinline__ void tile(const Args& P, int tid, int wave, u32 lane, char* smem, u64 tstart) {
  const int r = P.rank;
  char* const myws = P.my;
  const int g = r >> 2, b = (r >> 1) & 1, a = r & 1;
  const int bid = blockIdx.x;

  if (PF && (tid >> 6) == PF) {
    const int h = ((tid >> 3) & 7) * 4 + wave, u = (tid & 7) >> 1, kf = tid & 1;
    const char* const pb = h < 16 ? P.x + (size_t)((u32)((r ^ (2 + (u >> 1))) * 512 + b * 256 + (u & 1) * 128 + (h & 1) * 64) * 1024u + (u32)(h >> 1) * 128u)
                                  : P.w + (size_t)((u32)((h >> 1) - 8) * 64u * 8192u + (u32)(g * 16 + (a ? 15 - kf : kf)) * 256u + (u32)(h & 1) * 128u);
    if (PFD) sleep_until(tstart, PFD);
    asm volatile("s_mov_b32 m0, %0\n\ts_nop 3\n\tglobal_load_lds_dword %1, %2"
                 :: "s"((u32)(uintptr_t)smem + 66048u), "v"(lane << (h < 16 ? 10 : 13)), "s"(pb) : "memory");
  }
  const u32 cnt = bump_issue(CNT_OFF + (u32)(bid * 4 + wave) * 64u, myws);

  if (LPT && (bid >> 3) == 25 && wave == 0) {
    sleep_until(tstart, (u32)LPT);
    if (lane < 28u) probe(lane * 2u * MIB + MIB + (u32)(bid & 7) * 128u, myws);
  }
  if (RPT && (bid >> 3) == RPB && wave == 1) {
    const u32 dd = 1u + lane % 7u;
    const u64 pp = ((const GLBP u64*)(uintptr_t)__builtin_amdgcn_kernarg_segment_ptr())[6u + 2u * dd];
    const u32 cl = bump_get<0>(cnt);
    const u32 o = (lane < 7u && dd > 1u ? (dd - 2u) * IN_SRC + 2u * MIB : PIN_OFF + (cl % 3u) * PIN_PAR + (dd > 1u ? dd * 16u : 30u) * PIECE) + (u32)(bid & 7) * 128u;
    sleep_until(tstart, (u32)RPT);
    if (lane < 14u) asm volatile("global_atomic_add %0, %1, off sc1" :: "v"(pp + o), "v"(0u) : "memory");
  }
  static_assert(!SUB || NS == 2, "first tiles");
  bool dead = false;
  u32 dbgv = 0;
  int ra = P.rank;
#pragma unroll 1
  for (int job = 0; job < 2; ++job) {
  asm volatile("" : "+s"(tid), "+s"(wave), "+s"(ra), "+v"(lane));
  const int r = ra, g = r >> 2, b = (r >> 1) & 1, a = r & 1;
  const int hi = tid < N_MAIN_WG ? (tid >> 3) * 3 : (tid - N_MAIN_WG) >> 3;
  const int sq = tid < N_MAIN_WG ? 0 : 1 + hi % 3;
  const bool sub = SUB && job == 0 && hi < ((tid & 4) ? 12 : 9);
  int row0, col0, nrem, tskip, j, e, k, rb;
  int pi0 = -1, pi1 = -1;
  bool fin;
  u32 dz = 0;
  if (tid < N_MAIN_WG || sub) {
    const int xcd = tid & 7, u = xcd >> 1, o = HRM && tid >= N_MAIN_WG && u >= 2 ? (hi / 3 + 1) & 3 : hi / 3;
    int rbi, cbi;
    if (u < 2) { rbi = o < 24 ? 1 + o % 3 : 0; cbi = o < 24 ? o / 3 : o - 24; }
    else { rbi = o & 3; cbi = o >> 2; }
    rb = u + 4 * rbi; k = cbi * 2 + (xcd & 1);
    j = rb >> 1; e = rb & 1;
    row0 = (r ^ j) * 512 + b * 256 + e * 128; col0 = g * 2048 + (a ? 15 - k : k) * 128;
    nrem = (j >= 2 ? 40 : 48) >> (sub ? 1 : 0); tskip = j >= 2 ? (j - 2) * (sub ? 4 : 8) : 99;
    fin = j == 0;
    if (j == 0) pi1 = 6 * NS + 15 - k;
    else if (j >= 2 && k >= 16 - NS) pi0 = ((j ^ 1) - 2) * NS + 15 - k;
    else if (j >= 2 && k == 15 - NS && e * 4 + wave < NX) { pi0 = NOUT1; dz = (u32)(((j ^ 1) - 2) * NX) * 8192u; }
  } else {
    const int f = tid - N_MAIN_WG;
    const int qd = 1 + f / 32, rem = f & 31;
    e = (rem >> 3) & 1; k = (rem & 7) * 2 + (rem >> 4);
    const int h = b ^ (qd & 1), g2 = g ^ (qd >> 1);
    row0 = r * 512 + h * 256 + e * 128; col0 = g2 * 2048 + (a ? 15 - k : k) * 128;
    nrem = 0; tskip = 99; j = 0; rb = 0;
    fin = true;
    const int ns = e * 4 + wave < NX ? NS + 1 : NS;
    const int iA = k >= ns ? 32 + (2 * qd - 2) * 16 + k : -1;
    const int iB = 15 - k >= ns ? 32 + (2 * qd - 1) * 16 + 15 - k : -1;
    pi0 = k < 8 ? iA : iB;
    pi1 = k < 8 ? iB : iA;
  }

  const u32 sbase = (u32)(uintptr_t)smem;
  const int wm = wave >> 1, wn = wave & 1;
  const u32 hb = sub ? ((lane >> 3) ^ (lane >> 5)) & 1u : 0u;
  RdAddr A;
  {
    const u32 h = lane >> 5, m = lane & 31u, f = (lane >> 2) & 3u;
    const u32 xr = (u32)(sub ? wave : wm * 2) * 32u + m;
    A.xa[0] = sbase + xr * 64u + 16u * (h ^ f);
    A.xa[1] = sbase + xr * 64u + 16u * ((2u + h) ^ f);
    const u32 blk = (lane >> 4) & 1u, q = (lane & 15u) >> 2, pp = lane & 3u;
#pragma unroll
    for (int nt = 0; nt < 2; ++nt)
#pragma unroll
      for (int t = 0; t < 2; ++t) {
        const u32 row = 8u * h + 4u * t + q;
        const u32 ch = ((u32)(sub ? wm * 12 : wn * 8) ^ ((u32)nt * 4u)) + 2u * blk + (pp & 1u);
        A.wa[nt][t] = sbase + 256u * row + 16u * (ch ^ (((row & 3u) << 2) | ((row >> 2) & 3u))) + 8u * (pp >> 1);
      }
  }

  f32x16 acc[2][2];
#pragma unroll
  for (int i = 0; i < 2; ++i)
#pragma unroll
    for (int k = 0; k < 2; ++k)
#pragma unroll
      for (int z = 0; z < 16; ++z) acc[i][k][z] = 0.f;

  const u32 tile_off = (u32)e * 32768u + (sub ? (u32)(sq * 4 + wave) * 2048u : (u32)wave * 8192u);
  const u32 l16 = lane * 16u, l16b = lane * 16u + 4096u;
  const u32 tw = sbase + (u32)wave * 8192u + (lane & 31u) * 128u + (((lane >> 5) ^ (lane & 7u)) * 16u);
  const u32 tr = sbase + (u32)wave * 8192u + (lane >> 3) * 128u + (((lane & 7u) ^ (lane >> 3)) * 16u);
  const u32 oo = (lane >> 3) * 8192u + (lane & 7u) * 16u;
  char* const ob = uni(P.out + (size_t)((u32)((row0 & 511) + wm * 64) * 8192u + (u32)(col0 + wn * 64) * 2u));
  const int nph = nrem ? 2 : 1;
  const int qin = max(slot_w(k), slot_x(j, e, j == 7 ? 2 : 7));
  const u32 tin = (u32)(EIN + TQ(qin));
  const int addph = fin ? nph - 1 : 0;

  if (SPR) { if (sub || (PJ1 && j == 1)) __builtin_amdgcn_s_setprio(SPR); else __builtin_amdgcn_s_setprio(0); }
  if (PD0 && !sub && (u32)(tid - PDL) < (u32)(200 - PDL)) {
    const int h1 = (tid - 104) >> 3;
    sleep_until(tstart, (u32)(PD1 && (u32)(h1 - (!HRM && (tid & 4) ? 3 : 0)) < 3u ? PD1 : PD0));
  }
  Loader L;
  L.m0base = sbase + (u32)wave * 4096u;
  u32 call = 0;
  for (int ph = 0; ph < nph; ++ph) {
    int n, ts;
    bool skip = false;
    if (ph == 0) {
      n = sub ? 8 : 16; ts = 99;
      if (wave < 2) {
        L.base = P.x + (size_t)((u32)row0 * 1024u + (sub ? (u32)(sq >> 1) * 65536u + (u32)wave * 64u : (u32)wave * 65536u));
        L.stride[0] = L.stride[1] = sub ? 128u : 64u;
        const u32 c = (lane & 3u) ^ ((lane >> 4) & 3u);
#pragma unroll
        for (int k = 0; k < 4; ++k) L.v[k] = (lane >> 2) * 1024u + c * 16u + (u32)k * (16384u - 1024u);
      } else {
        L.base = P.w + (size_t)((u32)(wave - 2) * 16u * 8192u + (u32)col0 * 2u + (sub ? (u32)(sq & 1) * 128u : 0u));
        L.stride[0] = L.stride[1] = sub ? 64u * 8192u : 32u * 8192u;
        const u32 q = lane >> 4;
#pragma unroll
        for (int k = 0; k < 4; ++k)
          L.v[k] = q * 8192u + (((lane & 15u) ^ ((q << 2) | (u32)k)) * 16u) + (u32)k * (4u * 8192u - 1024u) + hb * (32u * 8192u - 128u);
      }
    } else {
      n = nrem; ts = tskip;
      STAMP(myws, 1);
      if (wave == 0) {
        const int sl = (int)min(lane, 5u);
        const u32 fo = fin_off(sl, max(slot_w(k), slot_x(j, e, sl + 2)));
        bool pend = lane < 6u && sl + 2 != j;
        sleep_until(tstart, tin);
        const u64 t0 = now();
        u32 st = 0;
        for (;;) {
          if (pend) pend = atom_rd(fo, myws) < 16u * call;
          if (__builtin_amdgcn_ballot_w64(pend) == 0ull) break;
          if (give_up(t0, myws, call)) {
            if (lane == 0) st_agt(myws + TMO_OFF, call);
            st = 1;
            break;
          }
          __builtin_amdgcn_s_sleep(PSL);
          if (PSL2 && !sub && j != 1) __builtin_amdgcn_s_sleep(PSL2);
        }
        if (lane == 0) *(LDSP u32*)(uintptr_t)(sbase + 4u * STAGE_B) = st;
        __builtin_amdgcn_s_waitcnt(0);
      }
      asm volatile("s_barrier" ::: "memory");
      const bool wgdead = lds_rd(sbase + 4u * STAGE_B) != 0;
      STAMP(myws, 2);
      dead = dead || wgdead;
      skip = wgdead;
      const u32 par = call & 1u;
      static_assert(!XCT, "layout of the x pieces");
      L.base = myws + (size_t)(((u32)(wave < 2 ? cont_x(rb) : cont_w(k)) * 2u + par) * PIECE
                               + (u32)(sub && wave < 2 ? (sq >> 1) + wave * 2 : wave & 1) * 4096u);
      L.stride[0] = L.stride[1] = sub ? 16384u : 8192u;
#pragma unroll
      for (int k = 0; k < 4; ++k) L.v[k] = sub && wave >= 2 ? (l16 ^ ((hb ^ (u32)(sq & 1)) << 7)) + hb * 8192u : l16;
    }
    if (!skip) gemm_phase(acc, A, L, n, ts, sub ? 4 : 0, ph ? IN_SRC - PIECE : 0u, dbgv, ph * 8);
    if (ph == nph - 1) STAMP(myws, 3);
    if (ph == 0) {
      call = bump_get<0>(cnt);
      if (ZT0 && max(pi0, pi1) >= 0) {
        char* const pz = myws + (size_t)(PIN_OFF + ((call + 2u) % 3u) * PIN_PAR + tile_off + dz);
        sleep_until(tstart, (u32)ZT0 + (u32)((tid >> 3) & 15) * (u32)ZDT);
        if (pi0 >= 0) zero8k(l16, l16b, uni(pz + (size_t)((u32)pi0 * PIECE)));
        if (pi1 >= 0) zero8k(l16, l16b, uni(pz + (size_t)((u32)pi1 * PIECE)));
      }
    }
    const u32 pin = PIN_OFF + (call % 3u) * PIN_PAR + tile_off + dz;
#pragma unroll 1
    for (int s = 0; s < 2; ++s) {
      const int pi = ph != addph ? -1 : (s == 0 ? pi0 : pi1);
      if (pi < 0 && !(fin && ph == 0 && s == 0)) continue;
      u32x4 pv[8];
#pragma unroll
      for (int i = 0; i < 8; ++i) pv[i] = u32x4{0u, 0u, 0u, 0u};
      if (pi >= 0) {
        char* const pb = uni(myws + (size_t)(pin + (u32)pi * PIECE));
        const bool first = pi == 0 || (pi >= 32 && (pi & 15) == NS);
        const u32 wo = pin + (u32)(pi - (((lane & 2u) && !first) ? 1 : 0)) * PIECE + (lane & 1u) * 4096u + 4092u;
        const bool lazy = pi >= 32 ? (pi & 15) < LZ : (!fin || pi < NOUT1 - LZ1);
        sleep_until(tstart, pi >= 32 ? (u32)(EARLY + TQ(NIN - 2 + (pi & 15) - NS) + (lazy ? LZD : 0)) : (fin ? 0u : tin - (u32)PAT));
        const u64 t0 = now();
        while (!dead) {
          u32 v = 0u;
          if (lane < (lazy ? 2u : 4u)) v = atom_rd(wo, myws);
          if (__builtin_amdgcn_ballot_w64(v != 0u) >= (lazy ? 3ull : 1ull)) break;
          if (give_up(t0, myws, call)) { dead = true; break; }
          if (lazy) __builtin_amdgcn_s_sleep(LZS);
          __builtin_amdgcn_s_sleep(WSL);
        }
        if (s == 1) STAMP(myws, 4);
        for (;;) {
          if (__builtin_expect(dead, 0)) {
            if (lane == 0) st_agt(myws + TMO_OFF, call);
#pragma unroll
            for (int i = 0; i < 8; ++i) pv[i] = u32x4{0x7fc07fc0u, 0x7fc07fc0u, 0x7fc07fc0u, 0x7fc07fc0u};
            break;
          }
          asm volatile("s_nop 4\n\t"
                       "global_load_dwordx4 %0, %8, %10 sc0 sc1\n\t"
                       "global_load_dwordx4 %1, %8, %10 offset:1024 sc0 sc1\n\t"
                       "global_load_dwordx4 %2, %8, %10 offset:2048 sc0 sc1\n\t"
                       "global_load_dwordx4 %3, %8, %10 offset:3072 sc0 sc1\n\t"
                       "global_load_dwordx4 %4, %9, %10 sc0 sc1\n\t"
                       "global_load_dwordx4 %5, %9, %10 offset:1024 sc0 sc1\n\t"
                       "global_load_dwordx4 %6, %9, %10 offset:2048 sc0 sc1\n\t"
                       "global_load_dwordx4 %7, %9, %10 offset:3072 sc0 sc1"
                       : "=&v"(pv[0]), "=&v"(pv[1]), "=&v"(pv[2]), "=&v"(pv[3]), "=&v"(pv[4]), "=&v"(pv[5]), "=&v"(pv[6]), "=&v"(pv[7])
                       : "v"(l16), "v"(l16b), "s"(pb) : "memory");
          const bool late = __builtin_amdgcn_readfirstlane((int)(now() - t0 > 50000ull)) != 0;
          asm volatile("s_waitcnt vmcnt(0)"
                       : "+v"(pv[0]), "+v"(pv[1]), "+v"(pv[2]), "+v"(pv[3]), "+v"(pv[4]), "+v"(pv[5]), "+v"(pv[6]), "+v"(pv[7]) :: "memory");
          u32 mn = 0xffffffffu;
#pragma unroll
          for (int i = 0; i < 8; ++i)
#pragma unroll
            for (int z = 0; z < 4; ++z) mn = min(mn, pv[i][z]);
          if (__builtin_amdgcn_ballot_w64(mn == 0u) == 0ull) break;
          if (__builtin_expect(late, 0)) dead = give_up(t0, myws, call);
        }
      }
#pragma unroll
      for (int nt = 0; nt < 2; ++nt)
#pragma unroll
        for (int mt = 0; mt < 2; ++mt)
#pragma unroll
          for (int sx = 0; sx < 2; ++sx)
#pragma unroll
            for (int z = 0; z < 4; ++z) {
              const u32 u = pv[(nt * 2 + mt) * 2 + sx][z];
              acc[nt][mt][8 * sx + 2 * z] += bf_lo(u);
              acc[nt][mt][8 * sx + 2 * z + 1] += bf_hi(u);
            }
      if (!fin) continue;
      if (s == 1) STAMP(myws, 6);
#pragma unroll
      for (int nt = 0; nt < 2; ++nt)
#pragma unroll
        for (int mt = 0; mt < 2; ++mt)
#pragma unroll
          for (int sx = 0; sx < 2; ++sx) {
            u32x4 v;
#pragma unroll
            for (int z = 0; z < 4; ++z) v[z] = relu2(pack2(acc[nt][mt][8 * sx + 2 * z], acc[nt][mt][8 * sx + 2 * z + 1]));
            *(volatile LDSP u32x4*)(uintptr_t)((tw ^ (u32)((nt * 4 + sx * 2) * 16)) + (u32)mt * 4096u) = v;
          }
      u32 o1, o2, o3, o4, o5, o6, o7;
      asm volatile("ds_read_b128 %0, %15\n\t"
                   "ds_read_b128 %1, %15 offset:1024\n\t"
                   "ds_read_b128 %2, %15 offset:2048\n\t"
                   "ds_read_b128 %3, %15 offset:3072\n\t"
                   "ds_read_b128 %4, %15 offset:4096\n\t"
                   "ds_read_b128 %5, %15 offset:5120\n\t"
                   "ds_read_b128 %6, %15 offset:6144\n\t"
                   "ds_read_b128 %7, %15 offset:7168\n\t"
                   "v_add_u32 %8, 0x10000, %16\n\t"
                   "v_add_u32 %9, 0x20000, %16\n\t"
                   "v_add_u32 %10, 0x30000, %16\n\t"
                   "v_add_u32 %11, 0x40000, %16\n\t"
                   "v_add_u32 %12, 0x50000, %16\n\t"
                   "v_add_u32 %13, 0x60000, %16\n\t"
                   "v_add_u32 %14, 0x70000, %16\n\t"
                   "s_mov_b64 exec, %18\n\t"
                   "s_waitcnt lgkmcnt(7)\n\tglobal_store_dwordx4 %16, %0, %17 sc0 sc1\n\t"
                   "s_waitcnt lgkmcnt(6)\n\tglobal_store_dwordx4 %8, %1, %17 sc0 sc1\n\t"
                   "s_waitcnt lgkmcnt(5)\n\tglobal_store_dwordx4 %9, %2, %17 sc0 sc1\n\t"
                   "s_waitcnt lgkmcnt(4)\n\tglobal_store_dwordx4 %10, %3, %17 sc0 sc1\n\t"
                   "s_waitcnt lgkmcnt(3)\n\tglobal_store_dwordx4 %11, %4, %17 sc0 sc1\n\t"
                   "s_waitcnt lgkmcnt(2)\n\tglobal_store_dwordx4 %12, %5, %17 sc0 sc1\n\t"
                   "s_waitcnt lgkmcnt(1)\n\tglobal_store_dwordx4 %13, %6, %17 sc0 sc1\n\t"
                   "s_waitcnt lgkmcnt(0)\n\tglobal_store_dwordx4 %14, %7, %17 sc0 sc1\n\t"
                   "s_cmp_lg_u64 %18, -1\n\t"
                   "s_cbranch_scc1 1f\n\t"
                   "s_endpgm\n"
                   "1:\n\t"
                   "s_mov_b64 exec, -1\n\t"
                   "s_nop 1"
                   : "=&v"(pv[0]), "=&v"(pv[1]), "=&v"(pv[2]), "=&v"(pv[3]), "=&v"(pv[4]), "=&v"(pv[5]), "=&v"(pv[6]), "=&v"(pv[7]),
                     "=&v"(o1), "=&v"(o2), "=&v"(o3), "=&v"(o4), "=&v"(o5), "=&v"(o6), "=&v"(o7)
                   : "v"(tr), "v"(oo), "s"(ob), "s"(s == 1 ? ~0ull : (u64)WMK) : "memory", "scc");
      if (s == 0) waitv<0>();
    }
  }

  if (!fin) {
    STAMP(myws, 6);
    const bool dir = DIR && j >= 2 && k < NS;
    char* dst = uni(dir ? P.peer + (size_t)(PIN_OFF + (call % 3u) * PIN_PAR + (u32)((j - 2) * NS + k) * PIECE + tile_off)
                        : myws + (size_t)(POUT_OFF + (u32)((j - 1) * 16 + k) * PIECE + tile_off));
    char* fl = myws + (size_t)(FLOC_OFF + (u32)((j - 1) * 8 + e * 4 + (sub ? sq : wave)) * 128u + (u32)k * 4u);
    if (sub) {
      typedef float f32x4 __attribute__((ext_vector_type(4)));
#pragma unroll
      for (int i = 0; i < 4; ++i)
        *(volatile LDSP f32x4*)(uintptr_t)(L.m0base + l16 + (u32)i * 1024u) =
            f32x4{acc[1][0][4 * i], acc[1][0][4 * i + 1], acc[1][0][4 * i + 2], acc[1][0][4 * i + 3]};
      __builtin_amdgcn_s_waitcnt(15 | (7 << 4) | (0 << 8) | (3 << 14));
      asm volatile("s_barrier" ::: "memory");
#pragma unroll
      for (int i = 0; i < 4; ++i) {
        const f32x4 t = *(volatile LDSP f32x4*)(uintptr_t)((sbase + (u32)(wave ^ 2) * 4096u) + l16 + (u32)i * 1024u);
#pragma unroll
        for (int z = 0; z < 4; ++z) acc[0][0][4 * i + z] += t[z];
      }
    }
    u32 least = 1u;
    if (__builtin_expect(dead, 0)) {
      least = 0x7fc07fc0u;
#pragma unroll
      for (int i = 0; i < 64; ++i) acc[i >> 5][(i >> 4) & 1][i & 15] = 0.f;
    }
    asm volatile("s_nop 4" ::: "memory");
#pragma unroll
    for (int nt = 0; nt < 2; ++nt)
#pragma unroll
      for (int mt = 0; mt < 2; ++mt) {
        if (sub && nt + mt) continue;
        u32x4 v0, v1;
#pragma unroll
        for (int z = 0; z < 4; ++z) {
          v0[z] = max(pack2(acc[nt][mt][2 * z], acc[nt][mt][2 * z + 1]), least);
          v1[z] = max(pack2(acc[nt][mt][8 + 2 * z], acc[nt][mt][8 + 2 * z + 1]), least);
        }
        const u32 o = nt ? l16b : l16;
        if (mt == 0) st16<0>(o, v0, dst); else st16<2048>(o, v0, dst);
        if (dir) __builtin_amdgcn_s_sleep(DSL);
        if (mt == 0) st16<1024>(o, v1, dst); else st16<3072>(o, v1, dst);
        if (dir) __builtin_amdgcn_s_sleep(DSL);
      }
    waitv<0>();
    if (sub) asm volatile("s_barrier" ::: "memory");
    if (lane == 0) st_sys(fl, call);
  }
  if (tid < N_MAIN_WG || !sub) break;
  }
  STAMP(myws, 5);
  STAMPV(myws, 0, tstart);
}

__global__ __launch_bounds__(256, 2) void fused_gemm_rs(Params P) {
  extern __shared__ __attribute__((aligned(16))) char smem[];
  const int bid = blockIdx.x;
  typedef u64 u64x4 __attribute__((ext_vector_type(4)));
  u64x4 a4;
  u64 tstart, pe;
  int rk, d;
  u32 tis, t0;
  u32x16 f16;
  u32x2 co;
  static_assert(N_PUSH_WG == 56 && PROT == 0 && PWN == 1 && CPB >= N_PUSH_WG, "entry code");
  asm volatile("v_readfirstlane_b32 s46, %[tid]\n\t"
               "s_load_dwordx8 s[36:43], %[ka], 0x0\n\t"
               "s_lshr_b32 s47, %[bid], 3\n\t"
               "s_cmp_gt_u32 %[bid], 55\n\t"
               "s_cbranch_scc1 2f\n\t"
               "s_lshr_b32 s48, s46, 7\n\t"
               "s_mul_i32 s49, s48, 3\n\t"
               "s_add_i32 s47, s47, s49\n\t"
               "s_add_i32 s49, s47, -7\n\t"
               "s_min_u32 s47, s47, s49\n\t"
               "s_lshl_b32 s49, s47, 4\n\t"
               "s_load_dwordx4 s[52:55], %[ka], s49 offset:0x40\n\t"
               "s_memrealtime s[44:45]\n\t"
               "s_load_dword s62, %[ka], 0x20\n\t"
               "s_setprio 3\n\t"
               "s_add_i32 s63, s47, 1\n\t"
               "v_lshlrev_b32 v1, 2, %[tid]\n\t"
               "v_mov_b32 v27, 0\n\t"
               "v_add_u32 v1, %[ctl], v1\n\t"
               "s_bitcmp1_b32 s46, 6\n\t"
               "ds_write_b32 v1, v27\n\t"
               "s_cbranch_scc0 1f\n\t"
               "s_and_b32 s49, %[bid], 7\n\t"
               "s_lshl1_add_u32 s48, s49, s48\n\t"
               "v_bfe_u32 v1, %[tid], 4, 2\n\t"
               "v_and_b32 v2, 15, %[tid]\n\t"
               "v_lshlrev_b32 v3, 2, v1\n\t"
               "v_lshlrev_b32 v1, 13, v1\n\t"
               "v_xor_b32 v2, v3, v2\n\t"
               "v_lshl_or_b32 v1, s48, 17, v1\n\t"
               "v_lshl_or_b32 v20, v2, 4, v1\n\t"
               "v_xor_b32 v21, 16, v20\n\t"
               "v_xor_b32 v22, 32, v20\n\t"
               "v_xor_b32 v23, 48, v20\n\t"
               "v_add_u32 v21, 0x7c00, v21\n\t"
               "v_add_u32 v22, 0xf800, v22\n\t"
               "v_add_u32 v23, 0x17400, v23\n\t"
               "s_lshl_b32 s49, s63, 7\n\t"
               "s_add_i32 s49, s49, %[lnk]\n\t"
               "v_mov_b32 v28, s49\n\t"
               "v_mov_b32 v25, 1\n\t"
               "s_waitcnt lgkmcnt(0)\n\t"
               "s_memrealtime s[58:59]\n\t"
               "global_load_dwordx4 v[144:147], v20, s[54:55] " MLP "\n\t"
               "global_load_dwordx4 v[148:151], v21, s[54:55] offset:1024 " MLP "\n\t"
               "global_load_dwordx4 v[152:155], v22, s[54:55] offset:2048 " MLP "\n\t"
               "global_load_dwordx4 v[156:159], v23, s[54:55] offset:3072 " MLP "\n\t"
               "s_mov_b64 exec, 1\n\t"
               "global_atomic_add v160, v28, v25, s[40:41] sc0 sc1\n\t"
               "s_add_i32 s50, s47, -1\n\t"
               "s_max_i32 s50, s50, 0\n\t"
               "s_mul_i32 s51, s50, %[fsl]\n\t"
               "s_lshl_b32 s49, s48, 7\n\t"
               "s_add_i32 s51, s51, s49\n\t"
               "s_add_i32 s51, s51, %[f0]\n\t"
               "v_mov_b32 v29, s51\n\t"
               "s_waitcnt lgkmcnt(0)\n\t"
               "s_or_b32 s66, s58, 1\n\t"
               "v_mov_b32 v26, s66\n\t"
               "global_atomic_cmpswap v161, v28, v[26:27], s[40:41] offset:4 sc0 sc1\n\t"
               "global_atomic_add v29, v27, s[52:53] sc1\n\t"
               "s_mov_b64 exec, -1\n"
               "1:\n\t"
               "s_waitcnt lgkmcnt(0)\n\t"
               "s_barrier\n\t"
               "s_branch 5f\n"
               "2:\n\t"
               "s_memrealtime s[44:45]\n\t"
               "s_load_dword s62, %[ka], 0x20\n\t"
               "s_load_dwordx2 s[52:53], %[ka], 0x40\n\t"
               "s_mov_b32 s63, 1\n\t"
               "s_add_i32 s47, %[bid], %[cpb]\n\t"
               "s_cmp_lt_u32 s47, 16\n\t"
               "s_cbranch_scc0 5f\n\t"
               "s_cmpk_lt_u32 s46, 0xc0\n\t"
               "s_cbranch_scc0 5f\n\t"
               "s_getpc_b64 s[56:57]\n\t"
               "s_mov_b32 m0, %[cpl]\n\t"
               "v_lshlrev_b32 v1, 7, %[tid]\n\t"
               "v_min_u32 v1, %[cpf], v1\n\t"
               "global_load_lds_dword v1, s[56:57]\n"
               "5:\n\t"
               "s_waitcnt lgkmcnt(0)"
               : "={s[36:43]}"(a4), "={s[44:45]}"(tstart), "={s46}"(t0), "={s[52:53]}"(pe), "={s62}"(rk), "={s63}"(d),
                 "={s66}"(tis)
               : [ka] "s"(__builtin_amdgcn_kernarg_segment_ptr()), [bid] "s"(bid), [tid] "v"(threadIdx.x),
                 [ctl] "i"(65536), [lnk] "i"(LNK_OFF), [fsl] "i"(IN_SRC + 4u * PIECE), [f0] "i"(NS * 2u * PIECE + 4096u),
                 [cpb] "i"(-CPB), [cpl] "i"(66048), [cpf] "i"(CPF - 128)
               : "memory", "scc", "s47", "s48", "s49", "s50", "s51", "s54", "s55", "s56", "s57", "s58", "s59",
                 "v1", "v2", "v3", "v20", "v21", "v22", "v23", "v25", "v26", "v27", "v28", "v29");
  asm volatile("" : "={v[144:159]}"(f16), "={v[160:161]}"(co));
  const u32 lane = threadIdx.x & 63u;
  const int wave = (int)(t0 >> 6);
  const Args A = {(const char*)a4[0], (const char*)a4[1], (char*)a4[2], (char*)a4[3], (char*)pe, rk, tis, f16, co};
  if (bid < N_PUSH_WG) mover(A, d, wave, lane, smem, tstart);
  else if (bid >= 256 && bid < 256 + N_PUSH_WG) {
    for (int i = 0; i < 6; ++i) __builtin_amdgcn_s_sleep(16);
  } else tile(A, bid < 256 - ROT ? bid - N_PUSH_WG + ROT : (bid < 256 ? bid - (256 - ROT) : bid - 2 * N_PUSH_WG), wave, lane, smem, tstart);
}

}

extern "C" void kernel_launch(int rank, int nranks, ncclComm_t comm,
                              hipStream_t stream,
                              void* const* d_in, const int* in_sizes, int n_in,
                              void* d_out, int out_size,
                              const int* in_dtypes, int out_dtype,
                              void* const* d_out_all,
                              void* const* d_ws_all, size_t ws_bytes) {
  if (nranks != NR || n_in < 2 || ws_bytes < (size_t)WS_NEED || (unsigned)rank >= (unsigned)NR) {
    fprintf(stderr, "unsupported configuration\n");
    return;
  }
  Params P;
  P.x = (const char*)d_in[0];
  P.w = (const char*)d_in[1];
  P.out = (char*)d_out;
  P.my = (char*)d_ws_all[rank];
  for (int i = 0; i < NR; ++i) {
    const int r = rank ^ i;
    P.lk[i].peer = (char*)d_ws_all[r];
    P.lk[i].w0 = P.w + (size_t)((r & 1) * (256 * 8192 + 15 * 256) + (r >> 2) * 4096);
  }
  P.rank = rank;
  P.pad[0] = P.pad[1] = P.pad[2] = 0;
  HIP_CHECK(hipFuncSetAttribute((const void*)fused_gemm_rs, hipFuncAttributeMaxDynamicSharedMemorySize, (int)LDS_BYTES));
  hipLaunchKernelGGL(fused_gemm_rs, dim3(N_WG), dim3(256), LDS_BYTES, stream, P);
  HIP_CHECK(hipGetLastError());
}
```

```cpp
#include <hip/hip_runtime.h>
#include <rccl/rccl.h>
#include <cstdio>
#include <cstdint>
#include <cstddef>

#define HIP_CHECK(expr) do { hipError_t _e = (expr); if (_e != hipSuccess) { \
    fprintf(stderr, "HIP error %s at %s:%d\n", hipGetErrorString(_e), __FILE__, __LINE__); \
    return; } } while (0)

namespace {

typedef unsigned char u8;
typedef unsigned short u16;
typedef unsigned u32;
typedef unsigned long long u64;
typedef u32 u32x4 __attribute__((ext_vector_type(4)));
typedef u32 u32x2 __attribute__((ext_vector_type(2)));
typedef u32 u32x16 __attribute__((ext_vector_type(16)));
typedef float f32x2 __attribute__((ext_vector_type(2)));
typedef float f32x16 __attribute__((ext_vector_type(16)));
typedef __bf16 bf16x2 __attribute__((ext_vector_type(2)));
typedef __bf16 bf16x8 __attribute__((ext_vector_type(8)));
typedef short s16x4 __attribute__((ext_vector_type(4)));
typedef short s16x8 __attribute__((ext_vector_type(8)));
#define LDSP __attribute__((address_space(3)))
#define GLBP __attribute__((address_space(1)))
#define CSTP __attribute__((address_space(4)))

constexpr int NR = 8;
constexpr u32 PIECE = 65536u;
constexpr u32 MIB = 1u << 20;
#define TAU 112
#define T0 20
#define QB 20
#define DB 0
#define TQ(q) (T0 + TAU * (q) - DB * ((q) > QB ? (q) - QB : 0))
#define EARLY 150
#define TAU1 111
#define TAU2 108
#define QM 22
#define STGD 2
#define PSL 4
#define LSL 16
#define WSL 8
#define CPF 19072
#define CPB 56
#define LZ 13
#define LZ1 4
#define LZD 200
#define LZS 24
#define LSQ 2
#define EIN 310
#define PHI 170
#define SPR 2
#define RK 1
#define OC 34
#define OG 10
#define OSP 7
#define OD 20
#define CLT 84
#define CKT 44
#define CLD 68
#define LPT 0
#define PQ1 99
#define PQ2 99
#define PQ3 0
#define L1A 1150
#define PWIN 30
#define PW1 30
#define FLS 128
#define FCS 64
#define LS1 0
#define PLD 0
#define EFL 8
#define EFW 5
#define PXM 0x75316420
#define DIR 8
#define DMX 0x30000ull
#define DMK (((1ull << (2 * DIR)) - 1ull) | DMX)
#define TOK 1
#define DTAU 114
#define DRL 80
#define DSP 2
#define DRT 40
#define WCH 1
#define WT0 380
#define WNOM 480
#define WMAX 1500
#define WMIN 6
#define WSP 2
#define OL 40
#define FST 16
#define REC 1
#define WTOP 112
#define SKP 1
#define WFAR 0
#define WD 16
#define LZE 4300
#define PR3 0
#define TWO 0
#define DBO 0
#define DBH 0
#define BAT 1
#define QD 21
#define BGN 49
#define BDZ 2
#define BCT 40
#define NOFL 0
#define DCP 1
#define RDM 0
#define NOS 0
#define BFK 7
#define QE2 3
#define SHD 5
#define SHM 13
#define SHR 1
#define PXH 0
#define AQ 0
#define AE 0
#define AGD 60
#define XR 32
#define ZT0 900
#define ZDT 100
#define RPT 400
#define RPB 25
#define L1T 850
#define SXT 6
#define WMK 1ull
#define WIN 3
#define PDU 100
#define GLP "sc0 sc1"
#define MLP "sc0 sc1"
constexpr int NT = 43;
constexpr int N1 = 31;
constexpr u32 IN_SRC = 4u * MIB;
constexpr u32 PIN_OFF = 24u * MIB;
constexpr u32 PIN_PAR = 8u * MIB;
constexpr u32 POUT_OFF = 48u * MIB;
constexpr u32 FLG_OFF = 55u * MIB;
constexpr u32 FLOC_OFF = FLG_OFF + 131072u;
constexpr u32 TMO_OFF = FLG_OFF + 49152u;
constexpr u32 LNK_OFF = FLG_OFF + MIB + 262144u + 131072u;
constexpr u32 DUMMY_OFF = FLG_OFF + 65536u;
constexpr u32 AGG_OFF = FLG_OFF + 262144u;
constexpr u32 TOK_OFF = FLG_OFF + 53248u;
constexpr u32 ORG_OFF = FLG_OFF + 57344u;
constexpr u32 DEC_OFF = FLG_OFF + 61440u;
constexpr u32 DBG_OFF = 58u * MIB;
constexpr u32 DBU_OFF = 59u * MIB;
constexpr u32 CNT_OFF = FLG_OFF + MIB + 262144u;
constexpr u32 WS_NEED = FLG_OFF + 2u * MIB;
constexpr u32 FIN0 = 20u * PIECE, FINS = IN_SRC + 2u * PIECE;

constexpr int N_PUSH_WG = 56;
#define N_WG 512
constexpr u32 STAGE_B = 16384u;
constexpr u32 LDS_BYTES = 4u * STAGE_B + 1024u;
constexpr u64 DEADLINE = 100000000ull;
constexpr u32 DEAD = 0x7fffffffu;

constexpr u64 RB_J = 0x0007654765432111ull;
constexpr u32 RB_BLK = 0x915403e4u;
__host__ __device__ __forceinline__ int cont_w(int k) { return k < 7 ? k : 16 + k; }
__host__ __device__ __forceinline__ int cont_x(int rb) { return 7 + rb; }
__host__ __device__ __forceinline__ int col_x(int s) { return ((s & 1) ? 15 : 0) ^ ((s & 2) * 4); }

struct Params {
  const char* x;
  const char* w;
  char* my;
  char* out;
  int rank;
  int pad[7];
  struct { char* peer; const char* w0; u64 f4; u32 meta[2]; } lk[NR];
  u64 arr[32];
  u8 ot[7][64];
  u16 it[6][40];
  u32 wg[N_WG - N_PUSH_WG];
};
struct Args { const char* x; const char* w; char* my; char* out; char* peer; int rank; u32 tis; u32x16 f16; u32x2 co; u32x4 m; };

#define SB() __builtin_amdgcn_sched_barrier(0)
template <int N> __device__ __forceinline__ void waitv() { asm volatile("s_waitcnt vmcnt(%0)" :: "i"(N) : "memory"); }

__device__ __forceinline__ u64 now() { return __builtin_amdgcn_s_memrealtime(); }
template <typename T> __device__ __forceinline__ T* uni(T* p) {
  const u64 v = (u64)(uintptr_t)p;
  const u32 lo = __builtin_amdgcn_readfirstlane((u32)v), hi = __builtin_amdgcn_readfirstlane((u32)(v >> 32));
  return (T*)(uintptr_t)(((u64)hi << 32) | lo);
}
__device__ __forceinline__ u32 ld_sys(const void* p) {
  return __hip_atomic_load((const GLBP u32*)(uintptr_t)p, __ATOMIC_RELAXED, __HIP_MEMORY_SCOPE_SYSTEM);
}
__device__ __forceinline__ void st_sys(void* p, u32 v) {
  __hip_atomic_store((GLBP u32*)(uintptr_t)p, v, __ATOMIC_RELAXED, __HIP_MEMORY_SCOPE_SYSTEM);
}
__device__ __forceinline__ void st_agt(void* p, u32 v) {
  __hip_atomic_store((GLBP u32*)(uintptr_t)p, v, __ATOMIC_RELAXED, __HIP_MEMORY_SCOPE_AGENT);
}
__device__ __forceinline__ bool give_up(u64 t0, const char* ws, u32 call) {
  const u64 dt = now() - t0;
  if (!__builtin_amdgcn_readfirstlane((int)(dt > 50000ull))) return false;
  __builtin_amdgcn_s_sleep(64);
  if (__builtin_amdgcn_readfirstlane((int)(dt > DEADLINE))) return true;
  return __builtin_amdgcn_readfirstlane(ld_sys(ws + TMO_OFF)) == call;
}
__device__ __forceinline__ void sleep_until(u64 tstart, u32 t) {
  for (;;) {
    const u32 el = (u32)__builtin_amdgcn_readfirstlane((u32)(now() - tstart));
    if (el >= t) break;
    if (t - el > 40u) __builtin_amdgcn_s_sleep(12); else __builtin_amdgcn_s_sleep(1);
  }
}
__device__ __forceinline__ void sleep_exact(u64 tstart, u32 t) {
  for (;;) {
    const u32 el = (u32)__builtin_amdgcn_readfirstlane((u32)(now() - tstart));
    if (el >= t) break;
    if (t - el > 60u) __builtin_amdgcn_s_sleep(12);
    else if (t - el > 20u) __builtin_amdgcn_s_sleep(3);
    else if (t - el > (u32)SXT) __builtin_amdgcn_s_sleep(1);
  }
}
template <int IMM> __device__ __forceinline__ void st16(u32 off, u32x4 v, char* base) {
  asm volatile("global_store_dwordx4 %0, %1, %2 offset:%3 sc0 sc1\n\ts_nop 1" :: "v"(off), "v"(v), "s"(base), "i"(IMM) : "memory");
}
template <int IMM> __device__ __forceinline__ void st16p(u32 off, u32x4 v, char* base) {
  asm volatile("global_store_dwordx4 %0, %1, %2 offset:%3\n\ts_nop 1" :: "v"(off), "v"(v), "s"(base), "i"(IMM) : "memory");
}
#define OST st16
__device__ __forceinline__ void dma16x4(u32 lds, const u32 (&o)[4], const char* base) {
  asm volatile("s_mov_b32 m0, %0\n\ts_nop 3\n\t"
               "global_load_lds_dwordx4 %1, %5 " MLP "\n\t"
               "global_load_lds_dwordx4 %2, %5 offset:1024 " MLP "\n\t"
               "global_load_lds_dwordx4 %3, %5 offset:2048 " MLP "\n\t"
               "global_load_lds_dwordx4 %4, %5 offset:3072 " MLP
               :: "s"(lds), "v"(o[0]), "v"(o[1]), "v"(o[2]), "v"(o[3]), "s"(base) : "memory");
}
__device__ __forceinline__ void probe(u32 off, char* base) {
  asm volatile("s_nop 4\n\tglobal_atomic_add %0, %1, %2 sc1" :: "v"(off), "v"(0u), "s"(base) : "memory");
}
__device__ __forceinline__ void probe0(u32 off, char* base) {
  asm volatile("s_mov_b64 exec, 1\n\ts_nop 3\n\tglobal_atomic_add %0, %1, %2 sc1\n\ts_mov_b64 exec, -1"
               :: "v"(off), "v"(0u), "s"(base) : "memory");
}
__device__ __forceinline__ u32 atom_rd(u32 off, const char* base) {
  u32 r;
  asm volatile("global_atomic_add %0, %1, %2, %3 sc0 sc1\n\ts_waitcnt vmcnt(0)" : "=&v"(r) : "v"(off), "v"(0u), "s"(base) : "memory");
  return r;
}
__device__ __forceinline__ u32 bump_issue(u32 off, const char* base) {
  u32 r;
  asm volatile("s_mov_b64 exec, 1\n\ts_nop 3\n\tglobal_atomic_add %0, %1, %2, %3 sc0 sc1\n\ts_mov_b64 exec, -1"
               : "=&v"(r) : "v"(off), "v"(1u), "s"(base) : "memory");
  return r;
}
template <int N> __device__ __forceinline__ u32 bump_get(u32 r) {
  u32 v;
  asm volatile("s_waitcnt vmcnt(%2)\n\tv_readfirstlane_b32 %0, %1\n\ts_nop 4" : "=s"(v) : "v"(r), "i"(N) : "memory");
  return v + 1u;
}
__device__ __forceinline__ void add1_lane0(u32 off, char* base) {
  asm volatile("s_mov_b64 exec, 1\n\ts_nop 3\n\tglobal_atomic_add %0, %1, %2 sc1\n\ts_mov_b64 exec, -1"
               :: "v"(off), "v"(1u), "s"(base) : "memory");
}
__device__ __forceinline__ void addn_lane0(u32 off, u32 n, char* base) {
  asm volatile("s_mov_b64 exec, 1\n\ts_nop 3\n\tglobal_atomic_add %0, %1, %2 sc1\n\ts_mov_b64 exec, -1"
               :: "v"(off), "v"(n), "s"(base) : "memory");
}
__device__ __forceinline__ void st4_lane0(u32 off, u32 val, char* base) {
  asm volatile("s_mov_b64 exec, 1\n\ts_nop 3\n\tglobal_store_dword %0, %1, %2 sc0 sc1\n\ts_mov_b64 exec, -1"
               :: "v"(off), "v"(val), "s"(base) : "memory");
}

#define STAMP(ws, idx) do {} while (0)
#define STAMPV(ws, idx, v) do {} while (0)

__device__ __forceinline__ u32 fin_off(u32 pl) { return FIN0 + (pl >> 5) * FINS + (pl & 31u) * FCS; }
__device__ __forceinline__ u32 dec_off(u32 s) { return FIN0 + s * FINS + 8192u; }
__device__ __forceinline__ u32 lds_rd(u32 a) { return __builtin_amdgcn_readfirstlane(*(volatile LDSP u32*)(uintptr_t)a); }
__device__ __forceinline__ void lds_wr(u32 a, u32 v) { *(volatile LDSP u32*)(uintptr_t)a = v; }
__device__ __forceinline__ u32 rd_lane0(u32 off, const char* base) {
  u32 r, v;
  asm volatile("s_mov_b64 exec, 1\n\ts_nop 3\n\tglobal_atomic_add %0, %2, %3, %4 sc0 sc1\n\ts_mov_b64 exec, -1\n\t"
               "s_waitcnt vmcnt(0)\n\tv_readfirstlane_b32 %1, %0\n\ts_nop 4" : "=&v"(r), "=s"(v) : "v"(off), "v"(0u), "s"(base) : "memory");
  return v;
}
__device__ __forceinline__ const char* kargs() { return (const char*)(uintptr_t)__builtin_amdgcn_kernarg_segment_ptr(); }

__device__ __forceinline__ void mover(const Args& P, int d, int wave, u32 lane, char* smem, u64 tstart) {
  char* const myws = P.my;
  const int pair = wave >> 1;
  const int p = (int)(((u32)PXM >> ((blockIdx.x & 7) * 4)) & 7u) * (PXH ? 1 : 2) + pair * (PXH ? 8 : 1);
  const u32 meta = P.m[2];
  int nin = (int)(meta & 255u);
  int np = d == 1 ? N1 - __builtin_popcountll((DMK >> (p >> 3)) & 0x5555555555555555ull) : 99;
  const u32 ring = (u32)(uintptr_t)smem + (u32)pair * 32768u;
  const u32 ctl = (u32)(uintptr_t)smem + 65536u + (u32)pair * 512u;
  constexpr u32 XC = 256u / XR;
  const u32 xrow = ((u32)p % (128u / XR)) * XR, xks = (u32)p / (128u / XR) * (XC / 4u);

  if ((wave & 1) == 0) {
    const u32 cnt = bump_issue(CNT_OFF + (u32)(blockIdx.x * 4 + wave) * 64u, myws);
    const u32 kh = (meta >> 24) & 1u;
    const char* const xin = P.x + (size_t)(kh * 512u);
    const char* const win = P.w + (size_t)(kh * 256u * 8192u);
    u32 wo[4], xo[4], oo[4];
#pragma unroll
    for (int k = 0; k < 4; ++k) {
      const u32 q4 = lane >> 4, c = (u32)k * 64u + lane;
      wo[k] = ((u32)p * 16u + (u32)k * 4u + q4) * 8192u + (((lane & 15u) ^ ((q4 << 2) | (u32)k)) * 16u) - (u32)k * 1024u;
      xo[k] = (xrow + c / XC) * 1024u + xks * 64u + (c % XC) * 16u - (u32)k * 1024u;
      oo[k] = lane * 16u;
    }
#define ISSUE_IN(QQ, EN) do { \
      const u32 dst_ = ring + (u32)((QQ) & 7) * 4096u, ix_ = ((EN) >> 8) & 31u; \
      if ((EN) & 8192u) dma16x4(dst_, xo, xin + (size_t)(ix_ * 131072u)); \
      else              dma16x4(dst_, wo, win + (size_t)(ix_ * 256u)); \
    } while (0)

    int q0 = 0;
    u32 pub = 0, call;
    u32 dbg1 = 0, dbg2 = 0;
    if (nin) {
      const u32 e1 = P.m[0] >> 16, e2 = P.m[1] & 0xffffu, e3 = P.m[1] >> 16;
      ISSUE_IN(1, e1); ISSUE_IN(2, e2); ISSUE_IN(3, e3);
      lds_wr(ctl + 36u, e1); lds_wr(ctl + 40u, e2); lds_wr(ctl + 44u, e3);
    }
    u32 tab, ob;
    asm volatile("s_nop 4\n\tglobal_load_ushort %0, %2, %4 offset:%5\n\t"
                 "global_load_ubyte %1, %3, %4 offset:%6"
                 : "=&v"(tab), "=&v"(ob)
                 : "v"((u32)max(d - 2, 0) * 80u + min(lane, 39u) * 2u),
                   "v"((u32)(d - 1) * 64u + (lane & 31u) * 2u + (u32)(p >> 3)), "s"(kargs()),
                   "i"(offsetof(Params, it)), "i"(offsetof(Params, ot)) : "memory");
    if (nin) {
      call = bump_get<10>(cnt);
      lds_wr(ctl, 2u);
      q0 = 4;
    } else {
      call = bump_get<0>(cnt);
    }
    asm volatile("s_waitcnt vmcnt(0)" : "+v"(tab), "+v"(ob) :: "memory");
    pub = (u32)q0;
    lds_wr(ctl, pub);
    const bool isf = d != 1 && lane >= 16u;
    const u32 poll_off = isf ? fin_off(ob) : FLOC_OFF + ((ob >> 4) * 4u + ((u32)(p & 7) >> 1)) * FLS + (ob & 15u) * 4u;
    const u32 src_off = isf ? (ob * 2u + (call & 1u)) * PIECE + (u32)p * 4096u : POUT_OFF + ob * 32768u + (u32)(p & 7) * 4096u;
    const u32 thr = isf ? 16u * call : call;
    u64 todo = __builtin_amdgcn_ballot_w64(!isf && lane < 32u && ob < 208u && (d > 1 || !((DMK >> (2u * (lane & 31u) + (u32)(p >> 3))) & 1ull)));
    if (!nin) sleep_until(tstart, (u32)L1T);
    u64 ready = 0;
    u32 cons = 0;
    bool dead = false;
    u64 left = ~0ull << q0;
    const u64 drm = __builtin_amdgcn_ballot_w64(tab < 16384u && (int)lane < nin);
    for (int q = q0; q < np; ++q) {
      if (nin && q == QD) {
        const u64 t0 = now();
        u32 wd;
        for (;;) {
          wd = rd_lane0(dec_off((u32)d - 2u), myws);
          pub = (u32)q;
          lds_wr(ctl, pub);
          if ((wd >> 12) == (call & 0xfffffu)) break;
          __builtin_amdgcn_s_sleep(8);
          if (give_up(t0, myws, call)) { dead = true; break; }
        }
        if (dead) break;
        u64 sk = 0;
#pragma unroll
        for (int e = 0; e < 4; ++e)
          if (((u32)(p + (e & 1) * 16) < ((wd >> (e >> 1) * 6) & 63u)) != (d < 4)) sk |= 1ull << ((P.m[3] >> (8 * e)) & 63u);
        sk &= ~(1ull << 63);
        left &= ~sk;
        nin -= __builtin_popcountll(sk);
        np = nin + __builtin_popcountll(todo);
        lds_wr(ctl + 12u, (u32)(nin | np << 8) | wd << 16);
      }
      int ix = SKP ? __builtin_ctzll(left) : q;
      u32 en = (u32)__builtin_amdgcn_readlane((int)tab, ix);
      u64 want = q < nin ? (u64)(en >> 14) << ((en >> 8) & 31u) : todo;
      bool rdy = want == 0ull || (ready & want) != 0ull;
      if (cons + 8u <= (u32)q) cons = lds_rd(ctl + 4u);
      if (!rdy || cons + 8u <= (u32)q) {
        if (rdy) waitv<0>();
        pub = (u32)q;
        if (rdy) lds_wr(ctl, pub);
        const u64 t0 = now();
        const u64 look = q < nin ? 0x3f0000ull : todo & ((1ull << (__builtin_ctzll(todo) + (nin ? PWIN : PW1))) - 1ull);
        for (;;) {
          if (!rdy) {
            u32 pv = 0;
            asm volatile("s_mov_b64 exec, %4\n\ts_nop 3\n\t"
                         "global_atomic_add %0, %1, %2, %3 sc0 sc1\n\t"
                         "s_mov_b64 exec, -1\n\t"
                         "s_waitcnt vmcnt(1)\n\t"
                         "ds_write_b32 %5, %6\n\t"
                         "s_waitcnt vmcnt(0)"
                         : "+&v"(pv) : "v"(poll_off), "v"(0u), "s"(myws), "s"(look), "v"(ctl), "v"(pub) : "memory");
            ready |= __builtin_amdgcn_ballot_w64(pv >= thr);
            rdy = (ready & want) != 0ull;
            if (LSL && !rdy && nin && cons + (u32)LSQ <= (u32)q) __builtin_amdgcn_s_sleep(LSL);
            if (LS1 && !rdy && !nin && cons + (u32)LSQ <= (u32)q) __builtin_amdgcn_s_sleep(LS1);
          } else {
            __builtin_amdgcn_s_sleep(2);
          }
          cons = lds_rd(ctl + 4u);
          if (SKP && !rdy && cons + (u32)SKP > (u32)q && (left & drm)) {
            ix = __builtin_ctzll(left & drm);
            en = (u32)__builtin_amdgcn_readlane((int)tab, ix);
            want = 0ull;
            rdy = true;
          }
          if (rdy && cons + 8u > (u32)q) break;
          if (give_up(t0, myws, call)) { dead = true; break; }
        }
        if (dead) break;
      }
      left &= ~(1ull << ix);
      if (want) {
        const int id = __builtin_ctzll(ready & want);
        todo &= ~(1ull << id);
        if (q >= nin) en = (u32)id;
        dma16x4(ring + (u32)(q & 7) * 4096u, oo, myws + (size_t)(u32)__builtin_amdgcn_readlane((int)src_off, id));
      } else {
        ISSUE_IN(q, en);
      }
      lds_wr(ctl + 32u + (u32)(q & 7) * 4u, en);
      if (TOK) lds_wr(ctl + 8u, (u32)q + 1u);
      waitv<8>();
      if (nin && q == PQ1) probe0((u32)(d - 2) * IN_SRC + 2u * MIB + (u32)p * 4096u, P.peer);
      if (nin && q == PQ2) probe0(PIN_OFF + (call % 3u) * PIN_PAR + (u32)(d * 16 + 8) * PIECE + (u32)p * 4096u, P.peer);
      if (PQ3 && q == PQ3 && d < 4 && nin && lane < 40u && (tab >> 14)) {
        probe((tab & 255u) * 2u * PIECE + (u32)p * 4096u, P.peer);
        probe(fin_off(tab & 224u) + 4096u + (u32)p * 128u, P.peer);
      }
      if (q - 1 > (int)pub) { pub = (u32)(q - 1); lds_wr(ctl, pub); }
      if (q == nin - 1) STAMP(myws, 1);
    }
    waitv<0>();
    if (dead) st4_lane0(TMO_OFF, call, myws);
    lds_wr(ctl, dead ? DEAD : (u32)np);
    STAMP(myws, 3);
    STAMPV(myws, 0, tstart);
    return;
  }

  char* const rws = P.peer;
  const int slot = d - 2;
  const u32 dl = (u32)p * 4096u + lane * 16u;
  u32 xl[4], xd[4];
#pragma unroll
  for (int k = 0; k < 4; ++k) {
    const u32 c = (u32)k * 64u + lane, t = c / (4u * XR), row = (c >> 2) % XR;
    xl[k] = (row * XC + t * 4u + ((c ^ ((xrow + row) >> 2)) & 3u)) * 16u;
    xd[k] = (xks + t) * 8192u + (xrow + row) * 64u + (c & 3u) * 16u - (u32)k * 1024u;
  }
  u32 tick, ts0, ta;
  int to;
  {
    u32 tg, tm;
    asm volatile("" : "+v"(xl[0]), "+v"(xl[1]), "+v"(xl[2]), "+v"(xl[3]), "+v"(xd[0]), "+v"(xd[1]), "+v"(xd[2]), "+v"(xd[3]));
    asm volatile("s_waitcnt vmcnt(1)\n\t"
                 "v_readfirstlane_b32 %[tick], v160\n\t"
                 "v_readfirstlane_b32 %[tg], v161\n\t"
                 "s_memrealtime s[58:59]\n\t"
                 "s_sub_i32 %[tm], %[tg], %[tis]\n\t"
                 "s_addk_i32 %[tm], 500\n\t"
                 "s_cmpk_gt_u32 %[tm], 1000\n\t"
                 "s_cselect_b32 %[tg], %[tis], %[tg]\n\t"
                 "s_addk_i32 %[tg], %[oc]\n\t"
                 "s_sub_i32 %[to], %[tg], %[tst]\n\t"
                 "s_and_b32 %[tm], %[tick], 15\n\t"
                 "s_mul_i32 %[tm], %[tm], %[osp]\n\t"
                 "s_add_i32 %[tg], %[tg], %[tm]\n\t"
                 "s_bitcmp0_b32 %[tick], 4\n\t"
                 "s_cselect_b32 %[tm], 0x10000, 0\n\t"
                 "v_add_u32 v160, %[tm], %[dl]\n\t"
                 "s_cmp_eq_u32 %[d], 1\n\t"
                 "s_waitcnt lgkmcnt(0)\n\t"
                 "s_mov_b32 %[ta], s58\n\t"
                 "s_cbranch_scc1 5f\n"
                 "4:\n\t"
                 "s_waitcnt lgkmcnt(0)\n\t"
                 "s_sub_i32 %[tm], s58, %[tg]\n\t"
                 "s_mov_b32 %[ts0], s58\n\t"
                 "s_memrealtime s[58:59]\n\t"
                 "s_cmp_lt_i32 %[tm], 0\n\t"
                 "s_cbranch_scc1 4b\n\t"
                 "global_store_dwordx4 v160, v[144:147], %[dst] sc0 sc1\n\t"
                 "global_store_dwordx4 v160, v[148:151], %[dst] offset:1024 sc0 sc1\n\t"
                 "global_store_dwordx4 v160, v[152:155], %[dst] offset:2048 sc0 sc1\n\t"
                 "global_store_dwordx4 v160, v[156:159], %[dst] offset:3072 sc0 sc1\n"
                 "5:\n\t"
                 "s_waitcnt lgkmcnt(0)"
                 : [tick] "=&s"(tick), [ts0] "=&s"(ts0), [ta] "=&s"(ta), [to] "=&s"(to), [tg] "=&s"(tg), [tm] "=&s"(tm)
                 : "{v[144:159]}"(P.f16), "{v[160:161]}"(P.co), [tis] "s"(P.tis), [tst] "s"((u32)tstart), [d] "s"(d),
                   [dl] "v"(dl), [dst] "s"(rws + (size_t)((u32)max(slot, 0) * IN_SRC)), [oc] "i"(OC), [osp] "i"(OSP)
                 : "memory", "scc", "s58", "s59");
  }
  const u32 call = (tick >> 4) + 1u;
  u32 dbg1 = 0, dbg2 = 0;
  const u32 pin0 = PIN_OFF + (call % 3u) * PIN_PAR + (u32)(d == 1 ? 0 : d * 16) * PIECE;
  u32 fpend = 0;
#define FLAG() do { \
    add1_lane0(fpend ? fpend : DUMMY_OFF + (u32)(blockIdx.x * 4 + wave) * 128u, uni(fpend ? rws : myws)); \
    fpend = 0; \
  } while (0)

  u32 tgrid = 0, ph = 0;
  int qsw = 99, qsx = 99, ph0 = 0, ad0 = 0, ad1 = 0;
  if (nin) {
    if (!EFL) FLAG();
    fpend = fin_off((u32)slot * 32u);
    const int cold = ta - P.tis > (u32)CLT || P.tis - (u32)tstart > (u32)CKT ? CLD : 0;
    const u32 tb = (u32)max(to + TAU + OG + cold, (int)(ts0 - (u32)tstart) + OD) + (u32)(TAU - 1);
    tgrid = tb - ((u32)tstart + tb - ((u32)(((p + STGD * d) & 15) * TAU) >> 4)) % (u32)TAU;
    ph = (u32)min((int)tgrid - (to + TAU + OG + cold), TAU) * (u32)FST >> 4;
    ph0 = (int)tgrid - (to + TAU + OG + cold);
  } else {
    sleep_until(tstart, 600u);
    probe0(pin0 + (u32)p * 4096u, rws);
  }
  u32 full = 0, tnext = 0, vlast = 0;
  u32 apend = 0, agg = 0, na = 0;
  const int qh = 99;
#define SETTLE() do { \
    u32 o_; \
    asm volatile("v_readfirstlane_b32 %0, v161\n\ts_nop 4" : "=s"(o_) :: "memory"); \
    if ((o_ & 15u) == 15u) { addn_lane0(apend, 16u, rws); ++vlast; } \
    apend = 0; \
  } while (0)
#define PUBLISH() do { \
    if (agg) { \
      asm volatile("s_mov_b64 exec, 1\n\ts_nop 3\n\tglobal_atomic_add v161, %0, %1, %2 sc0 sc1\n\ts_mov_b64 exec, -1" \
                   :: "v"(agg), "v"(1u), "s"(myws) : "memory", "v161"); \
      apend = fpend; na = 0; \
    } else add1_lane0(fpend, rws); \
    fpend = 0; \
  } while (0)
#define L2S()
  u32 tsh = 0;
#define SHED() do { \
    if (SHD && (u32)__builtin_amdgcn_readfirstlane((u32)(now() - tstart)) > tsent + (u32)SHM) { tgrid += (u32)SHD; tsh += (u32)SHD; } \
  } while (0)
  u32 res = 0, rn = 1, have = 0;
#define RESERVE(T) asm volatile("s_mov_b64 exec, 1\n\ts_nop 3\n\t" \
                   "global_atomic_umax_x2 %0, %1, %3 sc1\n\t" \
                   "global_atomic_add_x2 v[160:161], %0, %2, %3 sc0 sc1\n\t" \
                   "s_mov_b64 exec, -1" \
                   :: "v"(TOK_OFF), "v"((now() + (u64)(T)) << 5), "v"((u64)(rn * (u32)(2 * TAU1))), "s"(myws) : "memory", "v160", "v161")
#define STEP() do { \
    u32 vm_; \
    asm volatile("s_getreg_b32 %0, hwreg(HW_REG_IB_STS, 0, 4)" : "=s"(vm_) :: "memory"); \
    if (apend) { \
      if (vm_ <= na) { \
        if (na) waitv<4>(); else waitv<0>(); \
        SETTLE(); \
      } \
    } else if (vm_ <= vlast) { \
      PUBLISH(); \
      L2S(); \
    } \
  } while (0)
  for (int q = nin ? 1 : 0; q < np; ++q) {
    tnext = max(tnext, tgrid);
    u32 tw = 0;
    if (full <= (u32)q) {
      full = lds_rd(ctl);
      if (full <= (u32)q) {
        u64 el;
        for (;;) {
          __builtin_amdgcn_s_sleep(1);
          full = lds_rd(ctl);
          el = now() - tstart;
          if (full > (u32)q) break;
          if (TOK && d == 1 && !res && !have && lds_rd(ctl + 8u) > (u32)q) { rn = 1; RESERVE(DRL); res = 3; }
          if (AQ) {
            if (fpend | apend) STEP();
          } else if (fpend && __builtin_amdgcn_readfirstlane((int)(el > (u64)(tnext + 30u)))) {
            waitv<0>();
            FLAG();
          }
          if (__builtin_amdgcn_readfirstlane((int)(el > 2ull * DEADLINE - 1000ull))) { full = DEAD; break; }
        }
        tw = (u32)__builtin_amdgcn_readfirstlane((u32)el);
        tnext = max(tnext, tw);
      }
      if (full == DEAD) break;
    }
    if (nin && q == QD) {
      const u32 w = lds_rd(ctl + 12u);
      nin = (int)(w & 255u);
      np = (int)(w >> 8) & 255;
      const int a0 = (int)(w >> 16) & 15, a1 = (int)(w >> 22) & 15;
      const int n0 = d < 4 ? -a0 & 15 : a0, n1 = d < 4 ? -a1 & 15 : a1;
      const int p0 = d < 4 ? (p - a0) & 15 : p, p1 = d < 4 ? (p - a1) & 15 : p;
      qsw = (int)((meta >> 16) & 255u) + (ph0 < ((p0 + 1) * TAU >> 4) ? 1 : 0);
      qsx = (int)((meta >> 16) & 255u) + QE2 + (ph0 < ((p1 + 1) * TAU >> 4) ? 1 : 0) - (p0 < n0 ? 1 : 0);
      ad0 = (p0 < n0 ? TAU : 0) - (n0 * TAU >> 4);
      ad1 = (p1 < n1 ? TAU : 0) - (n1 * TAU >> 4);
    }
    if (TOK && d == 1 && !res && !have) { rn = 1; RESERVE(DRL); res = 3; }
    const u32 id = lds_rd(ctl + 32u + (u32)(q & 7) * 4u);
    const bool isx = (id & 8192u) != 0u;
    if (FST && (id & 16384u) && tw + ph > tgrid) {
      tsh += tw + ph - tgrid;
      tgrid = tw + ph;
      tnext = max(tnext, tgrid);
    }
    const u32 la = ring + (u32)(q & 7) * 4096u;
    const u32x4 d0 = *(volatile LDSP u32x4*)(uintptr_t)(la + (isx ? xl[0] : lane * 16u));
    const u32x4 d1 = *(volatile LDSP u32x4*)(uintptr_t)(la + (isx ? xl[1] : lane * 16u + 1024u));
    const u32x4 d2 = *(volatile LDSP u32x4*)(uintptr_t)(la + (isx ? xl[2] : lane * 16u + 2048u));
    const u32x4 d3 = *(volatile LDSP u32x4*)(uintptr_t)(la + (isx ? xl[3] : lane * 16u + 3072u));
    lds_wr(ctl + 4u, (u32)q + 1u);
    char* const dst = rws + (size_t)(q < nin ? ((id & 255u) * 2u + (call & 1u)) * PIECE : pin0 + (id & 31u) * PIECE);
    if (AQ) {
      while (fpend | apend) {
        STEP();
        if ((u32)__builtin_amdgcn_readfirstlane((u32)(now() - tstart)) + (u32)EFL >= tnext) break;
      }
    } else if (EFL) {
      while (fpend) {
        u32 vm;
        asm volatile("s_getreg_b32 %0, hwreg(HW_REG_IB_STS, 0, 4)" : "=s"(vm) :: "memory");
        if (vm <= vlast) {
          if (fpend) add1_lane0(fpend, rws);
          fpend = 0;
        }
        if ((u32)__builtin_amdgcn_readfirstlane((u32)(now() - tstart)) + (u32)EFL >= tnext) break;
      }
    }
    if (TOK && d == 1 && have) {
      tnext = have;
      have = 0;
    } else if (TOK && d == 1) {
      if (res == 1u) waitv<4>(); else if (res == 2u) waitv<8>(); else waitv<0>();
      u32 lo, hi;
      asm volatile("v_readfirstlane_b32 %0, v160\n\tv_readfirstlane_b32 %1, v161\n\ts_nop 4" : "=s"(lo), "=s"(hi) :: "memory");
      tnext = min((u32)((((u64)hi << 32 | lo) >> 5) - tstart), 20000u);
      if (rn == 2u) have = tnext + (u32)(TAU1 / 16);
      sleep_exact(tstart, tnext - 12u);
      const u32 av = lds_rd(ctl + 8u) - (u32)q - rn;
      res = av ? rn : 0u;
      rn = BAT && q >= BAT - 1 && av > 1u ? 2u : 1u;
      if (res) RESERVE(DRL);
    }
    sleep_exact(tstart, tnext);
    const u32 tsent = tnext;
    if (nin) {
      tgrid += (u32)(TAU - (q >= QB ? DB : 0) - (q >= nin ? DBO : (DBH && q >= DBH ? q & 1 : 0)));
      if (q == qsw) tgrid += (u32)ad0;
      if (q == qsx) tgrid += (u32)ad1;
    }
    tnext += (u32)(d == 1 ? TAU1 : (q >= QM && q != qsw && q != qsx ? TAU2 : 0));
    st16<0>(isx ? xd[0] : dl, d0, dst);
    st16<1024>(isx ? xd[1] : dl, d1, dst);
    st16<2048>(isx ? xd[2] : dl, d2, dst);
    st16<3072>(isx ? xd[3] : dl, d3, dst);
    vlast = 0;
    if (AQ) {
      if (fpend | (apend ? na : 0u)) {
        waitv<4>();
        SHED();
        if (apend) SETTLE();
        if (fpend) {
          PUBLISH();
          ++vlast;
        }
      } else {
        if (SHR && tsh) { --tgrid; --tsh; }
        na = 4;
        if (q <= nin && nin) waitv<EFW>(); else waitv<4 * WIN>();
      }
      fpend = q < nin ? fin_off(id & 255u) : 0u;
      agg = q >= AQ && q < nin - AE && q != qh ? AGG_OFF + ((u32)d * 256u + (id & 255u)) * 64u : 0u;
    } else if (EFL && nin && q <= nin) {
      waitv<EFW>();
      if (fpend) {
        waitv<4>();
        add1_lane0(fpend, rws);
        SHED();
        vlast = 1;
      } else if (SHR && tsh) { --tgrid; --tsh; }
      fpend = q < nin ? fin_off(id & 255u) : 0u;
    } else if (nin && q <= nin) {
      waitv<5>();
      FLAG();
      if (q < nin) fpend = fin_off(id & 255u);
    } else {
      if (REC && tsh) { const u32 rc = min(tsh, (u32)REC); tgrid -= rc; tsh -= rc; }
      waitv<4 * WIN>();
    }
    if (q == nin - 1) STAMP(myws, 2);
    if (q == np - 1) STAMP(myws, 4);
  }
  if ((tick & 15u) == 0u) st4_lane0(LNK_OFF + (u32)d * 128u + 4u, 0u, myws);
  waitv<0>();
  STAMP(myws, 3);
  STAMPV(myws, 0, tstart);
}

struct Frag {
  bf16x8 x[2][2];
  bf16x8 w[2][2];
};

struct Loader {
  const char* base;
  u32 stride[2];
  u32 v[4];
  u32 m0base;
};

template <int STAGE> __device__ __forceinline__ void glds_pair(const Loader& L, int half) {
  const u32 m0v = L.m0base + STAGE * STAGE_B;
  if (half == 0)
    asm volatile("s_mov_b32 m0, %0\n\ts_nop 0\n\t"
                 "global_load_lds_dwordx4 %1, %3 " GLP "\n\t"
                 "global_load_lds_dwordx4 %2, %3 offset:1024 " GLP
                 :: "s"(m0v), "v"(L.v[0]), "v"(L.v[1]), "s"(L.base) : "memory");
  else
    asm volatile("s_mov_b32 m0, %0\n\ts_nop 0\n\t"
                 "global_load_lds_dwordx4 %1, %3 offset:2048 " GLP "\n\t"
                 "global_load_lds_dwordx4 %2, %3 offset:3072 " GLP
                 :: "s"(m0v), "v"(L.v[2]), "v"(L.v[3]), "s"(L.base) : "memory");
}

struct RdAddr { u32 xa[2]; u32 wa[2][2]; };

template <int STAGE> __device__ __forceinline__ bf16x8 rd_x(const RdAddr& A, int mt, int ks) {
  return *(const LDSP bf16x8*)(uintptr_t)(A.xa[ks] + (u32)(STAGE * STAGE_B + mt * 2048));
}
template <int STAGE> __device__ __forceinline__ bf16x8 rd_w(const RdAddr& A, int nt, int ks) {
  s16x4 lo = __builtin_amdgcn_ds_read_tr16_b64_v4i16((LDSP s16x4*)(uintptr_t)(A.wa[nt][0] + (u32)(STAGE * STAGE_B + 8192 + ks * 4096)));
  s16x4 hi = __builtin_amdgcn_ds_read_tr16_b64_v4i16((LDSP s16x4*)(uintptr_t)(A.wa[nt][1] + (u32)(STAGE * STAGE_B + 8192 + ks * 4096)));
  s16x8 v = __builtin_shufflevector(lo, hi, 0, 1, 2, 3, 4, 5, 6, 7);
  return __builtin_bit_cast(bf16x8, v);
}
#define MFMA(nt, mt, ks) acc[nt][mt] = __builtin_amdgcn_mfma_f32_32x32x16_bf16(cur.w[nt][ks], cur.x[mt][ks], acc[nt][mt], 0, 0, 0)

template <int S, int VM, bool LOAD, bool READ>
__device__ __forceinline__ void kstep(f32x16 (&acc)[2][2], const Frag& cur, Frag& nxt, const RdAddr& A, Loader& L) {
  constexpr int SN = (S + 1) & 3;
  if (READ) {
    __builtin_amdgcn_s_waitcnt((VM & 15) | (7 << 4) | (0 << 8) | ((VM >> 4) << 14));
    asm volatile("s_barrier" ::: "memory");
  } else {
    __builtin_amdgcn_s_waitcnt(15 | (7 << 4) | (0 << 8) | (3 << 14));
  }
  SB();
  MFMA(0, 0, 0); SB();
  if (READ) { nxt.w[0][0] = rd_w<SN>(A, 0, 0); nxt.x[0][0] = rd_x<SN>(A, 0, 0); } SB();
  MFMA(0, 1, 0); SB();
  if (READ) { nxt.x[1][0] = rd_x<SN>(A, 1, 0); nxt.w[1][0] = rd_w<SN>(A, 1, 0); } SB();
  MFMA(1, 0, 0); SB();
  if (LOAD) glds_pair<S>(L, 0);
  SB();
  MFMA(1, 1, 0); SB();
  if (READ) { nxt.w[0][1] = rd_w<SN>(A, 0, 1); nxt.x[0][1] = rd_x<SN>(A, 0, 1); } SB();
  MFMA(0, 0, 1); SB();
  if (READ) { nxt.x[1][1] = rd_x<SN>(A, 1, 1); nxt.w[1][1] = rd_w<SN>(A, 1, 1); } SB();
  MFMA(0, 1, 1); SB();
  if (LOAD) { glds_pair<S>(L, 1); L.base += L.stride[S & 1]; }
  SB();
  MFMA(1, 0, 1); SB();
  MFMA(1, 1, 1); SB();
}

template <int S> __device__ __forceinline__ void stage_load(Loader& L) {
  glds_pair<S>(L, 0); glds_pair<S>(L, 1); L.base += L.stride[S & 1];
}

__device__ __forceinline__ void gemm_phase(f32x16 (&acc)[2][2], const RdAddr& A, Loader& L, int n, u32 msk, int jm, int sstr) {
  Frag f0, f1;
  const char* const b0 = uni(L.base);
  L.base = b0 + (ptrdiff_t)(__builtin_ctz(msk) * sstr);
  L.stride[0] = __builtin_amdgcn_readfirstlane(L.stride[0]);
  L.stride[1] = __builtin_amdgcn_readfirstlane(L.stride[1]);
  asm volatile("s_nop 4" ::: "memory");
  stage_load<0>(L); stage_load<1>(L); stage_load<2>(L); stage_load<3>(L);
  waitv<12>();
  asm volatile("s_barrier" ::: "memory");
#pragma unroll
  for (int ks = 0; ks < 2; ++ks)
#pragma unroll
    for (int i = 0; i < 2; ++i) { f0.w[i][ks] = rd_w<0>(A, i, ks); f0.x[i][ks] = rd_x<0>(A, i, ks); }
  SB();
  for (int t = 0; t < n - 4; t += 4) {
    if ((t & jm) == (jm & ~3)) { msk &= msk - 1u; L.base = b0 + (ptrdiff_t)(__builtin_ctz(msk) * sstr); }
    kstep<0, 8, true, true>(acc, f0, f1, A, L);
    kstep<1, 8, true, true>(acc, f1, f0, A, L);
    kstep<2, 8, true, true>(acc, f0, f1, A, L);
    kstep<3, 8, true, true>(acc, f1, f0, A, L);
  }
  kstep<0, 8, false, true>(acc, f0, f1, A, L);
  kstep<1, 4, false, true>(acc, f1, f0, A, L);
  kstep<2, 0, false, true>(acc, f0, f1, A, L);
  kstep<3, 0, false, false>(acc, f1, f0, A, L);
}

__device__ __forceinline__ u32 pack2(float a, float b) {
  f32x2 t = {a, b};
  return __builtin_bit_cast(u32, __builtin_convertvector(t, bf16x2));
}
__device__ __forceinline__ u32 relu2(u32 v) {
  typedef short s16x2 __attribute__((ext_vector_type(2)));
  return __builtin_bit_cast(u32, __builtin_elementwise_max(__builtin_bit_cast(s16x2, v), s16x2{0, 0}));
}
__device__ __forceinline__ float bf_lo(u32 v) { return __builtin_bit_cast(float, v << 16); }
__device__ __forceinline__ float bf_hi(u32 v) { return __builtin_bit_cast(float, v & 0xffff0000u); }

__device__ __forceinline__ void zero8k(u32 l16, u32 l16b, char* pb) {
  const u32x4 z4 = {0u, 0u, 0u, 0u};
  asm volatile("s_nop 4" ::: "memory");
  st16<0>(l16, z4, pb); st16<1024>(l16, z4, pb); st16<2048>(l16, z4, pb); st16<3072>(l16, z4, pb);
  st16<0>(l16b, z4, pb); st16<1024>(l16b, z4, pb); st16<2048>(l16b, z4, pb); st16<3072>(l16b, z4, pb);
}

__device__ __forceinline__ void watch(char* myws, u32 lane, u64 tstart, int rk) {
  const u32 call = bump_get<0>(bump_issue(CNT_OFF + (u32)(blockIdx.x * 4) * 64u, myws));
  const u32 j = lane & 7u, s = min(lane >> 3, 5u);
  const u32 fo = fin_off(s * 32u + j + (j >= 5u ? 2u : 1u));
  const u64 pw = ((const GLBP u64*)(uintptr_t)kargs())[16u + 4u * s];
  bool told = false, sent = false;
  u32 it = 0, old = 0;
  sleep_until(tstart, (u32)WT0);
  for (;;) {
    const u32 t = (u32)now();
    u32 c = min(atom_rd(fo, myws) + 16u - 16u * call, 16u);
    c += (u32)__builtin_amdgcn_update_dpp(0, (int)c, 0xb1, 15, 15, true);
    c += (u32)__builtin_amdgcn_update_dpp(0, (int)c, 0x4e, 15, 15, true);
    c += (u32)__builtin_amdgcn_update_dpp(0, (int)c, 0x141, 15, 15, true);
    const u32 e = t - (c * (u32)TAU >> 4);
    if (c >= (u32)WMIN && (!told || (c < (u32)WTOP && e - old + (u32)WD > (u32)(2 * WD)))) {
      old = told ? old + (u32)((int)(e - old) >> 1) : e;
      told = true;
      if (lane < 48u) st_sys(myws + ORG_OFF + j * 256u + s * 4u, old);
    }
    const bool over = (u32)__builtin_amdgcn_readfirstlane(t - (u32)tstart) > (u32)(WNOM + WMAX);
    if (!sent && (over || __builtin_amdgcn_ballot_w64(c < (u32)BCT) == 0ull)) {
      const u32 h = s < 2u ? s ^ (u32)(rk & 1) : s & 1u, o = told ? old : t;
      const u32 of = (u32)__builtin_amdgcn_ds_bpermute((int)((h ^ (u32)(rk & 1)) * 32u), (int)o);
      int da = (int)((u32)__builtin_amdgcn_ds_bpermute((int)(64u + h * 32u), (int)o) - of);
      int db = (int)((u32)__builtin_amdgcn_ds_bpermute((int)(128u + h * 32u), (int)o) - of);
      const int hc = max(max(da, db), 0) * BFK >> 4;
      da = (da - hc) * BGN;
      db = (db - hc) * BGN;
      int ya = min(max(2 * da - db, -8192), 24576);
      int yb = min(max((3 * db - ya) >> 1, -16384), 16384);
      ya = min(max((3 * da - yb) >> 1, -8192), 24576);
      ya = (ya + 512) >> 10;
      yb = (yb + 512) >> 10;
      ya = 8 + (__builtin_abs(ya) > BDZ ? ya : 0);
      yb = 16 + (__builtin_abs(yb) > BDZ ? yb : 0);
      const u32 wd = (u32)(s < 2u ? ya | yb << 6 : (s < 4u ? ya : yb | 512)) | call << 12;
      if (!NOS && lane < 48u && j == 0u) st_sys((void*)(uintptr_t)(pw + dec_off(s)), wd);
      sent = true;
    }
    if (over || __builtin_amdgcn_ballot_w64(c < (u32)WTOP) == 0ull) break;
    __builtin_amdgcn_s_sleep(WSP);
  }
}

__device__ __forceinline__ void tile(const Args& P, int wave, u32 lane, char* smem, u64 tstart) {
  char* const myws = P.my;
  const int bid = blockIdx.x;
  const u32 cnt = bump_issue(CNT_OFF + (u32)(bid * 4 + wave) * 64u, myws);

  if (LPT && (bid >> 3) == 25 && wave == 0) {
    sleep_until(tstart, (u32)LPT);
    if (lane < 28u) probe(lane * 2u * MIB + MIB + (u32)(bid & 7) * 128u, myws);
  }
  if (RPT && (bid >> 3) == RPB && wave == 1) {
    const u32 dd = lane < 14u ? 1u + lane % 7u : 2u + (lane & 1u);
    const u64 pp = ((const GLBP u64*)(uintptr_t)kargs())[8u + 4u * dd];
    const u32 cl = bump_get<0>(cnt);
    u32 o = lane < 7u && dd > 1u ? (dd - 2u) * IN_SRC + 2u * MIB : PIN_OFF + (cl % 3u) * PIN_PAR + dd * 16u * PIECE;
    if (lane >= 16u) o = ((lane >> 1) - 4u) * 2u * MIB;
    sleep_until(tstart, (u32)RPT);
    if (lane < 32u) asm volatile("global_atomic_add %0, %1, off sc1" :: "v"(pp + o + (u32)(bid & 7) * 128u), "v"(0u) : "memory");
  }
  bool dead = false;
  int ra = P.rank;
  u32 jobs = P.m[0];
  int jo = 0;
#pragma unroll 1
  for (;;) {
  asm volatile("" : "+s"(jobs), "+s"(wave), "+s"(ra), "+v"(lane));
  const int r = ra;
  const int id = (int)(jobs & 255u), rb = id >> 4, k = id & 15;
  const bool sub = (jobs & 1024u) != 0u, own = (jobs & 0x700u) == 0x200u;
  const int sq = (int)(jobs >> 8) & 3;
  const int j = (int)(RB_J >> (4 * rb)) & 7;
  const int sr = own ? r ^ j : r;
  const int row0 = (sr ^ j) * 512 + ((int)((RB_BLK >> (2 * rb)) & 3u) + (j >= 4 ? sr & 2 : 0)) * 128;
  const int col0 = ((sr >> 2) * 16 + (k ^ col_x(sr))) * 128;
  const bool fin = own || j == 0;
  u32 need = own ? 0u : 63u & ~((1u << j) >> 2);
  const u64 tx = *(const CSTP u64*)(uintptr_t)(kargs() + offsetof(Params, arr) + (u32)cont_x(rb) * 8u);
  const u64 tw8 = *(const CSTP u64*)(uintptr_t)(kargs() + offsetof(Params, arr) + (u32)cont_w(k) * 8u);
  STAMPV(myws, jo + 7, (u64)(jobs & 0xffffu));

  const u32 sbase = (u32)(uintptr_t)smem;
  const int wm = wave >> 1, wn = wave & 1;
  const u32 hb = sub ? ((lane >> 3) ^ (lane >> 5)) & 1u : 0u;
  RdAddr A;
  {
    const u32 h = lane >> 5, m = lane & 31u, f = (lane >> 2) & 3u;
    const u32 xr = (u32)(sub ? wave : wm * 2) * 32u + m;
    A.xa[0] = sbase + xr * 64u + 16u * (h ^ f);
    A.xa[1] = sbase + xr * 64u + 16u * ((2u + h) ^ f);
    const u32 blk = (lane >> 4) & 1u, q = (lane & 15u) >> 2, pp = lane & 3u;
#pragma unroll
    for (int nt = 0; nt < 2; ++nt)
#pragma unroll
      for (int t = 0; t < 2; ++t) {
        const u32 row = 8u * h + 4u * t + q;
        const u32 ch = ((u32)(sub ? wm * 12 : wn * 8) ^ ((u32)nt * 4u)) + 2u * blk + (pp & 1u);
        A.wa[nt][t] = sbase + 256u * row + 16u * (ch ^ (((row & 3u) << 2) | ((row >> 2) & 3u))) + 8u * (pp >> 1);
      }
  }

  f32x16 acc[2][2];
#pragma unroll
  for (int i = 0; i < 2; ++i)
#pragma unroll
    for (int k = 0; k < 2; ++k)
#pragma unroll
      for (int z = 0; z < 16; ++z) acc[i][k][z] = 0.f;

  const u32 l16 = lane * 16u, l16b = lane * 16u + 4096u;
  const u32 tw = sbase + (u32)wave * 8192u + (lane & 31u) * 128u + (((lane >> 5) ^ (lane & 7u)) * 16u);
  const u32 tr = sbase + (u32)wave * 8192u + (lane >> 3) * 128u + (((lane & 7u) ^ (lane >> 3)) * 16u);
  const u32 oo = (lane >> 3) * 8192u + (lane & 7u) * 16u;
  char* const ob = uni(P.out + (size_t)((u32)((row0 & 511) + wm * 64) * 8192u + (u32)(col0 + wn * 64) * 2u));

  if (jobs & 2048u) __builtin_amdgcn_s_setprio(SPR);
  else if (PR3 && !fin) __builtin_amdgcn_s_setprio(1);
  else __builtin_amdgcn_s_setprio(0);
  sleep_until(tstart, ((jobs >> 12) & 15u) * (u32)PDU);
  Loader L;
  L.m0base = sbase + (u32)wave * 4096u;
  if (wave < 2) {
    L.base = P.x + (size_t)((u32)row0 * 1024u + (sub ? (u32)(sq >> 1) * 65536u + (u32)wave * 64u : (u32)wave * 65536u));
    L.stride[0] = L.stride[1] = sub ? 128u : 64u;
    const u32 c = (lane & 3u) ^ ((lane >> 4) & 3u);
#pragma unroll
    for (int k = 0; k < 4; ++k) L.v[k] = (lane >> 2) * 1024u + c * 16u + (u32)k * (16384u - 1024u);
  } else {
    L.base = P.w + (size_t)((u32)(wave - 2) * 16u * 8192u + (u32)col0 * 2u + (sub ? (u32)(sq & 1) * 128u : 0u));
    L.stride[0] = L.stride[1] = sub ? 64u * 8192u : 32u * 8192u;
    const u32 q = lane >> 4;
#pragma unroll
    for (int k = 0; k < 4; ++k)
      L.v[k] = q * 8192u + (((lane & 15u) ^ ((q << 2) | (u32)k)) * 16u) + (u32)k * (4u * 8192u - 1024u) + hb * (32u * 8192u - 128u);
  }
  u32 call = 0, msk = 1u, fo = 0, tl = 0, thr = 0, tq = 0;
  const u32 org = ORG_OFF + (u32)(bid & 7) * 256u + (lane & 8u ? (lane ^ (u32)r) & 1u : lane & 7u) * 4u;
#define EXPECT(OG, EL) do { \
    const u32 dl_ = (OG) - (u32)tstart; \
    unk = dl_ + 1000u >= 4000u; \
    const u32 o_ = !unk ? dl_ : (u32)(EIN + T0 + OL) + min(max((EL), (u32)WNOM) - (u32)WNOM, (u32)WMAX); \
    tl = (u32)max((int)(o_ + tq), 0); \
    unk = unk && (EL) < (u32)(WNOM + WMAX + 100); \
  } while (0)
  int n = sub ? 8 : 16, jm = 63, sstr = 0;
  int pi0 = -1, pi1 = -1;
  bool pend = false, early = false, unk = false;
  u32 dir = 0;
#pragma unroll 1
  for (;;) {
    if (n) gemm_phase(acc, A, L, n, msk, jm, sstr);
    const bool first = jm == 63;
    if (first) {
      STAMP(myws, jo + 1);
      const u32 og = WCH ? ld_sys(myws + org) : 0u;
      call = bump_get<0>(cnt);
      const u32 ov = *(const GLBP u8*)(uintptr_t)(kargs() + offsetof(Params, ot) + (own ? (u32)(j - 1) * 64u + lane + (lane & 32u) : lane));
      if (DIR && !fin) {
        const u64 m = __builtin_amdgcn_ballot_w64(ov == (u32)id) & DMK;
        if (m) {
          dir = PIN_OFF + (call % 3u) * PIN_PAR + (u32)__builtin_ctzll(m) * 32768u;
          probe0(dir + (u32)wave * 8192u, *(char* const CSTP*)(uintptr_t)(kargs() + offsetof(Params, lk) + 32u));
        }
      }
      if (!sub) {
        const int pr = rb >= 13 ? rb - 13 : (rb < 3 ? 14 : ((rb + 1) ^ 1) - 1);
        const u64 m = __builtin_amdgcn_ballot_w64(ov == (u32)(own && lane < 32u ? id : pr * 16 + (k ^ 15)));
        if (m) {
          const int u = __builtin_ctzll(m), v = 63 - __builtin_clzll(m), bs = own ? j * 32 : 0;
          const bool sw = (u & 31) > (v & 31);
          pi1 = bs + (sw ? u : v);
          if (u != v) pi0 = bs + (sw ? v : u);
          char* const pz = myws + (size_t)(PIN_OFF + ((call + 2u) % 3u) * PIN_PAR + (u32)wave * 8192u);
          sleep_until(tstart, (u32)ZT0 + (u32)((bid >> 3) & 15) * (u32)ZDT);
          if (pi0 >= 0) zero8k(l16, l16b, uni(pz + (size_t)((u32)pi0 * 32768u)));
          zero8k(l16, l16b, uni(pz + (size_t)((u32)pi1 * 32768u)));
        }
      }
      const u32 sl = lane & 7u, qx = (u32)(tx >> (8u * sl)) & 255u, qf = (u32)(tw8 >> (8u * sl)) & 255u, qw = qf & 63u;
      const bool usew = (lane & 8u) || (qf < 128u && qw > qx);
      pend = lane < 16u && ((need >> sl) & 1u) && (lane < 8u || qf >= 128u);
      fo = fin_off(sl * 32u + (u32)(usew ? cont_w(k) : cont_x(rb)));
      const u32 qq = usew ? qw : qx;
      tl = (u32)EIN + TQ(qq) + (AQ && qq >= AQ ? AGD : 0);
      tq = (u32)(TQ(qq) - T0 - OL);
      if (WCH) EXPECT(og, 0u);
      thr = 16u * call;
      if (pi1 >= 0 && !own) {
        need |= 64u;
        if ((lane >> 3) == 2u) {
          pend = true;
          unk = false;
          fo = PIN_OFF + (call % 3u) * PIN_PAR + (u32)pi1 * 32768u + sl * 4096u + 4092u;
          tl = (u32)(L1A + (pi1 >> 1) * TAU1);
          thr = 1u;
        }
      }
      jm = sub ? 3 : 7;
#pragma unroll
      for (int k = 0; k < 4; ++k) L.v[k] = sub && wave >= 2 ? (l16 ^ ((hb ^ (u32)(sq & 1)) << 7)) + hb * 8192u : l16;
    }
    const bool last = (need & 63u) == 0u;
    if (last) STAMP(myws, jo + 3);
    const u32 pin = PIN_OFF + (call % 3u) * PIN_PAR + (u32)wave * 8192u;
#pragma unroll 1
    for (int s = 0; s < 2; ++s) {
      if (s == 1 && !last) break;
      const int pi = s == 0 && !early ? pi0 : pi1;
      if (pi < 0 && !(fin && (first || s == 1))) continue;
      if (s == 0 && early) pi1 = -1;
      u32x4 pv[8];
#pragma unroll
      for (int i = 0; i < 8; ++i) pv[i] = u32x4{0u, 0u, 0u, 0u};
      if (pi >= 0 || dead) {
        char* const pb = uni(myws + (size_t)(pin + (u32)pi * 32768u));
        const u32 wo = pin + (u32)(pi - (int)(lane & 2u)) * 32768u + (lane & 1u) * 4096u + 4092u;
        const int tn = (pi & 31) >> 1;
        const bool lazy = own ? tn < LZ : (!fin || pi < 2 * (N1 - LZ1));
        if (own) sleep_until(tstart, (u32)(EARLY + TQ(NT - 18 + tn) + (lazy ? LZD : 0)));
        const u64 t0 = now();
        while (!dead && !early) {
          u32 v = 0u;
          if (lane < (lazy ? 2u : 4u)) v = atom_rd(wo, myws);
          if (__builtin_amdgcn_ballot_w64(v != 0u) >= (lazy ? 3ull : 1ull)) break;
          if (give_up(t0, myws, call)) { dead = true; break; }
          if (lazy && (u32)__builtin_amdgcn_readfirstlane((u32)(now() - tstart)) < (u32)LZE) __builtin_amdgcn_s_sleep(LZS);
          __builtin_amdgcn_s_sleep(WSL);
        }
        if (s == 1) STAMP(myws, jo + 4);
        const bool two = fin && s == 1 && !lazy;
        for (;;) {
          if (__builtin_expect(dead, 0)) {
            if (lane == 0) st_agt(myws + TMO_OFF, call);
#pragma unroll
            for (int i = 0; i < 8; ++i) pv[i] = u32x4{0x7fc07fc0u, 0x7fc07fc0u, 0x7fc07fc0u, 0x7fc07fc0u};
            break;
          }
          asm volatile("s_nop 4\n\t"
                       "global_load_dwordx4 %0, %8, %10 sc0 sc1\n\t"
                       "global_load_dwordx4 %1, %8, %10 offset:1024 sc0 sc1\n\t"
                       "global_load_dwordx4 %2, %8, %10 offset:2048 sc0 sc1\n\t"
                       "global_load_dwordx4 %3, %8, %10 offset:3072 sc0 sc1\n\t"
                       "global_load_dwordx4 %4, %9, %10 sc0 sc1\n\t"
                       "global_load_dwordx4 %5, %9, %10 offset:1024 sc0 sc1\n\t"
                       "global_load_dwordx4 %6, %9, %10 offset:2048 sc0 sc1\n\t"
                       "global_load_dwordx4 %7, %9, %10 offset:3072 sc0 sc1"
                       : "=&v"(pv[0]), "=&v"(pv[1]), "=&v"(pv[2]), "=&v"(pv[3]), "=&v"(pv[4]), "=&v"(pv[5]), "=&v"(pv[6]), "=&v"(pv[7])
                       : "v"(l16), "v"(l16b), "s"(pb) : "memory");
          const bool late = __builtin_amdgcn_readfirstlane((int)(now() - t0 > 50000ull)) != 0;
          asm volatile("s_waitcnt vmcnt(0)"
                       : "+v"(pv[0]), "+v"(pv[1]), "+v"(pv[2]), "+v"(pv[3]), "+v"(pv[4]), "+v"(pv[5]), "+v"(pv[6]), "+v"(pv[7]) :: "memory");
          u32 mn = 0xffffffffu;
#pragma unroll
          for (int i = 0; i < 8; ++i)
#pragma unroll
            for (int z = 0; z < 4; ++z) mn = min(mn, pv[i][z]);
          if (__builtin_amdgcn_ballot_w64(mn == 0u) == 0ull) break;
          if (__builtin_expect(late, 0)) dead = give_up(t0, myws, call);
        }
      }
#pragma unroll
      for (int nt = 0; nt < 2; ++nt)
#pragma unroll
        for (int mt = 0; mt < 2; ++mt)
#pragma unroll
          for (int sx = 0; sx < 2; ++sx)
#pragma unroll
            for (int z = 0; z < 4; ++z) {
              const u32 u = pv[(nt * 2 + mt) * 2 + sx][z];
              acc[nt][mt][8 * sx + 2 * z] += bf_lo(u);
              acc[nt][mt][8 * sx + 2 * z + 1] += bf_hi(u);
            }
      if (!fin || (s == 0 && !first)) continue;
      if (s == 1) STAMP(myws, jo + 6);
#pragma unroll
      for (int nt = 0; nt < 2; ++nt)
#pragma unroll
        for (int mt = 0; mt < 2; ++mt)
#pragma unroll
          for (int sx = 0; sx < 2; ++sx) {
            u32x4 v;
#pragma unroll
            for (int z = 0; z < 4; ++z) v[z] = relu2(pack2(acc[nt][mt][8 * sx + 2 * z], acc[nt][mt][8 * sx + 2 * z + 1]));
            *(volatile LDSP u32x4*)(uintptr_t)((tw ^ (u32)((nt * 4 + sx * 2) * 16)) + (u32)mt * 4096u) = v;
          }
      u32 o1, o2, o3, o4, o5, o6, o7;
      asm volatile("ds_read_b128 %0, %15\n\t"
                   "ds_read_b128 %1, %15 offset:1024\n\t"
                   "ds_read_b128 %2, %15 offset:2048\n\t"
                   "ds_read_b128 %3, %15 offset:3072\n\t"
                   "ds_read_b128 %4, %15 offset:4096\n\t"
                   "ds_read_b128 %5, %15 offset:5120\n\t"
                   "ds_read_b128 %6, %15 offset:6144\n\t"
                   "ds_read_b128 %7, %15 offset:7168\n\t"
                   "v_add_u32 %8, 0x10000, %16\n\t"
                   "v_add_u32 %9, 0x20000, %16\n\t"
                   "v_add_u32 %10, 0x30000, %16\n\t"
                   "v_add_u32 %11, 0x40000, %16\n\t"
                   "v_add_u32 %12, 0x50000, %16\n\t"
                   "v_add_u32 %13, 0x60000, %16\n\t"
                   "v_add_u32 %14, 0x70000, %16\n\t"
                   "s_mov_b64 exec, %18\n\t"
                   "s_waitcnt lgkmcnt(7)\n\tglobal_store_dwordx4 %16, %0, %17 sc0 sc1\n\t"
                   "s_waitcnt lgkmcnt(6)\n\tglobal_store_dwordx4 %8, %1, %17 sc0 sc1\n\t"
                   "s_waitcnt lgkmcnt(5)\n\tglobal_store_dwordx4 %9, %2, %17 sc0 sc1\n\t"
                   "s_waitcnt lgkmcnt(4)\n\tglobal_store_dwordx4 %10, %3, %17 sc0 sc1\n\t"
                   "s_waitcnt lgkmcnt(3)\n\tglobal_store_dwordx4 %11, %4, %17 sc0 sc1\n\t"
                   "s_waitcnt lgkmcnt(2)\n\tglobal_store_dwordx4 %12, %5, %17 sc0 sc1\n\t"
                   "s_waitcnt lgkmcnt(1)\n\tglobal_store_dwordx4 %13, %6, %17 sc0 sc1\n\t"
                   "s_waitcnt lgkmcnt(0)\n\tglobal_store_dwordx4 %14, %7, %17 sc0 sc1\n\t"
                   "s_cmp_lg_u64 %18, -1\n\t"
                   "s_cbranch_scc1 1f\n\t"
                   "s_endpgm\n"
                   "1:\n\t"
                   "s_mov_b64 exec, -1\n\t"
                   "s_nop 1"
                   : "=&v"(pv[0]), "=&v"(pv[1]), "=&v"(pv[2]), "=&v"(pv[3]), "=&v"(pv[4]), "=&v"(pv[5]), "=&v"(pv[6]), "=&v"(pv[7]),
                     "=&v"(o1), "=&v"(o2), "=&v"(o3), "=&v"(o4), "=&v"(o5), "=&v"(o6), "=&v"(o7)
                   : "v"(tr), "v"(oo), "s"(ob), "s"(s == 1 ? ~0ull : (u64)WMK) : "memory", "scc");
      if (s == 0) waitv<0>();
    }
    if (last) break;
    if (wave == 0) {
      const u64 t0 = now();
      u32 st;
      for (;;) {
        const u32 el = (u32)__builtin_amdgcn_readfirstlane((u32)(now() - tstart));
        const bool ask = WCH && unk;
        if (pend && (ask || el >= tl)) {
          const u32 v = atom_rd(ask ? org : fo, myws);
          if (ask) EXPECT(v, el); else pend = v < thr;
        }
        const u32 pm = (u32)__builtin_amdgcn_ballot_w64(pend);
        st = need & ~(pm | ((pm >> 8) & 63u) | (pm >> 16 ? 64u : 0u));
        if ((st & 64u) || (st && (__builtin_amdgcn_ballot_w64(pend && el + (u32)PLD - tl < (u32)(PHI + PLD)) & 0xffffull) == 0ull)) break;
        if (give_up(t0, myws, call)) {
          if (lane == 0) st_agt(myws + TMO_OFF, call);
          st = 0;
          break;
        }
        if (__builtin_amdgcn_ballot_w64(pend && el + 40u >= tl) == 0ull) __builtin_amdgcn_s_sleep(8);
        __builtin_amdgcn_s_sleep(PSL);
      }
      if (lane == 0) *(LDSP u32*)(uintptr_t)(sbase + 4u * STAGE_B + ((need >> 4) & 4u)) = st;
      __builtin_amdgcn_s_waitcnt(0);
    }
    asm volatile("s_barrier" ::: "memory");
    msk = lds_rd(sbase + 4u * STAGE_B + ((need >> 4) & 4u));
    if (!msk) { dead = true; need = 0u; }
    early = msk >= 64u;
    need &= ~msk;
    n = __builtin_popcount(msk & 63u) * (jm + 1);
    L.base = myws + (size_t)(((u32)(wave < 2 ? cont_x(rb) : cont_w(k)) * 2u + (call & 1u)) * PIECE
                             + (u32)(sub && wave < 2 ? (sq >> 1) + wave * 2 : wave & 1) * 4096u);
    L.stride[0] = L.stride[1] = sub ? 16384u : 8192u;
    sstr = (int)IN_SRC;
    if (RK && sub && ((sq ^ (sq >> 1)) & 1)) {
      msk = __builtin_bitreverse32(msk) >> 26;
      L.base += 5u * IN_SRC;
      sstr = -(int)IN_SRC;
    }
  }

  if (!fin) {
    STAMP(myws, jo + 6);
    char* dst = myws + (size_t)(POUT_OFF + (u32)id * 32768u);
    u64 slot = 0;
    if (DIR && dir) {
      dst = *(char* const CSTP*)(uintptr_t)(kargs() + offsetof(Params, lk) + 32u) + (size_t)dir;
      asm volatile("s_mov_b64 exec, 1\n\ts_nop 3\n\t"
                   "global_atomic_umax_x2 %0, %1, %3 sc1\n\t"
                   "global_atomic_add_x2 v[200:201], %0, %2, %3 sc0 sc1\n\t"
                   "s_mov_b64 exec, -1"
                   :: "v"(TOK_OFF), "v"((now() + (u64)DRT) << 5), "v"((u64)(sub ? DTAU : 4 * DTAU)), "s"(myws) : "memory", "v200", "v201");
    }
    dst = uni(dst + (size_t)(sub ? (u32)(sq * 4 + wave) * 2048u : (u32)wave * 8192u));
    char* fl = myws + (size_t)(FLOC_OFF + (u32)(rb * 4 + (sub ? sq : wave)) * FLS + (u32)k * 4u);
    if (sub) {
      typedef float f32x4 __attribute__((ext_vector_type(4)));
#pragma unroll
      for (int i = 0; i < 4; ++i)
        *(volatile LDSP f32x4*)(uintptr_t)(L.m0base + l16 + (u32)i * 1024u) =
            f32x4{acc[1][0][4 * i], acc[1][0][4 * i + 1], acc[1][0][4 * i + 2], acc[1][0][4 * i + 3]};
      __builtin_amdgcn_s_waitcnt(15 | (7 << 4) | (0 << 8) | (3 << 14));
      asm volatile("s_barrier" ::: "memory");
#pragma unroll
      for (int i = 0; i < 4; ++i) {
        const f32x4 t = *(volatile LDSP f32x4*)(uintptr_t)((sbase + (u32)(wave ^ 2) * 4096u) + l16 + (u32)i * 1024u);
#pragma unroll
        for (int z = 0; z < 4; ++z) acc[0][0][4 * i + z] += t[z];
      }
    }
    u32 least = 1u;
    if (__builtin_expect(dead, 0)) {
      least = 0x7fc07fc0u;
#pragma unroll
      for (int i = 0; i < 64; ++i) acc[i >> 5][(i >> 4) & 1][i & 15] = 0.f;
    }
    u32x4 pk[8];
#pragma unroll
    for (int i = 0; i < 8; ++i)
#pragma unroll
      for (int z = 0; z < 4; ++z)
        pk[i][z] = max(pack2(acc[i >> 2][(i >> 1) & 1][8 * (i & 1) + 2 * z], acc[i >> 2][(i >> 1) & 1][8 * (i & 1) + 2 * z + 1]), least);
    if (DIR && dir) {
      u32 lo, hi;
      asm volatile("s_setprio 3\n\ts_waitcnt vmcnt(0)\n\tv_readfirstlane_b32 %0, v200\n\tv_readfirstlane_b32 %1, v201\n\ts_nop 4"
                   : "=s"(lo), "=s"(hi), "+v"(pk[0]), "+v"(pk[1]), "+v"(pk[2]), "+v"(pk[3]), "+v"(pk[4]), "+v"(pk[5]), "+v"(pk[6]), "+v"(pk[7]) :: "memory");
      slot = ((u64)hi << 32 | lo) >> 5;
      STAMPV(myws, jo + 2, slot);
      STAMP(myws, jo + 4);
      sleep_exact(tstart, min((u32)(slot - tstart), 20000u));
    }
    asm volatile("s_nop 4" ::: "memory");
    st16<0>(l16, pk[0], dst);
    st16<1024>(l16, pk[1], dst);
    if (!sub) {
      st16<2048>(l16, pk[2], dst);
      st16<3072>(l16, pk[3], dst);
      if (DIR && dir) __builtin_amdgcn_s_sleep(DSP);
      st16<0>(l16b, pk[4], dst);
      st16<1024>(l16b, pk[5], dst);
      st16<2048>(l16b, pk[6], dst);
      st16<3072>(l16b, pk[7], dst);
    }
    if (!(DIR && dir)) {
      waitv<0>();
      if (sub) asm volatile("s_barrier" ::: "memory");
      if (lane == 0) st_sys(fl, call);
    }
    asm volatile("s_barrier" ::: "memory");
  }
  STAMP(myws, jo + 5);
  STAMPV(myws, jo + 0, tstart);
  jobs >>= 16;
  jo = 8;
  if (!jobs) break;
  }
}

__global__ __launch_bounds__(256, 2) void fused_gemm_rs(Params P) {
  extern __shared__ __attribute__((aligned(16))) char smem[];
  const int bid = blockIdx.x;
  typedef u64 u64x4 __attribute__((ext_vector_type(4)));
  u64x4 a4;
  u64 tstart, pe;
  int rk, d;
  u32 tis, t0;
  u32x16 f16;
  u32x2 co;
  u32x4 mt;
  static_assert(N_PUSH_WG == 56 && CPB >= N_PUSH_WG && offsetof(Params, lk) == 64 && sizeof(Params) <= 4096, "entry code");
  asm volatile("v_readfirstlane_b32 s46, %[tid]\n\t"
               "s_load_dwordx8 s[36:43], %[ka], 0x0\n\t"
               "s_lshr_b32 s47, %[bid], 3\n\t"
               "s_cmp_gt_u32 %[bid], 55\n\t"
               "s_cbranch_scc1 2f\n\t"
               "s_lshr_b32 s48, s46, 7\n\t"
               "s_mul_i32 s49, s48, 3\n\t"
               "s_add_i32 s47, s47, s49\n\t"
               "s_add_i32 s49, s47, -7\n\t"
               "s_min_u32 s47, s47, s49\n\t"
               "s_lshl_b32 s49, s47, 5\n\t"
               "s_load_dwordx8 s[68:75], %[ka], s49 offset:0x60\n\t"
               "s_memrealtime s[44:45]\n\t"
               "s_setprio 3\n\t"
               "s_add_i32 s63, s47, 1\n\t"
               "v_lshlrev_b32 v1, 2, %[tid]\n\t"
               "v_mov_b32 v27, 0\n\t"
               "v_add_u32 v1, %[ctl], v1\n\t"
               "s_bitcmp1_b32 s46, 6\n\t"
               "ds_write_b32 v1, v27\n\t"
               "s_cbranch_scc0 1f\n\t"
               "s_lshl_b32 s49, %[bid], 2\n\t"
               "s_lshr_b32 s49, %[pxm], s49\n\t"
               "s_and_b32 s49, s49, 7\n\t"
               "s_lshl1_add_u32 s48, s49, s48\n\t"
               "v_bfe_u32 v1, %[tid], 4, 2\n\t"
               "v_and_b32 v2, 15, %[tid]\n\t"
               "v_lshlrev_b32 v3, 2, v1\n\t"
               "v_lshlrev_b32 v1, 13, v1\n\t"
               "v_xor_b32 v2, v3, v2\n\t"
               "v_lshl_or_b32 v1, s48, 17, v1\n\t"
               "v_lshl_or_b32 v20, v2, 4, v1\n\t"
               "v_xor_b32 v21, 16, v20\n\t"
               "v_xor_b32 v22, 32, v20\n\t"
               "v_xor_b32 v23, 48, v20\n\t"
               "v_add_u32 v21, 0x7c00, v21\n\t"
               "v_add_u32 v22, 0xf800, v22\n\t"
               "v_add_u32 v23, 0x17400, v23\n\t"
               "s_lshl_b32 s49, s63, 7\n\t"
               "s_add_i32 s49, s49, %[lnk]\n\t"
               "v_mov_b32 v28, s49\n\t"
               "v_mov_b32 v25, 1\n\t"
               "s_waitcnt lgkmcnt(0)\n\t"
               "s_memrealtime s[58:59]\n\t"
               "global_load_dwordx4 v[144:147], v20, s[70:71] " MLP "\n\t"
               "global_load_dwordx4 v[148:151], v21, s[70:71] offset:1024 " MLP "\n\t"
               "global_load_dwordx4 v[152:155], v22, s[70:71] offset:2048 " MLP "\n\t"
               "global_load_dwordx4 v[156:159], v23, s[70:71] offset:3072 " MLP "\n\t"
               "s_mov_b64 exec, 1\n\t"
               "global_atomic_add v160, v28, v25, s[40:41] sc0 sc1\n\t"
               "s_add_i32 s50, s47, -1\n\t"
               "s_max_i32 s50, s50, 0\n\t"
               "s_mul_i32 s51, s50, %[fsl]\n\t"
               "s_lshl_b32 s49, s48, 7\n\t"
               "s_add_i32 s51, s51, s49\n\t"
               "s_add_i32 s51, s51, %[f0]\n\t"
               "v_mov_b32 v29, s51\n\t"
               "s_waitcnt lgkmcnt(0)\n\t"
               "s_or_b32 s66, s58, 1\n\t"
               "v_mov_b32 v26, s66\n\t"
               "global_atomic_cmpswap v161, v28, v[26:27], s[40:41] offset:4 sc0 sc1\n\t"
               "global_atomic_add v29, v27, s[68:69] sc1\n\t"
               "s_mov_b64 exec, -1\n"
               "1:\n\t"
               "s_waitcnt lgkmcnt(0)\n\t"
               "s_barrier\n\t"
               "s_branch 5f\n"
               "2:\n\t"
               "s_lshl_b32 s49, %[bid], 2\n\t"
               "s_memrealtime s[44:45]\n\t"
               "s_load_dword s62, %[ka], 0x20\n\t"
               "s_load_dword s72, %[ka], s49 offset:%[wg]\n\t"
               "s_add_i32 s47, %[bid], %[cpb]\n\t"
               "s_cmp_lt_u32 s47, 16\n\t"
               "s_cbranch_scc0 5f\n\t"
               "s_cmpk_lt_u32 s46, 0xc0\n\t"
               "s_cbranch_scc0 5f\n\t"
               "s_getpc_b64 s[56:57]\n\t"
               "s_mov_b32 m0, %[cpl]\n\t"
               "v_lshlrev_b32 v1, 7, %[tid]\n\t"
               "v_min_u32 v1, %[cpf], v1\n\t"
               "global_load_lds_dword v1, s[56:57]\n"
               "5:\n\t"
               "s_waitcnt lgkmcnt(0)"
               : "={s[36:43]}"(a4), "={s[44:45]}"(tstart), "={s46}"(t0), "={s[68:69]}"(pe), "={s62}"(rk), "={s63}"(d),
                 "={s66}"(tis), "={s[72:75]}"(mt)
               : [ka] "s"(__builtin_amdgcn_kernarg_segment_ptr()), [bid] "s"(bid), [tid] "v"(threadIdx.x),
                 [ctl] "i"(65536), [pxm] "i"(PXM), [lnk] "i"(LNK_OFF), [fsl] "i"(FINS), [f0] "i"(FIN0 + 4096u),
                 [cpb] "i"(-CPB), [cpl] "i"(66048), [cpf] "i"(CPF - 128), [wg] "i"(offsetof(Params, wg) - 4 * N_PUSH_WG)
               : "memory", "scc", "s47", "s48", "s49", "s50", "s51", "s56", "s57", "s58", "s59", "s70", "s71",
                 "v1", "v2", "v3", "v20", "v21", "v22", "v23", "v25", "v26", "v27", "v28", "v29");
  asm volatile("" : "={v[144:159]}"(f16), "={v[160:161]}"(co));
  const u32 lane = threadIdx.x & 63u;
  const int wave = (int)(t0 >> 6);
  const Args A = {(const char*)a4[0], (const char*)a4[1], (char*)a4[2], (char*)a4[3], (char*)pe, rk, tis, f16, co, mt};
  if (bid < N_PUSH_WG) mover(A, d, wave, lane, smem, tstart);
  else if (mt[0] == 0u) {
    if (WCH && bid == 256 && wave == 0) watch(A.my, lane, tstart, rk);
    for (int i = 0; i < 6; ++i) __builtin_amdgcn_s_sleep(16);
  } else tile(A, wave, lane, smem, tstart);
}

}

extern "C" void kernel_launch(int rank, int nranks, ncclComm_t comm,
                              hipStream_t stream,
                              void* const* d_in, const int* in_sizes, int n_in,
                              void* d_out, int out_size,
                              const int* in_dtypes, int out_dtype,
                              void* const* d_out_all,
                              void* const* d_ws_all, size_t ws_bytes) {
  if (nranks != NR || n_in < 2 || ws_bytes < (size_t)WS_NEED || (unsigned)rank >= (unsigned)NR) {
    fprintf(stderr, "unsupported configuration\n");
    return;
  }
  const u8 ORD[3][39] = {
      {0, 18, 16, 2, 1, 17, 5, 4, 3, 24, 28, 21, 22, 27, 26, 11, 23, 8, 25, 9, 7, 20, 10, 36, 37, 35, 19, 34, 12, 33, 6, 32, 14, 15, 13, 30, 29, 31, 255},
      {0, 18, 16, 2, 1, 17, 5, 4, 3, 22, 23, 21, 24, 11, 26, 28, 25, 8, 27, 7, 19, 10, 13, 12, 20, 6, 9, 14, 15, 30, 31, 29, 255},
      {0, 18, 16, 2, 1, 17, 5, 4, 3, 21, 28, 27, 26, 13, 22, 24, 23, 25, 11, 10, 20, 19, 12, 7, 8, 6, 9, 14, 15, 30, 31, 29, 255},
  };
  const int OM[4] = {13, 15, 13, 15}, QS[3] = {26, 26, 26};
  const u8 OT[7 * 64] = {
      32, 0, 2, 34, 1, 33, 16, 17, 5, 21, 37, 36, 3, 19, 35, 18, 80, 81, 4, 20, 96, 97, 192, 160, 128, 129, 112, 113, 11, 176, 161, 144,
      27, 43, 145, 8, 40, 7, 23, 10, 24, 39, 26, 42, 25, 28, 44, 9, 41, 38, 12, 30, 46, 6, 22, 15, 13, 45, 14, 31, 47, 29, 255, 255,
      255, 255, 255, 255, 255, 255, 255, 255, 255, 255, 255, 255, 255, 255, 255, 255, 48, 49, 50, 51, 52, 53, 59, 55, 56, 58, 60, 57, 54, 62, 63, 61,
      255, 255, 255, 255, 255, 255, 255, 255, 255, 255, 255, 255, 255, 255, 255, 255, 255, 255, 255, 255, 255, 255, 255, 255, 255, 255, 255, 255, 255, 255, 255, 255,
      255, 255, 255, 255, 255, 255, 255, 255, 255, 255, 255, 255, 255, 255, 255, 255, 64, 65, 66, 67, 68, 69, 75, 74, 71, 76, 72, 73, 70, 78, 79, 77,
      255, 255, 255, 255, 255, 255, 255, 255, 255, 255, 255, 255, 255, 255, 255, 255, 255, 255, 255, 255, 255, 255, 255, 255, 255, 255, 255, 255, 255, 255, 255, 255,
      255, 255, 255, 255, 82, 83, 84, 85, 91, 146, 147, 148, 149, 155, 88, 152, 90, 154, 87, 151, 89, 153, 156, 92, 86, 150, 94, 158, 95, 159, 93, 157,
      255, 255, 255, 255, 255, 255, 255, 255, 255, 255, 255, 255, 255, 255, 255, 255, 255, 255, 255, 255, 255, 255, 255, 255, 255, 255, 255, 255, 255, 255, 255, 255,
      255, 255, 255, 255, 98, 99, 100, 101, 162, 163, 164, 165, 107, 171, 104, 168, 106, 170, 103, 167, 105, 169, 108, 172, 102, 166, 110, 174, 111, 175, 109, 173,
      255, 255, 255, 255, 255, 255, 255, 255, 255, 255, 255, 255, 255, 255, 255, 255, 255, 255, 255, 255, 255, 255, 255, 255, 255, 255, 255, 255, 255, 255, 255, 255,
      255, 255, 255, 114, 115, 116, 117, 177, 178, 179, 180, 181, 123, 187, 120, 119, 183, 184, 122, 186, 121, 185, 124, 188, 118, 182, 126, 190, 127, 191, 125, 189,
      255, 255, 255, 255, 255, 255, 255, 255, 255, 255, 255, 255, 255, 255, 255, 255, 255, 255, 255, 255, 255, 255, 255, 255, 255, 255, 255, 255, 255, 255, 255, 255,
      255, 255, 255, 130, 131, 132, 133, 193, 194, 195, 196, 197, 139, 203, 136, 200, 199, 135, 138, 202, 137, 201, 140, 204, 134, 198, 142, 206, 143, 207, 141, 205,
      255, 255, 255, 255, 255, 255, 255, 255, 255, 255, 255, 255, 255, 255, 255, 255, 255, 255, 255, 255, 255, 255, 255, 255, 255, 255, 255, 255, 255, 255, 255, 255,};
  const u32 WG[N_WG - N_PUSH_WG] = {
      0xd90, 0xf0a, 0x914, 0x1c1e, 0x11bb, 0xb2b50c21, 0x1c8, 0x114d,
      0x194, 0x991, 0x164, 0x32530d13, 0x5f280d24, 0x51be, 0x1cb, 0x182,
      0x1154, 0xd1d30c02, 0x169, 0x1aa, 0x11ff, 0x117d, 0x29e, 0x113a,
      0x92701e00, 0x19e, 0xda0, 0x1f17, 0x1174, 0x1fa, 0x27e, 0x232,
      0x1d5, 0x15a, 0x1e8, 0x21e1, 0x1f9, 0x2d71, 0x1980, 0x0,
      0x19b, 0x11dd, 0x116b, 0x216c, 0x1c28, 0xe2c, 0x1189, 0x18d,
      0x11d4, 0xd02, 0x1d10, 0x6d1e, 0x4c70, 0x1ba, 0xfc0, 0x277,
      0xe90, 0x157, 0x1f10, 0x2f1e, 0x2e70, 0x2f2c, 0x138, 0x236,
      0x909, 0xae0d1c03, 0x21e6, 0x225b1e61, 0xe2bc0e20, 0x17e, 0x1c5, 0x131,
      0xc158, 0x82590d51, 0x627a2c60, 0xe17, 0xf70, 0x21bd, 0x81482ec0, 0x1ce11c3,
      0x1d8, 0x1de, 0xea0, 0x329b0d11, 0x175, 0xc2e, 0x13b, 0x187,
      0xf90, 0x1f07, 0x191b, 0xe1e, 0xfb0, 0xc2d, 0x18f, 0x1c1,
      0x904, 0xd07, 0x32b20c15, 0x1e2, 0x1d28, 0x629d0e22, 0x123d, 0x297,
      0x61990f00, 0x19c, 0x191f, 0x1ec, 0xe29c2f24, 0x2172, 0x296, 0x234,
      0x1d0, 0x1152, 0xf19, 0x1ed, 0x1b6, 0xf2e, 0x18b, 0x142,
      0x3e08, 0x92991e03, 0xd2ba0e60, 0x1e11, 0x92b, 0x1f2d, 0x11c6, 0x1cd,
      0x1156, 0x12b13c51, 0x1918, 0x327b0c11, 0xe2940f25, 0x1b2, 0x149, 0x23f,
      0xf2500c50, 0x92512e02, 0xe10, 0x273, 0x92f, 0x1e2d, 0x1233, 0x114e,
      0x1df, 0xd0d, 0xc10, 0x1a7, 0xe28, 0xd2c, 0x146, 0x981,
      0xd0f, 0x1197, 0x1e9, 0x62bb0f13, 0xedb00d20, 0x1fd, 0x1c4, 0x29f,
      0x1db, 0x15d, 0xd19, 0xf1c, 0xe25, 0x72551c23, 0x1144, 0x257,
      0xc08, 0x15e, 0xa1eb0f60, 0xd61, 0x17b, 0xd2d, 0x1dc0, 0x13c,
      0x15f, 0x15c, 0x11e4, 0xe1c, 0x225c0c25, 0x1f3, 0x1185, 0x183,
      0x298, 0xe01, 0x2d15, 0x1c1c, 0x1f6, 0x1fc, 0x2145, 0x1cc,
      0x19f, 0x90e, 0x1ab, 0x912, 0x11b9, 0x92a, 0x1130, 0x23b,
      0x0, 0x0, 0x0, 0x0, 0x0, 0x0, 0x0, 0x0,
      0x0, 0x0, 0x0, 0x0, 0x0, 0x0, 0x0, 0x0,
      0x0, 0x0, 0x0, 0x0, 0x0, 0x0, 0x0, 0x0,
      0x0, 0x0, 0x0, 0x0, 0x0, 0x0, 0x0, 0x0,
      0x0, 0x0, 0x0, 0x0, 0x0, 0x0, 0x0, 0x0,
      0x0, 0x0, 0x0, 0x0, 0x0, 0x0, 0x0, 0x0,
      0x0, 0x0, 0x0, 0x0, 0x0, 0x0, 0x0, 0x0,
      0x91590e05, 0x11da, 0x168, 0x3c17, 0x1f4, 0x117c, 0x237, 0x0,
      0x906, 0x1c0d, 0x16f, 0x216e, 0x1bf, 0xe23, 0x238, 0x1132,
      0xc0f, 0x119a, 0x165, 0x11e7, 0x1b4, 0x1b1, 0x11cf, 0x125f,
      0x3f08, 0xc0a, 0x11e0, 0x16d, 0x2274, 0x1c2c, 0x1186, 0x1141,
      0x195, 0x1d7, 0xe19, 0x167, 0xd25, 0x1bc, 0x1140, 0x118a,
      0xf05, 0x52910d03, 0x292, 0xf61, 0x21f5, 0x72bd1c22, 0x4230, 0x2b7,
      0xd0b, 0xaf0d0193, 0xc19, 0x1a3, 0x32542c20, 0x62950f22, 0x136, 0x13e,
      0x31d6, 0x1f03, 0x11a4, 0x2162, 0xc26, 0x71f10d23, 0x123e, 0x1ca,
      0xf50, 0xc07, 0xf29a0d60, 0x1ee, 0x7f264d70, 0x427d0d21, 0x1c9, 0x27f,
      0x22b81c00, 0x1d2, 0x21af, 0x1ae, 0x179, 0x1d22, 0x2b6, 0x23a,
      0xd2900d50, 0xc2710f51, 0x11e5, 0xd17, 0x1fb, 0x11b3, 0x184, 0x13d,
      0x1d9, 0x72790f01, 0x916, 0x12b31e13, 0x929, 0x173, 0x125e, 0x18e,
      0x198, 0x1153, 0x1a8, 0x16a, 0x21f0, 0xe2e, 0x13f, 0x1137,
      0x4e0b, 0x72b90c01, 0x82720f15, 0x9a1, 0x17f, 0x1fe, 0x114b, 0x18c,
      0xf0b, 0xf02, 0x1a5, 0x1c61, 0x1b5, 0x22750f21, 0x14f, 0x235,
      0x9f0f0d00, 0x90c, 0x166, 0x3d1c0163, 0x2e26, 0xbf711e21, 0x276, 0x0,
      0x15b, 0x1dc, 0x252, 0x1ac, 0x1b8, 0xd2e, 0x12be, 0x143,
      0xe0f, 0x1e0a, 0x11ef, 0x91d, 0xc2b40c24, 0xb25d0f23, 0x2188, 0x2bf,
      0x1d08, 0xe07, 0x1a9, 0x11e3, 0xf20, 0x11f70e71, 0x1256, 0x14c,
      0x12580d05, 0x1d01, 0xca0, 0x11ad, 0x0, 0x927, 0x135, 0x1c7,
      0xc0b, 0x1e51, 0x0, 0x11a2, 0x1d260eb0, 0x177, 0xcc0, 0x133,
      0x22b00c05, 0x192, 0x0, 0x1ea, 0x1176, 0x1c71, 0x139, 0x239,
      0x1c90, 0x1d1, 0x425a0e15, 0x1f11, 0x51f80cb0, 0x17a, 0x1134, 0x14a,
      0x61961155, 0x19d, 0xfa0, 0x22930c13, 0x178, 0x11b7, 0x23c, 0x11c2,
      0xc2781e50, 0xd0a, 0x31a6, 0x191a, 0x27c0e24, 0x31f2, 0x1231, 0x2147,
  };
  Params P = {};
  P.x = (const char*)d_in[0];
  P.w = (const char*)d_in[1];
  P.out = (char*)d_out;
  P.my = (char*)d_ws_all[rank];
  P.rank = rank;
  for (int i = 0; i < 7 * 64; ++i) P.ot[i >> 6][i & 63] = OT[i];
  for (int i = 0; i < N_WG - N_PUSH_WG; ++i) P.wg[i] = WG[i];
  u8 arr[32][8] = {};
  for (int d = 0; d < NR; ++d) {
    const int r = rank ^ d, c = d < 4 ? 0 : d / 2 - 1;
    const int kh = (r ^ ((r >> 1) & rank)) & 1;
    P.lk[d].peer = (char*)d_ws_all[r];
    P.lk[d].w0 = P.w + (size_t)(kh * 256 * 8192 + ((r >> 2) * 16 + col_x(r)) * 256);
    if (d < 2) continue;
    int q = 0, qn = 0;
    u32 fx = 0x3f3f3f3fu;
    for (int i = 0; ORD[c][i] != 255; ++i) {
      const int e = ORD[c][i], v = e & 15, j = (int)(RB_J >> (4 * v)) & 7;
      const int sb = v < 2 || v == 4 ? 4 : 6;
      int en, f = -1;
      bool on = true;
      if (e < 16) {
        if (c && v == OM[2 * c - 2]) continue;
        if (c && v == OM[2 * c - 1]) { f = 0; on = c == 1; }
        if (c && v == (rank & 1 ? 6 : 9)) f = 1;
        en = ((d - 2) * 32 + cont_w(v)) | ((r >> 2) * 16 + (v ^ col_x(r))) << 8;
      } else if (e < 32) {
        if (j == d) continue;
        en = ((d - 2) * 32 + cont_x(v)) | ((r ^ j) * 4 + (int)((RB_BLK >> (2 * v)) & 3u) + (j >= 4 ? r & 2 : 0)) << 8 | 8192;
      } else {
        const int dr = sb + (rank & 1), dn = dr ^ d, om = v < 4 ? OM[v] : (d == 2 ? 9 : 6);
        f = v >= 4 ? 2 * v - 7 : (v & 1 ? v - 1 : -1);
        on = v < 4;
        en = ((dr - 2) * 32 + cont_w(om)) | (16 + v) << 8 | 16384;
        P.ot[d - 1][32 + 2 * v] = P.ot[d - 1][33 + 2 * v] = (u8)((dn - 2) * 32 + cont_w(om ^ col_x(d)));
      }
      u8& a = e < 32 ? arr[en & 31][d - 2] : arr[en & 31][sb - 2 + (r & 1)];
      a = (u8)((a & 128) | (e < 32 || (NOFL && !on) ? 0 : 128) | ((a & 63) > qn || !on ? a & 63 : qn));
      if (f >= 0) fx ^= (u32)(63 ^ q) << (8 * f);
      P.it[d - 2][q++] = (u16)en;
      qn += on;
    }
    P.lk[d].f4 = (u64)P.it[d - 2][0] | (u64)P.it[d - 2][1] << 16 | (u64)P.it[d - 2][2] << 32 | (u64)P.it[d - 2][3] << 48;
    P.lk[d].meta[0] = (u32)(q | QS[c] << 16 | kh << 24);
    P.lk[d].meta[1] = fx;
  }
  for (int i = 0; i < 32; ++i)
    for (int s = 0; s < 8; ++s) P.arr[i] |= (u64)arr[i][s] << (8 * s);
  HIP_CHECK(hipFuncSetAttribute((const void*)fused_gemm_rs, hipFuncAttributeMaxDynamicSharedMemorySize, (int)LDS_BYTES));
  hipLaunchKernelGGL(fused_gemm_rs, dim3(N_WG), dim3(256), LDS_BYTES, stream, P);
  HIP_CHECK(hipGetLastError());
}
```
